```python
import numpy as np

kernel_name = "gpu_gemm_ar_m4096_k4096_n32768_bf16_none"


ABSMAX_RTOL = 0.008

COMPUTE_DTYPE = "bfloat16"
EPILOGUE = "none"
IN_QUANT = None
OUT_QUANT = None
IN_DTYPES = {"x": "bfloat16", "w_mat": "bfloat16"}
WORLD_SIZE = 8

def _to_bf16(x):
    u = np.ascontiguousarray(x, dtype=np.float32).view(np.uint32)
    return ((u + 0x7FFF + ((u >> 16) & 1)) & 0xFFFF0000).view(np.float32)

def setup_inputs():

    x = _to_bf16(np.random.randn(4096, 4096).astype(np.float32))
    w_mat = _to_bf16((np.random.randn(4096, 32768) / np.sqrt(4096)).astype(np.float32))
    return {"x": x, "w_mat": w_mat}

def _epilogue(y):
    return y

def reference(x, w_mat):
    y = (x @ w_mat).astype(np.float32)
    return _epilogue(y)
```

```cpp
#include <hip/hip_runtime.h>
#include <rccl/rccl.h>
#include <cstdio>
#include <cstdint>
#pragma clang diagnostic ignored "-Wint-to-pointer-cast"

#define HIP_CHECK(expr) do { hipError_t _e = (expr); if (_e != hipSuccess) { \
    fprintf(stderr, "HIP error %s at %s:%d\n", hipGetErrorString(_e), __FILE__, __LINE__); \
    return; } } while (0)

namespace gar {
#define LAS __attribute__((address_space(3)))
#define GAS __attribute__((address_space(1)))
typedef unsigned u32;
typedef unsigned short u16;
typedef unsigned long long u64;
typedef unsigned u32x4 __attribute__((ext_vector_type(4)));
typedef unsigned u32x2 __attribute__((ext_vector_type(2)));
typedef short s16x4 __attribute__((ext_vector_type(4)));
typedef short bf16x8 __attribute__((ext_vector_type(8)));
typedef float f32x4 __attribute__((ext_vector_type(4)));

constexpr int NR = 8, MM = 4096, NN = 32768, KS = 512;
#define GAR_DP 16
constexpr int DP = GAR_DP;
constexpr int RP = 128 - NR * DP;
constexpr int LP = DP + RP;
constexpr int NWG = 256;
#define GAR_PW 8
#define GAR_WIN 8
constexpr int PW = GAR_PW;
constexpr int NPULL = 7 * PW;
constexpr int WIN = GAR_WIN;
constexpr int GEMM0 = (NPULL + 7) / 8 * 8;
constexpr int NGEMM = NWG - GEMM0;
constexpr int NENC = NWG - NPULL;
constexpr int GX = 0, GW = 32, GY = 288;
constexpr int NGID = GY + 16 * DP;
constexpr int NS = 32 + 2 * DP + 2 * RP + 16 * DP;
constexpr int NTILE = 16 * LP;
constexpr int ROUNDS = (NTILE + NGEMM - 1) / NGEMM;
constexpr int GPW = (NS + PW - 1) / PW;
constexpr u64 DEADLINE = 30000000ull;

constexpr size_t WS_CALL = 0;
constexpr size_t WS_TMO = 32768;
constexpr size_t WS_CNT = 36864;
constexpr size_t WS_FLAG = 65536;
constexpr size_t FLAG_SRC = 544 * 32, FLAG_PAR = 8 * FLAG_SRC;
constexpr size_t WS_DBG = 524288;
constexpr size_t GRP_STRIDE = 131072;
constexpr size_t WS_STG = 2u << 20;
constexpr size_t STG_PAR = 544 * GRP_STRIDE;
constexpr size_t WS_XI = WS_STG + 2 * STG_PAR;
constexpr size_t PANEL = 64 * 32768;
constexpr size_t WS_WI = WS_XI + 16 * PANEL;
constexpr size_t WS_END = WS_WI + (size_t)LP * PANEL;
static_assert(WS_FLAG + 2 * FLAG_PAR <= WS_DBG && NGID <= 544, "layout");

constexpr u32 LDS_CTL = 131072;
constexpr u32 LDS_BYTES = LDS_CTL + 1024;
constexpr u32 CTL_CALL = LDS_CTL, CTL_SIZE = LDS_CTL + 64  , CTL_BAD = LDS_CTL + 128;
constexpr u32 CTL_NREADY = LDS_CTL + 192, CTL_LANDED = LDS_CTL + 256, CTL_ABORT = LDS_CTL + 320, CTL_NEXT = LDS_CTL + 384  ;
constexpr u32 E_HIST = 0, E_LUT = 1024, E_L2 = 1152, E_L3 = E_L2 + 4160, E_VAR = E_L3 + 2112, E_END = E_VAR + 2112;
static_assert(E_END <= 10240 && E_L3 % 16 == 0 && E_VAR % 16 == 0, "packer scratch");
constexpr u32 CAP2 = 4080, CAP3 = 2032, CAPV = 2032, FIXB = 10240, RAWB = 16384, RAWCODE = 255;
constexpr u32 RING = 65536, RMASK = RING - 1, ND = 6;
constexpr u32 D_SCR = RING, D_SCRB = 6400, D_L2 = 0, D_L3 = 4160;
constexpr u32 D_DESC = D_SCR + ND * D_SCRB, D_GTOT = D_DESC + GPW * 32;
static_assert(D_L3 + 2112 <= D_SCRB && D_GTOT + GPW * 4 <= LDS_CTL && D_DESC % 16 == 0, "puller LDS");

struct Params {
    char* ws[NR];
    const u16* x; const u16* w; u16* out;
    int rank, pad;
    u32 stream[(NS + 1) / 2];
    u32 enc[144];
    u32 tiles[ROUNDS * NGEMM / 2];
};
__device__ __forceinline__ u32 tab16(const u32* t, u32 i) { return (t[i >> 1] >> ((i & 1) * 16)) & 0xffffu; }
static_assert(sizeof(Params) <= 4096, "kernarg");

__device__ __forceinline__ u32 lds32(u32 a) { return *(LAS u32*)a; }
__device__ __forceinline__ u32 lds32v(u32 a) { return *(volatile LAS u32*)a; }
__device__ __forceinline__ void lds32vw(u32 a, u32 x) { *(volatile LAS u32*)a = x; }
__device__ __forceinline__ u32x4 lds128(u32 a) { return *(LAS u32x4*)a; }
__device__ __forceinline__ void lds128w(u32 a, u32x4 x) { *(LAS u32x4*)a = x; }
__device__ __forceinline__ void lds_or(u32 a, u32 x) { __hip_atomic_fetch_or((LAS u32*)a, x, __ATOMIC_RELAXED, __HIP_MEMORY_SCOPE_WORKGROUP); }
__device__ __forceinline__ void lds_add(u32 a, u32 x) { __hip_atomic_fetch_add((LAS u32*)a, x, __ATOMIC_RELAXED, __HIP_MEMORY_SCOPE_WORKGROUP); }
__device__ __forceinline__ u32 rfl(u32 x) { return (u32)__builtin_amdgcn_readfirstlane((int)x); }
#define CBAR() asm volatile("" ::: "memory")
#define WAIT_LGKM0() asm volatile("s_waitcnt lgkmcnt(0)" ::: "memory")
#define WAIT_VM(n) asm volatile("s_waitcnt vmcnt(%0)" :: "i"(n) : "memory")
#define BAR() do { CBAR(); __builtin_amdgcn_s_barrier(); CBAR(); } while (0)
__device__ __forceinline__ void st_sys(void* p, u32x4 x) { asm volatile("global_store_dwordx4 %0, %1, off sc0 sc1\n\ts_nop 1" :: "v"(p), "v"(x) : "memory"); }
__device__ __forceinline__ void st_dev(void* p, u32x4 x) { asm volatile("global_store_dwordx4 %0, %1, off sc1\n\ts_nop 1" :: "v"(p), "v"(x) : "memory"); }
__device__ __forceinline__ void glds_sys(const void* g, u32 dst) {
    u32 keep;
    asm volatile("s_mov_b32 %0, m0\n\ts_mov_b32 m0, %2\n\ts_nop 0\n\tglobal_load_lds_dwordx4 %1, off sc0 sc1\n\ts_mov_b32 m0, %0" : "=&s"(keep) : "v"(g), "s"(dst) : "memory");
}
__device__ __forceinline__ void glds_plain(const void* g, u32 dst) {
    u32 keep;
    asm volatile("s_mov_b32 %0, m0\n\ts_mov_b32 m0, %2\n\ts_nop 0\n\tglobal_load_lds_dwordx4 %1, off\n\ts_mov_b32 m0, %0" : "=&s"(keep) : "v"(g), "s"(dst) : "memory");
}
__device__ __forceinline__ u64 now() { return __builtin_amdgcn_s_memrealtime(); }
template <int CTRL, int RM, bool BC> __device__ __forceinline__ u32 dpp(u32 v) { return (u32)__builtin_amdgcn_update_dpp(0, (int)v, CTRL, RM, 0xf, BC); }
__device__ __forceinline__ u32 scan_incl(u32 v) {
    v += dpp<0x111, 0xf, true>(v); v += dpp<0x112, 0xf, true>(v); v += dpp<0x114, 0xf, true>(v); v += dpp<0x118, 0xf, true>(v);
    v += dpp<0x142, 0xa, false>(v); v += dpp<0x143, 0xc, false>(v); return v;
}
__device__ __forceinline__ u32 umax(u32 a, u32 b) { return a > b ? a : b; }
__device__ __forceinline__ u32 umin(u32 a, u32 b) { return a < b ? a : b; }
__device__ __forceinline__ u32 wave_max(u32 v) {
    v = umax(v, dpp<0x111, 0xf, true>(v)); v = umax(v, dpp<0x112, 0xf, true>(v)); v = umax(v, dpp<0x114, 0xf, true>(v)); v = umax(v, dpp<0x118, 0xf, true>(v));
    v = umax(v, dpp<0x142, 0xa, false>(v)); v = umax(v, dpp<0x143, 0xc, false>(v)); return (u32)__builtin_amdgcn_readlane((int)v, 63);
}
__device__ __forceinline__ u32 popc(u32 x) { return (u32)__builtin_popcount(x); }
__device__ __forceinline__ u32 perm(u32 hi, u32 lo, u32 sel) { return __builtin_amdgcn_perm(hi, lo, sel); }
__device__ __forceinline__ u32 swz(u32 ob) { return ob ^ (((ob >> 9) & 1) << 5); }

#define STAMP(P, slot) do {} while (0)

__device__ __forceinline__ u32 nzb(u32 z) { return ((z & 0x7f7f7f7fu) + 0x7f7f7f7fu) | z; }
__device__ __forceinline__ u32 classify(u32 e, u32 s1, u32 s2, u32 s3) {
    const u32 a1 = nzb(e ^ s1), a2 = nzb(e ^ s2), a3 = nzb(e ^ s3);
    return ((~(a1 & a3) >> 7) & 0x01010101u) | ((~(a2 & a3) >> 6) & 0x02020202u);
}
__device__ __forceinline__ u32 escbits(u32 c) { return ~(c | (c >> 1)) & 0x55555555u; }
__device__ __forceinline__ void compact(u32 e, u32 m, u32& off, u32 lut) {
    const u32 sel = lds32(lut + __builtin_amdgcn_udot4(m, 0x20100804u, 0u, false));
    const u64 sh = (u64)perm(0u, e, sel) << ((off & 3u) * 8u);
    const u32 a = off & ~3u;
    lds_or(a, (u32)sh); lds_or(a + 4, (u32)(sh >> 32));
    off += popc(m);
}
__device__ __forceinline__ u32 pack_level(u32 src, u32 n, u32 t, u32 cdst, u32 edst, u32 cap, u32 lut, u32 lane) {
    const u32 s1 = ((t >> 8) & 255u) * 0x01010101u, s2 = ((t >> 16) & 255u) * 0x01010101u, s3 = (t >> 24) * 0x01010101u;
    if (lane < 16) *(LAS unsigned char*)(src + n + lane) = (unsigned char)s1;
    u32 ne = 0;
    for (u32 r = 0; r * 1024 < n; ++r) {
        const u32 p0 = r * 1024 + lane * 16; const bool act = p0 < n;
        CBAR();
        const u32x4 q = lds128(src + p0);
        u32 c = classify(q.x, s1, s2, s3) | (classify(q.y, s1, s2, s3) << 2) | (classify(q.z, s1, s2, s3) << 4) | (classify(q.w, s1, s2, s3) << 6);
        if (!act) c = 0x55555555u;
        const u32 e = escbits(c), cnt = popc(e), inc = scan_incl(cnt), tot = (u32)__builtin_amdgcn_readlane((int)inc, 63);
        if (ne + tot > cap) return ~0u;
        u32 off = edst + ne + inc - cnt;
        compact(q.x, e & 0x01010101u, off, lut); compact(q.y, (e >> 2) & 0x01010101u, off, lut);
        compact(q.z, (e >> 4) & 0x01010101u, off, lut); compact(q.w, (e >> 6) & 0x01010101u, off, lut);
        if (act) *(LAS u32*)(cdst + (r * 64 + lane) * 4) = c;
        ne += tot;
    }
    CBAR();
    return ne;
}
__device__ __forceinline__ u32 pack_chunk(const u32 (&v)[64], u32 (&Mr)[32], u32 (&Er)[32], u32 (&Cr)[8], u32 scr, u32 lane) {
    CBAR();
#pragma unroll
    for (int j = 0; j < 10; ++j) lds128w(scr + j * 1024 + lane * 16, (u32x4){0u, 0u, 0u, 0u});
    CBAR();
    if (lane < 16) { u32 sel = 0x0c0c0c0cu, k = 0;
#pragma unroll
        for (u32 b = 0; b < 4; ++b) if ((lane >> b) & 1) { sel = (sel & ~(255u << k)) | (b << k); k += 8; }
        *(LAS u32*)(scr + E_LUT + lane * 4) = sel; }
#pragma unroll
    for (int g = 0; g < 32; ++g) {
        const u32 a = __builtin_amdgcn_alignbit(v[2 * g], v[2 * g], 31), b = __builtin_amdgcn_alignbit(v[2 * g + 1], v[2 * g + 1], 31);
        Er[g] = perm(b, a, 0x07050301u); Mr[g] = perm(b, a, 0x06040200u);
    }
#pragma unroll
    for (int g = 0; g < 32; g += 4) {
        const u32 e = Er[g + ((g >> 2) & 3)];
        lds_add(scr + E_HIST + (e & 255u) * 4, 1u); lds_add(scr + E_HIST + ((e >> 8) & 255u) * 4, 1u);
        lds_add(scr + E_HIST + ((e >> 16) & 255u) * 4, 1u); lds_add(scr + E_HIST + (e >> 24) * 4, 1u);
    }
    CBAR();
    const u32x4 h = lds128(scr + E_HIST + lane * 16);
    u32 k0 = ((h.x + 1) << 8) | (lane * 4), k1 = ((h.y + 1) << 8) | (lane * 4 + 1), k2 = ((h.z + 1) << 8) | (lane * 4 + 2), k3 = ((h.w + 1) << 8) | (lane * 4 + 3);
    u32 T[3] = {0u, 0u, 0u};
#pragma unroll
    for (int i = 0; i < 9; ++i) {
        const u32 best = wave_max(umax(umax(k0, k1), umax(k2, k3)));
        T[i / 3] |= (best & 255u) << (8 * (i % 3 + 1));
        k0 = k0 == best ? 0u : k0; k1 = k1 == best ? 0u : k1; k2 = k2 == best ? 0u : k2; k3 = k3 == best ? 0u : k3;
    }
    const u32 s1 = ((T[0] >> 8) & 255u) * 0x01010101u, s2 = ((T[0] >> 16) & 255u) * 0x01010101u, s3 = (T[0] >> 24) * 0x01010101u;
    u32 cnt = 0;
#pragma unroll
    for (int k = 0; k < 8; ++k) {
        Cr[k] = classify(Er[4 * k], s1, s2, s3) | (classify(Er[4 * k + 1], s1, s2, s3) << 2) | (classify(Er[4 * k + 2], s1, s2, s3) << 4) | (classify(Er[4 * k + 3], s1, s2, s3) << 6);
        cnt += popc(escbits(Cr[k]));
    }
    const u32 inc = scan_incl(cnt), n2 = (u32)__builtin_amdgcn_readlane((int)inc, 63);
    if (n2 > CAP2) return RAWCODE;
    const u32 lut = scr + E_LUT;
    u32 off = scr + E_L2 + inc - cnt;
#pragma unroll
    for (int k = 0; k < 8; ++k) {
        const u32 e = escbits(Cr[k]);
#pragma unroll
        for (int j = 0; j < 4; ++j) compact(Er[4 * k + j], (e >> (2 * j)) & 0x01010101u, off, lut);
    }
    CBAR();
    const u32 d2 = (n2 + 15) >> 4;
    const u32 n3 = pack_level(scr + E_L2, n2, T[1], scr + E_VAR + 12, scr + E_L3, CAP3, lut, lane);
    if (n3 == ~0u) return RAWCODE;
    const u32 d3 = (n3 + 15) >> 4, b4 = 12 + 4 * (d2 + d3);
    const u32 n4 = pack_level(scr + E_L3, n3, T[2], scr + E_VAR + 12 + 4 * d2, scr + E_VAR + b4, CAPV - b4, lut, lane);
    if (n4 == ~0u) return RAWCODE;
    if (lane == 0) { *(LAS u32*)(scr + E_VAR) = T[0] | d2; *(LAS u32*)(scr + E_VAR + 4) = T[1] | d3; *(LAS u32*)(scr + E_VAR + 8) = T[2]; }
    CBAR();
    return (b4 + n4 + 15) >> 4;
}
__device__ __forceinline__ u32 chunk_bytes(u32 code) { return code == RAWCODE ? RAWB : FIXB + 16 * code; }

__device__ __forceinline__ void pack_group(const Params& P, const u32 (&v)[64], u32 gid, u32 peers, u32 call, u32 wid, u32 lane) {
    u32 Mr[32], Er[32], Cr[8];
    const u32 scr = wid * 16384;
    const u32 code = rfl(pack_chunk(v, Mr, Er, Cr, scr, lane));
    if (lane == 0) lds32vw(CTL_SIZE + wid * 4, code);
    WAIT_LGKM0(); BAR();
    const u32 mine = lds32v(CTL_SIZE + (lane & 7) * 4);
    u32 off = 0;
#pragma unroll
    for (u32 j = 0; j < 7; ++j) { const u32 c = (u32)__builtin_amdgcn_readlane((int)mine, j); if (j < wid) off += chunk_bytes(c); }
    char* dst = P.ws[P.rank] + WS_STG + (call & 1) * STG_PAR + (size_t)gid * GRP_STRIDE + off + lane * 16;
#pragma unroll
    for (int j = 0; j < 8; ++j) st_sys(dst + j * 1024, (u32x4){Mr[4 * j], Mr[4 * j + 1], Mr[4 * j + 2], Mr[4 * j + 3]});
    if (code == RAWCODE) {
#pragma unroll
        for (int j = 0; j < 8; ++j) st_sys(dst + 8192 + j * 1024, (u32x4){Er[4 * j], Er[4 * j + 1], Er[4 * j + 2], Er[4 * j + 3]});
    } else {
        st_sys(dst + 8192, (u32x4){Cr[0], Cr[1], Cr[2], Cr[3]}); st_sys(dst + 9216, (u32x4){Cr[4], Cr[5], Cr[6], Cr[7]});
        if (lane < code) st_sys(dst + FIXB, lds128(scr + E_VAR + lane * 16));
        if (lane + 64 < code) st_sys(dst + FIXB + 1024, lds128(scr + E_VAR + 1024 + lane * 16));
    }
    WAIT_VM(0); WAIT_LGKM0(); BAR();
    if (wid == 0) {
        const u32 q = lane >> 3;
        if ((peers >> q) & 1) {
            char* wq = P.ws[0];
#pragma unroll
            for (u32 j = 1; j < NR; ++j) wq = q == j ? P.ws[j] : wq;
            u32* f = (u32*)(wq + WS_FLAG + (call & 1) * FLAG_PAR + (size_t)P.rank * FLAG_SRC + gid * 32) + (lane & 7);
            __hip_atomic_store(f, (call << 8) | mine, __ATOMIC_RELAXED, __HIP_MEMORY_SCOPE_SYSTEM);
        }
    }
}

__device__ __forceinline__ u32 expand(u32 c, u32 t, u32 win) {
    const u32 mk = perm(0u, 0xffu, c);
    const u32 p = __umul24(mk & 0x02020202u, 0x808080u) + 0x04040404u;
    return perm(win, t, (mk & p) | c);
}
template <bool WRAP> __device__ __forceinline__ u32 window(u32 off) {
    const u32 a = off & ~3u;
    const u32 lo = lds32(WRAP ? (a & RMASK) : a), hi = lds32(WRAP ? ((a + 4) & RMASK) : a + 4);
    return __builtin_amdgcn_alignbyte(hi, lo, off & 3u);
}
template <bool WRAP> __device__ __forceinline__ void unpack4(u32 c, u32 t, u32& off, u32& o0, u32& o1, u32& o2, u32& o3) {
    const u32 e = escbits(c);
    const u32 f1 = off + popc(e & 0x01010101u), f2 = f1 + popc(e & 0x04040404u), f3 = f2 + popc(e & 0x10101010u);
    const u32 w0 = window<WRAP>(off), w1 = window<WRAP>(f1), w2 = window<WRAP>(f2), w3 = window<WRAP>(f3);
    off = f3 + popc(e & 0x40404040u);
    o0 = expand(c & 0x03030303u, t, w0); o1 = expand((c >> 2) & 0x03030303u, t, w1);
    o2 = expand((c >> 4) & 0x03030303u, t, w2); o3 = expand((c >> 6) & 0x03030303u, t, w3);
}
template <bool WRAP> __device__ __forceinline__ u32 unpack_round(u32 c, u32 t, u32 src, u32 dst) {
    const u32 cnt = popc(escbits(c)), inc = scan_incl(cnt);
    u32 off = src + inc - cnt; u32x4 o; u32 a, b, d, e;
    unpack4<WRAP>(c, t, off, a, b, d, e); o.x = a; o.y = b; o.z = d; o.w = e;
    lds128w(dst, o);
    return (u32)__builtin_amdgcn_readlane((int)inc, 63);
}
__device__ __forceinline__ void unpack_chunk(u32 S, u32 code, u32 scr, u32 lane, u32 rel_addr, u32 rel_val, u32 (&Mr)[32], u32 (&Er)[32]) {
    const u32 a0 = S + lane * 16;
#pragma unroll
    for (int j = 0; j < 8; ++j) { const u32x4 q = lds128((a0 + j * 1024) & RMASK); Mr[4 * j] = q.x; Mr[4 * j + 1] = q.y; Mr[4 * j + 2] = q.z; Mr[4 * j + 3] = q.w; }
    if (code == RAWCODE) {
#pragma unroll
        for (int j = 0; j < 8; ++j) { const u32x4 q = lds128((a0 + 8192 + j * 1024) & RMASK); Er[4 * j] = q.x; Er[4 * j + 1] = q.y; Er[4 * j + 2] = q.z; Er[4 * j + 3] = q.w; }
        WAIT_LGKM0(); lds32vw(rel_addr, rel_val); CBAR();
        return;
    }
    const u32x4 ca = lds128((a0 + 8192) & RMASK), cb = lds128((a0 + 9216) & RMASK);
    const u32 V = S + FIXB;
    const u32x4 hd = lds128(V & RMASK);
    const u32 t1 = rfl(hd.x), t2 = rfl(hd.y), t3 = rfl(hd.z), d2 = t1 & 255u, d3 = t2 & 127u;
    u32 c2[4], c3[2];
#pragma unroll
    for (int r = 0; r < 4; ++r) { const u32 i = r * 64 + lane; const u32 x = lds32((V + 12 + 4 * i) & RMASK); c2[r] = i < d2 ? x : 0x55555555u; }
#pragma unroll
    for (int r = 0; r < 2; ++r) { const u32 i = r * 64 + lane; const u32 x = lds32((V + 12 + 4 * (d2 + i)) & RMASK); c3[r] = i < d3 ? x : 0x55555555u; }
    u32 s4 = V + 12 + 4 * (d2 + d3);
    s4 += unpack_round<true>(c3[0], t3, s4, scr + D_L3 + lane * 16);
    if (d3 > 64) unpack_round<true>(c3[1], t3, s4, scr + D_L3 + 1024 + lane * 16);
    WAIT_LGKM0(); lds32vw(rel_addr, rel_val); CBAR();
    u32 s3 = scr + D_L3;
    s3 += unpack_round<false>(c2[0], t2, s3, scr + D_L2 + lane * 16);
    s3 += unpack_round<false>(c2[1], t2, s3, scr + D_L2 + 1024 + lane * 16);
    if (d2 > 128) {
        s3 += unpack_round<false>(c2[2], t2, s3, scr + D_L2 + 2048 + lane * 16);
        if (d2 > 192) unpack_round<false>(c2[3], t2, s3, scr + D_L2 + 3072 + lane * 16);
    }
    CBAR();
    const u32 C[8] = {ca.x, ca.y, ca.z, ca.w, cb.x, cb.y, cb.z, cb.w};
    u32 cnt = 0;
#pragma unroll
    for (int k = 0; k < 8; ++k) cnt += popc(escbits(C[k]));
    u32 off = scr + D_L2 + scan_incl(cnt) - cnt;
#pragma unroll
    for (int k = 0; k < 8; ++k) unpack4<false>(C[k], t1, off, Er[4 * k], Er[4 * k + 1], Er[4 * k + 2], Er[4 * k + 3]);
    CBAR();
}
__device__ __forceinline__ u32x4 piece(const u32 (&Mr)[32], const u32 (&Er)[32], int p) {
    const u32 a = perm(Er[2 * p], Mr[2 * p], 0x05010400u), b = perm(Er[2 * p], Mr[2 * p], 0x07030602u);
    const u32 c = perm(Er[2 * p + 1], Mr[2 * p + 1], 0x05010400u), d = perm(Er[2 * p + 1], Mr[2 * p + 1], 0x07030602u);
    return (u32x4){__builtin_amdgcn_alignbit(a, a, 1), __builtin_amdgcn_alignbit(b, b, 1), __builtin_amdgcn_alignbit(c, c, 1), __builtin_amdgcn_alignbit(d, d, 1)};
}

__device__ __forceinline__ u32 x_piece_off(u32 half, int p) { return half * 16384 + p * 1024; }
__device__ __forceinline__ u32 w_piece_off(u32 h, int p) {
    const u32 g32 = p >> 2, n = (p >> 1) & 1, kb = p & 1;
    return (g32 & 1) * 16384 + ((2 * (2 * h + (g32 >> 1)) + n) * 2 + kb) * 1024;
}
__device__ __forceinline__ u32 local_panel(u32 j, u32 rank) { return j >= (u32)(NR * DP) ? j - (NR - 1) * DP : j - rank * DP; }
__device__ __forceinline__ u32 global_panel(u32 jj, u32 rank) { return jj >= (u32)DP ? jj + (NR - 1) * DP : jj + rank * DP; }
__device__ __forceinline__ u32* counter(const Params& P, u32 i) { return (u32*)(P.ws[P.rank] + WS_CNT + i * 128); }
__device__ __forceinline__ u32* tmo_word(const Params& P) { return (u32*)(P.ws[P.rank] + WS_TMO); }
__device__ __forceinline__ bool expired(const Params& P, u64 t0, u32 call) {
    if (__hip_atomic_load(tmo_word(P), __ATOMIC_RELAXED, __HIP_MEMORY_SCOPE_AGENT) == call || now() - t0 > DEADLINE) {
        __hip_atomic_store(tmo_word(P), call, __ATOMIC_RELAXED, __HIP_MEMORY_SCOPE_AGENT); return true; }
    return false;
}

__device__ __forceinline__ void puller(const Params& P, u32 call, u32 wid, u32 lane) {
    const u32 sub = blockIdx.x / 7, q = (P.rank + 1 + blockIdx.x % 7) & 7;
    const u32 ng = (NS - sub + PW - 1) / PW;
    if (wid == 0) {
        __builtin_amdgcn_s_setprio(3);
        const char* base = P.ws[q] + WS_STG + (call & 1) * STG_PAR + lane * 16;
        u32 pos = 0, pub = 0, limit = RING / 1024;
        for (u32 gi = 0; gi < ng; ++gi) {
            if (rfl(lds32v(CTL_NREADY)) <= gi) {
                WAIT_VM(0); lds32vw(CTL_LANDED, pos); pub = pos;
                bool dead = false;
                while (rfl(lds32v(CTL_NREADY)) <= gi) { if (rfl(lds32v(CTL_ABORT))) { dead = true; break; } __builtin_amdgcn_s_sleep(1); }
                if (dead) break;
            }
            CBAR();
            const u32 gt = rfl(lds32v(D_GTOT + gi * 4)), total = gt & 0x3ffffu;
            const char* src = base + (size_t)(gt >> 18) * GRP_STRIDE;
            for (u32 o = 0; o < total; o += 1024) {
                if (pos >= limit) {
                    bool drained = false;
                    for (;;) {
                        const u32 nx = lds32v(CTL_NEXT + (lane & 7) * 4);
                        u32 f = (u32)__builtin_amdgcn_readlane((int)nx, 0);
#pragma unroll
                        for (u32 j = 1; j < ND; ++j) f = umin(f, (u32)__builtin_amdgcn_readlane((int)nx, j));
                        f = umin(f, gi * 8 + 7);
                        limit = (rfl(lds32v(D_DESC + f * 4)) >> 14) + RING / 1024;
                        if (pos < limit) break;
                        if (!drained) { WAIT_VM(0); lds32vw(CTL_LANDED, pos); pub = pos; drained = true; }
                        __builtin_amdgcn_s_sleep(1);
                    }
                }
                if (o + lane * 16 < total) glds_sys(src + o, (pos * 1024) & RMASK);
                ++pos;
                WAIT_VM(WIN);
                if (pos > pub + WIN) { pub = pos - WIN; lds32vw(CTL_LANDED, pub); }
            }
            STAMP(P, 128 + (gi & 127));
        }
        WAIT_VM(0); lds32vw(CTL_LANDED, pos);
    } else if (wid == 1) {
        u32 basek = 0;
        for (u32 gi = 0; gi < ng; ++gi) {
            const u32 gid = tab16(P.stream, sub + gi * PW);
            const u32* f = (const u32*)(P.ws[P.rank] + WS_FLAG + (call & 1) * FLAG_PAR + (size_t)q * FLAG_SRC + gid * 32) + (lane & 7);
            const u64 t0 = now(); u32 val; bool dead = false;
            for (;;) {
                val = __hip_atomic_load(f, __ATOMIC_RELAXED, __HIP_MEMORY_SCOPE_SYSTEM);
                if (__all((val >> 8) == (call & 0xffffffu))) break;
                if (__any(expired(P, t0, call))) { dead = true; break; }
                __builtin_amdgcn_s_sleep(2);
            }
            if (dead) {
                lds32vw(CTL_ABORT, 1u);
                const u32 t = tab16(P.stream, sub + (ng - 1) * PW) - GY;
                if (lane == 0 && t < (u32)(16 * DP)) P.out[(size_t)(t & 15) * 256 * NN + (q * DP + (t >> 4)) * 256] = 0x7fc0;
                break;
            }
            u32 code = val & 255u; if (code > 127u) code = RAWCODE;
            const u32 sz = lane < 8 ? chunk_bytes(code) : 0u;
            u32 inc = sz; inc += dpp<0x111, 0xf, true>(inc); inc += dpp<0x112, 0xf, true>(inc); inc += dpp<0x114, 0xf, true>(inc);
            const u32 total = (u32)__builtin_amdgcn_readlane((int)inc, 7);
            if (lane < 8) lds32vw(D_DESC + (gi * 8 + lane) * 4, ((basek * 1024 + inc - sz) << 4) | code);
            if (lane == 0) lds32vw(D_GTOT + gi * 4, total | (gid << 18));
            WAIT_LGKM0();
            lds32vw(CTL_NREADY, gi + 1);
            basek += (total + 1023) >> 10;
            STAMP(P, (gi & 127));
        }
    } else {
        const u32 dw = wid - 2, scr = D_SCR + dw * D_SCRB;
        const u32 loff = swz((lane & 15) * 64 + (lane >> 4) * 16);
        for (u32 c = dw; c < ng * 8; c += ND) {
            const u32 gi = c >> 3, w = c & 7; bool dead = false;
            while (rfl(lds32v(CTL_NREADY)) <= gi) { if (rfl(lds32v(CTL_ABORT))) { dead = true; break; } __builtin_amdgcn_s_sleep(2); }
            if (dead) { lds32vw(CTL_NEXT + dw * 4, 0x7fffffffu); break; }
            CBAR();
            const u32 d = rfl(lds32v(D_DESC + c * 4)), code = d & 255u, S = (d >> 8) << 4;
            const u32 need = (S + chunk_bytes(code) + 1023) >> 10;
            const u32 gid = rfl(lds32v(D_GTOT + gi * 4)) >> 18;
            while (rfl(lds32v(CTL_LANDED)) < need) __builtin_amdgcn_s_sleep(1);
            CBAR();
            u32 Mr[32], Er[32];
            unpack_chunk(S, code, scr, lane, CTL_NEXT + dw * 4, c + ND, Mr, Er);
            if (gid >= (u32)GY) {
                const u32 t = gid - GY;
                u16* o = P.out + (size_t)((t & 15) * 256 + (w >> 2) * 64 + (lane & 15)) * NN + (q * DP + (t >> 4)) * 256 + (w & 3) * 64 + (lane >> 4) * 8;
#pragma unroll
                for (int p = 0; p < 16; ++p) *(u32x4*)(o + (size_t)((p >> 3) * 128 + ((p >> 1) & 3) * 16) * NN + (p & 1) * 32) = piece(Mr, Er, p);
            } else {
                const bool isx = gid < (u32)GW;
                const u32 pn = isx ? gid >> 1 : local_panel((gid - GW) >> 1, P.rank);
                char* o = P.ws[P.rank] + (isx ? WS_XI : WS_WI) + (size_t)pn * PANEL + (size_t)(q * 8 + (gid & 1) * 4 + (w >> 1)) * 32768 + loff;
                if (isx) {
#pragma unroll
                    for (int p = 0; p < 16; ++p) st_dev(o + x_piece_off(w & 1, p), piece(Mr, Er, p));
                } else {
#pragma unroll
                    for (int p = 0; p < 16; ++p) st_dev(o + w_piece_off(w & 1, p), piece(Mr, Er, p));
                }
                WAIT_VM(0);
                if (lane == 0) __hip_atomic_fetch_add(counter(P, isx ? pn : 16 + pn), 1u, __ATOMIC_RELAXED, __HIP_MEMORY_SCOPE_AGENT);
            }
        }
    }
}

__device__ __forceinline__ void pack_inputs(const Params& P, u32 gid, u32 call, u32 wid, u32 lane) {
    u32 v[64];
    const u32 ktl = (gid & 1) * 4 + (wid >> 1), half = wid & 1;
    const u32 loff = swz((lane & 15) * 64 + (lane >> 4) * 16);
    u32 peers = 255u & ~(1u << P.rank); bool own = true; u32 pn;
    char* o;
    if (gid < (u32)GW) {
        pn = gid >> 1;
        const u16* s = P.x + (size_t)(pn * 256 + half * 128 + (lane & 15)) * KS + ktl * 64 + (lane >> 4) * 8;
#pragma unroll
        for (int p = 0; p < 16; ++p) { const u32x4 t = *(const u32x4*)(s + (p >> 1) * 16 * KS + (p & 1) * 32); v[4 * p] = t.x; v[4 * p + 1] = t.y; v[4 * p + 2] = t.z; v[4 * p + 3] = t.w; }
        o = P.ws[P.rank] + WS_XI + (size_t)pn * PANEL + (size_t)(P.rank * 8 + ktl) * 32768 + loff;
#pragma unroll
        for (int p = 0; p < 16; ++p) st_dev(o + x_piece_off(half, p), (u32x4){v[4 * p], v[4 * p + 1], v[4 * p + 2], v[4 * p + 3]});
    } else {
        const u32 j = (gid - GW) >> 1;
        if (j < (u32)(NR * DP)) { const u32 dest = j / DP; own = dest == (u32)P.rank; peers = own ? 0u : 1u << dest; }
        const u16* s = P.w + (size_t)(ktl * 64 + (lane >> 4)) * NN + j * 256 + half * 128 + (lane & 15) * 8;
        const u32 scr = wid * 16384;
#pragma unroll
        for (int i = 0; i < 16; ++i) glds_plain(s + (size_t)i * 4 * NN, scr + i * 1024);
        WAIT_VM(0);
        const u32 ta = scr + ((lane >> 4) * 8 + ((lane & 15) >> 2)) * 256 + (lane & 3) * 16;
#pragma unroll
        for (int p = 0; p < 16; ++p) {
            const u32 a = ta + (p & 1) * 32 * 256 + (p >> 2) * 64 + ((p >> 1) & 1) * 8;
            const u32x2 lo = __builtin_bit_cast(u32x2, __builtin_amdgcn_ds_read_tr16_b64_v4i16((LAS s16x4*)a));
            const u32x2 hi = __builtin_bit_cast(u32x2, __builtin_amdgcn_ds_read_tr16_b64_v4i16((LAS s16x4*)(a + 4 * 256)));
            v[4 * p] = lo.x; v[4 * p + 1] = lo.y; v[4 * p + 2] = hi.x; v[4 * p + 3] = hi.y;
        }
        WAIT_LGKM0();
        pn = 16;
        if (own) {
            const u32 jj = local_panel(j, P.rank); pn = 16 + jj;
            o = P.ws[P.rank] + WS_WI + (size_t)jj * PANEL + (size_t)(P.rank * 8 + ktl) * 32768 + loff;
#pragma unroll
            for (int p = 0; p < 16; ++p) st_dev(o + w_piece_off(half, p), (u32x4){v[4 * p], v[4 * p + 1], v[4 * p + 2], v[4 * p + 3]});
        }
    }
    if (peers) pack_group(P, v, gid, peers, call, wid, lane);
    if (own) { WAIT_VM(0); if (lane == 0) __hip_atomic_fetch_add(counter(P, pn), 1u, __ATOMIC_RELAXED, __HIP_MEMORY_SCOPE_AGENT); }
}

__device__ __forceinline__ u32 cvt_pk(float lo, float hi) { u32 r; asm("v_cvt_pk_bf16_f32 %0, %1, %2" : "=v"(r) : "v"(lo), "v"(hi)); return r; }
__device__ __forceinline__ void wait_panels(const Params& P, u32 tile, u32 call, u32 wid, u32 lane) {
    if (wid == 0) {
        const u32* c = counter(P, (lane & 1) ? 16 + (tile >> 4) : (tile & 15));
        const u64 t0 = now();
        for (;;) {
            const u32 x = __hip_atomic_load(c, __ATOMIC_RELAXED, __HIP_MEMORY_SCOPE_AGENT);
            if (__all((int)(x - call * 128u) >= 0)) break;
            if (__any(expired(P, t0, call))) { asm volatile("ds_write_b32 %0, %1\n\ts_waitcnt lgkmcnt(0)" :: "v"(CTL_BAD), "v"(1u) : "memory"); break; }
            __builtin_amdgcn_s_sleep(4);
        }
        __builtin_amdgcn_fence(__ATOMIC_ACQUIRE, "agent");
        WAIT_VM(0);
    }
    BAR();
}
__device__ __forceinline__ void gemm_role(const Params& P, unsigned char* lds_generic, u32 call, u32 wid, u32 lane) {
    LAS unsigned char* lds = (LAS unsigned char*)lds_generic;
    const u32 gw = blockIdx.x - GEMM0, tid = threadIdx.x;
    const u32 wr = wid >> 2, wc = wid & 3, fr = lane & 15, fq = lane >> 4;
    constexpr int HTB = 16384;
    const u32 voff = tid * 16, ldsw = wid * 1024;
    const u32 fo = swz(fr * 64 + fq * 16);
    const u32 aoff = wr * 8192 + fo, boff = wc * 4096 + fo;
#define G_SA(b, h) (((b) * 2 + (h)) * HTB)
#define G_SB(b, h) ((4 + (b) * 2 + (h)) * HTB)
#define G_STAGE(bufoff, gbase) do { _Pragma("unroll") for (int _i = 0; _i < 2; ++_i) \
        __builtin_amdgcn_global_load_lds((const u32*)((const char*)(gbase) + voff + _i * 8192), (LAS u32*)(lds + (bufoff) + ldsw + _i * 8192), 16, 0, 0); } while (0)
#define G_LDA(dst, b, h) do { _Pragma("unroll") for (int m = 0; m < 4; ++m) _Pragma("unroll") for (int k = 0; k < 2; ++k) dst[m][k] = *(const LAS bf16x8*)(lds + G_SA(b, h) + aoff + m * 2048 + k * 1024); } while (0)
#define G_LDB(dst, b, h) do { _Pragma("unroll") for (int n = 0; n < 2; ++n) _Pragma("unroll") for (int k = 0; k < 2; ++k) dst[n][k] = *(const LAS bf16x8*)(lds + G_SB(b, h) + boff + n * 2048 + k * 1024); } while (0)
#define G_MMA(ai, bj, At, Bt) do { __builtin_amdgcn_s_setprio(1); _Pragma("unroll") for (int m = 0; m < 4; ++m) _Pragma("unroll") for (int n = 0; n < 2; ++n) _Pragma("unroll") for (int k = 0; k < 2; ++k) \
        acc[ai][bj][m][n] = __builtin_amdgcn_mfma_f32_16x16x32_bf16(Bt[n][k], At[m][k], acc[ai][bj][m][n], 0, 0, 0); __builtin_amdgcn_s_setprio(0); } while (0)
#define G_WV(n) asm volatile("s_waitcnt vmcnt(" #n ")" ::: "memory")
#define G_WL(n) asm volatile("s_waitcnt lgkmcnt(" #n ")" ::: "memory")
#define G_BAR __builtin_amdgcn_s_barrier()
#define G_SCHED __builtin_amdgcn_sched_barrier(0)
    const char* XI = P.ws[P.rank] + WS_XI; const char* WI = P.ws[P.rank] + WS_WI;
    constexpr size_t kstep = 32768, hstep = 16384;
    u32 cur = tab16(P.tiles, gw);
    if (cur == 0xffffu) return;
    bool fresh = true;
    bf16x8 At[4][2], B0[2][2], B1[2][2];
    for (u32 ui = 0;; ++ui) {
        const u32 nxt = ui + 1 < (u32)ROUNDS ? tab16(P.tiles, (ui + 1) * NGEMM + gw) : 0xffffu;
        const bool has_next = nxt != 0xffffu, dist = (cur >> 4) < (u32)DP, chain = has_next && !dist;
        const char* cA = XI + (size_t)(cur & 15) * PANEL; const char* cB = WI + (size_t)(cur >> 4) * PANEL;
        if (fresh) {
            wait_panels(P, cur, call, wid, lane);
            STAMP(P, 8 + ui * 4);
            G_STAGE(G_SB(0, 0), cB); G_STAGE(G_SA(0, 0), cA); G_STAGE(G_SB(0, 1), cB + hstep); G_STAGE(G_SA(0, 1), cA + hstep);
            if (wr == 1) G_BAR;
            G_WV(4); G_BAR;
            G_STAGE(G_SB(1, 0), cB + kstep); G_STAGE(G_SA(1, 0), cA + kstep); G_STAGE(G_SB(1, 1), cB + hstep + kstep);
            G_WV(6); G_BAR;
        }
        f32x4 acc[2][2][4][2];
#pragma unroll
        for (int a = 0; a < 2; ++a)
#pragma unroll
            for (int b = 0; b < 2; ++b)
#pragma unroll
                for (int m = 0; m < 4; ++m)
#pragma unroll
                    for (int n = 0; n < 2; ++n) acc[a][b][m][n] = (f32x4){0.f, 0.f, 0.f, 0.f};
        const char* nA = chain ? XI + (size_t)(nxt & 15) * PANEL : cA; const char* nB = chain ? WI + (size_t)(nxt >> 4) * PANEL : cB;
        for (int t = 0; t < 64; t += 2) {
            const bool last = (t == 62);
            const char* a1 = cA + (size_t)(t + 1) * kstep;
            const char* a2 = last ? nA : cA + (size_t)(t + 2) * kstep; const char* b2 = last ? nB : cB + (size_t)(t + 2) * kstep;
            const char* a3 = a2 + kstep; const char* b3 = b2 + kstep;
            if (last && chain) wait_panels(P, nxt, call, wid, lane);
            G_LDB(B0, 0, 0); G_SCHED; G_LDA(At, 0, 0); G_STAGE(G_SA(1, 1), a1 + hstep);
            G_WL(8); G_BAR; G_WL(0); G_MMA(0, 0, At, B0); G_BAR; G_SCHED;
            G_LDB(B1, 0, 1); G_STAGE(G_SB(0, 0), b2);
            G_BAR; G_WL(0); G_MMA(0, 1, At, B1); G_BAR;
            G_LDA(At, 0, 1); G_STAGE(G_SA(0, 0), a2);
            G_BAR; G_WL(0); G_MMA(1, 0, At, B0); G_BAR; G_SCHED;
            G_STAGE(G_SB(0, 1), b2 + hstep);
            G_WV(6); G_BAR; G_MMA(1, 1, At, B1); G_BAR;
            G_LDB(B0, 1, 0); G_SCHED; G_LDA(At, 1, 0); G_STAGE(G_SA(0, 1), a2 + hstep);
            G_WL(8); G_BAR; G_WL(0); G_MMA(0, 0, At, B0); G_BAR; G_SCHED;
            G_LDB(B1, 1, 1); G_STAGE(G_SB(1, 0), b3);
            G_BAR; G_WL(0); G_MMA(0, 1, At, B1); G_BAR;
            G_LDA(At, 1, 1); G_STAGE(G_SA(1, 0), a3);
            G_BAR; G_WL(0); G_MMA(1, 0, At, B0); G_BAR; G_SCHED;
            G_STAGE(G_SB(1, 1), b3 + hstep);
            G_WV(6); G_BAR; G_MMA(1, 1, At, B1); G_BAR;
        }
        STAMP(P, 9 + ui * 4);
        u32 v[64];
        {
            int tz = threadIdx.x; asm volatile("" : "+v"(tz));
            const u32 ln = tz & 63;
            u32 bad; asm volatile("ds_read_b32 %0, %1\n\ts_waitcnt lgkmcnt(0)" : "=v"(bad) : "v"(CTL_BAD) : "memory");
            u16* o = P.out + (size_t)((cur & 15) * 256 + wr * 64 + (ln & 15)) * NN + global_panel(cur >> 4, P.rank) * 256 + wc * 64 + (ln >> 4) * 8;
#pragma unroll
            for (int p = 0; p < 16; ++p) {
                const int ai = p >> 3, m = (p >> 1) & 3, bj = p & 1;
                const f32x4 x0 = acc[ai][bj][m][0], x1 = acc[ai][bj][m][1];
                v[4 * p] = cvt_pk(x0[0], x0[1]); v[4 * p + 1] = cvt_pk(x0[2], x0[3]); v[4 * p + 2] = cvt_pk(x1[0], x1[1]); v[4 * p + 3] = cvt_pk(x1[2], x1[3]);
                if (bad) v[4 * p] = 0x7fc07fc0u;
                *(u32x4*)(o + (size_t)(ai * 128 + m * 16) * NN + bj * 32) = (u32x4){v[4 * p], v[4 * p + 1], v[4 * p + 2], v[4 * p + 3]};
            }
            if (!chain) {
                G_WV(0);
                if (wr == 0) G_BAR;
                BAR();
                if (dist) pack_group(P, v, GY + cur, 255u & ~(1u << P.rank), call, wid, ln);
                STAMP(P, 10 + ui * 4);
                if (!has_next) break;
            }
        }
        fresh = !chain; cur = nxt;
    }
}

__global__ void __launch_bounds__(512, 2) fused(const Params P) {
    extern __shared__ __attribute__((aligned(16))) unsigned char lds[];
    const u32 wid = rfl(threadIdx.x >> 6), lane = threadIdx.x & 63;
    if (threadIdx.x == 0) {
        const u32 c = __hip_atomic_fetch_add((u32*)(P.ws[P.rank] + WS_CALL + blockIdx.x * 128), 1u, __ATOMIC_RELAXED, __HIP_MEMORY_SCOPE_AGENT) + 1;
        lds32vw(CTL_CALL, c); lds32vw(CTL_NREADY, 0u); lds32vw(CTL_LANDED, 0u); lds32vw(CTL_ABORT, 0u); lds32vw(CTL_BAD, 0u);
    }
    if (wid == 1 && lane < 8) lds32vw(CTL_NEXT + lane * 4, lane);
    WAIT_LGKM0(); BAR();
    const u32 call = rfl(lds32v(CTL_CALL));
    if (blockIdx.x < (u32)NPULL) { puller(P, call, wid, lane); return; }
    STAMP(P, 0);
    for (u32 e = blockIdx.x - NPULL; e < 288u; e += NENC) { pack_inputs(P, tab16(P.enc, e), call, wid, lane); WAIT_LGKM0(); BAR(); STAMP(P, e >= (u32)NENC ? 2 : 1); }
    if (blockIdx.x >= (u32)GEMM0) gemm_role(P, lds, call, wid, lane);
}

static void build_tables(Params& P) {
    u16 order[NTILE]; int n = 0;
    u16* tiles = (u16*)P.tiles; u16* stream = (u16*)P.stream; u16* enc = (u16*)P.enc;
    for (int s = 0; s < 16; ++s) {
        for (int jj = 0; jj < s && jj < DP; ++jj) order[n++] = (u16)(s | jj << 4);
        if (s < DP) for (int pm = 0; pm <= s; ++pm) order[n++] = (u16)(pm | s << 4);
    }
    for (int jj = DP; jj < LP; ++jj) for (int pm = 0; pm < 16; ++pm) order[n++] = (u16)(pm | jj << 4);
    for (int i = 0; i < ROUNDS * NGEMM; ++i) tiles[i] = i < NTILE ? order[i] : (u16)0xffff;
    int k = 0;
    for (int s = 0; s < 16; ++s) {
        stream[k++] = (u16)(GX + 2 * s); stream[k++] = (u16)(GX + 2 * s + 1);
        if (s < DP) { const int j = P.rank * DP + s; stream[k++] = (u16)(GW + 2 * j); stream[k++] = (u16)(GW + 2 * j + 1); }
    }
    for (int j = NR * DP; j < 128; ++j) { stream[k++] = (u16)(GW + 2 * j); stream[k++] = (u16)(GW + 2 * j + 1); }
    for (int i = 0; i < 16 * DP; ++i) stream[k++] = (u16)(GY + order[i]);
    k = 0;
    for (int s = 0; s < 16; ++s) {
        enc[k++] = (u16)(GX + 2 * s); enc[k++] = (u16)(GX + 2 * s + 1);
        if (s < DP) for (int d = 1; d <= NR; ++d) { const int j = ((P.rank + d) & 7) * DP + s; enc[k++] = (u16)(GW + 2 * j); enc[k++] = (u16)(GW + 2 * j + 1); }
    }
    for (int j = NR * DP; j < 128; ++j) { enc[k++] = (u16)(GW + 2 * j); enc[k++] = (u16)(GW + 2 * j + 1); }
}
}

extern "C" void kernel_launch(int rank, int nranks, ncclComm_t comm, hipStream_t stream,
                              void* const* d_in, const int* in_sizes, int n_in,
                              void* d_out, int out_size, const int* in_dtypes, int out_dtype,
                              void* const* d_out_all, void* const* d_ws_all, size_t ws_bytes) {
    using namespace gar;
    if (nranks != NR || (unsigned)rank >= (unsigned)NR || n_in != 2 || ws_bytes < WS_END || in_sizes[0] != MM * KS || in_sizes[1] != KS * NN || out_size != MM * NN) {
        fprintf(stderr, "kernel_launch: unexpected shapes\n"); return; }
    Params P = {};
    for (int p = 0; p < NR; ++p) P.ws[p] = (char*)d_ws_all[p];
    P.x = (const u16*)d_in[0]; P.w = (const u16*)d_in[1]; P.out = (u16*)d_out; P.rank = rank; P.pad = 0;
    build_tables(P);
    HIP_CHECK(hipFuncSetAttribute((const void*)fused, hipFuncAttributeMaxDynamicSharedMemorySize, LDS_BYTES));
    hipLaunchKernelGGL(fused, dim3(NWG), dim3(512), LDS_BYTES, stream, P);
    HIP_CHECK(hipGetLastError());
}
```

```cpp
#include <hip/hip_runtime.h>
#include <rccl/rccl.h>
#include <cstdio>
#include <cstdint>
#pragma clang diagnostic ignored "-Wint-to-pointer-cast"

#define HIP_CHECK(expr) do { hipError_t _e = (expr); if (_e != hipSuccess) { \
    fprintf(stderr, "HIP error %s at %s:%d\n", hipGetErrorString(_e), __FILE__, __LINE__); \
    return; } } while (0)

namespace gar {
#define LAS __attribute__((address_space(3)))
#define GAS __attribute__((address_space(1)))
typedef unsigned u32;
typedef unsigned short u16;
typedef unsigned long long u64;
typedef unsigned u32x4 __attribute__((ext_vector_type(4)));
typedef unsigned u32x2 __attribute__((ext_vector_type(2)));
typedef short s16x4 __attribute__((ext_vector_type(4)));
typedef short bf16x8 __attribute__((ext_vector_type(8)));
typedef float f32x4 __attribute__((ext_vector_type(4)));

constexpr int NR = 8, MM = 4096, NN = 32768, KS = 512;
#define GAR_DP 11
constexpr int DP = GAR_DP;
constexpr int RP = 128 - NR * DP;
constexpr int LP = DP + RP;
constexpr int NWG = 256;
#define GAR_PW 9
#define GAR_WIN 6
constexpr int PW = GAR_PW;
constexpr int NPULL = 7 * PW;
constexpr int WIN = GAR_WIN;
constexpr int GEMM0 = 64;
constexpr int NGEMM = NWG - GEMM0;
static_assert(NPULL <= GEMM0, "roles");
#define GAR_RAWG 4
#define GAR_LEAD 32
#define GAR_HOLD 600
constexpr int RAWG = GAR_RAWG;
constexpr int LEAD = GAR_LEAD;
constexpr int URGENT = 16 - RAWG;
static_assert(8 * RAWG <= NPULL, "raw starters");
constexpr int NENC = NWG - NPULL;
constexpr int GX = 0, GW = 32, GY = 288;
constexpr int NGID = GY + 16 * DP;
constexpr int NS = 32 + 2 * DP + 2 * RP + 16 * DP;
constexpr int NTILE = 16 * LP;
constexpr int ROUNDS = (NTILE + NGEMM - 1) / NGEMM;
constexpr int NUMAX = 512;
constexpr int NPACK = 288 - RAWG;
constexpr u64 DEADLINE = 30000000ull;

constexpr size_t WS_CALL = 0;
constexpr size_t WS_TMO = 32768;
constexpr size_t WS_CNT = 36864;
constexpr size_t WS_CLAIM = 57344;
constexpr size_t WS_FLAG = 65536;
constexpr size_t FLAG_SRC = 544 * 32, FLAG_PAR = 8 * FLAG_SRC;
constexpr size_t WS_DBG = 524288;
constexpr size_t GRP_STRIDE = 131072;
constexpr size_t WS_STG = 2u << 20;
constexpr size_t STG_PAR = 544 * GRP_STRIDE;
constexpr size_t WS_XI = WS_STG + 2 * STG_PAR;
constexpr size_t PANEL = 64 * 32768;
constexpr size_t WS_WI = WS_XI + 16 * PANEL;
constexpr size_t WS_END = WS_WI + (size_t)LP * PANEL;
static_assert(WS_FLAG + 2 * FLAG_PAR <= WS_DBG && NGID <= 544, "layout");

constexpr u32 LDS_CTL = 131072;
constexpr u32 LDS_LUT = LDS_CTL + 512, LDS_HIST = LDS_CTL + 1024;
constexpr u32 LDS_BYTES = LDS_HIST + 4096;
constexpr u32 CTL_CALL = LDS_CTL, CTL_SIZE = LDS_CTL + 64  , CTL_BAD = LDS_CTL + 128;
constexpr u32 CTL_NUNIT = LDS_CTL + 192, CTL_LANDED = LDS_CTL + 256, CTL_NCHUNK = LDS_CTL + 320, CTL_NEXT = LDS_CTL + 384  ;
constexpr u32 E_L2 = 0, E_L3 = E_L2 + 4160, E_VAR = E_L3 + 2112, E_END = E_VAR + 2112;
static_assert(E_END <= 9216 && E_L3 % 16 == 0 && E_VAR % 16 == 0, "packer scratch");
constexpr u32 CAP2 = 4080, CAP3 = 2032, CAPV = 2032, FIXB = 10240, RAWB = 16384, RAWCODE = 255;
constexpr u32 RING = 65536, RMASK = RING - 1, ND = 6;
constexpr u32 D_SCR = RING, D_SCRB = 6400, D_L2 = 0, D_L3 = 4160;
constexpr u32 D_DESC = D_SCR + ND * D_SCRB, D_UNIT = D_DESC + NS * 64, DONE = 0x80000000u;
static_assert(D_L3 + 2112 <= D_SCRB && D_UNIT + NUMAX * 8 <= LDS_CTL && D_DESC % 16 == 0, "puller LDS");

struct Params {
    char* ws[NR];
    const u16* x; const u16* w; u16* out;
    int rank, nunits;
    u32 stream[NUMAX / 2];
    u32 enc[144];
    u32 tiles[ROUNDS * NGEMM / 2];
};
__device__ __forceinline__ u32 tab16(const u32* t, u32 i) { return (t[i >> 1] >> ((i & 1) * 16)) & 0xffffu; }
static_assert(sizeof(Params) <= 4096, "kernarg");

__device__ __forceinline__ u32 lds32(u32 a) { return *(LAS u32*)a; }
__device__ __forceinline__ u32 lds32v(u32 a) { return *(volatile LAS u32*)a; }
__device__ __forceinline__ void lds32vw(u32 a, u32 x) { *(volatile LAS u32*)a = x; }
__device__ __forceinline__ u32x2 lds64v(u32 a) { return *(volatile LAS u32x2*)a; }
__device__ __forceinline__ void lds64vw(u32 a, u32x2 x) { *(volatile LAS u32x2*)a = x; }
__device__ __forceinline__ u32x4 lds128(u32 a) { return *(LAS u32x4*)a; }
__device__ __forceinline__ void lds128w(u32 a, u32x4 x) { *(LAS u32x4*)a = x; }
__device__ __forceinline__ void lds_or(u32 a, u32 x) { __hip_atomic_fetch_or((LAS u32*)a, x, __ATOMIC_RELAXED, __HIP_MEMORY_SCOPE_WORKGROUP); }
__device__ __forceinline__ void lds_add(u32 a, u32 x) { __hip_atomic_fetch_add((LAS u32*)a, x, __ATOMIC_RELAXED, __HIP_MEMORY_SCOPE_WORKGROUP); }
__device__ __forceinline__ u32 rfl(u32 x) { return (u32)__builtin_amdgcn_readfirstlane((int)x); }
#define CBAR() asm volatile("" ::: "memory")
#define WAIT_LGKM0() asm volatile("s_waitcnt lgkmcnt(0)" ::: "memory")
#define WAIT_VM(n) asm volatile("s_waitcnt vmcnt(%0)" :: "i"(n) : "memory")
#define BAR() do { CBAR(); __builtin_amdgcn_s_barrier(); CBAR(); } while (0)
__device__ __forceinline__ void st_sys(void* p, u32x4 x) { asm volatile("global_store_dwordx4 %0, %1, off sc0 sc1\n\ts_nop 1" :: "v"(p), "v"(x) : "memory"); }
__device__ __forceinline__ void st_dev(void* p, u32x4 x) { asm volatile("global_store_dwordx4 %0, %1, off sc1\n\ts_nop 1" :: "v"(p), "v"(x) : "memory"); }
#define GAR_OST 3
#define GAR_GST 3
template <int MODE> __device__ __forceinline__ void st_out(void* p, u32x4 x) {
    if (MODE == 0) *(u32x4*)p = x;
    else if (MODE == 1) __builtin_nontemporal_store(x, (u32x4*)p);
    else if (MODE == 2) st_dev(p, x);
    else asm volatile("global_store_dwordx4 %0, %1, off sc1 nt\n\ts_nop 1" :: "v"(p), "v"(x) : "memory");
}
__device__ __forceinline__ void glds_sys(const void* g, u32 dst) {
    u32 keep;
    asm volatile("s_mov_b32 %0, m0\n\ts_mov_b32 m0, %2\n\ts_nop 0\n\tglobal_load_lds_dwordx4 %1, off sc0 sc1\n\ts_mov_b32 m0, %0" : "=&s"(keep) : "v"(g), "s"(dst) : "memory");
}
__device__ __forceinline__ void glds_plain(const void* g, u32 dst) {
    u32 keep;
    asm volatile("s_mov_b32 %0, m0\n\ts_mov_b32 m0, %2\n\ts_nop 0\n\tglobal_load_lds_dwordx4 %1, off\n\ts_mov_b32 m0, %0" : "=&s"(keep) : "v"(g), "s"(dst) : "memory");
}
__device__ __forceinline__ u64 now() { return __builtin_amdgcn_s_memrealtime(); }
template <int CTRL, int RM, bool BC> __device__ __forceinline__ u32 dpp(u32 v) { return (u32)__builtin_amdgcn_update_dpp(0, (int)v, CTRL, RM, 0xf, BC); }
__device__ __forceinline__ u32 scan_incl(u32 v) {
    v += dpp<0x111, 0xf, true>(v); v += dpp<0x112, 0xf, true>(v); v += dpp<0x114, 0xf, true>(v); v += dpp<0x118, 0xf, true>(v);
    v += dpp<0x142, 0xa, false>(v); v += dpp<0x143, 0xc, false>(v); return v;
}
__device__ __forceinline__ u32 umax(u32 a, u32 b) { return a > b ? a : b; }
__device__ __forceinline__ u32 umin(u32 a, u32 b) { return a < b ? a : b; }
__device__ __forceinline__ u32 wave_max(u32 v) {
    v = umax(v, dpp<0x111, 0xf, true>(v)); v = umax(v, dpp<0x112, 0xf, true>(v)); v = umax(v, dpp<0x114, 0xf, true>(v)); v = umax(v, dpp<0x118, 0xf, true>(v));
    v = umax(v, dpp<0x142, 0xa, false>(v)); v = umax(v, dpp<0x143, 0xc, false>(v)); return (u32)__builtin_amdgcn_readlane((int)v, 63);
}
__device__ __forceinline__ u32 popc(u32 x) { return (u32)__builtin_popcount(x); }
__device__ __forceinline__ u32 perm(u32 hi, u32 lo, u32 sel) { return __builtin_amdgcn_perm(hi, lo, sel); }
__device__ __forceinline__ u32 swz(u32 ob) { return ob ^ (((ob >> 9) & 1) << 5); }

#define STAMP(P, slot) do {} while (0)
#define TICK(t) do {} while (0)
#define ACC(sum, t) do {} while (0)
#define DUMP(P, slot, val) do {} while (0)

#define CSTAMP(i) do {} while (0)
#define CS_ARG
#define CS_DECL
#define CS_PASS

__device__ __forceinline__ u32 nzb(u32 z) { return ((z & 0x7f7f7f7fu) + 0x7f7f7f7fu) | z; }
typedef unsigned short u16x2 __attribute__((ext_vector_type(2)));
struct Classifier {
    u32 s1, s2, s3, nb, tlo, thi; bool win;
    __device__ __forceinline__ void init(u32 t) {
        const u32 a = (t >> 8) & 255u, b = (t >> 16) & 255u, c = t >> 24, lo = umin(a, umin(b, c));
        win = umax(a, umax(b, c)) - lo <= 6u;
        const u64 l = (1ull << (((a - lo) & 7u) * 8)) | (2ull << (((b - lo) & 7u) * 8)) | (3ull << (((c - lo) & 7u) * 8));
        tlo = (u32)l; thi = (u32)(l >> 32); nb = ((0u - lo) & 255u) * 0x01000100u;
        s1 = a * 0x01010101u; s2 = b * 0x01010101u; s3 = c * 0x01010101u;
    }
    __device__ __forceinline__ u32 any(u32 e) const {
        const u32 a1 = nzb(e ^ s1), a2 = nzb(e ^ s2), a3 = nzb(e ^ s3);
        return ((~(a1 & a3) >> 7) & 0x01010101u) | ((~(a2 & a3) >> 6) & 0x02020202u);
    }
    __device__ __forceinline__ u32 near(u32 e) const {
        const u16x2 x = __builtin_bit_cast(u16x2, e), n = __builtin_bit_cast(u16x2, nb), top = {0x07ff, 0x07ff};
        u32 sh; asm("v_pk_mad_u16 %0, %1, %2, %3" : "=v"(sh) : "v"(e), "v"(0x01000100u), "s"(nb));
        const u16x2 hi = __builtin_elementwise_min((u16x2)(x + n), top), lo = __builtin_elementwise_min(__builtin_bit_cast(u16x2, sh), top);
        return perm(thi, tlo, perm(__builtin_bit_cast(u32, hi), __builtin_bit_cast(u32, lo), 0x07030501u));
    }
    template <int N> __device__ __forceinline__ void run(const u32* e, u32* c) const {
        if (win) {
#pragma unroll
            for (int i = 0; i < N; ++i) c[i] = near(e[i]);
        } else {
#pragma unroll
            for (int i = 0; i < N; ++i) c[i] = any(e[i]);
        }
    }
};
__device__ __forceinline__ u32 escbits(u32 c) { return ~(c | (c >> 1)) & 0x55555555u; }
struct Appender {
    u32 lo, fill, addr;
    __device__ __forceinline__ void init(u32 off) { lo = 0; fill = (off & 3u) * 8u; addr = off & ~3u; }
    __device__ __forceinline__ void push(u32 k, u32 mk) {
        const u64 t = (u64)k << fill;
        lo |= (u32)t; fill += popc(mk);
        if (fill >= 32u) { lds_or(addr, lo); lo = (u32)(t >> 32); addr += 4; }
        fill &= 31u;
    }
    __device__ __forceinline__ void flush() { if (lo) lds_or(addr, lo); }
};
__device__ __forceinline__ u32 escmask(u32 c) { return perm(0u, 0xffu, c); }
__device__ __forceinline__ u32 selector(u32 mk) { return lds32(__builtin_amdgcn_udot4(mk & 0x20100804u, 0x01010101u, LDS_LUT, false)); }
template <int R> __device__ __forceinline__ u32 pack_level(u32 src, u32 n, u32 t, u32 cdst, u32 edst, u32 cap, u32 lane) {
    Classifier K; K.init(t);
    if (lane < 16) *(LAS unsigned char*)(src + n + lane) = (unsigned char)K.s1;
    u32 ne = 0;
    for (u32 r = 0; r * 1024 < n; r += R) {
        CBAR();
        u32 q[4 * R], c[4 * R], m[4 * R], l[4 * R], cnt = 0;
#pragma unroll
        for (int i = 0; i < R; ++i) { const u32 p0 = (r + i) * 1024 + lane * 16; const u32x4 x = lds128(src + p0); const bool act = p0 < n;
            q[4 * i] = act ? x.x : K.s1; q[4 * i + 1] = act ? x.y : K.s1; q[4 * i + 2] = act ? x.z : K.s1; q[4 * i + 3] = act ? x.w : K.s1; }
        K.run<4 * R>(q, c);
#pragma unroll
        for (int i = 0; i < 4 * R; ++i) { m[i] = escmask(c[i]); l[i] = selector(m[i]); }
#pragma unroll
        for (int i = 0; i < R; ++i) cnt |= ((popc(m[4 * i]) + popc(m[4 * i + 1]) + popc(m[4 * i + 2]) + popc(m[4 * i + 3])) >> 3) << (16 * i);
        const u32 inc = scan_incl(cnt), tot = (u32)__builtin_amdgcn_readlane((int)inc, 63), exc = inc - cnt;
        if (ne + (tot & 0xffffu) + (tot >> 16) > cap) return ~0u;
#pragma unroll
        for (int i = 0; i < R; ++i) {
            Appender A; A.init(edst + ne + (i ? (tot & 0xffffu) + (exc >> 16) : (exc & 0xffffu)));
#pragma unroll
            for (int j = 0; j < 4; ++j) A.push(perm(0u, q[4 * i + j], l[4 * i + j]), m[4 * i + j]);
            A.flush();
            if ((r + i) * 1024 + lane * 16 < n) *(LAS u32*)(cdst + ((r + i) * 64 + lane) * 4) = c[4 * i] | (c[4 * i + 1] << 2) | (c[4 * i + 2] << 4) | (c[4 * i + 3] << 6);
        }
        ne += (tot & 0xffffu) + (tot >> 16);
    }
    CBAR();
    return ne;
}
template <int N> __device__ __forceinline__ u32 rank_ror(u32 key) { return (dpp<0x120 + N, 0xf, true>(key) > key ? 1u : 0u) + rank_ror<N - 1>(key); }
template <> __device__ __forceinline__ u32 rank_ror<0>(u32) { return 0u; }
__device__ __forceinline__ u32 row_or(u32 x) { x |= dpp<0x128, 0xf, true>(x); x |= dpp<0x124, 0xf, true>(x); x |= dpp<0x122, 0xf, true>(x); x |= dpp<0x121, 0xf, true>(x); return rfl(x); }
__device__ __forceinline__ void split(const u32 (&v)[64], u32 (&Mr)[32], u32 (&Er)[32]) {
#pragma unroll
    for (int g = 0; g < 32; ++g) {
        const u32 a = __builtin_amdgcn_alignbit(v[2 * g], v[2 * g], 31), b = __builtin_amdgcn_alignbit(v[2 * g + 1], v[2 * g + 1], 31);
        Er[g] = perm(b, a, 0x07050301u); Mr[g] = perm(b, a, 0x06040200u);
    }
}
__device__ __forceinline__ u32 pack_chunk(const u32 (&v)[64], u32 (&Mr)[32], u32 (&Er)[32], u32 (&Cr)[8], u32 scr, u32 lane CS_ARG) {
    CBAR(); CSTAMP(0);
#pragma unroll
    for (int i = 0; i < 4; ++i) lds_add(LDS_HIST + (((v[i * 17 + 5] >> ((i & 1) * 16 + 3)) & 0xff0u) | ((lane & 3) * 4)), 1u);
#pragma unroll
    for (int j = 0; j < 9; ++j) lds128w(scr + j * 1024 + lane * 16, (u32x4){0u, 0u, 0u, 0u});
    WAIT_LGKM0(); BAR();
    u32 key = 0;
#pragma unroll
    for (int i = 0; i < 4; ++i) { const u32 bin = i * 64 + lane; const u32x4 h = lds128(LDS_HIST + bin * 16); key = umax(key, ((h.x + h.y + h.z + h.w) << 8) | bin); }
    split(v, Mr, Er);
    CSTAMP(1);
    { const auto r = __builtin_amdgcn_permlane32_swap(key, key, false, false); key = umax(r[0], r[1]); }
    { const auto r = __builtin_amdgcn_permlane16_swap(key, key, false, false); key = umax(r[0], r[1]); }
    const u32 rank = rank_ror<15>(key), lev = (rank * 11u) >> 5, sym = (key & 255u) << (8 * (rank - 3 * lev + 1));
    const u32 T[3] = {row_or(lev == 0 ? sym : 0u), row_or(lev == 1 ? sym : 0u), row_or(lev == 2 ? sym : 0u)};
    CSTAMP(3);
    u32 cnt = 0, mk[32], sl[32];
    {
        Classifier K; K.init(T[0]);
        u32 c[32]; K.run<32>(Er, c);
#pragma unroll
        for (int g = 0; g < 32; ++g) { mk[g] = escmask(c[g]); sl[g] = selector(mk[g]); cnt += popc(mk[g]); }
#pragma unroll
        for (int k = 0; k < 8; ++k) Cr[k] = c[4 * k] | (c[4 * k + 1] << 2) | (c[4 * k + 2] << 4) | (c[4 * k + 3] << 6);
    }
    cnt >>= 3;
    const u32 inc = scan_incl(cnt), n2 = (u32)__builtin_amdgcn_readlane((int)inc, 63);
    CSTAMP(4);
    if (n2 > CAP2) return RAWCODE;
    Appender A; A.init(scr + E_L2 + inc - cnt);
#pragma unroll
    for (int g = 0; g < 32; ++g) A.push(perm(0u, Er[g], sl[g]), mk[g]);
    A.flush();
    CBAR(); CSTAMP(5);
    const u32 d2 = (n2 + 15) >> 4;
    const u32 n3 = pack_level<2>(scr + E_L2, n2, T[1], scr + E_VAR + 12, scr + E_L3, CAP3, lane);
    CSTAMP(6);
    if (n3 == ~0u) return RAWCODE;
    const u32 d3 = (n3 + 15) >> 4, b4 = 12 + 4 * (d2 + d3);
    const u32 n4 = pack_level<1>(scr + E_L3, n3, T[2], scr + E_VAR + 12 + 4 * d2, scr + E_VAR + b4, CAPV - b4, lane);
    if (n4 == ~0u) return RAWCODE;
    if (lane == 0) { *(LAS u32*)(scr + E_VAR) = T[0] | d2; *(LAS u32*)(scr + E_VAR + 4) = T[1] | d3; *(LAS u32*)(scr + E_VAR + 8) = T[2]; }
    CBAR();
    return (b4 + n4 + 15) >> 4;
}
__device__ __forceinline__ void pack_init(u32 tid) {
    if (tid < 256) lds128w(LDS_HIST + tid * 16, (u32x4){0u, 0u, 0u, 0u});
    else if (tid < 272) { u32 sel = 0x0c0c0c0cu, k = 0;
#pragma unroll
        for (u32 b = 0; b < 4; ++b) if ((tid >> b) & 1) { sel = (sel & ~(255u << k)) | (b << k); k += 8; }
        *(LAS u32*)(LDS_LUT + (tid & 15) * 4) = sel; }
}
__device__ __forceinline__ u32 chunk_bytes(u32 code) { return code == RAWCODE ? RAWB : FIXB + 16 * code; }

__device__ __forceinline__ void pack_store(char* dst, u32 code, const u32 (&Mr)[32], const u32 (&Er)[32], const u32 (&Cr)[8], u32 scr, u32 lane) {
#pragma unroll
    for (int j = 0; j < 8; ++j) st_sys(dst + j * 1024, (u32x4){Mr[4 * j], Mr[4 * j + 1], Mr[4 * j + 2], Mr[4 * j + 3]});
    if (code == RAWCODE) {
#pragma unroll
        for (int j = 0; j < 8; ++j) st_sys(dst + 8192 + j * 1024, (u32x4){Er[4 * j], Er[4 * j + 1], Er[4 * j + 2], Er[4 * j + 3]});
    } else {
        st_sys(dst + 8192, (u32x4){Cr[0], Cr[1], Cr[2], Cr[3]}); st_sys(dst + 9216, (u32x4){Cr[4], Cr[5], Cr[6], Cr[7]});
        if (lane < code) st_sys(dst + FIXB, lds128(scr + E_VAR + lane * 16));
        if (lane + 64 < code) st_sys(dst + FIXB + 1024, lds128(scr + E_VAR + 1024 + lane * 16));
    }
}
__device__ __forceinline__ void raise_flag(const Params& P, u32 q, u32 gid, u32 w, u32 call, u32 code) {
    char* wq = P.ws[0];
#pragma unroll
    for (u32 j = 1; j < NR; ++j) wq = q == j ? P.ws[j] : wq;
    u32* f = (u32*)(wq + WS_FLAG + (call & 1) * FLAG_PAR + (size_t)P.rank * FLAG_SRC + gid * 32) + w;
    __hip_atomic_store(f, (call << 8) | code, __ATOMIC_RELAXED, __HIP_MEMORY_SCOPE_SYSTEM);
}
__device__ __forceinline__ void pack_group(const Params& P, const u32 (&v)[64], u32 gid, u32 peers, u32 call, u32 wid, u32 lane) {
    u32 Mr[32], Er[32], Cr[8];
    const u32 scr = wid * 16384;
    CS_DECL
    if (gid < 64u) STAMP(P, 40 + wid);
    const u32 code = rfl(pack_chunk(v, Mr, Er, Cr, scr, lane CS_PASS));
    if (gid < 64u) STAMP(P, 48 + wid);
    if (lane == 0) lds32vw(CTL_SIZE + wid * 4, code);
    WAIT_LGKM0(); BAR();
    const u32 mine = lds32v(CTL_SIZE + (lane & 7) * 4);
    if (lane < 32) lds128w(LDS_HIST + wid * 512 + lane * 16, (u32x4){0u, 0u, 0u, 0u});
    u32 off = 0;
#pragma unroll
    for (u32 j = 0; j < 7; ++j) { const u32 c = (u32)__builtin_amdgcn_readlane((int)mine, j); if (j < wid) off += chunk_bytes(c); }
    char* dst = P.ws[P.rank] + WS_STG + (call & 1) * STG_PAR + (size_t)gid * GRP_STRIDE + off + lane * 16;
    pack_store(dst, code, Mr, Er, Cr, scr, lane);
    if (gid < 64u) STAMP(P, 56 + wid);
    WAIT_VM(0);
    if (gid < 64u) STAMP(P, 64 + wid);
    WAIT_LGKM0(); BAR();
    if (wid == 0 && ((peers >> (lane >> 3)) & 1)) raise_flag(P, lane >> 3, gid, lane & 7, call, mine);
}

#define GAR_WINMODE 1
template <bool WRAP> struct Unp {
    u32 sel[4], lo[4], hi[4], of[4];
    __device__ __forceinline__ void fetch(u32 c, u32& off) {
#pragma unroll
        for (int j = 0; j < 4; ++j) {
            const u32 cj = (c >> (2 * j)) & 0x03030303u, mk = escmask(cj), m2 = mk & 0x02020202u;
            if (WRAP || GAR_WINMODE != 2) { const u32 a = off & ~3u; lo[j] = lds32(WRAP ? (a & RMASK) : a); hi[j] = lds32(WRAP ? ((a + 4) & RMASK) : a + 4); of[j] = off; }
            else lo[j] = lds32(off);
            u32 p; asm("v_mad_u32_u24 %0, %1, %2, %3" : "=v"(p) : "v"(m2), "s"(0x808080u), "v"(0x04040404u));
            sel[j] = (mk & p) | cj;
            off += popc(m2);
        }
    }
    __device__ __forceinline__ u32 finish(int j, u32 t) const {
        u32 w;
        if (WRAP || GAR_WINMODE == 0) w = __builtin_amdgcn_alignbyte(hi[j], lo[j], of[j] & 3u);
        else if (GAR_WINMODE == 1) w = __builtin_amdgcn_alignbyte(hi[j], lo[j], of[j]);
        else w = lo[j];
        return perm(w, t, sel[j]);
    }
    __device__ __forceinline__ u32x4 finish4(u32 t) const { return (u32x4){finish(0, t), finish(1, t), finish(2, t), finish(3, t)}; }
};
#define SCHED() __builtin_amdgcn_sched_barrier(0)
template <bool WRAP> __device__ __forceinline__ u32 unpack_round(u32 c, u32 t, u32 src, u32 dst) {
    const u32 cnt = popc(escbits(c)), inc = scan_incl(cnt);
    u32 off = src + inc - cnt;
    Unp<WRAP> u; u.fetch(c, off); SCHED();
    lds128w(dst, u.finish4(t));
    return (u32)__builtin_amdgcn_readlane((int)inc, 63);
}
__device__ __forceinline__ void unpack_chunk(u32 S, u32 code, u32 scr, u32 lane, u32 rel_addr, u32 rel_val, u32 (&Mr)[32], u32 (&Er)[32] CS_ARG) {
    const u32 a0 = S + lane * 16;
#pragma unroll
    for (int j = 0; j < 8; ++j) { const u32x4 q = lds128((a0 + j * 1024) & RMASK); Mr[4 * j] = q.x; Mr[4 * j + 1] = q.y; Mr[4 * j + 2] = q.z; Mr[4 * j + 3] = q.w; }
    if (code == RAWCODE) {
#pragma unroll
        for (int j = 0; j < 8; ++j) { const u32x4 q = lds128((a0 + 8192 + j * 1024) & RMASK); Er[4 * j] = q.x; Er[4 * j + 1] = q.y; Er[4 * j + 2] = q.z; Er[4 * j + 3] = q.w; }
        WAIT_LGKM0(); lds32vw(rel_addr, rel_val); CBAR();
        return;
    }
    const u32x4 ca = lds128((a0 + 8192) & RMASK), cb = lds128((a0 + 9216) & RMASK);
    const u32 V = S + FIXB;
    const u32x4 hd = lds128(V & RMASK);
    const u32 t1 = rfl(hd.x), t2 = rfl(hd.y), t3 = rfl(hd.z), d2 = t1 & 255u, d3 = t2 & 127u;
    u32 c2[4], c3[2];
#pragma unroll
    for (int r = 0; r < 4; ++r) { const u32 i = r * 64 + lane; const u32 x = lds32((V + 12 + 4 * i) & RMASK); c2[r] = i < d2 ? x : 0x55555555u; }
#pragma unroll
    for (int r = 0; r < 2; ++r) { const u32 i = r * 64 + lane; const u32 x = lds32((V + 12 + 4 * (d2 + i)) & RMASK); c3[r] = i < d3 ? x : 0x55555555u; }
    CSTAMP(1);
    u32 s4 = V + 12 + 4 * (d2 + d3);
    s4 += unpack_round<true>(c3[0], t3, s4, scr + D_L3 + lane * 16);
    if (d3 > 64) unpack_round<true>(c3[1], t3, s4, scr + D_L3 + 1024 + lane * 16);
    WAIT_LGKM0(); lds32vw(rel_addr, rel_val); CBAR();
    CSTAMP(2);
    const u32 C[8] = {ca.x, ca.y, ca.z, ca.w, cb.x, cb.y, cb.z, cb.w};
    u32 cnt = 0;
#pragma unroll
    for (int k = 0; k < 8; ++k) cnt += popc(escbits(C[k]));
    {
        const u32 n0 = popc(escbits(c2[0])), n1 = popc(escbits(c2[1]));
        const u32 i01 = scan_incl(n0 | (n1 << 16)), tot = (u32)__builtin_amdgcn_readlane((int)i01, 63);
        cnt = scan_incl(cnt) - cnt;
        u32 o0 = scr + D_L3 + (i01 & 0xffffu) - n0, o1 = scr + D_L3 + (tot & 0xffffu) + (i01 >> 16) - n1;
        Unp<false> u0, u1; u0.fetch(c2[0], o0); u1.fetch(c2[1], o1); SCHED();
        lds128w(scr + D_L2 + lane * 16, u0.finish4(t2)); lds128w(scr + D_L2 + 1024 + lane * 16, u1.finish4(t2));
        if (d2 > 128) {
            u32 s3 = scr + D_L3 + (tot & 0xffffu) + (tot >> 16);
            s3 += unpack_round<false>(c2[2], t2, s3, scr + D_L2 + 2048 + lane * 16);
            if (d2 > 192) unpack_round<false>(c2[3], t2, s3, scr + D_L2 + 3072 + lane * 16);
        }
    }
    CBAR(); CSTAMP(3);
    u32 off = scr + D_L2 + cnt;
    Unp<false> u[8];
    u[0].fetch(C[0], off); u[1].fetch(C[1], off);
#pragma unroll
    for (int k = 0; k < 8; ++k) {
        SCHED();
        if (k + 2 < 8) u[k + 2].fetch(C[k + 2], off);
        SCHED();
#pragma unroll
        for (int j = 0; j < 4; ++j) Er[4 * k + j] = u[k].finish(j, t1);
    }
    CBAR();
}
__device__ __forceinline__ u32x4 piece(const u32 (&Mr)[32], const u32 (&Er)[32], int p) {
    const u32 a = perm(Er[2 * p], Mr[2 * p], 0x05010400u), b = perm(Er[2 * p], Mr[2 * p], 0x07030602u);
    const u32 c = perm(Er[2 * p + 1], Mr[2 * p + 1], 0x05010400u), d = perm(Er[2 * p + 1], Mr[2 * p + 1], 0x07030602u);
    return (u32x4){__builtin_amdgcn_alignbit(a, a, 1), __builtin_amdgcn_alignbit(b, b, 1), __builtin_amdgcn_alignbit(c, c, 1), __builtin_amdgcn_alignbit(d, d, 1)};
}

__device__ __forceinline__ u32 x_piece_off(u32 half, int p) { return half * 16384 + p * 1024; }
__device__ __forceinline__ u32 w_piece_off(u32 h, int p) {
    const u32 g32 = p >> 2, n = (p >> 1) & 1, kb = p & 1;
    return (g32 & 1) * 16384 + ((2 * (2 * h + (g32 >> 1)) + n) * 2 + kb) * 1024;
}
__device__ __forceinline__ u16* y_base(const Params& P, u32 pm, u32 j, u32 w, u32 lane) {
    return P.out + (size_t)(pm * 256 + (w >> 2) * 64 + (lane & 7)) * NN + j * 256 + (w & 3) * 64 + (lane & 8) * 4 + (lane >> 4) * 8;
}
__device__ __forceinline__ size_t y_piece_off(int p) { return (size_t)((p >> 3) * 128 + (p & 7) * 8) * NN; }
__device__ __forceinline__ u32 local_panel(u32 j, u32 rank) { return j >= (u32)(NR * DP) ? j - (NR - 1) * DP : j - rank * DP; }
__device__ __forceinline__ u32 global_panel(u32 jj, u32 rank) { return jj >= (u32)DP ? jj + (NR - 1) * DP : jj + rank * DP; }
__device__ __forceinline__ u32* counter(const Params& P, u32 i) { return (u32*)(P.ws[P.rank] + WS_CNT + i * 128); }
__device__ __forceinline__ u32* tmo_word(const Params& P) { return (u32*)(P.ws[P.rank] + WS_TMO); }
__device__ __forceinline__ bool expired(const Params& P, u64 t0, u32 call) {
    if (__hip_atomic_load(tmo_word(P), __ATOMIC_RELAXED, __HIP_MEMORY_SCOPE_AGENT) == call || now() - t0 > DEADLINE) {
        __hip_atomic_store(tmo_word(P), call, __ATOMIC_RELAXED, __HIP_MEMORY_SCOPE_AGENT); return true; }
    return false;
}

__device__ __forceinline__ void load_x(const Params& P, u32 gid, u32 w, u32 lane, u32 (&v)[64]) {
    const u16* s = P.x + (size_t)((gid >> 1) * 256 + (w & 1) * 128 + (lane & 15)) * KS + ((gid & 1) * 4 + (w >> 1)) * 64 + (lane >> 4) * 8;
#pragma unroll
    for (int p = 0; p < 16; ++p) { const u32x4 t = *(const u32x4*)(s + (p >> 1) * 16 * KS + (p & 1) * 32); v[4 * p] = t.x; v[4 * p + 1] = t.y; v[4 * p + 2] = t.z; v[4 * p + 3] = t.w; }
}
__device__ __forceinline__ void own_x(const Params& P, u32 gid, u32 w, u32 lane, const u32 (&v)[64]) {
    char* o = P.ws[P.rank] + WS_XI + (size_t)(gid >> 1) * PANEL + (size_t)(P.rank * 8 + (gid & 1) * 4 + (w >> 1)) * 32768 + swz((lane & 15) * 64 + (lane >> 4) * 16);
#pragma unroll
    for (int p = 0; p < 16; ++p) st_dev(o + x_piece_off(w & 1, p), (u32x4){v[4 * p], v[4 * p + 1], v[4 * p + 2], v[4 * p + 3]});
}
__device__ __forceinline__ void puller(const Params& P, u32 call, u32 wid, u32 lane) {
    const u32 q = (P.rank + 1 + blockIdx.x % 7) & 7;
    if (wid == 0) {
        __builtin_amdgcn_s_setprio(3);
        const char* base = P.ws[q] + WS_STG + (call & 1) * STG_PAR;
        const char* dummy = P.ws[P.rank] + WS_TMO + 1024 + lane * 16;
        const u32 voff = lane * 16;
        u32 pos = 0, pub = 0, limit = RING / 1024;
        u64 tk = 0, c_full = 0, c_idle = 0, n_full = 0, n_ref = 0; (void)tk; (void)c_full; (void)c_idle; (void)n_full; (void)n_ref;
        for (u32 ui = 0;; ++ui) {
            u32 nu = rfl(lds32v(CTL_NUNIT));
            if ((nu & ~DONE) <= ui) {
                TICK(tk);
                WAIT_VM(0); lds32vw(CTL_LANDED, pos); pub = pos;
                while (nu <= ui) { __builtin_amdgcn_s_sleep(1); nu = rfl(lds32v(CTL_NUNIT)); }
                if ((nu & ~DONE) <= ui) break;
                ACC(c_idle, tk);
            }
            CBAR();
            const u32x2 un = lds64v(D_UNIT + ui * 8);
            const u32 ut = rfl(un.y), total = ut & 0x3ffffu, lastc = ut >> 18;
            const char* src = base + rfl(un.x);
            for (u32 o = 0; o < total; o += 4096) {
                if (pos + 4 > limit) {
                    bool drained = false; TICK(tk); ++n_ref;
                    for (;;) {
                        const u32 nx = lds32v(CTL_NEXT + (lane & 7) * 4);
                        u32 f = lastc;
#pragma unroll
                        for (u32 j = 0; j < ND; ++j) f = umin(f, (u32)__builtin_amdgcn_readlane((int)nx, j));
                        limit = (rfl(lds32v(D_DESC + f * 8)) >> 14) + RING / 1024;
                        if (pos + 4 <= limit) break;
                        if (!drained) { WAIT_VM(0); lds32vw(CTL_LANDED, pos); pub = pos; drained = true; }
                        __builtin_amdgcn_s_sleep(1);
                    }
                    ACC(c_full, tk); n_full += drained;
                }
                const u32 dst = (pos * 1024) & RMASK;
                if (o + 4096 <= total) {
                    u32 keep;
                    asm volatile("s_mov_b32 %0, m0\n\ts_mov_b32 m0, %3\n\ts_nop 4\n\t"
                                 "global_load_lds_dwordx4 %1, %2 sc0 sc1\n\ts_waitcnt vmcnt(%4)\n\t"
                                 "global_load_lds_dwordx4 %1, %2 offset:1024 sc0 sc1\n\ts_waitcnt vmcnt(%4)\n\t"
                                 "global_load_lds_dwordx4 %1, %2 offset:2048 sc0 sc1\n\ts_waitcnt vmcnt(%4)\n\t"
                                 "global_load_lds_dwordx4 %1, %2 offset:3072 sc0 sc1\n\ts_waitcnt vmcnt(%4)\n\t"
                                 "s_mov_b32 m0, %0" : "=&s"(keep) : "v"(voff), "s"(src + o), "s"(dst), "i"(WIN) : "memory");
                } else {
#pragma unroll
                    for (u32 j = 0; j < 4; ++j) { const u32 b = o + j * 1024 + voff; glds_sys(b < total ? src + b : dummy, dst + j * 1024); WAIT_VM(WIN); }
                }
                pos += 4;
                if (pos > pub + WIN) { pub = pos - WIN; lds32vw(CTL_LANDED, pub); }
            }
            STAMP(P, 128 + umin(ui, 127)); DUMP(P, 64 + umin(ui, 127), total);
        }
        DUMP(P, 0, c_full); DUMP(P, 1, n_full); DUMP(P, 2, n_ref); DUMP(P, 3, c_idle);
    } else if (wid == 1) {
        u32 basek = 0, nch = 0, ui = 0;
        u32* ticket = (u32*)(P.ws[P.rank] + WS_CLAIM + q * 128);
        for (;; ++ui) {
            while (rfl(lds32v(CTL_LANDED)) + LEAD < basek) __builtin_amdgcn_s_sleep(4);
            u32 idx = 0;
            if (lane == 0) idx = __hip_atomic_fetch_add(ticket, 1u, __ATOMIC_RELAXED, __HIP_MEMORY_SCOPE_AGENT);
            idx = rfl(idx) - (call - 1) * (u32)(P.nunits + PW);
            if (idx >= (u32)P.nunits) break;
            const u32 e = tab16(P.stream, idx), gid = e & 1023u, part = e >> 10, lv = 31 - __builtin_clz(part), n = 8u >> lv, first = (part - (1u << lv)) * n;
            const u32* f = (const u32*)(P.ws[P.rank] + WS_FLAG + (call & 1) * FLAG_PAR + (size_t)q * FLAG_SRC + gid * 32) + (lane & 7);
            const u64 t0 = now(); u32 val; bool dead = false;
            for (;;) {
                val = __hip_atomic_load(f, __ATOMIC_RELAXED, __HIP_MEMORY_SCOPE_SYSTEM);
                if (__all((val >> 8) == (call & 0xffffffu) || (lane & 7) >= first + n)) break;
                if (__any(expired(P, t0, call))) { dead = true; break; }
                __builtin_amdgcn_s_sleep(2);
            }
            if (dead) {
                const u32 t = gid - GY;
                if (lane == 0 && t < (u32)(16 * DP)) P.out[(size_t)((t & 15) * 256 + (first >> 2) * 64) * NN + (q * DP + (t >> 4)) * 256 + (first & 3) * 64] = 0x7fc0;
                break;
            }
            u32 code = val & 255u; if (code > 127u) code = RAWCODE;
            const u32 sz = lane < first + n ? chunk_bytes(code) : 0u;
            u32 inc = sz; inc += dpp<0x111, 0xf, true>(inc); inc += dpp<0x112, 0xf, true>(inc); inc += dpp<0x114, 0xf, true>(inc);
            const u32 a0 = (u32)__builtin_amdgcn_readlane((int)(inc - sz), (int)first) & ~127u, total = (u32)__builtin_amdgcn_readlane((int)inc, 7) - a0;
            if (lane >= first && lane < first + n) lds64vw(D_DESC + (nch + lane - first) * 8, (u32x2){((basek * 1024 + inc - sz - a0) << 4) | code, gid * 8 + lane});
            nch += n;
            if (lane == 0) lds64vw(D_UNIT + ui * 8, (u32x2){gid * (u32)GRP_STRIDE + a0, total | ((nch - 1) << 18)});
            WAIT_LGKM0();
            lds32vw(CTL_NUNIT, ui + 1); lds32vw(CTL_NCHUNK, nch);
            basek += ((total + 4095) >> 12) << 2;
            STAMP(P, umin(ui, 127));
        }
        lds32vw(CTL_NUNIT, ui | DONE); lds32vw(CTL_NCHUNK, nch | DONE);
    } else {
        const u32 dw = wid - 2, scr = D_SCR + dw * D_SCRB;
        const u32 loff = swz((lane & 15) * 64 + (lane >> 4) * 16);
        if (dw == ND - 1 && blockIdx.x < (u32)(8 * RAWG)) {
            const u32 gid = blockIdx.x >> 3, w = blockIdx.x & 7;
            u32 v[64], Mr[32], Er[32]; const u32 Cr[8] = {};
            load_x(P, gid, w, lane, v); split(v, Mr, Er);
            pack_store(P.ws[P.rank] + WS_STG + (call & 1) * STG_PAR + (size_t)gid * GRP_STRIDE + w * RAWB + lane * 16, RAWCODE, Mr, Er, Cr, 0u, lane);
            WAIT_VM(0);
            if (lane < (u32)NR && lane != (u32)P.rank) raise_flag(P, lane, gid, w, call, RAWCODE);
            own_x(P, gid, w, lane, v);
            WAIT_VM(0);
            if (lane == 0) __hip_atomic_fetch_add(counter(P, gid >> 1), 1u, __ATOMIC_RELAXED, __HIP_MEMORY_SCOPE_AGENT);
        }
        u64 tk = 0, c_wait = 0, c_dec = 0, c_st = 0; (void)c_wait; (void)c_dec; (void)c_st; TICK(tk);
        for (u32 c = dw;; c += ND) {
            u32 nc = rfl(lds32v(CTL_NCHUNK));
            while (nc <= c) { __builtin_amdgcn_s_sleep(2); nc = rfl(lds32v(CTL_NCHUNK)); }
            if ((nc & ~DONE) <= c) break;
            CBAR();
            const u32x2 de = lds64v(D_DESC + c * 8);
            const u32 d = rfl(de.x), code = d & 255u, S = (d >> 8) << 4, gid = rfl(de.y) >> 3, w = rfl(de.y) & 7;
            const u32 need = (S + chunk_bytes(code) + 1023) >> 10;
            while (rfl(lds32v(CTL_LANDED)) < need) __builtin_amdgcn_s_sleep(1);
            CBAR();
            u32 Mr[32], Er[32];
            CS_DECL
            ACC(c_wait, tk);
            unpack_chunk(S, code, scr, lane, CTL_NEXT + dw * 4, c + ND, Mr, Er CS_PASS);
            ACC(c_dec, tk);
            if (gid >= (u32)GY) {
                const u32 t = gid - GY;
                u16* o = y_base(P, t & 15, q * DP + (t >> 4), w, lane);
#pragma unroll
                for (int p = 0; p < 16; ++p) st_out<GAR_OST>(o + y_piece_off(p), piece(Mr, Er, p));
            } else {
                const bool isx = gid < (u32)GW;
                const u32 pn = isx ? gid >> 1 : local_panel((gid - GW) >> 1, P.rank);
                char* o = P.ws[P.rank] + (isx ? WS_XI : WS_WI) + (size_t)pn * PANEL + (size_t)(q * 8 + (gid & 1) * 4 + (w >> 1)) * 32768 + loff;
                if (isx) {
#pragma unroll
                    for (int p = 0; p < 16; ++p) st_dev(o + x_piece_off(w & 1, p), piece(Mr, Er, p));
                } else {
#pragma unroll
                    for (int p = 0; p < 16; ++p) st_dev(o + w_piece_off(w & 1, p), piece(Mr, Er, p));
                }
                WAIT_VM(0);
                if (lane == 0) __hip_atomic_fetch_add(counter(P, isx ? pn : 16 + pn), 1u, __ATOMIC_RELAXED, __HIP_MEMORY_SCOPE_AGENT);
            }
            ACC(c_st, tk);
        }
        DUMP(P, wid * 4, c_wait); DUMP(P, wid * 4 + 1, c_dec); DUMP(P, wid * 4 + 2, c_st); DUMP(P, wid * 4 + 3, now());
    }
}

__device__ __forceinline__ void pack_inputs(const Params& P, u32 gid, u32 call, u32 wid, u32 lane) {
    u32 v[64];
    const u32 ktl = (gid & 1) * 4 + (wid >> 1), half = wid & 1;
    const u32 loff = swz((lane & 15) * 64 + (lane >> 4) * 16);
    u32 peers = 255u & ~(1u << P.rank); bool own = true; u32 pn;
    if (gid < (u32)GW) {
        pn = gid >> 1;
        load_x(P, gid, wid, lane, v);
        WAIT_VM(0); STAMP(P, 32 + wid);
        own_x(P, gid, wid, lane, v);
    } else {
        const u32 j = (gid - GW) >> 1;
        if (j < (u32)(NR * DP)) { const u32 dest = j / DP; own = dest == (u32)P.rank; peers = own ? 0u : 1u << dest; }
        const u16* s = P.w + (size_t)(ktl * 64 + (lane >> 4)) * NN + j * 256 + half * 128 + (lane & 15) * 8;
        const u32 scr = wid * 16384;
#pragma unroll
        for (int i = 0; i < 16; ++i) glds_plain(s + (size_t)i * 4 * NN, scr + i * 1024);
        WAIT_VM(0);
        const u32 ta = scr + ((lane >> 4) * 8 + ((lane & 15) >> 2)) * 256 + (lane & 3) * 16;
#pragma unroll
        for (int p = 0; p < 16; ++p) {
            const u32 a = ta + (p & 1) * 32 * 256 + (p >> 2) * 64 + ((p >> 1) & 1) * 8;
            const u32x2 lo = __builtin_bit_cast(u32x2, __builtin_amdgcn_ds_read_tr16_b64_v4i16((LAS s16x4*)a));
            const u32x2 hi = __builtin_bit_cast(u32x2, __builtin_amdgcn_ds_read_tr16_b64_v4i16((LAS s16x4*)(a + 4 * 256)));
            v[4 * p] = lo.x; v[4 * p + 1] = lo.y; v[4 * p + 2] = hi.x; v[4 * p + 3] = hi.y;
        }
        WAIT_LGKM0();
        pn = 16;
        if (own) {
            const u32 jj = local_panel(j, P.rank); pn = 16 + jj;
            char* o = P.ws[P.rank] + WS_WI + (size_t)jj * PANEL + (size_t)(P.rank * 8 + ktl) * 32768 + loff;
#pragma unroll
            for (int p = 0; p < 16; ++p) st_dev(o + w_piece_off(half, p), (u32x4){v[4 * p], v[4 * p + 1], v[4 * p + 2], v[4 * p + 3]});
        }
    }
    if (peers) pack_group(P, v, gid, peers, call, wid, lane);
    if (own) { WAIT_VM(0); if (lane == 0) __hip_atomic_fetch_add(counter(P, pn), 1u, __ATOMIC_RELAXED, __HIP_MEMORY_SCOPE_AGENT); }
}

typedef __bf16 bf16x2 __attribute__((ext_vector_type(2)));
typedef float f32x2 __attribute__((ext_vector_type(2)));
__device__ __forceinline__ u32 cvt_pk(float lo, float hi) { return __builtin_bit_cast(u32, __builtin_convertvector((f32x2){lo, hi}, bf16x2)); }
__device__ __forceinline__ void wait_panels(const Params& P, u32 tile, u32 call, u32 wid, u32 lane) {
    if (wid == 0) {
        const u32* c = counter(P, (lane & 1) ? 16 + (tile >> 4) : (tile & 15));
        const u64 t0 = now();
        for (;;) {
            const u32 x = __hip_atomic_load(c, __ATOMIC_RELAXED, __HIP_MEMORY_SCOPE_AGENT);
            if (__all((int)(x - call * 128u) >= 0)) break;
            if (__any(expired(P, t0, call))) { asm volatile("ds_write_b32 %0, %1\n\ts_waitcnt lgkmcnt(0)" :: "v"(CTL_BAD), "v"(1u) : "memory"); break; }
            __builtin_amdgcn_s_sleep(4);
        }
        __builtin_amdgcn_fence(__ATOMIC_ACQUIRE, "agent");
        WAIT_VM(0);
    }
    BAR();
}
__device__ __forceinline__ void gemm_role(const Params& P, unsigned char* lds_generic, u32 call, u32 wid, u32 lane) {
    LAS unsigned char* lds = (LAS unsigned char*)lds_generic;
    const u32 gw = blockIdx.x - GEMM0, tid = threadIdx.x;
    const u32 wr = wid >> 2, wc = wid & 3, fr = lane & 15, fq = lane >> 4;
    constexpr int HTB = 16384;
    const u32 voff = tid * 16, ldsw = wid * 1024;
    const u32 fo = swz(fr * 64 + fq * 16);
    const u32 aoff = wr * 8192 + fo, boff = wc * 4096 + fo;
#define G_SA(b, h) (((b) * 2 + (h)) * HTB)
#define G_SB(b, h) ((4 + (b) * 2 + (h)) * HTB)
#define G_STAGE(bufoff, gbase) do { _Pragma("unroll") for (int _i = 0; _i < 2; ++_i) \
        __builtin_amdgcn_global_load_lds((const u32*)((const char*)(gbase) + voff + _i * 8192), (LAS u32*)(lds + (bufoff) + ldsw + _i * 8192), 16, 0, 0); } while (0)
#define G_LDA(dst, b, h) do { _Pragma("unroll") for (int m = 0; m < 4; ++m) _Pragma("unroll") for (int k = 0; k < 2; ++k) dst[m][k] = *(const LAS bf16x8*)(lds + G_SA(b, h) + aoff + m * 2048 + k * 1024); } while (0)
#define G_LDB(dst, b, h) do { _Pragma("unroll") for (int n = 0; n < 2; ++n) _Pragma("unroll") for (int k = 0; k < 2; ++k) dst[n][k] = *(const LAS bf16x8*)(lds + G_SB(b, h) + boff + n * 2048 + k * 1024); } while (0)
#define G_MMA(ai, bj, At, Bt) do { __builtin_amdgcn_s_setprio(1); _Pragma("unroll") for (int m = 0; m < 4; ++m) _Pragma("unroll") for (int n = 0; n < 2; ++n) _Pragma("unroll") for (int k = 0; k < 2; ++k) \
        acc[ai][bj][m][n] = __builtin_amdgcn_mfma_f32_16x16x32_bf16(Bt[n][k], At[m][k], acc[ai][bj][m][n], 0, 0, 0); __builtin_amdgcn_s_setprio(0); } while (0)
#define G_WV(n) asm volatile("s_waitcnt vmcnt(" #n ")" ::: "memory")
#define G_WL(n) asm volatile("s_waitcnt lgkmcnt(" #n ")" ::: "memory")
#define G_BAR __builtin_amdgcn_s_barrier()
#define G_SCHED __builtin_amdgcn_sched_barrier(0)
    const char* XI = P.ws[P.rank] + WS_XI; const char* WI = P.ws[P.rank] + WS_WI;
    constexpr size_t kstep = 32768, hstep = 16384;
    u32 cur = tab16(P.tiles, gw);
    if (cur == 0xffffu) return;
    bool fresh = true;
    bf16x8 At[4][2], B0[2][2], B1[2][2];
    for (u32 ui = 0;; ++ui) {
        const u32 nxt = ui + 1 < (u32)ROUNDS ? tab16(P.tiles, (ui + 1) * NGEMM + gw) : 0xffffu;
        const bool has_next = nxt != 0xffffu, dist = (cur >> 4) < (u32)DP, chain = has_next && !dist;
        const char* cA = XI + (size_t)(cur & 15) * PANEL; const char* cB = WI + (size_t)(cur >> 4) * PANEL;
        if (fresh) {
            wait_panels(P, cur, call, wid, lane);
            STAMP(P, 8 + ui * 4);
            G_STAGE(G_SB(0, 0), cB); G_STAGE(G_SA(0, 0), cA); G_STAGE(G_SB(0, 1), cB + hstep); G_STAGE(G_SA(0, 1), cA + hstep);
            if (wr == 1) G_BAR;
            G_WV(4); G_BAR;
            G_STAGE(G_SB(1, 0), cB + kstep); G_STAGE(G_SA(1, 0), cA + kstep); G_STAGE(G_SB(1, 1), cB + hstep + kstep);
            G_WV(6); G_BAR;
        }
        f32x4 acc[2][2][4][2];
#pragma unroll
        for (int a = 0; a < 2; ++a)
#pragma unroll
            for (int b = 0; b < 2; ++b)
#pragma unroll
                for (int m = 0; m < 4; ++m)
#pragma unroll
                    for (int n = 0; n < 2; ++n) acc[a][b][m][n] = (f32x4){0.f, 0.f, 0.f, 0.f};
        const char* nA = chain ? XI + (size_t)(nxt & 15) * PANEL : cA; const char* nB = chain ? WI + (size_t)(nxt >> 4) * PANEL : cB;
        for (int t = 0; t < 64; t += 2) {
            const bool last = (t == 62);
            const char* a1 = cA + (size_t)(t + 1) * kstep;
            const char* a2 = last ? nA : cA + (size_t)(t + 2) * kstep; const char* b2 = last ? nB : cB + (size_t)(t + 2) * kstep;
            const char* a3 = a2 + kstep; const char* b3 = b2 + kstep;
            if (last && chain) wait_panels(P, nxt, call, wid, lane);
            G_LDB(B0, 0, 0); G_SCHED; G_LDA(At, 0, 0); G_STAGE(G_SA(1, 1), a1 + hstep);
            G_WL(8); G_BAR; G_WL(0); G_MMA(0, 0, At, B0); G_BAR; G_SCHED;
            G_LDB(B1, 0, 1); G_STAGE(G_SB(0, 0), b2);
            G_BAR; G_WL(0); G_MMA(0, 1, At, B1); G_BAR;
            G_LDA(At, 0, 1); G_STAGE(G_SA(0, 0), a2);
            G_BAR; G_WL(0); G_MMA(1, 0, At, B0); G_BAR; G_SCHED;
            G_STAGE(G_SB(0, 1), b2 + hstep);
            G_WV(6); G_BAR; G_MMA(1, 1, At, B1); G_BAR;
            G_LDB(B0, 1, 0); G_SCHED; G_LDA(At, 1, 0); G_STAGE(G_SA(0, 1), a2 + hstep);
            G_WL(8); G_BAR; G_WL(0); G_MMA(0, 0, At, B0); G_BAR; G_SCHED;
            G_LDB(B1, 1, 1); G_STAGE(G_SB(1, 0), b3);
            G_BAR; G_WL(0); G_MMA(0, 1, At, B1); G_BAR;
            G_LDA(At, 1, 1); G_STAGE(G_SA(1, 0), a3);
            G_BAR; G_WL(0); G_MMA(1, 0, At, B0); G_BAR; G_SCHED;
            G_STAGE(G_SB(1, 1), b3 + hstep);
            G_WV(6); G_BAR; G_MMA(1, 1, At, B1); G_BAR;
        }
        STAMP(P, 9 + ui * 4);
        u32 v[64];
        {
            int tz = threadIdx.x; asm volatile("" : "+v"(tz));
            const u32 ln = tz & 63;
            u32 bad; asm volatile("ds_read_b32 %0, %1\n\ts_waitcnt lgkmcnt(0)" : "=v"(bad) : "v"(CTL_BAD) : "memory");
            u16* o = y_base(P, cur & 15, global_panel(cur >> 4, P.rank), wid, ln);
#pragma unroll
            for (int p = 0; p < 16; p += 2) {
                const int ai = p >> 3, m = (p >> 1) & 3;
#pragma unroll
                for (int i = 0; i < 4; ++i) {
                    const f32x4 xa = acc[ai][0][m][i >> 1], xb = acc[ai][1][m][i >> 1];
                    u32 a = cvt_pk(xa[(i & 1) * 2], xa[(i & 1) * 2 + 1]), b = cvt_pk(xb[(i & 1) * 2], xb[(i & 1) * 2 + 1]);
                    if (bad) a = b = 0x7fc07fc0u;
                    v[4 * p + i] = (u32)__builtin_amdgcn_update_dpp((int)a, (int)b, 0x128, 0xf, 0xc, false);
                    v[4 * p + 4 + i] = (u32)__builtin_amdgcn_update_dpp((int)b, (int)a, 0x128, 0xf, 0x3, false);
                }
                st_out<GAR_GST>(o + y_piece_off(p), (u32x4){v[4 * p], v[4 * p + 1], v[4 * p + 2], v[4 * p + 3]});
                st_out<GAR_GST>(o + y_piece_off(p + 1), (u32x4){v[4 * p + 4], v[4 * p + 5], v[4 * p + 6], v[4 * p + 7]});
            }
            if (!chain) {
                G_WV(0);
                if (wr == 0) G_BAR;
                BAR();
                if (dist) pack_group(P, v, GY + cur, 255u & ~(1u << P.rank), call, wid, ln);
                STAMP(P, 10 + ui * 4);
                if (!has_next) break;
            }
        }
        fresh = !chain; cur = nxt;
    }
}

__global__ void __launch_bounds__(512, 2) fused(const Params P) {
    extern __shared__ __attribute__((aligned(16))) unsigned char lds[];
    const u32 wid = rfl(threadIdx.x >> 6), lane = threadIdx.x & 63;
    if (threadIdx.x == 0) {
        const u32 c = __hip_atomic_fetch_add((u32*)(P.ws[P.rank] + WS_CALL + blockIdx.x * 128), 1u, __ATOMIC_RELAXED, __HIP_MEMORY_SCOPE_AGENT) + 1;
        lds32vw(CTL_CALL, c); lds32vw(CTL_NUNIT, 0u); lds32vw(CTL_LANDED, 0u); lds32vw(CTL_NCHUNK, 0u); lds32vw(CTL_BAD, 0u);
    }
    if (wid == 1 && lane < 8) lds32vw(CTL_NEXT + lane * 4, lane);
    pack_init(threadIdx.x);
    WAIT_LGKM0(); BAR();
    const u32 call = rfl(lds32v(CTL_CALL));
    if (blockIdx.x < (u32)NPULL) { puller(P, call, wid, lane); return; }
    STAMP(P, 0);
    if (blockIdx.x - NPULL >= (u32)URGENT) { const u64 t0 = now(); while (now() - t0 < GAR_HOLD) __builtin_amdgcn_s_sleep(2); }
    for (u32 e = blockIdx.x - NPULL; e < (u32)NPACK; e += NENC) { pack_inputs(P, tab16(P.enc, e), call, wid, lane); WAIT_LGKM0(); BAR(); STAMP(P, e >= (u32)NENC ? 2 : 1); }
    if (blockIdx.x >= (u32)GEMM0) gemm_role(P, lds, call, wid, lane);
}

static void build_tables(Params& P) {
    u16* tiles = (u16*)P.tiles; u16* stream = (u16*)P.stream; u16* enc = (u16*)P.enc;
    static const int RG[8] = {0, 1, 0, 1, 2, 2, 3, 3}, CG[8] = {0, 0, 1, 1, 0, 1, 0, 1};
    for (int i = 0; i < ROUNDS * NGEMM; ++i) tiles[i] = 0xffff;
    const int full = LP / 12, rest = LP - full * 12;
    for (int r = 0; r < full; ++r) for (int l = 0; l < 8; ++l) for (int i = 0; i < 24; ++i)
        tiles[r * NGEMM + i * 8 + l] = (u16)((RG[l] * 4 + (i & 3)) | (r * 12 + CG[l] * 6 + (i >> 2)) << 4);
    {
        int l = 0, i = 0;
        for (int c0 = 0; c0 < rest; c0 += 6) for (int rg = 0; rg < 4; ++rg) for (int c = c0; c < c0 + 6 && c < rest; ++c) for (int m = 0; m < 4; ++m) {
            tiles[full * NGEMM + i * 8 + l] = (u16)((rg * 4 + m) | (full * 12 + c) << 4);
            if (++i == 24) { i = 0; ++l; } }
    }
    int in[16 + LP], n = 0;
    for (int i = 0; i < 8; ++i) in[n++] = i;
    for (int i = 0; i < 12; ++i) in[n++] = 16 + i;
    for (int i = 8; i < 16; ++i) in[n++] = i;
    for (int i = 12; i < LP; ++i) in[n++] = 16 + i;
    int g[NS], ng = 0, k = 0;
    for (int i = 0; i < n; ++i) for (int h = 0; h < 2; ++h) {
        if (in[i] < 16) { g[ng++] = GX + 2 * in[i] + h; if (2 * in[i] + h >= RAWG) enc[k++] = (u16)(GX + 2 * in[i] + h); continue; }
        const int jj = in[i] - 16;
        if (jj >= DP) { g[ng++] = GW + 2 * (jj + (NR - 1) * DP) + h; enc[k++] = (u16)g[ng - 1]; continue; }
        g[ng++] = GW + 2 * (P.rank * DP + jj) + h;
        for (int d = 1; d <= NR; ++d) enc[k++] = (u16)(GW + 2 * (((P.rank + d) & 7) * DP + jj) + h);
    }
    for (int r = 0; r < ROUNDS; ++r) for (int l = 0; l < 8; ++l) for (int i = 0; i < 24; ++i) { const int t = tiles[r * NGEMM + i * 8 + l]; if (t != 0xffff && (t >> 4) < DP) g[ng++] = GY + t; }
    k = 0;
    for (int i = 0; i < NS; ++i) {
        const int left = NS - i; int lv = 0;
        if (i < RAWG) lv = 3; else if (i < RAWG + 4) lv = 2; else if (i < RAWG + 8) lv = 1;
        if (left <= 2) lv = 3; else if (left <= 6) lv = 2; else if (left <= 14) lv = 1;
        for (int j = 0; j < (1 << lv); ++j) stream[k++] = (u16)(g[i] | ((1 << lv) + j) << 10);
    }
    P.nunits = k;
}
}

extern "C" void kernel_launch(int rank, int nranks, ncclComm_t comm, hipStream_t stream,
                              void* const* d_in, const int* in_sizes, int n_in,
                              void* d_out, int out_size, const int* in_dtypes, int out_dtype,
                              void* const* d_out_all, void* const* d_ws_all, size_t ws_bytes) {
    using namespace gar;
    if (nranks != NR || (unsigned)rank >= (unsigned)NR || n_in != 2 || ws_bytes < WS_END || in_sizes[0] != MM * KS || in_sizes[1] != KS * NN || out_size != MM * NN) {
        fprintf(stderr, "kernel_launch: unexpected shapes\n"); return; }
    Params P = {};
    for (int p = 0; p < NR; ++p) P.ws[p] = (char*)d_ws_all[p];
    P.x = (const u16*)d_in[0]; P.w = (const u16*)d_in[1]; P.out = (u16*)d_out; P.rank = rank;
    build_tables(P);
    HIP_CHECK(hipFuncSetAttribute((const void*)fused, hipFuncAttributeMaxDynamicSharedMemorySize, LDS_BYTES));
    hipLaunchKernelGGL(fused, dim3(NWG), dim3(512), LDS_BYTES, stream, P);
    HIP_CHECK(hipGetLastError());
}
```

```cpp
#include <hip/hip_runtime.h>
#include <rccl/rccl.h>
#include <cstdio>
#include <cstdint>
#pragma clang diagnostic ignored "-Wint-to-pointer-cast"

#define HIP_CHECK(expr) do { hipError_t _e = (expr); if (_e != hipSuccess) { \
    fprintf(stderr, "HIP error %s at %s:%d\n", hipGetErrorString(_e), __FILE__, __LINE__); \
    return; } } while (0)

namespace gar {
#define LAS __attribute__((address_space(3)))
#define GAS __attribute__((address_space(1)))
typedef unsigned u32;
typedef unsigned short u16;
typedef unsigned long long u64;
typedef unsigned u32x4 __attribute__((ext_vector_type(4)));
typedef unsigned u32x2 __attribute__((ext_vector_type(2)));
typedef short s16x4 __attribute__((ext_vector_type(4)));
typedef short bf16x8 __attribute__((ext_vector_type(8)));
typedef float f32x4 __attribute__((ext_vector_type(4)));

constexpr int NR = 8, MM = 4096, NN = 32768, KS = 512;
#define GAR_DP 12
constexpr int DP = GAR_DP;
constexpr int RP = 128 - NR * DP;
constexpr int LP = DP + RP;
constexpr int NWG = 256;
#define GAR_PW 9
#define GAR_WIN 6
constexpr int PW = GAR_PW;
constexpr int NPULL = 7 * PW;
constexpr int WIN = GAR_WIN;
constexpr int GEMM0 = 64;
constexpr int NGEMM = NWG - GEMM0;
static_assert(NPULL <= GEMM0, "roles");
#define GAR_NRAW 18
#define GAR_SOLOG 16
#define GAR_LEAD 32
#define GAR_HOLD 200
constexpr int SOLOG = GAR_SOLOG, NRAW = GAR_NRAW;
constexpr int LEAD = GAR_LEAD;
static_assert(NRAW <= NPULL && NRAW <= 8 * SOLOG && 8 * SOLOG - NRAW <= NWG - NPULL && SOLOG <= 32, "solo chunks");
constexpr int NENC = NWG - NPULL;
constexpr int GX = 0, GW = 32, GY = 288;
constexpr int NGID = GY + 16 * DP;
constexpr int NS = 32 + 2 * DP + 2 * RP + 16 * DP;
constexpr int NTILE = 16 * LP;
constexpr int ROUNDS = (NTILE + NGEMM - 1) / NGEMM;
constexpr int NUMAX = 512;
constexpr int NSTAT = 3;
constexpr int NPACK = 288 - SOLOG;
constexpr u64 DEADLINE = 30000000ull;

constexpr size_t WS_CALL = 0;
constexpr size_t WS_TMO = 32768;
constexpr size_t WS_CNT = 36864;
constexpr size_t WS_CLAIM = 57344;
constexpr size_t WS_FLAG = 65536;
constexpr size_t FLAG_SRC = 544 * 32, FLAG_PAR = 8 * FLAG_SRC;
constexpr size_t WS_DBG = 524288;
constexpr size_t GRP_STRIDE = 131072;
constexpr size_t WS_STG = 2u << 20;
constexpr size_t STG_PAR = 544 * GRP_STRIDE;
constexpr size_t WS_XI = WS_STG + 2 * STG_PAR;
constexpr size_t PANEL = 64 * 32768;
constexpr size_t WS_WI = WS_XI + 16 * PANEL;
constexpr size_t WS_END = WS_WI + (size_t)LP * PANEL;
static_assert(WS_FLAG + 2 * FLAG_PAR <= WS_DBG && NGID <= 544, "layout");

constexpr u32 LDS_CTL = 131072;
constexpr u32 LDS_LUT = LDS_CTL + 512, LDS_HIST = LDS_CTL + 1024;
constexpr u32 LDS_BYTES = LDS_HIST + 4096;
constexpr u32 CTL_CALL = LDS_CTL  , CTL_SIZE = LDS_CTL + 64  , CTL_BAD = LDS_CTL + 128;
constexpr u32 CTL_NUNIT = LDS_CTL + 192, CTL_LANDED = LDS_CTL + 256, CTL_NCHUNK = LDS_CTL + 320, CTL_NEXT = LDS_CTL + 384  ;
constexpr u32 E_L2 = 0, E_L3 = E_L2 + 4160, E_VAR = E_L3 + 2112, E_END = E_VAR + 2112;
static_assert(E_END <= 9216 && E_L3 % 16 == 0 && E_VAR % 16 == 0, "packer scratch");
constexpr u32 CAP2 = 4080, CAP3 = 2032, CAPV = 2032, FIXB = 10240, RAWB = 16384, RAWCODE = 255;
constexpr u32 RING = 65536, RMASK = RING - 1, ND = 6;
constexpr u32 D_SCR = RING, D_SCRB = 6400, D_L2 = 0, D_L3 = 4160;
constexpr u32 D_DESC = D_SCR + ND * D_SCRB, D_UNIT = D_DESC + NS * 64, DONE = 0x80000000u;
static_assert(D_L3 + 2112 <= D_SCRB && D_UNIT + (NUMAX + 8 * SOLOG) * 8 <= LDS_CTL && D_DESC % 16 == 0, "puller LDS");

struct Params {
    char* ws[NR];
    const u16* x; const u16* w; u16* out;
    int rank, nunits;
    u32 stream[NUMAX / 2];
    u32 enc[144];
    u32 tiles[ROUNDS * NGEMM / 2];
};
__device__ __forceinline__ u32 tab16(const u32* t, u32 i) { return (t[i >> 1] >> ((i & 1) * 16)) & 0xffffu; }
static_assert(sizeof(Params) <= 4096, "kernarg");

__device__ __forceinline__ u32 lds32(u32 a) { return *(LAS u32*)a; }
__device__ __forceinline__ u32 lds32v(u32 a) { return *(volatile LAS u32*)a; }
__device__ __forceinline__ void lds32vw(u32 a, u32 x) { *(volatile LAS u32*)a = x; }
__device__ __forceinline__ u32x2 lds64v(u32 a) { return *(volatile LAS u32x2*)a; }
__device__ __forceinline__ void lds64vw(u32 a, u32x2 x) { *(volatile LAS u32x2*)a = x; }
__device__ __forceinline__ u32x4 lds128(u32 a) { return *(LAS u32x4*)a; }
__device__ __forceinline__ void lds128w(u32 a, u32x4 x) { *(LAS u32x4*)a = x; }
__device__ __forceinline__ void lds_or(u32 a, u32 x) { __hip_atomic_fetch_or((LAS u32*)a, x, __ATOMIC_RELAXED, __HIP_MEMORY_SCOPE_WORKGROUP); }
__device__ __forceinline__ void lds_add(u32 a, u32 x) { __hip_atomic_fetch_add((LAS u32*)a, x, __ATOMIC_RELAXED, __HIP_MEMORY_SCOPE_WORKGROUP); }
__device__ __forceinline__ u32 rfl(u32 x) { return (u32)__builtin_amdgcn_readfirstlane((int)x); }
#define CBAR() asm volatile("" ::: "memory")
#define WAIT_LGKM0() asm volatile("s_waitcnt lgkmcnt(0)" ::: "memory")
#define WAIT_VM(n) asm volatile("s_waitcnt vmcnt(%0)" :: "i"(n) : "memory")
#define BAR() do { CBAR(); __builtin_amdgcn_s_barrier(); CBAR(); } while (0)
__device__ __forceinline__ void st_sys(void* p, u32x4 x) { asm volatile("global_store_dwordx4 %0, %1, off sc0 sc1\n\ts_nop 1" :: "v"(p), "v"(x) : "memory"); }
__device__ __forceinline__ void st_dev(void* p, u32x4 x) { asm volatile("global_store_dwordx4 %0, %1, off sc1\n\ts_nop 1" :: "v"(p), "v"(x) : "memory"); }
#define GAR_OST 3
#define GAR_GST 3
template <int MODE> __device__ __forceinline__ void st_out(void* p, u32x4 x) {
    if (MODE == 0) *(u32x4*)p = x;
    else if (MODE == 1) __builtin_nontemporal_store(x, (u32x4*)p);
    else if (MODE == 2) st_dev(p, x);
    else if (MODE == 9) asm volatile("" :: "v"(p), "v"(x) : "memory");
    else asm volatile("global_store_dwordx4 %0, %1, off sc1 nt\n\ts_nop 1" :: "v"(p), "v"(x) : "memory");
}
__device__ __forceinline__ void glds_sys(const void* g, u32 dst) {
    u32 keep;
    asm volatile("s_mov_b32 %0, m0\n\ts_mov_b32 m0, %2\n\ts_nop 0\n\tglobal_load_lds_dwordx4 %1, off sc0 sc1\n\ts_mov_b32 m0, %0" : "=&s"(keep) : "v"(g), "s"(dst) : "memory");
}
__device__ __forceinline__ void glds_plain(const void* g, u32 dst) {
    u32 keep;
    asm volatile("s_mov_b32 %0, m0\n\ts_mov_b32 m0, %2\n\ts_nop 0\n\tglobal_load_lds_dwordx4 %1, off\n\ts_mov_b32 m0, %0" : "=&s"(keep) : "v"(g), "s"(dst) : "memory");
}
__device__ __forceinline__ u64 now() { return __builtin_amdgcn_s_memrealtime(); }
template <int CTRL, int RM, bool BC> __device__ __forceinline__ u32 dpp(u32 v) { return (u32)__builtin_amdgcn_update_dpp(0, (int)v, CTRL, RM, 0xf, BC); }
__device__ __forceinline__ u32 scan_incl(u32 v) {
    v += dpp<0x111, 0xf, true>(v); v += dpp<0x112, 0xf, true>(v); v += dpp<0x114, 0xf, true>(v); v += dpp<0x118, 0xf, true>(v);
    v += dpp<0x142, 0xa, false>(v); v += dpp<0x143, 0xc, false>(v); return v;
}
__device__ __forceinline__ u32 umax(u32 a, u32 b) { return a > b ? a : b; }
__device__ __forceinline__ u32 umin(u32 a, u32 b) { return a < b ? a : b; }
__device__ __forceinline__ u32 wave_max(u32 v) {
    v = umax(v, dpp<0x111, 0xf, true>(v)); v = umax(v, dpp<0x112, 0xf, true>(v)); v = umax(v, dpp<0x114, 0xf, true>(v)); v = umax(v, dpp<0x118, 0xf, true>(v));
    v = umax(v, dpp<0x142, 0xa, false>(v)); v = umax(v, dpp<0x143, 0xc, false>(v)); return (u32)__builtin_amdgcn_readlane((int)v, 63);
}
__device__ __forceinline__ u32 popc(u32 x) { return (u32)__builtin_popcount(x); }
__device__ __forceinline__ u32 perm(u32 hi, u32 lo, u32 sel) { return __builtin_amdgcn_perm(hi, lo, sel); }
__device__ __forceinline__ u32 swz(u32 ob) { return ob ^ (((ob >> 9) & 1) << 5); }

#define STAMP(P, slot) do {} while (0)
#define TICK(t) do {} while (0)
#define ACC(sum, t) do {} while (0)
#define DUMP(P, slot, val) do {} while (0)

#define CSTAMP(i) do {} while (0)
#define CS_ARG
#define CS_DECL
#define CS_PASS

__device__ __forceinline__ u32 nzb(u32 z) { return ((z & 0x7f7f7f7fu) + 0x7f7f7f7fu) | z; }
typedef unsigned short u16x2 __attribute__((ext_vector_type(2)));
struct Classifier {
    u32 s1, s2, s3, nb, tlo, thi; bool win;
    __device__ __forceinline__ void init(u32 t) {
        const u32 a = (t >> 8) & 255u, b = (t >> 16) & 255u, c = t >> 24, lo = umin(a, umin(b, c));
        win = umax(a, umax(b, c)) - lo <= 6u;
        const u64 l = (1ull << (((a - lo) & 7u) * 8)) | (2ull << (((b - lo) & 7u) * 8)) | (3ull << (((c - lo) & 7u) * 8));
        tlo = (u32)l; thi = (u32)(l >> 32); nb = ((0u - lo) & 255u) * 0x01000100u;
        s1 = a * 0x01010101u; s2 = b * 0x01010101u; s3 = c * 0x01010101u;
    }
    __device__ __forceinline__ u32 any(u32 e) const {
        const u32 a1 = nzb(e ^ s1), a2 = nzb(e ^ s2), a3 = nzb(e ^ s3);
        return ((~(a1 & a3) >> 7) & 0x01010101u) | ((~(a2 & a3) >> 6) & 0x02020202u);
    }
    __device__ __forceinline__ u32 near(u32 e) const {
        const u16x2 x = __builtin_bit_cast(u16x2, e), n = __builtin_bit_cast(u16x2, nb), top = {0x07ff, 0x07ff};
        u32 sh; asm("v_pk_mad_u16 %0, %1, %2, %3" : "=v"(sh) : "v"(e), "v"(0x01000100u), "s"(nb));
        const u16x2 hi = __builtin_elementwise_min((u16x2)(x + n), top), lo = __builtin_elementwise_min(__builtin_bit_cast(u16x2, sh), top);
        return perm(thi, tlo, perm(__builtin_bit_cast(u32, hi), __builtin_bit_cast(u32, lo), 0x07030501u));
    }
    template <int N> __device__ __forceinline__ void run(const u32* e, u32* c) const {
        if (win) {
#pragma unroll
            for (int i = 0; i < N; ++i) c[i] = near(e[i]);
        } else {
#pragma unroll
            for (int i = 0; i < N; ++i) c[i] = any(e[i]);
        }
    }
};
__device__ __forceinline__ u32 escbits(u32 c) { return ~(c | (c >> 1)) & 0x55555555u; }
struct Appender {
    u32 lo, fill, addr;
    __device__ __forceinline__ void init(u32 off) { lo = 0; fill = (off & 3u) * 8u; addr = off & ~3u; }
    __device__ __forceinline__ void push(u32 k, u32 mk) {
        const u64 t = (u64)k << fill;
        lo |= (u32)t; fill += popc(mk);
        if (fill >= 32u) { lds_or(addr, lo); lo = (u32)(t >> 32); addr += 4; }
        fill &= 31u;
    }
    __device__ __forceinline__ void flush() { if (lo) lds_or(addr, lo); }
};
__device__ __forceinline__ u32 escmask(u32 c) { return perm(0u, 0xffu, c); }
__device__ __forceinline__ u32 selector(u32 mk) { return lds32(__builtin_amdgcn_udot4(mk & 0x20100804u, 0x01010101u, LDS_LUT, false)); }
template <int R> __device__ __forceinline__ u32 pack_level(u32 src, u32 n, u32 t, u32 cdst, u32 edst, u32 cap, u32 lane) {
    Classifier K; K.init(t);
    if (lane < 16) *(LAS unsigned char*)(src + n + lane) = (unsigned char)K.s1;
    u32 ne = 0;
    for (u32 r = 0; r * 1024 < n; r += R) {
        CBAR();
        u32 q[4 * R], c[4 * R], m[4 * R], l[4 * R], cnt = 0;
#pragma unroll
        for (int i = 0; i < R; ++i) { const u32 p0 = (r + i) * 1024 + lane * 16; const u32x4 x = lds128(src + p0); const bool act = p0 < n;
            q[4 * i] = act ? x.x : K.s1; q[4 * i + 1] = act ? x.y : K.s1; q[4 * i + 2] = act ? x.z : K.s1; q[4 * i + 3] = act ? x.w : K.s1; }
        K.run<4 * R>(q, c);
#pragma unroll
        for (int i = 0; i < 4 * R; ++i) { m[i] = escmask(c[i]); l[i] = selector(m[i]); }
#pragma unroll
        for (int i = 0; i < R; ++i) cnt |= ((popc(m[4 * i]) + popc(m[4 * i + 1]) + popc(m[4 * i + 2]) + popc(m[4 * i + 3])) >> 3) << (16 * i);
        const u32 inc = scan_incl(cnt), tot = (u32)__builtin_amdgcn_readlane((int)inc, 63), exc = inc - cnt;
        if (ne + (tot & 0xffffu) + (tot >> 16) > cap) return ~0u;
#pragma unroll
        for (int i = 0; i < R; ++i) {
            Appender A; A.init(edst + ne + (i ? (tot & 0xffffu) + (exc >> 16) : (exc & 0xffffu)));
#pragma unroll
            for (int j = 0; j < 4; ++j) A.push(perm(0u, q[4 * i + j], l[4 * i + j]), m[4 * i + j]);
            A.flush();
            if ((r + i) * 1024 + lane * 16 < n) *(LAS u32*)(cdst + ((r + i) * 64 + lane) * 4) = c[4 * i] | (c[4 * i + 1] << 2) | (c[4 * i + 2] << 4) | (c[4 * i + 3] << 6);
        }
        ne += (tot & 0xffffu) + (tot >> 16);
    }
    CBAR();
    return ne;
}
template <int N> __device__ __forceinline__ u32 rank_ror(u32 key) { return (dpp<0x120 + N, 0xf, true>(key) > key ? 1u : 0u) + rank_ror<N - 1>(key); }
template <> __device__ __forceinline__ u32 rank_ror<0>(u32) { return 0u; }
__device__ __forceinline__ u32 row_or(u32 x) { x |= dpp<0x128, 0xf, true>(x); x |= dpp<0x124, 0xf, true>(x); x |= dpp<0x122, 0xf, true>(x); x |= dpp<0x121, 0xf, true>(x); return rfl(x); }
__device__ __forceinline__ void split(const u32 (&v)[64], u32 (&Mr)[32], u32 (&Er)[32]) {
#pragma unroll
    for (int g = 0; g < 32; ++g) {
        const u32 a = __builtin_amdgcn_alignbit(v[2 * g], v[2 * g], 31), b = __builtin_amdgcn_alignbit(v[2 * g + 1], v[2 * g + 1], 31);
        Er[g] = perm(b, a, 0x07050301u); Mr[g] = perm(b, a, 0x06040200u);
    }
}
__device__ __forceinline__ u32 pack_chunk(const u32 (&v)[64], u32 (&Mr)[32], u32 (&Er)[32], u32 (&Cr)[8], u32 scr, u32 lane, bool solo CS_ARG) {
    CBAR(); CSTAMP(0);
    const u32 hist = solo ? scr + 10240 : LDS_HIST;
    if (solo) {
#pragma unroll
        for (int j = 0; j < 4; ++j) lds128w(hist + j * 1024 + lane * 16, (u32x4){0u, 0u, 0u, 0u});
        CBAR();
#pragma unroll
        for (int i = 0; i < 12; ++i) lds_add(hist + (((v[i * 5 + 2] >> ((i & 1) * 16 + 3)) & 0xff0u) | ((lane & 3) * 4)), 1u);
    }
#pragma unroll
    for (int i = 0; i < 4; ++i) lds_add(hist + (((v[i * 15 + 5] >> ((i & 1) * 16 + 3)) & 0xff0u) | ((lane & 3) * 4)), 1u);
#pragma unroll
    for (int j = 0; j < 9; ++j) lds128w(scr + j * 1024 + lane * 16, (u32x4){0u, 0u, 0u, 0u});
    WAIT_LGKM0();
    if (!solo) BAR();
    CBAR();
    u32 key = 0;
#pragma unroll
    for (int i = 0; i < 4; ++i) { const u32 bin = i * 64 + lane; const u32x4 h = lds128(hist + bin * 16); key = umax(key, ((h.x + h.y + h.z + h.w) << 8) | bin); }
    split(v, Mr, Er);
    CSTAMP(1);
    { const auto r = __builtin_amdgcn_permlane32_swap(key, key, false, false); key = umax(r[0], r[1]); }
    { const auto r = __builtin_amdgcn_permlane16_swap(key, key, false, false); key = umax(r[0], r[1]); }
    const u32 rank = rank_ror<15>(key), lev = (rank * 11u) >> 5, sym = (key & 255u) << (8 * (rank - 3 * lev + 1));
    const u32 T[3] = {row_or(lev == 0 ? sym : 0u), row_or(lev == 1 ? sym : 0u), row_or(lev == 2 ? sym : 0u)};
    CSTAMP(3);
    u32 cnt = 0, mk[32], sl[32];
    {
        Classifier K; K.init(T[0]);
        u32 c[32]; K.run<32>(Er, c);
#pragma unroll
        for (int g = 0; g < 32; ++g) { mk[g] = escmask(c[g]); sl[g] = selector(mk[g]); cnt += popc(mk[g]); }
#pragma unroll
        for (int k = 0; k < 8; ++k) Cr[k] = c[4 * k] | (c[4 * k + 1] << 2) | (c[4 * k + 2] << 4) | (c[4 * k + 3] << 6);
    }
    cnt >>= 3;
    const u32 inc = scan_incl(cnt), n2 = (u32)__builtin_amdgcn_readlane((int)inc, 63);
    CSTAMP(4);
    if (n2 > CAP2) return RAWCODE;
    Appender A; A.init(scr + E_L2 + inc - cnt);
#pragma unroll
    for (int g = 0; g < 32; ++g) A.push(perm(0u, Er[g], sl[g]), mk[g]);
    A.flush();
    CBAR(); CSTAMP(5);
    const u32 d2 = (n2 + 15) >> 4;
    const u32 n3 = pack_level<2>(scr + E_L2, n2, T[1], scr + E_VAR + 12, scr + E_L3, CAP3, lane);
    CSTAMP(6);
    if (n3 == ~0u) return RAWCODE;
    const u32 d3 = (n3 + 15) >> 4, b4 = 12 + 4 * (d2 + d3);
    const u32 n4 = pack_level<1>(scr + E_L3, n3, T[2], scr + E_VAR + 12 + 4 * d2, scr + E_VAR + b4, CAPV - b4, lane);
    if (n4 == ~0u) return RAWCODE;
    if (lane == 0) { *(LAS u32*)(scr + E_VAR) = T[0] | d2; *(LAS u32*)(scr + E_VAR + 4) = T[1] | d3; *(LAS u32*)(scr + E_VAR + 8) = T[2]; }
    CBAR();
    return (b4 + n4 + 15) >> 4;
}
__device__ __forceinline__ void pack_init(u32 wid, u32 lane) {
    if (wid >= 4) lds128w(LDS_HIST + (wid - 4) * 1024 + lane * 16, (u32x4){0u, 0u, 0u, 0u});
    if (wid == 1 && lane < 16) { u32 sel = 0x0c0c0c0cu, k = 0;
#pragma unroll
        for (u32 b = 0; b < 4; ++b) if ((lane >> b) & 1) { sel = (sel & ~(255u << k)) | (b << k); k += 8; }
        *(LAS u32*)(LDS_LUT + lane * 4) = sel; }
}
__device__ __forceinline__ u32 chunk_bytes(u32 code) { return code == RAWCODE ? RAWB : FIXB + 16 * code; }

__device__ __forceinline__ void pack_store(char* dst, u32 code, const u32 (&Mr)[32], const u32 (&Er)[32], const u32 (&Cr)[8], u32 scr, u32 lane) {
#pragma unroll
    for (int j = 0; j < 8; ++j) st_sys(dst + j * 1024, (u32x4){Mr[4 * j], Mr[4 * j + 1], Mr[4 * j + 2], Mr[4 * j + 3]});
    if (code == RAWCODE) {
#pragma unroll
        for (int j = 0; j < 8; ++j) st_sys(dst + 8192 + j * 1024, (u32x4){Er[4 * j], Er[4 * j + 1], Er[4 * j + 2], Er[4 * j + 3]});
    } else {
        st_sys(dst + 8192, (u32x4){Cr[0], Cr[1], Cr[2], Cr[3]}); st_sys(dst + 9216, (u32x4){Cr[4], Cr[5], Cr[6], Cr[7]});
        if (lane < code) st_sys(dst + FIXB, lds128(scr + E_VAR + lane * 16));
        if (lane + 64 < code) st_sys(dst + FIXB + 1024, lds128(scr + E_VAR + 1024 + lane * 16));
    }
}
__device__ __forceinline__ void raise_flag(const Params& P, u32 q, u32 gid, u32 w, u32 call, u32 code) {
    char* wq = P.ws[0];
#pragma unroll
    for (u32 j = 1; j < NR; ++j) wq = q == j ? P.ws[j] : wq;
    u32* f = (u32*)(wq + WS_FLAG + (call & 1) * FLAG_PAR + (size_t)P.rank * FLAG_SRC + gid * 32) + w;
    __hip_atomic_store(f, (call << 8) | code, __ATOMIC_RELAXED, __HIP_MEMORY_SCOPE_SYSTEM);
}
__device__ __forceinline__ void pack_group(const Params& P, const u32 (&v)[64], u32 (&Mr)[32], u32 (&Er)[32], u32 gid, u32 w, bool solo, u32 peers, u32 call, u32 wid, u32 lane) {
    u32 Cr[8];
    const u32 scr = wid * 16384;
    CS_DECL
    if (gid < 64u) STAMP(P, 40 + wid);
    const u32 code = rfl(pack_chunk(v, Mr, Er, Cr, scr, lane, solo CS_PASS));
    if (gid < 64u) STAMP(P, 48 + wid);
    u32 off = w * RAWB, mine = code;
    if (!solo) {
        if (lane == 0) lds32vw(CTL_SIZE + wid * 4, code);
        WAIT_LGKM0(); BAR();
        mine = lds32v(CTL_SIZE + (lane & 7) * 4);
        if (lane < 32) lds128w(LDS_HIST + wid * 512 + lane * 16, (u32x4){0u, 0u, 0u, 0u});
        off = 0;
#pragma unroll
        for (u32 j = 0; j < 7; ++j) { const u32 c = (u32)__builtin_amdgcn_readlane((int)mine, j); if (j < wid) off += chunk_bytes(c); }
    }
    pack_store(P.ws[P.rank] + WS_STG + (call & 1) * STG_PAR + (size_t)gid * GRP_STRIDE + off + lane * 16, code, Mr, Er, Cr, scr, lane);
    if (gid < 64u) STAMP(P, 56 + wid);
    WAIT_VM(0);
    if (gid < 64u) STAMP(P, 64 + wid);
    WAIT_LGKM0();
    if (!solo) BAR();
    if ((solo ? (lane & 7) == w : wid == 0) && ((peers >> (lane >> 3)) & 1)) raise_flag(P, lane >> 3, gid, lane & 7, call, mine);
}

#define GAR_WINMODE 1
template <bool WRAP> struct Unp {
    u32 sel[4], lo[4], hi[4], of[4];
    __device__ __forceinline__ void fetch(u32 c, u32& off) {
#pragma unroll
        for (int j = 0; j < 4; ++j) {
            const u32 cj = (c >> (2 * j)) & 0x03030303u, mk = escmask(cj), m2 = mk & 0x02020202u;
            if (WRAP || GAR_WINMODE != 2) { const u32 a = off & ~3u; lo[j] = lds32(WRAP ? (a & RMASK) : a); hi[j] = lds32(WRAP ? ((a + 4) & RMASK) : a + 4); of[j] = off; }
            else lo[j] = lds32(off);
            u32 p; asm("v_mad_u32_u24 %0, %1, %2, %3" : "=v"(p) : "v"(m2), "s"(0x808080u), "v"(0x04040404u));
            sel[j] = (mk & p) | cj;
            off += popc(m2);
        }
    }
    __device__ __forceinline__ u32 finish(int j, u32 t) const {
        u32 w;
        if (WRAP || GAR_WINMODE == 0) w = __builtin_amdgcn_alignbyte(hi[j], lo[j], of[j] & 3u);
        else if (GAR_WINMODE == 1) w = __builtin_amdgcn_alignbyte(hi[j], lo[j], of[j]);
        else w = lo[j];
        return perm(w, t, sel[j]);
    }
    __device__ __forceinline__ u32x4 finish4(u32 t) const { return (u32x4){finish(0, t), finish(1, t), finish(2, t), finish(3, t)}; }
};
#define SCHED() __builtin_amdgcn_sched_barrier(0)
template <bool WRAP> __device__ __forceinline__ u32 unpack_round(u32 c, u32 t, u32 src, u32 dst) {
    const u32 cnt = popc(escbits(c)), inc = scan_incl(cnt);
    u32 off = src + inc - cnt;
    Unp<WRAP> u; u.fetch(c, off); SCHED();
    lds128w(dst, u.finish4(t));
    return (u32)__builtin_amdgcn_readlane((int)inc, 63);
}
__device__ __forceinline__ void unpack_chunk(u32 S, u32 code, u32 scr, u32 lane, u32 rel_addr, u32 rel_val, u32 (&Mr)[32], u32 (&Er)[32] CS_ARG) {
    const u32 a0 = S + lane * 16;
#pragma unroll
    for (int j = 0; j < 8; ++j) { const u32x4 q = lds128((a0 + j * 1024) & RMASK); Mr[4 * j] = q.x; Mr[4 * j + 1] = q.y; Mr[4 * j + 2] = q.z; Mr[4 * j + 3] = q.w; }
    if (code == RAWCODE) {
#pragma unroll
        for (int j = 0; j < 8; ++j) { const u32x4 q = lds128((a0 + 8192 + j * 1024) & RMASK); Er[4 * j] = q.x; Er[4 * j + 1] = q.y; Er[4 * j + 2] = q.z; Er[4 * j + 3] = q.w; }
        WAIT_LGKM0(); lds32vw(rel_addr, rel_val); CBAR();
        return;
    }
    const u32x4 ca = lds128((a0 + 8192) & RMASK), cb = lds128((a0 + 9216) & RMASK);
    const u32 V = S + FIXB;
    const u32x4 hd = lds128(V & RMASK);
    const u32 t1 = rfl(hd.x), t2 = rfl(hd.y), t3 = rfl(hd.z), d2 = t1 & 255u, d3 = t2 & 127u;
    u32 c2[4], c3[2];
#pragma unroll
    for (int r = 0; r < 4; ++r) { const u32 i = r * 64 + lane; const u32 x = lds32((V + 12 + 4 * i) & RMASK); c2[r] = i < d2 ? x : 0x55555555u; }
#pragma unroll
    for (int r = 0; r < 2; ++r) { const u32 i = r * 64 + lane; const u32 x = lds32((V + 12 + 4 * (d2 + i)) & RMASK); c3[r] = i < d3 ? x : 0x55555555u; }
    CSTAMP(1);
    u32 s4 = V + 12 + 4 * (d2 + d3);
    s4 += unpack_round<true>(c3[0], t3, s4, scr + D_L3 + lane * 16);
    if (d3 > 64) unpack_round<true>(c3[1], t3, s4, scr + D_L3 + 1024 + lane * 16);
    WAIT_LGKM0(); lds32vw(rel_addr, rel_val); CBAR();
    CSTAMP(2);
    const u32 C[8] = {ca.x, ca.y, ca.z, ca.w, cb.x, cb.y, cb.z, cb.w};
    u32 cnt = 0;
#pragma unroll
    for (int k = 0; k < 8; ++k) cnt += popc(escbits(C[k]));
    {
        const u32 n0 = popc(escbits(c2[0])), n1 = popc(escbits(c2[1]));
        const u32 i01 = scan_incl(n0 | (n1 << 16)), tot = (u32)__builtin_amdgcn_readlane((int)i01, 63);
        cnt = scan_incl(cnt) - cnt;
        u32 o0 = scr + D_L3 + (i01 & 0xffffu) - n0, o1 = scr + D_L3 + (tot & 0xffffu) + (i01 >> 16) - n1;
        Unp<false> u0, u1; u0.fetch(c2[0], o0); u1.fetch(c2[1], o1); SCHED();
        lds128w(scr + D_L2 + lane * 16, u0.finish4(t2)); lds128w(scr + D_L2 + 1024 + lane * 16, u1.finish4(t2));
        if (d2 > 128) {
            u32 s3 = scr + D_L3 + (tot & 0xffffu) + (tot >> 16);
            s3 += unpack_round<false>(c2[2], t2, s3, scr + D_L2 + 2048 + lane * 16);
            if (d2 > 192) unpack_round<false>(c2[3], t2, s3, scr + D_L2 + 3072 + lane * 16);
        }
    }
    CBAR(); CSTAMP(3);
    u32 off = scr + D_L2 + cnt;
    Unp<false> u[8];
    u[0].fetch(C[0], off); u[1].fetch(C[1], off);
#pragma unroll
    for (int k = 0; k < 8; ++k) {
        SCHED();
        if (k + 2 < 8) u[k + 2].fetch(C[k + 2], off);
        SCHED();
#pragma unroll
        for (int j = 0; j < 4; ++j) Er[4 * k + j] = u[k].finish(j, t1);
    }
    CBAR();
}
__device__ __forceinline__ u32x4 piece(const u32 (&Mr)[32], const u32 (&Er)[32], int p) {
    const u32 a = perm(Er[2 * p], Mr[2 * p], 0x05010400u), b = perm(Er[2 * p], Mr[2 * p], 0x07030602u);
    const u32 c = perm(Er[2 * p + 1], Mr[2 * p + 1], 0x05010400u), d = perm(Er[2 * p + 1], Mr[2 * p + 1], 0x07030602u);
    return (u32x4){__builtin_amdgcn_alignbit(a, a, 1), __builtin_amdgcn_alignbit(b, b, 1), __builtin_amdgcn_alignbit(c, c, 1), __builtin_amdgcn_alignbit(d, d, 1)};
}

__device__ __forceinline__ u32 x_piece_off(u32 half, int p) { return half * 16384 + p * 1024; }
__device__ __forceinline__ u32 w_piece_off(u32 h, int p) {
    const u32 g32 = p >> 2, n = (p >> 1) & 1, kb = p & 1;
    return (g32 & 1) * 16384 + ((2 * (2 * h + (g32 >> 1)) + n) * 2 + kb) * 1024;
}
__device__ __forceinline__ u16* y_base(const Params& P, u32 pm, u32 j, u32 w, u32 lane) {
    return P.out + (size_t)(pm * 256 + (w >> 2) * 64 + (lane & 7)) * NN + j * 256 + (w & 3) * 64 + (lane & 8) * 4 + (lane >> 4) * 8;
}
__device__ __forceinline__ size_t y_piece_off(int p) { return (size_t)((p >> 3) * 128 + (p & 7) * 8) * NN; }
__device__ __forceinline__ u32 local_panel(u32 j, u32 rank) { return j >= (u32)(NR * DP) ? j - (NR - 1) * DP : j - rank * DP; }
__device__ __forceinline__ u32 global_panel(u32 jj, u32 rank) { return jj >= (u32)DP ? jj + (NR - 1) * DP : jj + rank * DP; }
__device__ __forceinline__ u32* counter(const Params& P, u32 i) { return (u32*)(P.ws[P.rank] + WS_CNT + i * 128); }
__device__ __forceinline__ u32* tmo_word(const Params& P) { return (u32*)(P.ws[P.rank] + WS_TMO); }
__device__ __forceinline__ bool expired(const Params& P, u64 t0, u32 call) {
    if (__hip_atomic_load(tmo_word(P), __ATOMIC_RELAXED, __HIP_MEMORY_SCOPE_AGENT) == call || now() - t0 > DEADLINE) {
        __hip_atomic_store(tmo_word(P), call, __ATOMIC_RELAXED, __HIP_MEMORY_SCOPE_AGENT); return true; }
    return false;
}

__device__ __forceinline__ void store_image(const Params& P, u32 gid, u32 w, u32 q, u32 lane, const u32 (&Mr)[32], const u32 (&Er)[32]) {
    const bool isx = gid < (u32)GW;
    const u32 pn = isx ? gid >> 1 : local_panel((gid - GW) >> 1, P.rank);
    char* o = P.ws[P.rank] + (isx ? WS_XI : WS_WI) + (size_t)pn * PANEL + (size_t)(q * 8 + (gid & 1) * 4 + (w >> 1)) * 32768 + swz((lane & 15) * 64 + (lane >> 4) * 16);
#pragma unroll
    for (int p = 0; p < 16; ++p) st_dev(o + (isx ? x_piece_off(w & 1, p) : w_piece_off(w & 1, p)), piece(Mr, Er, p));
    WAIT_VM(0);
    if (lane == 0) __hip_atomic_fetch_add(counter(P, isx ? pn : 16 + pn), 1u, __ATOMIC_RELAXED, __HIP_MEMORY_SCOPE_AGENT);
}
__device__ __forceinline__ u32* call_word(const Params& P, u32 wid) { return (u32*)(P.ws[P.rank] + WS_CALL + blockIdx.x * 128 + wid * 4); }
__device__ __forceinline__ void call_load(const Params& P, u32 wid, u32 lane) {
    u32 keep;
    if (lane == 0) asm volatile("s_mov_b32 %0, m0\n\ts_mov_b32 m0, %2\n\ts_nop 0\n\tglobal_load_lds_dword %1, off sc1\n\ts_mov_b32 m0, %0" : "=&s"(keep) : "v"(call_word(P, wid)), "s"(CTL_CALL + wid * 4) : "memory");
}
__device__ __forceinline__ u32 call_take(u32 wid) { WAIT_VM(0); return rfl(lds32v(CTL_CALL + wid * 4)) + 1; }
__device__ __forceinline__ void call_bump(const Params& P, u32 wid, u32 lane, u32 call) { if (lane == 0) __hip_atomic_store(call_word(P, wid), call, __ATOMIC_RELAXED, __HIP_MEMORY_SCOPE_AGENT); }

__device__ __forceinline__ void load_x(const Params& P, u32 gid, u32 w, u32 lane, u32 (&v)[64]) {
    const u16* s = P.x + (size_t)((gid >> 1) * 256 + (w & 1) * 128 + (lane & 15)) * KS + ((gid & 1) * 4 + (w >> 1)) * 64 + (lane >> 4) * 8;
#pragma unroll
    for (int p = 0; p < 16; ++p) { const u32x4 t = *(const u32x4*)(s + (p >> 1) * 16 * KS + (p & 1) * 32); v[4 * p] = t.x; v[4 * p + 1] = t.y; v[4 * p + 2] = t.z; v[4 * p + 3] = t.w; }
}
__device__ __forceinline__ void puller(const Params& P, u32 wid, u32 lane) {
    const u32 q = (P.rank + 1 + blockIdx.x % 7) & 7;
    u32 call;
    if (wid == 7 && blockIdx.x < (u32)NRAW) {
        const u32 gid = blockIdx.x >> 3, w = blockIdx.x & 7;
        u32 v[64], Mr[32], Er[32]; const u32 Cr[8] = {};
        load_x(P, gid, w, lane, v); call = call_take(wid); split(v, Mr, Er);
        pack_store(P.ws[P.rank] + WS_STG + (call & 1) * STG_PAR + (size_t)gid * GRP_STRIDE + w * RAWB + lane * 16, RAWCODE, Mr, Er, Cr, 0u, lane);
        WAIT_VM(0);
        if (lane < (u32)NR && lane != (u32)P.rank) raise_flag(P, lane, gid, w, call, RAWCODE);
        store_image(P, gid, w, P.rank, lane, Mr, Er);
    }
    call = call_take(wid);
    call_bump(P, wid, lane, call);
    if (wid == 0) {
        __builtin_amdgcn_s_setprio(3);
        const u64 b64 = (u64)(P.ws[q] + WS_STG + (call & 1) * STG_PAR);
        const char* base = (const char*)(((u64)rfl((u32)(b64 >> 32)) << 32) | rfl((u32)b64));
        const char* dummy = P.ws[P.rank] + WS_TMO + 1024 + lane * 16;
        const u32 voff = lane * 16;
        u32 pos = 0, pub = 0, limit = RING / 1024;
        u64 tk = 0, c_full = 0, c_idle = 0, n_full = 0, n_ref = 0; (void)tk; (void)c_full; (void)c_idle; (void)n_full; (void)n_ref;
        for (u32 ui = 0;; ++ui) {
            u32 nu = rfl(lds32v(CTL_NUNIT));
            if ((nu & ~DONE) <= ui) {
                TICK(tk);
                WAIT_VM(0); lds32vw(CTL_LANDED, pos); pub = pos;
                while (nu <= ui) { __builtin_amdgcn_s_sleep(1); nu = rfl(lds32v(CTL_NUNIT)); }
                if ((nu & ~DONE) <= ui) break;
                ACC(c_idle, tk);
            }
            CBAR();
            const u32x2 un = lds64v(D_UNIT + ui * 8);
            const u32 ut = rfl(un.y), total = ut & 0x3ffffu, lastc = ut >> 18;
            const char* src = base + rfl(un.x);
            for (u32 o = 0; o < total; o += 4096) {
                if (pos + 4 > limit) {
                    bool drained = false; TICK(tk); ++n_ref;
                    for (;;) {
                        const u32 nx = lds32v(CTL_NEXT + (lane & 7) * 4);
                        u32 f = lastc;
#pragma unroll
                        for (u32 j = 0; j < ND; ++j) f = umin(f, (u32)__builtin_amdgcn_readlane((int)nx, j));
                        limit = (rfl(lds32v(D_DESC + f * 8)) >> 14) + RING / 1024;
                        if (pos + 4 <= limit) break;
                        if (!drained) { WAIT_VM(0); lds32vw(CTL_LANDED, pos); pub = pos; drained = true; }
                        __builtin_amdgcn_s_sleep(1);
                    }
                    ACC(c_full, tk); n_full += drained;
                }
                const u32 dst = (pos * 1024) & RMASK;
                if (o + 4096 <= total) {
                    u32 keep;
                    asm volatile("s_mov_b32 %0, m0\n\ts_mov_b32 m0, %3\n\ts_nop 4\n\t"
                                 "global_load_lds_dwordx4 %1, %2 sc0 sc1\n\ts_waitcnt vmcnt(%4)\n\t"
                                 "global_load_lds_dwordx4 %1, %2 offset:1024 sc0 sc1\n\ts_waitcnt vmcnt(%4)\n\t"
                                 "global_load_lds_dwordx4 %1, %2 offset:2048 sc0 sc1\n\ts_waitcnt vmcnt(%4)\n\t"
                                 "global_load_lds_dwordx4 %1, %2 offset:3072 sc0 sc1\n\ts_waitcnt vmcnt(%4)\n\t"
                                 "s_mov_b32 m0, %0" : "=&s"(keep) : "v"(voff), "s"(src + o), "s"(dst), "i"(WIN) : "memory");
                } else {
#pragma unroll
                    for (u32 j = 0; j < 4; ++j) { const u32 b = o + j * 1024 + voff; glds_sys(b < total ? src + b : dummy, dst + j * 1024); WAIT_VM(WIN); }
                }
                pos += 4;
                if (pos > pub + WIN) { pub = pos - WIN; lds32vw(CTL_LANDED, pub); }
            }
            STAMP(P, 128 + umin(ui, 127)); DUMP(P, 64 + umin(ui, 127), total);
        }
        DUMP(P, 0, c_full); DUMP(P, 1, n_full); DUMP(P, 2, n_ref); DUMP(P, 3, c_idle);
    } else if (wid == 1) {
        u32 basek = 0, nch = 0, nun = 0;
        u32* ticket = (u32*)(P.ws[P.rank] + WS_CLAIM + q * 128);
        for (u32 ui = 0;; ++ui) {
            while (rfl(lds32v(CTL_LANDED)) + LEAD < basek) __builtin_amdgcn_s_sleep(4);
            u32 idx = ui * PW + blockIdx.x / 7;
            if (ui >= (u32)NSTAT) {
                if (lane == 0) idx = __hip_atomic_fetch_add(ticket, 1u, __ATOMIC_RELAXED, __HIP_MEMORY_SCOPE_AGENT);
                idx = rfl(idx) - (call - 1) * (u32)(P.nunits - NSTAT * PW + PW) + NSTAT * PW;
                if (idx >= (u32)P.nunits) break;
            }
            const u32 e = tab16(P.stream, idx), gid = e & 1023u, part = e >> 10, lv = 31 - __builtin_clz(part), n = 8u >> lv, first = (part - (1u << lv)) * n;
            const bool solo = gid < (u32)SOLOG, mine = lane >= first && lane < first + n;
            const u32* f = (const u32*)(P.ws[P.rank] + WS_FLAG + (call & 1) * FLAG_PAR + (size_t)q * FLAG_SRC + gid * 32) + (lane & 7);
            const u64 t0 = now(); u32 val; bool dead = false;
            for (;;) {
                val = __hip_atomic_load(f, __ATOMIC_RELAXED, __HIP_MEMORY_SCOPE_SYSTEM);
                if (__all((val >> 8) == (call & 0xffffffu) || lane >= first + n || (solo && lane < first))) break;
                if (__any(expired(P, t0, call))) { dead = true; break; }
                __builtin_amdgcn_s_sleep(1);
            }
            if (dead) {
                const u32 t = gid - GY;
                if (lane == 0 && t < (u32)(16 * DP)) *y_base(P, t & 15, q * DP + (t >> 4), first, 0u) = 0x7fc0;
                break;
            }
            u32 code = val & 255u; if (code > 127u) code = RAWCODE;
            const u32 cb = chunk_bytes(code), sz = solo ? (mine ? (cb + 4095u) & ~4095u : 0u) : (lane < first + n ? cb : 0u);
            u32 inc = sz; inc += dpp<0x111, 0xf, true>(inc); inc += dpp<0x112, 0xf, true>(inc); inc += dpp<0x114, 0xf, true>(inc);
            const u32 a0 = (u32)__builtin_amdgcn_readlane((int)(inc - sz), (int)first) & ~127u, total = (u32)__builtin_amdgcn_readlane((int)inc, 7) - a0;
            if (mine) lds64vw(D_DESC + (nch + lane - first) * 8, (u32x2){((basek * 1024 + inc - sz - a0) << 4) | code, gid * 8 + lane});
            if (solo) { if (mine) lds64vw(D_UNIT + (nun + lane - first) * 8, (u32x2){gid * (u32)GRP_STRIDE + lane * RAWB, cb | ((nch + lane - first) << 18)}); nun += n; }
            else { if (lane == 0) lds64vw(D_UNIT + nun * 8, (u32x2){gid * (u32)GRP_STRIDE + a0, total | ((nch + n - 1) << 18)}); nun += 1; }
            nch += n;
            WAIT_LGKM0();
            lds32vw(CTL_NUNIT, nun); lds32vw(CTL_NCHUNK, nch);
            basek += ((total + 4095) >> 12) << 2;
            STAMP(P, umin(ui, 127));
        }
        lds32vw(CTL_NUNIT, nun | DONE); lds32vw(CTL_NCHUNK, nch | DONE);
    } else {
        const u32 dw = wid - 2, scr = D_SCR + dw * D_SCRB;
        u64 tk = 0, c_wait = 0, c_dec = 0, c_st = 0; (void)c_wait; (void)c_dec; (void)c_st; TICK(tk);
        for (u32 c = dw;; c += ND) {
            u32 nc = rfl(lds32v(CTL_NCHUNK));
            while (nc <= c) { __builtin_amdgcn_s_sleep(2); nc = rfl(lds32v(CTL_NCHUNK)); }
            if ((nc & ~DONE) <= c) break;
            CBAR();
            const u32x2 de = lds64v(D_DESC + c * 8);
            const u32 d = rfl(de.x), code = d & 255u, S = (d >> 8) << 4, gid = rfl(de.y) >> 3, w = rfl(de.y) & 7;
            const u32 need = (S + chunk_bytes(code) + 1023) >> 10;
            while (rfl(lds32v(CTL_LANDED)) < need) __builtin_amdgcn_s_sleep(1);
            CBAR();
            u32 Mr[32], Er[32];
            CS_DECL
            ACC(c_wait, tk);
            unpack_chunk(S, code, scr, lane, CTL_NEXT + dw * 4, c + ND, Mr, Er CS_PASS);
            ACC(c_dec, tk);
            if (gid >= (u32)GY) {
                const u32 t = gid - GY;
                u16* o = y_base(P, t & 15, q * DP + (t >> 4), w, lane);
#pragma unroll
                for (int p = 0; p < 16; ++p) st_out<GAR_OST>(o + y_piece_off(p), piece(Mr, Er, p));
            } else store_image(P, gid, w, q, lane, Mr, Er);
            ACC(c_st, tk);
        }
        DUMP(P, wid * 4, c_wait); DUMP(P, wid * 4 + 1, c_dec); DUMP(P, wid * 4 + 2, c_st); DUMP(P, wid * 4 + 3, now());
    }
}

__device__ __forceinline__ void pack_inputs(const Params& P, u32 gid, u32 w, bool solo, u32 wid, u32 lane) {
    asm volatile("" : "+v"(lane));
    u32 v[64], Mr[32], Er[32];
    u32 peers = 255u & ~(1u << P.rank); bool own = true;
    if (gid < (u32)GW) load_x(P, gid, w, lane, v);
    else {
        const u32 j = (gid - GW) >> 1, ktl = (gid & 1) * 4 + (w >> 1);
        if (j < (u32)(NR * DP)) { const u32 dest = j / DP; own = dest == (u32)P.rank; peers = own ? 0u : 1u << dest; }
        const u16* s = P.w + (size_t)(ktl * 64 + (lane >> 4)) * NN + j * 256 + (w & 1) * 128 + (lane & 15) * 8;
        const u32 scr = wid * 16384;
#pragma unroll
        for (int i = 0; i < 16; ++i) glds_plain(s + (size_t)i * 4 * NN, scr + i * 1024);
        WAIT_VM(0);
        const u32 ta = scr + ((lane >> 4) * 8 + ((lane & 15) >> 2)) * 256 + (lane & 3) * 16;
#pragma unroll
        for (int p = 0; p < 16; ++p) {
            const u32 a = ta + (p & 1) * 32 * 256 + (p >> 2) * 64 + ((p >> 1) & 1) * 8;
            const u32x2 lo = __builtin_bit_cast(u32x2, __builtin_amdgcn_ds_read_tr16_b64_v4i16((LAS s16x4*)a));
            const u32x2 hi = __builtin_bit_cast(u32x2, __builtin_amdgcn_ds_read_tr16_b64_v4i16((LAS s16x4*)(a + 4 * 256)));
            v[4 * p] = lo.x; v[4 * p + 1] = lo.y; v[4 * p + 2] = hi.x; v[4 * p + 3] = hi.y;
        }
        WAIT_LGKM0();
    }
    const u32 call = call_take(wid);
    if (peers) pack_group(P, v, Mr, Er, gid, w, solo, peers, call, wid, lane); else split(v, Mr, Er);
    if (own) store_image(P, gid, w, P.rank, lane, Mr, Er);
}

typedef __bf16 bf16x2 __attribute__((ext_vector_type(2)));
typedef float f32x2 __attribute__((ext_vector_type(2)));
__device__ __forceinline__ u32 cvt_pk(float lo, float hi) { return __builtin_bit_cast(u32, __builtin_convertvector((f32x2){lo, hi}, bf16x2)); }
__device__ __forceinline__ void wait_panels(const Params& P, u32 tile, u32 call, u32 wid, u32 lane) {
    if (wid == 0) {
        const u32* c = counter(P, (lane & 1) ? 16 + (tile >> 4) : (tile & 15));
        const u64 t0 = now();
        for (;;) {
            const u32 x = __hip_atomic_load(c, __ATOMIC_RELAXED, __HIP_MEMORY_SCOPE_AGENT);
            if (__all((int)(x - call * 128u) >= 0)) break;
            if (__any(expired(P, t0, call))) { asm volatile("ds_write_b32 %0, %1\n\ts_waitcnt lgkmcnt(0)" :: "v"(CTL_BAD), "v"(1u) : "memory"); break; }
            __builtin_amdgcn_s_sleep(4);
        }
        __builtin_amdgcn_fence(__ATOMIC_ACQUIRE, "agent");
        WAIT_VM(0);
    }
    BAR();
}
__device__ __forceinline__ void gemm_role(const Params& P, unsigned char* lds_generic, u32 call, u32 wid, u32 lane) {
    LAS unsigned char* lds = (LAS unsigned char*)lds_generic;
    const u32 gw = blockIdx.x - GEMM0, tid = threadIdx.x;
    const u32 wr = wid >> 2, wc = wid & 3, fr = lane & 15, fq = lane >> 4;
    constexpr int HTB = 16384;
    const u32 voff = tid * 16, ldsw = wid * 1024;
    const u32 fo = swz(fr * 64 + fq * 16);
    const u32 aoff = wr * 8192 + fo, boff = wc * 4096 + fo;
#define G_SA(b, h) (((b) * 2 + (h)) * HTB)
#define G_SB(b, h) ((4 + (b) * 2 + (h)) * HTB)
#define G_STAGE(bufoff, gbase) do { _Pragma("unroll") for (int _i = 0; _i < 2; ++_i) \
        __builtin_amdgcn_global_load_lds((const u32*)((const char*)(gbase) + voff + _i * 8192), (LAS u32*)(lds + (bufoff) + ldsw + _i * 8192), 16, 0, 0); } while (0)
#define G_LDA(dst, b, h) do { _Pragma("unroll") for (int m = 0; m < 4; ++m) _Pragma("unroll") for (int k = 0; k < 2; ++k) dst[m][k] = *(const LAS bf16x8*)(lds + G_SA(b, h) + aoff + m * 2048 + k * 1024); } while (0)
#define G_LDB(dst, b, h) do { _Pragma("unroll") for (int n = 0; n < 2; ++n) _Pragma("unroll") for (int k = 0; k < 2; ++k) dst[n][k] = *(const LAS bf16x8*)(lds + G_SB(b, h) + boff + n * 2048 + k * 1024); } while (0)
#define G_MMA(ai, bj, At, Bt) do { __builtin_amdgcn_s_setprio(1); _Pragma("unroll") for (int m = 0; m < 4; ++m) _Pragma("unroll") for (int n = 0; n < 2; ++n) _Pragma("unroll") for (int k = 0; k < 2; ++k) \
        acc[ai][bj][m][n] = __builtin_amdgcn_mfma_f32_16x16x32_bf16(Bt[n][k], At[m][k], acc[ai][bj][m][n], 0, 0, 0); __builtin_amdgcn_s_setprio(0); } while (0)
#define G_WV(n) asm volatile("s_waitcnt vmcnt(" #n ")" ::: "memory")
#define G_WL(n) asm volatile("s_waitcnt lgkmcnt(" #n ")" ::: "memory")
#define G_BAR __builtin_amdgcn_s_barrier()
#define G_SCHED __builtin_amdgcn_sched_barrier(0)
    const char* XI = P.ws[P.rank] + WS_XI; const char* WI = P.ws[P.rank] + WS_WI;
    constexpr size_t kstep = 32768, hstep = 16384;
    u32 cur = tab16(P.tiles, gw);
    if (cur == 0xffffu) return;
    bool fresh = true;
    bf16x8 At[4][2], B0[2][2], B1[2][2];
    for (u32 ui = 0;; ++ui) {
        const u32 nxt = ui + 1 < (u32)ROUNDS ? tab16(P.tiles, (ui + 1) * NGEMM + gw) : 0xffffu;
        const bool has_next = nxt != 0xffffu, dist = (cur >> 4) < (u32)DP, chain = has_next && !dist;
        const char* cA = XI + (size_t)(cur & 15) * PANEL; const char* cB = WI + (size_t)(cur >> 4) * PANEL;
        if (fresh) {
            wait_panels(P, cur, call, wid, lane);
            STAMP(P, 8 + ui * 4); if (ui == 0) DUMP(P, 1, __builtin_amdgcn_s_memtime());
            G_STAGE(G_SB(0, 0), cB); G_STAGE(G_SA(0, 0), cA); G_STAGE(G_SB(0, 1), cB + hstep); G_STAGE(G_SA(0, 1), cA + hstep);
            if (wr == 1) G_BAR;
            G_WV(4); G_BAR;
            G_STAGE(G_SB(1, 0), cB + kstep); G_STAGE(G_SA(1, 0), cA + kstep); G_STAGE(G_SB(1, 1), cB + hstep + kstep);
            G_WV(6); G_BAR;
        }
        f32x4 acc[2][2][4][2];
#pragma unroll
        for (int a = 0; a < 2; ++a)
#pragma unroll
            for (int b = 0; b < 2; ++b)
#pragma unroll
                for (int m = 0; m < 4; ++m)
#pragma unroll
                    for (int n = 0; n < 2; ++n) acc[a][b][m][n] = (f32x4){0.f, 0.f, 0.f, 0.f};
        const char* nA = chain ? XI + (size_t)(nxt & 15) * PANEL : cA; const char* nB = chain ? WI + (size_t)(nxt >> 4) * PANEL : cB;
        for (int t = 0; t < 64; t += 2) {
            const bool last = (t == 62);
            const char* a1 = cA + (size_t)(t + 1) * kstep;
            const char* a2 = last ? nA : cA + (size_t)(t + 2) * kstep; const char* b2 = last ? nB : cB + (size_t)(t + 2) * kstep;
            const char* a3 = a2 + kstep; const char* b3 = b2 + kstep;
            if (last && chain) wait_panels(P, nxt, call, wid, lane);
            G_LDB(B0, 0, 0); G_SCHED; G_LDA(At, 0, 0); G_STAGE(G_SA(1, 1), a1 + hstep);
            G_WL(8); G_BAR; G_WL(0); G_MMA(0, 0, At, B0); G_BAR; G_SCHED;
            G_LDB(B1, 0, 1); G_STAGE(G_SB(0, 0), b2);
            G_BAR; G_WL(0); G_MMA(0, 1, At, B1); G_BAR;
            G_LDA(At, 0, 1); G_STAGE(G_SA(0, 0), a2);
            G_BAR; G_WL(0); G_MMA(1, 0, At, B0); G_BAR; G_SCHED;
            G_STAGE(G_SB(0, 1), b2 + hstep);
            G_WV(6); G_BAR; G_MMA(1, 1, At, B1); G_BAR;
            G_LDB(B0, 1, 0); G_SCHED; G_LDA(At, 1, 0); G_STAGE(G_SA(0, 1), a2 + hstep);
            G_WL(8); G_BAR; G_WL(0); G_MMA(0, 0, At, B0); G_BAR; G_SCHED;
            G_LDB(B1, 1, 1); G_STAGE(G_SB(1, 0), b3);
            G_BAR; G_WL(0); G_MMA(0, 1, At, B1); G_BAR;
            G_LDA(At, 1, 1); G_STAGE(G_SA(1, 0), a3);
            G_BAR; G_WL(0); G_MMA(1, 0, At, B0); G_BAR; G_SCHED;
            G_STAGE(G_SB(1, 1), b3 + hstep);
            G_WV(6); G_BAR; G_MMA(1, 1, At, B1); G_BAR;
        }
        STAMP(P, 9 + ui * 4); DUMP(P, 2 + ui, __builtin_amdgcn_s_memtime());
        u32 v[64];
        {
            int tz = threadIdx.x; asm volatile("" : "+v"(tz));
            const u32 ln = tz & 63;
            u32 bad; asm volatile("ds_read_b32 %0, %1\n\ts_waitcnt lgkmcnt(0)" : "=v"(bad) : "v"(CTL_BAD) : "memory");
            u16* o = y_base(P, cur & 15, global_panel(cur >> 4, P.rank), wid, ln);
#pragma unroll
            for (int p = 0; p < 16; p += 2) {
                const int ai = p >> 3, m = (p >> 1) & 3;
#pragma unroll
                for (int i = 0; i < 4; ++i) {
                    const f32x4 xa = acc[ai][0][m][i >> 1], xb = acc[ai][1][m][i >> 1];
                    u32 a = cvt_pk(xa[(i & 1) * 2], xa[(i & 1) * 2 + 1]), b = cvt_pk(xb[(i & 1) * 2], xb[(i & 1) * 2 + 1]);
                    if (bad) a = b = 0x7fc07fc0u;
                    v[4 * p + i] = (u32)__builtin_amdgcn_update_dpp((int)a, (int)b, 0x128, 0xf, 0xc, false);
                    v[4 * p + 4 + i] = (u32)__builtin_amdgcn_update_dpp((int)b, (int)a, 0x128, 0xf, 0x3, false);
                }
                st_out<GAR_GST>(o + y_piece_off(p), (u32x4){v[4 * p], v[4 * p + 1], v[4 * p + 2], v[4 * p + 3]});
                st_out<GAR_GST>(o + y_piece_off(p + 1), (u32x4){v[4 * p + 4], v[4 * p + 5], v[4 * p + 6], v[4 * p + 7]});
            }
            if (!chain) {
                G_WV(0);
                if (wr == 0) G_BAR;
                BAR();
                if (dist) { u32 Mr[32], Er[32]; pack_group(P, v, Mr, Er, GY + cur, wid, false, 255u & ~(1u << P.rank), call, wid, ln); }
                STAMP(P, 10 + ui * 4);
                if (!has_next) break;
            }
        }
        fresh = !chain; cur = nxt;
    }
}

__global__ void __launch_bounds__(512, 2) fused(const Params P) {
    extern __shared__ __attribute__((aligned(16))) unsigned char lds[];
    const u32 wid = rfl(threadIdx.x >> 6), lane = threadIdx.x & 63;
    call_load(P, wid, lane);
    if (blockIdx.x < (u32)NPULL) {
        if (threadIdx.x == 0) { lds32vw(CTL_NUNIT, 0u); lds32vw(CTL_LANDED, 0u); lds32vw(CTL_NCHUNK, 0u); }
        if (wid == 1 && lane < 8) lds32vw(CTL_NEXT + lane * 4, lane);
        WAIT_LGKM0(); BAR();
        puller(P, wid, lane); return;
    }
    STAMP(P, 0); DUMP(P, 0, __builtin_amdgcn_s_memtime());
    if (threadIdx.x == 0) lds32vw(CTL_BAD, 0u);
    pack_init(wid, lane);
    const u32 sc = NRAW + blockIdx.x - NPULL;
    if (sc >= (u32)(8 * SOLOG)) { const u64 t0 = now(); while (now() - t0 < GAR_HOLD) __builtin_amdgcn_s_sleep(2); }
    for (u32 it = 0;; ++it) {
        const u32 e = NWG - 1 - blockIdx.x + (it - 1) * NENC;
        if (it && e >= (u32)NPACK) break;
        if (it || (wid == 1 && sc < (u32)(8 * SOLOG))) pack_inputs(P, it ? tab16(P.enc, e) : sc >> 3, it ? wid : sc & 7, !it, wid, lane);
        WAIT_LGKM0(); BAR(); STAMP(P, it ? it : 3);
    }
    const u32 call = call_take(wid);
    call_bump(P, wid, lane, call);
    if (blockIdx.x >= (u32)GEMM0) gemm_role(P, lds, call, wid, lane);
}

static void build_tables(Params& P) {
    u16* tiles = (u16*)P.tiles; u16* stream = (u16*)P.stream; u16* enc = (u16*)P.enc;
    static const int RG[8] = {0, 1, 0, 1, 2, 2, 3, 3}, CG[8] = {0, 0, 1, 1, 0, 1, 0, 1};
    for (int i = 0; i < ROUNDS * NGEMM; ++i) tiles[i] = 0xffff;
    const int full = LP / 12, rest = LP - full * 12;
    for (int r = 0; r < full; ++r) for (int l = 0; l < 8; ++l) for (int i = 0; i < 24; ++i)
        tiles[r * NGEMM + i * 8 + l] = (u16)((RG[l] * 4 + (i & 3)) | (r * 12 + CG[l] * 6 + (i >> 2)) << 4);
    {
        int l = 0, i = 0;
        for (int c0 = 0; c0 < rest; c0 += 6) for (int rg = 0; rg < 4; ++rg) for (int c = c0; c < c0 + 6 && c < rest; ++c) for (int m = 0; m < 4; ++m) {
            tiles[full * NGEMM + i * 8 + l] = (u16)((rg * 4 + m) | (full * 12 + c) << 4);
            if (++i == 24) { i = 0; ++l; } }
    }
    int in[16 + LP], n = 0;
    for (int i = 0; i < 8; ++i) in[n++] = i;
    for (int i = 0; i < 12; ++i) in[n++] = 16 + i;
    for (int i = 8; i < 16; ++i) in[n++] = i;
    for (int i = 12; i < LP; ++i) in[n++] = 16 + i;
    int g[NS], ng = 0, k = 0;
    for (int i = 0; i < n; ++i) for (int h = 0; h < 2; ++h) {
        if (in[i] < 16) { g[ng++] = GX + 2 * in[i] + h; if (2 * in[i] + h >= SOLOG) enc[k++] = (u16)(GX + 2 * in[i] + h); continue; }
        const int jj = in[i] - 16;
        if (jj >= DP) { g[ng++] = GW + 2 * (jj + (NR - 1) * DP) + h; enc[k++] = (u16)g[ng - 1]; continue; }
        g[ng++] = GW + 2 * (P.rank * DP + jj) + h;
        for (int d = 1; d <= NR; ++d) enc[k++] = (u16)(GW + 2 * (((P.rank + d) & 7) * DP + jj) + h);
    }
    for (int r = 0; r < ROUNDS; ++r) for (int l = 0; l < 8; ++l) for (int i = 0; i < 24; ++i) { const int t = tiles[r * NGEMM + i * 8 + l]; if (t != 0xffff && (t >> 4) < DP) g[ng++] = GY + t; }
    k = 0;
    for (int i = 0; i < NS; ++i) {
        const int left = NS - i; int lv = 0;
        if (i < SOLOG) lv = 2; else if (i < SOLOG + 6) lv = 1;
        if (left <= 2) lv = 3; else if (left <= 6) lv = 2; else if (left <= 14) lv = 1;
        for (int j = 0; j < (1 << lv); ++j) stream[k++] = (u16)(g[i] | ((1 << lv) + j) << 10);
    }
    P.nunits = k;
}
}

extern "C" void kernel_launch(int rank, int nranks, ncclComm_t comm, hipStream_t stream,
                              void* const* d_in, const int* in_sizes, int n_in,
                              void* d_out, int out_size, const int* in_dtypes, int out_dtype,
                              void* const* d_out_all, void* const* d_ws_all, size_t ws_bytes) {
    using namespace gar;
    if (nranks != NR || (unsigned)rank >= (unsigned)NR || n_in != 2 || ws_bytes < WS_END || in_sizes[0] != MM * KS || in_sizes[1] != KS * NN || out_size != MM * NN) {
        fprintf(stderr, "kernel_launch: unexpected shapes\n"); return; }
    Params P = {};
    for (int p = 0; p < NR; ++p) P.ws[p] = (char*)d_ws_all[p];
    P.x = (const u16*)d_in[0]; P.w = (const u16*)d_in[1]; P.out = (u16*)d_out; P.rank = rank;
    build_tables(P);
    HIP_CHECK(hipFuncSetAttribute((const void*)fused, hipFuncAttributeMaxDynamicSharedMemorySize, LDS_BYTES));
    hipLaunchKernelGGL(fused, dim3(NWG), dim3(512), LDS_BYTES, stream, P);
    HIP_CHECK(hipGetLastError());
}
```

```cpp
#include <hip/hip_runtime.h>
#include <rccl/rccl.h>
#include <cstdio>
#include <cstdint>
#pragma clang diagnostic ignored "-Wint-to-pointer-cast"

#define HIP_CHECK(expr) do { hipError_t _e = (expr); if (_e != hipSuccess) { \
    fprintf(stderr, "HIP error %s at %s:%d\n", hipGetErrorString(_e), __FILE__, __LINE__); \
    return; } } while (0)

namespace gar {
#define LAS __attribute__((address_space(3)))
#define GAS __attribute__((address_space(1)))
typedef unsigned u32;
typedef unsigned short u16;
typedef unsigned long long u64;
typedef unsigned u32x4 __attribute__((ext_vector_type(4)));
typedef unsigned u32x2 __attribute__((ext_vector_type(2)));
typedef short s16x4 __attribute__((ext_vector_type(4)));
typedef short bf16x8 __attribute__((ext_vector_type(8)));
typedef float f32x4 __attribute__((ext_vector_type(4)));

constexpr int NR = 8, MM = 4096, NN = 32768, KS = 512;
#define GAR_DP 11
constexpr int DP = GAR_DP;
constexpr int RP = 128 - NR * DP;
constexpr int LP = DP + RP;
constexpr int NWG = 256;
#define GAR_PW 9
#define GAR_WIN 6
constexpr int PW = GAR_PW;
constexpr int NPULL = 7 * PW;
constexpr int WIN = GAR_WIN;
constexpr int GEMM0 = 64;
constexpr int NGEMM = NWG - GEMM0;
static_assert(NPULL <= GEMM0, "roles");
#define GAR_SP2 1
#define GAR_TAILRAW 0
#define GAR_ALIGN 1
#define GAR_NRAW 18
#define GAR_SOLOG 16
#define GAR_LEAD 60
#define GAR_HOLD 200
constexpr int SOLOG = GAR_SOLOG, NRAW = GAR_NRAW;
constexpr int LEAD = GAR_LEAD;
static_assert(NRAW <= NPULL && NRAW <= 8 * SOLOG && 8 * SOLOG - NRAW <= NWG - NPULL && SOLOG == 16, "solo chunks");
constexpr int NENC = NWG - NPULL;
constexpr int GX = 0, GW = 32, GY = 288;
constexpr int NGID = GY + 16 * DP;
constexpr int NS = 32 + 2 * DP + 2 * RP + 16 * DP;
constexpr int NTILE = 16 * LP;
constexpr int ROUNDS = (LP + 11) / 12;
constexpr int NUMAX = 544;
#define GAR_NLIGHT 80
constexpr int NLIGHT = GAR_NLIGHT;
constexpr int NSTAT = 3;
constexpr int NPACK = 288 - SOLOG;
constexpr u64 DEADLINE = 30000000ull;

constexpr size_t WS_CALL = 0;
constexpr size_t WS_TMO = 32768;
constexpr size_t WS_CNT = 36864;
constexpr size_t WS_CLAIM = 57344;
constexpr size_t WS_FLAG = 65536;
constexpr size_t FLAG_SRC = 544 * 32, FLAG_PAR = 8 * FLAG_SRC;
constexpr size_t WS_DBG = 524288;
constexpr size_t GRP_STRIDE = 131072;
constexpr size_t WS_STG = 2u << 20;
constexpr size_t STG_PAR = 544 * GRP_STRIDE;
constexpr size_t WS_XI = WS_STG + 2 * STG_PAR;
constexpr size_t PANEL = 64 * 32768;
constexpr size_t WS_WI = WS_XI + 16 * PANEL;
constexpr size_t WS_END = WS_WI + (size_t)LP * PANEL;
static_assert(WS_FLAG + 2 * FLAG_PAR <= WS_DBG && NGID <= 544, "layout");

constexpr u32 LDS_CTL = 131072;
constexpr u32 LDS_LUT = LDS_CTL + 512, LDS_HIST = LDS_CTL + 1024;
constexpr u32 LDS_BYTES = LDS_HIST + 4096;
constexpr u32 CTL_CALL = LDS_CTL  , CTL_SIZE = LDS_CTL + 64  , CTL_BAD = LDS_CTL + 128;
constexpr u32 CTL_NUNIT = LDS_CTL + 192, CTL_LANDED = LDS_CTL + 256, CTL_NCHUNK = LDS_CTL + 320, CTL_NEXT = LDS_CTL + 384  ;
constexpr u32 E_L2 = 0, E_L3 = E_L2 + 4160, E_VAR = E_L3 + 2112, E_END = E_VAR + 2112;
static_assert(E_END <= 9216 && E_L3 % 16 == 0 && E_VAR % 16 == 0, "packer scratch");
constexpr u32 CAP2 = 4080, CAP3 = 2032, CAPV = 2032, FIXB = 10240, RAWB = 16384, RAWCODE = 255;
constexpr u32 RING = 65536, RMASK = RING - 1, ND = 6;
constexpr u32 D_SCR = RING, D_SCRB = 6400, D_L2 = 0, D_L3 = 4160;
constexpr u32 D_DESC = D_SCR + ND * D_SCRB, D_UNIT = D_DESC + NS * 64, DONE = 0x80000000u;
static_assert(D_L3 + 2112 <= D_SCRB && D_UNIT + (NUMAX + 8 * SOLOG) * 8 <= LDS_CTL && D_DESC % 16 == 0, "puller LDS");

struct Params {
    char* ws[NR];
    const u16* x; const u16* w; u16* out;
    int rank, nunits;
    u32 stream[NUMAX / 2];
    u32 enc[144];
    u32 tiles[ROUNDS * NGEMM / 2];
};
__device__ __forceinline__ u32 tab16(const u32* t, u32 i) { return (t[i >> 1] >> ((i & 1) * 16)) & 0xffffu; }
static_assert(sizeof(Params) <= 4096, "kernarg");

__device__ __forceinline__ u32 lds32(u32 a) { return *(LAS u32*)a; }
__device__ __forceinline__ u32 lds32v(u32 a) { return *(volatile LAS u32*)a; }
__device__ __forceinline__ void lds32vw(u32 a, u32 x) { *(volatile LAS u32*)a = x; }
__device__ __forceinline__ u32x2 lds64v(u32 a) { return *(volatile LAS u32x2*)a; }
__device__ __forceinline__ void lds64vw(u32 a, u32x2 x) { *(volatile LAS u32x2*)a = x; }
__device__ __forceinline__ u32x4 lds128(u32 a) { return *(LAS u32x4*)a; }
__device__ __forceinline__ void lds128w(u32 a, u32x4 x) { *(LAS u32x4*)a = x; }
__device__ __forceinline__ void lds_or(u32 a, u32 x) { __hip_atomic_fetch_or((LAS u32*)a, x, __ATOMIC_RELAXED, __HIP_MEMORY_SCOPE_WORKGROUP); }
__device__ __forceinline__ void lds_add(u32 a, u32 x) { __hip_atomic_fetch_add((LAS u32*)a, x, __ATOMIC_RELAXED, __HIP_MEMORY_SCOPE_WORKGROUP); }
__device__ __forceinline__ u32 rfl(u32 x) { return (u32)__builtin_amdgcn_readfirstlane((int)x); }
#define CBAR() asm volatile("" ::: "memory")
#define WAIT_LGKM0() asm volatile("s_waitcnt lgkmcnt(0)" ::: "memory")
#define WAIT_VM(n) asm volatile("s_waitcnt vmcnt(%0)" :: "i"(n) : "memory")
#define BAR() do { CBAR(); __builtin_amdgcn_s_barrier(); CBAR(); } while (0)
__device__ __forceinline__ void st_sys(void* p, u32x4 x) { asm volatile("global_store_dwordx4 %0, %1, off sc0 sc1\n\ts_nop 1" :: "v"(p), "v"(x) : "memory"); }
__device__ __forceinline__ void st_dev(void* p, u32x4 x) { asm volatile("global_store_dwordx4 %0, %1, off sc1\n\ts_nop 1" :: "v"(p), "v"(x) : "memory"); }
#define GAR_OST 3
#define GAR_GST 1
template <int MODE> __device__ __forceinline__ void st_out(void* p, u32x4 x) {
    if (MODE == 0) *(u32x4*)p = x;
    else if (MODE == 1) __builtin_nontemporal_store(x, (u32x4*)p);
    else if (MODE == 2) st_dev(p, x);
    else if (MODE == 9) asm volatile("" :: "v"(p), "v"(x) : "memory");
    else asm volatile("global_store_dwordx4 %0, %1, off sc1 nt\n\ts_nop 1" :: "v"(p), "v"(x) : "memory");
}
__device__ __forceinline__ void glds_sys(const void* g, u32 dst) {
    u32 keep;
    asm volatile("s_mov_b32 %0, m0\n\ts_mov_b32 m0, %2\n\ts_nop 0\n\tglobal_load_lds_dwordx4 %1, off sc0 sc1\n\ts_mov_b32 m0, %0" : "=&s"(keep) : "v"(g), "s"(dst) : "memory");
}
__device__ __forceinline__ void glds_plain(const void* g, u32 dst) {
    u32 keep;
    asm volatile("s_mov_b32 %0, m0\n\ts_mov_b32 m0, %2\n\ts_nop 0\n\tglobal_load_lds_dwordx4 %1, off\n\ts_mov_b32 m0, %0" : "=&s"(keep) : "v"(g), "s"(dst) : "memory");
}
__device__ __forceinline__ u64 now() { return __builtin_amdgcn_s_memrealtime(); }
template <int CTRL, int RM, bool BC> __device__ __forceinline__ u32 dpp(u32 v) { return (u32)__builtin_amdgcn_update_dpp(0, (int)v, CTRL, RM, 0xf, BC); }
__device__ __forceinline__ u32 scan_incl(u32 v) {
    v += dpp<0x111, 0xf, true>(v); v += dpp<0x112, 0xf, true>(v); v += dpp<0x114, 0xf, true>(v); v += dpp<0x118, 0xf, true>(v);
    v += dpp<0x142, 0xa, false>(v); v += dpp<0x143, 0xc, false>(v); return v;
}
__device__ __forceinline__ u32 umax(u32 a, u32 b) { return a > b ? a : b; }
__device__ __forceinline__ u32 umin(u32 a, u32 b) { return a < b ? a : b; }
__device__ __forceinline__ u32 wave_max(u32 v) {
    v = umax(v, dpp<0x111, 0xf, true>(v)); v = umax(v, dpp<0x112, 0xf, true>(v)); v = umax(v, dpp<0x114, 0xf, true>(v)); v = umax(v, dpp<0x118, 0xf, true>(v));
    v = umax(v, dpp<0x142, 0xa, false>(v)); v = umax(v, dpp<0x143, 0xc, false>(v)); return (u32)__builtin_amdgcn_readlane((int)v, 63);
}
__device__ __forceinline__ u32 popc(u32 x) { return (u32)__builtin_popcount(x); }
__device__ __forceinline__ u32 perm(u32 hi, u32 lo, u32 sel) { return __builtin_amdgcn_perm(hi, lo, sel); }
__device__ __forceinline__ u32 swz(u32 ob) { return ob ^ (((ob >> 9) & 1) << 5); }

#define STAMP(P, slot) do {} while (0)
#define TICK(t) do {} while (0)
#define ACC(sum, t) do {} while (0)
#define DUMP(P, slot, val) do {} while (0)

#define CSTAMP(i) do {} while (0)
#define CS_ARG
#define CS_DECL
#define CS_PASS

__device__ __forceinline__ u32 nzb(u32 z) { return ((z & 0x7f7f7f7fu) + 0x7f7f7f7fu) | z; }
typedef unsigned short u16x2 __attribute__((ext_vector_type(2)));
struct Classifier {
    u32 s1, s2, s3, nb, tlo, thi; bool win;
    __device__ __forceinline__ void init(u32 t) {
        const u32 a = (t >> 8) & 255u, b = (t >> 16) & 255u, c = t >> 24, lo = umin(a, umin(b, c));
        win = umax(a, umax(b, c)) - lo <= 6u;
        const u64 l = (1ull << (((a - lo) & 7u) * 8)) | (2ull << (((b - lo) & 7u) * 8)) | (3ull << (((c - lo) & 7u) * 8));
        tlo = (u32)l; thi = (u32)(l >> 32); nb = ((0u - lo) & 255u) * 0x01000100u;
        s1 = a * 0x01010101u; s2 = b * 0x01010101u; s3 = c * 0x01010101u;
    }
    __device__ __forceinline__ u32 any(u32 e) const {
        const u32 a1 = nzb(e ^ s1), a2 = nzb(e ^ s2), a3 = nzb(e ^ s3);
        return ((~(a1 & a3) >> 7) & 0x01010101u) | ((~(a2 & a3) >> 6) & 0x02020202u);
    }
    __device__ __forceinline__ u32 near(u32 e) const {
        const u16x2 x = __builtin_bit_cast(u16x2, e), n = __builtin_bit_cast(u16x2, nb), top = {0x07ff, 0x07ff};
        u32 sh; asm("v_pk_mad_u16 %0, %1, %2, %3" : "=v"(sh) : "v"(e), "v"(0x01000100u), "s"(nb));
        const u16x2 hi = __builtin_elementwise_min((u16x2)(x + n), top), lo = __builtin_elementwise_min(__builtin_bit_cast(u16x2, sh), top);
        return perm(thi, tlo, perm(__builtin_bit_cast(u32, hi), __builtin_bit_cast(u32, lo), 0x07030501u));
    }
    template <int N> __device__ __forceinline__ void run(const u32* e, u32* c) const {
        if (win) {
#pragma unroll
            for (int i = 0; i < N; ++i) c[i] = near(e[i]);
        } else {
#pragma unroll
            for (int i = 0; i < N; ++i) c[i] = any(e[i]);
        }
    }
};
__device__ __forceinline__ u32 escbits(u32 c) { return ~(c | (c >> 1)) & 0x55555555u; }
struct Appender {
    u32 lo, fill, addr;
    __device__ __forceinline__ void init(u32 off) { lo = 0; fill = (off & 3u) * 8u; addr = off & ~3u; }
    __device__ __forceinline__ void push(u32 k, u32 mk) {
        const u64 t = (u64)k << fill;
        lo |= (u32)t; fill += popc(mk);
        if (fill >= 32u) { lds_or(addr, lo); lo = (u32)(t >> 32); addr += 4; }
        fill &= 31u;
    }
    __device__ __forceinline__ void flush() { if (lo) lds_or(addr, lo); }
};
__device__ __forceinline__ u32 escmask(u32 c) { return perm(0u, 0xffu, c); }
__device__ __forceinline__ u32 selector(u32 mk) { return lds32(__builtin_amdgcn_udot4(mk & 0x20100804u, 0x01010101u, LDS_LUT, false)); }
template <int R> __device__ __forceinline__ u32 pack_level(u32 src, u32 n, u32 t, u32 cdst, u32 edst, u32 cap, u32 lane) {
    Classifier K; K.init(t);
    if (lane < 16) *(LAS unsigned char*)(src + n + lane) = (unsigned char)K.s1;
    u32 ne = 0;
    for (u32 r = 0; r * 1024 < n; r += R) {
        CBAR();
        u32 q[4 * R], c[4 * R], m[4 * R], l[4 * R], cnt = 0;
#pragma unroll
        for (int i = 0; i < R; ++i) { const u32 p0 = (r + i) * 1024 + lane * 16; const u32x4 x = lds128(src + p0); const bool act = p0 < n;
            q[4 * i] = act ? x.x : K.s1; q[4 * i + 1] = act ? x.y : K.s1; q[4 * i + 2] = act ? x.z : K.s1; q[4 * i + 3] = act ? x.w : K.s1; }
        K.run<4 * R>(q, c);
#pragma unroll
        for (int i = 0; i < 4 * R; ++i) { m[i] = escmask(c[i]); l[i] = selector(m[i]); }
#pragma unroll
        for (int i = 0; i < R; ++i) cnt |= ((popc(m[4 * i]) + popc(m[4 * i + 1]) + popc(m[4 * i + 2]) + popc(m[4 * i + 3])) >> 3) << (16 * i);
        const u32 inc = scan_incl(cnt), tot = (u32)__builtin_amdgcn_readlane((int)inc, 63), exc = inc - cnt;
        if (ne + (tot & 0xffffu) + (tot >> 16) > cap) return ~0u;
#pragma unroll
        for (int i = 0; i < R; ++i) {
            Appender A; A.init(edst + ne + (i ? (tot & 0xffffu) + (exc >> 16) : (exc & 0xffffu)));
#pragma unroll
            for (int j = 0; j < 4; ++j) A.push(perm(0u, q[4 * i + j], l[4 * i + j]), m[4 * i + j]);
            A.flush();
            if ((r + i) * 1024 + lane * 16 < n) *(LAS u32*)(cdst + ((r + i) * 64 + lane) * 4) = c[4 * i] | (c[4 * i + 1] << 2) | (c[4 * i + 2] << 4) | (c[4 * i + 3] << 6);
        }
        ne += (tot & 0xffffu) + (tot >> 16);
    }
    CBAR();
    return ne;
}
template <int N> __device__ __forceinline__ u32 rank_ror(u32 key) { return (dpp<0x120 + N, 0xf, true>(key) > key ? 1u : 0u) + rank_ror<N - 1>(key); }
template <> __device__ __forceinline__ u32 rank_ror<0>(u32) { return 0u; }
__device__ __forceinline__ u32 row_or(u32 x) { x |= dpp<0x128, 0xf, true>(x); x |= dpp<0x124, 0xf, true>(x); x |= dpp<0x122, 0xf, true>(x); x |= dpp<0x121, 0xf, true>(x); return rfl(x); }
__device__ __forceinline__ void split(const u32 (&v)[64], u32 (&Mr)[32], u32 (&Er)[32]) {
#pragma unroll
    for (int g = 0; g < 32; ++g) {
        const u32 a = __builtin_amdgcn_alignbit(v[2 * g], v[2 * g], 31), b = __builtin_amdgcn_alignbit(v[2 * g + 1], v[2 * g + 1], 31);
        Er[g] = perm(b, a, 0x07050301u); Mr[g] = perm(b, a, 0x06040200u);
    }
}
__device__ __forceinline__ u32 pack_chunk(const u32 (&v)[64], u32 (&Mr)[32], u32 (&Er)[32], u32 (&Cr)[8], u32 scr, u32 lane, bool solo CS_ARG) {
    CBAR(); CSTAMP(0);
    const u32 hist = solo ? scr + 10240 : LDS_HIST;
    if (solo) {
#pragma unroll
        for (int j = 0; j < 4; ++j) lds128w(hist + j * 1024 + lane * 16, (u32x4){0u, 0u, 0u, 0u});
        CBAR();
#pragma unroll
        for (int i = 0; i < 12; ++i) lds_add(hist + (((v[i * 5 + 2] >> ((i & 1) * 16 + 3)) & 0xff0u) | ((lane & 3) * 4)), 1u);
    }
#pragma unroll
    for (int i = 0; i < 4; ++i) lds_add(hist + (((v[i * 15 + 5] >> ((i & 1) * 16 + 3)) & 0xff0u) | ((lane & 3) * 4)), 1u);
#pragma unroll
    for (int j = 0; j < 9; ++j) lds128w(scr + j * 1024 + lane * 16, (u32x4){0u, 0u, 0u, 0u});
    WAIT_LGKM0();
    if (!solo) BAR();
    CBAR();
    u32 key = 0;
#pragma unroll
    for (int i = 0; i < 4; ++i) { const u32 bin = i * 64 + lane; const u32x4 h = lds128(hist + bin * 16); key = umax(key, ((h.x + h.y + h.z + h.w) << 8) | bin); }
    split(v, Mr, Er);
    CSTAMP(1);
    { const auto r = __builtin_amdgcn_permlane32_swap(key, key, false, false); key = umax(r[0], r[1]); }
    { const auto r = __builtin_amdgcn_permlane16_swap(key, key, false, false); key = umax(r[0], r[1]); }
    const u32 rank = rank_ror<15>(key), lev = (rank * 11u) >> 5, sym = (key & 255u) << (8 * (rank - 3 * lev + 1));
    const u32 T[3] = {row_or(lev == 0 ? sym : 0u), row_or(lev == 1 ? sym : 0u), row_or(lev == 2 ? sym : 0u)};
    CSTAMP(3);
    u32 cnt = 0, mk[32], sl[32];
    {
        Classifier K; K.init(T[0]);
        u32 c[32]; K.run<32>(Er, c);
#pragma unroll
        for (int g = 0; g < 32; ++g) { mk[g] = escmask(c[g]); sl[g] = selector(mk[g]); cnt += popc(mk[g]); }
#pragma unroll
        for (int k = 0; k < 8; ++k) Cr[k] = c[4 * k] | (c[4 * k + 1] << 2) | (c[4 * k + 2] << 4) | (c[4 * k + 3] << 6);
    }
    cnt >>= 3;
    const u32 inc = scan_incl(cnt), n2 = (u32)__builtin_amdgcn_readlane((int)inc, 63);
    CSTAMP(4);
    if (n2 > CAP2) return RAWCODE;
    Appender A; A.init(scr + E_L2 + inc - cnt);
#pragma unroll
    for (int g = 0; g < 32; ++g) A.push(perm(0u, Er[g], sl[g]), mk[g]);
    A.flush();
    CBAR(); CSTAMP(5);
    const u32 d2 = (n2 + 15) >> 4;
    const u32 n3 = pack_level<2>(scr + E_L2, n2, T[1], scr + E_VAR + 12, scr + E_L3, CAP3, lane);
    CSTAMP(6);
    if (n3 == ~0u) return RAWCODE;
    const u32 d3 = (n3 + 15) >> 4, b4 = 12 + 4 * (d2 + d3);
    const u32 n4 = pack_level<1>(scr + E_L3, n3, T[2], scr + E_VAR + 12 + 4 * d2, scr + E_VAR + b4, CAPV - b4, lane);
    if (n4 == ~0u) return RAWCODE;
    if (lane == 0) { *(LAS u32*)(scr + E_VAR) = T[0] | d2; *(LAS u32*)(scr + E_VAR + 4) = T[1] | d3; *(LAS u32*)(scr + E_VAR + 8) = T[2]; }
    CBAR();
    return (b4 + n4 + 15) >> 4;
}
__device__ __forceinline__ void pack_init(u32 wid, u32 lane) {
    if (wid >= 4) lds128w(LDS_HIST + (wid - 4) * 1024 + lane * 16, (u32x4){0u, 0u, 0u, 0u});
    if (wid == 1 && lane < 16) { u32 sel = 0x0c0c0c0cu, k = 0;
#pragma unroll
        for (u32 b = 0; b < 4; ++b) if ((lane >> b) & 1) { sel = (sel & ~(255u << k)) | (b << k); k += 8; }
        *(LAS u32*)(LDS_LUT + lane * 4) = sel; }
}
__device__ __forceinline__ u32 chunk_bytes(u32 code) { return code == RAWCODE ? RAWB : FIXB + 16 * code; }

__device__ __forceinline__ void pack_store(char* dst, u32 code, const u32 (&Mr)[32], const u32 (&Er)[32], const u32 (&Cr)[8], u32 scr, u32 lane) {
#pragma unroll
    for (int j = 0; j < 8; ++j) st_sys(dst + j * 1024, (u32x4){Mr[4 * j], Mr[4 * j + 1], Mr[4 * j + 2], Mr[4 * j + 3]});
    if (code == RAWCODE) {
#pragma unroll
        for (int j = 0; j < 8; ++j) st_sys(dst + 8192 + j * 1024, (u32x4){Er[4 * j], Er[4 * j + 1], Er[4 * j + 2], Er[4 * j + 3]});
    } else {
        st_sys(dst + 8192, (u32x4){Cr[0], Cr[1], Cr[2], Cr[3]}); st_sys(dst + 9216, (u32x4){Cr[4], Cr[5], Cr[6], Cr[7]});
        if (lane < code) st_sys(dst + FIXB, lds128(scr + E_VAR + lane * 16));
        if (lane + 64 < code) st_sys(dst + FIXB + 1024, lds128(scr + E_VAR + 1024 + lane * 16));
    }
}
__device__ __forceinline__ void raise_flag(const Params& P, u32 q, u32 gid, u32 w, u32 call, u32 code) {
    char* wq = P.ws[0];
#pragma unroll
    for (u32 j = 1; j < NR; ++j) wq = q == j ? P.ws[j] : wq;
    u32* f = (u32*)(wq + WS_FLAG + (call & 1) * FLAG_PAR + (size_t)P.rank * FLAG_SRC + gid * 32) + w;
    __hip_atomic_store(f, (call << 8) | code, __ATOMIC_RELAXED, __HIP_MEMORY_SCOPE_SYSTEM);
}
__device__ __forceinline__ void pack_group(const Params& P, const u32 (&v)[64], u32 (&Mr)[32], u32 (&Er)[32], u32 gid, u32 w, bool solo, u32 peers, u32 call, u32 wid, u32 lane) {
    u32 Cr[8];
    const u32 scr = wid * 16384;
    CS_DECL
    if (gid < 64u) STAMP(P, 40 + wid);
    u32 code = rfl(pack_chunk(v, Mr, Er, Cr, scr, lane, solo CS_PASS));
    if (gid == (u32)(NGID - 1) && w >= (u32)(8 - GAR_TAILRAW)) code = RAWCODE;
    if (gid < 64u) STAMP(P, 48 + wid);
    u32 off = w * RAWB, mine = code;
    if (!solo) {
        if (lane == 0) lds32vw(CTL_SIZE + wid * 4, code);
        WAIT_LGKM0(); BAR();
        mine = lds32v(CTL_SIZE + (lane & 7) * 4);
        if (lane < 32) lds128w(LDS_HIST + wid * 512 + lane * 16, (u32x4){0u, 0u, 0u, 0u});
        off = 0;
#pragma unroll
        for (u32 j = 0; j < 7; ++j) { const u32 c = (u32)__builtin_amdgcn_readlane((int)mine, j); if (j < wid) off += chunk_bytes(c); }
    }
    pack_store(P.ws[P.rank] + WS_STG + (call & 1) * STG_PAR + (size_t)gid * GRP_STRIDE + off + lane * 16, code, Mr, Er, Cr, scr, lane);
    if (gid < 64u) STAMP(P, 56 + wid);
    WAIT_VM(0);
    if (gid < 64u) STAMP(P, 64 + wid);
    WAIT_LGKM0();
    if (!solo) BAR();
    if ((solo ? (lane & 7) == w : wid == 0) && ((peers >> (lane >> 3)) & 1)) raise_flag(P, lane >> 3, gid, lane & 7, call, mine);
}

#define GAR_WINMODE 1
template <bool WRAP> struct Unp {
    u32 sel[4], lo[4], hi[4], of[4];
    __device__ __forceinline__ void fetch(u32 c, u32& off) {
#pragma unroll
        for (int j = 0; j < 4; ++j) {
            const u32 cj = (c >> (2 * j)) & 0x03030303u, mk = escmask(cj), m2 = mk & 0x02020202u;
            if (WRAP || GAR_WINMODE != 2) { const u32 a = off & ~3u; lo[j] = lds32(WRAP ? (a & RMASK) : a); hi[j] = lds32(WRAP ? ((a + 4) & RMASK) : a + 4); of[j] = off; }
            else lo[j] = lds32(off);
            u32 p; asm("v_mad_u32_u24 %0, %1, %2, %3" : "=v"(p) : "v"(m2), "s"(0x808080u), "v"(0x04040404u));
            sel[j] = (mk & p) | cj;
            off += popc(m2);
        }
    }
    __device__ __forceinline__ u32 finish(int j, u32 t) const {
        u32 w;
        if (WRAP || GAR_WINMODE == 0) w = __builtin_amdgcn_alignbyte(hi[j], lo[j], of[j] & 3u);
        else if (GAR_WINMODE == 1) w = __builtin_amdgcn_alignbyte(hi[j], lo[j], of[j]);
        else w = lo[j];
        return perm(w, t, sel[j]);
    }
    __device__ __forceinline__ u32x4 finish4(u32 t) const { return (u32x4){finish(0, t), finish(1, t), finish(2, t), finish(3, t)}; }
};
#define SCHED() __builtin_amdgcn_sched_barrier(0)
template <bool WRAP> __device__ __forceinline__ u32 unpack_round(u32 c, u32 t, u32 src, u32 dst) {
    const u32 cnt = popc(escbits(c)), inc = scan_incl(cnt);
    u32 off = src + inc - cnt;
    Unp<WRAP> u; u.fetch(c, off); SCHED();
    lds128w(dst, u.finish4(t));
    return (u32)__builtin_amdgcn_readlane((int)inc, 63);
}
__device__ __forceinline__ void unpack_chunk(u32 S, u32 code, u32 scr, u32 lane, u32 rel_addr, u32 rel_val, u32 (&Mr)[32], u32 (&Er)[32] CS_ARG) {
    const u32 a0 = S + lane * 16;
#pragma unroll
    for (int j = 0; j < 8; ++j) { const u32x4 q = lds128((a0 + j * 1024) & RMASK); Mr[4 * j] = q.x; Mr[4 * j + 1] = q.y; Mr[4 * j + 2] = q.z; Mr[4 * j + 3] = q.w; }
    if (code == RAWCODE) {
#pragma unroll
        for (int j = 0; j < 8; ++j) { const u32x4 q = lds128((a0 + 8192 + j * 1024) & RMASK); Er[4 * j] = q.x; Er[4 * j + 1] = q.y; Er[4 * j + 2] = q.z; Er[4 * j + 3] = q.w; }
        WAIT_LGKM0(); lds32vw(rel_addr, rel_val); CBAR();
        return;
    }
    const u32x4 ca = lds128((a0 + 8192) & RMASK), cb = lds128((a0 + 9216) & RMASK);
    const u32 V = S + FIXB;
    const u32x4 hd = lds128(V & RMASK);
    const u32 t1 = rfl(hd.x), t2 = rfl(hd.y), t3 = rfl(hd.z), d2 = t1 & 255u, d3 = t2 & 127u;
    u32 c2[4], c3[2];
#pragma unroll
    for (int r = 0; r < 4; ++r) { const u32 i = r * 64 + lane; const u32 x = lds32((V + 12 + 4 * i) & RMASK); c2[r] = i < d2 ? x : 0x55555555u; }
#pragma unroll
    for (int r = 0; r < 2; ++r) { const u32 i = r * 64 + lane; const u32 x = lds32((V + 12 + 4 * (d2 + i)) & RMASK); c3[r] = i < d3 ? x : 0x55555555u; }
    CSTAMP(1);
    u32 s4 = V + 12 + 4 * (d2 + d3);
    s4 += unpack_round<true>(c3[0], t3, s4, scr + D_L3 + lane * 16);
    if (d3 > 64) unpack_round<true>(c3[1], t3, s4, scr + D_L3 + 1024 + lane * 16);
    WAIT_LGKM0(); lds32vw(rel_addr, rel_val); CBAR();
    CSTAMP(2);
    const u32 C[8] = {ca.x, ca.y, ca.z, ca.w, cb.x, cb.y, cb.z, cb.w};
    u32 cnt = 0;
#pragma unroll
    for (int k = 0; k < 8; ++k) cnt += popc(escbits(C[k]));
    {
        const u32 n0 = popc(escbits(c2[0])), n1 = popc(escbits(c2[1]));
        const u32 i01 = scan_incl(n0 | (n1 << 16)), tot = (u32)__builtin_amdgcn_readlane((int)i01, 63);
        cnt = scan_incl(cnt) - cnt;
        u32 o0 = scr + D_L3 + (i01 & 0xffffu) - n0, o1 = scr + D_L3 + (tot & 0xffffu) + (i01 >> 16) - n1;
        Unp<false> u0, u1; u0.fetch(c2[0], o0); u1.fetch(c2[1], o1); SCHED();
        lds128w(scr + D_L2 + lane * 16, u0.finish4(t2)); lds128w(scr + D_L2 + 1024 + lane * 16, u1.finish4(t2));
        if (d2 > 128) {
            u32 s3 = scr + D_L3 + (tot & 0xffffu) + (tot >> 16);
            s3 += unpack_round<false>(c2[2], t2, s3, scr + D_L2 + 2048 + lane * 16);
            if (d2 > 192) unpack_round<false>(c2[3], t2, s3, scr + D_L2 + 3072 + lane * 16);
        }
    }
    CBAR(); CSTAMP(3);
    u32 off = scr + D_L2 + cnt;
    Unp<false> u[8];
    u[0].fetch(C[0], off); u[1].fetch(C[1], off);
#pragma unroll
    for (int k = 0; k < 8; ++k) {
        SCHED();
        if (k + 2 < 8) u[k + 2].fetch(C[k + 2], off);
        SCHED();
#pragma unroll
        for (int j = 0; j < 4; ++j) Er[4 * k + j] = u[k].finish(j, t1);
    }
    CBAR();
}
__device__ __forceinline__ u32x4 piece(const u32 (&Mr)[32], const u32 (&Er)[32], int p) {
    const u32 a = perm(Er[2 * p], Mr[2 * p], 0x05010400u), b = perm(Er[2 * p], Mr[2 * p], 0x07030602u);
    const u32 c = perm(Er[2 * p + 1], Mr[2 * p + 1], 0x05010400u), d = perm(Er[2 * p + 1], Mr[2 * p + 1], 0x07030602u);
    return (u32x4){__builtin_amdgcn_alignbit(a, a, 1), __builtin_amdgcn_alignbit(b, b, 1), __builtin_amdgcn_alignbit(c, c, 1), __builtin_amdgcn_alignbit(d, d, 1)};
}

__device__ __forceinline__ u32 x_piece_off(u32 half, int p) { return half * 16384 + p * 1024; }
__device__ __forceinline__ u32 w_piece_off(u32 h, int p) {
    const u32 g32 = p >> 2, n = (p >> 1) & 1, kb = p & 1;
    return (g32 & 1) * 16384 + ((2 * (2 * h + (g32 >> 1)) + n) * 2 + kb) * 1024;
}
__device__ __forceinline__ u16* y_base(const Params& P, u32 pm, u32 j, u32 w, u32 lane) {
    return P.out + (size_t)(pm * 256 + (w >> 2) * 64 + (lane & 7)) * NN + j * 256 + (w & 3) * 64 + (lane & 8) * 4 + (lane >> 4) * 8;
}
__device__ __forceinline__ size_t y_piece_off(int p) { return (size_t)((p >> 3) * 128 + (p & 7) * 8) * NN; }
__device__ __forceinline__ u32 local_panel(u32 j, u32 rank) { return j >= (u32)(NR * DP) ? j - (NR - 1) * DP : j - rank * DP; }
__device__ __forceinline__ u32 global_panel(u32 jj, u32 rank) { return jj >= (u32)DP ? jj + (NR - 1) * DP : jj + rank * DP; }
__device__ __forceinline__ u32* counter(const Params& P, u32 i) { return (u32*)(P.ws[P.rank] + WS_CNT + i * 128); }
__device__ __forceinline__ u32* tmo_word(const Params& P) { return (u32*)(P.ws[P.rank] + WS_TMO); }
__device__ __forceinline__ bool expired(const Params& P, u64 t0, u32 call) {
    if (__hip_atomic_load(tmo_word(P), __ATOMIC_RELAXED, __HIP_MEMORY_SCOPE_AGENT) == call || now() - t0 > DEADLINE) {
        __hip_atomic_store(tmo_word(P), call, __ATOMIC_RELAXED, __HIP_MEMORY_SCOPE_AGENT); return true; }
    return false;
}

__device__ __forceinline__ void store_image(const Params& P, u32 gid, u32 w, u32 q, u32 lane, const u32 (&Mr)[32], const u32 (&Er)[32]) {
    const bool isx = gid < (u32)GW;
    const u32 pn = isx ? gid & 15 : local_panel((gid - GW) >> 1, P.rank), ktl = (isx ? gid >> 4 : gid & 1) * 4 + (w >> 1);
    const u32 qi = (q - P.rank - 1) & 7, stage = qi == 7 ? 0u : 1 + (ktl >> 1);
    const u32 slot = stage == 0 ? ktl : stage * 14 - 6 + qi * 2 + (ktl & 1);
    char* o = P.ws[P.rank] + (isx ? WS_XI : WS_WI) + (size_t)pn * PANEL + (size_t)slot * 32768 + swz((lane & 15) * 64 + (lane >> 4) * 16);
#pragma unroll
    for (int p = 0; p < 16; ++p) st_dev(o + (isx ? x_piece_off(w & 1, p) : w_piece_off(w & 1, p)), piece(Mr, Er, p));
    WAIT_VM(0);
    if (lane == 0) __hip_atomic_fetch_add(counter(P, isx ? pn : 16 + pn) + stage, 1u, __ATOMIC_RELAXED, __HIP_MEMORY_SCOPE_AGENT);
}
__device__ __forceinline__ u32* call_word(const Params& P, u32 wid) { return (u32*)(P.ws[P.rank] + WS_CALL + blockIdx.x * 128 + wid * 4); }
__device__ __forceinline__ void call_load(const Params& P, u32 wid, u32 lane) {
    u32 keep;
    if (lane == 0) asm volatile("s_mov_b32 %0, m0\n\ts_mov_b32 m0, %2\n\ts_nop 0\n\tglobal_load_lds_dword %1, off sc1\n\ts_mov_b32 m0, %0" : "=&s"(keep) : "v"(call_word(P, wid)), "s"(CTL_CALL + wid * 4) : "memory");
}
__device__ __forceinline__ u32 call_take(u32 wid) { WAIT_VM(0); return rfl(lds32v(CTL_CALL + wid * 4)) + 1; }
__device__ __forceinline__ void call_bump(const Params& P, u32 wid, u32 lane, u32 call) { if (lane == 0) __hip_atomic_store(call_word(P, wid), call, __ATOMIC_RELAXED, __HIP_MEMORY_SCOPE_AGENT); }

__device__ __forceinline__ u32 solo_gid(u32 c) { return (c >> 2) & 15; }
__device__ __forceinline__ u32 solo_w(u32 c) { return (c & 3) | (c >> 6) << 2; }
__device__ __forceinline__ void load_x(const Params& P, u32 gid, u32 w, u32 lane, u32 (&v)[64]) {
    const u16* s = P.x + (size_t)((gid & 15) * 256 + (w & 1) * 128 + (lane & 15)) * KS + ((gid >> 4) * 4 + (w >> 1)) * 64 + (lane >> 4) * 8;
#pragma unroll
    for (int p = 0; p < 16; ++p) { const u32x4 t = *(const u32x4*)(s + (p >> 1) * 16 * KS + (p & 1) * 32); v[4 * p] = t.x; v[4 * p + 1] = t.y; v[4 * p + 2] = t.z; v[4 * p + 3] = t.w; }
}
__device__ __forceinline__ void puller(const Params& P, u32 wid, u32 lane) {
    const u32 q = (P.rank + 1 + blockIdx.x % 7) & 7;
    u32 call;
    if (wid == 7 && blockIdx.x < (u32)NRAW) {
        const u32 gid = solo_gid(blockIdx.x), w = solo_w(blockIdx.x);
        u32 v[64], Mr[32], Er[32]; const u32 Cr[8] = {};
        load_x(P, gid, w, lane, v); call = call_take(wid); split(v, Mr, Er);
        pack_store(P.ws[P.rank] + WS_STG + (call & 1) * STG_PAR + (size_t)gid * GRP_STRIDE + w * RAWB + lane * 16, RAWCODE, Mr, Er, Cr, 0u, lane);
        WAIT_VM(0);
        if (lane < (u32)NR && lane != (u32)P.rank) raise_flag(P, lane, gid, w, call, RAWCODE);
        store_image(P, gid, w, P.rank, lane, Mr, Er);
    }
    call = call_take(wid);
    call_bump(P, wid, lane, call);
    if (wid == 0) {
        __builtin_amdgcn_s_setprio(3);
        const u64 b64 = (u64)(P.ws[q] + WS_STG + (call & 1) * STG_PAR);
        const char* base = (const char*)(((u64)rfl((u32)(b64 >> 32)) << 32) | rfl((u32)b64));
        const char* dummy = P.ws[P.rank] + WS_TMO + 1024 + lane * 16;
        const u32 voff = lane * 16;
        u32 pos = 0, pub = 0, limit = RING / 1024;
        u64 tk = 0, c_full = 0, c_idle = 0, n_full = 0, n_ref = 0; (void)tk; (void)c_full; (void)c_idle; (void)n_full; (void)n_ref;
        for (u32 ui = 0;; ++ui) {
            u32 nu = rfl(lds32v(CTL_NUNIT));
            if ((nu & ~DONE) <= ui) {
                TICK(tk);
                WAIT_VM(0); lds32vw(CTL_LANDED, pos); pub = pos;
                while (nu <= ui) { __builtin_amdgcn_s_sleep(1); nu = rfl(lds32v(CTL_NUNIT)); }
                if ((nu & ~DONE) <= ui) break;
                ACC(c_idle, tk);
            }
            CBAR();
            const u32x2 un = lds64v(D_UNIT + ui * 8);
            const u32 ut = rfl(un.y), total = ut & 0x3ffffu, lastc = ut >> 18;
            const char* src = base + rfl(un.x);
            for (u32 o = 0; o < total; o += 4096) {
                if (pos + 4 > limit) {
                    bool drained = false; TICK(tk); ++n_ref;
                    for (;;) {
                        const u32 nx = lds32v(CTL_NEXT + (lane & 7) * 4);
                        u32 f = lastc;
#pragma unroll
                        for (u32 j = 0; j < ND; ++j) f = umin(f, (u32)__builtin_amdgcn_readlane((int)nx, j));
                        limit = (rfl(lds32v(D_DESC + f * 8)) >> 14) + RING / 1024;
                        if (pos + 4 <= limit) break;
                        if (!drained) { WAIT_VM(0); lds32vw(CTL_LANDED, pos); pub = pos; drained = true; }
                        __builtin_amdgcn_s_sleep(1);
                    }
                    ACC(c_full, tk); n_full += drained;
                }
                const u32 dst = (pos * 1024) & RMASK;
                if (o + 4096 <= total) {
                    u32 keep;
                    asm volatile("s_mov_b32 %0, m0\n\ts_mov_b32 m0, %3\n\ts_nop 4\n\t"
                                 "global_load_lds_dwordx4 %1, %2 sc0 sc1\n\ts_waitcnt vmcnt(%4)\n\t"
                                 "global_load_lds_dwordx4 %1, %2 offset:1024 sc0 sc1\n\ts_waitcnt vmcnt(%4)\n\t"
                                 "global_load_lds_dwordx4 %1, %2 offset:2048 sc0 sc1\n\ts_waitcnt vmcnt(%4)\n\t"
                                 "global_load_lds_dwordx4 %1, %2 offset:3072 sc0 sc1\n\ts_waitcnt vmcnt(%4)\n\t"
                                 "s_mov_b32 m0, %0" : "=&s"(keep) : "v"(voff), "s"(src + o), "s"(dst), "i"(WIN) : "memory");
                } else {
#pragma unroll
                    for (u32 j = 0; j < 4; ++j) { const u32 b = o + j * 1024 + voff; glds_sys(b < total ? src + b : dummy, dst + j * 1024); WAIT_VM(WIN); }
                }
                pos += 4;
                if (pos > pub + WIN) { pub = pos - WIN; lds32vw(CTL_LANDED, pub); }
            }
            STAMP(P, 128 + umin(ui, 127)); DUMP(P, 64 + umin(ui, 127), total);
        }
        DUMP(P, 0, c_full); DUMP(P, 1, n_full); DUMP(P, 2, n_ref); DUMP(P, 3, c_idle);
    } else if (wid == 1) {
        u32 basek = 0, nch = 0, nun = 0;
        u32* ticket = (u32*)(P.ws[P.rank] + WS_CLAIM + q * 128);
        for (u32 ui = 0;; ++ui) {
            while (rfl(lds32v(CTL_LANDED)) + LEAD < basek) __builtin_amdgcn_s_sleep(4);
            u32 idx = ui * PW + blockIdx.x / 7;
            if (ui >= (u32)NSTAT) {
                if (lane == 0) idx = __hip_atomic_fetch_add(ticket, 1u, __ATOMIC_RELAXED, __HIP_MEMORY_SCOPE_AGENT);
                idx = rfl(idx) - (call - 1) * (u32)(P.nunits - NSTAT * PW + PW) + NSTAT * PW;
                if (idx >= (u32)P.nunits) break;
            }
            const u32 e = tab16(P.stream, idx), gid = e & 1023u, part = e >> 10, lv = 31 - __builtin_clz(part), n = 8u >> lv, first = (part - (1u << lv)) * n;
            const bool solo = gid < (u32)SOLOG, mine = lane >= first && lane < first + n;
            const u32* f = (const u32*)(P.ws[P.rank] + WS_FLAG + (call & 1) * FLAG_PAR + (size_t)q * FLAG_SRC + gid * 32) + (lane & 7);
            const u64 t0 = now(); u32 val; bool dead = false;
            for (;;) {
                val = __hip_atomic_load(f, __ATOMIC_RELAXED, __HIP_MEMORY_SCOPE_SYSTEM);
                if (__all((val >> 8) == (call & 0xffffffu) || lane >= first + n || (solo && lane < first))) break;
                if (__any(expired(P, t0, call))) { dead = true; break; }
                __builtin_amdgcn_s_sleep(1);
            }
            if (dead) {
                const u32 t = gid - GY;
                if (lane == 0 && t < (u32)(16 * DP)) *y_base(P, t & 15, q * DP + (t >> 4), first, 0u) = 0x7fc0;
                break;
            }
            u32 code = val & 255u; if (code > 127u) code = RAWCODE;
            const u32 cb = chunk_bytes(code), sz = solo ? (mine ? (cb + 4095u) & ~4095u : 0u) : (lane < first + n ? cb : 0u);
            u32 inc = sz; inc += dpp<0x111, 0xf, true>(inc); inc += dpp<0x112, 0xf, true>(inc); inc += dpp<0x114, 0xf, true>(inc);
            const u32 a0 = (u32)__builtin_amdgcn_readlane((int)(inc - sz), (int)first) & ~127u, total = (u32)__builtin_amdgcn_readlane((int)inc, 7) - a0;
            if (mine) lds64vw(D_DESC + (nch + lane - first) * 8, (u32x2){((basek * 1024 + inc - sz - a0) << 4) | code, gid * 8 + lane});
            if (solo) { if (mine) lds64vw(D_UNIT + (nun + lane - first) * 8, (u32x2){gid * (u32)GRP_STRIDE + lane * RAWB, cb | ((nch + lane - first) << 18)}); nun += n; }
            else { if (lane == 0) lds64vw(D_UNIT + nun * 8, (u32x2){gid * (u32)GRP_STRIDE + a0, total | ((nch + n - 1) << 18)}); nun += 1; }
            nch += n;
            WAIT_LGKM0();
            lds32vw(CTL_NUNIT, nun); lds32vw(CTL_NCHUNK, nch);
            basek += ((total + 4095) >> 12) << 2;
            STAMP(P, umin(ui, 127));
        }
        lds32vw(CTL_NUNIT, nun | DONE); lds32vw(CTL_NCHUNK, nch | DONE);
    } else {
        const u32 dw = wid - 2, scr = D_SCR + dw * D_SCRB;
        u64 tk = 0, c_wait = 0, c_dec = 0, c_st = 0; (void)c_wait; (void)c_dec; (void)c_st; TICK(tk);
        for (u32 c = dw;; c += ND) {
            u32 nc = rfl(lds32v(CTL_NCHUNK));
            while (nc <= c) { __builtin_amdgcn_s_sleep(2); nc = rfl(lds32v(CTL_NCHUNK)); }
            if ((nc & ~DONE) <= c) break;
            CBAR();
            const u32x2 de = lds64v(D_DESC + c * 8);
            const u32 d = rfl(de.x), code = d & 255u, S = (d >> 8) << 4, gid = rfl(de.y) >> 3, w = rfl(de.y) & 7;
            const u32 need = (S + chunk_bytes(code) + 1023) >> 10;
            while (rfl(lds32v(CTL_LANDED)) < need) __builtin_amdgcn_s_sleep(1);
            CBAR();
            u32 Mr[32], Er[32];
            CS_DECL
            ACC(c_wait, tk);
            unpack_chunk(S, code, scr, lane, CTL_NEXT + dw * 4, c + ND, Mr, Er CS_PASS);
            ACC(c_dec, tk);
            if (gid >= (u32)GY) {
                const u32 t = gid - GY;
                u16* o = y_base(P, t & 15, q * DP + (t >> 4), w, lane);
#pragma unroll
                for (int p = 0; p < 16; ++p) st_out<GAR_OST>(o + y_piece_off(p), piece(Mr, Er, p));
            } else store_image(P, gid, w, q, lane, Mr, Er);
            ACC(c_st, tk);
        }
        DUMP(P, wid * 4, c_wait); DUMP(P, wid * 4 + 1, c_dec); DUMP(P, wid * 4 + 2, c_st); DUMP(P, wid * 4 + 3, now());
    }
}

__device__ __forceinline__ void pack_inputs(const Params& P, u32 gid, u32 w, bool solo, u32 wid, u32 lane) {
    asm volatile("" : "+v"(lane));
    u32 v[64], Mr[32], Er[32];
    u32 peers = 255u & ~(1u << P.rank); bool own = true;
    if (gid < (u32)GW) load_x(P, gid, w, lane, v);
    else {
        const u32 j = (gid - GW) >> 1, ktl = (gid & 1) * 4 + (w >> 1);
        if (j < (u32)(NR * DP)) { const u32 dest = j / DP; own = dest == (u32)P.rank; peers = own ? 0u : 1u << dest; }
        const u16* s = P.w + (size_t)(ktl * 64 + (lane >> 4)) * NN + j * 256 + (w & 1) * 128 + (lane & 15) * 8;
        const u32 scr = wid * 16384;
#pragma unroll
        for (int i = 0; i < 16; ++i) glds_plain(s + (size_t)i * 4 * NN, scr + i * 1024);
        WAIT_VM(0);
        const u32 ta = scr + ((lane >> 4) * 8 + ((lane & 15) >> 2)) * 256 + (lane & 3) * 16;
#pragma unroll
        for (int p = 0; p < 16; ++p) {
            const u32 a = ta + (p & 1) * 32 * 256 + (p >> 2) * 64 + ((p >> 1) & 1) * 8;
            const u32x2 lo = __builtin_bit_cast(u32x2, __builtin_amdgcn_ds_read_tr16_b64_v4i16((LAS s16x4*)a));
            const u32x2 hi = __builtin_bit_cast(u32x2, __builtin_amdgcn_ds_read_tr16_b64_v4i16((LAS s16x4*)(a + 4 * 256)));
            v[4 * p] = lo.x; v[4 * p + 1] = lo.y; v[4 * p + 2] = hi.x; v[4 * p + 3] = hi.y;
        }
        WAIT_LGKM0();
    }
    const u32 call = call_take(wid);
    if (peers) pack_group(P, v, Mr, Er, gid, w, solo, peers, call, wid, lane); else split(v, Mr, Er);
    if (own) store_image(P, gid, w, P.rank, lane, Mr, Er);
}

typedef __bf16 bf16x2 __attribute__((ext_vector_type(2)));
typedef float f32x2 __attribute__((ext_vector_type(2)));
__device__ __forceinline__ u32 cvt_pk(float lo, float hi) { return __builtin_bit_cast(u32, __builtin_convertvector((f32x2){lo, hi}, bf16x2)); }
__device__ __forceinline__ void wait_panels(const Params& P, u32 tile, u32 upto, u32 call, u32 wid, u32 lane) {
    if (wid == 0) {
        const u32 st = umin(lane & 7, upto), need = call * (st == 0 ? 16u : 28u);
        const u32* c = counter(P, (lane & 8) ? 16 + (tile >> 4) : (tile & 15)) + st;
        const u64 t0 = now();
        for (;;) {
            const u32 x = __hip_atomic_load(c, __ATOMIC_RELAXED, __HIP_MEMORY_SCOPE_AGENT);
            if (__all((int)(x - need) >= 0)) break;
            if (__any(expired(P, t0, call))) { asm volatile("ds_write_b32 %0, %1\n\ts_waitcnt lgkmcnt(0)" :: "v"(CTL_BAD), "v"(1u) : "memory"); break; }
            __builtin_amdgcn_s_sleep(4);
        }
        __builtin_amdgcn_fence(__ATOMIC_ACQUIRE, "agent");
        WAIT_VM(0);
    }
    BAR();
}
__device__ __forceinline__ void gemm_role(const Params& P, unsigned char* lds_generic, u32 call, u32 wid, u32 lane) {
    LAS unsigned char* lds = (LAS unsigned char*)lds_generic;
    const u32 gw = blockIdx.x - GEMM0, tid = threadIdx.x;
    const u32 wr = wid >> 2, wc = wid & 3, fr = lane & 15, fq = lane >> 4;
    constexpr int HTB = 16384;
    const u32 voff = tid * 16, ldsw = wid * 1024;
    const u32 fo = swz(fr * 64 + fq * 16);
    const u32 aoff = wr * 8192 + fo, boff = wc * 4096 + fo;
#define G_SA(b, h) (((b) * 2 + (h)) * HTB)
#define G_SB(b, h) ((4 + (b) * 2 + (h)) * HTB)
#define G_STAGE(bufoff, gbase) do { _Pragma("unroll") for (int _i = 0; _i < 2; ++_i) \
        __builtin_amdgcn_global_load_lds((const u32*)((const char*)(gbase) + voff + _i * 8192), (LAS u32*)(lds + (bufoff) + ldsw + _i * 8192), 16, 0, 0); } while (0)
#define G_LDA(dst, b, h) do { _Pragma("unroll") for (int m = 0; m < 4; ++m) _Pragma("unroll") for (int k = 0; k < 2; ++k) dst[m][k] = *(const LAS bf16x8*)(lds + G_SA(b, h) + aoff + m * 2048 + k * 1024); } while (0)
#define G_LDB(dst, b, h) do { _Pragma("unroll") for (int n = 0; n < 2; ++n) _Pragma("unroll") for (int k = 0; k < 2; ++k) dst[n][k] = *(const LAS bf16x8*)(lds + G_SB(b, h) + boff + n * 2048 + k * 1024); } while (0)
#define G_MMA(ai, bj, At, Bt) do { __builtin_amdgcn_s_setprio(1); _Pragma("unroll") for (int m = 0; m < 4; ++m) _Pragma("unroll") for (int n = 0; n < 2; ++n) _Pragma("unroll") for (int k = 0; k < 2; ++k) \
        acc[ai][bj][m][n] = __builtin_amdgcn_mfma_f32_16x16x32_bf16(Bt[n][k], At[m][k], acc[ai][bj][m][n], 0, 0, 0); __builtin_amdgcn_s_setprio(0); } while (0)
#define G_WV(n) asm volatile("s_waitcnt vmcnt(" #n ")" ::: "memory")
#define G_WL(n) asm volatile("s_waitcnt lgkmcnt(" #n ")" ::: "memory")
#define G_BAR __builtin_amdgcn_s_barrier()
#define G_SCHED __builtin_amdgcn_sched_barrier(0)
    const char* XI = P.ws[P.rank] + WS_XI; const char* WI = P.ws[P.rank] + WS_WI;
    constexpr size_t kstep = 32768, hstep = 16384;
    u32 cur = tab16(P.tiles, gw);
    if (cur == 0xffffu) return;
    bool fresh = true;
    bf16x8 At[4][2], B0[2][2], B1[2][2];
    for (u32 ui = 0;; ++ui) {
        const u32 nxt = ui + 1 < (u32)ROUNDS ? tab16(P.tiles, (ui + 1) * NGEMM + gw) : 0xffffu;
        const bool has_next = nxt != 0xffffu, dist = (cur >> 4) < (u32)DP, chain = has_next && !dist;
        const char* cA = XI + (size_t)(cur & 15) * PANEL; const char* cB = WI + (size_t)(cur >> 4) * PANEL;
        if (fresh) {
            wait_panels(P, cur, ui ? 4u : 0u, call, wid, lane);
            STAMP(P, 8 + ui * 4); if (ui == 0) DUMP(P, 1, __builtin_amdgcn_s_memtime());
            G_STAGE(G_SB(0, 0), cB); G_STAGE(G_SB(0, 1), cB + hstep); G_STAGE(G_SA(0, 0), cA); G_STAGE(G_SA(0, 1), cA + hstep);
            if (wr == 1) G_BAR;
            G_WV(2); G_BAR;
            G_STAGE(G_SB(1, 0), cB + kstep); G_STAGE(G_SA(1, 0), cA + kstep); G_STAGE(G_SB(1, 1), cB + hstep + kstep);
            G_WV(6); G_BAR;
        }
        f32x4 acc[2][2][4][2];
#pragma unroll
        for (int a = 0; a < 2; ++a)
#pragma unroll
            for (int b = 0; b < 2; ++b)
#pragma unroll
                for (int m = 0; m < 4; ++m)
#pragma unroll
                    for (int n = 0; n < 2; ++n) acc[a][b][m][n] = (f32x4){0.f, 0.f, 0.f, 0.f};
        const char* nA = chain ? XI + (size_t)(nxt & 15) * PANEL : cA; const char* nB = chain ? WI + (size_t)(nxt >> 4) * PANEL : cB;
        for (int t = 0; t < 64; t += 2) {
            const bool last = (t == 62);
            const char* a1 = cA + (size_t)(t + 1) * kstep;
            const char* a2 = last ? nA : cA + (size_t)(t + 2) * kstep; const char* b2 = last ? nB : cB + (size_t)(t + 2) * kstep;
            const char* a3 = a2 + kstep; const char* b3 = b2 + kstep;
            {
                const bool staged = ui == 0 && (t == 6 || t == 20 || t == 34 || t == 48);
                if (staged || (last && chain)) wait_panels(P, staged ? cur : nxt, t == 6 ? 1u : t == 20 ? 2u : t == 34 ? 3u : 4u, call, wid, lane);
                if (staged) STAMP(P, t == 6 ? 4 : t == 20 ? 5 : t == 34 ? 6 : 7);
            }
            G_LDB(B0, 0, 0); G_LDB(B1, 0, 1); G_SCHED; G_LDA(At, 0, 0); G_STAGE(G_SA(1, 1), a1 + hstep);
            G_WV(8); G_WL(0); G_BAR; G_MMA(0, 0, At, B0); G_MMA(0, 1, At, B1); G_BAR; G_SCHED;
            G_LDA(At, 0, 1); G_STAGE(G_SB(0, 0), b2); G_STAGE(G_SB(0, 1), b2 + hstep); G_STAGE(G_SA(0, 0), a2);
            G_WV(8); G_WL(0); G_BAR; G_MMA(1, 0, At, B0); G_MMA(1, 1, At, B1); G_BAR; G_SCHED;
            G_LDB(B0, 1, 0); G_LDB(B1, 1, 1); G_SCHED; G_LDA(At, 1, 0); G_STAGE(G_SA(0, 1), a2 + hstep);
            G_WV(8); G_WL(0); G_BAR; G_MMA(0, 0, At, B0); G_MMA(0, 1, At, B1); G_BAR; G_SCHED;
            G_LDA(At, 1, 1); G_STAGE(G_SB(1, 0), b3); G_STAGE(G_SB(1, 1), b3 + hstep); G_STAGE(G_SA(1, 0), a3);
            G_WV(8); G_WL(0); G_BAR; G_MMA(1, 0, At, B0); G_MMA(1, 1, At, B1); G_BAR; G_SCHED;
        }
        STAMP(P, 9 + ui * 4); DUMP(P, 2 + ui, __builtin_amdgcn_s_memtime());
        if (!chain) G_WV(0);
        if (GAR_ALIGN && wr == 0) G_BAR;
        u32 v[64];
        {
            int tz = threadIdx.x; asm volatile("" : "+v"(tz));
            const u32 ln = tz & 63;
            u32 bad; asm volatile("ds_read_b32 %0, %1\n\ts_waitcnt lgkmcnt(0)" : "=v"(bad) : "v"(CTL_BAD) : "memory");
            u16* o = y_base(P, cur & 15, global_panel(cur >> 4, P.rank), wid, ln);
#pragma unroll
            for (int p = 0; p < 16; p += 2) {
                const int ai = p >> 3, m = (p >> 1) & 3;
#pragma unroll
                for (int i = 0; i < 4; ++i) {
                    const f32x4 xa = acc[ai][0][m][i >> 1], xb = acc[ai][1][m][i >> 1];
                    u32 a = cvt_pk(xa[(i & 1) * 2], xa[(i & 1) * 2 + 1]), b = cvt_pk(xb[(i & 1) * 2], xb[(i & 1) * 2 + 1]);
                    if (bad) a = b = 0x7fc07fc0u;
                    v[4 * p + i] = (u32)__builtin_amdgcn_update_dpp((int)a, (int)b, 0x128, 0xf, 0xc, false);
                    v[4 * p + 4 + i] = (u32)__builtin_amdgcn_update_dpp((int)b, (int)a, 0x128, 0xf, 0x3, false);
                }
                st_out<GAR_GST>(o + y_piece_off(p), (u32x4){v[4 * p], v[4 * p + 1], v[4 * p + 2], v[4 * p + 3]});
                st_out<GAR_GST>(o + y_piece_off(p + 1), (u32x4){v[4 * p + 4], v[4 * p + 5], v[4 * p + 6], v[4 * p + 7]});
            }
            if (chain) { if (GAR_ALIGN && wr == 1) G_BAR; }
            else {
                if (!GAR_ALIGN && wr == 0) G_BAR;
                BAR();
                if (dist) { u32 Mr[32], Er[32]; pack_group(P, v, Mr, Er, GY + cur, wid, false, 255u & ~(1u << P.rank), call, wid, ln); }
                STAMP(P, 10 + ui * 4);
                if (!has_next) break;
            }
        }
        fresh = !chain; cur = nxt;
    }
}

__global__ void __launch_bounds__(512, 2) fused(const Params P) {
    extern __shared__ __attribute__((aligned(16))) unsigned char lds[];
    const u32 wid = rfl(threadIdx.x >> 6), lane = threadIdx.x & 63;
    call_load(P, wid, lane);
    if (blockIdx.x < (u32)NPULL) {
        if (threadIdx.x == 0) { lds32vw(CTL_NUNIT, 0u); lds32vw(CTL_LANDED, 0u); lds32vw(CTL_NCHUNK, 0u); }
        if (wid == 1 && lane < 8) lds32vw(CTL_NEXT + lane * 4, lane);
        WAIT_LGKM0(); BAR();
        puller(P, wid, lane); return;
    }
    STAMP(P, 0); DUMP(P, 0, __builtin_amdgcn_s_memtime());
    if (threadIdx.x == 0) lds32vw(CTL_BAD, 0u);
    pack_init(wid, lane);
    const u32 sc = NRAW + blockIdx.x - NPULL;
    if (sc >= (u32)(8 * SOLOG)) { const u64 t0 = now(); while (now() - t0 < GAR_HOLD) __builtin_amdgcn_s_sleep(2); }
    for (u32 it = 0;; ++it) {
        const u32 rb = NWG - 1 - blockIdx.x, e = it > 1 ? NENC + rb + (it - 2) * NLIGHT : rb;
        if (it > 1 && (e >= (u32)NPACK || rb >= (u32)NLIGHT)) break;
        if (it || (wid == 1 && sc < (u32)(8 * SOLOG))) pack_inputs(P, it ? tab16(P.enc, e) : solo_gid(sc), it ? wid : solo_w(sc), !it, wid, lane);
        WAIT_LGKM0(); BAR(); STAMP(P, it ? umin(it, 2) : 3);
    }
    const u32 call = call_take(wid);
    call_bump(P, wid, lane, call);
    if (blockIdx.x >= (u32)GEMM0) gemm_role(P, lds, call, wid, lane);
}

#define GAR_T3 2
#define GAR_T2 6
#define GAR_T1 14
#define GAR_SPARE 1
#define GAR_QTAIL 6
constexpr int QTAIL = GAR_QTAIL;
static void build_tables(Params& P) {
    u16* tiles = (u16*)P.tiles; u16* stream = (u16*)P.stream; u16* enc = (u16*)P.enc;
    int col[LP], nc = 0;
    {
        const int rest = LP - (ROUNDS - 1) * 12, r0 = (rest + 1) / 2; const bool spare = GAR_SPARE && DP <= 2 * (12 - rest);
        int nd = 0, nr = DP;
        for (int r = 0; r < ROUNDS; ++r) for (int c = 0; c < 6; ++c) for (int cg = 0; cg < 2; ++cg) {
            const bool heavy = c < (cg ? rest - r0 : r0); int jj = -1;
            if (r + 1 < ROUNDS || heavy) col[nc++] = jj = nd < DP && !(spare && heavy) ? nd++ : nr++;
            for (int x = 0; x < 16; ++x) tiles[r * NGEMM + (c * 4 + (x & 3)) * 8 + cg * 4 + (x >> 2)] = jj < 0 ? (u16)0xffff : (u16)(x | jj << 4);
        }
    }
    auto wgid = [&](int jj, int dest, int h) { return GW + 2 * (jj >= DP ? jj + (NR - 1) * DP : dest * DP + jj) + h; };
    int k = 0;
    auto put = [&](int gid, int lv) { for (int j = 0; j < (1 << lv); ++j) stream[k++] = (u16)(gid | ((1 << lv) + j) << 10); };
    for (int st = 0; st < 4; ++st) {
        const int h = st >> 1, half = st & 1;
        for (int i = 0; i < 28; ++i) {
            const int gid = i < 16 ? GX + 16 * h + i : wgid(col[i - 16], P.rank, h);
            if ((i < 16 && h == 0) || i >= 28 - QTAIL) { stream[k++] = (u16)(gid | (4 + 2 * half) << 10); stream[k++] = (u16)(gid | (5 + 2 * half) << 10); }
            else stream[k++] = (u16)(gid | (2 + half) << 10);
        }
    }
    for (int i = 12; i < LP; ++i) for (int h = 0; h < 2; ++h) put(wgid(col[i], P.rank, h), i < 24 ? 1 : 0);
    for (int t = 0; t < 16 * DP; ++t) { const int left = 16 * DP - t; put(GY + t, left <= GAR_T3 ? 3 : left <= GAR_T2 ? 2 : left <= GAR_T1 ? 1 : 0); }
    P.nunits = k;
    k = 0;
    for (int i = 0; i < 12; ++i) for (int d = 1; d < (col[i] < DP ? NR : 2); ++d) enc[k++] = (u16)wgid(col[i], (P.rank + d) & 7, 0);
    for (int i = SOLOG; i < 32; ++i) enc[k++] = (u16)(GX + i);
    for (int i = 0; i < 12; ++i) { if (col[i] < DP) enc[k++] = (u16)wgid(col[i], P.rank, 0); enc[k++] = (u16)wgid(col[i], P.rank, 1); }
    for (int i = 0; i < 12; ++i) if (col[i] < DP) for (int d = 1; d < NR; ++d) enc[k++] = (u16)wgid(col[i], (P.rank + d) & 7, 1);
    for (int i = 12; i < LP; ++i) for (int h = 0; h < 2; ++h) for (int d = 0; d < (col[i] < DP ? NR : 1); ++d) enc[k++] = (u16)wgid(col[i], (P.rank + d) & 7, h);
}
}

extern "C" void kernel_launch(int rank, int nranks, ncclComm_t comm, hipStream_t stream,
                              void* const* d_in, const int* in_sizes, int n_in,
                              void* d_out, int out_size, const int* in_dtypes, int out_dtype,
                              void* const* d_out_all, void* const* d_ws_all, size_t ws_bytes) {
    using namespace gar;
    if (nranks != NR || (unsigned)rank >= (unsigned)NR || n_in != 2 || ws_bytes < WS_END || in_sizes[0] != MM * KS || in_sizes[1] != KS * NN || out_size != MM * NN) {
        fprintf(stderr, "kernel_launch: unexpected shapes\n"); return; }
    Params P = {};
    for (int p = 0; p < NR; ++p) P.ws[p] = (char*)d_ws_all[p];
    P.x = (const u16*)d_in[0]; P.w = (const u16*)d_in[1]; P.out = (u16*)d_out; P.rank = rank;
    build_tables(P);
    HIP_CHECK(hipFuncSetAttribute((const void*)fused, hipFuncAttributeMaxDynamicSharedMemorySize, LDS_BYTES));
    hipLaunchKernelGGL(fused, dim3(NWG), dim3(512), LDS_BYTES, stream, P);
    HIP_CHECK(hipGetLastError());
}
```

```cpp
#include <hip/hip_runtime.h>
#include <rccl/rccl.h>
#include <cstdio>
#include <cstdint>
#include <cstddef>
#pragma clang diagnostic ignored "-Wint-to-pointer-cast"

#define HIP_CHECK(expr) do { hipError_t _e = (expr); if (_e != hipSuccess) { \
    fprintf(stderr, "HIP error %s at %s:%d\n", hipGetErrorString(_e), __FILE__, __LINE__); \
    return; } } while (0)

namespace gar {
#define LAS __attribute__((address_space(3)))
#define GAS __attribute__((address_space(1)))
typedef unsigned u32;
typedef unsigned short u16;
typedef unsigned long long u64;
typedef unsigned u32x4 __attribute__((ext_vector_type(4)));
typedef unsigned u32x2 __attribute__((ext_vector_type(2)));
typedef short s16x4 __attribute__((ext_vector_type(4)));
typedef short bf16x8 __attribute__((ext_vector_type(8)));
typedef float f32x4 __attribute__((ext_vector_type(4)));

constexpr int NR = 8, MM = 4096, NN = 32768, KS = 512;
#define GAR_DP 11
constexpr int DP = GAR_DP;
constexpr int RP = 128 - NR * DP;
constexpr int LP = DP + RP;
constexpr int NWG = 256;
#define GAR_NL 9
#define GAR_FL 0
#define GAR_NH 8
#define GAR_FH 1
#define GAR_CL 6
#define GAR_WIN 6
constexpr int NL = GAR_NL, FL = GAR_FL, NH = GAR_NH, FH = GAR_FH, CL = GAR_CL, CH = 6, CPR = CL + CH;
constexpr int NPULL = 3 * (NL + FL) + 4 * (NH + FH);
constexpr int WIN = GAR_WIN;
static_assert(3 * NL + 4 * FH + 16 * CL <= 128 && 4 * NH + 3 * FL + 16 * CH <= 128, "roles");
#define GAR_SP2 1
#define GAR_TAILRAW 0
#define GAR_ALIGN 1
#define GAR_NRAW 18
#define GAR_SOLOG 16
#define GAR_LEAD 60
#define GAR_HOLD 200
constexpr int SOLOG = GAR_SOLOG, NRAW = GAR_NRAW;
constexpr int LEAD = GAR_LEAD;
static_assert(NRAW <= 24 && NRAW % 2 == 0 && 8 * SOLOG <= NWG - NPULL && SOLOG == 16, "solo chunks");
constexpr int NENC = NWG - NPULL;
constexpr int GX = 0, GW = 32, GY = 288;
constexpr int NGID = GY + 16 * DP;
constexpr int NS = 32 + 2 * DP + 2 * RP + 16 * DP;
constexpr int ROUNDS = (LP + CPR - 1) / CPR;
constexpr int NUMAX = 544;
#define GAR_NLIGHT 80
constexpr int NLIGHT = GAR_NLIGHT;
constexpr int NSTAT = 3;
constexpr int NPACK = 288 - SOLOG;
constexpr u64 DEADLINE = 30000000ull;

constexpr size_t WS_CALL = 0;
constexpr size_t WS_TMO = 32768;
constexpr size_t WS_CNT = 36864;
constexpr size_t WS_CLAIM = 57344;
constexpr size_t WS_FLAG = 65536;
constexpr size_t FLAG_SRC = 544 * 32, FLAG_PAR = 8 * FLAG_SRC;
constexpr size_t WS_DBG = 524288;
#define GAR_SKEW 0
constexpr size_t GRP_STRIDE = 131072 + GAR_SKEW;
constexpr size_t WS_STG = 2u << 20;
constexpr size_t STG_PAR = (544 * GRP_STRIDE + 0x1fffff) & ~(size_t)0x1fffff;
constexpr size_t WS_XI = WS_STG + 2 * STG_PAR;
constexpr size_t PANEL = 64 * 32768;
constexpr size_t WS_WI = WS_XI + 32 * PANEL;
constexpr size_t WS_END = WS_WI + (size_t)LP * PANEL;
static_assert(WS_FLAG + 2 * FLAG_PAR <= WS_DBG && NGID <= 544, "layout");

constexpr u32 LDS_CTL = 131072;
constexpr u32 LDS_LUT = LDS_CTL + 512, LDS_HIST = LDS_CTL + 1024;
constexpr u32 LDS_TAB = LDS_HIST + 4096, LDS_BYTES = LDS_TAB + 3072;
constexpr u32 CTL_CALL = LDS_CTL  , CTL_SIZE = LDS_CTL + 64  , CTL_BAD = LDS_CTL + 128;
constexpr u32 CTL_NUNIT = LDS_CTL + 192, CTL_LANDED = LDS_CTL + 256, CTL_NCHUNK = LDS_CTL + 320, CTL_NEXT = LDS_CTL + 384  ;
constexpr u32 E_L2 = 0, E_L3 = E_L2 + 4160, E_VAR = E_L3 + 2112, E_END = E_VAR + 2112;
static_assert(E_END <= 9216 && E_L3 % 16 == 0 && E_VAR % 16 == 0, "packer scratch");
constexpr u32 CAP2 = 4080, CAP3 = 2032, CAPV = 2032, FIXB = 10240, RAWB = 16384, RAWCODE = 255;
constexpr u32 RING = 65536, RMASK = RING - 1, ND = 6;
constexpr u32 D_SCR = RING, D_SCRB = 6400, D_L2 = 0, D_L3 = 4160;
constexpr u32 D_DESC = D_SCR + ND * D_SCRB, D_UNIT = D_DESC + NS * 64, DONE = 0x80000000u;
static_assert(D_L3 + 2112 <= D_SCRB && D_UNIT + (NUMAX + 8 * SOLOG) * 8 <= LDS_CTL && D_DESC % 16 == 0, "puller LDS");

struct Args {
    char* ws[NR];
    char* me; const u16* x; const u16* w; u16* out;
    int rank, nunits;
    u32 role[NWG / 2];
    u32 stream[NUMAX / 2];
    u32 enc[144];
    u32 cols[64];
    u32 pad[(3072 - 104) / 4 - NWG / 2 - NUMAX / 2 - 144 - 64];
};
static_assert(sizeof(Args) == 3072 && ROUNDS < 8, "kernarg");
constexpr u32 TAB_STREAM = LDS_TAB + offsetof(Args, stream), TAB_ENC = LDS_TAB + offsetof(Args, enc), TAB_COLS = LDS_TAB + offsetof(Args, cols);
struct Params { char* me; char* peer; const u16* x; const u16* w; u16* out; u32 rank, nunits; };
template <class T> __device__ __forceinline__ T* pin(T* p) { u64 v = (u64)p; asm volatile("" : "+s"(v)); return (T*)(GAS T*)v; }
__device__ __forceinline__ u32 pin(u32 v) { asm volatile("" : "+s"(v)); return v; }

__device__ __forceinline__ u32 lds32(u32 a) { return *(LAS u32*)a; }
__device__ __forceinline__ u32 lds32v(u32 a) { return *(volatile LAS u32*)a; }
__device__ __forceinline__ void lds32vw(u32 a, u32 x) { *(volatile LAS u32*)a = x; }
__device__ __forceinline__ u32x2 lds64v(u32 a) { return *(volatile LAS u32x2*)a; }
__device__ __forceinline__ void lds64vw(u32 a, u32x2 x) { *(volatile LAS u32x2*)a = x; }
__device__ __forceinline__ u32x4 lds128(u32 a) { return *(LAS u32x4*)a; }
__device__ __forceinline__ void lds128w(u32 a, u32x4 x) { *(LAS u32x4*)a = x; }
__device__ __forceinline__ void lds_or(u32 a, u32 x) { __hip_atomic_fetch_or((LAS u32*)a, x, __ATOMIC_RELAXED, __HIP_MEMORY_SCOPE_WORKGROUP); }
__device__ __forceinline__ void lds_add(u32 a, u32 x) { __hip_atomic_fetch_add((LAS u32*)a, x, __ATOMIC_RELAXED, __HIP_MEMORY_SCOPE_WORKGROUP); }
__device__ __forceinline__ u32 rfl(u32 x) { return (u32)__builtin_amdgcn_readfirstlane((int)x); }
__device__ __forceinline__ u32 tab16(u32 t, u32 i) { return rfl(*(volatile LAS u16*)(t + i * 2)); }
#define CBAR() asm volatile("" ::: "memory")
#define WAIT_LGKM0() asm volatile("s_waitcnt lgkmcnt(0)" ::: "memory")
#define WAIT_VM(n) asm volatile("s_waitcnt vmcnt(%0)" :: "i"(n) : "memory")
#define BAR() do { CBAR(); __builtin_amdgcn_s_barrier(); CBAR(); } while (0)
__device__ __forceinline__ void st_sys(void* p, u32x4 x) { asm volatile("global_store_dwordx4 %0, %1, off sc0 sc1\n\ts_nop 1" :: "v"(p), "v"(x) : "memory"); }
__device__ __forceinline__ void st_dev(void* p, u32x4 x) { asm volatile("global_store_dwordx4 %0, %1, off sc1\n\ts_nop 1" :: "v"(p), "v"(x) : "memory"); }
#define GAR_OST 3
#define GAR_GST 1
template <int MODE> __device__ __forceinline__ void st_out(void* p, u32x4 x) {
    if (MODE == 0) *(u32x4*)p = x;
    else if (MODE == 1) __builtin_nontemporal_store(x, (u32x4*)p);
    else if (MODE == 2) st_dev(p, x);
    else if (MODE == 9) asm volatile("" :: "v"(p), "v"(x) : "memory");
    else asm volatile("global_store_dwordx4 %0, %1, off sc1 nt\n\ts_nop 1" :: "v"(p), "v"(x) : "memory");
}
__device__ __forceinline__ void glds_sys(const void* g, u32 dst) {
    u32 keep;
    asm volatile("s_mov_b32 %0, m0\n\ts_mov_b32 m0, %2\n\ts_nop 0\n\tglobal_load_lds_dwordx4 %1, off sc0 sc1\n\ts_mov_b32 m0, %0" : "=&s"(keep) : "v"(g), "s"(dst) : "memory");
}
__device__ __forceinline__ void glds_plain(const void* g, u32 dst) {
    u32 keep;
    asm volatile("s_mov_b32 %0, m0\n\ts_mov_b32 m0, %2\n\ts_nop 0\n\tglobal_load_lds_dwordx4 %1, off\n\ts_mov_b32 m0, %0" : "=&s"(keep) : "v"(g), "s"(dst) : "memory");
}
__device__ __forceinline__ u64 now() { return __builtin_amdgcn_s_memrealtime(); }
template <int CTRL, int RM, bool BC> __device__ __forceinline__ u32 dpp(u32 v) { return (u32)__builtin_amdgcn_update_dpp(0, (int)v, CTRL, RM, 0xf, BC); }
__device__ __forceinline__ u32 scan_incl(u32 v) {
    v += dpp<0x111, 0xf, true>(v); v += dpp<0x112, 0xf, true>(v); v += dpp<0x114, 0xf, true>(v); v += dpp<0x118, 0xf, true>(v);
    v += dpp<0x142, 0xa, false>(v); v += dpp<0x143, 0xc, false>(v); return v;
}
__device__ __forceinline__ u32 umax(u32 a, u32 b) { return a > b ? a : b; }
__device__ __forceinline__ u32 umin(u32 a, u32 b) { return a < b ? a : b; }
__device__ __forceinline__ u32 wave_max(u32 v) {
    v = umax(v, dpp<0x111, 0xf, true>(v)); v = umax(v, dpp<0x112, 0xf, true>(v)); v = umax(v, dpp<0x114, 0xf, true>(v)); v = umax(v, dpp<0x118, 0xf, true>(v));
    v = umax(v, dpp<0x142, 0xa, false>(v)); v = umax(v, dpp<0x143, 0xc, false>(v)); return (u32)__builtin_amdgcn_readlane((int)v, 63);
}
__device__ __forceinline__ u32 popc(u32 x) { return (u32)__builtin_popcount(x); }
__device__ __forceinline__ u32 perm(u32 hi, u32 lo, u32 sel) { return __builtin_amdgcn_perm(hi, lo, sel); }

#define STAMP(P, slot) do {} while (0)
#define TICK(t) do {} while (0)
#define ACC(sum, t) do {} while (0)
#define DUMP(P, slot, val) do {} while (0)

#define CSTAMP(i) do {} while (0)
#define CS_ARG
#define CS_DECL
#define CS_PASS

__device__ __forceinline__ u32 nzb(u32 z) { return ((z & 0x7f7f7f7fu) + 0x7f7f7f7fu) | z; }
typedef unsigned short u16x2 __attribute__((ext_vector_type(2)));
struct Classifier {
    u32 s1, s2, s3, nb, tlo, thi; bool win;
    __device__ __forceinline__ void init(u32 t) {
        const u32 a = (t >> 8) & 255u, b = (t >> 16) & 255u, c = t >> 24, lo = umin(a, umin(b, c));
        win = umax(a, umax(b, c)) - lo <= 6u;
        const u64 l = (1ull << (((a - lo) & 7u) * 8)) | (2ull << (((b - lo) & 7u) * 8)) | (3ull << (((c - lo) & 7u) * 8));
        tlo = (u32)l; thi = (u32)(l >> 32); nb = ((0u - lo) & 255u) * 0x01000100u;
        s1 = a * 0x01010101u; s2 = b * 0x01010101u; s3 = c * 0x01010101u;
    }
    __device__ __forceinline__ u32 any(u32 e) const {
        const u32 a1 = nzb(e ^ s1), a2 = nzb(e ^ s2), a3 = nzb(e ^ s3);
        return ((~(a1 & a3) >> 7) & 0x01010101u) | ((~(a2 & a3) >> 6) & 0x02020202u);
    }
    __device__ __forceinline__ u32 near(u32 e) const {
        const u16x2 x = __builtin_bit_cast(u16x2, e), n = __builtin_bit_cast(u16x2, nb), top = {0x07ff, 0x07ff};
        u32 sh; asm("v_pk_mad_u16 %0, %1, %2, %3" : "=v"(sh) : "v"(e), "v"(0x01000100u), "s"(nb));
        const u16x2 hi = __builtin_elementwise_min((u16x2)(x + n), top), lo = __builtin_elementwise_min(__builtin_bit_cast(u16x2, sh), top);
        return perm(thi, tlo, perm(__builtin_bit_cast(u32, hi), __builtin_bit_cast(u32, lo), 0x07030501u));
    }
    template <int N> __device__ __forceinline__ void run(const u32* e, u32* c) const {
        if (win) {
#pragma unroll
            for (int i = 0; i < N; ++i) c[i] = near(e[i]);
        } else {
#pragma unroll
            for (int i = 0; i < N; ++i) c[i] = any(e[i]);
        }
    }
};
__device__ __forceinline__ u32 escbits(u32 c) { return ~(c | (c >> 1)) & 0x55555555u; }
struct Appender {
    u32 lo, fill, addr;
    __device__ __forceinline__ void init(u32 off) { lo = 0; fill = (off & 3u) * 8u; addr = off & ~3u; }
    __device__ __forceinline__ void push(u32 k, u32 mk) {
        const u64 t = (u64)k << fill;
        lo |= (u32)t; fill += popc(mk);
        if (fill >= 32u) { lds_or(addr, lo); lo = (u32)(t >> 32); addr += 4; }
        fill &= 31u;
    }
    __device__ __forceinline__ void flush() { if (lo) lds_or(addr, lo); }
};
__device__ __forceinline__ u32 escmask(u32 c) { return perm(0u, 0xffu, c); }
__device__ __forceinline__ u32 selector(u32 mk) { return lds32(__builtin_amdgcn_udot4(mk & 0x20100804u, 0x01010101u, LDS_LUT, false)); }
template <int R> __device__ __forceinline__ u32 pack_level(u32 src, u32 n, u32 t, u32 cdst, u32 edst, u32 cap, u32 lane) {
    Classifier K; K.init(t);
    if (lane < 16) *(LAS unsigned char*)(src + n + lane) = (unsigned char)K.s1;
    u32 ne = 0;
    for (u32 r = 0; r * 1024 < n; r += R) {
        CBAR();
        u32 q[4 * R], c[4 * R], m[4 * R], l[4 * R], cnt = 0;
#pragma unroll
        for (int i = 0; i < R; ++i) { const u32 p0 = (r + i) * 1024 + lane * 16; const u32x4 x = lds128(src + p0); const bool act = p0 < n;
            q[4 * i] = act ? x.x : K.s1; q[4 * i + 1] = act ? x.y : K.s1; q[4 * i + 2] = act ? x.z : K.s1; q[4 * i + 3] = act ? x.w : K.s1; }
        K.run<4 * R>(q, c);
#pragma unroll
        for (int i = 0; i < 4 * R; ++i) { m[i] = escmask(c[i]); l[i] = selector(m[i]); }
#pragma unroll
        for (int i = 0; i < R; ++i) cnt |= ((popc(m[4 * i]) + popc(m[4 * i + 1]) + popc(m[4 * i + 2]) + popc(m[4 * i + 3])) >> 3) << (16 * i);
        const u32 inc = scan_incl(cnt), tot = (u32)__builtin_amdgcn_readlane((int)inc, 63), exc = inc - cnt;
        if (ne + (tot & 0xffffu) + (tot >> 16) > cap) return ~0u;
#pragma unroll
        for (int i = 0; i < R; ++i) {
            Appender A; A.init(edst + ne + (i ? (tot & 0xffffu) + (exc >> 16) : (exc & 0xffffu)));
#pragma unroll
            for (int j = 0; j < 4; ++j) A.push(perm(0u, q[4 * i + j], l[4 * i + j]), m[4 * i + j]);
            A.flush();
            if ((r + i) * 1024 + lane * 16 < n) *(LAS u32*)(cdst + ((r + i) * 64 + lane) * 4) = c[4 * i] | (c[4 * i + 1] << 2) | (c[4 * i + 2] << 4) | (c[4 * i + 3] << 6);
        }
        ne += (tot & 0xffffu) + (tot >> 16);
    }
    CBAR();
    return ne;
}
template <int N> __device__ __forceinline__ u32 rank_ror(u32 key) { return (dpp<0x120 + N, 0xf, true>(key) > key ? 1u : 0u) + rank_ror<N - 1>(key); }
template <> __device__ __forceinline__ u32 rank_ror<0>(u32) { return 0u; }
__device__ __forceinline__ u32 row_or(u32 x) { x |= dpp<0x128, 0xf, true>(x); x |= dpp<0x124, 0xf, true>(x); x |= dpp<0x122, 0xf, true>(x); x |= dpp<0x121, 0xf, true>(x); return rfl(x); }
__device__ __forceinline__ void split(const u32 (&v)[64], u32 (&Mr)[32], u32 (&Er)[32]) {
#pragma unroll
    for (int g = 0; g < 32; ++g) {
        const u32 a = __builtin_amdgcn_alignbit(v[2 * g], v[2 * g], 31), b = __builtin_amdgcn_alignbit(v[2 * g + 1], v[2 * g + 1], 31);
        Er[g] = perm(b, a, 0x07050301u); Mr[g] = perm(b, a, 0x06040200u);
        asm volatile("" : "+v"(Er[g]), "+v"(Mr[g]));
    }
}
__device__ __forceinline__ u32 pack_chunk(const u32 (&v)[64], u32 (&Mr)[32], u32 (&Er)[32], u32 (&Cr)[8], u32 scr, u32 lane, bool solo CS_ARG) {
    CBAR(); CSTAMP(0);
    const u32 hist = solo ? scr + 10240 : LDS_HIST;
    if (solo) {
#pragma unroll
        for (int j = 0; j < 4; ++j) lds128w(hist + j * 1024 + lane * 16, (u32x4){0u, 0u, 0u, 0u});
        CBAR();
#pragma unroll
        for (int i = 0; i < 12; ++i) lds_add(hist + (((v[i * 5 + 2] >> ((i & 1) * 16 + 3)) & 0xff0u) | ((lane & 3) * 4)), 1u);
    }
#pragma unroll
    for (int i = 0; i < 4; ++i) lds_add(hist + (((v[i * 15 + 5] >> ((i & 1) * 16 + 3)) & 0xff0u) | ((lane & 3) * 4)), 1u);
#pragma unroll
    for (int j = 0; j < 9; ++j) lds128w(scr + j * 1024 + lane * 16, (u32x4){0u, 0u, 0u, 0u});
    WAIT_LGKM0();
    if (!solo) BAR();
    CBAR();
    u32 key = 0;
#pragma unroll
    for (int i = 0; i < 4; ++i) { const u32 bin = i * 64 + lane; const u32x4 h = lds128(hist + bin * 16); key = umax(key, ((h.x + h.y + h.z + h.w) << 8) | bin); }
    split(v, Mr, Er);
    CSTAMP(1);
    { const auto r = __builtin_amdgcn_permlane32_swap(key, key, false, false); key = umax(r[0], r[1]); }
    { const auto r = __builtin_amdgcn_permlane16_swap(key, key, false, false); key = umax(r[0], r[1]); }
    const u32 rank = rank_ror<15>(key), lev = (rank * 11u) >> 5, sym = (key & 255u) << (8 * (rank - 3 * lev + 1));
    const u32 T[3] = {row_or(lev == 0 ? sym : 0u), row_or(lev == 1 ? sym : 0u), row_or(lev == 2 ? sym : 0u)};
    CSTAMP(3);
    u32 cnt = 0, mk[32], sl[32];
    {
        Classifier K; K.init(T[0]);
        u32 c[32]; K.run<32>(Er, c);
#pragma unroll
        for (int g = 0; g < 32; ++g) { mk[g] = escmask(c[g]); sl[g] = selector(mk[g]); cnt += popc(mk[g]); }
#pragma unroll
        for (int k = 0; k < 8; ++k) Cr[k] = c[4 * k] | (c[4 * k + 1] << 2) | (c[4 * k + 2] << 4) | (c[4 * k + 3] << 6);
    }
    cnt >>= 3;
    const u32 inc = scan_incl(cnt), n2 = (u32)__builtin_amdgcn_readlane((int)inc, 63);
    CSTAMP(4);
    if (n2 > CAP2) return RAWCODE;
    Appender A; A.init(scr + E_L2 + inc - cnt);
#pragma unroll
    for (int g = 0; g < 32; ++g) A.push(perm(0u, Er[g], sl[g]), mk[g]);
    A.flush();
    CBAR(); CSTAMP(5);
    const u32 d2 = (n2 + 15) >> 4;
    const u32 n3 = pack_level<2>(scr + E_L2, n2, T[1], scr + E_VAR + 12, scr + E_L3, CAP3, lane);
    CSTAMP(6);
    if (n3 == ~0u) return RAWCODE;
    const u32 d3 = (n3 + 15) >> 4, b4 = 12 + 4 * (d2 + d3);
    const u32 n4 = pack_level<1>(scr + E_L3, n3, T[2], scr + E_VAR + 12 + 4 * d2, scr + E_VAR + b4, CAPV - b4, lane);
    if (n4 == ~0u) return RAWCODE;
    if (lane == 0) { *(LAS u32*)(scr + E_VAR) = T[0] | d2; *(LAS u32*)(scr + E_VAR + 4) = T[1] | d3; *(LAS u32*)(scr + E_VAR + 8) = T[2]; }
    CBAR();
    return (b4 + n4 + 15) >> 4;
}
__device__ __forceinline__ void pack_init(u32 wid, u32 lane) {
    if (wid >= 4) lds128w(LDS_HIST + (wid - 4) * 1024 + lane * 16, (u32x4){0u, 0u, 0u, 0u});
    if (wid == 1 && lane < 16) { u32 sel = 0x0c0c0c0cu, k = 0;
#pragma unroll
        for (u32 b = 0; b < 4; ++b) if ((lane >> b) & 1) { sel = (sel & ~(255u << k)) | (b << k); k += 8; }
        *(LAS u32*)(LDS_LUT + lane * 4) = sel; }
}
__device__ __forceinline__ u32 chunk_bytes(u32 code) { return code == RAWCODE ? RAWB : FIXB + 16 * code; }

__device__ __forceinline__ void pack_store(char* dst, u32 code, const u32 (&Mr)[32], const u32 (&Er)[32], const u32 (&Cr)[8], u32 scr, u32 lane) {
#pragma unroll
    for (int j = 0; j < 8; ++j) st_sys(dst + j * 1024, (u32x4){Mr[4 * j], Mr[4 * j + 1], Mr[4 * j + 2], Mr[4 * j + 3]});
    if (code == RAWCODE) {
#pragma unroll
        for (int j = 0; j < 8; ++j) st_sys(dst + 8192 + j * 1024, (u32x4){Er[4 * j], Er[4 * j + 1], Er[4 * j + 2], Er[4 * j + 3]});
    } else {
        st_sys(dst + 8192, (u32x4){Cr[0], Cr[1], Cr[2], Cr[3]}); st_sys(dst + 9216, (u32x4){Cr[4], Cr[5], Cr[6], Cr[7]});
        if (lane < code) st_sys(dst + FIXB, lds128(scr + E_VAR + lane * 16));
        if (lane + 64 < code) st_sys(dst + FIXB + 1024, lds128(scr + E_VAR + 1024 + lane * 16));
    }
}
__device__ __forceinline__ void raise_flag(const Params& P, u32 q, u32 gid, u32 w, u32 call, u32 code) {
    char* wq = (char*)(GAS char*)__builtin_bit_cast(u64, lds64v(LDS_TAB + q * 8));
    u32* f = (u32*)(wq + WS_FLAG + (call & 1) * FLAG_PAR + (size_t)P.rank * FLAG_SRC + gid * 32) + w;
    __hip_atomic_store(f, (call << 8) | code, __ATOMIC_RELAXED, __HIP_MEMORY_SCOPE_SYSTEM);
}
__device__ __forceinline__ void pack_group(const Params& P, const u32 (&v)[64], u32 (&Mr)[32], u32 (&Er)[32], u32 gid, u32 w, bool solo, u32 peers, u32 call, u32 wid, u32 lane) {
    u32 Cr[8];
    const u32 scr = wid * 16384;
    CS_DECL
    if (gid < 64u) STAMP(P, 40 + wid);
    u32 code = rfl(pack_chunk(v, Mr, Er, Cr, scr, lane, solo CS_PASS));
    if (gid == (u32)(NGID - 1) && w >= (u32)(8 - GAR_TAILRAW)) code = RAWCODE;
    if (gid < 64u) STAMP(P, 48 + wid);
    u32 off = w * RAWB, mine = code;
    if (!solo) {
        if (lane == 0) lds32vw(CTL_SIZE + wid * 4, code);
        WAIT_LGKM0(); BAR();
        mine = lds32v(CTL_SIZE + (lane & 7) * 4);
        if (lane < 32) lds128w(LDS_HIST + wid * 512 + lane * 16, (u32x4){0u, 0u, 0u, 0u});
        off = 0;
#pragma unroll
        for (u32 j = 0; j < 7; ++j) { const u32 c = (u32)__builtin_amdgcn_readlane((int)mine, j); if (j < wid) off += chunk_bytes(c); }
    }
    pack_store(P.me + WS_STG + (call & 1) * STG_PAR + (size_t)gid * GRP_STRIDE + off + lane * 16, code, Mr, Er, Cr, scr, lane);
    if (gid < 64u) STAMP(P, 56 + wid);
    WAIT_VM(0);
    if (gid < 64u) STAMP(P, 64 + wid);
    WAIT_LGKM0();
    if (!solo) BAR();
    if ((solo ? (lane & 7) == w : wid == 0) && ((peers >> (lane >> 3)) & 1)) raise_flag(P, lane >> 3, gid, lane & 7, call, mine);
}

#define GAR_WINMODE 1
template <bool WRAP> struct Unp {
    u32 sel[4], lo[4], hi[4], of[4];
    __device__ __forceinline__ void fetch(u32 c, u32& off) {
#pragma unroll
        for (int j = 0; j < 4; ++j) {
            const u32 cj = (c >> (2 * j)) & 0x03030303u, mk = escmask(cj), m2 = mk & 0x02020202u;
            if (WRAP || GAR_WINMODE != 2) { const u32 a = off & ~3u; lo[j] = lds32(WRAP ? (a & RMASK) : a); hi[j] = lds32(WRAP ? ((a + 4) & RMASK) : a + 4); of[j] = off; }
            else lo[j] = lds32(off);
            u32 p; asm("v_mad_u32_u24 %0, %1, %2, %3" : "=v"(p) : "v"(m2), "s"(0x808080u), "v"(0x04040404u));
            sel[j] = (mk & p) | cj;
            off += popc(m2);
        }
    }
    __device__ __forceinline__ u32 finish(int j, u32 t) const {
        u32 w;
        if (WRAP || GAR_WINMODE == 0) w = __builtin_amdgcn_alignbyte(hi[j], lo[j], of[j] & 3u);
        else if (GAR_WINMODE == 1) w = __builtin_amdgcn_alignbyte(hi[j], lo[j], of[j]);
        else w = lo[j];
        return perm(w, t, sel[j]);
    }
    __device__ __forceinline__ u32x4 finish4(u32 t) const { return (u32x4){finish(0, t), finish(1, t), finish(2, t), finish(3, t)}; }
};
#define SCHED() __builtin_amdgcn_sched_barrier(0)
template <bool WRAP> __device__ __forceinline__ u32 unpack_round(u32 c, u32 t, u32 src, u32 dst) {
    const u32 cnt = popc(escbits(c)), inc = scan_incl(cnt);
    u32 off = src + inc - cnt;
    Unp<WRAP> u; u.fetch(c, off); SCHED();
    lds128w(dst, u.finish4(t));
    return (u32)__builtin_amdgcn_readlane((int)inc, 63);
}
__device__ __forceinline__ void unpack_chunk(u32 S, u32 code, u32 scr, u32 lane, u32 rel_addr, u32 rel_val, u32 (&Mr)[32], u32 (&Er)[32] CS_ARG) {
    const u32 a0 = S + lane * 16;
#pragma unroll
    for (int j = 0; j < 8; ++j) { const u32x4 q = lds128((a0 + j * 1024) & RMASK); Mr[4 * j] = q.x; Mr[4 * j + 1] = q.y; Mr[4 * j + 2] = q.z; Mr[4 * j + 3] = q.w; }
    if (code == RAWCODE) {
#pragma unroll
        for (int j = 0; j < 8; ++j) { const u32x4 q = lds128((a0 + 8192 + j * 1024) & RMASK); Er[4 * j] = q.x; Er[4 * j + 1] = q.y; Er[4 * j + 2] = q.z; Er[4 * j + 3] = q.w; }
        WAIT_LGKM0(); lds32vw(rel_addr, rel_val); CBAR();
        return;
    }
    const u32x4 ca = lds128((a0 + 8192) & RMASK), cb = lds128((a0 + 9216) & RMASK);
    const u32 V = S + FIXB;
    const u32x4 hd = lds128(V & RMASK);
    const u32 t1 = rfl(hd.x), t2 = rfl(hd.y), t3 = rfl(hd.z), d2 = t1 & 255u, d3 = t2 & 127u;
    u32 c2[4], c3[2];
#pragma unroll
    for (int r = 0; r < 4; ++r) { const u32 i = r * 64 + lane; const u32 x = lds32((V + 12 + 4 * i) & RMASK); c2[r] = i < d2 ? x : 0x55555555u; }
#pragma unroll
    for (int r = 0; r < 2; ++r) { const u32 i = r * 64 + lane; const u32 x = lds32((V + 12 + 4 * (d2 + i)) & RMASK); c3[r] = i < d3 ? x : 0x55555555u; }
    CSTAMP(1);
    u32 s4 = V + 12 + 4 * (d2 + d3);
    s4 += unpack_round<true>(c3[0], t3, s4, scr + D_L3 + lane * 16);
    if (d3 > 64) unpack_round<true>(c3[1], t3, s4, scr + D_L3 + 1024 + lane * 16);
    WAIT_LGKM0(); lds32vw(rel_addr, rel_val); CBAR();
    CSTAMP(2);
    const u32 C[8] = {ca.x, ca.y, ca.z, ca.w, cb.x, cb.y, cb.z, cb.w};
    u32 cnt = 0;
#pragma unroll
    for (int k = 0; k < 8; ++k) cnt += popc(escbits(C[k]));
    {
        const u32 n0 = popc(escbits(c2[0])), n1 = popc(escbits(c2[1]));
        const u32 i01 = scan_incl(n0 | (n1 << 16)), tot = (u32)__builtin_amdgcn_readlane((int)i01, 63);
        cnt = scan_incl(cnt) - cnt;
        u32 o0 = scr + D_L3 + (i01 & 0xffffu) - n0, o1 = scr + D_L3 + (tot & 0xffffu) + (i01 >> 16) - n1;
        Unp<false> u0, u1; u0.fetch(c2[0], o0); u1.fetch(c2[1], o1); SCHED();
        lds128w(scr + D_L2 + lane * 16, u0.finish4(t2)); lds128w(scr + D_L2 + 1024 + lane * 16, u1.finish4(t2));
        if (d2 > 128) {
            u32 s3 = scr + D_L3 + (tot & 0xffffu) + (tot >> 16);
            s3 += unpack_round<false>(c2[2], t2, s3, scr + D_L2 + 2048 + lane * 16);
            if (d2 > 192) unpack_round<false>(c2[3], t2, s3, scr + D_L2 + 3072 + lane * 16);
        }
    }
    CBAR(); CSTAMP(3);
    u32 off = scr + D_L2 + cnt;
    Unp<false> u[8];
    u[0].fetch(C[0], off); u[1].fetch(C[1], off);
#pragma unroll
    for (int k = 0; k < 8; ++k) {
        SCHED();
        if (k + 2 < 8) u[k + 2].fetch(C[k + 2], off);
        SCHED();
#pragma unroll
        for (int j = 0; j < 4; ++j) Er[4 * k + j] = u[k].finish(j, t1);
    }
    CBAR();
}
__device__ __forceinline__ u32x4 piece(const u32 (&Mr)[32], const u32 (&Er)[32], int p) {
    const u32 a = perm(Er[2 * p], Mr[2 * p], 0x05010400u), b = perm(Er[2 * p], Mr[2 * p], 0x07030602u);
    const u32 c = perm(Er[2 * p + 1], Mr[2 * p + 1], 0x05010400u), d = perm(Er[2 * p + 1], Mr[2 * p + 1], 0x07030602u);
    return (u32x4){__builtin_amdgcn_alignbit(a, a, 1), __builtin_amdgcn_alignbit(b, b, 1), __builtin_amdgcn_alignbit(c, c, 1), __builtin_amdgcn_alignbit(d, d, 1)};
}

__device__ __forceinline__ u32 x_piece_off(u32 half, int p) { return half * 16384 + p * 1024; }
__device__ __forceinline__ u32 w_piece_off(u32 h, int p) {
    const u32 g32 = p >> 2, n = (p >> 1) & 1, kb = p & 1;
    return (g32 & 1) * 16384 + ((2 * (2 * h + (g32 >> 1)) + n) * 2 + kb) * 1024;
}
__device__ __forceinline__ u16* y_base(const Params& P, u32 pm, u32 j, u32 w, u32 lane) {
    return P.out + (size_t)(pm * 256 + (w >> 2) * 64 + (lane & 7)) * NN + j * 256 + (w & 3) * 64 + (lane & 8) * 4 + (lane >> 4) * 8;
}
__device__ __forceinline__ u16* y_base_lin(const Params& P, u32 pm, u32 j, u32 w, u32 lane) {
    return P.out + (size_t)(pm * 256 + (w >> 2) * 64 + (lane >> 3)) * NN + j * 256 + (w & 3) * 64 + (lane & 7) * 8;
}
__device__ __forceinline__ size_t y_piece_off(int p) { return (size_t)((p >> 3) * 128 + (p & 7) * 8) * NN; }
__device__ __forceinline__ u32 local_panel(u32 j, u32 rank) { return j >= (u32)(NR * DP) ? j - (NR - 1) * DP : j - rank * DP; }
__device__ __forceinline__ u32 global_panel(u32 jj, u32 rank) { return jj >= (u32)DP ? jj + (NR - 1) * DP : jj + rank * DP; }
__device__ __forceinline__ u32* counter(const Params& P, u32 i) { return (u32*)(P.me + WS_CNT + i * 128); }
__device__ __forceinline__ u32* tmo_word(const Params& P) { return (u32*)(P.me + WS_TMO); }
__device__ __forceinline__ bool expired(const Params& P, u64 t0, u32 call) {
    if (__hip_atomic_load(tmo_word(P), __ATOMIC_RELAXED, __HIP_MEMORY_SCOPE_AGENT) == call || now() - t0 > DEADLINE) {
        __hip_atomic_store(tmo_word(P), call, __ATOMIC_RELAXED, __HIP_MEMORY_SCOPE_AGENT); return true; }
    return false;
}

__device__ __forceinline__ size_t image_off(u32 gid, u32 w, u32 q, u32 r, u32 call, u32 lane, u32& cnt) {
    const bool isx = gid < (u32)GW;
    const u32 pn = isx ? gid & 15 : local_panel((gid - GW) >> 1, r), ktl = (isx ? gid >> 4 : gid & 1) * 4 + (w >> 1);
    const u32 qi = (q - r - 1) & 7, stage = qi == 7 ? 0u : 1 + (ktl >> 1);
    const u32 slot = stage == 0 ? ktl : stage * 14 - 6 + qi * 2 + (ktl & 1);
    cnt = (isx ? pn : 16 + pn) * 32 + stage;
    return (isx ? WS_XI + (call & 1) * 16 * PANEL : WS_WI) + (size_t)pn * PANEL + (size_t)slot * 32768 + lane * 16;
}
__device__ __forceinline__ void store_image(const Params& P, u32 gid, u32 w, u32 q, u32 call, u32 lane, const u32 (&Mr)[32], const u32 (&Er)[32]) {
    u32 cnt; char* o = P.me + image_off(gid, w, q, P.rank, call, lane, cnt);
#pragma unroll
    for (int p = 0; p < 16; ++p) st_dev(o + (gid < (u32)GW ? x_piece_off(w & 1, p) : w_piece_off(w & 1, p)), piece(Mr, Er, p));
    WAIT_VM(0);
    if (lane == 0) __hip_atomic_fetch_add(counter(P, 0) + cnt, 1u, __ATOMIC_RELAXED, __HIP_MEMORY_SCOPE_AGENT);
}
__device__ __forceinline__ u32* call_word(const Params& P, u32 wid) { return (u32*)(P.me + WS_CALL + blockIdx.x * 128 + wid * 4); }
__device__ __forceinline__ void call_load(const Params& P, u32 wid, u32 lane) {
    u32 keep;
    if (lane == 0) asm volatile("s_mov_b32 %0, m0\n\ts_mov_b32 m0, %2\n\ts_nop 0\n\tglobal_load_lds_dword %1, off sc1\n\ts_mov_b32 m0, %0" : "=&s"(keep) : "v"(call_word(P, wid)), "s"(CTL_CALL + wid * 4) : "memory");
}
__device__ __forceinline__ u32 call_take(u32 wid) { WAIT_VM(0); return rfl(lds32v(CTL_CALL + wid * 4)) + 1; }
__device__ __forceinline__ void call_bump(const Params& P, u32 wid, u32 lane, u32 call) { if (lane == 0) __hip_atomic_store(call_word(P, wid), call, __ATOMIC_RELAXED, __HIP_MEMORY_SCOPE_AGENT); }

__device__ __forceinline__ u32 solo_gid(u32 c) { return (c >> 2) & 15; }
__device__ __forceinline__ u32 solo_w(u32 c) { return (c & 3) | (c >> 6) << 2; }
__device__ __forceinline__ void load_x(const Params& P, u32 gid, u32 w, u32 lane, u32 (&v)[64]) {
    const u16* s = P.x + (size_t)((gid & 15) * 256 + (w & 1) * 128 + (lane & 15)) * KS + ((gid >> 4) * 4 + (w >> 1)) * 64 + (lane >> 4) * 8;
#pragma unroll
    for (int p = 0; p < 16; ++p) { const u32x4 t = *(const u32x4*)(s + (p >> 1) * 16 * KS + (p & 1) * 32); v[4 * p] = t.x; v[4 * p + 1] = t.y; v[4 * p + 2] = t.z; v[4 * p + 3] = t.w; }
}
__device__ __forceinline__ void puller(const Params& P, u32 role, u32 wid, u32 lane) {
    const u32 q = role & 7, sub = (role >> 3) & 15, PW = (role >> 7) & 15;
    const u32 pc = sub + PW * (7 - wid);
    if (pc < (u32)NRAW) {
        const u32 gid = solo_gid(pc), w = solo_w(pc);
        u32 v[64], cnt;
        load_x(P, gid, w, lane, v);
        const u32 call = call_take(wid);
        for (u32 r = q;; r = P.rank) {
            char* wr = r == q ? P.peer : P.me;
            char* o = wr + image_off(gid, w, P.rank, r, call, lane, cnt);
#pragma unroll
            for (int p = 0; p < 16; ++p) st_sys(o + x_piece_off(w & 1, p), (u32x4){v[4 * p], v[4 * p + 1], v[4 * p + 2], v[4 * p + 3]});
            WAIT_VM(0); if (r == q) STAMP(P, 120 + wid);
            if (r != q) { if (lane == 0) __hip_atomic_fetch_add(counter(P, 0) + cnt, 1u, __ATOMIC_RELAXED, __HIP_MEMORY_SCOPE_AGENT); break; }
            if (lane == 0) __hip_atomic_fetch_add((u32*)(wr + WS_CNT) + gid * 32 + 5, 1u, __ATOMIC_RELAXED, __HIP_MEMORY_SCOPE_SYSTEM);
            if (((q - P.rank - 1) & 7) != pc % 7) break;
        }
    }
    const u32 call = call_take(wid);
    call_bump(P, wid, lane, call);
    if (wid == 0) {
        __builtin_amdgcn_s_setprio(3);
        const char* base = P.peer + WS_STG + (call & 1) * STG_PAR;
        const char* dummy = P.me + WS_TMO + 1024 + lane * 16;
        const u32 voff = lane * 16;
        u32 pos = 0, pub = 0, limit = RING / 1024;
        u64 tk = 0, c_full = 0, c_idle = 0, n_full = 0, n_ref = 0; (void)tk; (void)c_full; (void)c_idle; (void)n_full; (void)n_ref;
        for (u32 ui = 0;; ++ui) {
            u32 nu = rfl(lds32v(CTL_NUNIT));
            if ((nu & ~DONE) <= ui) {
                TICK(tk);
                WAIT_VM(0); lds32vw(CTL_LANDED, pos); pub = pos;
                while (nu <= ui) { __builtin_amdgcn_s_sleep(1); nu = rfl(lds32v(CTL_NUNIT)); }
                if ((nu & ~DONE) <= ui) break;
                ACC(c_idle, tk);
            }
            CBAR();
            const u32x2 un = lds64v(D_UNIT + ui * 8);
            const u32 ut = rfl(un.y), total = ut & 0x3ffffu, lastc = ut >> 18;
            const char* src = base + rfl(un.x);
            for (u32 o = 0; o < total; o += 4096) {
                if (pos + 4 > limit) {
                    bool drained = false; TICK(tk); ++n_ref;
                    for (;;) {
                        const u32 nx = lds32v(CTL_NEXT + (lane & 7) * 4);
                        u32 f = lastc;
#pragma unroll
                        for (u32 j = 0; j < ND; ++j) f = umin(f, (u32)__builtin_amdgcn_readlane((int)nx, j));
                        limit = (rfl(lds32v(D_DESC + f * 8)) >> 14) + RING / 1024;
                        if (pos + 4 <= limit) break;
                        if (!drained) { WAIT_VM(0); lds32vw(CTL_LANDED, pos); pub = pos; drained = true; }
                        __builtin_amdgcn_s_sleep(1);
                    }
                    ACC(c_full, tk); n_full += drained;
                }
                const u32 dst = (pos * 1024) & RMASK;
                if (o + 4096 <= total) {
                    u32 keep;
                    asm volatile("s_mov_b32 %0, m0\n\ts_mov_b32 m0, %3\n\ts_nop 4\n\t"
                                 "global_load_lds_dwordx4 %1, %2 sc0 sc1\n\ts_waitcnt vmcnt(%4)\n\t"
                                 "global_load_lds_dwordx4 %1, %2 offset:1024 sc0 sc1\n\ts_waitcnt vmcnt(%4)\n\t"
                                 "global_load_lds_dwordx4 %1, %2 offset:2048 sc0 sc1\n\ts_waitcnt vmcnt(%4)\n\t"
                                 "global_load_lds_dwordx4 %1, %2 offset:3072 sc0 sc1\n\ts_waitcnt vmcnt(%4)\n\t"
                                 "s_mov_b32 m0, %0" : "=&s"(keep) : "v"(voff), "s"(src + o), "s"(dst), "i"(WIN) : "memory");
                } else {
#pragma unroll
                    for (u32 j = 0; j < 4; ++j) { const u32 b = o + j * 1024 + voff; glds_sys(b < total ? src + b : dummy, dst + j * 1024); WAIT_VM(WIN); }
                }
                pos += 4;
                if (pos > pub + WIN) { pub = pos - WIN; lds32vw(CTL_LANDED, pub); }
            }
            STAMP(P, 128 + umin(ui, 127)); DUMP(P, 64 + umin(ui, 127), total);
        }
        DUMP(P, 0, c_full); DUMP(P, 1, n_full); DUMP(P, 2, n_ref); DUMP(P, 3, c_idle);
    } else if (wid == 1) {
        u32 basek = 0, nch = 0, nun = 0;
        u32* ticket = (u32*)(P.me + WS_CLAIM + q * 128);
        for (u32 ui = 0;; ++ui) {
            while (rfl(lds32v(CTL_LANDED)) + LEAD < basek) __builtin_amdgcn_s_sleep(4);
            u32 idx = ui * PW + sub;
            if (ui >= (u32)NSTAT) {
                if (lane == 0) idx = __hip_atomic_fetch_add(ticket, 1u, __ATOMIC_RELAXED, __HIP_MEMORY_SCOPE_AGENT);
                idx = rfl(idx) - (call - 1) * (u32)(P.nunits - NSTAT * PW + PW) + NSTAT * PW;
                if (idx >= (u32)P.nunits) break;
            }
            const u32 e = tab16(TAB_STREAM, idx), gid = e & 1023u, part = e >> 10, lv = 31 - __builtin_clz(part), n = 8u >> lv, first = (part - (1u << lv)) * n;
            const bool solo = gid < (u32)SOLOG, mine = lane >= first && lane < first + n;
            const u32* f = (const u32*)(P.me + WS_FLAG + (call & 1) * FLAG_PAR + (size_t)q * FLAG_SRC + gid * 32) + (lane & 7);
            const u64 t0 = now(); u32 val; bool dead = false;
            for (;;) {
                val = __hip_atomic_load(f, __ATOMIC_RELAXED, __HIP_MEMORY_SCOPE_SYSTEM);
                if (__all((val >> 8) == (call & 0xffffffu) || lane >= first + n || (solo && lane < first))) break;
                if (__any(expired(P, t0, call))) { dead = true; break; }
                __builtin_amdgcn_s_sleep(1);
            }
            if (dead) {
                const u32 t = gid - GY;
                if (lane == 0 && t < (u32)(16 * DP)) *y_base(P, t & 15, q * DP + (t >> 4), first, 0u) = 0x7fc0;
                break;
            }
            u32 code = val & 255u; if (code > 127u) code = RAWCODE;
            const u32 cb = chunk_bytes(code), sz = solo ? (mine ? (cb + 4095u) & ~4095u : 0u) : (lane < first + n ? cb : 0u);
            u32 inc = sz; inc += dpp<0x111, 0xf, true>(inc); inc += dpp<0x112, 0xf, true>(inc); inc += dpp<0x114, 0xf, true>(inc);
            const u32 a0 = (u32)__builtin_amdgcn_readlane((int)(inc - sz), (int)first) & ~127u, total = (u32)__builtin_amdgcn_readlane((int)inc, 7) - a0;
            if (mine) lds64vw(D_DESC + (nch + lane - first) * 8, (u32x2){((basek * 1024 + inc - sz - a0) << 4) | code, gid * 8 + lane});
            if (solo) { if (mine) lds64vw(D_UNIT + (nun + lane - first) * 8, (u32x2){gid * (u32)GRP_STRIDE + lane * RAWB, cb | ((nch + lane - first) << 18)}); nun += n; }
            else { if (lane == 0) lds64vw(D_UNIT + nun * 8, (u32x2){gid * (u32)GRP_STRIDE + a0, total | ((nch + n - 1) << 18)}); nun += 1; }
            nch += n;
            WAIT_LGKM0();
            lds32vw(CTL_NUNIT, nun); lds32vw(CTL_NCHUNK, nch);
            basek += ((total + 4095) >> 12) << 2;
            STAMP(P, umin(ui, 127));
        }
        lds32vw(CTL_NUNIT, nun | DONE); lds32vw(CTL_NCHUNK, nch | DONE);
    } else {
        const u32 dw = wid - 2, scr = D_SCR + dw * D_SCRB;
        u64 tk = 0, c_wait = 0, c_dec = 0, c_st = 0; (void)c_wait; (void)c_dec; (void)c_st; TICK(tk);
        for (u32 c = dw;; c += ND) {
            u32 nc = rfl(lds32v(CTL_NCHUNK));
            while (nc <= c) { __builtin_amdgcn_s_sleep(2); nc = rfl(lds32v(CTL_NCHUNK)); }
            if ((nc & ~DONE) <= c) break;
            CBAR();
            const u32x2 de = lds64v(D_DESC + c * 8);
            const u32 d = rfl(de.x), code = d & 255u, S = (d >> 8) << 4, gid = rfl(de.y) >> 3, w = rfl(de.y) & 7;
            const u32 need = (S + chunk_bytes(code) + 1023) >> 10;
            while (rfl(lds32v(CTL_LANDED)) < need) __builtin_amdgcn_s_sleep(1);
            CBAR();
            u32 Mr[32], Er[32];
            CS_DECL
            ACC(c_wait, tk);
            unpack_chunk(S, code, scr, lane, CTL_NEXT + dw * 4, c + ND, Mr, Er CS_PASS);
            ACC(c_dec, tk);
            if (gid >= (u32)GY) {
                const u32 t = gid - GY;
                u16* o = y_base_lin(P, t & 15, q * DP + (t >> 4), w, lane);
#pragma unroll
                for (int p = 0; p < 16; ++p) st_out<GAR_OST>(o + y_piece_off(p), piece(Mr, Er, p));
            } else store_image(P, gid, w, q, call, lane, Mr, Er);
            ACC(c_st, tk);
        }
        DUMP(P, wid * 4, c_wait); DUMP(P, wid * 4 + 1, c_dec); DUMP(P, wid * 4 + 2, c_st); DUMP(P, wid * 4 + 3, now());
    }
}

__device__ __forceinline__ void pack_inputs(const Params& P, u32 gid, u32 w, bool solo, u32 wid, u32 lane) {
    asm volatile("" : "+v"(lane));
    u32 v[64], Mr[32], Er[32];
    u32 peers = 255u & ~(1u << P.rank); bool own = true;
    if (gid < (u32)GW) load_x(P, gid, w, lane, v);
    else {
        const u32 j = (gid - GW) >> 1, ktl = (gid & 1) * 4 + (w >> 1);
        if (j < (u32)(NR * DP)) { const u32 dest = j / DP; own = dest == (u32)P.rank; peers = own ? 0u : 1u << dest; }
        const u16* s = P.w + (size_t)(ktl * 64 + (lane >> 4)) * NN + j * 256 + (w & 1) * 128 + (lane & 15) * 8;
        const u32 scr = wid * 16384;
#pragma unroll
        for (int i = 0; i < 16; ++i) glds_plain(s + (size_t)i * 4 * NN, scr + i * 1024);
        WAIT_VM(0);
        const u32 ta = scr + ((lane >> 4) * 8 + ((lane & 15) >> 2)) * 256 + (lane & 3) * 16;
#pragma unroll
        for (int p = 0; p < 16; ++p) {
            const u32 a = ta + (p & 1) * 32 * 256 + (p >> 2) * 64 + ((p >> 1) & 1) * 8;
            const u32x2 lo = __builtin_bit_cast(u32x2, __builtin_amdgcn_ds_read_tr16_b64_v4i16((LAS s16x4*)a));
            const u32x2 hi = __builtin_bit_cast(u32x2, __builtin_amdgcn_ds_read_tr16_b64_v4i16((LAS s16x4*)(a + 4 * 256)));
            v[4 * p] = lo.x; v[4 * p + 1] = lo.y; v[4 * p + 2] = hi.x; v[4 * p + 3] = hi.y;
        }
        WAIT_LGKM0();
    }
    const u32 call = call_take(wid);
    if (peers) pack_group(P, v, Mr, Er, gid, w, solo, peers, call, wid, lane); else split(v, Mr, Er);
    if (own) store_image(P, gid, w, P.rank, call, lane, Mr, Er);
}

typedef __bf16 bf16x2 __attribute__((ext_vector_type(2)));
typedef float f32x2 __attribute__((ext_vector_type(2)));
__device__ __forceinline__ u32 cvt_pk(float lo, float hi) { return __builtin_bit_cast(u32, __builtin_convertvector((f32x2){lo, hi}, bf16x2)); }
__device__ __forceinline__ void wait_panels(const Params& P, u32 tile, u32 upto, u32 call, u32 wid, u32 lane) {
    if (wid == 0) {
        const int left = NRAW - 4 * (int)(tile & 15); const u32 pushed = (lane & 8) ? 0u : 7u * (u32)(left < 0 ? 0 : left > 4 ? 4 : left);
        const u32 st = (lane & 7) == 5 && upto ? 5u : umin(lane & 7, upto), need = call * (st == 0 ? 16u : st == 1 ? 28u - pushed : st == 5 ? pushed : 28u);
        const u32* c = counter(P, (lane & 8) ? 16 + (tile >> 4) : (tile & 15)) + st;
        const u64 t0 = now();
        for (;;) {
            const u32 x = __hip_atomic_load(c, __ATOMIC_RELAXED, __HIP_MEMORY_SCOPE_SYSTEM);
            if (__all((int)(x - need) >= 0)) break;
            if (__any(expired(P, t0, call))) { asm volatile("ds_write_b32 %0, %1\n\ts_waitcnt lgkmcnt(0)" :: "v"(CTL_BAD), "v"(1u) : "memory"); break; }
            __builtin_amdgcn_s_sleep(4);
        }
        if (upto == 1) __builtin_amdgcn_fence(__ATOMIC_ACQUIRE, ""); else __builtin_amdgcn_fence(__ATOMIC_ACQUIRE, "agent");
        WAIT_VM(0);
    }
    BAR();
}
__device__ __forceinline__ void gemm_role(const Params& P, unsigned char* lds_generic, u32 place, u32 call, u32 wid, u32 lane) {
    LAS unsigned char* lds = (LAS unsigned char*)lds_generic;
    const u32 tid = threadIdx.x;
    const u32 wr = wid >> 2, wc = wid & 3;
    constexpr int HTB = 16384;
    const u32 voff = tid * 16, ldsw = wid * 1024;
    const u32 fo = lane * 16;
    const u32 aoff = wr * 8192 + fo, boff = wc * 4096 + fo;
#define G_SA(b, h) (((b) * 2 + (h)) * HTB)
#define G_SB(b, h) ((4 + (b) * 2 + (h)) * HTB)
#define G_STAGE(bufoff, gbase) do { _Pragma("unroll") for (int _i = 0; _i < 2; ++_i) \
        __builtin_amdgcn_global_load_lds((const u32*)((const char*)(gbase) + voff + _i * 8192), (LAS u32*)(lds + (bufoff) + ldsw + _i * 8192), 16, 0, 0); } while (0)
#define G_LDA(dst, b, h) do { _Pragma("unroll") for (int m = 0; m < 4; ++m) _Pragma("unroll") for (int k = 0; k < 2; ++k) dst[m][k] = *(const LAS bf16x8*)(lds + G_SA(b, h) + aoff + m * 2048 + k * 1024); } while (0)
#define G_LDB(dst, b, h) do { _Pragma("unroll") for (int n = 0; n < 2; ++n) _Pragma("unroll") for (int k = 0; k < 2; ++k) dst[n][k] = *(const LAS bf16x8*)(lds + G_SB(b, h) + boff + n * 2048 + k * 1024); } while (0)
#define G_MMA(ai, bj, At, Bt) do { __builtin_amdgcn_s_setprio(1); _Pragma("unroll") for (int m = 0; m < 4; ++m) _Pragma("unroll") for (int n = 0; n < 2; ++n) _Pragma("unroll") for (int k = 0; k < 2; ++k) \
        acc[ai][bj][m][n] = __builtin_amdgcn_mfma_f32_16x16x32_bf16(Bt[n][k], At[m][k], acc[ai][bj][m][n], 0, 0, 0); __builtin_amdgcn_s_setprio(0); } while (0)
#define G_WV(n) asm volatile("s_waitcnt vmcnt(" #n ")" ::: "memory")
#define G_WL(n) asm volatile("s_waitcnt lgkmcnt(" #n ")" ::: "memory")
#define G_BAR __builtin_amdgcn_s_barrier()
#define G_SCHED __builtin_amdgcn_sched_barrier(0)
    const char* XI = P.me + WS_XI + (call & 1) * 16 * PANEL; const char* WI = P.me + WS_WI;
    constexpr size_t kstep = 32768, hstep = 16384;
    const u32x2 cl = lds64v(TAB_COLS + ((blockIdx.x & 4) * 2 + (place >> 2)) * 8);
    const u64 mycols = (u64)rfl(cl.y) << 32 | rfl(cl.x); const u32 pm = (blockIdx.x & 3) * 4 + (place & 3);
    u32 cur = pm | ((u32)mycols & 255u) << 4;
    bool fresh = true;
    bf16x8 At[4][2], B0[2][2], B1[2][2];
    for (u32 ui = 0;; ++ui) {
        const u32 nxt = pm | ((u32)(mycols >> (8 * ui + 8)) & 255u) << 4;
        const bool has_next = (nxt >> 4) != 255u, dist = (cur >> 4) < (u32)DP, chain = has_next && !dist;
        const char* cA = XI + (size_t)(cur & 15) * PANEL; const char* cB = WI + (size_t)(cur >> 4) * PANEL;
        if (fresh) {
            wait_panels(P, cur, ui ? 4u : 0u, call, wid, lane);
            STAMP(P, 8 + ui * 4); if (ui == 0) DUMP(P, 1, __builtin_amdgcn_s_memtime());
            G_STAGE(G_SB(0, 0), cB); G_STAGE(G_SB(0, 1), cB + hstep); G_STAGE(G_SA(0, 0), cA); G_STAGE(G_SA(0, 1), cA + hstep);
            if (wr == 1) G_BAR;
            G_WV(2); G_BAR;
            G_STAGE(G_SB(1, 0), cB + kstep); G_STAGE(G_SA(1, 0), cA + kstep); G_STAGE(G_SB(1, 1), cB + hstep + kstep);
            G_WV(6); G_BAR;
        }
        f32x4 acc[2][2][4][2];
#pragma unroll
        for (int a = 0; a < 2; ++a)
#pragma unroll
            for (int b = 0; b < 2; ++b)
#pragma unroll
                for (int m = 0; m < 4; ++m)
#pragma unroll
                    for (int n = 0; n < 2; ++n) acc[a][b][m][n] = (f32x4){0.f, 0.f, 0.f, 0.f};
        const char* nA = chain ? XI + (size_t)(nxt & 15) * PANEL : cA; const char* nB = chain ? WI + (size_t)(nxt >> 4) * PANEL : cB;
        for (int t = 0; t < 64; t += 2) {
            const bool last = (t == 62);
            const char* a1 = cA + (size_t)(t + 1) * kstep;
            const char* a2 = last ? nA : cA + (size_t)(t + 2) * kstep; const char* b2 = last ? nB : cB + (size_t)(t + 2) * kstep;
            const char* a3 = a2 + kstep; const char* b3 = b2 + kstep;
            {
                const bool staged = ui == 0 && (t == 6 || t == 20 || t == 34 || t == 48);
                if (staged || (last && chain)) wait_panels(P, staged ? cur : nxt, t == 6 ? 1u : t == 20 ? 2u : t == 34 ? 3u : 4u, call, wid, lane);
                if (staged) STAMP(P, t == 6 ? 4 : t == 20 ? 5 : t == 34 ? 6 : 7);
            }
            G_LDB(B0, 0, 0); G_LDB(B1, 0, 1); G_SCHED; G_LDA(At, 0, 0); G_STAGE(G_SA(1, 1), a1 + hstep);
            G_WV(8); G_WL(0); G_BAR; G_MMA(0, 0, At, B0); G_MMA(0, 1, At, B1); G_BAR; G_SCHED;
            G_LDA(At, 0, 1); G_STAGE(G_SB(0, 0), b2); G_STAGE(G_SB(0, 1), b2 + hstep); G_STAGE(G_SA(0, 0), a2);
            G_WV(8); G_WL(0); G_BAR; G_MMA(1, 0, At, B0); G_MMA(1, 1, At, B1); G_BAR; G_SCHED;
            G_LDB(B0, 1, 0); G_LDB(B1, 1, 1); G_SCHED; G_LDA(At, 1, 0); G_STAGE(G_SA(0, 1), a2 + hstep);
            G_WV(8); G_WL(0); G_BAR; G_MMA(0, 0, At, B0); G_MMA(0, 1, At, B1); G_BAR; G_SCHED;
            G_LDA(At, 1, 1); G_STAGE(G_SB(1, 0), b3); G_STAGE(G_SB(1, 1), b3 + hstep); G_STAGE(G_SA(1, 0), a3);
            G_WV(8); G_WL(0); G_BAR; G_MMA(1, 0, At, B0); G_MMA(1, 1, At, B1); G_BAR; G_SCHED;
        }
        STAMP(P, 9 + ui * 4); DUMP(P, 2 + ui, __builtin_amdgcn_s_memtime());
        if (!chain) G_WV(0);
        if (GAR_ALIGN && wr == 0) G_BAR;
        u32 v[64];
        {
            int tz = threadIdx.x; asm volatile("" : "+v"(tz));
            const u32 ln = tz & 63;
            u32 bad; asm volatile("ds_read_b32 %0, %1\n\ts_waitcnt lgkmcnt(0)" : "=v"(bad) : "v"(CTL_BAD) : "memory");
            u16* o = y_base(P, cur & 15, global_panel(cur >> 4, P.rank), wid, ln);
#pragma unroll
            for (int p = 0; p < 16; p += 2) {
                const int ai = p >> 3, m = (p >> 1) & 3;
#pragma unroll
                for (int i = 0; i < 4; ++i) {
                    const f32x4 xa = acc[ai][0][m][i >> 1], xb = acc[ai][1][m][i >> 1];
                    u32 a = cvt_pk(xa[(i & 1) * 2], xa[(i & 1) * 2 + 1]), b = cvt_pk(xb[(i & 1) * 2], xb[(i & 1) * 2 + 1]);
                    if (bad) a = b = 0x7fc07fc0u;
                    v[4 * p + i] = (u32)__builtin_amdgcn_update_dpp((int)a, (int)b, 0x128, 0xf, 0xc, false);
                    v[4 * p + 4 + i] = (u32)__builtin_amdgcn_update_dpp((int)b, (int)a, 0x128, 0xf, 0x3, false);
                }
                st_out<GAR_GST>(o + y_piece_off(p), (u32x4){v[4 * p], v[4 * p + 1], v[4 * p + 2], v[4 * p + 3]});
                st_out<GAR_GST>(o + y_piece_off(p + 1), (u32x4){v[4 * p + 4], v[4 * p + 5], v[4 * p + 6], v[4 * p + 7]});
            }
            if (chain) { if (GAR_ALIGN && wr == 1) G_BAR; }
            else {
                if (!GAR_ALIGN && wr == 0) G_BAR;
                BAR();
                if (dist) {
                    const u32 scr = wid * 16384, to = scr + ((ln & 7) * 8 + (ln & 8) / 2 + (ln >> 4)) * 16;
#pragma unroll
                    for (int p = 0; p < 16; ++p) lds128w(to + p * 1024, (u32x4){v[4 * p], v[4 * p + 1], v[4 * p + 2], v[4 * p + 3]});
                    CBAR();
#pragma unroll
                    for (int p = 0; p < 16; ++p) { const u32x4 t = lds128(scr + ln * 16 + p * 1024); v[4 * p] = t.x; v[4 * p + 1] = t.y; v[4 * p + 2] = t.z; v[4 * p + 3] = t.w; }
                    WAIT_LGKM0();
                    u32 Mr[32], Er[32]; pack_group(P, v, Mr, Er, GY + cur, wid, false, 255u & ~(1u << P.rank), call, wid, ln); }
                STAMP(P, 10 + ui * 4);
                if (!has_next) break;
            }
        }
        fresh = !chain; cur = nxt;
    }
}

__global__ void __launch_bounds__(512, 2) fused(const Args A) {
    extern __shared__ __attribute__((aligned(16))) unsigned char lds[];
    const u32 wid = rfl(threadIdx.x >> 6), lane = threadIdx.x & 63;
    const u32 role = pin((A.role[blockIdx.x >> 1] >> ((blockIdx.x & 1) * 16)) & 0xffffu);
    char* peer = pin(A.ws[0]);
#pragma unroll
    for (u32 j = 1; j < NR; ++j) { char* c = pin(A.ws[j]); peer = (role & 7) == j ? c : peer; }
    const Params P = {pin(A.me), peer, pin(A.x), pin(A.w), pin(A.out), pin((u32)A.rank), pin((u32)A.nunits)};
    call_load(P, wid, lane);
    if (wid == 1 || wid == 2) {
        const char* ka = (const char*)(const GAS char*)(u64)__builtin_amdgcn_kernarg_segment_ptr() + lane * 16;
        if (wid == 1 && !(role & 0x8000u)) { if (lane < 4) glds_plain(ka, LDS_TAB); }
        else if ((wid == 1) == ((role & 0x8000u) != 0)) { glds_plain(ka, LDS_TAB); glds_plain(ka + 1024, LDS_TAB + 1024); glds_plain(ka + 2048, LDS_TAB + 2048); }
    }
    if (role & 0x8000u) {
        if (threadIdx.x == 0) { lds32vw(CTL_NUNIT, 0u); lds32vw(CTL_LANDED, 0u); lds32vw(CTL_NCHUNK, 0u); }
        if (wid == 1 && lane < 8) lds32vw(CTL_NEXT + lane * 4, lane);
        WAIT_LGKM0(); BAR();
        puller(P, role, wid, lane); return;
    }
    STAMP(P, 0); DUMP(P, 0, __builtin_amdgcn_s_memtime());
    if (threadIdx.x == 0) lds32vw(CTL_BAD, 0u);
    pack_init(wid, lane);
    const u32 sc = NRAW + (role & 255u);
    if (sc >= (u32)(8 * SOLOG)) { const u64 t0 = now(); while (now() - t0 < GAR_HOLD) __builtin_amdgcn_s_sleep(2); }
    for (u32 it = 0;; ++it) {
        const u32 rb = NENC - 1 - (role & 255u), e = it > 1 ? NENC + rb + (it - 2) * NLIGHT : rb;
        if (it > 1 && (e >= (u32)NPACK || rb >= (u32)NLIGHT)) break;
        if (it || (wid == 1 && sc < (u32)(8 * SOLOG))) pack_inputs(P, it ? tab16(TAB_ENC, e) : solo_gid(sc), it ? wid : solo_w(sc), !it, wid, lane);
        WAIT_VM(0); WAIT_LGKM0(); BAR(); STAMP(P, it ? umin(it, 2) : 3);
    }
    const u32 call = call_take(wid);
    call_bump(P, wid, lane, call);
    if (role >> 8) gemm_role(P, lds, (role >> 8) - 1, call, wid, lane);
}

#define GAR_T3 2
#define GAR_T2 6
#define GAR_T1 14
#define GAR_SPARE 1
#define GAR_XAHEAD 8
#define GAR_QTAIL 6
constexpr int QTAIL = GAR_QTAIL;
static void build_tables(Args& P) {
    unsigned char* cols = (unsigned char*)P.cols; u16* stream = (u16*)P.stream; u16* enc = (u16*)P.enc; u16* role = (u16*)P.role;
    {
        int peer[2][4], np[2] = {0, 0}, cnt[8] = {}, seen[2] = {0, 0}, n = 0;
        for (int d = 1; d < NR; ++d) { const int q = (P.rank + d) & 7, h = ((0xb2 >> q) ^ (0xb2 >> P.rank)) & 1; peer[h][np[h]++] = q; }
        for (int b = 0; b < NWG; ++b) {
            const int h = (b >> 2) & 1, i = b >> 3, g0 = 32 - 4 * (h ? CH : CL), near = h ? 4 * NH : 3 * NL, far = h ? 3 * FL : 4 * FH, k = seen[h]++;
            if (k >= near + far) { role[b] = (u16)(n++ | (i >= g0 ? i - g0 + 1 : 0) << 8); continue; }
            const int side = k < near ? h : !h, j = k < near ? k : k - near, q = peer[side][side ? (j + j / 4) & 3 : j % 3];
            role[b] = (u16)(0x8000 | q | cnt[q]++ << 3);
        }
        for (int b = 0; b < NWG; ++b) if (role[b] & 0x8000) role[b] |= (u16)(cnt[role[b] & 7] << 7);
    }
    int col[LP], nc = 0;
    {
        const int rest = LP - (ROUNDS - 1) * CPR, r0 = (rest + 1) / 2; const bool spare = GAR_SPARE && DP <= 2 * (CPR - rest);
        int nd = 0, nr = DP;
        for (int i = 0; i < 128; ++i) cols[i] = 0xff;
        for (int r = 0; r < ROUNDS; ++r) for (int c = 0; c < 8; ++c) for (int cg = 0; cg < 2; ++cg) {
            const bool heavy = c < (cg ? rest - r0 : r0);
            if (c < (cg ? CH : CL) && (r + 1 < ROUNDS || heavy)) cols[(cg * 8 + c) * 8 + r] = (unsigned char)(col[nc++] = nd < DP && !(spare && heavy) ? nd++ : nr++);
        }
    }
    auto wgid = [&](int jj, int dest, int h) { return GW + 2 * (jj >= DP ? jj + (NR - 1) * DP : dest * DP + jj) + h; };
    int k = 0;
    auto put = [&](int gid, int lv) { for (int j = 0; j < (1 << lv); ++j) stream[k++] = (u16)(gid | ((1 << lv) + j) << 10); };
    auto stage = [&](int st, int i0, int i1) {
        const int h = st >> 1, half = st & 1;
        for (int i = i0; i < i1; ++i) {
            const int gid = i < 16 ? GX + 16 * h + i : wgid(col[i - 16], P.rank, h);
            if ((i < 16 && h == 0) || i >= 16 + CPR - QTAIL) { for (int j = 0; j < 2; ++j) if (st || 4 * i + 2 * j >= NRAW) stream[k++] = (u16)(gid | (4 + 2 * half + j) << 10); }
            else stream[k++] = (u16)(gid | (2 + half) << 10);
        }
    };
    stage(0, 0, 16); stage(1, 0, GAR_XAHEAD); stage(0, 16, 16 + CPR); stage(1, GAR_XAHEAD, 16 + CPR); stage(2, 0, 16 + CPR); stage(3, 0, 16 + CPR);
    for (int i = CPR; i < LP; ++i) for (int h = 0; h < 2; ++h) put(wgid(col[i], P.rank, h), i < 2 * CPR ? 1 : 0);
    for (int t = 0; t < 16 * DP; ++t) { const int left = 16 * DP - t; put(GY + t, left <= GAR_T3 ? 3 : left <= GAR_T2 ? 2 : left <= GAR_T1 ? 1 : 0); }
    P.nunits = k;
    k = 0;
    for (int i = 0; i < CPR; ++i) for (int d = 1; d < (col[i] < DP ? NR : 2); ++d) enc[k++] = (u16)wgid(col[i], (P.rank + d) & 7, 0);
    for (int i = SOLOG; i < 32; ++i) enc[k++] = (u16)(GX + i);
    for (int i = 0; i < CPR; ++i) { if (col[i] < DP) enc[k++] = (u16)wgid(col[i], P.rank, 0); enc[k++] = (u16)wgid(col[i], P.rank, 1); }
    for (int i = 0; i < CPR; ++i) if (col[i] < DP) for (int d = 1; d < NR; ++d) enc[k++] = (u16)wgid(col[i], (P.rank + d) & 7, 1);
    for (int i = CPR; i < LP; ++i) for (int h = 0; h < 2; ++h) for (int d = 0; d < (col[i] < DP ? NR : 1); ++d) enc[k++] = (u16)wgid(col[i], (P.rank + d) & 7, h);
}
}

extern "C" void kernel_launch(int rank, int nranks, ncclComm_t comm, hipStream_t stream,
                              void* const* d_in, const int* in_sizes, int n_in,
                              void* d_out, int out_size, const int* in_dtypes, int out_dtype,
                              void* const* d_out_all, void* const* d_ws_all, size_t ws_bytes) {
    using namespace gar;
    if (nranks != NR || (unsigned)rank >= (unsigned)NR || n_in != 2 || ws_bytes < WS_END || in_sizes[0] != MM * KS || in_sizes[1] != KS * NN || out_size != MM * NN) {
        fprintf(stderr, "kernel_launch: unexpected shapes\n"); return; }
    Args P = {};
    for (int p = 0; p < NR; ++p) P.ws[p] = (char*)d_ws_all[p];
    P.me = P.ws[rank];
    P.x = (const u16*)d_in[0]; P.w = (const u16*)d_in[1]; P.out = (u16*)d_out; P.rank = rank;
    build_tables(P);
    HIP_CHECK(hipFuncSetAttribute((const void*)fused, hipFuncAttributeMaxDynamicSharedMemorySize, LDS_BYTES));
    hipLaunchKernelGGL(fused, dim3(NWG), dim3(512), LDS_BYTES, stream, P);
    HIP_CHECK(hipGetLastError());
}
```

```cpp
#include <hip/hip_runtime.h>
#include <rccl/rccl.h>
#include <cstdio>
#include <cstdint>
#include <cstddef>
#pragma clang diagnostic ignored "-Wint-to-pointer-cast"

#define HIP_CHECK(expr) do { hipError_t _e = (expr); if (_e != hipSuccess) { \
    fprintf(stderr, "HIP error %s at %s:%d\n", hipGetErrorString(_e), __FILE__, __LINE__); \
    return; } } while (0)

namespace gar {
#define LAS __attribute__((address_space(3)))
#define GAS __attribute__((address_space(1)))
typedef unsigned u32;
typedef unsigned short u16;
typedef unsigned long long u64;
typedef unsigned u32x4 __attribute__((ext_vector_type(4)));
typedef unsigned u32x2 __attribute__((ext_vector_type(2)));
typedef short s16x4 __attribute__((ext_vector_type(4)));
typedef short bf16x8 __attribute__((ext_vector_type(8)));
typedef float f32x4 __attribute__((ext_vector_type(4)));

constexpr int NR = 8, MM = 4096, NN = 32768, KS = 512;
#define GAR_DP 11
constexpr int DP = GAR_DP;
constexpr int RP = 128 - NR * DP;
constexpr int LP = DP + RP;
constexpr int NWG = 256;
#define GAR_NL 9
#define GAR_FL 0
#define GAR_NH 8
#define GAR_FH 1
#define GAR_CL 6
#define GAR_WIN 6
constexpr int NL = GAR_NL, FL = GAR_FL, NH = GAR_NH, FH = GAR_FH, CL = GAR_CL, CH = 6, CPR = CL + CH;
constexpr int NPULL = 3 * (NL + FL) + 4 * (NH + FH);
constexpr int WIN = GAR_WIN;
static_assert(3 * NL + 4 * FH + 16 * CL <= 128 && 4 * NH + 3 * FL + 16 * CH <= 128, "roles");
#define GAR_SP2 1
#define GAR_TAILRAW 0
#define GAR_ALIGN 1
#define GAR_NSAMP 6
#define GAR_NRAW 44
#define GAR_SOLOG 16
#define GAR_LEAD 60
#define GAR_HOLD 100
constexpr int SOLOG = GAR_SOLOG, NRAW = GAR_NRAW;
constexpr int LEAD = GAR_LEAD;
static_assert(NRAW <= 52 && NRAW % 4 == 0 && 8 * SOLOG <= NWG - NPULL && SOLOG == 16, "solo chunks");
constexpr int NENC = NWG - NPULL;
constexpr int GX = 0, GW = 32, GY = 288;
constexpr int NGID = GY + 16 * DP;
constexpr int NS = 32 + 2 * DP + 2 * RP + 16 * DP;
constexpr int ROUNDS = (LP + CPR - 1) / CPR;
constexpr int NUMAX = 544;
#define GAR_NLIGHT 80
constexpr int NLIGHT = GAR_NLIGHT;
constexpr int NSTAT = 3;
constexpr int NPACK = 288 - SOLOG;
constexpr u64 DEADLINE = 30000000ull;

constexpr size_t WS_CALL = 0;
constexpr size_t WS_TMO = 32768;
constexpr size_t WS_CNT = 36864;
constexpr size_t WS_CLAIM = 57344;
constexpr size_t WS_FLAG = 65536;
constexpr size_t FLAG_SRC = 544 * 32, FLAG_PAR = 8 * FLAG_SRC;
constexpr size_t WS_DBG = 524288;
#define GAR_SKEW 0
constexpr size_t GRP_STRIDE = 131072 + GAR_SKEW;
constexpr size_t WS_STG = 2u << 20;
constexpr size_t STG_PAR = (544 * GRP_STRIDE + 0x1fffff) & ~(size_t)0x1fffff;
constexpr size_t WS_INB = WS_STG + 2 * STG_PAR;
constexpr size_t WS_XI = WS_INB + 2 * 8 * 64 * 8192;
constexpr size_t PANEL = 64 * 32768;
constexpr size_t WS_WI = WS_XI + 16 * PANEL;
constexpr size_t WS_END = WS_WI + (size_t)LP * PANEL;
static_assert(WS_FLAG + 2 * FLAG_PAR <= WS_DBG && NGID <= 512, "layout");

constexpr u32 LDS_CTL = 131072;
constexpr u32 LDS_LUT = LDS_CTL + 512, LDS_HIST = LDS_CTL + 1024;
constexpr u32 LDS_TAB = LDS_HIST + 4096, D_UNIT = LDS_TAB + 3072, LDS_BYTES = D_UNIT + (NUMAX + 8 * SOLOG) * 8;
constexpr u32 CTL_CALL = LDS_CTL  , CTL_SIZE = LDS_CTL + 64  , CTL_BAD = LDS_CTL + 128;
constexpr u32 CTL_NUNIT = LDS_CTL + 192, CTL_LANDED = LDS_CTL + 256, CTL_NCHUNK = LDS_CTL + 320, CTL_NEXT = LDS_CTL + 384  ;
constexpr u32 E_L2 = 0, E_L3 = E_L2 + 4160, E_L4 = E_L3 + 2112, E_VAR = E_L4 + 1088, E_END = E_VAR + 2112;
static_assert(E_END <= 10240 && E_L4 % 16 == 0 && E_L3 % 16 == 0 && E_VAR % 16 == 0, "packer scratch");
constexpr u32 CAP2 = 4080, CAP3 = 2032, CAP4 = 1008, CAPV = 2032, FIXB = 10240, RAWB = 16384, RAWCODE = 255;
constexpr u32 RING = 65536, RMASK = RING - 1, ND = 6;
constexpr u32 D_SCR = RING, D_SCRB = 7360, D_L2 = 0, D_L3 = 4160, D_L4 = D_L3 + 2112;
constexpr u32 D_DESC = D_SCR + ND * D_SCRB, DONE = 0x80000000u;
static_assert(D_L4 + 1088 <= D_SCRB && D_DESC + NS * 64 <= LDS_CTL && LDS_BYTES <= 163840 && D_DESC % 16 == 0, "puller LDS");

struct Args {
    char* ws[NR];
    char* me; const u16* x; const u16* w; u16* out;
    int rank, nunits;
    u32 role[NWG / 2];
    u32 stream[NUMAX / 2];
    u32 enc[144];
    u32 cols[64];
    u32 pad[(3072 - 104) / 4 - NWG / 2 - NUMAX / 2 - 144 - 64];
};
static_assert(sizeof(Args) == 3072 && ROUNDS < 8, "kernarg");
constexpr u32 TAB_STREAM = LDS_TAB + offsetof(Args, stream), TAB_ENC = LDS_TAB + offsetof(Args, enc), TAB_COLS = LDS_TAB + offsetof(Args, cols);
struct Params { char* me; char* peer; const u16* x; const u16* w; u16* out; u32 rank, nunits; };
template <class T> __device__ __forceinline__ T* pin(T* p) { u64 v = (u64)p; asm volatile("" : "+s"(v)); return (T*)(GAS T*)v; }
__device__ __forceinline__ u32 pin(u32 v) { asm volatile("" : "+s"(v)); return v; }

__device__ __forceinline__ u32 lds32(u32 a) { return *(LAS u32*)a; }
__device__ __forceinline__ u32 lds32v(u32 a) { return *(volatile LAS u32*)a; }
__device__ __forceinline__ void lds32vw(u32 a, u32 x) { *(volatile LAS u32*)a = x; }
__device__ __forceinline__ u32x2 lds64v(u32 a) { return *(volatile LAS u32x2*)a; }
__device__ __forceinline__ void lds64vw(u32 a, u32x2 x) { *(volatile LAS u32x2*)a = x; }
__device__ __forceinline__ u32x4 lds128(u32 a) { return *(LAS u32x4*)a; }
__device__ __forceinline__ void lds128w(u32 a, u32x4 x) { *(LAS u32x4*)a = x; }
__device__ __forceinline__ void lds_or(u32 a, u32 x) { __hip_atomic_fetch_or((LAS u32*)a, x, __ATOMIC_RELAXED, __HIP_MEMORY_SCOPE_WORKGROUP); }
__device__ __forceinline__ void lds_add(u32 a, u32 x) { __hip_atomic_fetch_add((LAS u32*)a, x, __ATOMIC_RELAXED, __HIP_MEMORY_SCOPE_WORKGROUP); }
__device__ __forceinline__ u32 rfl(u32 x) { return (u32)__builtin_amdgcn_readfirstlane((int)x); }
__device__ __forceinline__ u32 tab16(u32 t, u32 i) { return rfl(*(volatile LAS u16*)(t + i * 2)); }
#define CBAR() asm volatile("" ::: "memory")
#define WAIT_LGKM0() asm volatile("s_waitcnt lgkmcnt(0)" ::: "memory")
#define WAIT_VM(n) asm volatile("s_waitcnt vmcnt(%0)" :: "i"(n) : "memory")
#define BAR() do { CBAR(); __builtin_amdgcn_s_barrier(); CBAR(); } while (0)
__device__ __forceinline__ void st_sys(void* p, u32x4 x) { asm volatile("global_store_dwordx4 %0, %1, off sc0 sc1\n\ts_nop 1" :: "v"(p), "v"(x) : "memory"); }
__device__ __forceinline__ void st_dev(void* p, u32x4 x) { asm volatile("global_store_dwordx4 %0, %1, off sc1\n\ts_nop 1" :: "v"(p), "v"(x) : "memory"); }
#define GAR_OST 3
#define GAR_GST 1
template <int MODE> __device__ __forceinline__ void st_out(void* p, u32x4 x) {
    if (MODE == 0) *(u32x4*)p = x;
    else if (MODE == 1) __builtin_nontemporal_store(x, (u32x4*)p);
    else if (MODE == 2) st_dev(p, x);
    else if (MODE == 9) asm volatile("" :: "v"(p), "v"(x) : "memory");
    else asm volatile("global_store_dwordx4 %0, %1, off sc1 nt\n\ts_nop 1" :: "v"(p), "v"(x) : "memory");
}
__device__ __forceinline__ void glds_sys(const void* g, u32 dst) {
    u32 keep;
    asm volatile("s_mov_b32 %0, m0\n\ts_mov_b32 m0, %2\n\ts_nop 0\n\tglobal_load_lds_dwordx4 %1, off sc0 sc1\n\ts_mov_b32 m0, %0" : "=&s"(keep) : "v"(g), "s"(dst) : "memory");
}
__device__ __forceinline__ void glds_plain(const void* g, u32 dst) {
    u32 keep;
    asm volatile("s_mov_b32 %0, m0\n\ts_mov_b32 m0, %2\n\ts_nop 0\n\tglobal_load_lds_dwordx4 %1, off\n\ts_mov_b32 m0, %0" : "=&s"(keep) : "v"(g), "s"(dst) : "memory");
}
__device__ __forceinline__ u64 now() { return __builtin_amdgcn_s_memrealtime(); }
template <int CTRL, int RM, bool BC> __device__ __forceinline__ u32 dpp(u32 v) { return (u32)__builtin_amdgcn_update_dpp(0, (int)v, CTRL, RM, 0xf, BC); }
__device__ __forceinline__ u32 scan_incl(u32 v) {
    v += dpp<0x111, 0xf, true>(v); v += dpp<0x112, 0xf, true>(v); v += dpp<0x114, 0xf, true>(v); v += dpp<0x118, 0xf, true>(v);
    v += dpp<0x142, 0xa, false>(v); v += dpp<0x143, 0xc, false>(v); return v;
}
__device__ __forceinline__ u32 umax(u32 a, u32 b) { return a > b ? a : b; }
__device__ __forceinline__ u32 umin(u32 a, u32 b) { return a < b ? a : b; }
__device__ __forceinline__ u32 wave_max(u32 v) {
    v = umax(v, dpp<0x111, 0xf, true>(v)); v = umax(v, dpp<0x112, 0xf, true>(v)); v = umax(v, dpp<0x114, 0xf, true>(v)); v = umax(v, dpp<0x118, 0xf, true>(v));
    v = umax(v, dpp<0x142, 0xa, false>(v)); v = umax(v, dpp<0x143, 0xc, false>(v)); return (u32)__builtin_amdgcn_readlane((int)v, 63);
}
__device__ __forceinline__ u32 popc(u32 x) { return (u32)__builtin_popcount(x); }
__device__ __forceinline__ u32 perm(u32 hi, u32 lo, u32 sel) { return __builtin_amdgcn_perm(hi, lo, sel); }

#define STAMP(P, slot) do {} while (0)
#define TICK(t) do {} while (0)
#define ACC(sum, t) do {} while (0)
#define DUMP(P, slot, val) do {} while (0)

#define CSTAMP(i) do {} while (0)
#define CS_ARG
#define CS_DECL
#define CS_PASS

__device__ __forceinline__ u32 nzb(u32 z) { return ((z & 0x7f7f7f7fu) + 0x7f7f7f7fu) | z; }
typedef unsigned short u16x2 __attribute__((ext_vector_type(2)));
struct Classifier {
    u32 s1, s2, s3, nb, tlo, thi; bool win;
    __device__ __forceinline__ void init(u32 t) {
        const u32 a = (t >> 8) & 255u, b = (t >> 16) & 255u, c = t >> 24, lo = umin(a, umin(b, c));
        win = umax(a, umax(b, c)) - lo <= 6u;
        const u64 l = (1ull << (((a - lo) & 7u) * 8)) | (2ull << (((b - lo) & 7u) * 8)) | (3ull << (((c - lo) & 7u) * 8));
        tlo = (u32)l; thi = (u32)(l >> 32); nb = ((0u - lo) & 255u) * 0x01000100u;
        s1 = a * 0x01010101u; s2 = b * 0x01010101u; s3 = c * 0x01010101u;
    }
    __device__ __forceinline__ u32 any(u32 e) const {
        const u32 a1 = nzb(e ^ s1), a2 = nzb(e ^ s2), a3 = nzb(e ^ s3);
        return ((~(a1 & a3) >> 7) & 0x01010101u) | ((~(a2 & a3) >> 6) & 0x02020202u);
    }
    __device__ __forceinline__ u32 near(u32 e) const {
        const u16x2 x = __builtin_bit_cast(u16x2, e), n = __builtin_bit_cast(u16x2, nb), top = {0x07ff, 0x07ff};
        u32 sh; asm("v_pk_mad_u16 %0, %1, %2, %3" : "=v"(sh) : "v"(e), "v"(0x01000100u), "s"(nb));
        const u16x2 hi = __builtin_elementwise_min((u16x2)(x + n), top), lo = __builtin_elementwise_min(__builtin_bit_cast(u16x2, sh), top);
        return perm(thi, tlo, perm(__builtin_bit_cast(u32, hi), __builtin_bit_cast(u32, lo), 0x07030501u));
    }
    template <int N> __device__ __forceinline__ void run(const u32* e, u32* c) const {
        if (win) {
#pragma unroll
            for (int i = 0; i < N; ++i) c[i] = near(e[i]);
        } else {
#pragma unroll
            for (int i = 0; i < N; ++i) c[i] = any(e[i]);
        }
    }
};
__device__ __forceinline__ u32 escbits(u32 c) { return ~(c | (c >> 1)) & 0x55555555u; }
struct Appender {
    u32 lo, fill, addr;
    __device__ __forceinline__ void init(u32 off) { lo = 0; fill = (off & 3u) * 8u; addr = off & ~3u; }
    __device__ __forceinline__ void push(u32 k, u32 mk) {
        const u64 t = (u64)k << fill;
        lo |= (u32)t; fill += popc(mk);
        if (fill >= 32u) { lds_or(addr, lo); lo = (u32)(t >> 32); addr += 4; }
        fill &= 31u;
    }
    __device__ __forceinline__ void flush() { if (lo) lds_or(addr, lo); }
};
__device__ __forceinline__ u32 escmask(u32 c) { return perm(0u, 0xffu, c); }
__device__ __forceinline__ u32 selector(u32 mk) { return lds32(__builtin_amdgcn_udot4(mk & 0x20100804u, 0x01010101u, LDS_LUT, false)); }
template <int R> __device__ __forceinline__ u32 pack_level(u32 src, u32 n, u32 t, u32 cdst, u32 edst, u32 cap, u32 lane) {
    Classifier K; K.init(t);
    if (lane < 16) *(LAS unsigned char*)(src + n + lane) = (unsigned char)K.s1;
    u32 ne = 0;
    for (u32 r = 0; r * 1024 < n; r += R) {
        CBAR();
        u32 q[4 * R], c[4 * R], m[4 * R], l[4 * R], cnt = 0;
#pragma unroll
        for (int i = 0; i < R; ++i) { const u32 p0 = (r + i) * 1024 + lane * 16; const u32x4 x = lds128(src + p0); const bool act = p0 < n;
            q[4 * i] = act ? x.x : K.s1; q[4 * i + 1] = act ? x.y : K.s1; q[4 * i + 2] = act ? x.z : K.s1; q[4 * i + 3] = act ? x.w : K.s1; }
        K.run<4 * R>(q, c);
#pragma unroll
        for (int i = 0; i < 4 * R; ++i) { m[i] = escmask(c[i]); l[i] = selector(m[i]); }
#pragma unroll
        for (int i = 0; i < R; ++i) cnt |= ((popc(m[4 * i]) + popc(m[4 * i + 1]) + popc(m[4 * i + 2]) + popc(m[4 * i + 3])) >> 3) << (16 * i);
        const u32 inc = scan_incl(cnt), tot = (u32)__builtin_amdgcn_readlane((int)inc, 63), exc = inc - cnt;
        if (ne + (tot & 0xffffu) + (tot >> 16) > cap) return ~0u;
#pragma unroll
        for (int i = 0; i < R; ++i) {
            Appender A; A.init(edst + ne + (i ? (tot & 0xffffu) + (exc >> 16) : (exc & 0xffffu)));
#pragma unroll
            for (int j = 0; j < 4; ++j) A.push(perm(0u, q[4 * i + j], l[4 * i + j]), m[4 * i + j]);
            A.flush();
            if ((r + i) * 1024 + lane * 16 < n) *(LAS u32*)(cdst + ((r + i) * 64 + lane) * 4) = c[4 * i] | (c[4 * i + 1] << 2) | (c[4 * i + 2] << 4) | (c[4 * i + 3] << 6);
        }
        ne += (tot & 0xffffu) + (tot >> 16);
    }
    CBAR();
    return ne;
}
template <int N> __device__ __forceinline__ u32 rank_ror(u32 key) { return (dpp<0x120 + N, 0xf, true>(key) > key ? 1u : 0u) + rank_ror<N - 1>(key); }
template <> __device__ __forceinline__ u32 rank_ror<0>(u32) { return 0u; }
__device__ __forceinline__ u32 row_or(u32 x) { x |= dpp<0x128, 0xf, true>(x); x |= dpp<0x124, 0xf, true>(x); x |= dpp<0x122, 0xf, true>(x); x |= dpp<0x121, 0xf, true>(x); return rfl(x); }
__device__ __forceinline__ void split(const u32 (&v)[64], u32 (&Mr)[32], u32 (&Er)[32]) {
#pragma unroll
    for (int g = 0; g < 32; ++g) {
        const u32 a = __builtin_amdgcn_alignbit(v[2 * g], v[2 * g], 31), b = __builtin_amdgcn_alignbit(v[2 * g + 1], v[2 * g + 1], 31);
        Er[g] = perm(b, a, 0x07050301u); Mr[g] = perm(b, a, 0x06040200u);
        asm volatile("" : "+v"(Er[g]), "+v"(Mr[g]));
    }
}
__device__ __forceinline__ u32 pack_chunk(const u32 (&v)[64], u32 (&Mr)[32], u32 (&Er)[32], u32 (&Cr)[8], u32 scr, u32 lane, bool solo CS_ARG) {
    CBAR(); CSTAMP(0);
    const u32 hist = solo ? scr + 10240 : LDS_HIST;
    if (solo) {
#pragma unroll
        for (int j = 0; j < 4; ++j) lds128w(hist + j * 1024 + lane * 16, (u32x4){0u, 0u, 0u, 0u});
        CBAR();
#pragma unroll
        for (int i = 0; i < 12; ++i) lds_add(hist + (((v[i * 5 + 2] >> ((i & 1) * 16 + 3)) & 0xff0u) | ((lane & 3) * 4)), 1u);
    }
#pragma unroll
    for (int i = 0; i < GAR_NSAMP; ++i) lds_add(hist + (((v[i * (60 / GAR_NSAMP) + 5] >> ((i & 1) * 16 + 3)) & 0xff0u) | ((lane & 3) * 4)), 1u);
#pragma unroll
    for (int j = 0; j < 10; ++j) lds128w(scr + j * 1024 + lane * 16, (u32x4){0u, 0u, 0u, 0u});
    WAIT_LGKM0();
    if (!solo) BAR();
    CBAR();
    u32 key = 0;
#pragma unroll
    for (int i = 0; i < 4; ++i) { const u32 bin = i * 64 + lane; const u32x4 h = lds128(hist + bin * 16); key = umax(key, ((h.x + h.y + h.z + h.w) << 8) | bin); }
    split(v, Mr, Er);
    CSTAMP(1);
    { const auto r = __builtin_amdgcn_permlane32_swap(key, key, false, false); key = umax(r[0], r[1]); }
    { const auto r = __builtin_amdgcn_permlane16_swap(key, key, false, false); key = umax(r[0], r[1]); }
    const u32 rank = rank_ror<15>(key), lev = (rank * 11u) >> 5, sym = (key & 255u) << (8 * (rank - 3 * lev + 1));
    const u32 T[4] = {row_or(lev == 0 ? sym : 0u), row_or(lev == 1 ? sym : 0u), row_or(lev == 2 ? sym : 0u), row_or(lev == 3 ? sym : 0u)};
    CSTAMP(3);
    u32 cnt = 0, mk[32], sl[32];
    {
        Classifier K; K.init(T[0]);
        u32 c[32]; K.run<32>(Er, c);
#pragma unroll
        for (int g = 0; g < 32; ++g) { mk[g] = escmask(c[g]); sl[g] = selector(mk[g]); cnt += popc(mk[g]); }
#pragma unroll
        for (int k = 0; k < 8; ++k) Cr[k] = c[4 * k] | (c[4 * k + 1] << 2) | (c[4 * k + 2] << 4) | (c[4 * k + 3] << 6);
    }
    cnt >>= 3;
    const u32 inc = scan_incl(cnt), n2 = (u32)__builtin_amdgcn_readlane((int)inc, 63);
    CSTAMP(4);
    if (n2 > CAP2) return RAWCODE;
    Appender A; A.init(scr + E_L2 + inc - cnt);
#pragma unroll
    for (int g = 0; g < 32; ++g) A.push(perm(0u, Er[g], sl[g]), mk[g]);
    A.flush();
    CBAR(); CSTAMP(5);
    const u32 d2 = (n2 + 15) >> 4;
    const u32 n3 = pack_level<2>(scr + E_L2, n2, T[1], scr + E_VAR + 16, scr + E_L3, CAP3, lane);
    CSTAMP(6);
    if (n3 == ~0u) return RAWCODE;
    const u32 d3 = (n3 + 15) >> 4;
    const u32 n4 = pack_level<1>(scr + E_L3, n3, T[2], scr + E_VAR + 16 + 4 * d2, scr + E_L4, CAP4, lane);
    CSTAMP(7);
    if (n4 == ~0u) return RAWCODE;
    const u32 d4 = (n4 + 15) >> 4, b5 = 16 + 4 * (d2 + d3 + d4);
    const u32 n5 = pack_level<1>(scr + E_L4, n4, T[3], scr + E_VAR + 16 + 4 * (d2 + d3), scr + E_VAR + b5, CAPV - b5, lane);
    if (n5 == ~0u) return RAWCODE;
    if (lane == 0) lds128w(scr + E_VAR, (u32x4){T[0] | d2, T[1] | d3, T[2] | d4, T[3]});
    CBAR();
    return (b5 + n5 + 15) >> 4;
}
__device__ __forceinline__ void pack_init(u32 wid, u32 lane) {
    if (wid >= 4) lds128w(LDS_HIST + (wid - 4) * 1024 + lane * 16, (u32x4){0u, 0u, 0u, 0u});
    if (wid == 1 && lane < 16) { u32 sel = 0x0c0c0c0cu, k = 0;
#pragma unroll
        for (u32 b = 0; b < 4; ++b) if ((lane >> b) & 1) { sel = (sel & ~(255u << k)) | (b << k); k += 8; }
        *(LAS u32*)(LDS_LUT + lane * 4) = sel; }
}
__device__ __forceinline__ u32 chunk_bytes(u32 code) { return code == RAWCODE ? RAWB : FIXB + 16 * code; }

__device__ __forceinline__ void pack_store(char* dst, u32 code, const u32 (&Mr)[32], const u32 (&Er)[32], const u32 (&Cr)[8], u32 scr, u32 lane) {
    char* dm = dst + (chunk_bytes(code) - 8192);
#pragma unroll
    for (int j = 0; j < 8; ++j) st_sys(dm + j * 1024, (u32x4){Mr[4 * j], Mr[4 * j + 1], Mr[4 * j + 2], Mr[4 * j + 3]});
    if (code == RAWCODE) {
#pragma unroll
        for (int j = 0; j < 8; ++j) st_sys(dst + j * 1024, (u32x4){Er[4 * j], Er[4 * j + 1], Er[4 * j + 2], Er[4 * j + 3]});
    } else {
        st_sys(dst, (u32x4){Cr[0], Cr[1], Cr[2], Cr[3]}); st_sys(dst + 1024, (u32x4){Cr[4], Cr[5], Cr[6], Cr[7]});
        if (lane < code) st_sys(dst + 2048, lds128(scr + E_VAR + lane * 16));
        if (lane + 64 < code) st_sys(dst + 3072, lds128(scr + E_VAR + 1024 + lane * 16));
    }
}
__device__ __forceinline__ void raise_flag(const Params& P, u32 q, u32 gid, u32 w, u32 call, u32 code) {
    char* wq = (char*)(GAS char*)__builtin_bit_cast(u64, lds64v(LDS_TAB + q * 8));
    u32* f = (u32*)(wq + WS_FLAG + (call & 1) * FLAG_PAR + (size_t)P.rank * FLAG_SRC + gid * 32) + w;
    __hip_atomic_store(f, (call << 8) | code, __ATOMIC_RELAXED, __HIP_MEMORY_SCOPE_SYSTEM);
}
__device__ __forceinline__ void pack_group(const Params& P, const u32 (&v)[64], u32 (&Mr)[32], u32 (&Er)[32], u32 gid, u32 w, bool solo, u32 peers, u32 call, u32 wid, u32 lane) {
    u32 Cr[8];
    const u32 scr = wid * 16384;
    CS_DECL
    if (gid < 64u) STAMP(P, 40 + wid);
    u32 code = rfl(pack_chunk(v, Mr, Er, Cr, scr, lane, solo CS_PASS));
    if (gid == (u32)(NGID - 1) && w >= (u32)(8 - GAR_TAILRAW)) code = RAWCODE;
    if (gid < 64u) STAMP(P, 48 + wid);
    u32 off = w * RAWB, mine = code;
    if (!solo) {
        if (lane == 0) lds32vw(CTL_SIZE + wid * 4, code);
        WAIT_LGKM0(); BAR();
        mine = lds32v(CTL_SIZE + (lane & 7) * 4);
        if (lane < 32) lds128w(LDS_HIST + wid * 512 + lane * 16, (u32x4){0u, 0u, 0u, 0u});
        off = 0;
#pragma unroll
        for (u32 j = 0; j < 7; ++j) { const u32 c = (u32)__builtin_amdgcn_readlane((int)mine, j); if (j < wid) off += chunk_bytes(c); }
    }
    pack_store(P.me + WS_STG + (call & 1) * STG_PAR + (size_t)gid * GRP_STRIDE + off + lane * 16, code, Mr, Er, Cr, scr, lane);
    if (gid < 64u) STAMP(P, 56 + wid);
    WAIT_VM(0);
    if (gid < 64u) STAMP(P, 64 + wid);
    WAIT_LGKM0();
    if (!solo) BAR();
    if ((solo ? (lane & 7) == w : wid == 0) && ((peers >> (lane >> 3)) & 1)) raise_flag(P, lane >> 3, gid, lane & 7, call, mine);
}

#define GAR_WINMODE 1
template <bool WRAP> struct Unp {
    u32 sel[4], lo[4], hi[4], of[4];
    __device__ __forceinline__ void fetch(u32 c, u32& off) {
#pragma unroll
        for (int j = 0; j < 4; ++j) {
            const u32 cj = (c >> (2 * j)) & 0x03030303u, mk = escmask(cj), m2 = mk & 0x02020202u;
            if (WRAP || GAR_WINMODE != 2) { const u32 a = off & ~3u; lo[j] = lds32(WRAP ? (a & RMASK) : a); hi[j] = lds32(WRAP ? ((a + 4) & RMASK) : a + 4); of[j] = off; }
            else lo[j] = lds32(off);
            u32 p; asm("v_mad_u32_u24 %0, %1, %2, %3" : "=v"(p) : "v"(m2), "s"(0x808080u), "v"(0x04040404u));
            sel[j] = (mk & p) | cj;
            off += popc(m2);
        }
    }
    __device__ __forceinline__ u32 finish(int j, u32 t) const {
        u32 w;
        if (WRAP || GAR_WINMODE == 0) w = __builtin_amdgcn_alignbyte(hi[j], lo[j], of[j] & 3u);
        else if (GAR_WINMODE == 1) w = __builtin_amdgcn_alignbyte(hi[j], lo[j], of[j]);
        else w = lo[j];
        return perm(w, t, sel[j]);
    }
    __device__ __forceinline__ u32x4 finish4(u32 t) const { return (u32x4){finish(0, t), finish(1, t), finish(2, t), finish(3, t)}; }
};
#define SCHED() __builtin_amdgcn_sched_barrier(0)
template <bool WRAP> __device__ __forceinline__ u32 unpack_round(u32 c, u32 t, u32 src, u32 dst) {
    const u32 cnt = popc(escbits(c)), inc = scan_incl(cnt);
    u32 off = src + inc - cnt;
    Unp<WRAP> u; u.fetch(c, off); SCHED();
    lds128w(dst, u.finish4(t));
    return (u32)__builtin_amdgcn_readlane((int)inc, 63);
}
__device__ __forceinline__ void ring_read8k(u32 a0, u32 (&r)[32]) {
#pragma unroll
    for (int j = 0; j < 8; ++j) { const u32x4 q = lds128((a0 + j * 1024) & RMASK); r[4 * j] = q.x; r[4 * j + 1] = q.y; r[4 * j + 2] = q.z; r[4 * j + 3] = q.w; }
}
__device__ __forceinline__ void unpack_exp(u32 S, u32 code, u32 scr, u32 lane, u32 (&Er)[32] CS_ARG) {
    const u32 a0 = S + lane * 16;
    if (code == RAWCODE) { ring_read8k(a0, Er); return; }
    const u32x4 ca = lds128(a0 & RMASK), cb = lds128((a0 + 1024) & RMASK);
    const u32 V = S + 2048;
    const u32x4 hd = lds128(V & RMASK);
    const u32 t1 = rfl(hd.x), t2 = rfl(hd.y), t3 = rfl(hd.z), t4 = rfl(hd.w), d2 = t1 & 255u, d3 = t2 & 127u, d4 = t3 & 63u;
    u32 c2[4], c3[2];
#pragma unroll
    for (int r = 0; r < 4; ++r) { const u32 i = r * 64 + lane; const u32 x = lds32((V + 16 + 4 * i) & RMASK); c2[r] = i < d2 ? x : 0x55555555u; }
#pragma unroll
    for (int r = 0; r < 2; ++r) { const u32 i = r * 64 + lane; const u32 x = lds32((V + 16 + 4 * (d2 + i)) & RMASK); c3[r] = i < d3 ? x : 0x55555555u; }
    const u32 x4 = lds32((V + 16 + 4 * (d2 + d3 + lane)) & RMASK), c4 = lane < d4 ? x4 : 0x55555555u;
    CSTAMP(1);
    unpack_round<true>(c4, t4, V + 16 + 4 * (d2 + d3 + d4), scr + D_L4 + lane * 16);
    CBAR();
    const u32 s4 = scr + D_L4 + unpack_round<false>(c3[0], t3, scr + D_L4, scr + D_L3 + lane * 16);
    if (d3 > 64) unpack_round<false>(c3[1], t3, s4, scr + D_L3 + 1024 + lane * 16);
    CBAR(); CSTAMP(2);
    const u32 C[8] = {ca.x, ca.y, ca.z, ca.w, cb.x, cb.y, cb.z, cb.w};
    u32 cnt = 0;
#pragma unroll
    for (int k = 0; k < 8; ++k) cnt += popc(escbits(C[k]));
    {
        const u32 n0 = popc(escbits(c2[0])), n1 = popc(escbits(c2[1]));
        const u32 i01 = scan_incl(n0 | (n1 << 16)), tot = (u32)__builtin_amdgcn_readlane((int)i01, 63);
        cnt = scan_incl(cnt) - cnt;
        u32 o0 = scr + D_L3 + (i01 & 0xffffu) - n0, o1 = scr + D_L3 + (tot & 0xffffu) + (i01 >> 16) - n1;
        Unp<false> u0, u1; u0.fetch(c2[0], o0); u1.fetch(c2[1], o1); SCHED();
        lds128w(scr + D_L2 + lane * 16, u0.finish4(t2)); lds128w(scr + D_L2 + 1024 + lane * 16, u1.finish4(t2));
        if (d2 > 128) {
            u32 s3 = scr + D_L3 + (tot & 0xffffu) + (tot >> 16);
            s3 += unpack_round<false>(c2[2], t2, s3, scr + D_L2 + 2048 + lane * 16);
            if (d2 > 192) unpack_round<false>(c2[3], t2, s3, scr + D_L2 + 3072 + lane * 16);
        }
    }
    CBAR(); CSTAMP(3);
    u32 off = scr + D_L2 + cnt;
    Unp<false> u[8];
    u[0].fetch(C[0], off); u[1].fetch(C[1], off);
#pragma unroll
    for (int k = 0; k < 8; ++k) {
        SCHED();
        if (k + 2 < 8) u[k + 2].fetch(C[k + 2], off);
        SCHED();
#pragma unroll
        for (int j = 0; j < 4; ++j) Er[4 * k + j] = u[k].finish(j, t1);
    }
    CBAR();
}
__device__ __forceinline__ u32x4 piece(const u32 (&Mr)[32], const u32 (&Er)[32], int p) {
    const u32 a = perm(Er[2 * p], Mr[2 * p], 0x05010400u), b = perm(Er[2 * p], Mr[2 * p], 0x07030602u);
    const u32 c = perm(Er[2 * p + 1], Mr[2 * p + 1], 0x05010400u), d = perm(Er[2 * p + 1], Mr[2 * p + 1], 0x07030602u);
    return (u32x4){__builtin_amdgcn_alignbit(a, a, 1), __builtin_amdgcn_alignbit(b, b, 1), __builtin_amdgcn_alignbit(c, c, 1), __builtin_amdgcn_alignbit(d, d, 1)};
}

__device__ __forceinline__ u32 x_piece_off(u32 half, int p) { return half * 16384 + p * 1024; }
__device__ __forceinline__ u32 w_piece_off(u32 h, int p) {
    const u32 g32 = p >> 2, n = (p >> 1) & 1, kb = p & 1;
    return (g32 & 1) * 16384 + ((2 * (2 * h + (g32 >> 1)) + n) * 2 + kb) * 1024;
}
__device__ __forceinline__ u16* y_base(const Params& P, u32 pm, u32 j, u32 w, u32 lane) {
    return P.out + (size_t)(pm * 256 + (w >> 2) * 64 + (lane & 7)) * NN + j * 256 + (w & 3) * 64 + (lane & 8) * 4 + (lane >> 4) * 8;
}
__device__ __forceinline__ u16* y_base_lin(const Params& P, u32 pm, u32 j, u32 w, u32 lane) {
    return P.out + (size_t)(pm * 256 + (w >> 2) * 64 + (lane >> 3)) * NN + j * 256 + (w & 3) * 64 + (lane & 7) * 8;
}
__device__ __forceinline__ size_t y_piece_off(int p) { return (size_t)((p >> 3) * 128 + (p & 7) * 8) * NN; }
__device__ __forceinline__ u32 local_panel(u32 j, u32 rank) { return j >= (u32)(NR * DP) ? j - (NR - 1) * DP : j - rank * DP; }
__device__ __forceinline__ u32 global_panel(u32 jj, u32 rank) { return jj >= (u32)DP ? jj + (NR - 1) * DP : jj + rank * DP; }
__device__ __forceinline__ u32* counter(const Params& P, u32 i) { return (u32*)(P.me + WS_CNT + i * 128); }
__device__ __forceinline__ u32* tmo_word(const Params& P) { return (u32*)(P.me + WS_TMO); }
__device__ __forceinline__ bool expired(const Params& P, u64 t0, u32 call) {
    if (__hip_atomic_load(tmo_word(P), __ATOMIC_RELAXED, __HIP_MEMORY_SCOPE_AGENT) == call || now() - t0 > DEADLINE) {
        __hip_atomic_store(tmo_word(P), call, __ATOMIC_RELAXED, __HIP_MEMORY_SCOPE_AGENT); return true; }
    return false;
}

__device__ __forceinline__ void store_image(const Params& P, u32 gid, u32 w, u32 q, u32 lane, const u32 (&Mr)[32], const u32 (&Er)[32]) {
    const bool isx = gid < (u32)GW;
    const u32 pn = isx ? gid & 15 : local_panel((gid - GW) >> 1, P.rank), ktl = (isx ? gid >> 4 : gid & 1) * 4 + (w >> 1);
    const u32 qi = (q - P.rank - 1) & 7, stage = qi == 7 ? 0u : 1 + (ktl >> 1);
    const u32 slot = stage == 0 ? ktl : stage * 14 - 6 + qi * 2 + (ktl & 1);
    char* o = P.me + (isx ? WS_XI : WS_WI) + (size_t)pn * PANEL + (size_t)slot * 32768 + lane * 16;
#pragma unroll
    for (int p = 0; p < 16; ++p) st_dev(o + (isx ? x_piece_off(w & 1, p) : w_piece_off(w & 1, p)), piece(Mr, Er, p));
    WAIT_VM(0);
    if (lane == 0) __hip_atomic_fetch_add(counter(P, isx ? pn : 16 + pn) + stage, 1u, __ATOMIC_RELAXED, __HIP_MEMORY_SCOPE_AGENT);
}
__device__ __forceinline__ u32* call_word(const Params& P, u32 wid) { return (u32*)(P.me + WS_CALL + blockIdx.x * 128 + wid * 4); }
__device__ __forceinline__ void call_load(const Params& P, u32 wid, u32 lane) {
    u32 keep;
    if (lane == 0) asm volatile("s_mov_b32 %0, m0\n\ts_mov_b32 m0, %2\n\ts_nop 0\n\tglobal_load_lds_dword %1, off sc1\n\ts_mov_b32 m0, %0" : "=&s"(keep) : "v"(call_word(P, wid)), "s"(CTL_CALL + wid * 4) : "memory");
}
__device__ __forceinline__ u32 call_take(u32 wid) { WAIT_VM(0); return rfl(lds32v(CTL_CALL + wid * 4)) + 1; }
__device__ __forceinline__ void call_bump(const Params& P, u32 wid, u32 lane, u32 call) { if (lane == 0) __hip_atomic_store(call_word(P, wid), call, __ATOMIC_RELAXED, __HIP_MEMORY_SCOPE_AGENT); }

__device__ __forceinline__ u32 solo_gid(u32 c) { return (c >> 2) & 15; }
__device__ __forceinline__ u32 solo_w(u32 c) { return (c & 3) | (c >> 6) << 2; }
__device__ __forceinline__ void load_x(const Params& P, u32 gid, u32 w, u32 lane, u32 (&v)[64]) {
    const u16* s = P.x + (size_t)((gid & 15) * 256 + (w & 1) * 128 + (lane & 15)) * KS + ((gid >> 4) * 4 + (w >> 1)) * 64 + (lane >> 4) * 8;
#pragma unroll
    for (int p = 0; p < 16; ++p) { const u32x4 t = *(const u32x4*)(s + (p >> 1) * 16 * KS + (p & 1) * 32); v[4 * p] = t.x; v[4 * p + 1] = t.y; v[4 * p + 2] = t.z; v[4 * p + 3] = t.w; }
}
__device__ __forceinline__ void puller(const Params& P, u32 role, u32 wid, u32 lane) {
    const u32 q = role & 7, sub = (role >> 3) & 15, PW = (role >> 7) & 15;
    const u32 pc = sub + PW * (7 - wid);
    if (pc < (u32)NRAW) {
        const u32 gid = solo_gid(pc), w = solo_w(pc);
        u32 v[64], Mr[32], Er[32];
        load_x(P, gid, w, lane, v);
        const u32 call = call_take(wid);
        split(v, Mr, Er);
        char* o = P.peer + WS_INB + (size_t)(((call & 1) * 8 + P.rank) * 64 + pc) * 8192 + lane * 16;
#pragma unroll
        for (int j = 0; j < 8; ++j) st_sys(o + j * 1024, (u32x4){Mr[4 * j], Mr[4 * j + 1], Mr[4 * j + 2], Mr[4 * j + 3]});
        WAIT_VM(0); STAMP(P, 120 + wid);
        if (lane == 0) __hip_atomic_store((u32*)(P.peer + WS_FLAG + (call & 1) * FLAG_PAR + (size_t)P.rank * FLAG_SRC + (512 + gid) * 32) + w, call << 8, __ATOMIC_RELAXED, __HIP_MEMORY_SCOPE_SYSTEM);
    }
    const u32 call = call_take(wid);
    call_bump(P, wid, lane, call);
    if (wid == 0) {
        __builtin_amdgcn_s_setprio(3);
        const char* base = P.peer + WS_STG + (call & 1) * STG_PAR;
        const char* dummy = P.me + WS_TMO + 1024 + lane * 16;
        const u32 voff = lane * 16;
        u32 pos = 0, pub = 0, limit = RING / 1024;
        u64 tk = 0, c_full = 0, c_idle = 0, n_full = 0, n_ref = 0; (void)tk; (void)c_full; (void)c_idle; (void)n_full; (void)n_ref;
        for (u32 ui = 0;; ++ui) {
            u32 nu = rfl(lds32v(CTL_NUNIT));
            if ((nu & ~DONE) <= ui) {
                TICK(tk);
                WAIT_VM(0); lds32vw(CTL_LANDED, pos); pub = pos;
                while (nu <= ui) { __builtin_amdgcn_s_sleep(1); nu = rfl(lds32v(CTL_NUNIT)); }
                if ((nu & ~DONE) <= ui) break;
                ACC(c_idle, tk);
            }
            CBAR();
            const u32x2 un = lds64v(D_UNIT + ui * 8);
            const u32 ut = rfl(un.y), total = ut & 0x3ffffu, lastc = ut >> 18;
            const char* src = base + rfl(un.x);
            for (u32 o = 0; o < total; o += 4096) {
                if (pos + 4 > limit) {
                    bool drained = false; TICK(tk); ++n_ref;
                    for (;;) {
                        const u32 nx = lds32v(CTL_NEXT + (lane & 7) * 4);
                        u32 f = lastc;
#pragma unroll
                        for (u32 j = 0; j < ND; ++j) f = umin(f, (u32)__builtin_amdgcn_readlane((int)nx, j));
                        limit = (rfl(lds32v(D_DESC + f * 8)) >> 14) + RING / 1024;
                        if (pos + 4 <= limit) break;
                        if (!drained) { WAIT_VM(0); lds32vw(CTL_LANDED, pos); pub = pos; drained = true; }
                        __builtin_amdgcn_s_sleep(1);
                    }
                    ACC(c_full, tk); n_full += drained;
                }
                const u32 dst = (pos * 1024) & RMASK;
                if (o + 4096 <= total) {
                    u32 keep;
                    asm volatile("s_mov_b32 %0, m0\n\ts_mov_b32 m0, %3\n\ts_nop 4\n\t"
                                 "global_load_lds_dwordx4 %1, %2 sc0 sc1\n\ts_waitcnt vmcnt(%4)\n\t"
                                 "global_load_lds_dwordx4 %1, %2 offset:1024 sc0 sc1\n\ts_waitcnt vmcnt(%4)\n\t"
                                 "global_load_lds_dwordx4 %1, %2 offset:2048 sc0 sc1\n\ts_waitcnt vmcnt(%4)\n\t"
                                 "global_load_lds_dwordx4 %1, %2 offset:3072 sc0 sc1\n\ts_waitcnt vmcnt(%4)\n\t"
                                 "s_mov_b32 m0, %0" : "=&s"(keep) : "v"(voff), "s"(src + o), "s"(dst), "i"(WIN) : "memory");
                } else {
#pragma unroll
                    for (u32 j = 0; j < 4; ++j) { const u32 b = o + j * 1024 + voff; glds_sys(b < total ? src + b : dummy, dst + j * 1024); WAIT_VM(WIN);
                        if (j < 3 && pos + j + 1 > pub + WIN) { pub = pos + j + 1 - WIN; lds32vw(CTL_LANDED, pub); } }
                }
                pos += 4;
                if (pos > pub + WIN) { pub = pos - WIN; lds32vw(CTL_LANDED, pub); }
            }
            STAMP(P, 128 + umin(ui, 127)); DUMP(P, 64 + umin(ui, 127), total);
        }
        DUMP(P, 0, c_full); DUMP(P, 1, n_full); DUMP(P, 2, n_ref); DUMP(P, 3, c_idle);
    } else if (wid == 1) {
        u32 basek = 0, nch = 0, nun = 0;
        u32* ticket = (u32*)(P.me + WS_CLAIM + q * 128);
        for (u32 ui = 0;; ++ui) {
            while (rfl(lds32v(CTL_LANDED)) + LEAD < basek) __builtin_amdgcn_s_sleep(4);
            u32 idx = ui * PW + sub;
            if (ui >= (u32)NSTAT) {
                if (lane == 0) idx = __hip_atomic_fetch_add(ticket, 1u, __ATOMIC_RELAXED, __HIP_MEMORY_SCOPE_AGENT);
                idx = rfl(idx) - (call - 1) * (u32)(P.nunits - NSTAT * PW + PW) + NSTAT * PW;
                if (idx >= (u32)P.nunits) break;
            }
            const u32 e = tab16(TAB_STREAM, idx), gid = e & 1023u, part = e >> 10, lv = 31 - __builtin_clz(part), n = 8u >> lv, first = (part - (1u << lv)) * n;
            const bool solo = gid < (u32)SOLOG, mine = lane >= first && lane < first + n, sent = gid < (u32)(NRAW / 4) && first < 4;
            const u32* f = (const u32*)(P.me + WS_FLAG + (call & 1) * FLAG_PAR + (size_t)q * FLAG_SRC + (gid + (sent && (lane & 8) ? 512 : 0)) * 32) + (lane & 7);
            const u64 t0 = now(); u32 val; bool dead = false;
            for (;;) {
                val = __hip_atomic_load(f, __ATOMIC_RELAXED, __HIP_MEMORY_SCOPE_SYSTEM);
                if (__all((val >> 8) == (call & 0xffffffu) || (lane & 7) >= first + n || (solo && (lane & 7) < first))) break;
                if (__any(expired(P, t0, call))) { dead = true; break; }
                __builtin_amdgcn_s_sleep(1);
            }
            if (dead) {
                const u32 t = gid - GY;
                if (lane == 0 && t < (u32)(16 * DP)) *y_base(P, t & 15, q * DP + (t >> 4), first, 0u) = 0x7fc0;
                break;
            }
            u32 code = val & 255u; if (code > 127u) code = RAWCODE;
            const u32 cb = chunk_bytes(code) - (sent ? 8192u : 0u), sz = solo ? (mine ? (cb + 4095u) & ~4095u : 0u) : (lane < first + n ? cb : 0u);
            u32 inc = sz; inc += dpp<0x111, 0xf, true>(inc); inc += dpp<0x112, 0xf, true>(inc); inc += dpp<0x114, 0xf, true>(inc);
            const u32 a0 = (u32)__builtin_amdgcn_readlane((int)(inc - sz), (int)first) & ~127u, total = (u32)__builtin_amdgcn_readlane((int)inc, 7) - a0;
            if (mine) lds64vw(D_DESC + (nch + lane - first) * 8, (u32x2){((basek * 1024 + inc - sz - a0) << 4) | code, gid * 8 + lane});
            if (solo) { if (mine) lds64vw(D_UNIT + (nun + lane - first) * 8, (u32x2){gid * (u32)GRP_STRIDE + lane * RAWB, cb | ((nch + lane - first) << 18)}); nun += n; }
            else { if (lane == 0) lds64vw(D_UNIT + nun * 8, (u32x2){gid * (u32)GRP_STRIDE + a0, total | ((nch + n - 1) << 18)}); nun += 1; }
            nch += n;
            WAIT_LGKM0();
            lds32vw(CTL_NUNIT, nun); lds32vw(CTL_NCHUNK, nch);
            basek += ((total + 4095) >> 12) << 2;
            STAMP(P, umin(ui, 127));
        }
        lds32vw(CTL_NUNIT, nun | DONE); lds32vw(CTL_NCHUNK, nch | DONE);
    } else {
        const u32 dw = wid - 2, scr = D_SCR + dw * D_SCRB;
        u64 tk = 0, c_wait = 0, c_dec = 0, c_st = 0; (void)c_wait; (void)c_dec; (void)c_st; TICK(tk);
        for (u32 c = dw;; c += ND) {
            u32 nc = rfl(lds32v(CTL_NCHUNK));
            while (nc <= c) { __builtin_amdgcn_s_sleep(2); nc = rfl(lds32v(CTL_NCHUNK)); }
            if ((nc & ~DONE) <= c) break;
            CBAR();
            const u32x2 de = lds64v(D_DESC + c * 8);
            const u32 d = rfl(de.x), code = d & 255u, S = (d >> 8) << 4, gid = rfl(de.y) >> 3, w = rfl(de.y) & 7;
            const u32 SM = S + chunk_bytes(code) - 8192;
            while (rfl(lds32v(CTL_LANDED)) < (SM + 1023) >> 10) __builtin_amdgcn_s_sleep(1);
            CBAR();
            u32 Mr[32], Er[32];
            CS_DECL
            ACC(c_wait, tk);
            unpack_exp(S, code, scr, lane, Er CS_PASS);
            ACC(c_dec, tk);
            if (gid < (u32)(NRAW / 4) && w < 4) {
                const char* m = P.me + WS_INB + (size_t)(((call & 1) * 8 + q) * 64 + gid * 4 + w) * 8192 + lane * 16;
                u32x4 r[8];
                asm volatile("global_load_dwordx4 %0, %8, off sc0 sc1\n\tglobal_load_dwordx4 %1, %8, off offset:1024 sc0 sc1\n\t"
                             "global_load_dwordx4 %2, %8, off offset:2048 sc0 sc1\n\tglobal_load_dwordx4 %3, %8, off offset:3072 sc0 sc1\n\t"
                             "global_load_dwordx4 %4, %9, off sc0 sc1\n\tglobal_load_dwordx4 %5, %9, off offset:1024 sc0 sc1\n\t"
                             "global_load_dwordx4 %6, %9, off offset:2048 sc0 sc1\n\tglobal_load_dwordx4 %7, %9, off offset:3072 sc0 sc1\n\ts_waitcnt vmcnt(0)"
                             : "=&v"(r[0]), "=&v"(r[1]), "=&v"(r[2]), "=&v"(r[3]), "=&v"(r[4]), "=&v"(r[5]), "=&v"(r[6]), "=&v"(r[7]) : "v"(m), "v"(m + 4096) : "memory");
#pragma unroll
                for (int j = 0; j < 8; ++j) { Mr[4 * j] = r[j].x; Mr[4 * j + 1] = r[j].y; Mr[4 * j + 2] = r[j].z; Mr[4 * j + 3] = r[j].w; }
            } else {
                while (rfl(lds32v(CTL_LANDED)) < (SM + 9215) >> 10) __builtin_amdgcn_s_sleep(1);
                CBAR();
                ring_read8k(SM + lane * 16, Mr);
            }
            WAIT_LGKM0(); lds32vw(CTL_NEXT + dw * 4, c + ND); CBAR();
            ACC(c_wait, tk);
            if (gid >= (u32)GY) {
                const u32 t = gid - GY;
                u16* o = y_base_lin(P, t & 15, q * DP + (t >> 4), w, lane);
#pragma unroll
                for (int p = 0; p < 16; ++p) st_out<GAR_OST>(o + y_piece_off(p), piece(Mr, Er, p));
            } else store_image(P, gid, w, q, lane, Mr, Er);
            ACC(c_st, tk);
        }
        DUMP(P, wid * 4, c_wait); DUMP(P, wid * 4 + 1, c_dec); DUMP(P, wid * 4 + 2, c_st); DUMP(P, wid * 4 + 3, now());
    }
}

__device__ __forceinline__ void pack_inputs(const Params& P, u32 gid, u32 w, bool solo, u32 wid, u32 lane) {
    asm volatile("" : "+v"(lane));
    u32 v[64], Mr[32], Er[32];
    u32 peers = 255u & ~(1u << P.rank); bool own = true;
    if (gid < (u32)GW) load_x(P, gid, w, lane, v);
    else {
        const u32 j = (gid - GW) >> 1, ktl = (gid & 1) * 4 + (w >> 1);
        if (j < (u32)(NR * DP)) { const u32 dest = j / DP; own = dest == (u32)P.rank; peers = own ? 0u : 1u << dest; }
        const u16* s = P.w + (size_t)(ktl * 64 + (lane >> 4)) * NN + j * 256 + (w & 1) * 128 + (lane & 15) * 8;
        const u32 scr = wid * 16384;
#pragma unroll
        for (int i = 0; i < 16; ++i) glds_plain(s + (size_t)i * 4 * NN, scr + i * 1024);
        WAIT_VM(0);
        const u32 ta = scr + ((lane >> 4) * 8 + ((lane & 15) >> 2)) * 256 + (lane & 3) * 16;
#pragma unroll
        for (int p = 0; p < 16; ++p) {
            const u32 a = ta + (p & 1) * 32 * 256 + (p >> 2) * 64 + ((p >> 1) & 1) * 8;
            const u32x2 lo = __builtin_bit_cast(u32x2, __builtin_amdgcn_ds_read_tr16_b64_v4i16((LAS s16x4*)a));
            const u32x2 hi = __builtin_bit_cast(u32x2, __builtin_amdgcn_ds_read_tr16_b64_v4i16((LAS s16x4*)(a + 4 * 256)));
            v[4 * p] = lo.x; v[4 * p + 1] = lo.y; v[4 * p + 2] = hi.x; v[4 * p + 3] = hi.y;
        }
        WAIT_LGKM0();
    }
    const u32 call = call_take(wid);
    if (peers) pack_group(P, v, Mr, Er, gid, w, solo, peers, call, wid, lane); else split(v, Mr, Er);
    if (own) store_image(P, gid, w, P.rank, lane, Mr, Er);
}

typedef __bf16 bf16x2 __attribute__((ext_vector_type(2)));
typedef float f32x2 __attribute__((ext_vector_type(2)));
__device__ __forceinline__ u32 cvt_pk(float lo, float hi) { return __builtin_bit_cast(u32, __builtin_convertvector((f32x2){lo, hi}, bf16x2)); }
__device__ __forceinline__ void wait_panels(const Params& P, u32 tile, u32 upto, u32 call, u32 wid, u32 lane) {
    if (wid == 0) {
        const u32 st = umin(lane & 7, upto), need = call * (st == 0 ? 16u : 28u);
        const u32* c = counter(P, (lane & 8) ? 16 + (tile >> 4) : (tile & 15)) + st;
        const u64 t0 = now();
        for (;;) {
            const u32 x = __hip_atomic_load(c, __ATOMIC_RELAXED, __HIP_MEMORY_SCOPE_AGENT);
            if (__all((int)(x - need) >= 0)) break;
            if (__any(expired(P, t0, call))) { asm volatile("ds_write_b32 %0, %1\n\ts_waitcnt lgkmcnt(0)" :: "v"(CTL_BAD), "v"(1u) : "memory"); break; }
            __builtin_amdgcn_s_sleep(4);
        }
        __builtin_amdgcn_fence(__ATOMIC_ACQUIRE, "agent");
        WAIT_VM(0);
    }
    BAR();
}
__device__ __forceinline__ void gemm_role(const Params& P, unsigned char* lds_generic, u32 place, u32 call, u32 wid, u32 lane) {
    LAS unsigned char* lds = (LAS unsigned char*)lds_generic;
    const u32 tid = threadIdx.x;
    const u32 wr = wid >> 2, wc = wid & 3;
    constexpr int HTB = 16384;
    const u32 voff = tid * 16, ldsw = wid * 1024;
    const u32 fo = lane * 16;
    const u32 aoff = wr * 8192 + fo, boff = wc * 4096 + fo;
#define G_SA(b, h) (((b) * 2 + (h)) * HTB)
#define G_SB(b, h) ((4 + (b) * 2 + (h)) * HTB)
#define G_STAGE(bufoff, gbase) do { _Pragma("unroll") for (int _i = 0; _i < 2; ++_i) \
        __builtin_amdgcn_global_load_lds((const u32*)((const char*)(gbase) + voff + _i * 8192), (LAS u32*)(lds + (bufoff) + ldsw + _i * 8192), 16, 0, 0); } while (0)
#define G_LDA(dst, b, h) do { _Pragma("unroll") for (int m = 0; m < 4; ++m) _Pragma("unroll") for (int k = 0; k < 2; ++k) dst[m][k] = *(const LAS bf16x8*)(lds + G_SA(b, h) + aoff + m * 2048 + k * 1024); } while (0)
#define G_LDB(dst, b, h) do { _Pragma("unroll") for (int n = 0; n < 2; ++n) _Pragma("unroll") for (int k = 0; k < 2; ++k) dst[n][k] = *(const LAS bf16x8*)(lds + G_SB(b, h) + boff + n * 2048 + k * 1024); } while (0)
#define G_MMA(ai, bj, At, Bt) do { __builtin_amdgcn_s_setprio(1); _Pragma("unroll") for (int m = 0; m < 4; ++m) _Pragma("unroll") for (int n = 0; n < 2; ++n) _Pragma("unroll") for (int k = 0; k < 2; ++k) \
        acc[ai][bj][m][n] = __builtin_amdgcn_mfma_f32_16x16x32_bf16(Bt[n][k], At[m][k], acc[ai][bj][m][n], 0, 0, 0); __builtin_amdgcn_s_setprio(0); } while (0)
#define G_WV(n) asm volatile("s_waitcnt vmcnt(" #n ")" ::: "memory")
#define G_WL(n) asm volatile("s_waitcnt lgkmcnt(" #n ")" ::: "memory")
#define G_BAR __builtin_amdgcn_s_barrier()
#define G_SCHED __builtin_amdgcn_sched_barrier(0)
    const char* XI = P.me + WS_XI; const char* WI = P.me + WS_WI;
    constexpr size_t kstep = 32768, hstep = 16384;
    const u32x2 cl = lds64v(TAB_COLS + ((blockIdx.x & 4) * 2 + (place >> 2)) * 8);
    const u64 mycols = (u64)rfl(cl.y) << 32 | rfl(cl.x); const u32 pm = (blockIdx.x & 3) * 4 + (place & 3);
    u32 cur = pm | ((u32)mycols & 255u) << 4;
    bool fresh = true;
    bf16x8 At[4][2], B0[2][2], B1[2][2];
    for (u32 ui = 0;; ++ui) {
        const u32 nxt = pm | ((u32)(mycols >> (8 * ui + 8)) & 255u) << 4;
        const bool has_next = (nxt >> 4) != 255u, dist = (cur >> 4) < (u32)DP, chain = has_next && !dist;
        const char* cA = XI + (size_t)(cur & 15) * PANEL; const char* cB = WI + (size_t)(cur >> 4) * PANEL;
        if (fresh) {
            wait_panels(P, cur, ui ? 4u : 0u, call, wid, lane);
            STAMP(P, 8 + ui * 4); if (ui == 0) DUMP(P, 1, __builtin_amdgcn_s_memtime());
            G_STAGE(G_SB(0, 0), cB); G_STAGE(G_SB(0, 1), cB + hstep); G_STAGE(G_SA(0, 0), cA); G_STAGE(G_SA(0, 1), cA + hstep);
            if (wr == 1) G_BAR;
            G_WV(2); G_BAR;
            G_STAGE(G_SB(1, 0), cB + kstep); G_STAGE(G_SA(1, 0), cA + kstep); G_STAGE(G_SB(1, 1), cB + hstep + kstep);
            G_WV(6); G_BAR;
        }
        f32x4 acc[2][2][4][2];
#pragma unroll
        for (int a = 0; a < 2; ++a)
#pragma unroll
            for (int b = 0; b < 2; ++b)
#pragma unroll
                for (int m = 0; m < 4; ++m)
#pragma unroll
                    for (int n = 0; n < 2; ++n) acc[a][b][m][n] = (f32x4){0.f, 0.f, 0.f, 0.f};
        const char* nA = chain ? XI + (size_t)(nxt & 15) * PANEL : cA; const char* nB = chain ? WI + (size_t)(nxt >> 4) * PANEL : cB;
        for (int t = 0; t < 64; t += 2) {
            const bool last = (t == 62);
            const char* a1 = cA + (size_t)(t + 1) * kstep;
            const char* a2 = last ? nA : cA + (size_t)(t + 2) * kstep; const char* b2 = last ? nB : cB + (size_t)(t + 2) * kstep;
            const char* a3 = a2 + kstep; const char* b3 = b2 + kstep;
            {
                const bool staged = ui == 0 && (t == 6 || t == 20 || t == 34 || t == 48);
                if (staged || (last && chain)) wait_panels(P, staged ? cur : nxt, t == 6 ? 1u : t == 20 ? 2u : t == 34 ? 3u : 4u, call, wid, lane);
                if (staged) STAMP(P, t == 6 ? 4 : t == 20 ? 5 : t == 34 ? 6 : 7);
            }
            G_LDB(B0, 0, 0); G_LDB(B1, 0, 1); G_SCHED; G_LDA(At, 0, 0); G_STAGE(G_SA(1, 1), a1 + hstep);
            G_WV(8); G_WL(0); G_BAR; G_MMA(0, 0, At, B0); G_MMA(0, 1, At, B1); G_BAR; G_SCHED;
            G_LDA(At, 0, 1); G_STAGE(G_SB(0, 0), b2); G_STAGE(G_SB(0, 1), b2 + hstep); G_STAGE(G_SA(0, 0), a2);
            G_WV(8); G_WL(0); G_BAR; G_MMA(1, 0, At, B0); G_MMA(1, 1, At, B1); G_BAR; G_SCHED;
            G_LDB(B0, 1, 0); G_LDB(B1, 1, 1); G_SCHED; G_LDA(At, 1, 0); G_STAGE(G_SA(0, 1), a2 + hstep);
            G_WV(8); G_WL(0); G_BAR; G_MMA(0, 0, At, B0); G_MMA(0, 1, At, B1); G_BAR; G_SCHED;
            G_LDA(At, 1, 1); G_STAGE(G_SB(1, 0), b3); G_STAGE(G_SB(1, 1), b3 + hstep); G_STAGE(G_SA(1, 0), a3);
            G_WV(8); G_WL(0); G_BAR; G_MMA(1, 0, At, B0); G_MMA(1, 1, At, B1); G_BAR; G_SCHED;
        }
        STAMP(P, 9 + ui * 4); DUMP(P, 2 + ui, __builtin_amdgcn_s_memtime());
        if (!chain) G_WV(0);
        if (GAR_ALIGN && wr == 0) G_BAR;
        u32 v[64];
        {
            int tz = threadIdx.x; asm volatile("" : "+v"(tz));
            const u32 ln = tz & 63;
            u32 bad; asm volatile("ds_read_b32 %0, %1\n\ts_waitcnt lgkmcnt(0)" : "=v"(bad) : "v"(CTL_BAD) : "memory");
            u16* o = y_base(P, cur & 15, global_panel(cur >> 4, P.rank), wid, ln);
#pragma unroll
            for (int p = 0; p < 16; p += 2) {
                const int ai = p >> 3, m = (p >> 1) & 3;
#pragma unroll
                for (int i = 0; i < 4; ++i) {
                    const f32x4 xa = acc[ai][0][m][i >> 1], xb = acc[ai][1][m][i >> 1];
                    u32 a = cvt_pk(xa[(i & 1) * 2], xa[(i & 1) * 2 + 1]), b = cvt_pk(xb[(i & 1) * 2], xb[(i & 1) * 2 + 1]);
                    if (bad) a = b = 0x7fc07fc0u;
                    v[4 * p + i] = (u32)__builtin_amdgcn_update_dpp((int)a, (int)b, 0x128, 0xf, 0xc, false);
                    v[4 * p + 4 + i] = (u32)__builtin_amdgcn_update_dpp((int)b, (int)a, 0x128, 0xf, 0x3, false);
                }
                st_out<GAR_GST>(o + y_piece_off(p), (u32x4){v[4 * p], v[4 * p + 1], v[4 * p + 2], v[4 * p + 3]});
                st_out<GAR_GST>(o + y_piece_off(p + 1), (u32x4){v[4 * p + 4], v[4 * p + 5], v[4 * p + 6], v[4 * p + 7]});
            }
            if (chain) { if (GAR_ALIGN && wr == 1) G_BAR; }
            else {
                if (!GAR_ALIGN && wr == 0) G_BAR;
                BAR();
                if (dist) {
                    const u32 scr = wid * 16384, to = scr + ((ln & 7) * 8 + (ln & 8) / 2 + (ln >> 4)) * 16;
#pragma unroll
                    for (int p = 0; p < 16; ++p) lds128w(to + p * 1024, (u32x4){v[4 * p], v[4 * p + 1], v[4 * p + 2], v[4 * p + 3]});
                    CBAR();
#pragma unroll
                    for (int p = 0; p < 16; ++p) { const u32x4 t = lds128(scr + ln * 16 + p * 1024); v[4 * p] = t.x; v[4 * p + 1] = t.y; v[4 * p + 2] = t.z; v[4 * p + 3] = t.w; }
                    WAIT_LGKM0();
                    u32 Mr[32], Er[32]; pack_group(P, v, Mr, Er, GY + cur, wid, false, 255u & ~(1u << P.rank), call, wid, ln); }
                STAMP(P, 10 + ui * 4);
                if (!has_next) break;
            }
        }
        fresh = !chain; cur = nxt;
    }
}

__global__ void __launch_bounds__(512, 2) fused(const Args A) {
    extern __shared__ __attribute__((aligned(16))) unsigned char lds[];
    const u32 wid = rfl(threadIdx.x >> 6), lane = threadIdx.x & 63;
    const u32 role = pin((A.role[blockIdx.x >> 1] >> ((blockIdx.x & 1) * 16)) & 0xffffu);
    char* peer = pin(A.ws[0]);
#pragma unroll
    for (u32 j = 1; j < NR; ++j) { char* c = pin(A.ws[j]); peer = (role & 7) == j ? c : peer; }
    const Params P = {pin(A.me), peer, pin(A.x), pin(A.w), pin(A.out), pin((u32)A.rank), pin((u32)A.nunits)};
    call_load(P, wid, lane);
    if (wid == 1 || wid == 2) {
        const char* ka = (const char*)(const GAS char*)(u64)__builtin_amdgcn_kernarg_segment_ptr() + lane * 16;
        if (wid == 1 && !(role & 0x8000u)) { if (lane < 4) glds_plain(ka, LDS_TAB); }
        else if ((wid == 1) == ((role & 0x8000u) != 0)) { glds_plain(ka, LDS_TAB); glds_plain(ka + 1024, LDS_TAB + 1024); glds_plain(ka + 2048, LDS_TAB + 2048); }
    }
    if (role & 0x8000u) {
        if (threadIdx.x == 0) { lds32vw(CTL_NUNIT, 0u); lds32vw(CTL_LANDED, 0u); lds32vw(CTL_NCHUNK, 0u); }
        if (wid == 1 && lane < 8) lds32vw(CTL_NEXT + lane * 4, lane);
        WAIT_LGKM0(); BAR();
        puller(P, role, wid, lane); return;
    }
    STAMP(P, 0); DUMP(P, 0, __builtin_amdgcn_s_memtime());
    if (threadIdx.x == 0) lds32vw(CTL_BAD, 0u);
    pack_init(wid, lane);
    const u32 sc = role & 255u;
    if (sc >= (u32)(8 * SOLOG)) { const u64 t0 = now(); while (now() - t0 < GAR_HOLD) __builtin_amdgcn_s_sleep(2); }
    for (u32 it = 0;; ++it) {
        const u32 rb = NENC - 1 - (role & 255u), e = it > 1 ? NENC + rb + (it - 2) * NLIGHT : rb;
        if (it > 1 && (e >= (u32)NPACK || rb >= (u32)NLIGHT)) break;
        if (it || (wid == 1 && sc < (u32)(8 * SOLOG))) pack_inputs(P, it ? tab16(TAB_ENC, e) : solo_gid(sc), it ? wid : solo_w(sc), !it, wid, lane);
        WAIT_VM(0); WAIT_LGKM0(); BAR(); STAMP(P, it ? umin(it, 2) : 3);
    }
    const u32 call = call_take(wid);
    call_bump(P, wid, lane, call);
    if (role >> 8) gemm_role(P, lds, (role >> 8) - 1, call, wid, lane);
}

#define GAR_T3 2
#define GAR_T2 6
#define GAR_T1 14
#define GAR_SPARE 1
#define GAR_XAHEAD 16
#define GAR_EPOS 18
#define GAR_QTAIL 6
constexpr int QTAIL = GAR_QTAIL;
static void build_tables(Args& P) {
    unsigned char* cols = (unsigned char*)P.cols; u16* stream = (u16*)P.stream; u16* enc = (u16*)P.enc; u16* role = (u16*)P.role;
    {
        int peer[2][4], np[2] = {0, 0}, cnt[8] = {}, seen[2] = {0, 0}, n = 0;
        for (int d = 1; d < NR; ++d) { const int q = (P.rank + d) & 7, h = ((0xb2 >> q) ^ (0xb2 >> P.rank)) & 1; peer[h][np[h]++] = q; }
        for (int b = 0; b < NWG; ++b) {
            const int h = (b >> 2) & 1, i = b >> 3, g0 = 32 - 4 * (h ? CH : CL), near = h ? 4 * NH : 3 * NL, far = h ? 3 * FL : 4 * FH, k = seen[h]++;
            if (k >= near + far) { role[b] = (u16)(n++ | (i >= g0 ? i - g0 + 1 : 0) << 8); continue; }
            const int side = k < near ? h : !h, j = k < near ? k : k - near, q = peer[side][side ? (j + j / 4) & 3 : j % 3];
            role[b] = (u16)(0x8000 | q | cnt[q]++ << 3);
        }
        for (int b = 0; b < NWG; ++b) if (role[b] & 0x8000) role[b] |= (u16)(cnt[role[b] & 7] << 7);
    }
    int col[LP], nc = 0;
    {
        const int rest = LP - (ROUNDS - 1) * CPR, r0 = (rest + 1) / 2; const bool spare = GAR_SPARE && DP <= 2 * (CPR - rest);
        int nd = 0, nr = DP;
        for (int i = 0; i < 128; ++i) cols[i] = 0xff;
        for (int r = 0; r < ROUNDS; ++r) for (int c = 0; c < 8; ++c) for (int cg = 0; cg < 2; ++cg) {
            const bool heavy = c < (cg ? rest - r0 : r0);
            if (c < (cg ? CH : CL) && (r + 1 < ROUNDS || heavy)) cols[(cg * 8 + c) * 8 + r] = (unsigned char)(col[nc++] = nd < DP && !(spare && heavy) ? nd++ : nr++);
        }
    }
    auto wgid = [&](int jj, int dest, int h) { return GW + 2 * (jj >= DP ? jj + (NR - 1) * DP : dest * DP + jj) + h; };
    int k = 0;
    auto put = [&](int gid, int lv) { for (int j = 0; j < (1 << lv); ++j) stream[k++] = (u16)(gid | ((1 << lv) + j) << 10); };
    auto stage = [&](int st, int i0, int i1) {
        const int h = st >> 1, half = st & 1;
        for (int i = i0; i < i1; ++i) {
            const int gid = i < 16 ? GX + 16 * h + i : wgid(col[i - 16], P.rank, h);
            if ((i < 16 && h == 0) || i >= 16 + CPR - QTAIL) { for (int j = 0; j < 2; ++j) stream[k++] = (u16)(gid | (4 + 2 * half + j) << 10); }
            else stream[k++] = (u16)(gid | (2 + half) << 10);
        }
    };
    stage(0, NRAW / 4, 16); stage(1, 0, GAR_XAHEAD);
    {
        u16 t[64]; const int n = k; int e = 0;
        for (int i = 0; i < n; ++i) t[i] = stream[i];
        k = 0;
        for (int i = 0; i < n || e < NRAW / 4; ++i) { if (i < n) stream[k++] = t[i]; if (i >= GAR_EPOS && e < NRAW / 4) stream[k++] = (u16)((GX + e++) | 2 << 10); }
    }
    stage(0, 16, 16 + CPR); stage(1, GAR_XAHEAD, 16 + CPR); stage(2, 0, 16 + CPR); stage(3, 0, 16 + CPR);
    for (int i = CPR; i < LP; ++i) for (int h = 0; h < 2; ++h) put(wgid(col[i], P.rank, h), i < 2 * CPR ? 1 : 0);
    for (int t = 0; t < 16 * DP; ++t) { const int left = 16 * DP - t; put(GY + t, left <= GAR_T3 ? 3 : left <= GAR_T2 ? 2 : left <= GAR_T1 ? 1 : 0); }
    P.nunits = k;
    k = 0;
    for (int i = 0; i < CPR; ++i) for (int d = 1; d < (col[i] < DP ? NR : 2); ++d) enc[k++] = (u16)wgid(col[i], (P.rank + d) & 7, 0);
    for (int i = SOLOG; i < 32; ++i) enc[k++] = (u16)(GX + i);
    for (int i = 0; i < CPR; ++i) { if (col[i] < DP) enc[k++] = (u16)wgid(col[i], P.rank, 0); enc[k++] = (u16)wgid(col[i], P.rank, 1); }
    for (int i = 0; i < CPR; ++i) if (col[i] < DP) for (int d = 1; d < NR; ++d) enc[k++] = (u16)wgid(col[i], (P.rank + d) & 7, 1);
    for (int i = CPR; i < LP; ++i) for (int h = 0; h < 2; ++h) for (int d = 0; d < (col[i] < DP ? NR : 1); ++d) enc[k++] = (u16)wgid(col[i], (P.rank + d) & 7, h);
}
}

extern "C" void kernel_launch(int rank, int nranks, ncclComm_t comm, hipStream_t stream,
                              void* const* d_in, const int* in_sizes, int n_in,
                              void* d_out, int out_size, const int* in_dtypes, int out_dtype,
                              void* const* d_out_all, void* const* d_ws_all, size_t ws_bytes) {
    using namespace gar;
    if (nranks != NR || (unsigned)rank >= (unsigned)NR || n_in != 2 || ws_bytes < WS_END || in_sizes[0] != MM * KS || in_sizes[1] != KS * NN || out_size != MM * NN) {
        fprintf(stderr, "kernel_launch: unexpected shapes\n"); return; }
    Args P = {};
    for (int p = 0; p < NR; ++p) P.ws[p] = (char*)d_ws_all[p];
    P.me = P.ws[rank];
    P.x = (const u16*)d_in[0]; P.w = (const u16*)d_in[1]; P.out = (u16*)d_out; P.rank = rank;
    build_tables(P);
    HIP_CHECK(hipFuncSetAttribute((const void*)fused, hipFuncAttributeMaxDynamicSharedMemorySize, LDS_BYTES));
    hipLaunchKernelGGL(fused, dim3(NWG), dim3(512), LDS_BYTES, stream, P);
    HIP_CHECK(hipGetLastError());
}
```

```cpp
#include <hip/hip_runtime.h>
#include <rccl/rccl.h>
#include <cstdio>
#include <cstdint>
#include <cstddef>
#pragma clang diagnostic ignored "-Wint-to-pointer-cast"

#define HIP_CHECK(expr) do { hipError_t _e = (expr); if (_e != hipSuccess) { \
    fprintf(stderr, "HIP error %s at %s:%d\n", hipGetErrorString(_e), __FILE__, __LINE__); \
    return; } } while (0)

namespace gar {
#define LAS __attribute__((address_space(3)))
#define GAS __attribute__((address_space(1)))
typedef unsigned u32;
typedef unsigned short u16;
typedef unsigned long long u64;
typedef unsigned u32x4 __attribute__((ext_vector_type(4)));
typedef unsigned u32x2 __attribute__((ext_vector_type(2)));
typedef short s16x4 __attribute__((ext_vector_type(4)));
typedef short bf16x8 __attribute__((ext_vector_type(8)));
typedef float f32x4 __attribute__((ext_vector_type(4)));

constexpr int NR = 8, MM = 4096, NN = 32768, KS = 512;
#define GAR_DP 11
constexpr int DP = GAR_DP;
constexpr int RP = 128 - NR * DP;
constexpr int LP = DP + RP;
constexpr int NWG = 256;
#define GAR_NL 9
#define GAR_FL 0
#define GAR_NH 8
#define GAR_FH 1
#define GAR_CL 6
#define GAR_WIN 6
constexpr int NL = GAR_NL, FL = GAR_FL, NH = GAR_NH, FH = GAR_FH, CL = GAR_CL, CH = 6, CPR = CL + CH;
constexpr int NPULL = 3 * (NL + FL) + 4 * (NH + FH);
constexpr int WIN = GAR_WIN;
static_assert(3 * NL + 4 * FH + 16 * CL <= 128 && 4 * NH + 3 * FL + 16 * CH <= 128, "roles");
#define GAR_SP2 1
#define GAR_ALIGN 1
#define GAR_NSAMP 6
#define GAR_NRAW 44
#define GAR_SOLOG 16
#define GAR_LEAD 48
#define GAR_HOLD 100
constexpr int SOLOG = GAR_SOLOG, NRAW = GAR_NRAW;
constexpr int LEAD = GAR_LEAD;
static_assert(NRAW <= 52 && NRAW % 4 == 0 && 8 * SOLOG <= NWG - NPULL && SOLOG == 16, "solo chunks");
constexpr int NENC = NWG - NPULL;
constexpr int GX = 0, GW = 32, GY = 288;
#define GAR_FX 3
constexpr int FX = GAR_FX, PK = DP + 8 * FX, LC = LP - FX;
constexpr int NGID = GY + 16 * PK, GM = NGID, NGT = GM + 16;
constexpr int NS = 32 + 2 * LC + 16 * DP + 16 * FX;
constexpr int ROUNDS = (LC + CPR - 1) / CPR;
constexpr int NUMAX = 704;
#define GAR_NLIGHT 80
constexpr int NLIGHT = GAR_NLIGHT;
constexpr int NSTAT = 3;
constexpr int NPACK = 288 - SOLOG;
constexpr u64 DEADLINE = 30000000ull;

constexpr size_t WS_CALL = 0;
constexpr size_t WS_TMO = 32768;
constexpr size_t WS_CNT = 36864;
constexpr size_t WS_CLAIM = 57344;
constexpr size_t WS_FLAG = 65536;
constexpr size_t FLAG_SRC = NGT * 32, FLAG_PAR = 8 * FLAG_SRC;
constexpr size_t WS_DBG = 1048576;
#define GAR_SKEW 0
constexpr size_t GRP_STRIDE = 131072 + GAR_SKEW;
constexpr size_t WS_STG = 2u << 20;
constexpr size_t STG_PAR = (NGID * GRP_STRIDE + 0x1fffff) & ~(size_t)0x1fffff;
constexpr size_t WS_INB = WS_STG + 2 * STG_PAR;
constexpr size_t WS_XI = WS_INB + 2 * 8 * 64 * 8192;
constexpr size_t PANEL = 64 * 32768;
constexpr size_t WS_WI = WS_XI + 16 * PANEL;
constexpr size_t WS_END = WS_WI + (size_t)LP * PANEL;
static_assert(WS_FLAG + 2 * FLAG_PAR <= WS_DBG && NGT <= 1024 && RP >= 8 * FX, "layout");

constexpr u32 LDS_CTL = 131072;
constexpr u32 LDS_LUT = LDS_CTL + 512, LDS_HIST = LDS_CTL + 1024;
constexpr u32 LDS_TAB = LDS_HIST + 4096, D_DESC = LDS_TAB + 3072, LDS_BYTES = D_DESC + NS * 64;
constexpr u32 CTL_CALL = LDS_CTL  , CTL_SIZE = LDS_CTL + 64  , CTL_BAD = LDS_CTL + 128;
constexpr u32 CTL_NUNIT = LDS_CTL + 192, CTL_LANDED = LDS_CTL + 256, CTL_NCHUNK = LDS_CTL + 320, CTL_NEXT = LDS_CTL + 384  ;
constexpr u32 E_L2 = 0, E_L3 = E_L2 + 4160, E_L4 = E_L3 + 2112, E_VAR = E_L4 + 1088, E_END = E_VAR + 2112;
static_assert(E_END <= 10240 && E_L4 % 16 == 0 && E_L3 % 16 == 0 && E_VAR % 16 == 0, "packer scratch");
constexpr u32 CAP2 = 4080, CAP3 = 2032, CAP4 = 1008, CAPV = 2032, FIXB = 10240, RAWB = 16384, RAWCODE = 255;
constexpr u32 RING = 65536, RMASK = RING - 1, ND = 6;
constexpr u32 D_SCR = RING, D_SCRB = 7360, D_L2 = 0, D_L3 = 4160, D_L4 = D_L3 + 2112;
constexpr u32 D_UNIT = D_SCR + ND * D_SCRB, DONE = 0x80000000u;
static_assert(D_L4 + 1088 <= D_SCRB && D_UNIT + (NUMAX + 8 * SOLOG) * 8 <= LDS_CTL && LDS_BYTES <= 163840 && D_DESC % 16 == 0, "puller LDS");

struct Args {
    char* ws[NR];
    char* me; const u16* x; const u16* w; u16* out;
    int rank, nunits;
    u32 role[NWG / 2];
    u32 stream[NUMAX / 2];
    u32 enc[144];
    u32 cols[64];
    u32 pad[(3072 - 104) / 4 - NWG / 2 - NUMAX / 2 - 144 - 64];
};
static_assert(sizeof(Args) == 3072 && ROUNDS < 8, "kernarg");
constexpr u32 TAB_STREAM = LDS_TAB + offsetof(Args, stream), TAB_ENC = LDS_TAB + offsetof(Args, enc), TAB_COLS = LDS_TAB + offsetof(Args, cols);
struct Params { char* me; char* peer; const u16* x; const u16* w; u16* out; u32 rank, nunits; };
template <class T> __device__ __forceinline__ T* pin(T* p) { u64 v = (u64)p; asm volatile("" : "+s"(v)); return (T*)(GAS T*)v; }
__device__ __forceinline__ u32 pin(u32 v) { asm volatile("" : "+s"(v)); return v; }

__device__ __forceinline__ u32 lds32(u32 a) { return *(LAS u32*)a; }
__device__ __forceinline__ u32 lds32v(u32 a) { return *(volatile LAS u32*)a; }
__device__ __forceinline__ void lds32vw(u32 a, u32 x) { *(volatile LAS u32*)a = x; }
__device__ __forceinline__ u32x2 lds64v(u32 a) { return *(volatile LAS u32x2*)a; }
__device__ __forceinline__ void lds64vw(u32 a, u32x2 x) { *(volatile LAS u32x2*)a = x; }
__device__ __forceinline__ u32x4 lds128(u32 a) { return *(LAS u32x4*)a; }
__device__ __forceinline__ void lds128w(u32 a, u32x4 x) { *(LAS u32x4*)a = x; }
__device__ __forceinline__ void lds_or(u32 a, u32 x) { __hip_atomic_fetch_or((LAS u32*)a, x, __ATOMIC_RELAXED, __HIP_MEMORY_SCOPE_WORKGROUP); }
__device__ __forceinline__ void lds_add(u32 a, u32 x) { __hip_atomic_fetch_add((LAS u32*)a, x, __ATOMIC_RELAXED, __HIP_MEMORY_SCOPE_WORKGROUP); }
__device__ __forceinline__ u32 rfl(u32 x) { return (u32)__builtin_amdgcn_readfirstlane((int)x); }
__device__ __forceinline__ u32 tab16(u32 t, u32 i) { return rfl(*(volatile LAS u16*)(t + i * 2)); }
#define CBAR() asm volatile("" ::: "memory")
#define WAIT_LGKM0() asm volatile("s_waitcnt lgkmcnt(0)" ::: "memory")
#define WAIT_VM(n) asm volatile("s_waitcnt vmcnt(%0)" :: "i"(n) : "memory")
#define BAR() do { CBAR(); __builtin_amdgcn_s_barrier(); CBAR(); } while (0)
#define GAR_SNT ""
#define GAR_INT ""
__device__ __forceinline__ void st_sys(void* p, u32x4 x) { asm volatile("global_store_dwordx4 %0, %1, off sc0 sc1" GAR_SNT "\n\ts_nop 1" :: "v"(p), "v"(x) : "memory"); }
__device__ __forceinline__ void st_dev(void* p, u32x4 x) { asm volatile("global_store_dwordx4 %0, %1, off sc1" GAR_INT "\n\ts_nop 1" :: "v"(p), "v"(x) : "memory"); }
#define GAR_OST 3
#define GAR_GST 3
template <int MODE> __device__ __forceinline__ void st_out(void* p, u32x4 x) {
    if (MODE == 0) *(u32x4*)p = x;
    else if (MODE == 1) __builtin_nontemporal_store(x, (u32x4*)p);
    else if (MODE == 2) st_dev(p, x);
    else if (MODE == 9) asm volatile("" :: "v"(p), "v"(x) : "memory");
    else asm volatile("global_store_dwordx4 %0, %1, off sc1 nt\n\ts_nop 1" :: "v"(p), "v"(x) : "memory");
}
__device__ __forceinline__ void glds_sys(const void* g, u32 dst) {
    u32 keep;
    asm volatile("s_mov_b32 %0, m0\n\ts_mov_b32 m0, %2\n\ts_nop 0\n\tglobal_load_lds_dwordx4 %1, off sc0 sc1\n\ts_mov_b32 m0, %0" : "=&s"(keep) : "v"(g), "s"(dst) : "memory");
}
__device__ __forceinline__ void glds_plain(const void* g, u32 dst) {
    u32 keep;
    asm volatile("s_mov_b32 %0, m0\n\ts_mov_b32 m0, %2\n\ts_nop 0\n\tglobal_load_lds_dwordx4 %1, off\n\ts_mov_b32 m0, %0" : "=&s"(keep) : "v"(g), "s"(dst) : "memory");
}
#define GAR_WNT " nt"
#define GAR_RNT ""
#define GAR_XNT 0
#define GAR_BAUX 0
__device__ __forceinline__ void glds_w(const void* g, u32 dst) {
    u32 keep;
    asm volatile("s_mov_b32 %0, m0\n\ts_mov_b32 m0, %2\n\ts_nop 0\n\tglobal_load_lds_dwordx4 %1, off" GAR_WNT "\n\ts_mov_b32 m0, %0" : "=&s"(keep) : "v"(g), "s"(dst) : "memory");
}
__device__ __forceinline__ u64 now() { return __builtin_amdgcn_s_memrealtime(); }
template <int CTRL, int RM, bool BC> __device__ __forceinline__ u32 dpp(u32 v) { return (u32)__builtin_amdgcn_update_dpp(0, (int)v, CTRL, RM, 0xf, BC); }
__device__ __forceinline__ u32 scan_incl(u32 v) {
    v += dpp<0x111, 0xf, true>(v); v += dpp<0x112, 0xf, true>(v); v += dpp<0x114, 0xf, true>(v); v += dpp<0x118, 0xf, true>(v);
    v += dpp<0x142, 0xa, false>(v); v += dpp<0x143, 0xc, false>(v); return v;
}
__device__ __forceinline__ u32 umax(u32 a, u32 b) { return a > b ? a : b; }
__device__ __forceinline__ u32 umin(u32 a, u32 b) { return a < b ? a : b; }
__device__ __forceinline__ u32 wave_max(u32 v) {
    v = umax(v, dpp<0x111, 0xf, true>(v)); v = umax(v, dpp<0x112, 0xf, true>(v)); v = umax(v, dpp<0x114, 0xf, true>(v)); v = umax(v, dpp<0x118, 0xf, true>(v));
    v = umax(v, dpp<0x142, 0xa, false>(v)); v = umax(v, dpp<0x143, 0xc, false>(v)); return (u32)__builtin_amdgcn_readlane((int)v, 63);
}
__device__ __forceinline__ u32 popc(u32 x) { return (u32)__builtin_popcount(x); }
__device__ __forceinline__ u32 perm(u32 hi, u32 lo, u32 sel) { return __builtin_amdgcn_perm(hi, lo, sel); }

#define STAMP(P, slot) do {} while (0)
#define TICK(t) do {} while (0)
#define ACC(sum, t) do {} while (0)
#define DUMP(P, slot, val) do {} while (0)

#define CSTAMP(i) do {} while (0)
#define CS_ARG
#define CS_DECL
#define CS_PASS

__device__ __forceinline__ u32 nzb(u32 z) { return ((z & 0x7f7f7f7fu) + 0x7f7f7f7fu) | z; }
typedef unsigned short u16x2 __attribute__((ext_vector_type(2)));
struct Classifier {
    u32 s1, s2, s3, nb, tlo, thi; bool win;
    __device__ __forceinline__ void init(u32 t) {
        const u32 a = (t >> 8) & 255u, b = (t >> 16) & 255u, c = t >> 24, lo = umin(a, umin(b, c));
        win = umax(a, umax(b, c)) - lo <= 6u;
        const u64 l = (1ull << (((a - lo) & 7u) * 8)) | (2ull << (((b - lo) & 7u) * 8)) | (3ull << (((c - lo) & 7u) * 8));
        tlo = (u32)l; thi = (u32)(l >> 32); nb = ((0u - lo) & 255u) * 0x01000100u;
        s1 = a * 0x01010101u; s2 = b * 0x01010101u; s3 = c * 0x01010101u;
    }
    __device__ __forceinline__ u32 any(u32 e) const {
        const u32 a1 = nzb(e ^ s1), a2 = nzb(e ^ s2), a3 = nzb(e ^ s3);
        return ((~(a1 & a3) >> 7) & 0x01010101u) | ((~(a2 & a3) >> 6) & 0x02020202u);
    }
    __device__ __forceinline__ u32 near(u32 e) const {
        const u16x2 x = __builtin_bit_cast(u16x2, e), n = __builtin_bit_cast(u16x2, nb), top = {0x07ff, 0x07ff};
        u32 sh; asm("v_pk_mad_u16 %0, %1, %2, %3" : "=v"(sh) : "v"(e), "v"(0x01000100u), "s"(nb));
        const u16x2 hi = __builtin_elementwise_min((u16x2)(x + n), top), lo = __builtin_elementwise_min(__builtin_bit_cast(u16x2, sh), top);
        return perm(thi, tlo, perm(__builtin_bit_cast(u32, hi), __builtin_bit_cast(u32, lo), 0x07030501u));
    }
    template <int N> __device__ __forceinline__ void run(const u32* e, u32* c) const {
        if (win) {
#pragma unroll
            for (int i = 0; i < N; ++i) c[i] = near(e[i]);
        } else {
#pragma unroll
            for (int i = 0; i < N; ++i) c[i] = any(e[i]);
        }
    }
};
__device__ __forceinline__ u32 escbits(u32 c) { return ~(c | (c >> 1)) & 0x55555555u; }
struct Appender {
    u32 lo, fill, addr;
    __device__ __forceinline__ void init(u32 off) { lo = 0; fill = (off & 3u) * 8u; addr = off & ~3u; }
    __device__ __forceinline__ void push(u32 k, u32 mk) {
        const u64 t = (u64)k << fill;
        lo |= (u32)t; fill += popc(mk);
        if (fill >= 32u) { lds_or(addr, lo); lo = (u32)(t >> 32); addr += 4; }
        fill &= 31u;
    }
    __device__ __forceinline__ void flush() { if (lo) lds_or(addr, lo); }
};
__device__ __forceinline__ u32 escmask(u32 c) { return perm(0u, 0xffu, c); }
__device__ __forceinline__ u32 selector(u32 mk) { return lds32(__builtin_amdgcn_udot4(mk & 0x20100804u, 0x01010101u, LDS_LUT, false)); }
template <int R> __device__ __forceinline__ u32 pack_level(u32 src, u32 n, u32 t, u32 cdst, u32 edst, u32 cap, u32 lane) {
    Classifier K; K.init(t);
    if (lane < 16) *(LAS unsigned char*)(src + n + lane) = (unsigned char)K.s1;
    u32 ne = 0;
    for (u32 r = 0; r * 1024 < n; r += R) {
        CBAR();
        u32 q[4 * R], c[4 * R], m[4 * R], l[4 * R], cnt = 0;
#pragma unroll
        for (int i = 0; i < R; ++i) { const u32 p0 = (r + i) * 1024 + lane * 16; const u32x4 x = lds128(src + p0); const bool act = p0 < n;
            q[4 * i] = act ? x.x : K.s1; q[4 * i + 1] = act ? x.y : K.s1; q[4 * i + 2] = act ? x.z : K.s1; q[4 * i + 3] = act ? x.w : K.s1; }
        K.run<4 * R>(q, c);
#pragma unroll
        for (int i = 0; i < 4 * R; ++i) { m[i] = escmask(c[i]); l[i] = selector(m[i]); }
#pragma unroll
        for (int i = 0; i < R; ++i) cnt |= ((popc(m[4 * i]) + popc(m[4 * i + 1]) + popc(m[4 * i + 2]) + popc(m[4 * i + 3])) >> 3) << (16 * i);
        const u32 inc = scan_incl(cnt), tot = (u32)__builtin_amdgcn_readlane((int)inc, 63), exc = inc - cnt;
        if (ne + (tot & 0xffffu) + (tot >> 16) > cap) return ~0u;
#pragma unroll
        for (int i = 0; i < R; ++i) {
            Appender A; A.init(edst + ne + (i ? (tot & 0xffffu) + (exc >> 16) : (exc & 0xffffu)));
#pragma unroll
            for (int j = 0; j < 4; ++j) A.push(perm(0u, q[4 * i + j], l[4 * i + j]), m[4 * i + j]);
            A.flush();
            if ((r + i) * 1024 + lane * 16 < n) *(LAS u32*)(cdst + ((r + i) * 64 + lane) * 4) = c[4 * i] | (c[4 * i + 1] << 2) | (c[4 * i + 2] << 4) | (c[4 * i + 3] << 6);
        }
        ne += (tot & 0xffffu) + (tot >> 16);
    }
    CBAR();
    return ne;
}
template <int N> __device__ __forceinline__ u32 rank_ror(u32 key) { return (dpp<0x120 + N, 0xf, true>(key) > key ? 1u : 0u) + rank_ror<N - 1>(key); }
template <> __device__ __forceinline__ u32 rank_ror<0>(u32) { return 0u; }
__device__ __forceinline__ u32 row_or(u32 x) { x |= dpp<0x128, 0xf, true>(x); x |= dpp<0x124, 0xf, true>(x); x |= dpp<0x122, 0xf, true>(x); x |= dpp<0x121, 0xf, true>(x); return rfl(x); }
__device__ __forceinline__ void split(const u32 (&v)[64], u32 (&Mr)[32], u32 (&Er)[32]) {
#pragma unroll
    for (int g = 0; g < 32; ++g) {
        const u32 a = __builtin_amdgcn_alignbit(v[2 * g], v[2 * g], 31), b = __builtin_amdgcn_alignbit(v[2 * g + 1], v[2 * g + 1], 31);
        Er[g] = perm(b, a, 0x07050301u); Mr[g] = perm(b, a, 0x06040200u);
        asm volatile("" : "+v"(Er[g]), "+v"(Mr[g]));
    }
}
__device__ __forceinline__ u32 pack_chunk(const u32 (&v)[64], u32 (&Mr)[32], u32 (&Er)[32], u32 (&Cr)[8], u32 scr, u32 lane, bool solo CS_ARG) {
    CBAR(); CSTAMP(0);
    const u32 hist = solo ? scr + 10240 : LDS_HIST;
    if (solo) {
#pragma unroll
        for (int j = 0; j < 4; ++j) lds128w(hist + j * 1024 + lane * 16, (u32x4){0u, 0u, 0u, 0u});
        CBAR();
#pragma unroll
        for (int i = 0; i < 12; ++i) lds_add(hist + (((v[i * 5 + 2] >> ((i & 1) * 16 + 3)) & 0xff0u) | ((lane & 3) * 4)), 1u);
    }
#pragma unroll
    for (int i = 0; i < GAR_NSAMP; ++i) lds_add(hist + (((v[i * (60 / GAR_NSAMP) + 5] >> ((i & 1) * 16 + 3)) & 0xff0u) | ((lane & 3) * 4)), 1u);
#pragma unroll
    for (int j = 0; j < 10; ++j) lds128w(scr + j * 1024 + lane * 16, (u32x4){0u, 0u, 0u, 0u});
    WAIT_LGKM0();
    if (!solo) BAR();
    CBAR();
    u32 key = 0;
#pragma unroll
    for (int i = 0; i < 4; ++i) { const u32 bin = i * 64 + lane; const u32x4 h = lds128(hist + bin * 16); key = umax(key, ((h.x + h.y + h.z + h.w) << 8) | bin); }
    split(v, Mr, Er);
    CSTAMP(1);
    { const auto r = __builtin_amdgcn_permlane32_swap(key, key, false, false); key = umax(r[0], r[1]); }
    { const auto r = __builtin_amdgcn_permlane16_swap(key, key, false, false); key = umax(r[0], r[1]); }
    const u32 rank = rank_ror<15>(key), lev = (rank * 11u) >> 5, sym = (key & 255u) << (8 * (rank - 3 * lev + 1));
    const u32 T[4] = {row_or(lev == 0 ? sym : 0u), row_or(lev == 1 ? sym : 0u), row_or(lev == 2 ? sym : 0u), row_or(lev == 3 ? sym : 0u)};
    CSTAMP(3);
    u32 cnt = 0, mk[32], sl[32];
    {
        Classifier K; K.init(T[0]);
        u32 c[32]; K.run<32>(Er, c);
#pragma unroll
        for (int g = 0; g < 32; ++g) { mk[g] = escmask(c[g]); sl[g] = selector(mk[g]); cnt += popc(mk[g]); }
#pragma unroll
        for (int k = 0; k < 8; ++k) Cr[k] = c[4 * k] | (c[4 * k + 1] << 2) | (c[4 * k + 2] << 4) | (c[4 * k + 3] << 6);
    }
    cnt >>= 3;
    const u32 inc = scan_incl(cnt), n2 = (u32)__builtin_amdgcn_readlane((int)inc, 63);
    CSTAMP(4);
    if (n2 > CAP2) return RAWCODE;
    Appender A; A.init(scr + E_L2 + inc - cnt);
#pragma unroll
    for (int g = 0; g < 32; ++g) A.push(perm(0u, Er[g], sl[g]), mk[g]);
    A.flush();
    CBAR(); CSTAMP(5);
    const u32 d2 = (n2 + 15) >> 4;
    const u32 n3 = pack_level<2>(scr + E_L2, n2, T[1], scr + E_VAR + 16, scr + E_L3, CAP3, lane);
    CSTAMP(6);
    if (n3 == ~0u) return RAWCODE;
    const u32 d3 = (n3 + 15) >> 4;
    const u32 n4 = pack_level<1>(scr + E_L3, n3, T[2], scr + E_VAR + 16 + 4 * d2, scr + E_L4, CAP4, lane);
    CSTAMP(7);
    if (n4 == ~0u) return RAWCODE;
    const u32 d4 = (n4 + 15) >> 4, b5 = 16 + 4 * (d2 + d3 + d4);
    const u32 n5 = pack_level<1>(scr + E_L4, n4, T[3], scr + E_VAR + 16 + 4 * (d2 + d3), scr + E_VAR + b5, CAPV - b5, lane);
    if (n5 == ~0u) return RAWCODE;
    if (lane == 0) lds128w(scr + E_VAR, (u32x4){T[0] | d2, T[1] | d3, T[2] | d4, T[3]});
    CBAR();
    return (b5 + n5 + 15) >> 4;
}
__device__ __forceinline__ void pack_init(u32 wid, u32 lane) {
    if (wid >= 4) lds128w(LDS_HIST + (wid - 4) * 1024 + lane * 16, (u32x4){0u, 0u, 0u, 0u});
    if (wid == 1 && lane < 16) { u32 sel = 0x0c0c0c0cu, k = 0;
#pragma unroll
        for (u32 b = 0; b < 4; ++b) if ((lane >> b) & 1) { sel = (sel & ~(255u << k)) | (b << k); k += 8; }
        *(LAS u32*)(LDS_LUT + lane * 4) = sel; }
}
__device__ __forceinline__ u32 chunk_bytes(u32 code) { return code == RAWCODE ? RAWB : FIXB + 16 * code; }

__device__ __forceinline__ void pack_store(char* dst, u32 code, const u32 (&Mr)[32], const u32 (&Er)[32], const u32 (&Cr)[8], u32 scr, u32 lane) {
    char* dm = dst + (chunk_bytes(code) - 8192);
#pragma unroll
    for (int j = 0; j < 8; ++j) st_sys(dm + j * 1024, (u32x4){Mr[4 * j], Mr[4 * j + 1], Mr[4 * j + 2], Mr[4 * j + 3]});
    if (code == RAWCODE) {
#pragma unroll
        for (int j = 0; j < 8; ++j) st_sys(dst + j * 1024, (u32x4){Er[4 * j], Er[4 * j + 1], Er[4 * j + 2], Er[4 * j + 3]});
    } else {
        st_sys(dst, (u32x4){Cr[0], Cr[1], Cr[2], Cr[3]}); st_sys(dst + 1024, (u32x4){Cr[4], Cr[5], Cr[6], Cr[7]});
        if (lane < code) st_sys(dst + 2048, lds128(scr + E_VAR + lane * 16));
        if (lane + 64 < code) st_sys(dst + 3072, lds128(scr + E_VAR + 1024 + lane * 16));
    }
}
__device__ __forceinline__ void raise_flag(const Params& P, u32 q, u32 gid, u32 w, u32 call, u32 code) {
    char* wq = (char*)(GAS char*)__builtin_bit_cast(u64, lds64v(LDS_TAB + q * 8));
    u32* f = (u32*)(wq + WS_FLAG + (call & 1) * FLAG_PAR + (size_t)P.rank * FLAG_SRC + gid * 32) + w;
    __hip_atomic_store(f, (call << 8) | code, __ATOMIC_RELAXED, __HIP_MEMORY_SCOPE_SYSTEM);
}
__device__ __forceinline__ void pack_group(const Params& P, const u32 (&v)[64], u32 (&Mr)[32], u32 (&Er)[32], u32 gid, u32 w, bool solo, u32 peers, u32 call, u32 wid, u32 lane) {
    u32 Cr[8];
    const u32 scr = wid * 16384;
    CS_DECL
    if (gid < 64u) STAMP(P, 40 + wid);
    const u32 code = rfl(pack_chunk(v, Mr, Er, Cr, scr, lane, solo CS_PASS));
    if (gid < 64u) STAMP(P, 48 + wid);
    u32 off = w * RAWB, mine = code;
    if (!solo) {
        if (lane == 0) lds32vw(CTL_SIZE + wid * 4, code);
        WAIT_LGKM0(); BAR();
        mine = lds32v(CTL_SIZE + (lane & 7) * 4);
        if (lane < 32) lds128w(LDS_HIST + wid * 512 + lane * 16, (u32x4){0u, 0u, 0u, 0u});
        off = 0;
#pragma unroll
        for (u32 j = 0; j < 7; ++j) { const u32 c = (u32)__builtin_amdgcn_readlane((int)mine, j); if (j < wid) off += chunk_bytes(c); }
    }
    pack_store(P.me + WS_STG + (call & 1) * STG_PAR + (size_t)gid * GRP_STRIDE + off + lane * 16, code, Mr, Er, Cr, scr, lane);
    if (gid < 64u) STAMP(P, 56 + wid);
    WAIT_VM(0);
    if (gid < 64u) STAMP(P, 64 + wid);
    WAIT_LGKM0();
    if (!solo) BAR();
    if ((solo ? (lane & 7) == w : wid == 0) && ((peers >> (lane >> 3)) & 1)) raise_flag(P, lane >> 3, gid, lane & 7, call, mine);
}

#define GAR_WINMODE 1
template <bool WRAP> struct Unp {
    u32 sel[4], lo[4], hi[4], of[4];
    __device__ __forceinline__ void fetch(u32 c, u32& off) {
#pragma unroll
        for (int j = 0; j < 4; ++j) {
            const u32 cj = (c >> (2 * j)) & 0x03030303u, mk = escmask(cj), m2 = mk & 0x02020202u;
            if (WRAP || GAR_WINMODE != 2) { const u32 a = off & ~3u; lo[j] = lds32(WRAP ? (a & RMASK) : a); hi[j] = lds32(WRAP ? ((a + 4) & RMASK) : a + 4); of[j] = off; }
            else lo[j] = lds32(off);
            u32 p; asm("v_mad_u32_u24 %0, %1, %2, %3" : "=v"(p) : "v"(m2), "s"(0x808080u), "v"(0x04040404u));
            sel[j] = (mk & p) | cj;
            off += popc(m2);
        }
    }
    __device__ __forceinline__ u32 finish(int j, u32 t) const {
        u32 w;
        if (WRAP || GAR_WINMODE == 0) w = __builtin_amdgcn_alignbyte(hi[j], lo[j], of[j] & 3u);
        else if (GAR_WINMODE == 1) w = __builtin_amdgcn_alignbyte(hi[j], lo[j], of[j]);
        else w = lo[j];
        return perm(w, t, sel[j]);
    }
    __device__ __forceinline__ u32x4 finish4(u32 t) const { return (u32x4){finish(0, t), finish(1, t), finish(2, t), finish(3, t)}; }
};
#define SCHED() __builtin_amdgcn_sched_barrier(0)
template <bool WRAP> __device__ __forceinline__ u32 unpack_round(u32 c, u32 t, u32 src, u32 dst) {
    const u32 cnt = popc(escbits(c)), inc = scan_incl(cnt);
    u32 off = src + inc - cnt;
    Unp<WRAP> u; u.fetch(c, off); SCHED();
    lds128w(dst, u.finish4(t));
    return (u32)__builtin_amdgcn_readlane((int)inc, 63);
}
__device__ __forceinline__ void ring_read8k(u32 a0, u32 (&r)[32]) {
#pragma unroll
    for (int j = 0; j < 8; ++j) { const u32x4 q = lds128((a0 + j * 1024) & RMASK); r[4 * j] = q.x; r[4 * j + 1] = q.y; r[4 * j + 2] = q.z; r[4 * j + 3] = q.w; }
}
__device__ __forceinline__ void unpack_exp(u32 S, u32 code, u32 scr, u32 lane, u32 (&Er)[32] CS_ARG) {
    const u32 a0 = S + lane * 16;
    if (code == RAWCODE) { ring_read8k(a0, Er); return; }
    const u32x4 ca = lds128(a0 & RMASK), cb = lds128((a0 + 1024) & RMASK);
    const u32 V = S + 2048;
    const u32x4 hd = lds128(V & RMASK);
    const u32 t1 = rfl(hd.x), t2 = rfl(hd.y), t3 = rfl(hd.z), t4 = rfl(hd.w), d2 = t1 & 255u, d3 = t2 & 127u, d4 = t3 & 63u;
    u32 c2[4], c3[2];
#pragma unroll
    for (int r = 0; r < 4; ++r) { const u32 i = r * 64 + lane; const u32 x = lds32((V + 16 + 4 * i) & RMASK); c2[r] = i < d2 ? x : 0x55555555u; }
#pragma unroll
    for (int r = 0; r < 2; ++r) { const u32 i = r * 64 + lane; const u32 x = lds32((V + 16 + 4 * (d2 + i)) & RMASK); c3[r] = i < d3 ? x : 0x55555555u; }
    const u32 x4 = lds32((V + 16 + 4 * (d2 + d3 + lane)) & RMASK), c4 = lane < d4 ? x4 : 0x55555555u;
    CSTAMP(1);
    unpack_round<true>(c4, t4, V + 16 + 4 * (d2 + d3 + d4), scr + D_L4 + lane * 16);
    CBAR();
    const u32 s4 = scr + D_L4 + unpack_round<false>(c3[0], t3, scr + D_L4, scr + D_L3 + lane * 16);
    if (d3 > 64) unpack_round<false>(c3[1], t3, s4, scr + D_L3 + 1024 + lane * 16);
    CBAR(); CSTAMP(2);
    const u32 C[8] = {ca.x, ca.y, ca.z, ca.w, cb.x, cb.y, cb.z, cb.w};
    u32 cnt = 0;
#pragma unroll
    for (int k = 0; k < 8; ++k) cnt += popc(escbits(C[k]));
    {
        const u32 n0 = popc(escbits(c2[0])), n1 = popc(escbits(c2[1]));
        const u32 i01 = scan_incl(n0 | (n1 << 16)), tot = (u32)__builtin_amdgcn_readlane((int)i01, 63);
        cnt = scan_incl(cnt) - cnt;
        u32 o0 = scr + D_L3 + (i01 & 0xffffu) - n0, o1 = scr + D_L3 + (tot & 0xffffu) + (i01 >> 16) - n1;
        Unp<false> u0, u1; u0.fetch(c2[0], o0); u1.fetch(c2[1], o1); SCHED();
        lds128w(scr + D_L2 + lane * 16, u0.finish4(t2)); lds128w(scr + D_L2 + 1024 + lane * 16, u1.finish4(t2));
        if (d2 > 128) {
            u32 s3 = scr + D_L3 + (tot & 0xffffu) + (tot >> 16);
            s3 += unpack_round<false>(c2[2], t2, s3, scr + D_L2 + 2048 + lane * 16);
            if (d2 > 192) unpack_round<false>(c2[3], t2, s3, scr + D_L2 + 3072 + lane * 16);
        }
    }
    CBAR(); CSTAMP(3);
    u32 off = scr + D_L2 + cnt;
    Unp<false> u[8];
    u[0].fetch(C[0], off); u[1].fetch(C[1], off);
#pragma unroll
    for (int k = 0; k < 8; ++k) {
        SCHED();
        if (k + 2 < 8) u[k + 2].fetch(C[k + 2], off);
        SCHED();
#pragma unroll
        for (int j = 0; j < 4; ++j) Er[4 * k + j] = u[k].finish(j, t1);
    }
    CBAR();
}
__device__ __forceinline__ u32x4 piece(const u32 (&Mr)[32], const u32 (&Er)[32], int p) {
    const u32 a = perm(Er[2 * p], Mr[2 * p], 0x05010400u), b = perm(Er[2 * p], Mr[2 * p], 0x07030602u);
    const u32 c = perm(Er[2 * p + 1], Mr[2 * p + 1], 0x05010400u), d = perm(Er[2 * p + 1], Mr[2 * p + 1], 0x07030602u);
    return (u32x4){__builtin_amdgcn_alignbit(a, a, 1), __builtin_amdgcn_alignbit(b, b, 1), __builtin_amdgcn_alignbit(c, c, 1), __builtin_amdgcn_alignbit(d, d, 1)};
}

__device__ __forceinline__ u32 x_piece_off(u32 half, int p) { return half * 16384 + p * 1024; }
__device__ __forceinline__ u32 w_piece_off(u32 h, int p) {
    const u32 g32 = p >> 2, n = (p >> 1) & 1, kb = p & 1;
    return (g32 & 1) * 16384 + ((2 * (2 * h + (g32 >> 1)) + n) * 2 + kb) * 1024;
}
__device__ __forceinline__ u16* y_base(const Params& P, u32 pm, u32 j, u32 w, u32 lane) {
    return P.out + (size_t)(pm * 256 + (w >> 2) * 64 + (lane & 7)) * NN + j * 256 + (w & 3) * 64 + (lane & 8) * 4 + (lane >> 4) * 8;
}
__device__ __forceinline__ u16* y_base_lin(const Params& P, u32 pm, u32 j, u32 w, u32 lane) {
    return P.out + (size_t)(pm * 256 + (w >> 2) * 64 + (lane >> 3)) * NN + j * 256 + (w & 3) * 64 + (lane & 7) * 8;
}
__device__ __forceinline__ size_t y_piece_off(int p) { return (size_t)((p >> 3) * 128 + (p & 7) * 8) * NN; }
__device__ __forceinline__ u32 local_panel(u32 j, u32 rank) { return j >= (u32)(NR * DP) ? j - (NR - 1) * DP : j - rank * DP; }
__device__ __forceinline__ u32 global_panel(u32 jj, u32 rank) { return jj >= (u32)DP ? jj + (NR - 1) * DP : jj + rank * DP; }
__device__ __forceinline__ u32* counter(const Params& P, u32 i) { return (u32*)(P.me + WS_CNT + i * 128); }
__device__ __forceinline__ u32* tmo_word(const Params& P) { return (u32*)(P.me + WS_TMO); }
__device__ __forceinline__ bool expired(const Params& P, u64 t0, u32 call) {
    if (__hip_atomic_load(tmo_word(P), __ATOMIC_RELAXED, __HIP_MEMORY_SCOPE_AGENT) == call || now() - t0 > DEADLINE) {
        __hip_atomic_store(tmo_word(P), call, __ATOMIC_RELAXED, __HIP_MEMORY_SCOPE_AGENT); return true; }
    return false;
}

__device__ __forceinline__ void store_image(const Params& P, u32 gid, u32 w, u32 q, u32 lane, const u32 (&Mr)[32], const u32 (&Er)[32]) {
    const bool isx = gid < (u32)GW;
    const u32 pn = isx ? gid & 15 : local_panel((gid - GW) >> 1, P.rank), ktl = (isx ? gid >> 4 : gid & 1) * 4 + (w >> 1);
    const u32 qi = (q - P.rank - 1) & 7, stage = qi == 7 ? 0u : 1 + (ktl >> 1);
    const u32 slot = stage == 0 ? ktl : stage * 14 - 6 + qi * 2 + (ktl & 1);
    char* o = P.me + (isx ? WS_XI : WS_WI) + (size_t)pn * PANEL + (size_t)slot * 32768 + lane * 16;
#pragma unroll
    for (int p = 0; p < 16; ++p) st_dev(o + (isx ? x_piece_off(w & 1, p) : w_piece_off(w & 1, p)), piece(Mr, Er, p));
    WAIT_VM(0);
    if (lane == 0) __hip_atomic_fetch_add(counter(P, isx ? pn : 16 + pn) + stage, 1u, __ATOMIC_RELAXED, __HIP_MEMORY_SCOPE_AGENT);
}
__device__ __forceinline__ u32* call_word(const Params& P, u32 wid) { return (u32*)(P.me + WS_CALL + blockIdx.x * 128 + wid * 4); }
__device__ __forceinline__ void call_load(const Params& P, u32 wid, u32 lane) {
    u32 keep;
    if (lane == 0) asm volatile("s_mov_b32 %0, m0\n\ts_mov_b32 m0, %2\n\ts_nop 0\n\tglobal_load_lds_dword %1, off sc1\n\ts_mov_b32 m0, %0" : "=&s"(keep) : "v"(call_word(P, wid)), "s"(CTL_CALL + wid * 4) : "memory");
}
__device__ __forceinline__ u32 call_take(u32 wid) { WAIT_VM(0); return rfl(lds32v(CTL_CALL + wid * 4)) + 1; }
__device__ __forceinline__ void call_bump(const Params& P, u32 wid, u32 lane, u32 call) { if (lane == 0) __hip_atomic_store(call_word(P, wid), call, __ATOMIC_RELAXED, __HIP_MEMORY_SCOPE_AGENT); }

__device__ __forceinline__ u32 solo_gid(u32 c) { return (c >> 2) & 15; }
__device__ __forceinline__ u32 solo_w(u32 c) { return (c & 3) | (c >> 6) << 2; }
__device__ __forceinline__ void load_x(const Params& P, u32 gid, u32 w, u32 lane, u32 (&v)[64]) {
    const u16* s = P.x + (size_t)((gid & 15) * 256 + (w & 1) * 128 + (lane & 15)) * KS + ((gid >> 4) * 4 + (w >> 1)) * 64 + (lane >> 4) * 8;
#pragma unroll
    for (int p = 0; p < 16; ++p) { const u32x4* a = (const u32x4*)(s + (p >> 1) * 16 * KS + (p & 1) * 32); const u32x4 t = GAR_XNT ? __builtin_nontemporal_load(a) : *a; v[4 * p] = t.x; v[4 * p + 1] = t.y; v[4 * p + 2] = t.z; v[4 * p + 3] = t.w; }
}
__device__ __forceinline__ void puller(const Params& P, u32 role, u32 wid, u32 lane) {
    const u32 q = role & 7, sub = (role >> 3) & 15, PW = (role >> 7) & 15;
    const u32 pc = sub + PW * (7 - wid);
    if (pc < (u32)NRAW) {
        const u32 gid = solo_gid(pc), w = solo_w(pc);
        u32 v[64], Mr[32], Er[32];
        load_x(P, gid, w, lane, v);
        const u32 call = call_take(wid);
        split(v, Mr, Er);
        char* o = P.peer + WS_INB + (size_t)(((call & 1) * 8 + P.rank) * 64 + pc) * 8192 + lane * 16;
#pragma unroll
        for (int j = 0; j < 8; ++j) st_sys(o + j * 1024, (u32x4){Mr[4 * j], Mr[4 * j + 1], Mr[4 * j + 2], Mr[4 * j + 3]});
        WAIT_VM(0); STAMP(P, 120 + wid);
        if (lane == 0) __hip_atomic_store((u32*)(P.peer + WS_FLAG + (call & 1) * FLAG_PAR + (size_t)P.rank * FLAG_SRC + (GM + gid) * 32) + w, call << 8, __ATOMIC_RELAXED, __HIP_MEMORY_SCOPE_SYSTEM);
    }
    const u32 call = call_take(wid);
    call_bump(P, wid, lane, call);
    if (wid == 0) {
        __builtin_amdgcn_s_setprio(3);
        const char* base = P.peer + WS_STG + (call & 1) * STG_PAR;
        const char* dummy = P.me + WS_TMO + 1024 + lane * 16;
        const u32 voff = lane * 16;
        u32 pos = 0, pub = 0, limit = RING / 1024;
        u64 tk = 0, c_full = 0, c_idle = 0, n_full = 0, n_ref = 0; (void)tk; (void)c_full; (void)c_idle; (void)n_full; (void)n_ref;
        for (u32 ui = 0;; ++ui) {
            u32 nu = rfl(lds32v(CTL_NUNIT));
            if ((nu & ~DONE) <= ui) {
                TICK(tk);
                WAIT_VM(0); lds32vw(CTL_LANDED, pos); pub = pos;
                while (nu <= ui) { __builtin_amdgcn_s_sleep(1); nu = rfl(lds32v(CTL_NUNIT)); }
                if ((nu & ~DONE) <= ui) break;
                ACC(c_idle, tk);
            }
            CBAR();
            const u32x2 un = lds64v(D_UNIT + ui * 8);
            const u32 ut = rfl(un.y), total = ut & 0x3ffffu, lastc = ut >> 18;
            const char* src = base + rfl(un.x);
            for (u32 o = 0; o < total; o += 4096) {
                if (pos + 4 > limit) {
                    bool drained = false; TICK(tk); ++n_ref;
                    for (;;) {
                        const u32 nx = lds32v(CTL_NEXT + (lane & 7) * 4);
                        u32 f = lastc;
#pragma unroll
                        for (u32 j = 0; j < ND; ++j) f = umin(f, (u32)__builtin_amdgcn_readlane((int)nx, j));
                        limit = (rfl(lds32v(D_DESC + f * 8)) >> 14) + RING / 1024;
                        if (pos + 4 <= limit) break;
                        if (!drained) { WAIT_VM(0); lds32vw(CTL_LANDED, pos); pub = pos; drained = true; }
                        __builtin_amdgcn_s_sleep(1);
                    }
                    ACC(c_full, tk); n_full += drained;
                }
                const u32 dst = (pos * 1024) & RMASK;
                if (o + 4096 <= total) {
                    u32 keep;
                    asm volatile("s_mov_b32 %0, m0\n\ts_mov_b32 m0, %3\n\ts_nop 4\n\t"
                                 "global_load_lds_dwordx4 %1, %2 sc0 sc1" GAR_RNT "\n\ts_waitcnt vmcnt(%4)\n\t"
                                 "global_load_lds_dwordx4 %1, %2 offset:1024 sc0 sc1" GAR_RNT "\n\ts_waitcnt vmcnt(%4)\n\t"
                                 "global_load_lds_dwordx4 %1, %2 offset:2048 sc0 sc1" GAR_RNT "\n\ts_waitcnt vmcnt(%4)\n\t"
                                 "global_load_lds_dwordx4 %1, %2 offset:3072 sc0 sc1" GAR_RNT "\n\ts_waitcnt vmcnt(%4)\n\t"
                                 "s_mov_b32 m0, %0" : "=&s"(keep) : "v"(voff), "s"(src + o), "s"(dst), "i"(WIN) : "memory");
                } else {
#pragma unroll
                    for (u32 j = 0; j < 4; ++j) { const u32 b = o + j * 1024 + voff; glds_sys(b < total ? src + b : dummy, dst + j * 1024); WAIT_VM(WIN);
                        if (j < 3 && pos + j + 1 > pub + WIN) { pub = pos + j + 1 - WIN; lds32vw(CTL_LANDED, pub); } }
                }
                pos += 4;
                if (pos > pub + WIN) { pub = pos - WIN; lds32vw(CTL_LANDED, pub); }
            }
            STAMP(P, 128 + umin(ui, 127)); DUMP(P, 64 + umin(ui, 127), total);
        }
        DUMP(P, 0, c_full); DUMP(P, 1, n_full); DUMP(P, 2, n_ref); DUMP(P, 3, c_idle);
    } else if (wid == 1) {
        u32 basek = 0, nch = 0, nun = 0;
        u32* ticket = (u32*)(P.me + WS_CLAIM + q * 128);
        const u32 nown = P.nunits & 0xffffu, nall = nown + (P.nunits >> 16);
        u32 tbase = NSTAT * PW - (call - 1) * (nown - NSTAT * PW + PW), tend = nown;
        for (u32 ui = 0;; ++ui) {
            while (rfl(lds32v(CTL_LANDED)) + LEAD < basek) __builtin_amdgcn_s_sleep(4);
            u32 idx = ui * PW + sub;
            if (ui >= (u32)NSTAT) {
                if (lane == 0) idx = __hip_atomic_fetch_add(ticket, 1u, __ATOMIC_RELAXED, __HIP_MEMORY_SCOPE_AGENT);
                idx = rfl(idx) + tbase;
                if (idx >= tend) {
                    if (tend == nall) break;
                    ticket = (u32*)(P.me + WS_CLAIM + P.rank * 128); tbase = nown - (call - 1) * (nall - nown + NPULL); tend = nall;
                    continue;
                }
            }
            const u32 e = tab16(TAB_STREAM, idx), gid = e & 1023u, part = e >> 10, lv = 31 - __builtin_clz(part), n = 8u >> lv, first = (part - (1u << lv)) * n;
            const bool solo = gid < (u32)SOLOG, mine = lane >= first && lane < first + n, sent = gid < (u32)(NRAW / 4) && first < 4;
            const u32* f = (const u32*)(P.me + WS_FLAG + (call & 1) * FLAG_PAR + (size_t)q * FLAG_SRC + (gid + (sent && (lane & 8) ? GM : 0)) * 32) + (lane & 7);
            const u64 t0 = now(); u32 val; bool dead = false;
            for (;;) {
                val = __hip_atomic_load(f, __ATOMIC_RELAXED, __HIP_MEMORY_SCOPE_SYSTEM);
                if (__all((val >> 8) == (call & 0xffffffu) || (lane & 7) >= first + n || (solo && (lane & 7) < first))) break;
                if (__any(expired(P, t0, call))) { dead = true; break; }
                __builtin_amdgcn_s_sleep(1);
            }
            if (dead) {
                const u32 t = gid - GY;
                if (lane == 0 && t < (u32)(16 * PK)) *y_base(P, t & 15, global_panel(t >> 4, q), first, 0u) = 0x7fc0;
                break;
            }
            u32 code = val & 255u; if (code > 127u) code = RAWCODE;
            const u32 cb = chunk_bytes(code) - (sent ? 8192u : 0u), sz = solo ? (mine ? (cb + 4095u) & ~4095u : 0u) : (lane < first + n ? cb : 0u);
            u32 inc = sz; inc += dpp<0x111, 0xf, true>(inc); inc += dpp<0x112, 0xf, true>(inc); inc += dpp<0x114, 0xf, true>(inc);
            const u32 a0 = (u32)__builtin_amdgcn_readlane((int)(inc - sz), (int)first) & ~127u, total = (u32)__builtin_amdgcn_readlane((int)inc, 7) - a0;
            if (mine) lds64vw(D_DESC + (nch + lane - first) * 8, (u32x2){((basek * 1024 + inc - sz - a0) << 4) | code, gid * 8 + lane});
            if (solo) { if (mine) lds64vw(D_UNIT + (nun + lane - first) * 8, (u32x2){gid * (u32)GRP_STRIDE + lane * RAWB, cb | ((nch + lane - first) << 18)}); nun += n; }
            else { if (lane == 0) lds64vw(D_UNIT + nun * 8, (u32x2){gid * (u32)GRP_STRIDE + a0, total | ((nch + n - 1) << 18)}); nun += 1; }
            nch += n;
            WAIT_LGKM0();
            lds32vw(CTL_NUNIT, nun); lds32vw(CTL_NCHUNK, nch);
            basek += ((total + 4095) >> 12) << 2;
            STAMP(P, umin(ui, 127));
        }
        lds32vw(CTL_NUNIT, nun | DONE); lds32vw(CTL_NCHUNK, nch | DONE);
    } else {
        const u32 dw = wid - 2, scr = D_SCR + dw * D_SCRB;
        u64 tk = 0, c_wait = 0, c_dec = 0, c_st = 0; (void)c_wait; (void)c_dec; (void)c_st; TICK(tk);
        for (u32 c = dw;; c += ND) {
            u32 nc = rfl(lds32v(CTL_NCHUNK));
            while (nc <= c) { __builtin_amdgcn_s_sleep(2); nc = rfl(lds32v(CTL_NCHUNK)); }
            if ((nc & ~DONE) <= c) break;
            CBAR();
            const u32x2 de = lds64v(D_DESC + c * 8);
            const u32 d = rfl(de.x), code = d & 255u, S = (d >> 8) << 4, gid = rfl(de.y) >> 3, w = rfl(de.y) & 7;
            const u32 SM = S + chunk_bytes(code) - 8192;
            while (rfl(lds32v(CTL_LANDED)) < (SM + 1023) >> 10) __builtin_amdgcn_s_sleep(1);
            CBAR();
            u32 Mr[32], Er[32];
            CS_DECL
            ACC(c_wait, tk);
            unpack_exp(S, code, scr, lane, Er CS_PASS);
            ACC(c_dec, tk);
            if (gid < (u32)(NRAW / 4) && w < 4) {
                const char* m = P.me + WS_INB + (size_t)(((call & 1) * 8 + q) * 64 + gid * 4 + w) * 8192 + lane * 16;
                u32x4 r[8];
                asm volatile("global_load_dwordx4 %0, %8, off sc0 sc1\n\tglobal_load_dwordx4 %1, %8, off offset:1024 sc0 sc1\n\t"
                             "global_load_dwordx4 %2, %8, off offset:2048 sc0 sc1\n\tglobal_load_dwordx4 %3, %8, off offset:3072 sc0 sc1\n\t"
                             "global_load_dwordx4 %4, %9, off sc0 sc1\n\tglobal_load_dwordx4 %5, %9, off offset:1024 sc0 sc1\n\t"
                             "global_load_dwordx4 %6, %9, off offset:2048 sc0 sc1\n\tglobal_load_dwordx4 %7, %9, off offset:3072 sc0 sc1\n\ts_waitcnt vmcnt(0)"
                             : "=&v"(r[0]), "=&v"(r[1]), "=&v"(r[2]), "=&v"(r[3]), "=&v"(r[4]), "=&v"(r[5]), "=&v"(r[6]), "=&v"(r[7]) : "v"(m), "v"(m + 4096) : "memory");
#pragma unroll
                for (int j = 0; j < 8; ++j) { Mr[4 * j] = r[j].x; Mr[4 * j + 1] = r[j].y; Mr[4 * j + 2] = r[j].z; Mr[4 * j + 3] = r[j].w; }
            } else {
                while (rfl(lds32v(CTL_LANDED)) < (SM + 9215) >> 10) __builtin_amdgcn_s_sleep(1);
                CBAR();
                ring_read8k(SM + lane * 16, Mr);
            }
            WAIT_LGKM0(); lds32vw(CTL_NEXT + dw * 4, c + ND); CBAR();
            ACC(c_wait, tk);
            if (gid >= (u32)GY) {
                const u32 t = gid - GY;
                u16* o = y_base_lin(P, t & 15, global_panel(t >> 4, q), w, lane);
#pragma unroll
                for (int p = 0; p < 16; ++p) st_out<GAR_OST>(o + y_piece_off(p), piece(Mr, Er, p));
            } else store_image(P, gid, w, q, lane, Mr, Er);
            ACC(c_st, tk);
        }
        DUMP(P, wid * 4, c_wait); DUMP(P, wid * 4 + 1, c_dec); DUMP(P, wid * 4 + 2, c_st); DUMP(P, wid * 4 + 3, now());
    }
}

__device__ __forceinline__ void pack_inputs(const Params& P, u32 gid, u32 w, bool solo, u32 wid, u32 lane) {
    asm volatile("" : "+v"(lane));
    u32 v[64], Mr[32], Er[32];
    u32 peers = 255u & ~(1u << P.rank); bool own = true;
    if (gid < (u32)GW) load_x(P, gid, w, lane, v);
    else {
        const u32 j = (gid - GW) >> 1, ktl = (gid & 1) * 4 + (w >> 1);
        if (j < (u32)(NR * DP)) { const u32 dest = j / DP; own = dest == (u32)P.rank; peers = own ? 0u : 1u << dest; }
        else if (j < (u32)(NR * (DP + FX))) own = (j & 7) != P.rank;
        const u16* s = P.w + (size_t)(ktl * 64 + (lane >> 4)) * NN + j * 256 + (w & 1) * 128 + (lane & 15) * 8;
        const u32 scr = wid * 16384;
#pragma unroll
        for (int i = 0; i < 16; ++i) glds_w(s + (size_t)i * 4 * NN, scr + i * 1024);
        WAIT_VM(0);
        const u32 ta = scr + ((lane >> 4) * 8 + ((lane & 15) >> 2)) * 256 + (lane & 3) * 16;
#pragma unroll
        for (int p = 0; p < 16; ++p) {
            const u32 a = ta + (p & 1) * 32 * 256 + (p >> 2) * 64 + ((p >> 1) & 1) * 8;
            const u32x2 lo = __builtin_bit_cast(u32x2, __builtin_amdgcn_ds_read_tr16_b64_v4i16((LAS s16x4*)a));
            const u32x2 hi = __builtin_bit_cast(u32x2, __builtin_amdgcn_ds_read_tr16_b64_v4i16((LAS s16x4*)(a + 4 * 256)));
            v[4 * p] = lo.x; v[4 * p + 1] = lo.y; v[4 * p + 2] = hi.x; v[4 * p + 3] = hi.y;
        }
        WAIT_LGKM0();
    }
    const u32 call = call_take(wid);
    if (peers) pack_group(P, v, Mr, Er, gid, w, solo, peers, call, wid, lane); else split(v, Mr, Er);
    if (own) store_image(P, gid, w, P.rank, lane, Mr, Er);
}

typedef __bf16 bf16x2 __attribute__((ext_vector_type(2)));
typedef float f32x2 __attribute__((ext_vector_type(2)));
__device__ __forceinline__ u32 cvt_pk(float lo, float hi) { return __builtin_bit_cast(u32, __builtin_convertvector((f32x2){lo, hi}, bf16x2)); }
__device__ __forceinline__ void wait_panels(const Params& P, u32 tile, u32 upto, u32 call, u32 wid, u32 lane) {
    if (wid == 0) {
        const u32 st = umin(lane & 7, upto), need = call * (st == 0 ? 16u : 28u);
        const u32* c = counter(P, (lane & 8) ? 16 + (tile >> 4) : (tile & 15)) + st;
        const u64 t0 = now();
        for (;;) {
            const u32 x = __hip_atomic_load(c, __ATOMIC_RELAXED, __HIP_MEMORY_SCOPE_AGENT);
            if (__all((int)(x - need) >= 0)) break;
            if (__any(expired(P, t0, call))) { asm volatile("ds_write_b32 %0, %1\n\ts_waitcnt lgkmcnt(0)" :: "v"(CTL_BAD), "v"(1u) : "memory"); break; }
            __builtin_amdgcn_s_sleep(4);
        }
        __builtin_amdgcn_fence(__ATOMIC_ACQUIRE, "agent");
        WAIT_VM(0);
    }
    BAR();
}
__device__ __forceinline__ void gemm_role(const Params& P, unsigned char* lds_generic, u32 place, u32 call, u32 wid, u32 lane) {
    LAS unsigned char* lds = (LAS unsigned char*)lds_generic;
    const u32 tid = threadIdx.x;
    const u32 wr = wid >> 2, wc = wid & 3;
    constexpr int HTB = 16384;
    const u32 voff = tid * 16, ldsw = wid * 1024;
    const u32 fo = lane * 16;
    const u32 aoff = wr * 8192 + fo, boff = wc * 4096 + fo;
#define G_SA(b, h) (((b) * 2 + (h)) * HTB)
#define G_SB(b, h) ((4 + (b) * 2 + (h)) * HTB)
#define G_STAGE(bufoff, gbase) do { _Pragma("unroll") for (int _i = 0; _i < 2; ++_i) \
        __builtin_amdgcn_global_load_lds((const u32*)((const char*)(gbase) + voff + _i * 8192), (LAS u32*)(lds + (bufoff) + ldsw + _i * 8192), 16, 0, (bufoff) >= 4 * HTB ? GAR_BAUX : 0); } while (0)
#define G_LDA(dst, b, h) do { _Pragma("unroll") for (int m = 0; m < 4; ++m) _Pragma("unroll") for (int k = 0; k < 2; ++k) dst[m][k] = *(const LAS bf16x8*)(lds + G_SA(b, h) + aoff + m * 2048 + k * 1024); } while (0)
#define G_LDB(dst, b, h) do { _Pragma("unroll") for (int n = 0; n < 2; ++n) _Pragma("unroll") for (int k = 0; k < 2; ++k) dst[n][k] = *(const LAS bf16x8*)(lds + G_SB(b, h) + boff + n * 2048 + k * 1024); } while (0)
#define G_MMA(ai, bj, At, Bt) do { __builtin_amdgcn_s_setprio(1); _Pragma("unroll") for (int m = 0; m < 4; ++m) _Pragma("unroll") for (int n = 0; n < 2; ++n) _Pragma("unroll") for (int k = 0; k < 2; ++k) \
        acc[ai][bj][m][n] = __builtin_amdgcn_mfma_f32_16x16x32_bf16(Bt[n][k], At[m][k], acc[ai][bj][m][n], 0, 0, 0); __builtin_amdgcn_s_setprio(0); } while (0)
#define G_WV(n) asm volatile("s_waitcnt vmcnt(" #n ")" ::: "memory")
#define G_WL(n) asm volatile("s_waitcnt lgkmcnt(" #n ")" ::: "memory")
#define G_BAR __builtin_amdgcn_s_barrier()
#define G_SCHED __builtin_amdgcn_sched_barrier(0)
    const char* XI = P.me + WS_XI; const char* WI = P.me + WS_WI;
    constexpr size_t kstep = 32768, hstep = 16384;
    const u32x2 cl = lds64v(TAB_COLS + ((blockIdx.x & 4) * 2 + (place >> 2)) * 8);
    const u64 mycols = (u64)rfl(cl.y) << 32 | rfl(cl.x); const u32 pm = (blockIdx.x & 3) * 4 + (place & 3);
    u32 cur = pm | ((u32)mycols & 255u) << 4;
    bool fresh = true;
    bf16x8 At[4][2], B0[2][2], B1[2][2];
    for (u32 ui = 0;; ++ui) {
        const u32 nxt = pm | ((u32)(mycols >> (8 * ui + 8)) & 255u) << 4;
        const bool has_next = (nxt >> 4) != 255u, dist = (cur >> 4) < (u32)PK, chain = has_next && !dist;
        const char* cA = XI + (size_t)(cur & 15) * PANEL; const char* cB = WI + (size_t)(cur >> 4) * PANEL;
        if (fresh) {
            wait_panels(P, cur, ui ? 4u : 0u, call, wid, lane);
            STAMP(P, 8 + ui * 4); if (ui == 0) DUMP(P, 1, __builtin_amdgcn_s_memtime());
            G_STAGE(G_SB(0, 0), cB); G_STAGE(G_SB(0, 1), cB + hstep); G_STAGE(G_SA(0, 0), cA); G_STAGE(G_SA(0, 1), cA + hstep);
            if (wr == 1) G_BAR;
            G_WV(2); G_BAR;
            G_STAGE(G_SB(1, 0), cB + kstep); G_STAGE(G_SA(1, 0), cA + kstep); G_STAGE(G_SB(1, 1), cB + hstep + kstep);
            G_WV(6); G_BAR;
        }
        f32x4 acc[2][2][4][2];
#pragma unroll
        for (int a = 0; a < 2; ++a)
#pragma unroll
            for (int b = 0; b < 2; ++b)
#pragma unroll
                for (int m = 0; m < 4; ++m)
#pragma unroll
                    for (int n = 0; n < 2; ++n) acc[a][b][m][n] = (f32x4){0.f, 0.f, 0.f, 0.f};
        const char* nA = chain ? XI + (size_t)(nxt & 15) * PANEL : cA; const char* nB = chain ? WI + (size_t)(nxt >> 4) * PANEL : cB;
        for (int t = 0; t < 64; t += 2) {
            const bool last = (t == 62);
            const char* a1 = cA + (size_t)(t + 1) * kstep;
            const char* a2 = last ? nA : cA + (size_t)(t + 2) * kstep; const char* b2 = last ? nB : cB + (size_t)(t + 2) * kstep;
            const char* a3 = a2 + kstep; const char* b3 = b2 + kstep;
            {
                const bool staged = ui == 0 && (t == 6 || t == 20 || t == 34 || t == 48);
                if (staged || (last && chain)) wait_panels(P, staged ? cur : nxt, t == 6 ? 1u : t == 20 ? 2u : t == 34 ? 3u : 4u, call, wid, lane);
                if (staged) STAMP(P, t == 6 ? 4 : t == 20 ? 5 : t == 34 ? 6 : 7);
            }
            G_LDB(B0, 0, 0); G_LDB(B1, 0, 1); G_SCHED; G_LDA(At, 0, 0); G_STAGE(G_SA(1, 1), a1 + hstep);
            G_WV(8); G_WL(0); G_BAR; G_MMA(0, 0, At, B0); G_MMA(0, 1, At, B1); G_BAR; G_SCHED;
            G_LDA(At, 0, 1); G_STAGE(G_SB(0, 0), b2); G_STAGE(G_SB(0, 1), b2 + hstep); G_STAGE(G_SA(0, 0), a2);
            G_WV(8); G_WL(0); G_BAR; G_MMA(1, 0, At, B0); G_MMA(1, 1, At, B1); G_BAR; G_SCHED;
            G_LDB(B0, 1, 0); G_LDB(B1, 1, 1); G_SCHED; G_LDA(At, 1, 0); G_STAGE(G_SA(0, 1), a2 + hstep);
            G_WV(8); G_WL(0); G_BAR; G_MMA(0, 0, At, B0); G_MMA(0, 1, At, B1); G_BAR; G_SCHED;
            G_LDA(At, 1, 1); G_STAGE(G_SB(1, 0), b3); G_STAGE(G_SB(1, 1), b3 + hstep); G_STAGE(G_SA(1, 0), a3);
            G_WV(8); G_WL(0); G_BAR; G_MMA(1, 0, At, B0); G_MMA(1, 1, At, B1); G_BAR; G_SCHED;
        }
        STAMP(P, 9 + ui * 4); DUMP(P, 2 + ui, __builtin_amdgcn_s_memtime());
        if (!chain) G_WV(0);
        if (GAR_ALIGN && wr == 0) G_BAR;
        u32 v[64];
        {
            int tz = threadIdx.x; asm volatile("" : "+v"(tz));
            const u32 ln = tz & 63;
            u32 bad; asm volatile("ds_read_b32 %0, %1\n\ts_waitcnt lgkmcnt(0)" : "=v"(bad) : "v"(CTL_BAD) : "memory");
            u16* o = y_base(P, cur & 15, global_panel(cur >> 4, P.rank), wid, ln);
#pragma unroll
            for (int p = 0; p < 16; p += 2) {
                const int ai = p >> 3, m = (p >> 1) & 3;
#pragma unroll
                for (int i = 0; i < 4; ++i) {
                    const f32x4 xa = acc[ai][0][m][i >> 1], xb = acc[ai][1][m][i >> 1];
                    u32 a = cvt_pk(xa[(i & 1) * 2], xa[(i & 1) * 2 + 1]), b = cvt_pk(xb[(i & 1) * 2], xb[(i & 1) * 2 + 1]);
                    if (bad) a = b = 0x7fc07fc0u;
                    v[4 * p + i] = (u32)__builtin_amdgcn_update_dpp((int)a, (int)b, 0x128, 0xf, 0xc, false);
                    v[4 * p + 4 + i] = (u32)__builtin_amdgcn_update_dpp((int)b, (int)a, 0x128, 0xf, 0x3, false);
                }
                st_out<GAR_GST>(o + y_piece_off(p), (u32x4){v[4 * p], v[4 * p + 1], v[4 * p + 2], v[4 * p + 3]});
                st_out<GAR_GST>(o + y_piece_off(p + 1), (u32x4){v[4 * p + 4], v[4 * p + 5], v[4 * p + 6], v[4 * p + 7]});
            }
            if (chain) { if (GAR_ALIGN && wr == 1) G_BAR; }
            else {
                if (!GAR_ALIGN && wr == 0) G_BAR;
                BAR();
                if (dist) {
                    const u32 scr = wid * 16384, to = scr + ((ln & 7) * 8 + (((ln & 8) / 2 + (ln >> 4)) ^ (ln & 7))) * 16, from = scr + (ln ^ (ln >> 3)) * 16;
#pragma unroll
                    for (int p = 0; p < 16; ++p) lds128w(to + p * 1024, (u32x4){v[4 * p], v[4 * p + 1], v[4 * p + 2], v[4 * p + 3]});
                    CBAR();
#pragma unroll
                    for (int p = 0; p < 16; ++p) { const u32x4 t = lds128(from + p * 1024); v[4 * p] = t.x; v[4 * p + 1] = t.y; v[4 * p + 2] = t.z; v[4 * p + 3] = t.w; }
                    WAIT_LGKM0();
                    u32 Mr[32], Er[32]; pack_group(P, v, Mr, Er, GY + cur, wid, false, (cur >> 4) < (u32)DP ? 255u & ~(1u << P.rank) : 1u << (((cur >> 4) - DP) & 7), call, wid, ln); }
                STAMP(P, 10 + ui * 4);
                if (!has_next) break;
            }
        }
        fresh = !chain; cur = nxt;
    }
}

__global__ void __launch_bounds__(512, 2) fused(const Args A) {
    extern __shared__ __attribute__((aligned(16))) unsigned char lds[];
    const u32 wid = rfl(threadIdx.x >> 6), lane = threadIdx.x & 63;
    const u32 role = pin((A.role[blockIdx.x >> 1] >> ((blockIdx.x & 1) * 16)) & 0xffffu);
    char* peer = pin(A.ws[0]);
#pragma unroll
    for (u32 j = 1; j < NR; ++j) { char* c = pin(A.ws[j]); peer = (role & 7) == j ? c : peer; }
    const Params P = {pin(A.me), peer, pin(A.x), pin(A.w), pin(A.out), pin((u32)A.rank), pin((u32)A.nunits)};
    call_load(P, wid, lane);
    if (wid == 1 || wid == 2) {
        const char* ka = (const char*)(const GAS char*)(u64)__builtin_amdgcn_kernarg_segment_ptr() + lane * 16;
        if (wid == 1 && !(role & 0x8000u)) { if (lane < 4) glds_plain(ka, LDS_TAB); }
        else if ((wid == 1) == ((role & 0x8000u) != 0)) { glds_plain(ka, LDS_TAB); glds_plain(ka + 1024, LDS_TAB + 1024); glds_plain(ka + 2048, LDS_TAB + 2048); }
    }
    if (role & 0x8000u) {
        if (threadIdx.x == 0) { lds32vw(CTL_NUNIT, 0u); lds32vw(CTL_LANDED, 0u); lds32vw(CTL_NCHUNK, 0u); }
        if (wid == 1 && lane < 8) lds32vw(CTL_NEXT + lane * 4, lane);
        WAIT_LGKM0(); BAR();
        puller(P, role, wid, lane); return;
    }
    STAMP(P, 0); DUMP(P, 0, __builtin_amdgcn_s_memtime());
    if (threadIdx.x == 0) lds32vw(CTL_BAD, 0u);
    pack_init(wid, lane);
    const u32 sc = role & 255u;
    if (sc >= (u32)(8 * SOLOG)) { const u64 t0 = now(); while (now() - t0 < GAR_HOLD) __builtin_amdgcn_s_sleep(2); }
    for (u32 it = 0;; ++it) {
        const u32 rb = NENC - 1 - (role & 255u), e = it > 1 ? NENC + rb + (it - 2) * NLIGHT : rb;
        if (it > 1 && (e >= (u32)NPACK || rb >= (u32)NLIGHT)) break;
        if (it || (wid == 1 && sc < (u32)(8 * SOLOG))) pack_inputs(P, it ? tab16(TAB_ENC, e) : solo_gid(sc), it ? wid : solo_w(sc), !it, wid, lane);
        WAIT_VM(0); WAIT_LGKM0(); BAR(); STAMP(P, it ? umin(it, 2) : 3);
    }
    const u32 call = call_take(wid);
    call_bump(P, wid, lane, call);
    if (role >> 8) gemm_role(P, lds, (role >> 8) - 1, call, wid, lane);
}

#define GAR_T3 0
#define GAR_T2 0
#define GAR_T1 8
#define GAR_SPARE 1
#define GAR_XAHEAD 16
#define GAR_EPOS 18
#define GAR_F3 10
#define GAR_BIG 0
#define GAR_QTAIL 0
constexpr int QTAIL = GAR_QTAIL;
static void build_tables(Args& P) {
    unsigned char* cols = (unsigned char*)P.cols; u16* stream = (u16*)P.stream; u16* enc = (u16*)P.enc; u16* role = (u16*)P.role;
    {
        int peer[2][4], np[2] = {0, 0}, cnt[8] = {}, seen[2] = {0, 0}, n = 0;
        for (int d = 1; d < NR; ++d) { const int q = (P.rank + d) & 7, h = ((0xb2 >> q) ^ (0xb2 >> P.rank)) & 1; peer[h][np[h]++] = q; }
        for (int b = 0; b < NWG; ++b) {
            const int h = (b >> 2) & 1, i = b >> 3, g0 = 32 - 4 * (h ? CH : CL), near = h ? 4 * NH : 3 * NL, far = h ? 3 * FL : 4 * FH, k = seen[h]++;
            if (k >= near + far) { role[b] = (u16)(n++ | (i >= g0 ? i - g0 + 1 : 0) << 8); continue; }
            const int side = k < near ? h : !h, j = k < near ? k : k - near, q = peer[side][side ? (j + j / 4) & 3 : j % 3];
            role[b] = (u16)(0x8000 | q | cnt[q]++ << 3);
        }
        for (int b = 0; b < NWG; ++b) if (role[b] & 0x8000) role[b] |= (u16)(cnt[role[b] & 7] << 7);
    }
    int col[LC], nc = 0;
    {
        const int rest = LC - (ROUNDS - 1) * CPR, r0 = (rest + 1) / 2; const bool spare = GAR_SPARE && PK - FX <= 3 * (CPR - rest);
        int nd = 0, nr = PK;
        for (int i = 0; i < 128; ++i) cols[i] = 0xff;
        for (int r = 0; r < ROUNDS; ++r) for (int c = 0; c < 8; ++c) for (int cg = 0; cg < 2; ++cg) {
            const bool heavy = c < (cg ? rest - r0 : r0);
            if (nd >= DP && nd < PK && ((nd - DP) & 7) == P.rank) ++nd;
            if (c < (cg ? CH : CL) && (r + 1 < ROUNDS || heavy)) cols[(cg * 8 + c) * 8 + r] = (unsigned char)(col[nc++] = nd < PK && !(spare && heavy) ? nd++ : nr++);
        }
    }
    auto wgid = [&](int jj, int dest, int h) { return GW + 2 * (jj >= DP ? jj + (NR - 1) * DP : dest * DP + jj) + h; };
    int k = 0;
    auto put = [&](int gid, int lv) { for (int j = 0; j < (1 << lv); ++j) stream[k++] = (u16)(gid | ((1 << lv) + j) << 10); };
    auto stage = [&](int st, int i0, int i1) {
        const int h = st >> 1, half = st & 1;
        for (int i = i0; i < i1; ++i) {
            const int gid = i < 16 ? GX + 16 * h + i : wgid(col[i - 16], P.rank, h);
            if ((i < 16 && h == 0) || i >= 16 + CPR - QTAIL) { for (int j = 0; j < 2; ++j) stream[k++] = (u16)(gid | (4 + 2 * half + j) << 10); }
            else if ((GAR_BIG >> (1 + h)) & 1) { if (!half) stream[k++] = (u16)(gid | 1 << 10); }
            else stream[k++] = (u16)(gid | (2 + half) << 10);
        }
    };
    stage(0, NRAW / 4, 16); stage(1, 0, GAR_XAHEAD);
    {
        u16 t[64]; const int n = k; int e = 0;
        for (int i = 0; i < n; ++i) t[i] = stream[i];
        k = 0;
        for (int i = 0; i < n || e < NRAW / 4; ++i) { if (i < n) stream[k++] = t[i]; if (i >= GAR_EPOS && e < NRAW / 4) stream[k++] = (u16)((GX + e++) | 2 << 10); }
    }
    stage(0, 16, 16 + CPR); stage(1, GAR_XAHEAD, 16 + CPR); stage(2, 0, 16 + CPR); stage(3, 0, 16 + CPR);
    for (int i = CPR; i < LC; ++i) for (int h = 0; h < 2; ++h) put(wgid(col[i], P.rank, h), i < 2 * CPR && !(GAR_BIG & 1) ? 1 : 0);
    for (int t = 0; t < 16 * DP; ++t) { const int left = 16 * DP - t; put(GY + t, left <= GAR_T3 ? 3 : left <= GAR_T2 ? 2 : left <= GAR_T1 ? 1 : 0); }
    const int nown = k;
    for (int t = 0; t < 16 * FX; ++t) put(GY + (DP + (t >> 4) * 8 + P.rank) * 16 + (t & 15), 16 * FX - t <= GAR_F3 ? 3 : 16 * FX - t <= GAR_F3 + 14 ? 2 : 1);
    P.nunits = nown | (k - nown) << 16;
    k = 0;
    for (int i = 0; i < CPR; ++i) for (int d = 1; d < (col[i] < DP ? NR : 2); ++d) enc[k++] = (u16)wgid(col[i], (P.rank + d) & 7, 0);
    for (int i = SOLOG; i < 32; ++i) enc[k++] = (u16)(GX + i);
    for (int i = 0; i < CPR; ++i) { if (col[i] < DP) enc[k++] = (u16)wgid(col[i], P.rank, 0); enc[k++] = (u16)wgid(col[i], P.rank, 1); }
    for (int i = 0; i < CPR; ++i) if (col[i] < DP) for (int d = 1; d < NR; ++d) enc[k++] = (u16)wgid(col[i], (P.rank + d) & 7, 1);
    for (int f = 0; f < FX; ++f) for (int h = 0; h < 2; ++h) enc[k++] = (u16)wgid(DP + f * 8 + P.rank, 0, h);
    for (int i = CPR; i < LC; ++i) for (int h = 0; h < 2; ++h) for (int d = 0; d < (col[i] < DP ? NR : 1); ++d) enc[k++] = (u16)wgid(col[i], (P.rank + d) & 7, h);
}
}

extern "C" void kernel_launch(int rank, int nranks, ncclComm_t comm, hipStream_t stream,
                              void* const* d_in, const int* in_sizes, int n_in,
                              void* d_out, int out_size, const int* in_dtypes, int out_dtype,
                              void* const* d_out_all, void* const* d_ws_all, size_t ws_bytes) {
    using namespace gar;
    if (nranks != NR || (unsigned)rank >= (unsigned)NR || n_in != 2 || ws_bytes < WS_END || in_sizes[0] != MM * KS || in_sizes[1] != KS * NN || out_size != MM * NN) {
        fprintf(stderr, "kernel_launch: unexpected shapes\n"); return; }
    Args P = {};
    for (int p = 0; p < NR; ++p) P.ws[p] = (char*)d_ws_all[p];
    P.me = P.ws[rank];
    P.x = (const u16*)d_in[0]; P.w = (const u16*)d_in[1]; P.out = (u16*)d_out; P.rank = rank;
    build_tables(P);
    HIP_CHECK(hipFuncSetAttribute((const void*)fused, hipFuncAttributeMaxDynamicSharedMemorySize, LDS_BYTES));
    hipLaunchKernelGGL(fused, dim3(NWG), dim3(512), LDS_BYTES, stream, P);
    HIP_CHECK(hipGetLastError());
}
```

```cpp
#include <hip/hip_runtime.h>
#include <rccl/rccl.h>
#include <cstdio>
#include <cstdint>
#include <cstddef>
#pragma clang diagnostic ignored "-Wint-to-pointer-cast"

#define HIP_CHECK(expr) do { hipError_t _e = (expr); if (_e != hipSuccess) { \
    fprintf(stderr, "HIP error %s at %s:%d\n", hipGetErrorString(_e), __FILE__, __LINE__); \
    return; } } while (0)

namespace gar {
#define LAS __attribute__((address_space(3)))
#define GAS __attribute__((address_space(1)))
typedef unsigned u32;
typedef unsigned short u16;
typedef unsigned long long u64;
typedef unsigned u32x4 __attribute__((ext_vector_type(4)));
typedef unsigned u32x2 __attribute__((ext_vector_type(2)));
typedef short s16x4 __attribute__((ext_vector_type(4)));
typedef short bf16x8 __attribute__((ext_vector_type(8)));
typedef float f32x4 __attribute__((ext_vector_type(4)));

constexpr int NR = 8, MM = 4096, NN = 32768, KS = 512;
#define GAR_DP 10
constexpr int DP = GAR_DP;
constexpr int RP = 128 - NR * DP;
constexpr int LP = DP + RP;
constexpr int NWG = 256;
#define GAR_NL 9
#define GAR_FL 0
#define GAR_NH 8
#define GAR_FH 1
#define GAR_CL 6
#define GAR_WIN 6
constexpr int NL = GAR_NL, FL = GAR_FL, NH = GAR_NH, FH = GAR_FH, CL = GAR_CL, CH = 6, CPR = CL + CH;
constexpr int NPULL = 3 * (NL + FL) + 4 * (NH + FH);
constexpr int WIN = GAR_WIN;
static_assert(3 * NL + 4 * FH + 16 * CL <= 128 && 4 * NH + 3 * FL + 16 * CH <= 128, "roles");
#define GAR_SLP 1
#define GAR_EARLY 2
#define GAR_NRAW 48
#define GAR_SOLOG 16
#define GAR_LEAD 48
#define GAR_HOLD 100
constexpr int SOLOG = GAR_SOLOG, NRAW = GAR_NRAW;
constexpr int LEAD = GAR_LEAD;
static_assert(NRAW <= 52 && NRAW % 4 == 0 && 8 * SOLOG <= NWG - NPULL && SOLOG == 16, "solo chunks");
constexpr int NENC = NWG - NPULL;
constexpr int GX = 0, GW = 32, GY = 288;
#define GAR_FX 2
constexpr int FX = GAR_FX, PK = DP + 8 * FX, LC = LP - FX;
constexpr int NGID = GY + 16 * PK, GM = NGID, NGT = GM + 16;
constexpr int NS = 32 + 2 * LC + 16 * DP + 16 * FX;
constexpr int ROUNDS = (LC + CPR - 1) / CPR;
constexpr int NUMAX = 704;
constexpr int NSTAT = 3;
constexpr int NPACK = 288 - SOLOG;
constexpr u64 DEADLINE = 30000000ull;

constexpr size_t WS_CALL = 0;
constexpr size_t WS_TMO = 32768;
constexpr size_t WS_CNT = 36864;
constexpr size_t WS_CLAIM = 57344;
constexpr size_t WS_FLAG = 65536;
constexpr size_t FLAG_SRC = NGT * 32, FLAG_PAR = 8 * FLAG_SRC;
constexpr size_t WS_DBG = 1048576;
#define GAR_SKEW 0
constexpr size_t GRP_STRIDE = 131072 + GAR_SKEW;
constexpr size_t WS_STG = 2u << 20;
constexpr size_t STG_PAR = (NGID * GRP_STRIDE + 0x1fffff) & ~(size_t)0x1fffff;
constexpr size_t WS_INB = WS_STG + 2 * STG_PAR;
constexpr size_t WS_XI = WS_INB + 2 * 8 * 64 * 8192;
constexpr size_t PANEL = 64 * 32768;
constexpr size_t WS_WI = WS_XI + 16 * PANEL;
constexpr size_t WS_END = WS_WI + (size_t)LP * PANEL;
static_assert(WS_FLAG + 2 * FLAG_PAR <= WS_DBG && NGT <= 1024 && RP >= 8 * FX, "layout");

constexpr u32 LDS_CTL = 131072;
constexpr u32 LDS_LUT = LDS_CTL + 512, LDS_HIST = LDS_CTL + 1024;
constexpr u32 LDS_TAB = LDS_HIST + 4096, D_DESC = LDS_TAB + 3072, LDS_BYTES = D_DESC + NS * 64;
constexpr u32 CTL_CALL = LDS_CTL  , CTL_SIZE = LDS_CTL + 64  , CTL_BAD = LDS_CTL + 128;
constexpr u32 CTL_NUNIT = LDS_CTL + 192, CTL_LANDED = LDS_CTL + 256, CTL_NCHUNK = LDS_CTL + 320, CTL_NEXT = LDS_CTL + 384  ;
constexpr u32 E_L2 = 0, E_L3 = E_L2 + 4160, E_L4 = E_L3 + 2112, E_VAR = E_L4 + 1088, E_END = E_VAR + 2112;
static_assert(E_END <= 10240 && E_L4 % 16 == 0 && E_L3 % 16 == 0 && E_VAR % 16 == 0, "packer scratch");
constexpr u32 CAP2 = 4080, CAP3 = 2032, CAP4 = 1008, CAPV = 2032, FIXB = 10240, RAWB = 16384, RAWCODE = 255;
constexpr u32 RING = 65536, RMASK = RING - 1, ND = 6;
constexpr u32 D_SCR = RING, D_SCRB = 7360, D_L2 = 0, D_L3 = 4160, D_L4 = D_L3 + 2112;
constexpr u32 D_UNIT = D_SCR + ND * D_SCRB, DONE = 0x80000000u;
static_assert(D_L4 + 1088 <= D_SCRB && D_UNIT + (NUMAX + 8 * SOLOG) * 8 <= LDS_CTL && LDS_BYTES <= 163840 && D_DESC % 128 == 0 && NS * 64 >= 16384, "puller LDS");

struct Args {
    char* ws[NR];
    char* me; const u16* x; const u16* w; u16* out;
    int rank, nunits;
    u32 role[NWG / 2];
    u32 stream[NUMAX / 2];
    u32 enc[NENC];
    u32 cols[32];
    u32 pad[(3072 - 104) / 4 - NWG / 2 - NUMAX / 2 - NENC - 32];
};
static_assert(sizeof(Args) == 3072 && ROUNDS < 8, "kernarg");
constexpr u32 TAB_STREAM = LDS_TAB + offsetof(Args, stream), TAB_ENC = LDS_TAB + offsetof(Args, enc), TAB_COLS = LDS_TAB + offsetof(Args, cols);
struct Params { char* me; char* peer; const u16* x; const u16* w; u16* out; u32 rank, nunits; };
template <class T> __device__ __forceinline__ T* pin(T* p) { u64 v = (u64)p; asm volatile("" : "+s"(v)); return (T*)(GAS T*)v; }
__device__ __forceinline__ u32 pin(u32 v) { asm volatile("" : "+s"(v)); return v; }

__device__ __forceinline__ u32 lds32(u32 a) { return *(LAS u32*)a; }
__device__ __forceinline__ u32 lds32v(u32 a) { return *(volatile LAS u32*)a; }
__device__ __forceinline__ void lds32vw(u32 a, u32 x) { *(volatile LAS u32*)a = x; }
__device__ __forceinline__ u32x2 lds64v(u32 a) { return *(volatile LAS u32x2*)a; }
__device__ __forceinline__ void lds64vw(u32 a, u32x2 x) { *(volatile LAS u32x2*)a = x; }
__device__ __forceinline__ u32x4 lds128(u32 a) { return *(LAS u32x4*)a; }
__device__ __forceinline__ void lds128w(u32 a, u32x4 x) { *(LAS u32x4*)a = x; }
__device__ __forceinline__ u32x4 zero4() { u32 z; asm volatile("v_mov_b32 %0, 0" : "=v"(z)); return (u32x4){z, z, z, z}; }
__device__ __forceinline__ void lds_or(u32 a, u32 x) { __hip_atomic_fetch_or((LAS u32*)a, x, __ATOMIC_RELAXED, __HIP_MEMORY_SCOPE_WORKGROUP); }
__device__ __forceinline__ void lds_add(u32 a, u32 x) { __hip_atomic_fetch_add((LAS u32*)a, x, __ATOMIC_RELAXED, __HIP_MEMORY_SCOPE_WORKGROUP); }
__device__ __forceinline__ u32 rfl(u32 x) { return (u32)__builtin_amdgcn_readfirstlane((int)x); }
__device__ __forceinline__ u32 tab16(u32 t, u32 i) { return rfl(*(volatile LAS u16*)(t + i * 2)); }
#define CBAR() asm volatile("" ::: "memory")
#define WAIT_LGKM0() asm volatile("s_waitcnt lgkmcnt(0)" ::: "memory")
#define WAIT_VM(n) asm volatile("s_waitcnt vmcnt(%0)" :: "i"(n) : "memory")
#define BAR() do { CBAR(); __builtin_amdgcn_s_barrier(); CBAR(); } while (0)
#define GAR_SNT ""
#define GAR_INT ""
__device__ __forceinline__ void st_sys(void* p, u32x4 x) { asm volatile("global_store_dwordx4 %0, %1, off sc0 sc1" GAR_SNT "\n\ts_nop 1" :: "v"(p), "v"(x) : "memory"); }
__device__ __forceinline__ void st_dev(void* p, u32x4 x) { asm volatile("global_store_dwordx4 %0, %1, off sc1" GAR_INT "\n\ts_nop 1" :: "v"(p), "v"(x) : "memory"); }
#define GAR_OST 3
#define GAR_GST 3
template <int MODE> __device__ __forceinline__ void st_out(void* p, u32x4 x) {
    if (MODE == 0) *(u32x4*)p = x;
    else if (MODE == 1) __builtin_nontemporal_store(x, (u32x4*)p);
    else if (MODE == 2) st_dev(p, x);
    else if (MODE == 9) asm volatile("" :: "v"(p), "v"(x) : "memory");
    else asm volatile("global_store_dwordx4 %0, %1, off sc1 nt\n\ts_nop 1" :: "v"(p), "v"(x) : "memory");
}
__device__ __forceinline__ void glds_sys(const void* g, u32 dst) {
    u32 keep;
    asm volatile("s_mov_b32 %0, m0\n\ts_mov_b32 m0, %2\n\ts_nop 0\n\tglobal_load_lds_dwordx4 %1, off sc0 sc1\n\ts_mov_b32 m0, %0" : "=&s"(keep) : "v"(g), "s"(dst) : "memory");
}
__device__ __forceinline__ void glds_plain(const void* g, u32 dst) {
    u32 keep;
    asm volatile("s_mov_b32 %0, m0\n\ts_mov_b32 m0, %2\n\ts_nop 0\n\tglobal_load_lds_dwordx4 %1, off\n\ts_mov_b32 m0, %0" : "=&s"(keep) : "v"(g), "s"(dst) : "memory");
}
#define GAR_WNT " nt"
#define GAR_RNT ""
#define GAR_XNT 0
__device__ __forceinline__ void glds_w(const void* g, u32 dst) {
    u32 keep;
    asm volatile("s_mov_b32 %0, m0\n\ts_mov_b32 m0, %2\n\ts_nop 0\n\tglobal_load_lds_dwordx4 %1, off" GAR_WNT "\n\ts_mov_b32 m0, %0" : "=&s"(keep) : "v"(g), "s"(dst) : "memory");
}
__device__ __forceinline__ u64 now() { return __builtin_amdgcn_s_memrealtime(); }
template <int CTRL, int RM, bool BC> __device__ __forceinline__ u32 dpp(u32 v) { return (u32)__builtin_amdgcn_update_dpp(0, (int)v, CTRL, RM, 0xf, BC); }
__device__ __forceinline__ u32 scan_incl(u32 v) {
    v += dpp<0x111, 0xf, true>(v); v += dpp<0x112, 0xf, true>(v); v += dpp<0x114, 0xf, true>(v); v += dpp<0x118, 0xf, true>(v);
    v += dpp<0x142, 0xa, false>(v); v += dpp<0x143, 0xc, false>(v); return v;
}
__device__ __forceinline__ u32 umax(u32 a, u32 b) { return a > b ? a : b; }
__device__ __forceinline__ u32 umin(u32 a, u32 b) { return a < b ? a : b; }
__device__ __forceinline__ u32 wave_max(u32 v) {
    v = umax(v, dpp<0x111, 0xf, true>(v)); v = umax(v, dpp<0x112, 0xf, true>(v)); v = umax(v, dpp<0x114, 0xf, true>(v)); v = umax(v, dpp<0x118, 0xf, true>(v));
    v = umax(v, dpp<0x142, 0xa, false>(v)); v = umax(v, dpp<0x143, 0xc, false>(v)); return (u32)__builtin_amdgcn_readlane((int)v, 63);
}
__device__ __forceinline__ u32 popc(u32 x) { return (u32)__builtin_popcount(x); }
__device__ __forceinline__ u32 perm(u32 hi, u32 lo, u32 sel) { return __builtin_amdgcn_perm(hi, lo, sel); }

#define STAMP(P, slot) do {} while (0)
#define TICK(t) do {} while (0)
#define ACC(sum, t) do {} while (0)
#define DUMP(P, slot, val) do {} while (0)

#define CSTAMP(i) do {} while (0)
#define CS_ARG
#define CS_DECL
#define CS_PASS

__device__ __forceinline__ u32 nzb(u32 z) { return ((z & 0x7f7f7f7fu) + 0x7f7f7f7fu) | z; }
typedef unsigned short u16x2 __attribute__((ext_vector_type(2)));
struct Classifier {
    u32 s1, s2, s3, nb, tlo, thi; bool win;
    __device__ __forceinline__ void init(u32 t) {
        const u32 a = (t >> 8) & 255u, b = (t >> 16) & 255u, c = t >> 24, lo = umin(a, umin(b, c));
        win = umax(a, umax(b, c)) - lo <= 6u;
        const u64 l = (1ull << (((a - lo) & 7u) * 8)) | (2ull << (((b - lo) & 7u) * 8)) | (3ull << (((c - lo) & 7u) * 8));
        tlo = (u32)l; thi = (u32)(l >> 32); nb = ((0u - lo) & 255u) * 0x01000100u;
        s1 = a * 0x01010101u; s2 = b * 0x01010101u; s3 = c * 0x01010101u;
    }
    __device__ __forceinline__ u32 any(u32 e) const {
        const u32 a1 = nzb(e ^ s1), a2 = nzb(e ^ s2), a3 = nzb(e ^ s3);
        return ((~(a1 & a3) >> 7) & 0x01010101u) | ((~(a2 & a3) >> 6) & 0x02020202u);
    }
    __device__ __forceinline__ u32 near(u32 e) const {
        const u16x2 x = __builtin_bit_cast(u16x2, e), n = __builtin_bit_cast(u16x2, nb), top = {0x07ff, 0x07ff};
        u32 sh; asm("v_pk_mad_u16 %0, %1, %2, %3" : "=v"(sh) : "v"(e), "v"(0x01000100u), "s"(nb));
        const u16x2 hi = __builtin_elementwise_min((u16x2)(x + n), top), lo = __builtin_elementwise_min(__builtin_bit_cast(u16x2, sh), top);
        return perm(thi, tlo, perm(__builtin_bit_cast(u32, hi), __builtin_bit_cast(u32, lo), 0x07030501u));
    }
    template <int N> __device__ __forceinline__ void run(const u32* e, u32* c) const {
        if (win) {
#pragma unroll
            for (int i = 0; i < N; ++i) c[i] = near(e[i]);
        } else {
#pragma unroll
            for (int i = 0; i < N; ++i) c[i] = any(e[i]);
        }
    }
};
__device__ __forceinline__ u32 escbits(u32 c) { return ~(c | (c >> 1)) & 0x55555555u; }
struct Appender {
    u32 lo, fill, addr;
    __device__ __forceinline__ void init(u32 off) { lo = 0; fill = (off & 3u) * 8u; addr = off & ~3u; }
    __device__ __forceinline__ void push(u32 k, u32 mk) {
        const u64 t = (u64)k << fill;
        lo |= (u32)t; fill += popc(mk);
        if (fill >= 32u) { lds_or(addr, lo); lo = (u32)(t >> 32); addr += 4; }
        fill &= 31u;
    }
    __device__ __forceinline__ void flush() { if (lo) lds_or(addr, lo); }
};
__device__ __forceinline__ u32 escmask(u32 c) { return perm(0u, 0xffu, c); }
__device__ __forceinline__ u32 selector(u32 mk) { return lds32(__builtin_amdgcn_udot4(mk & 0x20100804u, 0x01010101u, LDS_LUT, false)); }
template <int R> __device__ __forceinline__ u32 pack_level(u32 src, u32 n, u32 t, u32 cdst, u32 edst, u32 cap, u32 lane) {
    Classifier K; K.init(t);
    if (lane < 16) *(LAS unsigned char*)(src + n + lane) = (unsigned char)K.s1;
    u32 ne = 0;
    for (u32 r = 0; r * 1024 < n; r += R) {
        CBAR();
        u32 q[4 * R], c[4 * R], m[4 * R], l[4 * R], cnt = 0;
#pragma unroll
        for (int i = 0; i < R; ++i) { const u32 p0 = (r + i) * 1024 + lane * 16; const u32x4 x = lds128(src + p0); const bool act = p0 < n;
            q[4 * i] = act ? x.x : K.s1; q[4 * i + 1] = act ? x.y : K.s1; q[4 * i + 2] = act ? x.z : K.s1; q[4 * i + 3] = act ? x.w : K.s1; }
        K.run<4 * R>(q, c);
#pragma unroll
        for (int i = 0; i < 4 * R; ++i) { m[i] = escmask(c[i]); l[i] = selector(m[i]); }
#pragma unroll
        for (int i = 0; i < R; ++i) cnt |= ((popc(m[4 * i]) + popc(m[4 * i + 1]) + popc(m[4 * i + 2]) + popc(m[4 * i + 3])) >> 3) << (16 * i);
        const u32 inc = scan_incl(cnt), tot = (u32)__builtin_amdgcn_readlane((int)inc, 63), exc = inc - cnt;
        if (ne + (tot & 0xffffu) + (tot >> 16) > cap) return ~0u;
#pragma unroll
        for (int i = 0; i < R; ++i) {
            Appender A; A.init(edst + ne + (i ? (tot & 0xffffu) + (exc >> 16) : (exc & 0xffffu)));
#pragma unroll
            for (int j = 0; j < 4; ++j) A.push(perm(0u, q[4 * i + j], l[4 * i + j]), m[4 * i + j]);
            A.flush();
            if ((r + i) * 1024 + lane * 16 < n) *(LAS u32*)(cdst + ((r + i) * 64 + lane) * 4) = c[4 * i] | (c[4 * i + 1] << 2) | (c[4 * i + 2] << 4) | (c[4 * i + 3] << 6);
        }
        ne += (tot & 0xffffu) + (tot >> 16);
    }
    CBAR();
    return ne;
}
template <int N> __device__ __forceinline__ u32 rank_ror(u32 key) { return (dpp<0x120 + N, 0xf, true>(key) > key ? 1u : 0u) + rank_ror<N - 1>(key); }
template <> __device__ __forceinline__ u32 rank_ror<0>(u32) { return 0u; }
__device__ __forceinline__ u32 row_or(u32 x) { x |= dpp<0x128, 0xf, true>(x); x |= dpp<0x124, 0xf, true>(x); x |= dpp<0x122, 0xf, true>(x); x |= dpp<0x121, 0xf, true>(x); return rfl(x); }
__device__ __forceinline__ void split(const u32 (&v)[64], u32 (&Mr)[32], u32 (&Er)[32]) {
#pragma unroll
    for (int g = 0; g < 32; ++g) {
        const u32 a = __builtin_amdgcn_alignbit(v[2 * g], v[2 * g], 31), b = __builtin_amdgcn_alignbit(v[2 * g + 1], v[2 * g + 1], 31);
        Er[g] = perm(b, a, 0x07050301u); Mr[g] = perm(b, a, 0x06040200u);
        asm volatile("" : "+v"(Er[g]), "+v"(Mr[g]));
    }
}
__device__ __forceinline__ u32x4 piece(const u32 (&Mr)[32], const u32 (&Er)[32], int p) {
    const u32 a = perm(Er[2 * p], Mr[2 * p], 0x05010400u), b = perm(Er[2 * p], Mr[2 * p], 0x07030602u);
    const u32 c = perm(Er[2 * p + 1], Mr[2 * p + 1], 0x05010400u), d = perm(Er[2 * p + 1], Mr[2 * p + 1], 0x07030602u);
    return (u32x4){__builtin_amdgcn_alignbit(a, a, 1), __builtin_amdgcn_alignbit(b, b, 1), __builtin_amdgcn_alignbit(c, c, 1), __builtin_amdgcn_alignbit(d, d, 1)};
}
template <int R> __device__ __forceinline__ u32 pack_short(u32 src, u32 n, u32 t, u32 cdst, u32 edst, u32 cap, u32 lane) {
    Classifier K; K.init(t);
    if (lane < 16) *(LAS unsigned char*)(src + n + lane) = (unsigned char)K.s1;
    u32 ne = 0;
    for (u32 r = 0; r * 256 < n; r += R) {
        CBAR();
        u32 q[R], c[R], m[R], l[R], cnt = 0;
#pragma unroll
        for (int i = 0; i < R; ++i) { const u32 p0 = (r + i) * 256 + lane * 4, x = lds32(src + p0); q[i] = p0 < n ? x : K.s1; }
        K.run<R>(q, c);
#pragma unroll
        for (int i = 0; i < R; ++i) { m[i] = escmask(c[i]); l[i] = selector(m[i]); cnt |= (popc(m[i]) >> 3) << (16 * i); }
        const u32 inc = scan_incl(cnt), tot = (u32)__builtin_amdgcn_readlane((int)inc, 63), exc = inc - cnt;
        if (ne + (tot & 0xffffu) + (tot >> 16) > cap) return ~0u;
#pragma unroll
        for (int i = 0; i < R; ++i) {
            Appender A; A.init(edst + ne + (i ? (tot & 0xffffu) + (exc >> 16) : (exc & 0xffffu)));
            A.push(perm(0u, q[i], l[i]), m[i]); A.flush();
            u32 x = c[i] << (2 * (lane & 3)); x |= dpp<0xb1, 0xf, true>(x); x |= dpp<0x4e, 0xf, true>(x);
            if ((lane & 3) == 0 && (r + i) * 256 + lane * 4 < n) *(LAS u32*)(cdst + (r + i) * 64 + lane) = x;
        }
        ne += (tot & 0xffffu) + (tot >> 16);
    }
    CBAR();
    return ne;
}
template <int N, int STEP, int OFF> __device__ __forceinline__ void sample(const u32 (&v)[64], u32 hist, u32 lane) {
#pragma unroll
    for (int i = 0; i < N; ++i) lds_add(hist + (((v[i * STEP + OFF] >> ((i & 1) * 16 + 3)) & 0xff0u) | ((lane & 3) * 4)), 1u);
}
template <bool SAMPLED, class Side>
__device__ __forceinline__ u32 pack_chunk(const u32 (&v)[64], u32 (&Mr)[32], u32 (&Er)[32], u32 (&Cr)[8], u32 scr, u32 lane, bool solo, const Side& side CS_ARG) {
    CBAR(); CSTAMP(0);
#define SIDE4(p0) do { if (side.on && !solo) { _Pragma("unroll") for (int p = p0; p < p0 + 4; ++p) side(p, piece(Mr, Er, p)); } } while (0)
    if (side.on) {
#pragma unroll
        for (int p = 0; p < 4; ++p) side(p, (u32x4){v[4 * p], v[4 * p + 1], v[4 * p + 2], v[4 * p + 3]});
    }
    const u32 hist = solo ? scr + 10240 : LDS_HIST; const u32x4 z4 = zero4();
    if (solo) {
#pragma unroll
        for (int j = 0; j < 4; ++j) lds128w(hist + j * 1024 + lane * 16, z4);
        CBAR();
        sample<12, 5, 2>(v, hist, lane);
        if (side.on) {
#pragma unroll
            for (int p = 4; p < 16; ++p) side(p, (u32x4){v[4 * p], v[4 * p + 1], v[4 * p + 2], v[4 * p + 3]});
        }
    }
    if (!SAMPLED) sample<6, 10, 5>(v, hist, lane);
#pragma unroll
    for (int j = 0; j < 10; ++j) lds128w(scr + j * 1024 + lane * 16, z4);
    if (!SAMPLED) { WAIT_LGKM0(); if (!solo) BAR(); }
    CBAR();
    u32 key = 0;
#pragma unroll
    for (int i = 0; i < 4; ++i) { const u32 bin = i * 64 + lane; const u32x4 h = lds128(hist + bin * 16); key = umax(key, ((h.x + h.y + h.z + h.w) << 8) | bin); }
    split(v, Mr, Er);
    CSTAMP(1);
    { const auto r = __builtin_amdgcn_permlane32_swap(key, key, false, false); key = umax(r[0], r[1]); }
    { const auto r = __builtin_amdgcn_permlane16_swap(key, key, false, false); key = umax(r[0], r[1]); }
    const u32 rank = rank_ror<15>(key), lev = (rank * 11u) >> 5, sym = (key & 255u) << (8 * (rank - 3 * lev + 1));
    const u32 T[4] = {row_or(lev == 0 ? sym : 0u), row_or(lev == 1 ? sym : 0u), row_or(lev == 2 ? sym : 0u), row_or(lev == 3 ? sym : 0u)};
    CSTAMP(3);
    SIDE4(4);
    u32 cnt = 0, mk[32], sl[32];
    {
        Classifier K; K.init(T[0]);
        u32 c[32]; K.run<32>(Er, c);
#pragma unroll
        for (int g = 0; g < 32; ++g) { mk[g] = escmask(c[g]); sl[g] = selector(mk[g]); cnt += popc(mk[g]); }
#pragma unroll
        for (int k = 0; k < 8; ++k) Cr[k] = c[4 * k] | (c[4 * k + 1] << 2) | (c[4 * k + 2] << 4) | (c[4 * k + 3] << 6);
    }
    cnt >>= 3;
    const u32 inc = scan_incl(cnt), n2 = (u32)__builtin_amdgcn_readlane((int)inc, 63);
    CSTAMP(4);
    SIDE4(8);
    if (n2 <= CAP2) {
        Appender A; A.init(scr + E_L2 + inc - cnt);
#pragma unroll
        for (int g = 0; g < 32; ++g) A.push(perm(0u, Er[g], sl[g]), mk[g]);
        A.flush();
    }
    SIDE4(12);
    if (n2 > CAP2) return RAWCODE;
    CBAR(); CSTAMP(5);
    const u32 d2 = (n2 + 15) >> 4;
    const u32 n3 = pack_level<2>(scr + E_L2, n2, T[1], scr + E_VAR + 16, scr + E_L3, CAP3, lane);
    CSTAMP(6);
    if (n3 == ~0u) return RAWCODE;
    const u32 d3 = (n3 + 15) >> 4;
    const u32 n4 = pack_short<2>(scr + E_L3, n3, T[2], scr + E_VAR + 16 + 4 * d2, scr + E_L4, CAP4, lane);
    CSTAMP(7);
    if (n4 == ~0u) return RAWCODE;
    const u32 d4 = (n4 + 15) >> 4, b5 = 16 + 4 * (d2 + d3 + d4);
    const u32 n5 = pack_short<1>(scr + E_L4, n4, T[3], scr + E_VAR + 16 + 4 * (d2 + d3), scr + E_VAR + b5, CAPV - b5, lane);
    if (n5 == ~0u) return RAWCODE;
    if (lane == 0) lds128w(scr + E_VAR, (u32x4){T[0] | d2, T[1] | d3, T[2] | d4, T[3]});
    CBAR();
    return (b5 + n5 + 15) >> 4;
}
__device__ __forceinline__ void pack_init(u32 wid, u32 lane) {
    if (wid >= 4) lds128w(LDS_HIST + (wid - 4) * 1024 + lane * 16, zero4());
    if ((wid == 1 || wid == 2) && lane < 16) { u32 sel = 0x0c0c0c0cu, k = 0;
#pragma unroll
        for (u32 b = 0; b < 4; ++b) if ((lane >> b) & 1) { sel = (sel & ~(255u << k)) | (b << k); k += 8; }
        *(LAS u32*)(LDS_LUT + lane * 4) = sel; }
}
__device__ __forceinline__ u32 chunk_bytes(u32 code) { return code == RAWCODE ? RAWB : FIXB + 16 * code; }

__device__ __forceinline__ void pack_store(char* dst, u32 code, const u32 (&Mr)[32], const u32 (&Er)[32], const u32 (&Cr)[8], u32 scr, u32 lane) {
    char* dm = dst + (chunk_bytes(code) - 8192);
#pragma unroll
    for (int j = 0; j < 8; ++j) st_sys(dm + j * 1024, (u32x4){Mr[4 * j], Mr[4 * j + 1], Mr[4 * j + 2], Mr[4 * j + 3]});
    if (code == RAWCODE) {
#pragma unroll
        for (int j = 0; j < 8; ++j) st_sys(dst + j * 1024, (u32x4){Er[4 * j], Er[4 * j + 1], Er[4 * j + 2], Er[4 * j + 3]});
    } else {
        st_sys(dst, (u32x4){Cr[0], Cr[1], Cr[2], Cr[3]}); st_sys(dst + 1024, (u32x4){Cr[4], Cr[5], Cr[6], Cr[7]});
        if (lane < code) st_sys(dst + 2048, lds128(scr + E_VAR + lane * 16));
        if (lane + 64 < code) st_sys(dst + 3072, lds128(scr + E_VAR + 1024 + lane * 16));
    }
}
template <bool SAMPLED, class Side>
__device__ __forceinline__ void pack_group(const Params& P, const u32 (&v)[64], u32 gid, u32 w, bool solo, u32 peers, u32 call, u32 wid, u32 lane, const Side& side, u32*& fp, u32& fv) {
    u32 Mr[32], Er[32], Cr[8];
    const u32 scr = wid * 16384;
    CS_DECL
    if (gid < 64u) STAMP(P, 40 + wid); if (gid >= 288u) STAMP(P, 80 + wid);
    const u32 code = rfl(pack_chunk<SAMPLED>(v, Mr, Er, Cr, scr, lane, solo, side CS_PASS));
    if (gid < 64u) STAMP(P, 48 + wid); if (gid >= 288u) STAMP(P, 88 + wid);
    u32 off = w * RAWB, mine = code;
    if (!solo) {
        if (lane == 0) lds32vw(CTL_SIZE + wid * 4, code | gid << 8);
        for (;;) { mine = lds32v(CTL_SIZE + (lane & 7) * 4); if (__all((mine >> 8) == gid || (lane & 7) > wid)) break; __builtin_amdgcn_s_sleep(1); }
        mine &= 255u; off = 0;
#pragma unroll
        for (u32 j = 0; j < 7; ++j) { const u32 c = (u32)__builtin_amdgcn_readlane((int)mine, j); if (j < wid) off += chunk_bytes(c); }
        if (wid == 7) {
            const u32x4 z4 = zero4();
#pragma unroll
            for (int j = 0; j < 4; ++j) lds128w(LDS_HIST + j * 1024 + lane * 16, z4);
        }
    }
    char* wq = (char*)(GAS char*)__builtin_bit_cast(u64, lds64v(LDS_TAB + (lane >> 3) * 8));
    pack_store(P.me + WS_STG + (call & 1) * STG_PAR + (size_t)gid * GRP_STRIDE + off + lane * 16, code, Mr, Er, Cr, scr, lane);
    if (gid < 64u) STAMP(P, 56 + wid); if (gid >= 288u) STAMP(P, 96 + wid);
    fp = (solo ? (lane & 7) == w : wid == 7) && ((peers >> (lane >> 3)) & 1) ? (u32*)(wq + WS_FLAG + (call & 1) * FLAG_PAR + (size_t)P.rank * FLAG_SRC + gid * 32) + (lane & 7) : nullptr;
    fv = (call << 8) | mine;
    WAIT_LGKM0();
}
__device__ __forceinline__ void raise_flags(u32* fp, u32 fv) { if (fp) __hip_atomic_store((u32*)(GAS u32*)fp, fv, __ATOMIC_RELAXED, __HIP_MEMORY_SCOPE_SYSTEM); }

#define GAR_WINMODE 1
template <bool WRAP> struct Unp {
    u32 sel[4], lo[4], hi[4], of[4];
    __device__ __forceinline__ void fetch(u32 c, u32& off) {
#pragma unroll
        for (int j = 0; j < 4; ++j) {
            const u32 cj = (c >> (2 * j)) & 0x03030303u, mk = escmask(cj), m2 = mk & 0x02020202u;
            if (WRAP || GAR_WINMODE != 2) { const u32 a = off & ~3u; lo[j] = lds32(WRAP ? (a & RMASK) : a); hi[j] = lds32(WRAP ? ((a + 4) & RMASK) : a + 4); of[j] = off; }
            else lo[j] = lds32(off);
            u32 p; asm("v_mad_u32_u24 %0, %1, %2, %3" : "=v"(p) : "v"(m2), "s"(0x808080u), "v"(0x04040404u));
            sel[j] = (mk & p) | cj;
            off += popc(m2);
        }
    }
    __device__ __forceinline__ u32 finish(int j, u32 t) const {
        u32 w;
        if (WRAP || GAR_WINMODE == 0) w = __builtin_amdgcn_alignbyte(hi[j], lo[j], of[j] & 3u);
        else if (GAR_WINMODE == 1) w = __builtin_amdgcn_alignbyte(hi[j], lo[j], of[j]);
        else w = lo[j];
        return perm(w, t, sel[j]);
    }
    __device__ __forceinline__ u32x4 finish4(u32 t) const { return (u32x4){finish(0, t), finish(1, t), finish(2, t), finish(3, t)}; }
};
#define SCHED() __builtin_amdgcn_sched_barrier(0)
template <bool WRAP> __device__ __forceinline__ u32 unpack_round(u32 c, u32 t, u32 src, u32 dst) {
    const u32 cnt = popc(escbits(c)), inc = scan_incl(cnt);
    u32 off = src + inc - cnt;
    Unp<WRAP> u; u.fetch(c, off); SCHED();
    lds128w(dst, u.finish4(t));
    return (u32)__builtin_amdgcn_readlane((int)inc, 63);
}
__device__ __forceinline__ void ring_read8k(u32 a0, u32 (&r)[32]) {
#pragma unroll
    for (int j = 0; j < 8; ++j) { const u32x4 q = lds128((a0 + j * 1024) & RMASK); r[4 * j] = q.x; r[4 * j + 1] = q.y; r[4 * j + 2] = q.z; r[4 * j + 3] = q.w; }
}
__device__ __forceinline__ void unpack_exp(u32 S, u32 code, u32 scr, u32 lane, u32 (&Er)[32] CS_ARG) {
    const u32 a0 = S + lane * 16;
    if (code == RAWCODE) { ring_read8k(a0, Er); return; }
    const u32x4 ca = lds128(a0 & RMASK), cb = lds128((a0 + 1024) & RMASK);
    const u32 V = S + 2048;
    const u32x4 hd = lds128(V & RMASK);
    const u32 t1 = rfl(hd.x), t2 = rfl(hd.y), t3 = rfl(hd.z), t4 = rfl(hd.w), d2 = t1 & 255u, d3 = t2 & 127u, d4 = t3 & 63u;
    u32 c2[4], c3[2];
#pragma unroll
    for (int r = 0; r < 4; ++r) { const u32 i = r * 64 + lane; const u32 x = lds32((V + 16 + 4 * i) & RMASK); c2[r] = i < d2 ? x : 0x55555555u; }
#pragma unroll
    for (int r = 0; r < 2; ++r) { const u32 i = r * 64 + lane; const u32 x = lds32((V + 16 + 4 * (d2 + i)) & RMASK); c3[r] = i < d3 ? x : 0x55555555u; }
    const u32 x4 = lds32((V + 16 + 4 * (d2 + d3 + lane)) & RMASK), c4 = lane < d4 ? x4 : 0x55555555u;
    CSTAMP(1);
    unpack_round<true>(c4, t4, V + 16 + 4 * (d2 + d3 + d4), scr + D_L4 + lane * 16);
    CBAR();
    const u32 s4 = scr + D_L4 + unpack_round<false>(c3[0], t3, scr + D_L4, scr + D_L3 + lane * 16);
    if (d3 > 64) unpack_round<false>(c3[1], t3, s4, scr + D_L3 + 1024 + lane * 16);
    CBAR(); CSTAMP(2);
    const u32 C[8] = {ca.x, ca.y, ca.z, ca.w, cb.x, cb.y, cb.z, cb.w};
    u32 cnt = 0;
#pragma unroll
    for (int k = 0; k < 8; ++k) cnt += popc(escbits(C[k]));
    {
        const u32 n0 = popc(escbits(c2[0])), n1 = popc(escbits(c2[1]));
        const u32 i01 = scan_incl(n0 | (n1 << 16)), tot = (u32)__builtin_amdgcn_readlane((int)i01, 63);
        cnt = scan_incl(cnt) - cnt;
        u32 o0 = scr + D_L3 + (i01 & 0xffffu) - n0, o1 = scr + D_L3 + (tot & 0xffffu) + (i01 >> 16) - n1;
        Unp<false> u0, u1; u0.fetch(c2[0], o0); u1.fetch(c2[1], o1); SCHED();
        lds128w(scr + D_L2 + lane * 16, u0.finish4(t2)); lds128w(scr + D_L2 + 1024 + lane * 16, u1.finish4(t2));
        if (d2 > 128) {
            u32 s3 = scr + D_L3 + (tot & 0xffffu) + (tot >> 16);
            s3 += unpack_round<false>(c2[2], t2, s3, scr + D_L2 + 2048 + lane * 16);
            if (d2 > 192) unpack_round<false>(c2[3], t2, s3, scr + D_L2 + 3072 + lane * 16);
        }
    }
    CBAR(); CSTAMP(3);
    u32 off = scr + D_L2 + cnt;
    Unp<false> u[8];
    u[0].fetch(C[0], off); u[1].fetch(C[1], off);
#pragma unroll
    for (int k = 0; k < 8; ++k) {
        SCHED();
        if (k + 2 < 8) u[k + 2].fetch(C[k + 2], off);
        SCHED();
#pragma unroll
        for (int j = 0; j < 4; ++j) Er[4 * k + j] = u[k].finish(j, t1);
    }
    CBAR();
}
__device__ __forceinline__ u32 x_piece_off(u32 half, int p) { return half * 16384 + p * 1024; }
__device__ __forceinline__ u32 w_piece_off(u32 h, int p) {
    const u32 g32 = p >> 2, n = (p >> 1) & 1, kb = p & 1;
    return (g32 & 1) * 16384 + ((2 * (2 * h + (g32 >> 1)) + n) * 2 + kb) * 1024;
}
__device__ __forceinline__ u16* y_base(const Params& P, u32 pm, u32 j, u32 w, u32 lane) {
    return P.out + (size_t)(pm * 256 + (w >> 2) * 64 + (lane >> 3)) * NN + j * 256 + (w & 3) * 64 + (lane & 7) * 8;
}
__device__ __forceinline__ size_t y_piece_off(int p) { return (size_t)((p >> 3) * 128 + (p & 7) * 8) * NN; }
__device__ __forceinline__ u32 local_panel(u32 j, u32 rank) { return j >= (u32)(NR * DP) ? j - (NR - 1) * DP : j - rank * DP; }
__device__ __forceinline__ u32 global_panel(u32 jj, u32 rank) { return jj >= (u32)DP ? jj + (NR - 1) * DP : jj + rank * DP; }
__device__ __forceinline__ u32* counter(const Params& P, u32 i) { return (u32*)(P.me + WS_CNT + i * 128); }
__device__ __forceinline__ u32* tmo_word(const Params& P) { return (u32*)(P.me + WS_TMO); }
__device__ __forceinline__ bool expired(const Params& P, u64 t0, u32 call) {
    if (__hip_atomic_load(tmo_word(P), __ATOMIC_RELAXED, __HIP_MEMORY_SCOPE_AGENT) == call || now() - t0 > DEADLINE) {
        __hip_atomic_store(tmo_word(P), call, __ATOMIC_RELAXED, __HIP_MEMORY_SCOPE_AGENT); return true; }
    return false;
}

struct ImageStore {
    char* o; u32* cnt; u32 half; bool isx, on;
    __device__ __forceinline__ ImageStore(const Params& P, u32 gid, u32 w, u32 q, u32 lane, bool on_) {
        isx = gid < (u32)GW; half = w & 1; on = on_;
        const u32 pn = isx ? gid & 15 : local_panel((gid - GW) >> 1, P.rank), ktl = (isx ? gid >> 4 : gid & 1) * 4 + (w >> 1);
        const u32 qi = (q - P.rank - 1) & 7, stage = qi == 7 ? 0u : ktl == 7 ? 5u : 1 + (ktl >> 1);
        const u32 slot = stage == 0 ? ktl : 8 + ktl * 7 + qi;
        o = P.me + (isx ? WS_XI : WS_WI) + (size_t)pn * PANEL + (size_t)slot * 32768 + lane * 16;
        cnt = counter(P, isx ? pn : 16 + pn) + stage;
    }
    __device__ __forceinline__ void operator()(int p, u32x4 x) const { st_dev(o + (isx ? x_piece_off(half, p) : w_piece_off(half, p)), x); }
    __device__ __forceinline__ void done(u32 lane) const { WAIT_VM(0); if (lane == 0) __hip_atomic_fetch_add(cnt, 1u, __ATOMIC_RELAXED, __HIP_MEMORY_SCOPE_AGENT); }
};
struct OutStore {
    u16* o; static constexpr bool on = true;
    __device__ __forceinline__ void operator()(int p, u32x4 x) const { st_out<GAR_OST>(o + y_piece_off(p), x); }
};
__device__ __forceinline__ u32* call_word(const Params& P, u32 wid) { return (u32*)(P.me + WS_CALL + blockIdx.x * 128 + wid * 4); }
__device__ __forceinline__ void call_load(const Params& P, u32 wid, u32 lane) {
    u32 keep;
    if (lane == 0) asm volatile("s_mov_b32 %0, m0\n\ts_mov_b32 m0, %2\n\ts_nop 0\n\tglobal_load_lds_dword %1, off sc1\n\ts_mov_b32 m0, %0" : "=&s"(keep) : "v"(call_word(P, wid)), "s"(CTL_CALL + wid * 4) : "memory");
}
__device__ __forceinline__ u32 call_take(u32 wid) { WAIT_VM(0); __builtin_amdgcn_s_waitcnt(0x0f70);   return rfl(lds32v(CTL_CALL + wid * 4)) + 1; }
__device__ __forceinline__ void call_bump(const Params& P, u32 wid, u32 lane, u32 call) { if (lane == 0) __hip_atomic_store(call_word(P, wid), call, __ATOMIC_RELAXED, __HIP_MEMORY_SCOPE_AGENT); }

__device__ __forceinline__ u32 solo_gid(u32 c) { return (c >> 2) & 15; }
__device__ __forceinline__ u32 solo_w(u32 c) { return (c & 3) | (c >> 6) << 2; }
__device__ __forceinline__ void load_x(const Params& P, u32 gid, u32 w, u32 lane, u32 (&v)[64]) {
    const u16* s = P.x + (size_t)((gid & 15) * 256 + (w & 1) * 128 + (lane & 15)) * KS + ((gid >> 4) * 4 + (w >> 1)) * 64 + (lane >> 4) * 8;
#pragma unroll
    for (int p = 0; p < 16; ++p) { const u32x4* a = (const u32x4*)(s + (p >> 1) * 16 * KS + (p & 1) * 32); const u32x4 t = GAR_XNT ? __builtin_nontemporal_load(a) : *a; v[4 * p] = t.x; v[4 * p + 1] = t.y; v[4 * p + 2] = t.z; v[4 * p + 3] = t.w; }
}
__device__ __forceinline__ void puller(const Params& P, u32 role, u32 wid, u32 lane) {
    const u32 q = role & 7, sub = (role >> 3) & 15, PW = (role >> 7) & 15;
    const u32 pc = sub + PW * (7 - wid);
    if (pc < (u32)NRAW) {
        const u32 gid = solo_gid(pc), w = solo_w(pc);
        u32 v[64], Mr[32], Er[32];
        load_x(P, gid, w, lane, v);
        const u32 call = call_take(wid);
        split(v, Mr, Er);
        char* o = P.peer + WS_INB + (size_t)(((call & 1) * 8 + P.rank) * 64 + pc) * 8192 + lane * 16;
#pragma unroll
        for (int j = 0; j < 8; ++j) st_sys(o + j * 1024, (u32x4){Mr[4 * j], Mr[4 * j + 1], Mr[4 * j + 2], Mr[4 * j + 3]});
        WAIT_VM(0); STAMP(P, 120 + wid);
        if (lane == 0) __hip_atomic_store((u32*)(P.peer + WS_FLAG + (call & 1) * FLAG_PAR + (size_t)P.rank * FLAG_SRC + (GM + gid) * 32) + w, call << 8, __ATOMIC_RELAXED, __HIP_MEMORY_SCOPE_SYSTEM);
    }
    const u32 call = call_take(wid);
    call_bump(P, wid, lane, call);
    if (wid == 0) {
        __builtin_amdgcn_s_setprio(3);
        const char* base = P.peer + WS_STG + (call & 1) * STG_PAR;
        const char* dummy = P.me + WS_TMO + 1024 + lane * 16;
        const u32 voff = lane * 16;
        u32 pos = 0, pub = 0, limit = RING / 1024;
        u64 tk = 0, c_full = 0, c_idle = 0, n_full = 0, n_ref = 0; (void)tk; (void)c_full; (void)c_idle; (void)n_full; (void)n_ref;
        for (u32 ui = 0;; ++ui) {
            u32 nu = rfl(lds32v(CTL_NUNIT));
            if ((nu & ~DONE) <= ui) {
                TICK(tk);
                WAIT_VM(0); lds32vw(CTL_LANDED, pos); pub = pos;
                while (nu <= ui) { __builtin_amdgcn_s_sleep(1); nu = rfl(lds32v(CTL_NUNIT)); }
                if ((nu & ~DONE) <= ui) break;
                ACC(c_idle, tk);
            }
            CBAR();
            const u32x2 un = lds64v(D_UNIT + ui * 8);
            const u32 ut = rfl(un.y), total = ut & 0x3ffffu, lastc = ut >> 18;
            const char* src = base + rfl(un.x);
            for (u32 o = 0; o < total; o += 4096) {
                if (pos + 4 > limit) {
                    bool drained = false; TICK(tk); ++n_ref;
                    for (;;) {
                        const u32 nx = lds32v(CTL_NEXT + (lane & 7) * 4);
                        u32 f = lastc;
#pragma unroll
                        for (u32 j = 0; j < ND; ++j) f = umin(f, (u32)__builtin_amdgcn_readlane((int)nx, j));
                        limit = (rfl(lds32v(D_DESC + f * 8)) >> 14) + RING / 1024;
                        if (pos + 4 <= limit) break;
                        if (!drained) { WAIT_VM(0); lds32vw(CTL_LANDED, pos); pub = pos; drained = true; }
                        __builtin_amdgcn_s_sleep(1);
                    }
                    ACC(c_full, tk); n_full += drained;
                }
                const u32 dst = (pos * 1024) & RMASK;
                if (o + 4096 <= total) {
                    u32 keep;
                    asm volatile("s_mov_b32 %0, m0\n\ts_mov_b32 m0, %3\n\ts_nop 4\n\t"
                                 "global_load_lds_dwordx4 %1, %2 sc0 sc1" GAR_RNT "\n\ts_waitcnt vmcnt(%4)\n\t"
                                 "global_load_lds_dwordx4 %1, %2 offset:1024 sc0 sc1" GAR_RNT "\n\ts_waitcnt vmcnt(%4)\n\t"
                                 "global_load_lds_dwordx4 %1, %2 offset:2048 sc0 sc1" GAR_RNT "\n\ts_waitcnt vmcnt(%4)\n\t"
                                 "global_load_lds_dwordx4 %1, %2 offset:3072 sc0 sc1" GAR_RNT "\n\ts_waitcnt vmcnt(%4)\n\t"
                                 "s_mov_b32 m0, %0" : "=&s"(keep) : "v"(voff), "s"(src + o), "s"(dst), "i"(WIN) : "memory");
                } else {
#pragma unroll
                    for (u32 j = 0; j < 4; ++j) { const u32 b = o + j * 1024 + voff; glds_sys(b < total ? src + b : dummy, dst + j * 1024); WAIT_VM(WIN);
                        if (j < 3 && pos + j + 1 > pub + WIN) { pub = pos + j + 1 - WIN; lds32vw(CTL_LANDED, pub); } }
                }
                pos += 4;
                if (pos > pub + WIN) { pub = pos - WIN; lds32vw(CTL_LANDED, pub); }
            }
            STAMP(P, 128 + umin(ui, 127)); DUMP(P, 64 + umin(ui, 127), total);
        }
        DUMP(P, 0, c_full); DUMP(P, 1, n_full); DUMP(P, 2, n_ref); DUMP(P, 3, c_idle);
    } else if (wid == 1) {
        u32 basek = 0, nch = 0, nun = 0;
        u32* ticket = (u32*)(P.me + WS_CLAIM + q * 128);
        const u32 nown = P.nunits & 0xffffu, nall = nown + (P.nunits >> 16);
        u32 tbase = NSTAT * PW - (call - 1) * (nown - NSTAT * PW + PW), tend = nown;
        for (u32 ui = 0;; ++ui) {
            while (rfl(lds32v(CTL_LANDED)) + LEAD < basek) __builtin_amdgcn_s_sleep(GAR_SLP * 4);
            u32 idx = ui * PW + sub;
            if (ui >= (u32)NSTAT) {
                if (lane == 0) idx = __hip_atomic_fetch_add(ticket, 1u, __ATOMIC_RELAXED, __HIP_MEMORY_SCOPE_AGENT);
                idx = rfl(idx) + tbase;
                if (idx >= tend) {
                    if (tend == nall) break;
                    ticket = (u32*)(P.me + WS_CLAIM + P.rank * 128); tbase = nown - (call - 1) * (nall - nown + NPULL); tend = nall;
                    continue;
                }
            }
            const u32 e = tab16(TAB_STREAM, idx), gid = e & 1023u, part = e >> 10, lv = 31 - __builtin_clz(part), n = 8u >> lv, first = (part - (1u << lv)) * n;
            const bool solo = gid < (u32)SOLOG, mine = lane >= first && lane < first + n, sent = gid < (u32)(NRAW / 4) && first < 4;
            const u32* f = (const u32*)(P.me + WS_FLAG + (call & 1) * FLAG_PAR + (size_t)q * FLAG_SRC + (gid + (sent && (lane & 8) ? GM : 0)) * 32) + (lane & 7);
            const u64 t0 = now(); u32 val; bool dead = false;
            for (;;) {
                val = __hip_atomic_load(f, __ATOMIC_RELAXED, __HIP_MEMORY_SCOPE_SYSTEM);
                if (__all((val >> 8) == (call & 0xffffffu) || (lane & 7) >= first + n || (solo && (lane & 7) < first))) break;
                if (__any(expired(P, t0, call))) { dead = true; break; }
                __builtin_amdgcn_s_sleep(1);
            }
            if (dead) {
                const u32 t = gid - GY;
                if (lane == 0 && t < (u32)(16 * PK)) *y_base(P, t & 15, global_panel(t >> 4, q), first, 0u) = 0x7fc0;
                break;
            }
            u32 code = val & 255u; if (code > 127u) code = RAWCODE;
            const u32 cb = chunk_bytes(code) - (sent ? 8192u : 0u), sz = solo ? (mine ? (cb + 4095u) & ~4095u : 0u) : (lane < first + n ? cb : 0u);
            u32 inc = sz; inc += dpp<0x111, 0xf, true>(inc); inc += dpp<0x112, 0xf, true>(inc); inc += dpp<0x114, 0xf, true>(inc);
            const u32 a0 = (u32)__builtin_amdgcn_readlane((int)(inc - sz), (int)first) & ~127u, total = (u32)__builtin_amdgcn_readlane((int)inc, 7) - a0;
            if (mine) lds64vw(D_DESC + (nch + lane - first) * 8, (u32x2){((basek * 1024 + inc - sz - a0) << 4) | code, gid * 8 + lane});
            if (solo) { if (mine) lds64vw(D_UNIT + (nun + lane - first) * 8, (u32x2){gid * (u32)GRP_STRIDE + lane * RAWB, cb | ((nch + lane - first) << 18)}); nun += n; }
            else { if (lane == 0) lds64vw(D_UNIT + nun * 8, (u32x2){gid * (u32)GRP_STRIDE + a0, total | ((nch + n - 1) << 18)}); nun += 1; }
            nch += n;
            WAIT_LGKM0();
            lds32vw(CTL_NUNIT, nun); lds32vw(CTL_NCHUNK, nch);
            basek += ((total + 4095) >> 12) << 2;
            STAMP(P, umin(ui, 127));
        }
        lds32vw(CTL_NUNIT, nun | DONE); lds32vw(CTL_NCHUNK, nch | DONE);
    } else {
        const u32 dw = wid - 2, scr = D_SCR + dw * D_SCRB;
        u64 tk = 0, c_wait = 0, c_dec = 0, c_st = 0; (void)c_wait; (void)c_dec; (void)c_st; TICK(tk);
        for (u32 c = dw;; c += ND) {
            u32 nc = rfl(lds32v(CTL_NCHUNK));
            while (nc <= c) { __builtin_amdgcn_s_sleep(GAR_SLP * 2); nc = rfl(lds32v(CTL_NCHUNK)); }
            if ((nc & ~DONE) <= c) break;
            CBAR();
            const u32x2 de = lds64v(D_DESC + c * 8);
            const u32 d = rfl(de.x), code = d & 255u, S = (d >> 8) << 4, gid = rfl(de.y) >> 3, w = rfl(de.y) & 7;
            const u32 SM = S + chunk_bytes(code) - 8192;
            while (rfl(lds32v(CTL_LANDED)) < (SM + 1023) >> 10) __builtin_amdgcn_s_sleep(GAR_SLP);
            CBAR();
            u32 Mr[32], Er[32];
            CS_DECL
            ACC(c_wait, tk);
            unpack_exp(S, code, scr, lane, Er CS_PASS);
            ACC(c_dec, tk);
            if (gid < (u32)(NRAW / 4) && w < 4) {
                const char* m = P.me + WS_INB + (size_t)(((call & 1) * 8 + q) * 64 + gid * 4 + w) * 8192 + lane * 16;
                u32x4 r[8];
                asm volatile("global_load_dwordx4 %0, %8, off sc0 sc1\n\tglobal_load_dwordx4 %1, %8, off offset:1024 sc0 sc1\n\t"
                             "global_load_dwordx4 %2, %8, off offset:2048 sc0 sc1\n\tglobal_load_dwordx4 %3, %8, off offset:3072 sc0 sc1\n\t"
                             "global_load_dwordx4 %4, %9, off sc0 sc1\n\tglobal_load_dwordx4 %5, %9, off offset:1024 sc0 sc1\n\t"
                             "global_load_dwordx4 %6, %9, off offset:2048 sc0 sc1\n\tglobal_load_dwordx4 %7, %9, off offset:3072 sc0 sc1\n\ts_waitcnt vmcnt(0)"
                             : "=&v"(r[0]), "=&v"(r[1]), "=&v"(r[2]), "=&v"(r[3]), "=&v"(r[4]), "=&v"(r[5]), "=&v"(r[6]), "=&v"(r[7]) : "v"(m), "v"(m + 4096) : "memory");
#pragma unroll
                for (int j = 0; j < 8; ++j) { Mr[4 * j] = r[j].x; Mr[4 * j + 1] = r[j].y; Mr[4 * j + 2] = r[j].z; Mr[4 * j + 3] = r[j].w; }
            } else {
                while (rfl(lds32v(CTL_LANDED)) < (SM + 9215) >> 10) __builtin_amdgcn_s_sleep(1);
                CBAR();
                ring_read8k(SM + lane * 16, Mr);
            }
            WAIT_LGKM0(); lds32vw(CTL_NEXT + dw * 4, c + ND); CBAR();
            ACC(c_wait, tk);
            if (gid >= (u32)GY) {
                const u32 t = gid - GY;
                u16* o = y_base(P, t & 15, global_panel(t >> 4, q), w, lane);
#pragma unroll
                for (int p = 0; p < 16; ++p) st_out<GAR_OST>(o + y_piece_off(p), piece(Mr, Er, p));
            } else {
                const ImageStore im(P, gid, w, q, lane, true);
#pragma unroll
                for (int p = 0; p < 16; ++p) im(p, piece(Mr, Er, p));
                im.done(lane);
            }
            ACC(c_st, tk);
        }
        DUMP(P, wid * 4, c_wait); DUMP(P, wid * 4 + 1, c_dec); DUMP(P, wid * 4 + 2, c_st); DUMP(P, wid * 4 + 3, now());
    }
}

__device__ __forceinline__ void pack_inputs(const Params& P, u32 gid, u32 w, bool solo, u32 wid, u32 lane) {
    asm volatile("" : "+v"(lane));
    u32 v[64];
    u32 peers = 255u & ~(1u << P.rank); bool own = true;
    if (gid < (u32)GW) load_x(P, gid, w, lane, v);
    else {
        const u32 j = (gid - GW) >> 1, ktl = (gid & 1) * 4 + (w >> 1);
        if (j < (u32)(NR * DP)) { const u32 dest = j / DP; own = dest == (u32)P.rank; peers = own ? 0u : 1u << dest; }
        else if (j < (u32)(NR * (DP + FX))) own = (j & 7) != P.rank;
        const u16* s = P.w + (size_t)(ktl * 64 + (lane >> 4)) * NN + j * 256 + (w & 1) * 128 + (lane & 15) * 8;
        const u32 scr = wid * 16384;
#pragma unroll
        for (int i = 0; i < 16; ++i) glds_w(s + (size_t)i * 4 * NN, scr + i * 1024);
        WAIT_VM(0);
        const u32 ta = scr + ((lane >> 4) * 8 + ((lane & 15) >> 2)) * 256 + (lane & 3) * 16;
#pragma unroll
        for (int p = 0; p < 16; ++p) {
            const u32 a = ta + (p & 1) * 32 * 256 + (p >> 2) * 64 + ((p >> 1) & 1) * 8;
            const u32x2 lo = __builtin_bit_cast(u32x2, __builtin_amdgcn_ds_read_tr16_b64_v4i16((LAS s16x4*)a));
            const u32x2 hi = __builtin_bit_cast(u32x2, __builtin_amdgcn_ds_read_tr16_b64_v4i16((LAS s16x4*)(a + 4 * 256)));
            v[4 * p] = lo.x; v[4 * p + 1] = lo.y; v[4 * p + 2] = hi.x; v[4 * p + 3] = hi.y;
        }
        WAIT_LGKM0();
    }
    const u32 call = call_take(wid);
    const ImageStore im(P, gid, w, P.rank, lane, own);
    if (peers) {
        u32* fp; u32 fv;
        pack_group<false>(P, v, gid, w, solo, peers, call, wid, lane, im, fp, fv);
        WAIT_VM(0);
        if (gid < 64u) STAMP(P, 64 + wid);
        if (!solo) BAR();
        raise_flags(fp, fv);
        if (own && lane == 0) __hip_atomic_fetch_add(im.cnt, 1u, __ATOMIC_RELAXED, __HIP_MEMORY_SCOPE_AGENT);
    } else {
#pragma unroll
        for (int p = 0; p < 16; ++p) im(p, (u32x4){v[4 * p], v[4 * p + 1], v[4 * p + 2], v[4 * p + 3]});
        im.done(lane);
    }
}

typedef __bf16 bf16x2 __attribute__((ext_vector_type(2)));
typedef float f32x2 __attribute__((ext_vector_type(2)));
__device__ __forceinline__ u32 cvt_pk(float lo, float hi) { return __builtin_bit_cast(u32, __builtin_convertvector((f32x2){lo, hi}, bf16x2)); }
__device__ __forceinline__ void wait_panels(const Params& P, u32 tile, u32 upto, u32 call, u32 wid, u32 lane) {
    if (wid == 0) {
        const u32 st = umin(lane & 7, upto), need = call * (st == 0 ? 16u : st < 4 ? 28u : 14u);
        const u32* c = counter(P, (lane & 8) ? 16 + (tile >> 4) : (tile & 15)) + st;
        const u64 t0 = now();
        for (;;) {
            const u32 x = __hip_atomic_load(c, __ATOMIC_RELAXED, __HIP_MEMORY_SCOPE_AGENT);
            if (__all((int)(x - need) >= 0)) break;
            if (__any(expired(P, t0, call))) { asm volatile("ds_write_b32 %0, %1\n\ts_waitcnt lgkmcnt(0)" :: "v"(CTL_BAD), "v"(1u) : "memory"); break; }
            __builtin_amdgcn_s_sleep(4);
        }
        __builtin_amdgcn_fence(__ATOMIC_ACQUIRE, "agent");
        WAIT_VM(0);
    }
    BAR();
}
__device__ __forceinline__ void stage2(u32 dst, u32 voff, const char* src) {
    asm volatile("s_mov_b32 m0, %2\n\ts_nop 0\n\tglobal_load_lds_dwordx4 %0, %1\n\tglobal_load_lds_dwordx4 %0, %1 offset:1024" :: "v"(voff), "s"(src), "s"(dst) : "memory");
}
__device__ __forceinline__ void stage4(u32 dst, u32 voff, const char* src) {
    asm volatile("s_mov_b32 m0, %2\n\ts_nop 0\n\tglobal_load_lds_dwordx4 %0, %1\n\tglobal_load_lds_dwordx4 %0, %1 offset:1024\n\t"
                 "global_load_lds_dwordx4 %0, %1 offset:2048\n\tglobal_load_lds_dwordx4 %0, %1 offset:3072" :: "v"(voff), "s"(src), "s"(dst) : "memory");
}
__device__ __forceinline__ void gemm_role(const Params& P, u32 place, u32 call, u32 wid, u32 lane) {
    const u32 wr = wid >> 2, wc = wid & 3;
    u32 aoff = wr * 8192 + lane * 16, boff = 65536 + wc * 4096 + lane * 16;
    const u32 mA = wid * 2048, mB = 65536 + wid * 4096;
    u32 vA0 = mA + lane * 16, vA1 = vA0 + 16384, vA2 = vA0 + 32768, vA3 = vA0 + 49152, vB0 = wid * 4096 + lane * 16, vB1 = vB0 + 32768;
    asm volatile("" : "+v"(vA0), "+v"(vA1), "+v"(vA2), "+v"(vA3), "+v"(vB0), "+v"(vB1), "+v"(aoff), "+v"(boff));
#define G_LDA(b, h) do { _Pragma("unroll") for (int m = 0; m < 4; ++m) _Pragma("unroll") for (int k = 0; k < 2; ++k) At[m][k] = *(const LAS bf16x8*)(aoff + ((b) * 2 + (h)) * 16384 + m * 2048 + k * 1024); } while (0)
#define G_LDB(b) do { _Pragma("unroll") for (int j = 0; j < 2; ++j) _Pragma("unroll") for (int n = 0; n < 2; ++n) _Pragma("unroll") for (int k = 0; k < 2; ++k) \
        Bt[j][n][k] = *(const LAS bf16x8*)(boff + ((b) * 2 + j) * 16384 + n * 2048 + k * 1024); } while (0)
#define G_MMA(ai, STEP) do { G_WV(8); __builtin_amdgcn_s_waitcnt(0xc07f);   G_SCHED; G_BAR; __builtin_amdgcn_s_setprio(1); _Pragma("unroll") for (int i = 0; i < 32; ++i) { \
        const int bj = i >> 4, k = (i >> 3) & 1, m = (i >> 1) & 3, n = i & 1; \
        if (i == 8) { G_SCHED; STEP; G_SCHED; } \
        if (i == 32 - GAR_EARLY) { G_SCHED; __builtin_amdgcn_s_setprio(3); G_BAR; G_SCHED; } \
        acc[ai][bj][m][n] = __builtin_amdgcn_mfma_f32_16x16x32_bf16(Bt[bj][n][k], At[m][k], acc[ai][bj][m][n], 0, 0, 0); } \
        __builtin_amdgcn_s_setprio(0); if (GAR_EARLY == 0) G_BAR; G_SCHED; } while (0)
#define G_WV(n) asm volatile("s_waitcnt vmcnt(" #n ")" ::: "memory")
#define G_BAR __builtin_amdgcn_s_barrier()
#define G_SCHED __builtin_amdgcn_sched_barrier(0)
    const u32x2 cl = lds64v(TAB_COLS + ((blockIdx.x & 4) * 2 + (place >> 2)) * 8);
    const u64 mycols = (u64)rfl(cl.y) << 32 | rfl(cl.x); const u32 pm = (blockIdx.x & 3) * 4 + (place & 3);
    const char* const cA = P.me + WS_XI + (size_t)pm * PANEL; const char* const WI = P.me + WS_WI;
    u32 cur = pm | ((u32)mycols & 255u) << 4;
    bool fresh = true; u32* fp = nullptr; u32 fv = 0;
    bf16x8 At[4][2], Bt[2][2][2];
    for (u32 ui = 0;; ++ui) {
        const u32 nxt = pm | ((u32)(mycols >> (8 * ui + 8)) & 255u) << 4;
        const bool has_next = (nxt >> 4) != 255u, dist = (cur >> 4) < (u32)PK, chain = has_next && !dist;
        const char* cB = WI + (size_t)(cur >> 4) * PANEL;
        if (fresh) {
            wait_panels(P, cur, ui > 1 ? 5u : 0u, call, wid, lane);
            STAMP(P, 8 + ui * 4); if (ui == 0) DUMP(P, 1, __builtin_amdgcn_s_memtime());
            stage4(mB, vB0, cB); stage2(mA, vA0, cA); stage2(mA + 16384, vA1, cA);
            if (wr == 1) G_BAR;
            G_WV(2); G_BAR;
            stage4(mB + 32768, vB1, cB); stage2(mA + 32768, vA2, cA);
            G_WV(6); G_BAR;
            raise_flags(fp, fv); fp = nullptr; fv = 0;
        }
        f32x4 acc[2][2][4][2];
#pragma unroll
        for (int a = 0; a < 2; ++a)
#pragma unroll
            for (int b = 0; b < 2; ++b)
#pragma unroll
                for (int m = 0; m < 4; ++m)
#pragma unroll
                    for (int n = 0; n < 2; ++n) acc[a][b][m][n] = (f32x4){0.f, 0.f, 0.f, 0.f};
        const char *p0 = cA, *pA = cA + 65536, *pB = cB + 65536;
        u32 stops = 0x80000000u | (ui == 0 ? 0x09020408u : ui == 1 ? 0x00020008u : 0u), upto = 0;
        for (u32 trip = 0;;) {
            const u32 end = stops ? (u32)__builtin_ctz(stops) : 32u;
            u32 left = end - trip;
            do {
                G_LDB(0); G_SCHED; G_LDA(0, 0); stage2(mA + 49152, vA3, p0);
                G_MMA(0, );
                G_LDA(0, 1); stage4(mB, vB0, pB); stage2(mA, vA0, pA);
                G_MMA(1, );
                G_LDB(1); G_SCHED; G_LDA(1, 0); stage2(mA + 16384, vA1, pA);
                G_MMA(0, );
                G_LDA(1, 1); stage4(mB + 32768, vB1, pB); stage2(mA + 32768, vA2, pA);
                G_MMA(1, p0 = pA; pA += 65536; pB += 65536; --left; asm volatile("" : "+s"(p0), "+s"(pA), "+s"(pB), "+s"(left)));
            } while (left);
            if (!stops) break;
            trip = end; stops &= stops - 1;
            if (stops) { upto += ui ? (upto ? 3u : 2u) : 1u; wait_panels(P, cur, upto, call, wid, lane); if (ui == 0) STAMP(P, upto == 5 ? 30 : 3 + upto); }
            else {
                pA = cA; pB = cB;
                if (chain) { pB = WI + (size_t)(nxt >> 4) * PANEL; wait_panels(P, nxt, ui ? 5u : 0u, call, wid, lane); }
            }
        }
        STAMP(P, 9 + ui * 4); DUMP(P, 2 + ui, __builtin_amdgcn_s_memtime());
        if (!chain) G_WV(0);
        if (wr == 0) G_BAR;
        u32 v[64];
        {
            int tz = threadIdx.x; asm volatile("" : "+v"(tz));
            const u32 ln = tz & 63, bad = lds32v(CTL_BAD);
            const u32 ep = D_DESC + wid * 2048, to = ep + (ln & 15) * 128 + (((ln >> 4) ^ (ln & 7)) * 16), from = ep + (ln ^ (ln >> 3)) * 16;
            u16* o = y_base(P, cur & 15, global_panel(cur >> 4, P.rank), wid, ln);
#pragma unroll
            for (int p = 0; p < 16; p += 2) {
                const int ai = p >> 3, m = (p >> 1) & 3;
#pragma unroll
                for (int bj = 0; bj < 2; ++bj) {
                    const f32x4 x0 = acc[ai][bj][m][0], x1 = acc[ai][bj][m][1];
                    lds128w(to ^ (bj * 64), (u32x4){bad ? 0x7fc07fc0u : cvt_pk(x0[0], x0[1]), cvt_pk(x0[2], x0[3]), cvt_pk(x1[0], x1[1]), cvt_pk(x1[2], x1[3])});
                }
                CBAR();
#pragma unroll
                for (int j = 0; j < 2; ++j) { const u32x4 t = lds128(from + j * 1024); v[4 * (p + j)] = t.x; v[4 * (p + j) + 1] = t.y; v[4 * (p + j) + 2] = t.z; v[4 * (p + j) + 3] = t.w; }
                CBAR();
            }
            if (!dist) {
#pragma unroll
                for (int p = 0; p < 16; ++p) st_out<GAR_GST>(o + y_piece_off(p), (u32x4){v[4 * p], v[4 * p + 1], v[4 * p + 2], v[4 * p + 3]});
            }
            if (chain) { if (wr == 1) G_BAR; }
            else {
                if (dist) {
                    STAMP(P, 72 + wid);
                    sample<6, 10, 5>(v, LDS_HIST, ln);
                    WAIT_LGKM0(); BAR();
                    const OutStore os = {o};
                    pack_group<true>(P, v, GY + cur, wid, false, (cur >> 4) < (u32)DP ? 255u & ~(1u << P.rank) : 1u << (((cur >> 4) - DP) & 7), call, wid, ln, os, fp, fv);
                    if (!has_next) { G_WV(0); BAR(); raise_flags(fp, fv); }
                }
                STAMP(P, 10 + ui * 4);
                if (!has_next) break;
            }
        }
        fresh = !chain; cur = nxt;
    }
}

__global__ void __launch_bounds__(512, 2) fused(const Args A) {
    const u32 wid = rfl(threadIdx.x >> 6), lane = threadIdx.x & 63;
    const u32 role = pin((A.role[blockIdx.x >> 1] >> ((blockIdx.x & 1) * 16)) & 0xffffu);
    char* peer = pin(A.ws[0]);
#pragma unroll
    for (u32 j = 1; j < NR; ++j) { char* c = pin(A.ws[j]); peer = (role & 7) == j ? c : peer; }
    const Params P = {pin(A.me), peer, pin(A.x), pin(A.w), pin(A.out), pin((u32)A.rank), pin((u32)A.nunits)};
    call_load(P, wid, lane);
    if (wid == 1 || wid == 2) {
        const char* ka = (const char*)(const GAS char*)(u64)__builtin_amdgcn_kernarg_segment_ptr() + lane * 16;
        if (wid == 1 && !(role & 0x8000u)) { if (lane < 4) glds_plain(ka, LDS_TAB); }
        else if ((wid == 1) == ((role & 0x8000u) != 0)) { glds_plain(ka, LDS_TAB); glds_plain(ka + 1024, LDS_TAB + 1024); glds_plain(ka + 2048, LDS_TAB + 2048); }
    }
    if (role & 0x8000u) {
        if (threadIdx.x == 0) { lds32vw(CTL_NUNIT, 0u); lds32vw(CTL_LANDED, 0u); lds32vw(CTL_NCHUNK, 0u); }
        if (wid == 1 && lane < 8) lds32vw(CTL_NEXT + lane * 4, lane);
        WAIT_LGKM0(); BAR();
        puller(P, role, wid, lane); return;
    }
    STAMP(P, 0); DUMP(P, 0, __builtin_amdgcn_s_memtime());
    if (threadIdx.x == 0) lds32vw(CTL_BAD, 0u);
    if (threadIdx.x < 8) lds32vw(CTL_SIZE + threadIdx.x * 4, 0u);
    pack_init(wid, lane);
    const u32 rb = NENC - 1 - (role & 255u), sc = rb + (wid & 2) * 32;
    if (rb >= 64u) { const u64 t0 = now(); while (now() - t0 < GAR_HOLD) __builtin_amdgcn_s_sleep(2); }
    for (u32 it = 0; it < 3; ++it) {
        const u32 gid = it ? tab16(TAB_ENC, (role & 255u) * 2 + it - 1) : solo_gid(sc);
        if (gid == 0xffffu) break;
        if (it || ((wid == 1 || wid == 2) && rb < 64u)) pack_inputs(P, gid, it ? wid : solo_w(sc), !it, wid, lane);
        WAIT_VM(0); WAIT_LGKM0(); BAR(); STAMP(P, it ? umin(it, 2) : 3);
    }
    const u32 call = call_take(wid);
    call_bump(P, wid, lane, call);
    if (role >> 8) gemm_role(P, (role >> 8) - 1, call, wid, lane);
}

#define GAR_T3 0
#define GAR_T2 0
#define GAR_T1 8
#define GAR_XAHEAD 12
#define GAR_EPOS 18
#define GAR_F3 10
#define GAR_LAG 4
static void build_tables(Args& P) {
    unsigned char* cols = (unsigned char*)P.cols; u16* stream = (u16*)P.stream; u16 enc[NPACK]; u16* role = (u16*)P.role;
    {
        int peer[2][4], np[2] = {0, 0}, cnt[8] = {}, seen[2] = {0, 0}, n = 0, top = NENC;
        for (int d = 1; d < NR; ++d) { const int q = (P.rank + d) & 7, h = ((0xb2 >> q) ^ (0xb2 >> P.rank)) & 1; peer[h][np[h]++] = q; }
        for (int b = 0; b < NWG; ++b) {
            const int h = (b >> 2) & 1, i = b >> 3, g0 = 32 - 4 * (h ? CH : CL), near = h ? 4 * NH : 3 * NL, far = h ? 3 * FL : 4 * FH, k = seen[h]++;
            if (k >= near + far) { role[b] = i >= g0 ? (u16)(n++ | (i - g0 + 1) << 8) : (u16)--top; continue; }
            const int side = k < near ? h : !h, j = k < near ? k : k - near, q = peer[side][side ? (j + j / 4) & 3 : j % 3];
            role[b] = (u16)(0x8000 | q | cnt[q]++ << 3);
        }
        for (int b = 0; b < NWG; ++b) if (role[b] & 0x8000) role[b] |= (u16)(cnt[role[b] & 7] << 7);
    }
    int col[LC], nc = 0;
    {
        const int rest = LC - (ROUNDS - 1) * CPR, r0 = (rest + 1) / 2, light = rest < CPR ? CPR - rest : CPR;
        int over = PK - FX - light * (ROUNDS - 1), nd = 0, nr = PK;
        for (int i = 0; i < 128; ++i) cols[i] = 0xff;
        for (int r = 0; r < ROUNDS; ++r) for (int c = 0; c < 8; ++c) for (int cg = 0; cg < 2; ++cg) {
            const bool last = c < (cg ? rest - r0 : r0), heavy = last && rest < CPR;
            if (nd >= DP && nd < PK && ((nd - DP) & 7) == P.rank) ++nd;
            if (c >= (cg ? CH : CL) || (r + 1 == ROUNDS && !last)) continue;
            cols[(cg * 8 + c) * 8 + r] = (unsigned char)(col[nc++] = nd < PK && (!heavy || over-- > 0) ? nd++ : nr++);
        }
    }
    auto wgid = [&](int jj, int dest, int h) { return GW + 2 * (jj >= DP ? jj + (NR - 1) * DP : dest * DP + jj) + h; };
    int k = 0;
    auto put = [&](int gid, int lv) { for (int j = 0; j < (1 << lv); ++j) stream[k++] = (u16)(gid | ((1 << lv) + j) << 10); };
    auto opnd = [&](int i, int h) { return i < 16 ? GX + 16 * h + i : wgid(col[i - 16], P.rank, h); };
    auto stage = [&](int st, int i0, int i1) {
        const int h = st >> 1, half = st & 1;
        for (int i = i0; i < i1; ++i) {
            if (i < 16 && h == 0) { for (int j = 0; j < 2; ++j) stream[k++] = (u16)(opnd(i, h) | (4 + 2 * half + j) << 10); }
            else stream[k++] = (u16)(opnd(i, h) | (2 + half) << 10);
        }
    };
    stage(0, NRAW / 4, 16); stage(1, 0, GAR_XAHEAD);
    {
        u16 t[64]; const int n = k; int e = 0;
        for (int i = 0; i < n; ++i) t[i] = stream[i];
        k = 0;
        for (int i = 0; i < n || e < NRAW / 4; ++i) { if (i < n) stream[k++] = t[i]; if (i >= GAR_EPOS && e < NRAW / 4) stream[k++] = (u16)((GX + e++) | 2 << 10); }
    }
    const int first = 16 + CPR - GAR_LAG;
    stage(0, 16, first); stage(1, GAR_XAHEAD, first); stage(0, first, 16 + CPR); stage(2, 0, first); stage(1, first, 16 + CPR);
    for (int part = 6; part < 8; ++part) for (int i = 0; i < first; ++i) stream[k++] = (u16)(opnd(i, 1) | part << 10);
    stage(2, first, 16 + CPR); stage(3, first, 16 + CPR);
    for (int h = 0; h < 2; ++h) for (int i = CPR; i < 2 * CPR; ++i) put(wgid(col[i], P.rank, h), 0);
    for (int i = 2 * CPR; i < LC; ++i) for (int h = 0; h < 2; ++h) put(wgid(col[i], P.rank, h), 0);
    for (int t = 0; t < 16 * DP; ++t) { const int left = 16 * DP - t; put(GY + t, left <= GAR_T3 ? 3 : left <= GAR_T2 ? 2 : left <= GAR_T1 ? 1 : 0); }
    const int nown = k;
    for (int t = 0; t < 16 * FX; ++t) put(GY + (DP + (t >> 4) * 8 + P.rank) * 16 + (t & 15), 16 * FX - t <= GAR_F3 ? 3 : 16 * FX - t <= GAR_F3 + 14 ? 2 : 1);
    P.nunits = nown | (k - nown) << 16;
    k = 0;
    for (int i = 0; i < CPR; ++i) for (int d = 1; d < (col[i] < DP ? NR : 2); ++d) enc[k++] = (u16)wgid(col[i], (P.rank + d) & 7, 0);
    for (int i = SOLOG; i < 32; ++i) enc[k++] = (u16)(GX + i);
    for (int i = 0; i < CPR; ++i) { if (col[i] < DP) enc[k++] = (u16)wgid(col[i], P.rank, 0); enc[k++] = (u16)wgid(col[i], P.rank, 1); }
    for (int i = 0; i < CPR; ++i) if (col[i] < DP) for (int d = 1; d < NR; ++d) enc[k++] = (u16)wgid(col[i], (P.rank + d) & 7, 1);
    for (int f = 0; f < FX; ++f) for (int h = 0; h < 2; ++h) enc[k++] = (u16)wgid(DP + f * 8 + P.rank, 0, h);
    for (int i = CPR; i < LC; ++i) for (int h = 0; h < 2; ++h) for (int d = 0; d < (col[i] < DP ? NR : 1); ++d) enc[k++] = (u16)wgid(col[i], (P.rank + d) & 7, h);
    u16* job = (u16*)P.enc; int nx = 0, top = NENC;
    for (int i = 0; i < NENC - 64; ++i) { job[2 * i] = enc[nx++]; job[2 * i + 1] = 0xffff; }
    for (int i = 0; i < NENC - 64; ++i) if (job[2 * i] >= GW && (job[2 * i] - GW) / 2 / DP == P.rank) job[2 * i + 1] = enc[nx++];
    for (int i = NENC - 64; i < NENC; ++i) { job[2 * i] = enc[nx++]; job[2 * i + 1] = 0xffff; }
    while (nx < NPACK) { do --top; while (job[2 * top + 1] != 0xffff); job[2 * top + 1] = enc[nx++]; }
}
}

extern "C" void kernel_launch(int rank, int nranks, ncclComm_t comm, hipStream_t stream,
                              void* const* d_in, const int* in_sizes, int n_in,
                              void* d_out, int out_size, const int* in_dtypes, int out_dtype,
                              void* const* d_out_all, void* const* d_ws_all, size_t ws_bytes) {
    using namespace gar;
    if (nranks != NR || (unsigned)rank >= (unsigned)NR || n_in != 2 || ws_bytes < WS_END || in_sizes[0] != MM * KS || in_sizes[1] != KS * NN || out_size != MM * NN) {
        fprintf(stderr, "kernel_launch: unexpected shapes\n"); return; }
    Args P = {};
    for (int p = 0; p < NR; ++p) P.ws[p] = (char*)d_ws_all[p];
    P.me = P.ws[rank];
    P.x = (const u16*)d_in[0]; P.w = (const u16*)d_in[1]; P.out = (u16*)d_out; P.rank = rank;
    build_tables(P);
    HIP_CHECK(hipFuncSetAttribute((const void*)fused, hipFuncAttributeMaxDynamicSharedMemorySize, LDS_BYTES));
    hipLaunchKernelGGL(fused, dim3(NWG), dim3(512), LDS_BYTES, stream, P);
    HIP_CHECK(hipGetLastError());
}
```

```cpp
#include <hip/hip_runtime.h>
#include <rccl/rccl.h>
#include <cstdio>
#include <cstdint>
#include <cstddef>
#pragma clang diagnostic ignored "-Wint-to-pointer-cast"

#define HIP_CHECK(expr) do { hipError_t _e = (expr); if (_e != hipSuccess) { \
    fprintf(stderr, "HIP error %s at %s:%d\n", hipGetErrorString(_e), __FILE__, __LINE__); \
    return; } } while (0)

namespace gar {
#define LAS __attribute__((address_space(3)))
#define GAS __attribute__((address_space(1)))
typedef unsigned u32;
typedef unsigned short u16;
typedef unsigned long long u64;
typedef unsigned u32x4 __attribute__((ext_vector_type(4)));
typedef unsigned u32x2 __attribute__((ext_vector_type(2)));
typedef short s16x4 __attribute__((ext_vector_type(4)));
typedef short bf16x8 __attribute__((ext_vector_type(8)));
typedef float f32x4 __attribute__((ext_vector_type(4)));

constexpr int NR = 8, MM = 4096, NN = 32768, KS = 512;
constexpr int DP = 10;
constexpr int RP = 128 - NR * DP;
constexpr int LP = DP + RP;
constexpr int NWG = 256;
constexpr int NL = 9, FL = 0, NH = 8, FH = 1, CL = 6, CH = 6, CPR = CL + CH;
constexpr int NPULL = 3 * (NL + FL) + 4 * (NH + FH);
constexpr int WIN = 6;
static_assert(3 * NL + 4 * FH + 16 * CL <= 128 && 4 * NH + 3 * FL + 16 * CH <= 128, "roles");
constexpr int SOLOG = 16, NRAW = 52;
constexpr int LEAD = 44;
static_assert(NRAW <= 64 && NRAW % 4 == 0 && 8 * SOLOG <= NWG - NPULL && SOLOG == 16, "solo chunks");
constexpr int NENC = NWG - NPULL;
constexpr int GX = 0, GW = 32, GY = 288;
constexpr int FX = 2, PK = DP + 8 * FX, LC = LP - FX;
constexpr int NGID = GY + 16 * PK, GM = NGID, NGT = GM + 16;
constexpr int NS = 32 + 2 * LC + 16 * DP + 16 * FX;
constexpr int ROUNDS = (LC + CPR - 1) / CPR;
constexpr int NUMAX = 704;
constexpr int NSTAT = 3;
constexpr int NPACK = 288 - SOLOG;
constexpr u64 DEADLINE = 30000000ull;

constexpr size_t WS_CALL = 0;
constexpr size_t WS_TMO = 32768;
constexpr size_t WS_CNT = 36864;
constexpr size_t WS_CLAIM = 57344;
constexpr size_t WS_FLAG = 65536;
constexpr size_t FLAG_SRC = NGT * 32, FLAG_PAR = 8 * FLAG_SRC;
constexpr size_t GRP_STRIDE = 131072;
constexpr size_t WS_STG = 2u << 20;
constexpr size_t STG_PAR = (NGID * GRP_STRIDE + 0x1fffff) & ~(size_t)0x1fffff;
constexpr size_t WS_INB = WS_STG + 2 * STG_PAR;
constexpr size_t WS_XI = WS_INB + 2 * 8 * 64 * 8192;
constexpr size_t PANEL = 64 * 32768;
constexpr size_t WS_WI = WS_XI + 16 * PANEL;
constexpr size_t WS_END = WS_WI + (size_t)LP * PANEL;
static_assert(WS_FLAG + 2 * FLAG_PAR <= WS_STG && NGT <= 1024 && RP >= 8 * FX, "layout");

constexpr u32 LDS_CTL = 131072;
constexpr u32 LDS_LUT = LDS_CTL + 512, LDS_HIST = LDS_CTL + 1024;
constexpr u32 LDS_TAB = LDS_HIST + 4096, D_DESC = LDS_TAB + 3072, LDS_BYTES = D_DESC + NS * 64;
constexpr u32 CTL_CALL = LDS_CTL  , CTL_SIZE = LDS_CTL + 64  , CTL_BAD = LDS_CTL + 128;
constexpr u32 CTL_NUNIT = LDS_CTL + 192, CTL_LANDED = LDS_CTL + 256, CTL_NCHUNK = LDS_CTL + 320, CTL_NEXT = LDS_CTL + 384  ;
constexpr u32 E_L2 = 0, E_L3 = E_L2 + 4160, E_L4 = E_L3 + 2112, E_VAR = E_L4 + 1088, E_END = E_VAR + 2112;
static_assert(E_END <= 10240 && E_L4 % 16 == 0 && E_L3 % 16 == 0 && E_VAR % 16 == 0, "packer scratch");
constexpr u32 CAP2 = 4080, CAP3 = 2032, CAP4 = 1008, CAPV = 2032, FIXB = 10240, RAWB = 16384, RAWCODE = 255;
constexpr u32 RING = 65536, RMASK = RING - 1, ND = 6;
constexpr u32 D_SCR = RING, D_SCRB = 7360, D_L2 = 0, D_L3 = 4160, D_L4 = D_L3 + 2112;
constexpr u32 D_UNIT = D_SCR + ND * D_SCRB, DONE = 0x80000000u;
static_assert(D_L4 + 1088 <= D_SCRB && D_UNIT + (NUMAX + 8 * SOLOG) * 8 <= LDS_CTL && LDS_BYTES <= 163840 && D_DESC % 128 == 0 && NS * 64 >= 16384, "puller LDS");

struct Args {
    char* ws[NR];
    char* me; const u16* x; const u16* w; u16* out;
    int rank, nunits;
    u16 role[NWG];
    u16 stream[NUMAX];
    unsigned char cols[128];
    u16 enc[2 * NENC];
    u16 pad[(3072 - 104 - 128) / 2 - NWG - NUMAX - 2 * NENC];
};
static_assert(sizeof(Args) == 3072 && ROUNDS < 8 && offsetof(Args, cols) % 8 == 0 && offsetof(Args, role) % 4 == 0, "kernarg");
constexpr u32 TAB_STREAM = LDS_TAB + offsetof(Args, stream), TAB_ENC = LDS_TAB + offsetof(Args, enc), TAB_COLS = LDS_TAB + offsetof(Args, cols);
struct Params { char* me; char* peer; const u16* x; const u16* w; u16* out; u32 rank, nunits; };
template <class T> __device__ __forceinline__ T* pin(T* p) { u64 v = (u64)p; asm volatile("" : "+s"(v)); return (T*)(GAS T*)v; }
__device__ __forceinline__ u32 pin(u32 v) { asm volatile("" : "+s"(v)); return v; }

__device__ __forceinline__ u32 lds32(u32 a) { return *(LAS u32*)a; }
__device__ __forceinline__ u32 lds32v(u32 a) { return *(volatile LAS u32*)a; }
__device__ __forceinline__ void lds32vw(u32 a, u32 x) { *(volatile LAS u32*)a = x; }
__device__ __forceinline__ u32x2 lds64v(u32 a) { return *(volatile LAS u32x2*)a; }
__device__ __forceinline__ void lds64vw(u32 a, u32x2 x) { *(volatile LAS u32x2*)a = x; }
__device__ __forceinline__ u32x4 lds128(u32 a) { return *(LAS u32x4*)a; }
__device__ __forceinline__ void lds128w(u32 a, u32x4 x) { *(LAS u32x4*)a = x; }
__device__ __forceinline__ u32x4 zero4() { u32 z; asm volatile("v_mov_b32 %0, 0" : "=v"(z)); return (u32x4){z, z, z, z}; }
__device__ __forceinline__ void lds_or(u32 a, u32 x) { __hip_atomic_fetch_or((LAS u32*)a, x, __ATOMIC_RELAXED, __HIP_MEMORY_SCOPE_WORKGROUP); }
__device__ __forceinline__ void lds_add(u32 a, u32 x) { __hip_atomic_fetch_add((LAS u32*)a, x, __ATOMIC_RELAXED, __HIP_MEMORY_SCOPE_WORKGROUP); }
__device__ __forceinline__ u32 rfl(u32 x) { return (u32)__builtin_amdgcn_readfirstlane((int)x); }
__device__ __forceinline__ u32 tab16(u32 t, u32 i) { return rfl(*(volatile LAS u16*)(t + i * 2)); }
#define CBAR() asm volatile("" ::: "memory")
#define WAIT_LGKM0() asm volatile("s_waitcnt lgkmcnt(0)" ::: "memory")
#define WAIT_VM(n) asm volatile("s_waitcnt vmcnt(%0)" :: "i"(n) : "memory")
#define BAR() do { CBAR(); __builtin_amdgcn_s_barrier(); CBAR(); } while (0)
__device__ __forceinline__ void st_sys(void* p, u32x4 x) { asm volatile("global_store_dwordx4 %0, %1, off sc0 sc1\n\ts_nop 1" :: "v"(p), "v"(x) : "memory"); }
__device__ __forceinline__ void st_dev(void* p, u32x4 x) { asm volatile("global_store_dwordx4 %0, %1, off sc1\n\ts_nop 1" :: "v"(p), "v"(x) : "memory"); }
__device__ __forceinline__ void st_out(void* p, u32x4 x) { asm volatile("global_store_dwordx4 %0, %1, off sc1 nt\n\ts_nop 1" :: "v"(p), "v"(x) : "memory"); }
__device__ __forceinline__ void glds_sys(const void* g, u32 dst) {
    u32 keep;
    asm volatile("s_mov_b32 %0, m0\n\ts_mov_b32 m0, %2\n\ts_nop 0\n\tglobal_load_lds_dwordx4 %1, off sc0 sc1\n\ts_mov_b32 m0, %0" : "=&s"(keep) : "v"(g), "s"(dst) : "memory");
}
__device__ __forceinline__ void glds_plain(const void* g, u32 dst) {
    u32 keep;
    asm volatile("s_mov_b32 %0, m0\n\ts_mov_b32 m0, %2\n\ts_nop 0\n\tglobal_load_lds_dwordx4 %1, off\n\ts_mov_b32 m0, %0" : "=&s"(keep) : "v"(g), "s"(dst) : "memory");
}
__device__ __forceinline__ void glds_w(const void* g, u32 dst) {
    u32 keep;
    asm volatile("s_mov_b32 %0, m0\n\ts_mov_b32 m0, %2\n\ts_nop 0\n\tglobal_load_lds_dwordx4 %1, off nt\n\ts_mov_b32 m0, %0" : "=&s"(keep) : "v"(g), "s"(dst) : "memory");
}
__device__ __forceinline__ u64 now() { return __builtin_amdgcn_s_memrealtime(); }
template <int CTRL, int RM, bool BC> __device__ __forceinline__ u32 dpp(u32 v) { return (u32)__builtin_amdgcn_update_dpp(0, (int)v, CTRL, RM, 0xf, BC); }
__device__ __forceinline__ u32 scan_incl(u32 v) {
    v += dpp<0x111, 0xf, true>(v); v += dpp<0x112, 0xf, true>(v); v += dpp<0x114, 0xf, true>(v); v += dpp<0x118, 0xf, true>(v);
    v += dpp<0x142, 0xa, false>(v); v += dpp<0x143, 0xc, false>(v); return v;
}
__device__ __forceinline__ u32 umax(u32 a, u32 b) { return a > b ? a : b; }
__device__ __forceinline__ u32 umin(u32 a, u32 b) { return a < b ? a : b; }
__device__ __forceinline__ u32 popc(u32 x) { return (u32)__builtin_popcount(x); }
__device__ __forceinline__ u32 perm(u32 hi, u32 lo, u32 sel) { return __builtin_amdgcn_perm(hi, lo, sel); }


__device__ __forceinline__ u32 nzb(u32 z) { return ((z & 0x7f7f7f7fu) + 0x7f7f7f7fu) | z; }
typedef unsigned short u16x2 __attribute__((ext_vector_type(2)));
struct Classifier {
    u32 s1, s2, s3, nb, tlo, thi; bool win;
    __device__ __forceinline__ void init(u32 t) {
        const u32 a = (t >> 8) & 255u, b = (t >> 16) & 255u, c = t >> 24, lo = umin(a, umin(b, c));
        win = umax(a, umax(b, c)) - lo <= 6u;
        const u64 l = (1ull << (((a - lo) & 7u) * 8)) | (2ull << (((b - lo) & 7u) * 8)) | (3ull << (((c - lo) & 7u) * 8));
        tlo = (u32)l; thi = (u32)(l >> 32); nb = ((0u - lo) & 255u) * 0x01000100u;
        s1 = a * 0x01010101u; s2 = b * 0x01010101u; s3 = c * 0x01010101u;
    }
    __device__ __forceinline__ u32 any(u32 e) const {
        const u32 a1 = nzb(e ^ s1), a2 = nzb(e ^ s2), a3 = nzb(e ^ s3);
        return ((~(a1 & a3) >> 7) & 0x01010101u) | ((~(a2 & a3) >> 6) & 0x02020202u);
    }
    __device__ __forceinline__ u32 near(u32 e) const {
        const u16x2 x = __builtin_bit_cast(u16x2, e), n = __builtin_bit_cast(u16x2, nb), top = {0x07ff, 0x07ff};
        u32 sh; asm("v_pk_mad_u16 %0, %1, %2, %3" : "=v"(sh) : "v"(e), "v"(0x01000100u), "s"(nb));
        const u16x2 hi = __builtin_elementwise_min((u16x2)(x + n), top), lo = __builtin_elementwise_min(__builtin_bit_cast(u16x2, sh), top);
        return perm(thi, tlo, perm(__builtin_bit_cast(u32, hi), __builtin_bit_cast(u32, lo), 0x07030501u));
    }
    template <int N> __device__ __forceinline__ void run(const u32* e, u32* c) const {
        if (win) {
#pragma unroll
            for (int i = 0; i < N; ++i) c[i] = near(e[i]);
        } else {
#pragma unroll
            for (int i = 0; i < N; ++i) c[i] = any(e[i]);
        }
    }
};
__device__ __forceinline__ u32 escbits(u32 c) { return ~(c | (c >> 1)) & 0x55555555u; }
struct Appender {
    u32 lo, fill, addr;
    __device__ __forceinline__ void init(u32 off) { lo = 0; fill = (off & 3u) * 8u; addr = off & ~3u; }
    __device__ __forceinline__ void push(u32 k, u32 mk) {
        const u64 t = (u64)k << fill;
        lo |= (u32)t; fill += popc(mk);
        if (fill >= 32u) { lds_or(addr, lo); lo = (u32)(t >> 32); addr += 4; }
        fill &= 31u;
    }
    __device__ __forceinline__ void flush() { if (lo) lds_or(addr, lo); }
};
__device__ __forceinline__ u32 escmask(u32 c) { return perm(0u, 0xffu, c); }
__device__ __forceinline__ u32 selector(u32 mk) { return lds32(__builtin_amdgcn_udot4(mk & 0x20100804u, 0x01010101u, LDS_LUT, false)); }
template <int R> __device__ __forceinline__ u32 pack_level(u32 src, u32 n, u32 t, u32 cdst, u32 edst, u32 cap, u32 lane) {
    Classifier K; K.init(t);
    if (lane < 16) *(LAS unsigned char*)(src + n + lane) = (unsigned char)K.s1;
    u32 ne = 0;
    for (u32 r = 0; r * 1024 < n; r += R) {
        CBAR();
        u32 q[4 * R], c[4 * R], m[4 * R], l[4 * R], cnt = 0;
#pragma unroll
        for (int i = 0; i < R; ++i) { const u32 p0 = (r + i) * 1024 + lane * 16; const u32x4 x = lds128(src + p0); const bool act = p0 < n;
            q[4 * i] = act ? x.x : K.s1; q[4 * i + 1] = act ? x.y : K.s1; q[4 * i + 2] = act ? x.z : K.s1; q[4 * i + 3] = act ? x.w : K.s1; }
        K.run<4 * R>(q, c);
#pragma unroll
        for (int i = 0; i < 4 * R; ++i) { m[i] = escmask(c[i]); l[i] = selector(m[i]); }
#pragma unroll
        for (int i = 0; i < R; ++i) cnt |= ((popc(m[4 * i]) + popc(m[4 * i + 1]) + popc(m[4 * i + 2]) + popc(m[4 * i + 3])) >> 3) << (16 * i);
        const u32 inc = scan_incl(cnt), tot = (u32)__builtin_amdgcn_readlane((int)inc, 63), exc = inc - cnt;
        if (ne + (tot & 0xffffu) + (tot >> 16) > cap) return ~0u;
#pragma unroll
        for (int i = 0; i < R; ++i) {
            Appender A; A.init(edst + ne + (i ? (tot & 0xffffu) + (exc >> 16) : (exc & 0xffffu)));
#pragma unroll
            for (int j = 0; j < 4; ++j) A.push(perm(0u, q[4 * i + j], l[4 * i + j]), m[4 * i + j]);
            A.flush();
            if ((r + i) * 1024 + lane * 16 < n) *(LAS u32*)(cdst + ((r + i) * 64 + lane) * 4) = c[4 * i] | (c[4 * i + 1] << 2) | (c[4 * i + 2] << 4) | (c[4 * i + 3] << 6);
        }
        ne += (tot & 0xffffu) + (tot >> 16);
    }
    CBAR();
    return ne;
}
template <int N> __device__ __forceinline__ u32 rank_ror(u32 key) { return (dpp<0x120 + N, 0xf, true>(key) > key ? 1u : 0u) + rank_ror<N - 1>(key); }
template <> __device__ __forceinline__ u32 rank_ror<0>(u32) { return 0u; }
__device__ __forceinline__ u32 row_or(u32 x) { x |= dpp<0x128, 0xf, true>(x); x |= dpp<0x124, 0xf, true>(x); x |= dpp<0x122, 0xf, true>(x); x |= dpp<0x121, 0xf, true>(x); return rfl(x); }
__device__ __forceinline__ void split(const u32 (&v)[64], u32 (&Mr)[32], u32 (&Er)[32]) {
#pragma unroll
    for (int g = 0; g < 32; ++g) {
        const u32 a = __builtin_amdgcn_alignbit(v[2 * g], v[2 * g], 31), b = __builtin_amdgcn_alignbit(v[2 * g + 1], v[2 * g + 1], 31);
        Er[g] = perm(b, a, 0x07050301u); Mr[g] = perm(b, a, 0x06040200u);
        asm volatile("" : "+v"(Er[g]), "+v"(Mr[g]));
    }
}
__device__ __forceinline__ u32x4 piece(const u32 (&Mr)[32], const u32 (&Er)[32], int p) {
    const u32 a = perm(Er[2 * p], Mr[2 * p], 0x05010400u), b = perm(Er[2 * p], Mr[2 * p], 0x07030602u);
    const u32 c = perm(Er[2 * p + 1], Mr[2 * p + 1], 0x05010400u), d = perm(Er[2 * p + 1], Mr[2 * p + 1], 0x07030602u);
    return (u32x4){__builtin_amdgcn_alignbit(a, a, 1), __builtin_amdgcn_alignbit(b, b, 1), __builtin_amdgcn_alignbit(c, c, 1), __builtin_amdgcn_alignbit(d, d, 1)};
}
template <int R> __device__ __forceinline__ u32 pack_short(u32 src, u32 n, u32 t, u32 cdst, u32 edst, u32 cap, u32 lane) {
    Classifier K; K.init(t);
    if (lane < 16) *(LAS unsigned char*)(src + n + lane) = (unsigned char)K.s1;
    u32 ne = 0;
    for (u32 r = 0; r * 256 < n; r += R) {
        CBAR();
        u32 q[R], c[R], m[R], l[R], cnt = 0;
#pragma unroll
        for (int i = 0; i < R; ++i) { const u32 p0 = (r + i) * 256 + lane * 4, x = lds32(src + p0); q[i] = p0 < n ? x : K.s1; }
        K.run<R>(q, c);
#pragma unroll
        for (int i = 0; i < R; ++i) { m[i] = escmask(c[i]); l[i] = selector(m[i]); cnt |= (popc(m[i]) >> 3) << (16 * i); }
        const u32 inc = scan_incl(cnt), tot = (u32)__builtin_amdgcn_readlane((int)inc, 63), exc = inc - cnt;
        if (ne + (tot & 0xffffu) + (tot >> 16) > cap) return ~0u;
#pragma unroll
        for (int i = 0; i < R; ++i) {
            Appender A; A.init(edst + ne + (i ? (tot & 0xffffu) + (exc >> 16) : (exc & 0xffffu)));
            A.push(perm(0u, q[i], l[i]), m[i]); A.flush();
            u32 x = c[i] << (2 * (lane & 3)); x |= dpp<0xb1, 0xf, true>(x); x |= dpp<0x4e, 0xf, true>(x);
            if ((lane & 3) == 0 && (r + i) * 256 + lane * 4 < n) *(LAS u32*)(cdst + (r + i) * 64 + lane) = x;
        }
        ne += (tot & 0xffffu) + (tot >> 16);
    }
    CBAR();
    return ne;
}
template <int N, int STEP, int OFF> __device__ __forceinline__ void sample(const u32 (&v)[64], u32 hist, u32 lane) {
#pragma unroll
    for (int i = 0; i < N; ++i) lds_add(hist + (((v[i * STEP + OFF] >> ((i & 1) * 16 + 3)) & 0xff0u) | ((lane & 3) * 4)), 1u);
}
template <bool SAMPLED, class Side>
__device__ __forceinline__ u32 pack_chunk(const u32 (&v)[64], u32 (&Mr)[32], u32 (&Er)[32], u32 (&Cr)[8], u32 scr, u32 lane, bool solo, const Side& side) {
    CBAR();
#define SIDE4(p0) do { if (side.on && !solo) { _Pragma("unroll") for (int p = p0; p < p0 + 4; ++p) side(p, piece(Mr, Er, p)); } } while (0)
    if (side.on) {
#pragma unroll
        for (int p = 0; p < 4; ++p) side(p, (u32x4){v[4 * p], v[4 * p + 1], v[4 * p + 2], v[4 * p + 3]});
    }
    const u32 hist = solo ? scr + 10240 : LDS_HIST; const u32x4 z4 = zero4();
    if (solo) {
#pragma unroll
        for (int j = 0; j < 4; ++j) lds128w(hist + j * 1024 + lane * 16, z4);
        CBAR();
        sample<12, 5, 2>(v, hist, lane);
        if (side.on) {
#pragma unroll
            for (int p = 4; p < 16; ++p) side(p, (u32x4){v[4 * p], v[4 * p + 1], v[4 * p + 2], v[4 * p + 3]});
        }
    }
    if (!SAMPLED) sample<6, 10, 5>(v, hist, lane);
#pragma unroll
    for (int j = 0; j < 10; ++j) lds128w(scr + j * 1024 + lane * 16, z4);
    if (!SAMPLED) { WAIT_LGKM0(); if (!solo) BAR(); }
    CBAR();
    u32 key = 0;
#pragma unroll
    for (int i = 0; i < 4; ++i) { const u32 bin = i * 64 + lane; const u32x4 h = lds128(hist + bin * 16); key = umax(key, ((h.x + h.y + h.z + h.w) << 8) | bin); }
    split(v, Mr, Er);
    { const auto r = __builtin_amdgcn_permlane32_swap(key, key, false, false); key = umax(r[0], r[1]); }
    { const auto r = __builtin_amdgcn_permlane16_swap(key, key, false, false); key = umax(r[0], r[1]); }
    const u32 rank = rank_ror<15>(key), lev = (rank * 11u) >> 5, sym = (key & 255u) << (8 * (rank - 3 * lev + 1));
    const u32 T[4] = {row_or(lev == 0 ? sym : 0u), row_or(lev == 1 ? sym : 0u), row_or(lev == 2 ? sym : 0u), row_or(lev == 3 ? sym : 0u)};
    SIDE4(4);
    u32 cnt = 0, mk[32], sl[32];
    {
        Classifier K; K.init(T[0]);
        u32 c[32]; K.run<32>(Er, c);
#pragma unroll
        for (int g = 0; g < 32; ++g) { mk[g] = escmask(c[g]); sl[g] = selector(mk[g]); cnt += popc(mk[g]); }
#pragma unroll
        for (int k = 0; k < 8; ++k) Cr[k] = c[4 * k] | (c[4 * k + 1] << 2) | (c[4 * k + 2] << 4) | (c[4 * k + 3] << 6);
    }
    cnt >>= 3;
    const u32 inc = scan_incl(cnt), n2 = (u32)__builtin_amdgcn_readlane((int)inc, 63);
    SIDE4(8);
    if (n2 <= CAP2) {
        Appender A; A.init(scr + E_L2 + inc - cnt);
#pragma unroll
        for (int g = 0; g < 32; ++g) A.push(perm(0u, Er[g], sl[g]), mk[g]);
        A.flush();
    }
    SIDE4(12);
    if (n2 > CAP2) return RAWCODE;
    CBAR();
    const u32 d2 = (n2 + 15) >> 4;
    const u32 n3 = pack_level<2>(scr + E_L2, n2, T[1], scr + E_VAR + 16, scr + E_L3, CAP3, lane);
    if (n3 == ~0u) return RAWCODE;
    const u32 d3 = (n3 + 15) >> 4;
    const u32 n4 = pack_short<2>(scr + E_L3, n3, T[2], scr + E_VAR + 16 + 4 * d2, scr + E_L4, CAP4, lane);
    if (n4 == ~0u) return RAWCODE;
    const u32 d4 = (n4 + 15) >> 4, b5 = 16 + 4 * (d2 + d3 + d4);
    const u32 n5 = pack_short<1>(scr + E_L4, n4, T[3], scr + E_VAR + 16 + 4 * (d2 + d3), scr + E_VAR + b5, CAPV - b5, lane);
    if (n5 == ~0u) return RAWCODE;
    if (lane == 0) lds128w(scr + E_VAR, (u32x4){T[0] | d2, T[1] | d3, T[2] | d4, T[3]});
    CBAR();
    return (b5 + n5 + 15) >> 4;
}
__device__ __forceinline__ void pack_init(u32 wid, u32 lane) {
    if (wid >= 4) lds128w(LDS_HIST + (wid - 4) * 1024 + lane * 16, zero4());
    if ((wid == 1 || wid == 2) && lane < 16) { u32 sel = 0x0c0c0c0cu, k = 0;
#pragma unroll
        for (u32 b = 0; b < 4; ++b) if ((lane >> b) & 1) { sel = (sel & ~(255u << k)) | (b << k); k += 8; }
        *(LAS u32*)(LDS_LUT + lane * 4) = sel; }
}
__device__ __forceinline__ u32 chunk_bytes(u32 code) { return code == RAWCODE ? RAWB : FIXB + 16 * code; }

__device__ __forceinline__ void pack_store(char* dst, u32 code, const u32 (&Mr)[32], const u32 (&Er)[32], const u32 (&Cr)[8], u32 scr, u32 lane) {
    char* dm = dst + (chunk_bytes(code) - 8192);
#pragma unroll
    for (int j = 0; j < 8; ++j) st_sys(dm + j * 1024, (u32x4){Mr[4 * j], Mr[4 * j + 1], Mr[4 * j + 2], Mr[4 * j + 3]});
    if (code == RAWCODE) {
#pragma unroll
        for (int j = 0; j < 8; ++j) st_sys(dst + j * 1024, (u32x4){Er[4 * j], Er[4 * j + 1], Er[4 * j + 2], Er[4 * j + 3]});
    } else {
        st_sys(dst, (u32x4){Cr[0], Cr[1], Cr[2], Cr[3]}); st_sys(dst + 1024, (u32x4){Cr[4], Cr[5], Cr[6], Cr[7]});
        if (lane < code) st_sys(dst + 2048, lds128(scr + E_VAR + lane * 16));
        if (lane + 64 < code) st_sys(dst + 3072, lds128(scr + E_VAR + 1024 + lane * 16));
    }
}
template <bool SAMPLED, class Side>
__device__ __forceinline__ void pack_group(const Params& P, const u32 (&v)[64], u32 gid, u32 w, bool solo, u32 peers, u32 call, u32 wid, u32 lane, const Side& side, u32*& fp, u32& fv) {
    u32 Mr[32], Er[32], Cr[8];
    const u32 scr = wid * 16384;
    const u32 code = rfl(pack_chunk<SAMPLED>(v, Mr, Er, Cr, scr, lane, solo, side));
    u32 off = w * RAWB, mine = code;
    if (!solo) {
        if (lane == 0) lds32vw(CTL_SIZE + wid * 4, code | gid << 8);
        for (;;) { mine = lds32v(CTL_SIZE + (lane & 7) * 4); if (__all((mine >> 8) == gid || (lane & 7) > wid)) break; __builtin_amdgcn_s_sleep(1); }
        mine &= 255u; off = 0;
#pragma unroll
        for (u32 j = 0; j < 7; ++j) { const u32 c = (u32)__builtin_amdgcn_readlane((int)mine, j); if (j < wid) off += chunk_bytes(c); }
        if (wid == 7) {
            const u32x4 z4 = zero4();
#pragma unroll
            for (int j = 0; j < 4; ++j) lds128w(LDS_HIST + j * 1024 + lane * 16, z4);
        }
    }
    char* wq = (char*)(GAS char*)__builtin_bit_cast(u64, lds64v(LDS_TAB + (lane >> 3) * 8));
    pack_store(P.me + WS_STG + (call & 1) * STG_PAR + (size_t)gid * GRP_STRIDE + off + lane * 16, code, Mr, Er, Cr, scr, lane);
    fp = (solo ? (lane & 7) == w : wid == 7) && ((peers >> (lane >> 3)) & 1) ? (u32*)(wq + WS_FLAG + (call & 1) * FLAG_PAR + (size_t)P.rank * FLAG_SRC + gid * 32) + (lane & 7) : nullptr;
    fv = (call << 8) | mine;
    WAIT_LGKM0();
}
__device__ __forceinline__ void raise_flags(u32* fp, u32 fv) { if (fp) __hip_atomic_store((u32*)(GAS u32*)fp, fv, __ATOMIC_RELAXED, __HIP_MEMORY_SCOPE_SYSTEM); }

template <bool WRAP> struct Unp {
    u32 sel[4], lo[4], hi[4], of[4];
    __device__ __forceinline__ void fetch(u32 c, u32& off) {
#pragma unroll
        for (int j = 0; j < 4; ++j) {
            const u32 cj = (c >> (2 * j)) & 0x03030303u, mk = escmask(cj), m2 = mk & 0x02020202u;
            const u32 a = off & ~3u; lo[j] = lds32(WRAP ? (a & RMASK) : a); hi[j] = lds32(WRAP ? ((a + 4) & RMASK) : a + 4); of[j] = off;
            u32 p; asm("v_mad_u32_u24 %0, %1, %2, %3" : "=v"(p) : "v"(m2), "s"(0x808080u), "v"(0x04040404u));
            sel[j] = (mk & p) | cj;
            off += popc(m2);
        }
    }
    __device__ __forceinline__ u32 finish(int j, u32 t) const {
        const u32 w = __builtin_amdgcn_alignbyte(hi[j], lo[j], WRAP ? of[j] & 3u : of[j]);
        return perm(w, t, sel[j]);
    }
    __device__ __forceinline__ u32x4 finish4(u32 t) const { return (u32x4){finish(0, t), finish(1, t), finish(2, t), finish(3, t)}; }
};
#define SCHED() __builtin_amdgcn_sched_barrier(0)
template <bool WRAP> __device__ __forceinline__ u32 unpack_round(u32 c, u32 t, u32 src, u32 dst) {
    const u32 cnt = popc(escbits(c)), inc = scan_incl(cnt);
    u32 off = src + inc - cnt;
    Unp<WRAP> u; u.fetch(c, off); SCHED();
    lds128w(dst, u.finish4(t));
    return (u32)__builtin_amdgcn_readlane((int)inc, 63);
}
__device__ __forceinline__ void ring_read8k(u32 a0, u32 (&r)[32]) {
#pragma unroll
    for (int j = 0; j < 8; ++j) { const u32x4 q = lds128((a0 + j * 1024) & RMASK); r[4 * j] = q.x; r[4 * j + 1] = q.y; r[4 * j + 2] = q.z; r[4 * j + 3] = q.w; }
}
__device__ __forceinline__ void unpack_exp(u32 S, u32 code, u32 scr, u32 lane, u32 (&Er)[32]) {
    const u32 a0 = S + lane * 16;
    if (code == RAWCODE) { ring_read8k(a0, Er); return; }
    const u32x4 ca = lds128(a0 & RMASK), cb = lds128((a0 + 1024) & RMASK);
    const u32 V = S + 2048;
    const u32x4 hd = lds128(V & RMASK);
    const u32 t1 = rfl(hd.x), t2 = rfl(hd.y), t3 = rfl(hd.z), t4 = rfl(hd.w), d2 = t1 & 255u, d3 = t2 & 127u, d4 = t3 & 63u;
    u32 c2[4], c3[2];
#pragma unroll
    for (int r = 0; r < 4; ++r) { const u32 i = r * 64 + lane; const u32 x = lds32((V + 16 + 4 * i) & RMASK); c2[r] = i < d2 ? x : 0x55555555u; }
#pragma unroll
    for (int r = 0; r < 2; ++r) { const u32 i = r * 64 + lane; const u32 x = lds32((V + 16 + 4 * (d2 + i)) & RMASK); c3[r] = i < d3 ? x : 0x55555555u; }
    const u32 x4 = lds32((V + 16 + 4 * (d2 + d3 + lane)) & RMASK), c4 = lane < d4 ? x4 : 0x55555555u;
    unpack_round<true>(c4, t4, V + 16 + 4 * (d2 + d3 + d4), scr + D_L4 + lane * 16);
    CBAR();
    const u32 s4 = scr + D_L4 + unpack_round<false>(c3[0], t3, scr + D_L4, scr + D_L3 + lane * 16);
    if (d3 > 64) unpack_round<false>(c3[1], t3, s4, scr + D_L3 + 1024 + lane * 16);
    CBAR();
    const u32 C[8] = {ca.x, ca.y, ca.z, ca.w, cb.x, cb.y, cb.z, cb.w};
    u32 cnt = 0;
#pragma unroll
    for (int k = 0; k < 8; ++k) cnt += popc(escbits(C[k]));
    {
        const u32 n0 = popc(escbits(c2[0])), n1 = popc(escbits(c2[1]));
        const u32 i01 = scan_incl(n0 | (n1 << 16)), tot = (u32)__builtin_amdgcn_readlane((int)i01, 63);
        cnt = scan_incl(cnt) - cnt;
        u32 o0 = scr + D_L3 + (i01 & 0xffffu) - n0, o1 = scr + D_L3 + (tot & 0xffffu) + (i01 >> 16) - n1;
        Unp<false> u0, u1; u0.fetch(c2[0], o0); u1.fetch(c2[1], o1); SCHED();
        lds128w(scr + D_L2 + lane * 16, u0.finish4(t2)); lds128w(scr + D_L2 + 1024 + lane * 16, u1.finish4(t2));
        if (d2 > 128) {
            u32 s3 = scr + D_L3 + (tot & 0xffffu) + (tot >> 16);
            s3 += unpack_round<false>(c2[2], t2, s3, scr + D_L2 + 2048 + lane * 16);
            if (d2 > 192) unpack_round<false>(c2[3], t2, s3, scr + D_L2 + 3072 + lane * 16);
        }
    }
    CBAR();
    u32 off = scr + D_L2 + cnt;
    Unp<false> u[8];
    u[0].fetch(C[0], off); u[1].fetch(C[1], off);
#pragma unroll
    for (int k = 0; k < 8; ++k) {
        SCHED();
        if (k + 2 < 8) u[k + 2].fetch(C[k + 2], off);
        SCHED();
#pragma unroll
        for (int j = 0; j < 4; ++j) Er[4 * k + j] = u[k].finish(j, t1);
    }
    CBAR();
}
__device__ __forceinline__ u32 x_piece_off(u32 half, int p) { return half * 16384 + p * 1024; }
__device__ __forceinline__ u32 w_piece_off(u32 h, int p) {
    const u32 g32 = p >> 2, n = (p >> 1) & 1, kb = p & 1;
    return (g32 & 1) * 16384 + ((2 * (2 * h + (g32 >> 1)) + n) * 2 + kb) * 1024;
}
__device__ __forceinline__ u16* y_base(const Params& P, u32 pm, u32 j, u32 w, u32 lane) {
    return P.out + (size_t)(pm * 256 + (w >> 2) * 64 + (lane >> 3)) * NN + j * 256 + (w & 3) * 64 + (lane & 7) * 8;
}
__device__ __forceinline__ size_t y_piece_off(int p) { return (size_t)((p >> 3) * 128 + (p & 7) * 8) * NN; }
__device__ __forceinline__ u32 local_panel(u32 j, u32 rank) { return j >= (u32)(NR * DP) ? j - (NR - 1) * DP : j - rank * DP; }
__device__ __forceinline__ u32 global_panel(u32 jj, u32 rank) { return jj >= (u32)DP ? jj + (NR - 1) * DP : jj + rank * DP; }
__device__ __forceinline__ u32* counter(const Params& P, u32 i) { return (u32*)(P.me + WS_CNT + i * 128); }
__device__ __forceinline__ u32* tmo_word(const Params& P) { return (u32*)(P.me + WS_TMO); }
__device__ __forceinline__ bool expired(const Params& P, u64 t0, u32 call) {
    if (__hip_atomic_load(tmo_word(P), __ATOMIC_RELAXED, __HIP_MEMORY_SCOPE_AGENT) == call || now() - t0 > DEADLINE) {
        __hip_atomic_store(tmo_word(P), call, __ATOMIC_RELAXED, __HIP_MEMORY_SCOPE_AGENT); return true; }
    return false;
}

struct ImageStore {
    char* o; u32* cnt; u32 half; bool isx, on;
    __device__ __forceinline__ ImageStore(const Params& P, u32 gid, u32 w, u32 q, u32 lane, bool on_) {
        isx = gid < (u32)GW; half = w & 1; on = on_;
        const u32 pn = isx ? gid & 15 : local_panel((gid - GW) >> 1, P.rank), ktl = (isx ? gid >> 4 : gid & 1) * 4 + (w >> 1);
        const u32 qi = (q - P.rank - 1) & 7, stage = qi == 7 ? 0u : ktl == 7 ? 5u : 1 + (ktl >> 1);
        const u32 slot = stage == 0 ? ktl : 8 + ktl * 7 + qi;
        o = P.me + (isx ? WS_XI : WS_WI) + (size_t)pn * PANEL + (size_t)slot * 32768 + lane * 16;
        cnt = counter(P, isx ? pn : 16 + pn) + stage;
    }
    __device__ __forceinline__ void operator()(int p, u32x4 x) const { st_dev(o + (isx ? x_piece_off(half, p) : w_piece_off(half, p)), x); }
    __device__ __forceinline__ void done(u32 lane) const { WAIT_VM(0); if (lane == 0) __hip_atomic_fetch_add(cnt, 1u, __ATOMIC_RELAXED, __HIP_MEMORY_SCOPE_AGENT); }
};
struct OutStore {
    u16* o; static constexpr bool on = true;
    __device__ __forceinline__ void operator()(int p, u32x4 x) const { st_out(o + y_piece_off(p), x); }
};
__device__ __forceinline__ u32* call_word(const Params& P, u32 wid) { return (u32*)(P.me + WS_CALL + blockIdx.x * 128 + wid * 4); }
__device__ __forceinline__ void call_load(const Params& P, u32 wid, u32 lane) {
    u32 keep;
    if (lane == 0) asm volatile("s_mov_b32 %0, m0\n\ts_mov_b32 m0, %2\n\ts_nop 0\n\tglobal_load_lds_dword %1, off sc1\n\ts_mov_b32 m0, %0" : "=&s"(keep) : "v"(call_word(P, wid)), "s"(CTL_CALL + wid * 4) : "memory");
}
__device__ __forceinline__ u32 call_take(u32 wid) { WAIT_VM(0); __builtin_amdgcn_s_waitcnt(0x0f70);   return rfl(lds32v(CTL_CALL + wid * 4)) + 1; }
__device__ __forceinline__ void call_bump(const Params& P, u32 wid, u32 lane, u32 call) { if (lane == 0) __hip_atomic_store(call_word(P, wid), call, __ATOMIC_RELAXED, __HIP_MEMORY_SCOPE_AGENT); }

__device__ __forceinline__ u32 solo_gid(u32 c) { return (c >> 2) & 15; }
__device__ __forceinline__ u32 solo_w(u32 c) { return (c & 3) | (c >> 6) << 2; }
__device__ __forceinline__ void load_x(const Params& P, u32 gid, u32 w, u32 lane, u32 (&v)[64]) {
    const u16* s = P.x + (size_t)((gid & 15) * 256 + (w & 1) * 128 + (lane >> 3)) * KS + ((gid >> 4) * 4 + (w >> 1)) * 64 + ((lane ^ (lane >> 3)) & 7) * 8;
#pragma unroll
    for (int p = 0; p < 16; ++p) { const u32x4 t = *(const u32x4*)(s + p * 8 * KS); v[4 * p] = t.x; v[4 * p + 1] = t.y; v[4 * p + 2] = t.z; v[4 * p + 3] = t.w; }
}
__device__ __forceinline__ void puller(const Params& P, u32 role, u32 wid, u32 lane) {
    const u32 q = role & 7, sub = (role >> 3) & 15, PW = (role >> 7) & 15;
    if (wid >= 2) for (u32 pc = sub + PW * (7 - wid); pc < (u32)NRAW; pc += 6 * PW) {
        const u32 gid = solo_gid(pc), w = solo_w(pc);
        u32 v[64], Mr[32], Er[32];
        load_x(P, gid, w, lane, v);
        const u32 call = call_take(wid);
        split(v, Mr, Er);
        char* o = P.peer + WS_INB + (size_t)(((call & 1) * 8 + P.rank) * 64 + pc) * 8192 + lane * 16;
#pragma unroll
        for (int j = 0; j < 8; ++j) st_sys(o + j * 1024, (u32x4){Mr[4 * j], Mr[4 * j + 1], Mr[4 * j + 2], Mr[4 * j + 3]});
        WAIT_VM(0);
        if (lane == 0) __hip_atomic_store((u32*)(P.peer + WS_FLAG + (call & 1) * FLAG_PAR + (size_t)P.rank * FLAG_SRC + (GM + gid) * 32) + w, call << 8, __ATOMIC_RELAXED, __HIP_MEMORY_SCOPE_SYSTEM);
    }
    const u32 call = call_take(wid);
    call_bump(P, wid, lane, call);
    if (wid == 0) {
        __builtin_amdgcn_s_setprio(3);
        const char* base = P.peer + WS_STG + (call & 1) * STG_PAR;
        const u32 voff = lane * 16;
        u32 pos = 0, pub = 0, limit = RING / 1024;
        for (u32 ui = 0;; ++ui) {
            u32 nu = rfl(lds32v(CTL_NUNIT));
            if ((nu & ~DONE) <= ui) {
                WAIT_VM(0); lds32vw(CTL_LANDED, pos); pub = pos;
                while (nu <= ui) { __builtin_amdgcn_s_sleep(1); nu = rfl(lds32v(CTL_NUNIT)); }
                if ((nu & ~DONE) <= ui) break;
            }
            CBAR();
            const u32x2 un = lds64v(D_UNIT + ui * 8);
            const u32 ut = rfl(un.y), total = ut & 0x3ffffu, lastc = ut >> 18;
            const char* src = base + rfl(un.x);
            for (u32 o = 0; o < total;) {
                if (pos + 4 > limit) {
                    bool drained = false;
                    for (;;) {
                        const u32 nx = lds32v(CTL_NEXT + (lane & 7) * 4);
                        u32 f = lastc;
#pragma unroll
                        for (u32 j = 0; j < ND; ++j) f = umin(f, (u32)__builtin_amdgcn_readlane((int)nx, j));
                        limit = (rfl(lds32v(D_DESC + f * 8)) >> 14) + RING / 1024;
                        if (pos + 4 <= limit) break;
                        if (!drained) { WAIT_VM(0); lds32vw(CTL_LANDED, pos); pub = pos; drained = true; }
                        __builtin_amdgcn_s_sleep(1);
                    }
                }
                const u32 dst = (pos * 1024) & RMASK;
                if (o + 4096 <= total && dst + 4096 <= RING) {
                    u32 keep;
                    asm volatile("s_mov_b32 %0, m0\n\ts_mov_b32 m0, %3\n\ts_nop 4\n\t"
                                 "global_load_lds_dwordx4 %1, %2 sc0 sc1\n\ts_waitcnt vmcnt(%4)\n\t"
                                 "global_load_lds_dwordx4 %1, %2 offset:1024 sc0 sc1\n\ts_waitcnt vmcnt(%4)\n\t"
                                 "global_load_lds_dwordx4 %1, %2 offset:2048 sc0 sc1\n\ts_waitcnt vmcnt(%4)\n\t"
                                 "global_load_lds_dwordx4 %1, %2 offset:3072 sc0 sc1\n\ts_waitcnt vmcnt(%4)\n\t"
                                 "s_mov_b32 m0, %0" : "=&s"(keep) : "v"(voff), "s"(src + o), "s"(dst), "i"(WIN) : "memory");
                    pos += 4; o += 4096;
                } else {
                    if (o + voff < total) glds_sys(src + o + voff, dst);
                    WAIT_VM(WIN);
                    pos += 1; o += 1024;
                }
                if (pos > pub + WIN) { pub = pos - WIN; lds32vw(CTL_LANDED, pub); }
            }
        }
    } else if (wid == 1) {
        u32 basek = 0, nch = 0, nun = 0;
        u32* ticket = (u32*)(P.me + WS_CLAIM + q * 128);
        const u32 nown = P.nunits & 0xffffu, nall = nown + (P.nunits >> 16);
        u32 tbase = NSTAT * PW - (call - 1) * (nown - NSTAT * PW + PW), tend = nown;
        for (u32 ui = 0;; ++ui) {
            while (rfl(lds32v(CTL_LANDED)) + LEAD < basek) __builtin_amdgcn_s_sleep(4);
            u32 idx = ui * PW + sub;
            if (ui >= (u32)NSTAT) {
                if (lane == 0) idx = __hip_atomic_fetch_add(ticket, 1u, __ATOMIC_RELAXED, __HIP_MEMORY_SCOPE_AGENT);
                idx = rfl(idx) + tbase;
                if (idx >= tend) {
                    if (tend == nall) break;
                    ticket = (u32*)(P.me + WS_CLAIM + P.rank * 128); tbase = nown - (call - 1) * (nall - nown + NPULL); tend = nall;
                    continue;
                }
            }
            const u32 e = tab16(TAB_STREAM, idx), gid = e & 1023u, part = e >> 10, lv = 31 - __builtin_clz(part), n = 8u >> lv, first = (part - (1u << lv)) * n;
            if (nch + n > (u32)(NS * 8)) break;
            const bool solo = gid < (u32)SOLOG, mine = lane >= first && lane < first + n, sent = gid < (u32)(NRAW / 4) && first < 4;
            const u32* f = (const u32*)(P.me + WS_FLAG + (call & 1) * FLAG_PAR + (size_t)q * FLAG_SRC + (gid + (sent && (lane & 8) ? GM : 0)) * 32) + (lane & 7);
            const u64 t0 = now(); u32 val; bool dead = false;
            for (;;) {
                val = __hip_atomic_load(f, __ATOMIC_RELAXED, __HIP_MEMORY_SCOPE_SYSTEM);
                if (__all((val >> 8) == (call & 0xffffffu) || (lane & 7) >= first + n || (solo && (lane & 7) < first))) break;
                if (__any(expired(P, t0, call))) { dead = true; break; }
                __builtin_amdgcn_s_sleep(1);
            }
            if (dead) {
                const u32 t = gid - GY;
                if (lane == 0 && t < (u32)(16 * PK)) *y_base(P, t & 15, global_panel(t >> 4, q), first, 0u) = 0x7fc0;
                break;
            }
            u32 code = val & 255u; if (code > 127u) code = RAWCODE;
            const u32 cb = chunk_bytes(code) - (sent ? 8192u : 0u), sz = solo ? (mine ? (cb + 1023u) & ~1023u : 0u) : (lane < first + n ? cb : 0u);
            u32 inc = sz; inc += dpp<0x111, 0xf, true>(inc); inc += dpp<0x112, 0xf, true>(inc); inc += dpp<0x114, 0xf, true>(inc);
            const u32 a0 = (u32)__builtin_amdgcn_readlane((int)(inc - sz), (int)first) & ~127u, total = (u32)__builtin_amdgcn_readlane((int)inc, 7) - a0;
            if (mine) lds64vw(D_DESC + (nch + lane - first) * 8, (u32x2){((basek * 1024 + inc - sz - a0) << 4) | code, gid * 8 + lane});
            if (solo) { if (mine) lds64vw(D_UNIT + (nun + lane - first) * 8, (u32x2){gid * (u32)GRP_STRIDE + lane * RAWB, cb | ((nch + lane - first) << 18)}); nun += n; }
            else { if (lane == 0) lds64vw(D_UNIT + nun * 8, (u32x2){gid * (u32)GRP_STRIDE + a0, total | ((nch + n - 1) << 18)}); nun += 1; }
            nch += n;
            WAIT_LGKM0();
            lds32vw(CTL_NUNIT, nun); lds32vw(CTL_NCHUNK, nch);
            basek += (total + 1023) >> 10;
        }
        lds32vw(CTL_NUNIT, nun | DONE); lds32vw(CTL_NCHUNK, nch | DONE);
    } else {
        const u32 dw = wid - 2, scr = D_SCR + dw * D_SCRB;
        for (u32 c = dw;; c += ND) {
            u32 nc = rfl(lds32v(CTL_NCHUNK));
            while (nc <= c) { __builtin_amdgcn_s_sleep(2); nc = rfl(lds32v(CTL_NCHUNK)); }
            if ((nc & ~DONE) <= c) break;
            CBAR();
            const u32x2 de = lds64v(D_DESC + c * 8);
            const u32 d = rfl(de.x), code = d & 255u, S = (d >> 8) << 4, gid = rfl(de.y) >> 3, w = rfl(de.y) & 7;
            const u32 SM = S + chunk_bytes(code) - 8192;
            while (rfl(lds32v(CTL_LANDED)) < (SM + 1023) >> 10) __builtin_amdgcn_s_sleep(1);
            CBAR();
            u32 Mr[32], Er[32];
            unpack_exp(S, code, scr, lane, Er);
            if (gid < (u32)(NRAW / 4) && w < 4) {
                const char* m = P.me + WS_INB + (size_t)(((call & 1) * 8 + q) * 64 + gid * 4 + w) * 8192 + lane * 16;
                u32x4 r[8];
                asm volatile("global_load_dwordx4 %0, %8, off sc0 sc1\n\tglobal_load_dwordx4 %1, %8, off offset:1024 sc0 sc1\n\t"
                             "global_load_dwordx4 %2, %8, off offset:2048 sc0 sc1\n\tglobal_load_dwordx4 %3, %8, off offset:3072 sc0 sc1\n\t"
                             "global_load_dwordx4 %4, %9, off sc0 sc1\n\tglobal_load_dwordx4 %5, %9, off offset:1024 sc0 sc1\n\t"
                             "global_load_dwordx4 %6, %9, off offset:2048 sc0 sc1\n\tglobal_load_dwordx4 %7, %9, off offset:3072 sc0 sc1\n\ts_waitcnt vmcnt(0)"
                             : "=&v"(r[0]), "=&v"(r[1]), "=&v"(r[2]), "=&v"(r[3]), "=&v"(r[4]), "=&v"(r[5]), "=&v"(r[6]), "=&v"(r[7]) : "v"(m), "v"(m + 4096) : "memory");
#pragma unroll
                for (int j = 0; j < 8; ++j) { Mr[4 * j] = r[j].x; Mr[4 * j + 1] = r[j].y; Mr[4 * j + 2] = r[j].z; Mr[4 * j + 3] = r[j].w; }
            } else {
                while (rfl(lds32v(CTL_LANDED)) < (SM + 9215) >> 10) __builtin_amdgcn_s_sleep(1);
                CBAR();
                ring_read8k(SM + lane * 16, Mr);
            }
            WAIT_LGKM0(); lds32vw(CTL_NEXT + dw * 4, c + ND); CBAR();
            if (gid >= (u32)GY) {
                const u32 t = gid - GY;
                u16* o = y_base(P, t & 15, global_panel(t >> 4, q), w, lane);
#pragma unroll
                for (int p = 0; p < 16; ++p) st_out(o + y_piece_off(p), piece(Mr, Er, p));
            } else {
                const ImageStore im(P, gid, w, q, lane, true);
#pragma unroll
                for (int p = 0; p < 16; ++p) im(p, piece(Mr, Er, p));
                im.done(lane);
            }
        }
    }
}

__device__ __forceinline__ void pack_inputs(const Params& P, u32 gid, u32 w, bool solo, u32 wid, u32 lane) {
    asm volatile("" : "+v"(lane));
    u32 v[64];
    u32 peers = 255u & ~(1u << P.rank); bool own = true;
    if (gid < (u32)GW) load_x(P, gid, w, lane, v);
    else {
        const u32 j = (gid - GW) >> 1, ktl = (gid & 1) * 4 + (w >> 1);
        if (j < (u32)(NR * DP)) { const u32 dest = j / DP; own = dest == (u32)P.rank; peers = own ? 0u : 1u << dest; }
        else if (j < (u32)(NR * (DP + FX))) own = (j & 7) != P.rank;
        const u16* s = P.w + (size_t)(ktl * 64 + (lane >> 4)) * NN + j * 256 + (w & 1) * 128 + (lane & 15) * 8;
        const u32 scr = wid * 16384;
#pragma unroll
        for (int i = 0; i < 16; ++i) glds_w(s + (size_t)i * 4 * NN, scr + i * 1024);
        WAIT_VM(0);
        const u32 ta = scr + ((lane >> 4) * 8 + ((lane & 15) >> 2)) * 256 + (lane & 3) * 16;
#pragma unroll
        for (int p = 0; p < 16; ++p) {
            const u32 a = ta + (p & 1) * 32 * 256 + (p >> 2) * 64 + ((p >> 1) & 1) * 8;
            const u32x2 lo = __builtin_bit_cast(u32x2, __builtin_amdgcn_ds_read_tr16_b64_v4i16((LAS s16x4*)a));
            const u32x2 hi = __builtin_bit_cast(u32x2, __builtin_amdgcn_ds_read_tr16_b64_v4i16((LAS s16x4*)(a + 4 * 256)));
            v[4 * p] = lo.x; v[4 * p + 1] = lo.y; v[4 * p + 2] = hi.x; v[4 * p + 3] = hi.y;
        }
        WAIT_LGKM0();
    }
    const u32 call = call_take(wid);
    const ImageStore im(P, gid, w, P.rank, lane, own);
    if (peers) {
        u32* fp; u32 fv;
        pack_group<false>(P, v, gid, w, solo, peers, call, wid, lane, im, fp, fv);
        WAIT_VM(0);
        if (!solo) BAR();
        raise_flags(fp, fv);
        if (own && lane == 0) __hip_atomic_fetch_add(im.cnt, 1u, __ATOMIC_RELAXED, __HIP_MEMORY_SCOPE_AGENT);
    } else {
#pragma unroll
        for (int p = 0; p < 16; ++p) im(p, (u32x4){v[4 * p], v[4 * p + 1], v[4 * p + 2], v[4 * p + 3]});
        im.done(lane);
    }
}

typedef __bf16 bf16x2 __attribute__((ext_vector_type(2)));
typedef float f32x2 __attribute__((ext_vector_type(2)));
__device__ __forceinline__ u32 cvt_pk(float lo, float hi) { return __builtin_bit_cast(u32, __builtin_convertvector((f32x2){lo, hi}, bf16x2)); }
__device__ __forceinline__ void wait_panels(const Params& P, u32 tile, u32 upto, u32 call, u32 wid, u32 lane) {
    if (wid == 0) {
        const u32 st = umin(lane & 7, upto), need = call * (st == 0 ? 16u : st < 4 ? 28u : 14u);
        const u32* c = counter(P, (lane & 8) ? 16 + (tile >> 4) : (tile & 15)) + st;
        const u64 t0 = now();
        for (;;) {
            const u32 x = __hip_atomic_load(c, __ATOMIC_RELAXED, __HIP_MEMORY_SCOPE_AGENT);
            if (__all((int)(x - need) >= 0)) break;
            if (__any(expired(P, t0, call))) { asm volatile("ds_write_b32 %0, %1\n\ts_waitcnt lgkmcnt(0)" :: "v"(CTL_BAD), "v"(1u) : "memory"); break; }
            __builtin_amdgcn_s_sleep(4);
        }
        __builtin_amdgcn_fence(__ATOMIC_ACQUIRE, "agent");
        WAIT_VM(0);
    }
    BAR();
}
__device__ __forceinline__ void stage2(u32 dst, u32 voff, const char* src) {
    asm volatile("s_mov_b32 m0, %2\n\ts_nop 0\n\tglobal_load_lds_dwordx4 %0, %1\n\tglobal_load_lds_dwordx4 %0, %1 offset:1024" :: "v"(voff), "s"(src), "s"(dst) : "memory");
}
__device__ __forceinline__ void stage4(u32 dst, u32 voff, const char* src) {
    asm volatile("s_mov_b32 m0, %2\n\ts_nop 0\n\tglobal_load_lds_dwordx4 %0, %1\n\tglobal_load_lds_dwordx4 %0, %1 offset:1024\n\t"
                 "global_load_lds_dwordx4 %0, %1 offset:2048\n\tglobal_load_lds_dwordx4 %0, %1 offset:3072" :: "v"(voff), "s"(src), "s"(dst) : "memory");
}
__device__ __forceinline__ void gemm_role(const Params& P, u32 place, u32 call, u32 wid, u32 lane) {
    const u32 wr = wid >> 2, wc = wid & 3;
    u32 aoff = wr * 8192 + (lane & 15) * 128 + (((lane >> 4) ^ lane) & 7) * 16, aoff1 = aoff ^ 64, boff = 65536 + wc * 4096 + lane * 16;
    const u32 mA = wid * 2048, mB = 65536 + wid * 4096;
    u32 vA0 = mA + lane * 16, vA1 = vA0 + 16384, vA2 = vA0 + 32768, vA3 = vA0 + 49152, vB0 = wid * 4096 + lane * 16, vB1 = vB0 + 32768;
    asm volatile("" : "+v"(vA0), "+v"(vA1), "+v"(vA2), "+v"(vA3), "+v"(vB0), "+v"(vB1), "+v"(aoff), "+v"(aoff1), "+v"(boff));
#define G_LDA(b, h) do { _Pragma("unroll") for (int m = 0; m < 4; ++m) _Pragma("unroll") for (int k = 0; k < 2; ++k) At[m][k] = *(const LAS bf16x8*)((k ? aoff1 : aoff) + ((b) * 2 + (h)) * 16384 + m * 2048); } while (0)
#define G_LDB(b) do { _Pragma("unroll") for (int j = 0; j < 2; ++j) _Pragma("unroll") for (int n = 0; n < 2; ++n) _Pragma("unroll") for (int k = 0; k < 2; ++k) \
        Bt[j][n][k] = *(const LAS bf16x8*)(boff + ((b) * 2 + j) * 16384 + n * 2048 + k * 1024); } while (0)
#define G_MMA(ai, STEP) do { G_WV(8); __builtin_amdgcn_s_waitcnt(0xc07f);   G_SCHED; G_BAR; __builtin_amdgcn_s_setprio(1); _Pragma("unroll") for (int i = 0; i < 32; ++i) { \
        const int bj = i >> 4, k = (i >> 3) & 1, m = (i >> 1) & 3, n = i & 1; \
        if (i == 8) { G_SCHED; STEP; G_SCHED; } \
        if (i == 32 - 2) { G_SCHED; __builtin_amdgcn_s_setprio(3); G_BAR; G_SCHED; } \
        acc[ai][bj][m][n] = __builtin_amdgcn_mfma_f32_16x16x32_bf16(Bt[bj][n][k], At[m][k], acc[ai][bj][m][n], 0, 0, 0); } \
        __builtin_amdgcn_s_setprio(0); G_SCHED; } while (0)
#define G_WV(n) asm volatile("s_waitcnt vmcnt(" #n ")" ::: "memory")
#define G_BAR __builtin_amdgcn_s_barrier()
#define G_SCHED __builtin_amdgcn_sched_barrier(0)
    const u32x2 cl = lds64v(TAB_COLS + ((blockIdx.x & 4) * 2 + (place >> 2)) * 8);
    const u64 mycols = (u64)rfl(cl.y) << 32 | rfl(cl.x); const u32 pm = (blockIdx.x & 3) * 4 + (place & 3);
    const char* const cA = P.me + WS_XI + (size_t)pm * PANEL; const char* const WI = P.me + WS_WI;
    u32 cur = pm | ((u32)mycols & 255u) << 4;
    bool fresh = true; u32* fp = nullptr; u32 fv = 0;
    bf16x8 At[4][2], Bt[2][2][2];
    for (u32 ui = 0;; ++ui) {
        const u32 nxt = pm | ((u32)(mycols >> (8 * ui + 8)) & 255u) << 4;
        const bool has_next = (nxt >> 4) != 255u, dist = (cur >> 4) < (u32)PK, chain = has_next && !dist;
        const char* cB = WI + (size_t)(cur >> 4) * PANEL;
        if (fresh) {
            wait_panels(P, cur, ui > 1 ? 5u : 0u, call, wid, lane);
            stage4(mB, vB0, cB); stage2(mA, vA0, cA); stage2(mA + 16384, vA1, cA);
            if (wr == 1) G_BAR;
            G_WV(2); G_BAR;
            stage4(mB + 32768, vB1, cB); stage2(mA + 32768, vA2, cA);
            G_WV(6); G_BAR;
            raise_flags(fp, fv); fp = nullptr; fv = 0;
        }
        f32x4 acc[2][2][4][2];
#pragma unroll
        for (int a = 0; a < 2; ++a)
#pragma unroll
            for (int b = 0; b < 2; ++b)
#pragma unroll
                for (int m = 0; m < 4; ++m)
#pragma unroll
                    for (int n = 0; n < 2; ++n) acc[a][b][m][n] = (f32x4){0.f, 0.f, 0.f, 0.f};
        const char *p0 = cA, *pA = cA + 65536, *pB = cB + 65536;
        u32 stops = 0x80000000u | (ui == 0 ? 0x09020408u : ui == 1 ? 0x00020008u : 0u), upto = 0;
        for (u32 trip = 0;;) {
            const u32 end = stops ? (u32)__builtin_ctz(stops) : 32u;
            u32 left = end - trip;
            do {
                G_LDB(0); G_SCHED; G_LDA(0, 0); stage2(mA + 49152, vA3, p0);
                G_MMA(0, );
                G_LDA(0, 1); stage4(mB, vB0, pB); stage2(mA, vA0, pA);
                G_MMA(1, );
                G_LDB(1); G_SCHED; G_LDA(1, 0); stage2(mA + 16384, vA1, pA);
                G_MMA(0, );
                G_LDA(1, 1); stage4(mB + 32768, vB1, pB); stage2(mA + 32768, vA2, pA);
                G_MMA(1, p0 = pA; pA += 65536; pB += 65536; --left; asm volatile("" : "+s"(p0), "+s"(pA), "+s"(pB), "+s"(left)));
            } while (left);
            if (!stops) break;
            trip = end; stops &= stops - 1;
            if (stops) { upto += ui ? (upto ? 3u : 2u) : 1u; wait_panels(P, cur, upto, call, wid, lane); }
            else {
                pA = cA; pB = cB;
                if (chain) { pB = WI + (size_t)(nxt >> 4) * PANEL; wait_panels(P, nxt, ui ? 5u : 0u, call, wid, lane); }
            }
        }
        if (!chain) G_WV(0);
        if (wr == 0) G_BAR;
        u32 v[64];
        {
            int tz = threadIdx.x; asm volatile("" : "+v"(tz));
            const u32 ln = tz & 63, bad = lds32v(CTL_BAD);
            const u32 ep = D_DESC + wid * 2048, to = ep + (ln & 15) * 128 + (((ln >> 4) ^ (ln & 7)) * 16), from = ep + (ln ^ (ln >> 3)) * 16;
            u16* o = y_base(P, cur & 15, global_panel(cur >> 4, P.rank), wid, ln);
#pragma unroll
            for (int p = 0; p < 16; p += 2) {
                const int ai = p >> 3, m = (p >> 1) & 3;
#pragma unroll
                for (int bj = 0; bj < 2; ++bj) {
                    const f32x4 x0 = acc[ai][bj][m][0], x1 = acc[ai][bj][m][1];
                    lds128w(to ^ (bj * 64), (u32x4){bad ? 0x7fc07fc0u : cvt_pk(x0[0], x0[1]), cvt_pk(x0[2], x0[3]), cvt_pk(x1[0], x1[1]), cvt_pk(x1[2], x1[3])});
                }
                CBAR();
#pragma unroll
                for (int j = 0; j < 2; ++j) { const u32x4 t = lds128(from + j * 1024); v[4 * (p + j)] = t.x; v[4 * (p + j) + 1] = t.y; v[4 * (p + j) + 2] = t.z; v[4 * (p + j) + 3] = t.w; }
                CBAR();
            }
            if (!dist) {
#pragma unroll
                for (int p = 0; p < 16; ++p) st_out(o + y_piece_off(p), (u32x4){v[4 * p], v[4 * p + 1], v[4 * p + 2], v[4 * p + 3]});
            }
            if (chain) { if (wr == 1) G_BAR; }
            else {
                if (dist) {
                    sample<6, 10, 5>(v, LDS_HIST, ln);
                    WAIT_LGKM0(); BAR();
                    const OutStore os = {o};
                    pack_group<true>(P, v, GY + cur, wid, false, (cur >> 4) < (u32)DP ? 255u & ~(1u << P.rank) : 1u << (((cur >> 4) - DP) & 7), call, wid, ln, os, fp, fv);
                    if (!has_next) { G_WV(0); BAR(); raise_flags(fp, fv); }
                }
                if (!has_next) break;
            }
        }
        fresh = !chain; cur = nxt;
    }
}

__global__ void __launch_bounds__(512, 2) fused(const Args A) {
    const u32 wid = rfl(threadIdx.x >> 6), lane = threadIdx.x & 63;
    if (wid == 3) {
        u64 pc; u32 len;
        asm volatile("s_getpc_b64 %0\ngar_code0:\n\ts_mov_b32 %1, gar_code1-gar_code0" : "=s"(pc), "=s"(len));
        for (u32 o = (blockIdx.x >> 3) * 1024 + lane * 16; o < len; o += 32768) glds_plain((const char*)(const GAS char*)(pc & ~15ull) + o, LDS_CTL - 1024);
    }
    const u32 role = pin((((const u32*)A.role)[blockIdx.x >> 1] >> ((blockIdx.x & 1) * 16)) & 0xffffu);
    char* peer = pin(A.ws[0]);
#pragma unroll
    for (u32 j = 1; j < NR; ++j) { char* c = pin(A.ws[j]); peer = (role & 7) == j ? c : peer; }
    const Params P = {pin(A.me), peer, pin(A.x), pin(A.w), pin(A.out), pin((u32)A.rank), pin((u32)A.nunits)};
    call_load(P, wid, lane);
    if (wid == 1 || wid == 2) {
        const char* ka = (const char*)(const GAS char*)(u64)__builtin_amdgcn_kernarg_segment_ptr() + lane * 16;
        if (wid == 1 && !(role & 0x8000u)) { if (lane < 4) glds_plain(ka, LDS_TAB); }
        else if ((wid == 1) == ((role & 0x8000u) != 0)) { glds_plain(ka, LDS_TAB); glds_plain(ka + 1024, LDS_TAB + 1024); glds_plain(ka + 2048, LDS_TAB + 2048); }
    }
    if (role & 0x8000u) {
        if (threadIdx.x == 0) { lds32vw(CTL_NUNIT, 0u); lds32vw(CTL_LANDED, 0u); lds32vw(CTL_NCHUNK, 0u); }
        if (wid == 1 && lane < 8) lds32vw(CTL_NEXT + lane * 4, lane);
        WAIT_LGKM0(); BAR();
        puller(P, role, wid, lane); return;
    }
    if (threadIdx.x == 0) lds32vw(CTL_BAD, 0u);
    if (threadIdx.x < 8) lds32vw(CTL_SIZE + threadIdx.x * 4, 0u);
    pack_init(wid, lane);
    const u32 rb = NENC - 1 - (role & 255u), sc = rb + (wid & 2) * 32;
    if (rb >= 64u) { const u64 t0 = now(); while (now() - t0 < 100) __builtin_amdgcn_s_sleep(2); }
    for (u32 it = 0; it < 3; ++it) {
        const u32 gid = it ? tab16(TAB_ENC, (role & 255u) * 2 + it - 1) : solo_gid(sc);
        if (gid == 0xffffu) break;
        if (it || ((wid == 1 || wid == 2) && rb < 64u)) pack_inputs(P, gid, it ? wid : solo_w(sc), !it, wid, lane);
        WAIT_VM(0); WAIT_LGKM0(); BAR();
    }
    const u32 call = call_take(wid);
    call_bump(P, wid, lane, call);
    if (role >> 8) gemm_role(P, (role >> 8) - 1, call, wid, lane);
    asm volatile("gar_code1:");
}

static void build_tables(Args& P) {
    unsigned char* cols = P.cols; u16 *stream = P.stream, *role = P.role, *job = P.enc, enc[NPACK];
    {
        int peer[2][4], np[2] = {0, 0}, cnt[8] = {}, seen[2] = {0, 0}, n = 0, top = NENC;
        for (int d = 1; d < NR; ++d) { const int q = (P.rank + d) & 7, h = ((0xb2 >> q) ^ (0xb2 >> P.rank)) & 1; peer[h][np[h]++] = q; }
        for (int b = 0; b < NWG; ++b) {
            const int h = (b >> 2) & 1, i = b >> 3, g0 = 32 - 4 * (h ? CH : CL), near = h ? 4 * NH : 3 * NL, far = h ? 3 * FL : 4 * FH, k = seen[h]++;
            if (k >= near + far) { role[b] = i >= g0 ? (u16)(n++ | (i - g0 + 1) << 8) : (u16)--top; continue; }
            const int side = k < near ? h : !h, j = k < near ? k : k - near, q = peer[side][side ? (j + j / 4) & 3 : j % 3];
            role[b] = (u16)(0x8000 | q | cnt[q]++ << 3);
        }
        for (int b = 0; b < NWG; ++b) if (role[b] & 0x8000) role[b] |= (u16)(cnt[role[b] & 7] << 7);
    }
    int col[LC], nc = 0;
    {
        const int rest = LC - (ROUNDS - 1) * CPR, r0 = (rest + 1) / 2, light = rest < CPR ? CPR - rest : CPR;
        int over = PK - FX - light * (ROUNDS - 1), nd = 0, nr = PK;
        for (int i = 0; i < 128; ++i) cols[i] = 0xff;
        for (int r = 0; r < ROUNDS; ++r) for (int c = 0; c < 8; ++c) for (int cg = 0; cg < 2; ++cg) {
            const bool last = c < (cg ? rest - r0 : r0), heavy = last && rest < CPR;
            if (nd >= DP && nd < PK && ((nd - DP) & 7) == P.rank) ++nd;
            if (c >= (cg ? CH : CL) || (r + 1 == ROUNDS && !last)) continue;
            cols[(cg * 8 + c) * 8 + r] = (unsigned char)(col[nc++] = nd < PK && (!heavy || over-- > 0) ? nd++ : nr++);
        }
    }
    auto wgid = [&](int jj, int dest, int h) { return GW + 2 * (jj >= DP ? jj + (NR - 1) * DP : dest * DP + jj) + h; };
    int k = 0;
    auto put = [&](int gid, int lv) { for (int j = 0; j < (1 << lv); ++j) stream[k++] = (u16)(gid | ((1 << lv) + j) << 10); };
    auto opnd = [&](int i, int h) { return i < 16 ? GX + 16 * h + i : wgid(col[i - 16], P.rank, h); };
    auto stage = [&](int st, int i0, int i1) {
        const int h = st >> 1, half = st & 1;
        for (int i = i0; i < i1; ++i) {
            if (i < 16 && h == 0) { for (int j = 0; j < 2; ++j) stream[k++] = (u16)(opnd(i, h) | (4 + 2 * half + j) << 10); }
            else stream[k++] = (u16)(opnd(i, h) | (2 + half) << 10);
        }
    };
    stage(0, NRAW / 4, 16); stage(1, 0, 12);
    {
        u16 t[64]; const int n = k; int e = 0;
        for (int i = 0; i < n; ++i) t[i] = stream[i];
        k = 0;
        for (int i = 0; i < n || e < NRAW / 4; ++i) { if (i < n) stream[k++] = t[i]; if (i >= 18 && e < NRAW / 4) stream[k++] = (u16)((GX + e++) | 2 << 10); }
    }
    const int first = 16 + CPR - 4;
    stage(0, 16, first); stage(1, 12, first); stage(0, first, 16 + CPR); stage(2, 0, first); stage(1, first, 16 + CPR);
    for (int part = 6; part < 8; ++part) for (int i = 0; i < first; ++i) stream[k++] = (u16)(opnd(i, 1) | part << 10);
    stage(2, first, 16 + CPR); stage(3, first, 16 + CPR);
    for (int h = 0; h < 2; ++h) for (int i = CPR; i < 2 * CPR; ++i) put(wgid(col[i], P.rank, h), 0);
    for (int i = 2 * CPR; i < LC; ++i) for (int h = 0; h < 2; ++h) put(wgid(col[i], P.rank, h), 0);
    for (int t = 0; t < 16 * DP; ++t) { const int left = 16 * DP - t; put(GY + t, left <= 8 ? 1 : 0); }
    const int nown = k;
    for (int t = 0; t < 16 * FX; ++t) put(GY + (DP + (t >> 4) * 8 + P.rank) * 16 + (t & 15), 16 * FX - t <= 16 ? 3 : 16 * FX - t <= 30 ? 2 : 1);
    P.nunits = nown | (k - nown) << 16;
    k = 0;
    for (int i = 0; i < CPR; ++i) for (int d = 1; d < (col[i] < DP ? NR : 2); ++d) enc[k++] = (u16)wgid(col[i], (P.rank + d) & 7, 0);
    for (int i = SOLOG; i < 32; ++i) enc[k++] = (u16)(GX + i);
    for (int i = 0; i < CPR; ++i) { if (col[i] < DP) enc[k++] = (u16)wgid(col[i], P.rank, 0); enc[k++] = (u16)wgid(col[i], P.rank, 1); }
    for (int i = 0; i < CPR; ++i) if (col[i] < DP) for (int d = 1; d < NR; ++d) enc[k++] = (u16)wgid(col[i], (P.rank + d) & 7, 1);
    for (int f = 0; f < FX; ++f) for (int h = 0; h < 2; ++h) enc[k++] = (u16)wgid(DP + f * 8 + P.rank, 0, h);
    for (int i = CPR; i < LC; ++i) for (int h = 0; h < 2; ++h) for (int d = 0; d < (col[i] < DP ? NR : 1); ++d) enc[k++] = (u16)wgid(col[i], (P.rank + d) & 7, h);
    int nx = 0, top = NENC;
    for (int i = 0; i < NENC - 64; ++i) { job[2 * i] = enc[nx++]; job[2 * i + 1] = 0xffff; }
    for (int i = 0; i < NENC - 64; ++i) if (job[2 * i] >= GW && (job[2 * i] - GW) / 2 / DP == P.rank) job[2 * i + 1] = enc[nx++];
    for (int i = NENC - 64; i < NENC; ++i) { job[2 * i] = enc[nx++]; job[2 * i + 1] = 0xffff; }
    while (nx < NPACK) { do --top; while (job[2 * top + 1] != 0xffff); job[2 * top + 1] = enc[nx++]; }
}
}

extern "C" void kernel_launch(int rank, int nranks, ncclComm_t comm, hipStream_t stream,
                              void* const* d_in, const int* in_sizes, int n_in,
                              void* d_out, int out_size, const int* in_dtypes, int out_dtype,
                              void* const* d_out_all, void* const* d_ws_all, size_t ws_bytes) {
    using namespace gar;
    if (nranks != NR || (unsigned)rank >= (unsigned)NR || n_in != 2 || ws_bytes < WS_END || in_sizes[0] != MM * KS || in_sizes[1] != KS * NN || out_size != MM * NN) {
        fprintf(stderr, "kernel_launch: unexpected shapes\n"); return; }
    Args P = {};
    for (int p = 0; p < NR; ++p) P.ws[p] = (char*)d_ws_all[p];
    P.me = P.ws[rank];
    P.x = (const u16*)d_in[0]; P.w = (const u16*)d_in[1]; P.out = (u16*)d_out; P.rank = rank;
    build_tables(P);
    HIP_CHECK(hipFuncSetAttribute((const void*)fused, hipFuncAttributeMaxDynamicSharedMemorySize, LDS_BYTES));
    hipLaunchKernelGGL(fused, dim3(NWG), dim3(512), LDS_BYTES, stream, P);
    HIP_CHECK(hipGetLastError());
}
```

```cpp
#include <hip/hip_runtime.h>
#include <rccl/rccl.h>
#include <cstdio>
#include <cstdint>
#include <cstddef>
#pragma clang diagnostic ignored "-Wint-to-pointer-cast"

#define HIP_CHECK(expr) do { hipError_t _e = (expr); if (_e != hipSuccess) { \
    fprintf(stderr, "HIP error %s at %s:%d\n", hipGetErrorString(_e), __FILE__, __LINE__); \
    return; } } while (0)

namespace gar {
#define LAS __attribute__((address_space(3)))
#define GAS __attribute__((address_space(1)))
typedef unsigned u32;
typedef unsigned short u16;
typedef unsigned long long u64;
typedef unsigned u32x4 __attribute__((ext_vector_type(4)));
typedef unsigned u32x2 __attribute__((ext_vector_type(2)));
typedef short s16x4 __attribute__((ext_vector_type(4)));
typedef short bf16x8 __attribute__((ext_vector_type(8)));
typedef float f32x4 __attribute__((ext_vector_type(4)));

constexpr int NR = 8, MM = 4096, NN = 32768, KS = 512;
constexpr int DP = 10;
constexpr int RP = 128 - NR * DP;
constexpr int LP = DP + RP;
constexpr int NWG = 256;
constexpr int NL = 9, FL = 0, NH = 8, FH = 1, CL = 6, CH = 6, CPR = CL + CH;
constexpr int NPULL = 3 * (NL + FL) + 4 * (NH + FH);
constexpr int WIN = 6;
static_assert(3 * NL + 4 * FH + 16 * CL <= 128 && 4 * NH + 3 * FL + 16 * CH <= 128, "roles");
constexpr int SOLOG = 16, NRAW = 52;
constexpr int LEAD = 44;
static_assert(NRAW <= 64 && NRAW % 4 == 0 && 8 * SOLOG <= NWG - NPULL && SOLOG == 16, "solo chunks");
constexpr int NENC = NWG - NPULL;
constexpr int GX = 0, GW = 32, GY = 288;
constexpr int FX = 2, PK = DP + 8 * FX, LC = LP - FX;
constexpr int NGID = GY + 16 * PK, GM = NGID, NGT = GM + 16;
constexpr int NS = 32 + 2 * LC + 16 * DP + 16 * FX;
constexpr int ROUNDS = (LC + CPR - 1) / CPR;
constexpr int NUMAX = 704;
constexpr int NSTAT = 3;
constexpr int NPACK = 288 - SOLOG;
constexpr u64 DEADLINE = 30000000ull;

constexpr size_t WS_CALL = 0;
constexpr size_t WS_TMO = 32768;
constexpr size_t WS_CNT = 36864;
constexpr size_t WS_CLAIM = 57344;
constexpr size_t WS_FLAG = 65536;
constexpr size_t FLAG_SRC = NGT * 32, FLAG_PAR = 8 * FLAG_SRC;
constexpr size_t GRP_STRIDE = 131072;
constexpr size_t WS_STG = 2u << 20;
constexpr size_t STG_PAR = (NGID * GRP_STRIDE + 0x1fffff) & ~(size_t)0x1fffff;
constexpr size_t WS_INB = WS_STG + 2 * STG_PAR;
constexpr size_t WS_XI = WS_INB + 2 * 8 * 64 * 8192;
constexpr size_t PANEL = 64 * 32768;
constexpr size_t WS_WI = WS_XI + 16 * PANEL;
constexpr size_t WS_END = WS_WI + (size_t)LP * PANEL;
static_assert(WS_FLAG + 2 * FLAG_PAR <= WS_STG && NGT <= 1024 && RP >= 8 * FX, "layout");

constexpr u32 LDS_CTL = 131072;
constexpr u32 LDS_LUT = LDS_CTL + 512, LDS_HIST = LDS_CTL + 1024;
constexpr u32 LDS_TAB = LDS_HIST + 4096, D_DESC = LDS_TAB + 3072, LDS_BYTES = D_DESC + NS * 64;
constexpr u32 CTL_CALL = LDS_CTL  , CTL_SIZE = LDS_CTL + 64  , CTL_BAD = LDS_CTL + 128;
constexpr u32 CTL_NUNIT = LDS_CTL + 192, CTL_LANDED = LDS_CTL + 256, CTL_NCHUNK = LDS_CTL + 320, CTL_NEXT = LDS_CTL + 384  ;
constexpr u32 E_L2 = 0, E_L3 = E_L2 + 4160, E_L4 = E_L3 + 2112, E_VAR = E_L4 + 1088, E_END = E_VAR + 2112;
static_assert(E_END <= 10240 && E_L4 % 16 == 0 && E_L3 % 16 == 0 && E_VAR % 16 == 0, "packer scratch");
constexpr u32 CAP2 = 4080, CAP3 = 2032, CAP4 = 1008, CAPV = 2032, FIXB = 10240, RAWB = 16384, RAWCODE = 255;
constexpr u32 RING = 65536, RMASK = RING - 1, ND = 6;
constexpr u32 D_SCR = RING, D_SCRB = 7360, D_L2 = 0, D_L3 = 4160, D_L4 = D_L3 + 2112;
constexpr u32 D_UNIT = D_SCR + ND * D_SCRB, DONE = 0x80000000u;
static_assert(D_L4 + 1088 <= D_SCRB && D_UNIT + (2 * NUMAX + 8 * SOLOG) * 8 <= LDS_CTL - 2048 && LDS_BYTES <= 163840 && D_DESC % 128 == 0 && NS * 64 >= 16384, "puller LDS");

struct Args {
    char* ws[NR];
    char* me; const u16* x; const u16* w; u16* out;
    int rank, nunits;
    u16 role[NWG];
    u16 stream[NUMAX];
    unsigned char cols[128];
    u16 enc[2 * NENC];
    u16 pad[(3072 - 104 - 128) / 2 - NWG - NUMAX - 2 * NENC];
};
static_assert(sizeof(Args) == 3072 && ROUNDS < 8 && offsetof(Args, cols) % 8 == 0 && offsetof(Args, role) % 4 == 0, "kernarg");
constexpr u32 TAB_STREAM = LDS_TAB + offsetof(Args, stream), TAB_ENC = LDS_TAB + offsetof(Args, enc), TAB_COLS = LDS_TAB + offsetof(Args, cols);
struct Params { char* me; char* peer; const u16* x; const u16* w; u16* out; u32 rank, nunits; };
template <class T> __device__ __forceinline__ T* pin(T* p) { u64 v = (u64)p; asm volatile("" : "+s"(v)); return (T*)(GAS T*)v; }
__device__ __forceinline__ u32 pin(u32 v) { asm volatile("" : "+s"(v)); return v; }

__device__ __forceinline__ u32 lds32(u32 a) { return *(LAS u32*)a; }
__device__ __forceinline__ u32 lds32v(u32 a) { return *(volatile LAS u32*)a; }
__device__ __forceinline__ void lds32vw(u32 a, u32 x) { *(volatile LAS u32*)a = x; }
__device__ __forceinline__ u32x2 lds64v(u32 a) { return *(volatile LAS u32x2*)a; }
__device__ __forceinline__ void lds64vw(u32 a, u32x2 x) { *(volatile LAS u32x2*)a = x; }
__device__ __forceinline__ u32x4 lds128(u32 a) { return *(LAS u32x4*)a; }
__device__ __forceinline__ void lds128w(u32 a, u32x4 x) { *(LAS u32x4*)a = x; }
__device__ __forceinline__ u32x4 zero4() { u32 z; asm volatile("v_mov_b32 %0, 0" : "=v"(z)); return (u32x4){z, z, z, z}; }
__device__ __forceinline__ void lds_or(u32 a, u32 x) { __hip_atomic_fetch_or((LAS u32*)a, x, __ATOMIC_RELAXED, __HIP_MEMORY_SCOPE_WORKGROUP); }
__device__ __forceinline__ void lds_add(u32 a, u32 x) { __hip_atomic_fetch_add((LAS u32*)a, x, __ATOMIC_RELAXED, __HIP_MEMORY_SCOPE_WORKGROUP); }
__device__ __forceinline__ u32 rfl(u32 x) { return (u32)__builtin_amdgcn_readfirstlane((int)x); }
__device__ __forceinline__ u32 tab16(u32 t, u32 i) { return rfl(*(volatile LAS u16*)(t + i * 2)); }
#define CBAR() asm volatile("" ::: "memory")
#define WAIT_LGKM0() asm volatile("s_waitcnt lgkmcnt(0)" ::: "memory")
#define WAIT_VM(n) asm volatile("s_waitcnt vmcnt(%0)" :: "i"(n) : "memory")
#define BAR() do { CBAR(); __builtin_amdgcn_s_barrier(); CBAR(); } while (0)
__device__ __forceinline__ void st_sys(void* p, u32x4 x) { asm volatile("global_store_dwordx4 %0, %1, off sc0 sc1\n\ts_nop 1" :: "v"(p), "v"(x) : "memory"); }
__device__ __forceinline__ void st_dev(void* p, u32x4 x) { asm volatile("global_store_dwordx4 %0, %1, off sc1\n\ts_nop 1" :: "v"(p), "v"(x) : "memory"); }
__device__ __forceinline__ void st_out(void* p, u32x4 x) { asm volatile("global_store_dwordx4 %0, %1, off sc1 nt\n\ts_nop 1" :: "v"(p), "v"(x) : "memory"); }
__device__ __forceinline__ void glds_sys(const void* g, u32 dst) {
    u32 keep;
    asm volatile("s_mov_b32 %0, m0\n\ts_mov_b32 m0, %2\n\ts_nop 0\n\tglobal_load_lds_dwordx4 %1, off sc0 sc1\n\ts_mov_b32 m0, %0" : "=&s"(keep) : "v"(g), "s"(dst) : "memory");
}
__device__ __forceinline__ void glds_plain(const void* g, u32 dst) {
    u32 keep;
    asm volatile("s_mov_b32 %0, m0\n\ts_mov_b32 m0, %2\n\ts_nop 0\n\tglobal_load_lds_dwordx4 %1, off\n\ts_mov_b32 m0, %0" : "=&s"(keep) : "v"(g), "s"(dst) : "memory");
}
__device__ __forceinline__ void glds_w(const void* g, u32 dst) {
    u32 keep;
    asm volatile("s_mov_b32 %0, m0\n\ts_mov_b32 m0, %2\n\ts_nop 0\n\tglobal_load_lds_dwordx4 %1, off nt\n\ts_mov_b32 m0, %0" : "=&s"(keep) : "v"(g), "s"(dst) : "memory");
}
__device__ __forceinline__ u64 now() { return __builtin_amdgcn_s_memrealtime(); }
template <int CTRL, int RM, bool BC> __device__ __forceinline__ u32 dpp(u32 v) { return (u32)__builtin_amdgcn_update_dpp(0, (int)v, CTRL, RM, 0xf, BC); }
__device__ __forceinline__ u32 scan_incl(u32 v) {
    v += dpp<0x111, 0xf, true>(v); v += dpp<0x112, 0xf, true>(v); v += dpp<0x114, 0xf, true>(v); v += dpp<0x118, 0xf, true>(v);
    v += dpp<0x142, 0xa, false>(v); v += dpp<0x143, 0xc, false>(v); return v;
}
__device__ __forceinline__ u32 umax(u32 a, u32 b) { return a > b ? a : b; }
__device__ __forceinline__ u32 umin(u32 a, u32 b) { return a < b ? a : b; }
__device__ __forceinline__ u32 popc(u32 x) { return (u32)__builtin_popcount(x); }
__device__ __forceinline__ u32 perm(u32 hi, u32 lo, u32 sel) { return __builtin_amdgcn_perm(hi, lo, sel); }


__device__ __forceinline__ u32 nzb(u32 z) { return ((z & 0x7f7f7f7fu) + 0x7f7f7f7fu) | z; }
typedef unsigned short u16x2 __attribute__((ext_vector_type(2)));
struct Classifier {
    u32 s1, s2, s3, nb, tlo, thi; bool win;
    __device__ __forceinline__ void init(u32 t) {
        const u32 a = (t >> 8) & 255u, b = (t >> 16) & 255u, c = t >> 24, lo = umin(a, umin(b, c));
        win = umax(a, umax(b, c)) - lo <= 6u;
        const u64 l = (1ull << (((a - lo) & 7u) * 8)) | (2ull << (((b - lo) & 7u) * 8)) | (3ull << (((c - lo) & 7u) * 8));
        tlo = (u32)l; thi = (u32)(l >> 32); nb = ((0u - lo) & 255u) * 0x01000100u;
        s1 = a * 0x01010101u; s2 = b * 0x01010101u; s3 = c * 0x01010101u;
    }
    __device__ __forceinline__ u32 any(u32 e) const {
        const u32 a1 = nzb(e ^ s1), a2 = nzb(e ^ s2), a3 = nzb(e ^ s3);
        return ((~(a1 & a3) >> 7) & 0x01010101u) | ((~(a2 & a3) >> 6) & 0x02020202u);
    }
    __device__ __forceinline__ u32 near(u32 e) const {
        const u16x2 x = __builtin_bit_cast(u16x2, e), n = __builtin_bit_cast(u16x2, nb), top = {0x07ff, 0x07ff};
        u32 sh; asm("v_pk_mad_u16 %0, %1, %2, %3" : "=v"(sh) : "v"(e), "v"(0x01000100u), "s"(nb));
        const u16x2 hi = __builtin_elementwise_min((u16x2)(x + n), top), lo = __builtin_elementwise_min(__builtin_bit_cast(u16x2, sh), top);
        return perm(thi, tlo, perm(__builtin_bit_cast(u32, hi), __builtin_bit_cast(u32, lo), 0x07030501u));
    }
    template <int N> __device__ __forceinline__ void run(const u32* e, u32* c) const {
        if (win) {
#pragma unroll
            for (int i = 0; i < N; ++i) c[i] = near(e[i]);
        } else {
#pragma unroll
            for (int i = 0; i < N; ++i) c[i] = any(e[i]);
        }
    }
};
__device__ __forceinline__ u32 escbits(u32 c) { return ~(c | (c >> 1)) & 0x55555555u; }
struct Appender {
    u32 lo, fill, addr;
    __device__ __forceinline__ void init(u32 off) { lo = 0; fill = (off & 3u) * 8u; addr = off & ~3u; }
    __device__ __forceinline__ void push(u32 k, u32 mk) {
        const u64 t = (u64)k << fill;
        lo |= (u32)t; fill += popc(mk);
        if (fill >= 32u) { lds_or(addr, lo); lo = (u32)(t >> 32); addr += 4; }
        fill &= 31u;
    }
    __device__ __forceinline__ void flush() { if (lo) lds_or(addr, lo); }
};
__device__ __forceinline__ u32 escmask(u32 c) { return perm(0u, 0xffu, c); }
__device__ __forceinline__ u32 selector(u32 mk) { return lds32(__builtin_amdgcn_udot4(mk & 0x20100804u, 0x01010101u, LDS_LUT, false)); }
template <int R> __device__ __forceinline__ u32 pack_level(u32 src, u32 n, u32 t, u32 cdst, u32 edst, u32 cap, u32 lane) {
    Classifier K; K.init(t);
    if (lane < 16) *(LAS unsigned char*)(src + n + lane) = (unsigned char)K.s1;
    u32 ne = 0;
    for (u32 r = 0; r * 1024 < n; r += R) {
        CBAR();
        u32 q[4 * R], c[4 * R], m[4 * R], l[4 * R], cnt = 0;
#pragma unroll
        for (int i = 0; i < R; ++i) { const u32 p0 = (r + i) * 1024 + lane * 16; const u32x4 x = lds128(src + p0); const bool act = p0 < n;
            q[4 * i] = act ? x.x : K.s1; q[4 * i + 1] = act ? x.y : K.s1; q[4 * i + 2] = act ? x.z : K.s1; q[4 * i + 3] = act ? x.w : K.s1; }
        K.run<4 * R>(q, c);
#pragma unroll
        for (int i = 0; i < 4 * R; ++i) { m[i] = escmask(c[i]); l[i] = selector(m[i]); }
#pragma unroll
        for (int i = 0; i < R; ++i) cnt |= ((popc(m[4 * i]) + popc(m[4 * i + 1]) + popc(m[4 * i + 2]) + popc(m[4 * i + 3])) >> 3) << (16 * i);
        const u32 inc = scan_incl(cnt), tot = (u32)__builtin_amdgcn_readlane((int)inc, 63), exc = inc - cnt;
        if (ne + (tot & 0xffffu) + (tot >> 16) > cap) return ~0u;
#pragma unroll
        for (int i = 0; i < R; ++i) {
            Appender A; A.init(edst + ne + (i ? (tot & 0xffffu) + (exc >> 16) : (exc & 0xffffu)));
#pragma unroll
            for (int j = 0; j < 4; ++j) A.push(perm(0u, q[4 * i + j], l[4 * i + j]), m[4 * i + j]);
            A.flush();
            if ((r + i) * 1024 + lane * 16 < n) *(LAS u32*)(cdst + ((r + i) * 64 + lane) * 4) = c[4 * i] | (c[4 * i + 1] << 2) | (c[4 * i + 2] << 4) | (c[4 * i + 3] << 6);
        }
        ne += (tot & 0xffffu) + (tot >> 16);
    }
    CBAR();
    return ne;
}
template <int N> __device__ __forceinline__ u32 rank_ror(u32 key) { return (dpp<0x120 + N, 0xf, true>(key) > key ? 1u : 0u) + rank_ror<N - 1>(key); }
template <> __device__ __forceinline__ u32 rank_ror<0>(u32) { return 0u; }
__device__ __forceinline__ u32 row_or(u32 x) { x |= dpp<0x128, 0xf, true>(x); x |= dpp<0x124, 0xf, true>(x); x |= dpp<0x122, 0xf, true>(x); x |= dpp<0x121, 0xf, true>(x); return rfl(x); }
__device__ __forceinline__ void split(const u32 (&v)[64], u32 (&Mr)[32], u32 (&Er)[32]) {
#pragma unroll
    for (int g = 0; g < 32; ++g) {
        const u32 a = __builtin_amdgcn_alignbit(v[2 * g], v[2 * g], 31), b = __builtin_amdgcn_alignbit(v[2 * g + 1], v[2 * g + 1], 31);
        Er[g] = perm(b, a, 0x07050301u); Mr[g] = perm(b, a, 0x06040200u);
        asm volatile("" : "+v"(Er[g]), "+v"(Mr[g]));
    }
}
__device__ __forceinline__ u32x4 piece(const u32 (&Mr)[32], const u32 (&Er)[32], int p) {
    const u32 a = perm(Er[2 * p], Mr[2 * p], 0x05010400u), b = perm(Er[2 * p], Mr[2 * p], 0x07030602u);
    const u32 c = perm(Er[2 * p + 1], Mr[2 * p + 1], 0x05010400u), d = perm(Er[2 * p + 1], Mr[2 * p + 1], 0x07030602u);
    return (u32x4){__builtin_amdgcn_alignbit(a, a, 1), __builtin_amdgcn_alignbit(b, b, 1), __builtin_amdgcn_alignbit(c, c, 1), __builtin_amdgcn_alignbit(d, d, 1)};
}
template <int R> __device__ __forceinline__ u32 pack_short(u32 src, u32 n, u32 t, u32 cdst, u32 edst, u32 cap, u32 lane) {
    Classifier K; K.init(t);
    if (lane < 16) *(LAS unsigned char*)(src + n + lane) = (unsigned char)K.s1;
    u32 ne = 0;
    for (u32 r = 0; r * 256 < n; r += R) {
        CBAR();
        u32 q[R], c[R], m[R], l[R], cnt = 0;
#pragma unroll
        for (int i = 0; i < R; ++i) { const u32 p0 = (r + i) * 256 + lane * 4, x = lds32(src + p0); q[i] = p0 < n ? x : K.s1; }
        K.run<R>(q, c);
#pragma unroll
        for (int i = 0; i < R; ++i) { m[i] = escmask(c[i]); l[i] = selector(m[i]); cnt |= (popc(m[i]) >> 3) << (16 * i); }
        const u32 inc = scan_incl(cnt), tot = (u32)__builtin_amdgcn_readlane((int)inc, 63), exc = inc - cnt;
        if (ne + (tot & 0xffffu) + (tot >> 16) > cap) return ~0u;
#pragma unroll
        for (int i = 0; i < R; ++i) {
            Appender A; A.init(edst + ne + (i ? (tot & 0xffffu) + (exc >> 16) : (exc & 0xffffu)));
            A.push(perm(0u, q[i], l[i]), m[i]); A.flush();
            u32 x = c[i] << (2 * (lane & 3)); x |= dpp<0xb1, 0xf, true>(x); x |= dpp<0x4e, 0xf, true>(x);
            if ((lane & 3) == 0 && (r + i) * 256 + lane * 4 < n) *(LAS u32*)(cdst + (r + i) * 64 + lane) = x;
        }
        ne += (tot & 0xffffu) + (tot >> 16);
    }
    CBAR();
    return ne;
}
template <int N, int STEP, int OFF> __device__ __forceinline__ void sample(const u32 (&v)[64], u32 hist, u32 lane) {
#pragma unroll
    for (int i = 0; i < N; ++i) lds_add(hist + (((v[i * STEP + OFF] >> ((i & 1) * 16 + 3)) & 0xff0u) | ((lane & 3) * 4)), 1u);
}
template <bool SAMPLED, class Side>
__device__ __forceinline__ u32 pack_chunk(const u32 (&v)[64], u32 (&Mr)[32], u32 (&Er)[32], u32 (&Cr)[8], u32 scr, u32 lane, bool solo, const Side& side) {
    CBAR();
#define SIDE4(p0) do { if (side.on && !solo) { _Pragma("unroll") for (int p = p0; p < p0 + 4; ++p) side(p, piece(Mr, Er, p)); } } while (0)
    if (side.on && !solo) {
#pragma unroll
        for (int p = 0; p < 4; ++p) side(p, (u32x4){v[4 * p], v[4 * p + 1], v[4 * p + 2], v[4 * p + 3]});
    }
    const u32 hist = solo ? scr + 10240 : LDS_HIST; const u32x4 z4 = zero4();
    if (solo) {
#pragma unroll
        for (int j = 0; j < 4; ++j) lds128w(hist + j * 1024 + lane * 16, z4);
        CBAR();
        sample<12, 5, 2>(v, hist, lane);
    }
    if (!SAMPLED) sample<6, 10, 5>(v, hist, lane);
#pragma unroll
    for (int j = 0; j < 10; ++j) lds128w(scr + j * 1024 + lane * 16, z4);
    if (!SAMPLED) { WAIT_LGKM0(); if (!solo) BAR(); }
    CBAR();
    u32 key = 0;
#pragma unroll
    for (int i = 0; i < 4; ++i) { const u32 bin = i * 64 + lane; const u32x4 h = lds128(hist + bin * 16); key = umax(key, ((h.x + h.y + h.z + h.w) << 8) | bin); }
    split(v, Mr, Er);
    { const auto r = __builtin_amdgcn_permlane32_swap(key, key, false, false); key = umax(r[0], r[1]); }
    { const auto r = __builtin_amdgcn_permlane16_swap(key, key, false, false); key = umax(r[0], r[1]); }
    const u32 rank = rank_ror<15>(key), lev = (rank * 11u) >> 5, sym = (key & 255u) << (8 * (rank - 3 * lev + 1));
    const u32 T[4] = {row_or(lev == 0 ? sym : 0u), row_or(lev == 1 ? sym : 0u), row_or(lev == 2 ? sym : 0u), row_or(lev == 3 ? sym : 0u)};
    SIDE4(4);
    u32 cnt = 0, mk[32], sl[32];
    {
        Classifier K; K.init(T[0]);
        u32 c[32]; K.run<32>(Er, c);
#pragma unroll
        for (int g = 0; g < 32; ++g) { mk[g] = escmask(c[g]); sl[g] = selector(mk[g]); cnt += popc(mk[g]); }
#pragma unroll
        for (int k = 0; k < 8; ++k) Cr[k] = c[4 * k] | (c[4 * k + 1] << 2) | (c[4 * k + 2] << 4) | (c[4 * k + 3] << 6);
    }
    cnt >>= 3;
    const u32 inc = scan_incl(cnt), n2 = (u32)__builtin_amdgcn_readlane((int)inc, 63);
    SIDE4(8);
    if (n2 <= CAP2) {
        Appender A; A.init(scr + E_L2 + inc - cnt);
#pragma unroll
        for (int g = 0; g < 32; ++g) A.push(perm(0u, Er[g], sl[g]), mk[g]);
        A.flush();
    }
    SIDE4(12);
    if (n2 > CAP2) return RAWCODE;
    CBAR();
    const u32 d2 = (n2 + 15) >> 4;
    const u32 n3 = pack_level<2>(scr + E_L2, n2, T[1], scr + E_VAR + 16, scr + E_L3, CAP3, lane);
    if (n3 == ~0u) return RAWCODE;
    const u32 d3 = (n3 + 15) >> 4;
    const u32 n4 = pack_short<2>(scr + E_L3, n3, T[2], scr + E_VAR + 16 + 4 * d2, scr + E_L4, CAP4, lane);
    if (n4 == ~0u) return RAWCODE;
    const u32 d4 = (n4 + 15) >> 4, b5 = 16 + 4 * (d2 + d3 + d4);
    const u32 n5 = pack_short<1>(scr + E_L4, n4, T[3], scr + E_VAR + 16 + 4 * (d2 + d3), scr + E_VAR + b5, CAPV - b5, lane);
    if (n5 == ~0u) return RAWCODE;
    if (lane == 0) lds128w(scr + E_VAR, (u32x4){T[0] | d2, T[1] | d3, T[2] | d4, T[3]});
    CBAR();
    return (b5 + n5 + 15) >> 4;
}
__device__ __forceinline__ void pack_init(u32 wid, u32 lane) {
    if (wid >= 4) lds128w(LDS_HIST + (wid - 4) * 1024 + lane * 16, zero4());
    if ((wid == 1 || wid == 2) && lane < 16) { u32 sel = 0x0c0c0c0cu, k = 0;
#pragma unroll
        for (u32 b = 0; b < 4; ++b) if ((lane >> b) & 1) { sel = (sel & ~(255u << k)) | (b << k); k += 8; }
        *(LAS u32*)(LDS_LUT + lane * 4) = sel; }
}
__device__ __forceinline__ u32 chunk_bytes(u32 code) { return code == RAWCODE ? RAWB : FIXB + 16 * code; }

__device__ __forceinline__ void pack_store(char* dst, u32 code, const u32 (&Mr)[32], const u32 (&Er)[32], const u32 (&Cr)[8], u32 scr, u32 lane) {
    char* dm = dst + (chunk_bytes(code) - 8192);
#pragma unroll
    for (int j = 0; j < 8; ++j) st_sys(dm + j * 1024, (u32x4){Mr[4 * j], Mr[4 * j + 1], Mr[4 * j + 2], Mr[4 * j + 3]});
    if (code == RAWCODE) {
#pragma unroll
        for (int j = 0; j < 8; ++j) st_sys(dst + j * 1024, (u32x4){Er[4 * j], Er[4 * j + 1], Er[4 * j + 2], Er[4 * j + 3]});
    } else {
        st_sys(dst, (u32x4){Cr[0], Cr[1], Cr[2], Cr[3]}); st_sys(dst + 1024, (u32x4){Cr[4], Cr[5], Cr[6], Cr[7]});
        if (lane < code) st_sys(dst + 2048, lds128(scr + E_VAR + lane * 16));
        if (lane + 64 < code) st_sys(dst + 3072, lds128(scr + E_VAR + 1024 + lane * 16));
    }
}
__device__ __forceinline__ void raise_flags(u32* fp, u32 fv) { if (fp) __hip_atomic_store((u32*)(GAS u32*)fp, fv, __ATOMIC_RELAXED, __HIP_MEMORY_SCOPE_SYSTEM); }
template <bool SAMPLED, class Side>
__device__ __forceinline__ void pack_group(const Params& P, const u32 (&v)[64], u32 gid, u32 w, bool solo, u32 peers, u32 call, u32 wid, u32 lane, const Side& side, u32*& fp, u32& fv) {
    u32 Mr[32], Er[32], Cr[8];
    const u32 scr = wid * 16384;
    const u32 code = rfl(pack_chunk<SAMPLED>(v, Mr, Er, Cr, scr, lane, solo, side));
    u32 off = w * RAWB, mine = code;
    if (!solo) {
        if (lane == 0) lds32vw(CTL_SIZE + wid * 4, code | gid << 8);
        for (;;) { mine = lds32v(CTL_SIZE + (lane & 7) * 4); if (__all((mine >> 8) == gid || (lane & 7) > wid)) break; __builtin_amdgcn_s_sleep(1); }
        mine &= 255u; off = 0;
#pragma unroll
        for (u32 j = 0; j < 7; ++j) { const u32 c = (u32)__builtin_amdgcn_readlane((int)mine, j); if (j < wid) off += chunk_bytes(c); }
        if (wid == 7) {
            const u32x4 z4 = zero4();
#pragma unroll
            for (int j = 0; j < 4; ++j) lds128w(LDS_HIST + j * 1024 + lane * 16, z4);
        }
    }
    char* wq = (char*)(GAS char*)__builtin_bit_cast(u64, lds64v(LDS_TAB + (lane >> 3) * 8));
    pack_store(P.me + WS_STG + (call & 1) * STG_PAR + (size_t)gid * GRP_STRIDE + off + lane * 16, code, Mr, Er, Cr, scr, lane);
    fp = (solo ? (lane & 7) == w : wid == 7) && ((peers >> (lane >> 3)) & 1) ? (u32*)(wq + WS_FLAG + (call & 1) * FLAG_PAR + (size_t)P.rank * FLAG_SRC + gid * 32) + (lane & 7) : nullptr;
    fv = (call << 8) | mine;
    WAIT_LGKM0();
}

template <bool WRAP> struct Unp {
    u32 sel[4], lo[4], hi[4], of[4];
    __device__ __forceinline__ void fetch(u32 c, u32& off) {
#pragma unroll
        for (int j = 0; j < 4; ++j) {
            const u32 cj = (c >> (2 * j)) & 0x03030303u, mk = escmask(cj), m2 = mk & 0x02020202u;
            const u32 a = off & ~3u; lo[j] = lds32(WRAP ? (a & RMASK) : a); hi[j] = lds32(WRAP ? ((a + 4) & RMASK) : a + 4); of[j] = off;
            u32 p; asm("v_mad_u32_u24 %0, %1, %2, %3" : "=v"(p) : "v"(m2), "s"(0x808080u), "v"(0x04040404u));
            sel[j] = (mk & p) | cj;
            off += popc(m2);
        }
    }
    __device__ __forceinline__ u32 finish(int j, u32 t) const {
        const u32 w = __builtin_amdgcn_alignbyte(hi[j], lo[j], WRAP ? of[j] & 3u : of[j]);
        return perm(w, t, sel[j]);
    }
    __device__ __forceinline__ u32x4 finish4(u32 t) const { return (u32x4){finish(0, t), finish(1, t), finish(2, t), finish(3, t)}; }
};
#define SCHED() __builtin_amdgcn_sched_barrier(0)
template <bool WRAP> __device__ __forceinline__ u32 unpack_round(u32 c, u32 t, u32 src, u32 dst) {
    const u32 cnt = popc(escbits(c)), inc = scan_incl(cnt);
    u32 off = src + inc - cnt;
    Unp<WRAP> u; u.fetch(c, off); SCHED();
    lds128w(dst, u.finish4(t));
    return (u32)__builtin_amdgcn_readlane((int)inc, 63);
}
__device__ __forceinline__ void ring_read8k(u32 a0, u32 (&r)[32]) {
#pragma unroll
    for (int j = 0; j < 8; ++j) { const u32x4 q = lds128((a0 + j * 1024) & RMASK); r[4 * j] = q.x; r[4 * j + 1] = q.y; r[4 * j + 2] = q.z; r[4 * j + 3] = q.w; }
}
__device__ __forceinline__ void unpack_exp(u32 S, u32 code, u32 scr, u32 lane, u32 (&Er)[32]) {
    const u32 a0 = S + lane * 16;
    if (code == RAWCODE) { ring_read8k(a0, Er); return; }
    const u32x4 ca = lds128(a0 & RMASK), cb = lds128((a0 + 1024) & RMASK);
    const u32 V = S + 2048;
    const u32x4 hd = lds128(V & RMASK);
    const u32 t1 = rfl(hd.x), t2 = rfl(hd.y), t3 = rfl(hd.z), t4 = rfl(hd.w), d2 = t1 & 255u, d3 = t2 & 127u, d4 = t3 & 63u;
    u32 c2[4], c3[2];
#pragma unroll
    for (int r = 0; r < 4; ++r) { const u32 i = r * 64 + lane; const u32 x = lds32((V + 16 + 4 * i) & RMASK); c2[r] = i < d2 ? x : 0x55555555u; }
#pragma unroll
    for (int r = 0; r < 2; ++r) { const u32 i = r * 64 + lane; const u32 x = lds32((V + 16 + 4 * (d2 + i)) & RMASK); c3[r] = i < d3 ? x : 0x55555555u; }
    const u32 x4 = lds32((V + 16 + 4 * (d2 + d3 + lane)) & RMASK), c4 = lane < d4 ? x4 : 0x55555555u;
    unpack_round<true>(c4, t4, V + 16 + 4 * (d2 + d3 + d4), scr + D_L4 + lane * 16);
    CBAR();
    const u32 s4 = scr + D_L4 + unpack_round<false>(c3[0], t3, scr + D_L4, scr + D_L3 + lane * 16);
    if (d3 > 64) unpack_round<false>(c3[1], t3, s4, scr + D_L3 + 1024 + lane * 16);
    CBAR();
    const u32 C[8] = {ca.x, ca.y, ca.z, ca.w, cb.x, cb.y, cb.z, cb.w};
    u32 cnt = 0;
#pragma unroll
    for (int k = 0; k < 8; ++k) cnt += popc(escbits(C[k]));
    {
        const u32 n0 = popc(escbits(c2[0])), n1 = popc(escbits(c2[1]));
        const u32 i01 = scan_incl(n0 | (n1 << 16)), tot = (u32)__builtin_amdgcn_readlane((int)i01, 63);
        cnt = scan_incl(cnt) - cnt;
        u32 o0 = scr + D_L3 + (i01 & 0xffffu) - n0, o1 = scr + D_L3 + (tot & 0xffffu) + (i01 >> 16) - n1;
        Unp<false> u0, u1; u0.fetch(c2[0], o0); u1.fetch(c2[1], o1); SCHED();
        lds128w(scr + D_L2 + lane * 16, u0.finish4(t2)); lds128w(scr + D_L2 + 1024 + lane * 16, u1.finish4(t2));
        if (d2 > 128) {
            u32 s3 = scr + D_L3 + (tot & 0xffffu) + (tot >> 16);
            s3 += unpack_round<false>(c2[2], t2, s3, scr + D_L2 + 2048 + lane * 16);
            if (d2 > 192) unpack_round<false>(c2[3], t2, s3, scr + D_L2 + 3072 + lane * 16);
        }
    }
    CBAR();
    u32 off = scr + D_L2 + cnt;
    Unp<false> u[8];
    u[0].fetch(C[0], off); u[1].fetch(C[1], off);
#pragma unroll
    for (int k = 0; k < 8; ++k) {
        SCHED();
        if (k + 2 < 8) u[k + 2].fetch(C[k + 2], off);
        SCHED();
#pragma unroll
        for (int j = 0; j < 4; ++j) Er[4 * k + j] = u[k].finish(j, t1);
    }
    CBAR();
}
__device__ __forceinline__ u32 x_piece_off(u32 half, int p) { return half * 16384 + p * 1024; }
__device__ __forceinline__ u32 w_piece_off(u32 h, int p) {
    const u32 g32 = p >> 2, n = (p >> 1) & 1, kb = p & 1;
    return (g32 & 1) * 16384 + ((2 * (2 * h + (g32 >> 1)) + n) * 2 + kb) * 1024;
}
__device__ __forceinline__ u16* y_base(const Params& P, u32 pm, u32 j, u32 w, u32 lane) {
    return P.out + (size_t)(pm * 256 + (w >> 2) * 64 + (lane >> 3)) * NN + j * 256 + (w & 3) * 64 + (lane & 7) * 8;
}
__device__ __forceinline__ size_t y_piece_off(int p) { return (size_t)((p >> 3) * 128 + (p & 7) * 8) * NN; }
__device__ __forceinline__ u32 local_panel(u32 j, u32 rank) { return j >= (u32)(NR * DP) ? j - (NR - 1) * DP : j - rank * DP; }
__device__ __forceinline__ u32 global_panel(u32 jj, u32 rank) { return jj >= (u32)DP ? jj + (NR - 1) * DP : jj + rank * DP; }
__device__ __forceinline__ u32* counter(const Params& P, u32 i) { return (u32*)(P.me + WS_CNT + i * 128); }
__device__ __forceinline__ u32* tmo_word(const Params& P) { return (u32*)(P.me + WS_TMO); }
__device__ __forceinline__ bool expired(const Params& P, u64 t0, u32 call) {
    if (__hip_atomic_load(tmo_word(P), __ATOMIC_RELAXED, __HIP_MEMORY_SCOPE_AGENT) == call || now() - t0 > DEADLINE) {
        __hip_atomic_store(tmo_word(P), call, __ATOMIC_RELAXED, __HIP_MEMORY_SCOPE_AGENT); return true; }
    return false;
}

struct ImageStore {
    char* o; u32* cnt; u32 half; bool isx, on;
    __device__ __forceinline__ ImageStore(const Params& P, u32 gid, u32 w, u32 q, u32 lane, bool on_) {
        isx = gid < (u32)GW; half = w & 1; on = on_;
        const u32 pn = isx ? gid & 15 : local_panel((gid - GW) >> 1, P.rank), ktl = (isx ? gid >> 4 : gid & 1) * 4 + (w >> 1);
        const u32 qi = (q - P.rank - 1) & 7, stage = qi == 7 ? 0u : ktl == 7 ? 5u : 1 + (ktl >> 1);
        const u32 slot = stage == 0 ? ktl : 8 + ktl * 7 + qi;
        o = P.me + (isx ? WS_XI : WS_WI) + (size_t)pn * PANEL + (size_t)slot * 32768 + lane * 16;
        cnt = counter(P, isx ? pn : 16 + pn) + stage;
    }
    __device__ __forceinline__ void operator()(int p, u32x4 x) const { st_dev(o + (isx ? x_piece_off(half, p) : w_piece_off(half, p)), x); }
    __device__ __forceinline__ void done(u32 lane) const { WAIT_VM(0); if (lane == 0) __hip_atomic_fetch_add(cnt, 1u, __ATOMIC_RELAXED, __HIP_MEMORY_SCOPE_AGENT); }
};
struct OutStore {
    u16* o; static constexpr bool on = true;
    __device__ __forceinline__ void operator()(int p, u32x4 x) const { st_out(o + y_piece_off(p), x); }
};
__device__ __forceinline__ u32* call_word(const Params& P, u32 wid) { return (u32*)(P.me + WS_CALL + blockIdx.x * 128 + wid * 4); }
__device__ __forceinline__ void call_load(const Params& P, u32 wid, u32 lane) {
    u32 keep;
    if (lane == 0) asm volatile("s_mov_b32 %0, m0\n\ts_mov_b32 m0, %2\n\ts_nop 0\n\tglobal_load_lds_dword %1, off sc1\n\ts_mov_b32 m0, %0" : "=&s"(keep) : "v"(call_word(P, wid)), "s"(CTL_CALL + wid * 4) : "memory");
}
__device__ __forceinline__ u32 call_take(u32 wid) { WAIT_VM(0); __builtin_amdgcn_s_waitcnt(0x0f70);   return rfl(lds32v(CTL_CALL + wid * 4)) + 1; }
__device__ __forceinline__ void call_bump(const Params& P, u32 wid, u32 lane, u32 call) { if (lane == 0) __hip_atomic_store(call_word(P, wid), call, __ATOMIC_RELAXED, __HIP_MEMORY_SCOPE_AGENT); }

__device__ __forceinline__ void hold(u32 ticks) { const u64 t0 = now(); while (now() - t0 < ticks) __builtin_amdgcn_s_sleep(2); }
__device__ __forceinline__ void load_tables(u32 lane) {
    const char* ka = (const char*)(const GAS char*)(u64)__builtin_amdgcn_kernarg_segment_ptr() + lane * 16;
    glds_plain(ka, LDS_TAB); glds_plain(ka + 1024, LDS_TAB + 1024); glds_plain(ka + 2048, LDS_TAB + 2048);
}

__device__ __forceinline__ u32 solo_gid(u32 c) { return (c >> 2) & 15; }
__device__ __forceinline__ u32 solo_w(u32 c) { return (c & 3) | (c >> 6) << 2; }
__device__ __forceinline__ void load_x(const Params& P, u32 gid, u32 w, u32 lane, u32 (&v)[64]) {
    const u16* s = P.x + (size_t)((gid & 15) * 256 + (w & 1) * 128 + (lane >> 3)) * KS + ((gid >> 4) * 4 + (w >> 1)) * 64 + ((lane ^ (lane >> 3)) & 7) * 8;
#pragma unroll
    for (int p = 0; p < 16; ++p) { const u32x4 t = *(const u32x4*)(s + p * 8 * KS); v[4 * p] = t.x; v[4 * p + 1] = t.y; v[4 * p + 2] = t.z; v[4 * p + 3] = t.w; }
}
__device__ __forceinline__ void puller(const Params& P, u32 role, u32 wid, u32 lane) {
    const u32 q = role & 7, sub = (role >> 3) & 15, PW = (role >> 7) & 15;
    if (wid >= 2) for (u32 pc = sub + PW * (7 - wid); pc < (u32)NRAW; pc += 6 * PW) {
        const u32 gid = solo_gid(pc), w = solo_w(pc);
        u32 v[64], Mr[32], Er[32];
        load_x(P, gid, w, lane, v);
        const u32 call = call_take(wid);
        split(v, Mr, Er);
        char* o = P.peer + WS_INB + (size_t)(((call & 1) * 8 + P.rank) * 64 + pc) * 8192 + lane * 16;
#pragma unroll
        for (int j = 0; j < 8; ++j) st_sys(o + j * 1024, (u32x4){Mr[4 * j], Mr[4 * j + 1], Mr[4 * j + 2], Mr[4 * j + 3]});
        WAIT_VM(0);
        if (lane == 0) __hip_atomic_store((u32*)(P.peer + WS_FLAG + (call & 1) * FLAG_PAR + (size_t)P.rank * FLAG_SRC + (GM + gid) * 32) + w, call << 8, __ATOMIC_RELAXED, __HIP_MEMORY_SCOPE_SYSTEM);
    }
    if (wid == 1) { hold(100); load_tables(lane); }
    const u32 call = call_take(wid);
    call_bump(P, wid, lane, call);
    if (wid == 0) {
        __builtin_amdgcn_s_setprio(3);
        const char* base = P.peer + WS_STG + (call & 1) * STG_PAR;
        const u32 voff = lane * 16;
        u32 pos = 0, pub = 0, limit = RING / 1024;
        for (u32 ui = 0;; ++ui) {
            u32 nu = rfl(lds32v(CTL_NUNIT));
            if ((nu & ~DONE) <= ui) {
                WAIT_VM(0); lds32vw(CTL_LANDED, pos); pub = pos;
                while (nu <= ui) { __builtin_amdgcn_s_sleep(1); nu = rfl(lds32v(CTL_NUNIT)); }
                if ((nu & ~DONE) <= ui) break;
            }
            CBAR();
            const u32x2 un = lds64v(D_UNIT + ui * 8);
            const u32 ut = rfl(un.y), total = ut & 0x3ffffu, lastc = ut >> 18;
            const char* src = base + rfl(un.x);
            for (u32 o = 0; o < total;) {
                if (pos + 4 > limit) {
                    bool drained = false;
                    for (;;) {
                        const u32 nx = lds32v(CTL_NEXT + (lane & 7) * 4);
                        u32 f = lastc;
#pragma unroll
                        for (u32 j = 0; j < ND; ++j) f = umin(f, (u32)__builtin_amdgcn_readlane((int)nx, j));
                        limit = (rfl(lds32v(D_DESC + f * 8)) >> 14) + RING / 1024;
                        if (pos + 4 <= limit) break;
                        if (!drained) { WAIT_VM(0); lds32vw(CTL_LANDED, pos); pub = pos; drained = true; }
                        __builtin_amdgcn_s_sleep(1);
                    }
                }
                const u32 dst = (pos * 1024) & RMASK;
                if (o + 4096 <= total && dst + 4096 <= RING) {
                    u32 keep;
                    asm volatile("s_mov_b32 %0, m0\n\ts_mov_b32 m0, %3\n\ts_nop 4\n\t"
                                 "global_load_lds_dwordx4 %1, %2 sc0 sc1\n\ts_waitcnt vmcnt(%4)\n\t"
                                 "global_load_lds_dwordx4 %1, %2 offset:1024 sc0 sc1\n\ts_waitcnt vmcnt(%4)\n\t"
                                 "global_load_lds_dwordx4 %1, %2 offset:2048 sc0 sc1\n\ts_waitcnt vmcnt(%4)\n\t"
                                 "global_load_lds_dwordx4 %1, %2 offset:3072 sc0 sc1\n\ts_waitcnt vmcnt(%4)\n\t"
                                 "s_mov_b32 m0, %0" : "=&s"(keep) : "v"(voff), "s"(src + o), "s"(dst), "i"(WIN) : "memory");
                    pos += 4; o += 4096;
                } else {
                    if (o + voff < total) glds_sys(src + o + voff, dst);
                    WAIT_VM(WIN);
                    pos += 1; o += 1024;
                }
                if (pos > pub + WIN) { pub = pos - WIN; lds32vw(CTL_LANDED, pub); }
            }
        }
    } else if (wid == 1) {
        u32 basek = 0, nch = 0, nun = 0;
        u32 held = 0, heldw = 0, helds = 0;
        auto release = [&](u32 c) {
            const u32 a0 = held & ~127u, total = held - a0 + 8192;
            if (lane == 0) { lds64vw(D_UNIT + nun * 8, (u32x2){a0, total | (c << 18)}); lds32vw(D_DESC + c * 8 + 4, heldw | ((basek * 1024 + held - a0 - helds) >> 4) << 13); }
            nun += 1; basek += (total + 1023) >> 10; held = 0;
        };
        u32* ticket = (u32*)(P.me + WS_CLAIM + q * 128);
        const u32 nown = P.nunits & 0xffffu, nall = nown + (P.nunits >> 16);
        u32 tbase = NSTAT * PW - (call - 1) * (nown - NSTAT * PW + PW), tend = nown;
        for (u32 ui = 0;; ++ui) {
            while (rfl(lds32v(CTL_LANDED)) + LEAD < basek + (held ? 8u : 0u)) __builtin_amdgcn_s_sleep(4);
            u32 idx = ui * PW + sub;
            if (ui >= (u32)NSTAT) {
                if (lane == 0) idx = __hip_atomic_fetch_add(ticket, 1u, __ATOMIC_RELAXED, __HIP_MEMORY_SCOPE_AGENT);
                idx = rfl(idx) + tbase;
                if (idx >= tend) {
                    if (tend == nall) break;
                    ticket = (u32*)(P.me + WS_CLAIM + P.rank * 128); tbase = nown - (call - 1) * (nall - nown + NPULL); tend = nall;
                    continue;
                }
            }
            const u32 e = tab16(TAB_STREAM, idx), gid = e & 1023u, part = e >> 10, lv = 31 - __builtin_clz(part), n = 8u >> lv, first = (part - (1u << lv)) * n;
            if (nch + n > (u32)(NS * 8)) break;
            const bool solo = gid < (u32)SOLOG, mine = lane >= first && lane < first + n, sent = gid < (u32)(NRAW / 4) && first < 4;
            const u32* f = (const u32*)(P.me + WS_FLAG + (call & 1) * FLAG_PAR + (size_t)q * FLAG_SRC + (gid + (sent && (lane & 8) ? GM : 0)) * 32) + (lane & 7);
            const u64 t0 = now(); u32 val; bool dead = false;
            for (;;) {
                val = __hip_atomic_load(f, __ATOMIC_RELAXED, __HIP_MEMORY_SCOPE_SYSTEM);
                if (__all((val >> 8) == (call & 0xffffffu) || (lane & 7) >= first + n || (solo && (lane & 7) < first))) break;
                if (__any(expired(P, t0, call))) { dead = true; break; }
                __builtin_amdgcn_s_sleep(1);
            }
            if (dead) {
                const u32 t = gid - GY;
                if (lane == 0 && t < (u32)(16 * PK)) *y_base(P, t & 15, global_panel(t >> 4, q), first, 0u) = 0x7fc0;
                break;
            }
            u32 code = val & 255u; if (code > 127u) code = RAWCODE;
            const u32 cb = chunk_bytes(code) - (sent ? 8192u : 0u), sz = solo ? (mine ? (cb + 1023u) & ~1023u : 0u) : (lane < first + n ? cb : 0u);
            u32 inc = sz; inc += dpp<0x111, 0xf, true>(inc); inc += dpp<0x112, 0xf, true>(inc); inc += dpp<0x114, 0xf, true>(inc);
            const bool two = n == 1;
            const u32 a0 = (u32)__builtin_amdgcn_readlane((int)(inc - sz), (int)first) & ~127u, end = (u32)__builtin_amdgcn_readlane((int)inc, 7) - (two ? 8192u : 0u), total = end - a0;
            const u32 S = basek * 1024 + inc - sz - a0;
            if (mine) lds64vw(D_DESC + (nch + lane - first) * 8, (u32x2){(S << 4) | code, (gid * 8 + lane) | (two ? 0u : (chunk_bytes(code) - 8192) >> 4) << 13});
            if (solo) { if (mine) lds64vw(D_UNIT + (nun + lane - first) * 8, (u32x2){gid * (u32)GRP_STRIDE + lane * RAWB, cb | ((nch + lane - first) << 18)}); nun += n; }
            else { if (lane == 0) lds64vw(D_UNIT + nun * 8, (u32x2){gid * (u32)GRP_STRIDE + a0, total | ((nch + n - 1) << 18)}); nun += 1; }
            basek += (total + 1023) >> 10;
            if (held) release(nch - 1);
            if (two) { held = gid * (u32)GRP_STRIDE + end; heldw = gid * 8 + first; helds = (u32)__builtin_amdgcn_readlane((int)S, (int)first); }
            nch += n;
            WAIT_LGKM0();
            lds32vw(CTL_NUNIT, nun); lds32vw(CTL_NCHUNK, nch);
        }
        if (held) { release(nch - 1); WAIT_LGKM0(); }
        lds32vw(CTL_NUNIT, nun | DONE); lds32vw(CTL_NCHUNK, nch | DONE);
    } else {
        const u32 dw = wid - 2, scr = D_SCR + dw * D_SCRB;
        for (u32 c = dw;; c += ND) {
            u32 nc = rfl(lds32v(CTL_NCHUNK));
            while (nc <= c) { __builtin_amdgcn_s_sleep(2); nc = rfl(lds32v(CTL_NCHUNK)); }
            if ((nc & ~DONE) <= c) break;
            CBAR();
            const u32x2 de = lds64v(D_DESC + c * 8);
            const u32 d = rfl(de.x), code = d & 255u, S = (d >> 8) << 4, gid = (rfl(de.y) >> 3) & 1023u, w = rfl(de.y) & 7;
            while (rfl(lds32v(CTL_LANDED)) < (S + chunk_bytes(code) - 8192 + 1023) >> 10) __builtin_amdgcn_s_sleep(1);
            CBAR();
            u32 Mr[32], Er[32];
            unpack_exp(S, code, scr, lane, Er);
            if (gid < (u32)(NRAW / 4) && w < 4) {
                const char* m = P.me + WS_INB + (size_t)(((call & 1) * 8 + q) * 64 + gid * 4 + w) * 8192 + lane * 16;
                u32x4 r[8];
                asm volatile("global_load_dwordx4 %0, %8, off sc0 sc1\n\tglobal_load_dwordx4 %1, %8, off offset:1024 sc0 sc1\n\t"
                             "global_load_dwordx4 %2, %8, off offset:2048 sc0 sc1\n\tglobal_load_dwordx4 %3, %8, off offset:3072 sc0 sc1\n\t"
                             "global_load_dwordx4 %4, %9, off sc0 sc1\n\tglobal_load_dwordx4 %5, %9, off offset:1024 sc0 sc1\n\t"
                             "global_load_dwordx4 %6, %9, off offset:2048 sc0 sc1\n\tglobal_load_dwordx4 %7, %9, off offset:3072 sc0 sc1\n\ts_waitcnt vmcnt(0)"
                             : "=&v"(r[0]), "=&v"(r[1]), "=&v"(r[2]), "=&v"(r[3]), "=&v"(r[4]), "=&v"(r[5]), "=&v"(r[6]), "=&v"(r[7]) : "v"(m), "v"(m + 4096) : "memory");
#pragma unroll
                for (int j = 0; j < 8; ++j) { Mr[4 * j] = r[j].x; Mr[4 * j + 1] = r[j].y; Mr[4 * j + 2] = r[j].z; Mr[4 * j + 3] = r[j].w; }
            } else {
                u32 dm = rfl(de.y) >> 13;
                while (!dm) { __builtin_amdgcn_s_sleep(1); dm = rfl(lds32v(D_DESC + c * 8 + 4)) >> 13; }
                const u32 SM = S + dm * 16;
                while (rfl(lds32v(CTL_LANDED)) < (SM + 9215) >> 10) __builtin_amdgcn_s_sleep(1);
                CBAR();
                ring_read8k(SM + lane * 16, Mr);
            }
            WAIT_LGKM0(); lds32vw(CTL_NEXT + dw * 4, c + ND); CBAR();
            if (gid >= (u32)GY) {
                const u32 t = gid - GY;
                u16* o = y_base(P, t & 15, global_panel(t >> 4, q), w, lane);
#pragma unroll
                for (int p = 0; p < 16; ++p) st_out(o + y_piece_off(p), piece(Mr, Er, p));
            } else {
                const ImageStore im(P, gid, w, q, lane, true);
#pragma unroll
                for (int p = 0; p < 16; ++p) im(p, piece(Mr, Er, p));
                im.done(lane);
            }
        }
    }
}

__device__ __forceinline__ void pack_inputs(const Params& P, u32 gid, u32 w, bool solo, u32 wid, u32 lane) {
    asm volatile("" : "+v"(lane));
    u32 v[64];
    u32 peers = 255u & ~(1u << P.rank); bool own = true;
    if (gid < (u32)GW) load_x(P, gid, w, lane, v);
    else {
        const u32 j = (gid - GW) >> 1, ktl = (gid & 1) * 4 + (w >> 1);
        if (j < (u32)(NR * DP)) { const u32 dest = j / DP; own = dest == (u32)P.rank; peers = own ? 0u : 1u << dest; }
        else if (j < (u32)(NR * (DP + FX))) own = (j & 7) != P.rank;
        const u16* s = P.w + (size_t)(ktl * 64 + (lane >> 4)) * NN + j * 256 + (w & 1) * 128 + (lane & 15) * 8;
        const u32 scr = wid * 16384;
#pragma unroll
        for (int i = 0; i < 16; ++i) glds_w(s + (size_t)i * 4 * NN, scr + i * 1024);
        WAIT_VM(0);
        const u32 ta = scr + ((lane >> 4) * 8 + ((lane & 15) >> 2)) * 256 + (lane & 3) * 16;
#pragma unroll
        for (int p = 0; p < 16; ++p) {
            const u32 a = ta + (p & 1) * 32 * 256 + (p >> 2) * 64 + ((p >> 1) & 1) * 8;
            const u32x2 lo = __builtin_bit_cast(u32x2, __builtin_amdgcn_ds_read_tr16_b64_v4i16((LAS s16x4*)a));
            const u32x2 hi = __builtin_bit_cast(u32x2, __builtin_amdgcn_ds_read_tr16_b64_v4i16((LAS s16x4*)(a + 4 * 256)));
            v[4 * p] = lo.x; v[4 * p + 1] = lo.y; v[4 * p + 2] = hi.x; v[4 * p + 3] = hi.y;
        }
        WAIT_LGKM0();
    }
    const u32 call = call_take(wid);
    const ImageStore im(P, gid, w, P.rank, lane, own);
    if (peers) {
        u32* fp; u32 fv;
        pack_group<false>(P, v, gid, w, solo, peers, call, wid, lane, im, fp, fv);
        WAIT_VM(0);
        if (!solo) BAR();
        raise_flags(fp, fv);
        if (solo) { load_x(P, gid, w, lane, v); __builtin_amdgcn_s_waitcnt(0x0f70); }
        else if (own && lane == 0) __hip_atomic_fetch_add(im.cnt, 1u, __ATOMIC_RELAXED, __HIP_MEMORY_SCOPE_AGENT);
    }
    if (!peers || solo) {
#pragma unroll
        for (int p = 0; p < 16; ++p) im(p, (u32x4){v[4 * p], v[4 * p + 1], v[4 * p + 2], v[4 * p + 3]});
        im.done(lane);
    }
}

typedef __bf16 bf16x2 __attribute__((ext_vector_type(2)));
typedef float f32x2 __attribute__((ext_vector_type(2)));
__device__ __forceinline__ u32 cvt_pk(float lo, float hi) { return __builtin_bit_cast(u32, __builtin_convertvector((f32x2){lo, hi}, bf16x2)); }
__device__ __forceinline__ void wait_panels(const Params& P, u32 tile, u32 upto, u32 call, u32 wid, u32 lane) {
    if (wid == 0) {
        const u32 st = umin(lane & 7, upto), need = call * (st == 0 ? 16u : st < 4 ? 28u : 14u);
        const u32* c = counter(P, (lane & 8) ? 16 + (tile >> 4) : (tile & 15)) + st;
        const u64 t0 = now();
        for (;;) {
            const u32 x = __hip_atomic_load(c, __ATOMIC_RELAXED, __HIP_MEMORY_SCOPE_AGENT);
            if (__all((int)(x - need) >= 0)) break;
            if (__any(expired(P, t0, call))) { asm volatile("ds_write_b32 %0, %1\n\ts_waitcnt lgkmcnt(0)" :: "v"(CTL_BAD), "v"(1u) : "memory"); break; }
            __builtin_amdgcn_s_sleep(4);
        }
        __builtin_amdgcn_fence(__ATOMIC_ACQUIRE, "agent");
        WAIT_VM(0);
    }
    BAR();
}
__device__ __forceinline__ void stage2(u32 dst, u32 voff, const char* src) {
    asm volatile("s_mov_b32 m0, %2\n\ts_nop 0\n\tglobal_load_lds_dwordx4 %0, %1\n\tglobal_load_lds_dwordx4 %0, %1 offset:1024" :: "v"(voff), "s"(src), "s"(dst) : "memory");
}
__device__ __forceinline__ void stage4(u32 dst, u32 voff, const char* src) {
    asm volatile("s_mov_b32 m0, %2\n\ts_nop 0\n\tglobal_load_lds_dwordx4 %0, %1\n\tglobal_load_lds_dwordx4 %0, %1 offset:1024\n\t"
                 "global_load_lds_dwordx4 %0, %1 offset:2048\n\tglobal_load_lds_dwordx4 %0, %1 offset:3072" :: "v"(voff), "s"(src), "s"(dst) : "memory");
}
__device__ __forceinline__ void gemm_role(const Params& P, u32 place, u32 call, u32 wid, u32 lane) {
    const u32 wr = wid >> 2, wc = wid & 3;
    u32 aoff = wr * 8192 + (lane & 15) * 128 + (((lane >> 4) ^ lane) & 7) * 16, aoff1 = aoff ^ 64, boff = 65536 + wc * 4096 + lane * 16;
    const u32 mA = wid * 2048, mB = 65536 + wid * 4096;
    u32 vA0 = mA + lane * 16, vA1 = vA0 + 16384, vA2 = vA0 + 32768, vA3 = vA0 + 49152, vB0 = wid * 4096 + lane * 16, vB1 = vB0 + 32768;
    asm volatile("" : "+v"(vA0), "+v"(vA1), "+v"(vA2), "+v"(vA3), "+v"(vB0), "+v"(vB1), "+v"(aoff), "+v"(aoff1), "+v"(boff));
#define G_LDA(b, h) do { _Pragma("unroll") for (int m = 0; m < 4; ++m) _Pragma("unroll") for (int k = 0; k < 2; ++k) At[m][k] = *(const LAS bf16x8*)((k ? aoff1 : aoff) + ((b) * 2 + (h)) * 16384 + m * 2048); } while (0)
#define G_LDB(b) do { _Pragma("unroll") for (int j = 0; j < 2; ++j) _Pragma("unroll") for (int n = 0; n < 2; ++n) _Pragma("unroll") for (int k = 0; k < 2; ++k) \
        Bt[j][n][k] = *(const LAS bf16x8*)(boff + ((b) * 2 + j) * 16384 + n * 2048 + k * 1024); } while (0)
#define G_MMA(ai, STEP) do { G_WV(8); __builtin_amdgcn_s_waitcnt(0xc07f);   G_SCHED; G_BAR; __builtin_amdgcn_s_setprio(1); _Pragma("unroll") for (int i = 0; i < 32; ++i) { \
        const int bj = i >> 4, k = (i >> 3) & 1, m = (i >> 1) & 3, n = i & 1; \
        if (i == 8) { G_SCHED; STEP; G_SCHED; } \
        if (i == 32 - 2) { G_SCHED; __builtin_amdgcn_s_setprio(3); G_BAR; G_SCHED; } \
        acc[ai][bj][m][n] = __builtin_amdgcn_mfma_f32_16x16x32_bf16(Bt[bj][n][k], At[m][k], acc[ai][bj][m][n], 0, 0, 0); } \
        __builtin_amdgcn_s_setprio(0); G_SCHED; } while (0)
#define G_WV(n) asm volatile("s_waitcnt vmcnt(" #n ")" ::: "memory")
#define G_BAR __builtin_amdgcn_s_barrier()
#define G_SCHED __builtin_amdgcn_sched_barrier(0)
    const u32x2 cl = lds64v(TAB_COLS + ((blockIdx.x & 4) * 2 + (place >> 2)) * 8);
    const u64 mycols = (u64)rfl(cl.y) << 32 | rfl(cl.x); const u32 pm = (blockIdx.x & 3) * 4 + (place & 3);
    const char* const cA = P.me + WS_XI + (size_t)pm * PANEL; const char* const WI = P.me + WS_WI;
    u32 cur = pm | ((u32)mycols & 255u) << 4;
    bool fresh = true; u32* fp = nullptr; u32 fv = 0;
    bf16x8 At[4][2], Bt[2][2][2];
    for (u32 ui = 0;; ++ui) {
        const u32 nxt = pm | ((u32)(mycols >> (8 * ui + 8)) & 255u) << 4;
        const bool has_next = (nxt >> 4) != 255u, dist = (cur >> 4) < (u32)PK, chain = has_next && !dist;
        const char* cB = WI + (size_t)(cur >> 4) * PANEL;
        if (fresh) {
            wait_panels(P, cur, ui > 1 ? 5u : 0u, call, wid, lane);
            stage4(mB, vB0, cB); stage2(mA, vA0, cA); stage2(mA + 16384, vA1, cA);
            if (wr == 1) G_BAR;
            G_WV(2); G_BAR;
            stage4(mB + 32768, vB1, cB); stage2(mA + 32768, vA2, cA);
            G_WV(6); G_BAR;
            raise_flags(fp, fv); fp = nullptr; fv = 0;
        }
        f32x4 acc[2][2][4][2];
#pragma unroll
        for (int a = 0; a < 2; ++a)
#pragma unroll
            for (int b = 0; b < 2; ++b)
#pragma unroll
                for (int m = 0; m < 4; ++m)
#pragma unroll
                    for (int n = 0; n < 2; ++n) acc[a][b][m][n] = (f32x4){0.f, 0.f, 0.f, 0.f};
        const char *p0 = cA, *pA = cA + 65536, *pB = cB + 65536;
        u32 stops = 0x80000000u | (ui == 0 ? 0x09020408u : ui == 1 ? 0x00020008u : 0u), upto = 0;
        for (u32 trip = 0;;) {
            const u32 end = stops ? (u32)__builtin_ctz(stops) : 32u;
            u32 left = end - trip;
            do {
                G_LDB(0); G_SCHED; G_LDA(0, 0); stage2(mA + 49152, vA3, p0);
                G_MMA(0, );
                G_LDA(0, 1); stage4(mB, vB0, pB); stage2(mA, vA0, pA);
                G_MMA(1, );
                G_LDB(1); G_SCHED; G_LDA(1, 0); stage2(mA + 16384, vA1, pA);
                G_MMA(0, );
                G_LDA(1, 1); stage4(mB + 32768, vB1, pB); stage2(mA + 32768, vA2, pA);
                G_MMA(1, p0 = pA; pA += 65536; pB += 65536; --left; asm volatile("" : "+s"(p0), "+s"(pA), "+s"(pB), "+s"(left)));
            } while (left);
            if (!stops) break;
            trip = end; stops &= stops - 1;
            if (stops) { upto += ui ? (upto ? 3u : 2u) : 1u; wait_panels(P, cur, upto, call, wid, lane); }
            else {
                pA = cA; pB = cB;
                if (chain) { pB = WI + (size_t)(nxt >> 4) * PANEL; wait_panels(P, nxt, ui ? 5u : 0u, call, wid, lane); }
            }
        }
        if (!chain) G_WV(0);
        if (wr == 0) G_BAR;
        u32 v[64];
        {
            int tz = threadIdx.x; asm volatile("" : "+v"(tz));
            const u32 ln = tz & 63, bad = lds32v(CTL_BAD);
            const u32 ep = D_DESC + wid * 2048, to = ep + (ln & 15) * 128 + (((ln >> 4) ^ (ln & 7)) * 16), from = ep + (ln ^ (ln >> 3)) * 16;
            u16* o = y_base(P, cur & 15, global_panel(cur >> 4, P.rank), wid, ln);
#pragma unroll
            for (int p = 0; p < 16; p += 2) {
                const int ai = p >> 3, m = (p >> 1) & 3;
#pragma unroll
                for (int bj = 0; bj < 2; ++bj) {
                    const f32x4 x0 = acc[ai][bj][m][0], x1 = acc[ai][bj][m][1];
                    lds128w(to ^ (bj * 64), (u32x4){bad ? 0x7fc07fc0u : cvt_pk(x0[0], x0[1]), cvt_pk(x0[2], x0[3]), cvt_pk(x1[0], x1[1]), cvt_pk(x1[2], x1[3])});
                }
                CBAR();
#pragma unroll
                for (int j = 0; j < 2; ++j) { const u32x4 t = lds128(from + j * 1024); v[4 * (p + j)] = t.x; v[4 * (p + j) + 1] = t.y; v[4 * (p + j) + 2] = t.z; v[4 * (p + j) + 3] = t.w; }
                CBAR();
            }
            if (!dist) {
#pragma unroll
                for (int p = 0; p < 16; ++p) st_out(o + y_piece_off(p), (u32x4){v[4 * p], v[4 * p + 1], v[4 * p + 2], v[4 * p + 3]});
            }
            if (chain) { if (wr == 1) G_BAR; }
            else {
                if (dist) {
                    sample<6, 10, 5>(v, LDS_HIST, ln);
                    WAIT_LGKM0(); BAR();
                    const OutStore os = {o};
                    pack_group<true>(P, v, GY + cur, wid, false, (cur >> 4) < (u32)DP ? 255u & ~(1u << P.rank) : 1u << (((cur >> 4) - DP) & 7), call, wid, ln, os, fp, fv);
                    if (!has_next) { G_WV(0); BAR(); raise_flags(fp, fv); }
                }
                if (!has_next) break;
            }
        }
        fresh = !chain; cur = nxt;
    }
}

__global__ void __launch_bounds__(512, 2) fused(const Args A) {
    const u32 wid = rfl(threadIdx.x >> 6), lane = threadIdx.x & 63;
    asm volatile("gar_code0:");
    const u32 role = pin((((const u32*)A.role)[blockIdx.x >> 1] >> ((blockIdx.x & 1) * 16)) & 0xffffu);
    char* peer = pin(A.ws[0]); char* lanep = peer;
#pragma unroll
    for (u32 j = 1; j < NR; ++j) { char* c = pin(A.ws[j]); peer = (role & 7) == j ? c : peer; lanep = (lane & 7) == j ? c : lanep; }
    const Params P = {pin(A.me), peer, pin(A.x), pin(A.w), pin(A.out), pin((u32)A.rank), pin((u32)A.nunits)};
    call_load(P, wid, lane);
    if (role & 0x8000u) {
        if (threadIdx.x == 0) { lds32vw(CTL_NUNIT, 0u); lds32vw(CTL_LANDED, 0u); lds32vw(CTL_NCHUNK, 0u); }
        if (wid == 1 && lane < 8) lds32vw(CTL_NEXT + lane * 4, lane);
        WAIT_LGKM0(); BAR();
        puller(P, role, wid, lane); return;
    }
    if (threadIdx.x == 0) lds32vw(CTL_BAD, 0u);
    if (threadIdx.x < 8) lds32vw(CTL_SIZE + threadIdx.x * 4, 0u);
    pack_init(wid, lane);
    const u32 rb = NENC - 1 - (role & 255u), sc = rb + (wid & 2) * 32;
    const bool soloist = (wid == 1 || wid == 2) && rb < 64u;
    if (soloist) { if (lane < 8) lds64vw(LDS_TAB + lane * 8, __builtin_bit_cast(u32x2, (u64)lanep)); }
    else {
        if (rb >= 64u) {
            if (wid == 3) {
                u64 pc; u32 back, len;
                asm volatile("s_getpc_b64 %0\ngar_here:\n\ts_mov_b32 %1, gar_here-gar_code0\n\ts_mov_b32 %2, gar_code1-gar_code0" : "=s"(pc), "=s"(back), "=s"(len));
                for (u32 o = ((blockIdx.x >> 3) & 15) * 1024 + lane * 16; o < len; o += 16384) glds_plain((const char*)(const GAS char*)((pc - back) & ~15ull) + o, LDS_CTL - 1024);
            }
        } else hold(100);
        if (wid == 0) load_tables(lane);
        if (rb >= 64u) hold(150);
    }
    for (u32 it = 0; it < 3; ++it) {
        const u32 gid = it ? tab16(TAB_ENC, (role & 255u) * 2 + it - 1) : solo_gid(sc);
        if (gid == 0xffffu) break;
        if (it || soloist) pack_inputs(P, gid, it ? wid : solo_w(sc), !it, wid, lane);
        WAIT_VM(0); WAIT_LGKM0(); BAR();
    }
    const u32 call = call_take(wid);
    call_bump(P, wid, lane, call);
    if (role >> 8) gemm_role(P, (role >> 8) - 1, call, wid, lane);
    asm volatile("gar_code1:");
}

static void build_tables(Args& P) {
    unsigned char* cols = P.cols; u16 *stream = P.stream, *role = P.role, *job = P.enc, enc[NPACK];
    {
        int peer[2][4], np[2] = {0, 0}, cnt[8] = {}, seen[2] = {0, 0}, n = 0, top = NENC;
        for (int d = 1; d < NR; ++d) { const int q = (P.rank + d) & 7, h = ((0xb2 >> q) ^ (0xb2 >> P.rank)) & 1; peer[h][np[h]++] = q; }
        for (int b = 0; b < NWG; ++b) {
            const int h = (b >> 2) & 1, i = b >> 3, g0 = 32 - 4 * (h ? CH : CL), near = h ? 4 * NH : 3 * NL, far = h ? 3 * FL : 4 * FH, k = seen[h]++;
            if (k >= near + far) { role[b] = i >= g0 ? (u16)(n++ | (i - g0 + 1) << 8) : (u16)--top; continue; }
            const int side = k < near ? h : !h, j = k < near ? k : k - near, q = peer[side][side ? (j + j / 4) & 3 : j % 3];
            role[b] = (u16)(0x8000 | q | cnt[q]++ << 3);
        }
        for (int b = 0; b < NWG; ++b) if (role[b] & 0x8000) role[b] |= (u16)(cnt[role[b] & 7] << 7);
    }
    int col[LC], nc = 0;
    {
        const int rest = LC - (ROUNDS - 1) * CPR, r0 = (rest + 1) / 2, light = rest < CPR ? CPR - rest : CPR;
        int over = PK - FX - light * (ROUNDS - 1), nd = 0, nr = PK;
        for (int i = 0; i < 128; ++i) cols[i] = 0xff;
        for (int r = 0; r < ROUNDS; ++r) for (int c = 0; c < 8; ++c) for (int cg = 0; cg < 2; ++cg) {
            const bool last = c < (cg ? rest - r0 : r0), heavy = last && rest < CPR;
            if (nd >= DP && nd < PK && ((nd - DP) & 7) == P.rank) ++nd;
            if (c >= (cg ? CH : CL) || (r + 1 == ROUNDS && !last)) continue;
            cols[(cg * 8 + c) * 8 + r] = (unsigned char)(col[nc++] = nd < PK && (!heavy || over-- > 0) ? nd++ : nr++);
        }
    }
    auto wgid = [&](int jj, int dest, int h) { return GW + 2 * (jj >= DP ? jj + (NR - 1) * DP : dest * DP + jj) + h; };
    int k = 0;
    auto put = [&](int gid, int lv) { for (int j = 0; j < (1 << lv); ++j) stream[k++] = (u16)(gid | ((1 << lv) + j) << 10); };
    auto opnd = [&](int i, int h) { return i < 16 ? GX + 16 * h + i : wgid(col[i - 16], P.rank, h); };
    auto stage = [&](int st, int i0, int i1) {
        const int h = st >> 1, half = st & 1;
        for (int i = i0; i < i1; ++i) {
            if (i < 16 && h == 0) { for (int j = 0; j < 2; ++j) stream[k++] = (u16)(opnd(i, h) | (4 + 2 * half + j) << 10); }
            else stream[k++] = (u16)(opnd(i, h) | (2 + half) << 10);
        }
    };
    stage(0, NRAW / 4, 16); stage(1, 0, 12);
    {
        u16 t[64]; const int n = k; int e = 0;
        for (int i = 0; i < n; ++i) t[i] = stream[i];
        k = 0;
        for (int i = 0; i < n || e < NRAW / 4; ++i) { if (i < n) stream[k++] = t[i]; if (i >= 18 && e < NRAW / 4) stream[k++] = (u16)((GX + e++) | 2 << 10); }
    }
    const int first = 16 + CPR - 4;
    stage(0, 16, first); stage(1, 12, first); stage(0, first, 16 + CPR); stage(2, 0, first); stage(1, first, 16 + CPR);
    for (int part = 6; part < 8; ++part) for (int i = 0; i < first; ++i) stream[k++] = (u16)(opnd(i, 1) | part << 10);
    stage(2, first, 16 + CPR); stage(3, first, 16 + CPR);
    for (int h = 0; h < 2; ++h) for (int i = CPR; i < 2 * CPR; ++i) put(wgid(col[i], P.rank, h), 0);
    for (int i = 2 * CPR; i < LC; ++i) for (int h = 0; h < 2; ++h) put(wgid(col[i], P.rank, h), 0);
    for (int t = 0; t < 16 * DP; ++t) { const int left = 16 * DP - t; put(GY + t, left <= 8 ? 1 : 0); }
    const int nown = k;
    for (int t = 0; t < 16 * FX; ++t) put(GY + (DP + (t >> 4) * 8 + P.rank) * 16 + (t & 15), 16 * FX - t <= 16 ? 3 : 16 * FX - t <= 30 ? 2 : 1);
    P.nunits = nown | (k - nown) << 16;
    k = 0;
    for (int i = 0; i < CPR; ++i) for (int d = 1; d < (col[i] < DP ? NR : 2); ++d) enc[k++] = (u16)wgid(col[i], (P.rank + d) & 7, 0);
    for (int i = SOLOG; i < 32; ++i) enc[k++] = (u16)(GX + i);
    for (int i = 0; i < CPR; ++i) { if (col[i] < DP) enc[k++] = (u16)wgid(col[i], P.rank, 0); enc[k++] = (u16)wgid(col[i], P.rank, 1); }
    for (int i = 0; i < CPR; ++i) if (col[i] < DP) for (int d = 1; d < NR; ++d) enc[k++] = (u16)wgid(col[i], (P.rank + d) & 7, 1);
    for (int f = 0; f < FX; ++f) for (int h = 0; h < 2; ++h) enc[k++] = (u16)wgid(DP + f * 8 + P.rank, 0, h);
    for (int i = CPR; i < LC; ++i) for (int h = 0; h < 2; ++h) for (int d = 0; d < (col[i] < DP ? NR : 1); ++d) enc[k++] = (u16)wgid(col[i], (P.rank + d) & 7, h);
    int nx = 0, top = NENC;
    for (int i = 0; i < NENC - 64; ++i) { job[2 * i] = enc[nx++]; job[2 * i + 1] = 0xffff; }
    for (int i = 0; i < NENC - 64; ++i) if (job[2 * i] >= GW && (job[2 * i] - GW) / 2 / DP == P.rank) job[2 * i + 1] = enc[nx++];
    for (int i = NENC - 64; i < NENC; ++i) { job[2 * i] = enc[nx++]; job[2 * i + 1] = 0xffff; }
    while (nx < NPACK) { do --top; while (job[2 * top + 1] != 0xffff); job[2 * top + 1] = enc[nx++]; }
}
}

extern "C" void kernel_launch(int rank, int nranks, ncclComm_t comm, hipStream_t stream,
                              void* const* d_in, const int* in_sizes, int n_in,
                              void* d_out, int out_size, const int* in_dtypes, int out_dtype,
                              void* const* d_out_all, void* const* d_ws_all, size_t ws_bytes) {
    using namespace gar;
    if (nranks != NR || (unsigned)rank >= (unsigned)NR || n_in != 2 || ws_bytes < WS_END || in_sizes[0] != MM * KS || in_sizes[1] != KS * NN || out_size != MM * NN) {
        fprintf(stderr, "kernel_launch: unexpected shapes\n"); return; }
    Args P = {};
    for (int p = 0; p < NR; ++p) P.ws[p] = (char*)d_ws_all[p];
    P.me = P.ws[rank];
    P.x = (const u16*)d_in[0]; P.w = (const u16*)d_in[1]; P.out = (u16*)d_out; P.rank = rank;
    build_tables(P);
    HIP_CHECK(hipFuncSetAttribute((const void*)fused, hipFuncAttributeMaxDynamicSharedMemorySize, LDS_BYTES));
    hipLaunchKernelGGL(fused, dim3(NWG), dim3(512), LDS_BYTES, stream, P);
    HIP_CHECK(hipGetLastError());
}
```

```cpp
#include <hip/hip_runtime.h>
#include <rccl/rccl.h>
#include <cstdio>
#include <cstdint>
#include <cstddef>
#pragma clang diagnostic ignored "-Wint-to-pointer-cast"

#define HIP_CHECK(expr) do { hipError_t _e = (expr); if (_e != hipSuccess) { \
    fprintf(stderr, "HIP error %s at %s:%d\n", hipGetErrorString(_e), __FILE__, __LINE__); \
    return; } } while (0)

namespace gar {
#define LAS __attribute__((address_space(3)))
#define GAS __attribute__((address_space(1)))
typedef unsigned u32;
typedef unsigned short u16;
typedef unsigned long long u64;
typedef unsigned u32x4 __attribute__((ext_vector_type(4)));
typedef unsigned u32x2 __attribute__((ext_vector_type(2)));
typedef short s16x4 __attribute__((ext_vector_type(4)));
typedef short bf16x8 __attribute__((ext_vector_type(8)));
typedef float f32x4 __attribute__((ext_vector_type(4)));

constexpr int NR = 8, MM = 4096, NN = 32768, KS = 512;
constexpr int DP = 10;
constexpr int RP = 128 - NR * DP;
constexpr int LP = DP + RP;
constexpr int NWG = 256;
constexpr int NL = 9, FL = 0, NH = 8, FH = 1, CL = 6, CH = 6, CPR = CL + CH;
constexpr int NPULL = 3 * (NL + FL) + 4 * (NH + FH);
constexpr int WIN = 5;
static_assert(3 * NL + 4 * FH + 16 * CL <= 128 && 4 * NH + 3 * FL + 16 * CH <= 128, "roles");
constexpr int SOLOG = 16, NRAW = 52;
constexpr int LEAD = 44;
static_assert(NRAW <= 64 && NRAW % 4 == 0 && 8 * SOLOG <= NWG - NPULL && SOLOG == 16, "solo chunks");
constexpr int NENC = NWG - NPULL;
constexpr int GX = 0, GW = 32, GY = 288;
constexpr int FX = 2, PK = DP + 8 * FX, LC = LP - FX;
constexpr int NGID = GY + 16 * PK, GM = NGID, NGT = GM + 16;
constexpr int NS = 32 + 2 * LC + 16 * DP + 16 * FX;
constexpr int ROUNDS = (LC + CPR - 1) / CPR;
constexpr int NUMAX = 704;
constexpr int NSTAT = 3;
constexpr int NPACK = 288 - SOLOG;
constexpr u64 DEADLINE = 30000000ull;

constexpr size_t WS_CALL = 0;
constexpr size_t WS_TMO = 32768;
constexpr size_t WS_CNT = 36864;
constexpr size_t WS_CLAIM = 57344;
constexpr size_t WS_FLAG = 65536;
constexpr size_t FLAG_SRC = NGT * 32, FLAG_PAR = 8 * FLAG_SRC;
constexpr size_t GRP_STRIDE = 131072;
constexpr size_t WS_STG = 2u << 20;
constexpr size_t STG_PAR = (NGID * GRP_STRIDE + 0x1fffff) & ~(size_t)0x1fffff;
constexpr size_t WS_INB = WS_STG + 2 * STG_PAR;
constexpr size_t WS_XI = WS_INB + 2 * 8 * 64 * 8192;
constexpr size_t PANEL = 64 * 32768;
constexpr size_t WS_WI = WS_XI + 16 * PANEL;
constexpr size_t WS_END = WS_WI + (size_t)LP * PANEL;
static_assert(WS_FLAG + 2 * FLAG_PAR <= WS_STG && NGT <= 1024 && RP >= 8 * FX, "layout");

constexpr u32 LDS_CTL = 131072;
constexpr u32 LDS_LUT = LDS_CTL + 512, LDS_HIST = LDS_CTL + 1024;
constexpr u32 LDS_TAB = LDS_HIST + 4096, D_DESC = LDS_TAB + 3072, LDS_BYTES = D_DESC + NS * 64;
constexpr u32 CTL_CALL = LDS_CTL  , CTL_SIZE = LDS_CTL + 64  , CTL_BAD = LDS_CTL + 128;
constexpr u32 CTL_NUNIT = LDS_CTL + 192, CTL_LANDED = LDS_CTL + 256, CTL_NCHUNK = LDS_CTL + 320, CTL_NEXT = LDS_CTL + 384  ;
constexpr u32 E_L2 = 0, E_L3 = E_L2 + 4160, E_L4 = E_L3 + 2112, E_VAR = E_L4 + 1088, E_END = E_VAR + 2112;
static_assert(E_END <= 10240 && E_L4 % 16 == 0 && E_L3 % 16 == 0 && E_VAR % 16 == 0, "packer scratch");
constexpr u32 CAP2 = 4080, CAP3 = 2032, CAP4 = 1008, CAPV = 2032, FIXB = 10240, RAWB = 16384, RAWCODE = 255;
constexpr u32 RING = 65536, RMASK = RING - 1, ND = 6;
constexpr u32 D_SCR = RING, D_SCRB = 7360, D_L2 = 0, D_L3 = 4160, D_L4 = D_L3 + 2112;
constexpr u32 D_UNIT = D_SCR + ND * D_SCRB, DONE = 0x80000000u;
static_assert(D_L4 + 1088 <= D_SCRB && D_UNIT + (2 * NUMAX + 8 * SOLOG) * 8 <= LDS_CTL - 2048 && LDS_BYTES <= 163840 && D_DESC % 128 == 0 && NS * 64 >= 16384, "puller LDS");

struct Args {
    char* ws[NR];
    char* me; const u16* x; const u16* w; u16* out;
    int rank, nunits;
    u16 role[NWG];
    u16 stream[NUMAX];
    unsigned char cols[128];
    u16 enc[2 * NENC];
    u16 pad[(3072 - 104 - 128) / 2 - NWG - NUMAX - 2 * NENC];
};
static_assert(sizeof(Args) == 3072 && ROUNDS < 8 && offsetof(Args, cols) % 8 == 0 && offsetof(Args, role) % 4 == 0, "kernarg");
constexpr u32 TAB_STREAM = LDS_TAB + offsetof(Args, stream), TAB_ENC = LDS_TAB + offsetof(Args, enc), TAB_COLS = LDS_TAB + offsetof(Args, cols);
struct Params { char* me; char* peer; const u16* x; const u16* w; u16* out; u32 rank, nunits; };
template <class T> __device__ __forceinline__ T* pin(T* p) { u64 v = (u64)p; asm volatile("" : "+s"(v)); return (T*)(GAS T*)v; }
__device__ __forceinline__ u32 pin(u32 v) { asm volatile("" : "+s"(v)); return v; }

__device__ __forceinline__ u32 lds32(u32 a) { return *(LAS u32*)a; }
__device__ __forceinline__ u32 lds32v(u32 a) { return *(volatile LAS u32*)a; }
__device__ __forceinline__ void lds32vw(u32 a, u32 x) { *(volatile LAS u32*)a = x; }
__device__ __forceinline__ u32x2 lds64v(u32 a) { return *(volatile LAS u32x2*)a; }
__device__ __forceinline__ void lds64vw(u32 a, u32x2 x) { *(volatile LAS u32x2*)a = x; }
__device__ __forceinline__ u32x4 lds128(u32 a) { return *(LAS u32x4*)a; }
__device__ __forceinline__ void lds128w(u32 a, u32x4 x) { *(LAS u32x4*)a = x; }
__device__ __forceinline__ u32x4 zero4() { u32 z; asm volatile("v_mov_b32 %0, 0" : "=v"(z)); return (u32x4){z, z, z, z}; }
__device__ __forceinline__ void lds_or(u32 a, u32 x) { __hip_atomic_fetch_or((LAS u32*)a, x, __ATOMIC_RELAXED, __HIP_MEMORY_SCOPE_WORKGROUP); }
__device__ __forceinline__ void lds_add(u32 a, u32 x) { __hip_atomic_fetch_add((LAS u32*)a, x, __ATOMIC_RELAXED, __HIP_MEMORY_SCOPE_WORKGROUP); }
__device__ __forceinline__ u32 rfl(u32 x) { return (u32)__builtin_amdgcn_readfirstlane((int)x); }
__device__ __forceinline__ u32 tab16(u32 t, u32 i) { return rfl(*(volatile LAS u16*)(t + i * 2)); }
#define CBAR() asm volatile("" ::: "memory")
#define WAIT_LGKM0() asm volatile("s_waitcnt lgkmcnt(0)" ::: "memory")
#define WAIT_VM(n) asm volatile("s_waitcnt vmcnt(%0)" :: "i"(n) : "memory")
#define BAR() do { CBAR(); __builtin_amdgcn_s_barrier(); CBAR(); } while (0)
__device__ __forceinline__ void st_sys(void* p, u32x4 x) { asm volatile("global_store_dwordx4 %0, %1, off sc0 sc1\n\ts_nop 1" :: "v"(p), "v"(x) : "memory"); }
__device__ __forceinline__ void st_dev(void* p, u32x4 x) { asm volatile("global_store_dwordx4 %0, %1, off sc1\n\ts_nop 1" :: "v"(p), "v"(x) : "memory"); }
__device__ __forceinline__ void st_out(void* p, u32x4 x) { asm volatile("global_store_dwordx4 %0, %1, off sc1 nt\n\ts_nop 1" :: "v"(p), "v"(x) : "memory"); }
__device__ __forceinline__ void glds_plain(const void* g, u32 dst) {
    u32 keep;
    asm volatile("s_mov_b32 %0, m0\n\ts_mov_b32 m0, %2\n\ts_nop 0\n\tglobal_load_lds_dwordx4 %1, off\n\ts_mov_b32 m0, %0" : "=&s"(keep) : "v"(g), "s"(dst) : "memory");
}
__device__ __forceinline__ void glds_w(const void* g, u32 dst) {
    u32 keep;
    asm volatile("s_mov_b32 %0, m0\n\ts_mov_b32 m0, %2\n\ts_nop 0\n\tglobal_load_lds_dwordx4 %1, off nt\n\ts_mov_b32 m0, %0" : "=&s"(keep) : "v"(g), "s"(dst) : "memory");
}
__device__ __forceinline__ u64 now() { return __builtin_amdgcn_s_memrealtime(); }
template <int CTRL, int RM, bool BC> __device__ __forceinline__ u32 dpp(u32 v) { return (u32)__builtin_amdgcn_update_dpp(0, (int)v, CTRL, RM, 0xf, BC); }
__device__ __forceinline__ u32 scan_incl(u32 v) {
    v += dpp<0x111, 0xf, true>(v); v += dpp<0x112, 0xf, true>(v); v += dpp<0x114, 0xf, true>(v); v += dpp<0x118, 0xf, true>(v);
    v += dpp<0x142, 0xa, false>(v); v += dpp<0x143, 0xc, false>(v); return v;
}
__device__ __forceinline__ u32 umax(u32 a, u32 b) { return a > b ? a : b; }
__device__ __forceinline__ u32 umin(u32 a, u32 b) { return a < b ? a : b; }
__device__ __forceinline__ u32 popc(u32 x) { return (u32)__builtin_popcount(x); }
__device__ __forceinline__ u32 perm(u32 hi, u32 lo, u32 sel) { return __builtin_amdgcn_perm(hi, lo, sel); }


__device__ __forceinline__ u32 nzb(u32 z) { return ((z & 0x7f7f7f7fu) + 0x7f7f7f7fu) | z; }
typedef unsigned short u16x2 __attribute__((ext_vector_type(2)));
struct Classifier {
    u32 s1, s2, s3, nb, tlo, thi; bool win;
    __device__ __forceinline__ void init(u32 t) {
        const u32 a = (t >> 8) & 255u, b = (t >> 16) & 255u, c = t >> 24, lo = umin(a, umin(b, c));
        win = umax(a, umax(b, c)) - lo <= 6u;
        const u64 l = (1ull << (((a - lo) & 7u) * 8)) | (2ull << (((b - lo) & 7u) * 8)) | (3ull << (((c - lo) & 7u) * 8));
        tlo = (u32)l; thi = (u32)(l >> 32); nb = ((0u - lo) & 255u) * 0x01000100u;
        s1 = a * 0x01010101u; s2 = b * 0x01010101u; s3 = c * 0x01010101u;
    }
    __device__ __forceinline__ u32 any(u32 e) const {
        const u32 a1 = nzb(e ^ s1), a2 = nzb(e ^ s2), a3 = nzb(e ^ s3);
        return ((~(a1 & a3) >> 7) & 0x01010101u) | ((~(a2 & a3) >> 6) & 0x02020202u);
    }
    __device__ __forceinline__ u32 near(u32 e) const {
        const u16x2 x = __builtin_bit_cast(u16x2, e), n = __builtin_bit_cast(u16x2, nb), top = {0x07ff, 0x07ff};
        u32 sh; asm("v_pk_mad_u16 %0, %1, %2, %3" : "=v"(sh) : "v"(e), "v"(0x01000100u), "s"(nb));
        const u16x2 hi = __builtin_elementwise_min((u16x2)(x + n), top), lo = __builtin_elementwise_min(__builtin_bit_cast(u16x2, sh), top);
        return perm(thi, tlo, perm(__builtin_bit_cast(u32, hi), __builtin_bit_cast(u32, lo), 0x07030501u));
    }
    template <int N> __device__ __forceinline__ void run(const u32* e, u32* c) const {
        if (win) {
#pragma unroll
            for (int i = 0; i < N; ++i) c[i] = near(e[i]);
        } else {
#pragma unroll
            for (int i = 0; i < N; ++i) c[i] = any(e[i]);
        }
    }
};
__device__ __forceinline__ u32 escbits(u32 c) { return ~(c | (c >> 1)) & 0x55555555u; }
struct Appender {
    u32 lo, fill, addr;
    __device__ __forceinline__ void init(u32 off) { lo = 0; fill = (off & 3u) * 8u; addr = off & ~3u; }
    __device__ __forceinline__ void push(u32 k, u32 mk) {
        const u64 t = (u64)k << fill;
        lo |= (u32)t; fill += popc(mk);
        if (fill >= 32u) { lds_or(addr, lo); lo = (u32)(t >> 32); addr += 4; }
        fill &= 31u;
    }
    __device__ __forceinline__ void flush() { if (lo) lds_or(addr, lo); }
};
__device__ __forceinline__ u32 escmask(u32 c) { return perm(0u, 0xffu, c); }
__device__ __forceinline__ u32 selector(u32 mk) { return lds32(__builtin_amdgcn_udot4(mk & 0x20100804u, 0x01010101u, LDS_LUT, false)); }
template <int R> __device__ __forceinline__ u32 pack_level(u32 src, u32 n, u32 t, u32 cdst, u32 edst, u32 cap, u32 lane) {
    Classifier K; K.init(t);
    if (lane < 16) *(LAS unsigned char*)(src + n + lane) = (unsigned char)K.s1;
    u32 ne = 0;
    for (u32 r = 0; r * 1024 < n; r += R) {
        CBAR();
        u32 q[4 * R], c[4 * R], m[4 * R], l[4 * R], cnt = 0;
#pragma unroll
        for (int i = 0; i < R; ++i) { const u32 p0 = (r + i) * 1024 + lane * 16; const u32x4 x = lds128(src + p0); const bool act = p0 < n;
            q[4 * i] = act ? x.x : K.s1; q[4 * i + 1] = act ? x.y : K.s1; q[4 * i + 2] = act ? x.z : K.s1; q[4 * i + 3] = act ? x.w : K.s1; }
        K.run<4 * R>(q, c);
#pragma unroll
        for (int i = 0; i < 4 * R; ++i) { m[i] = escmask(c[i]); l[i] = selector(m[i]); }
#pragma unroll
        for (int i = 0; i < R; ++i) cnt |= ((popc(m[4 * i]) + popc(m[4 * i + 1]) + popc(m[4 * i + 2]) + popc(m[4 * i + 3])) >> 3) << (16 * i);
        const u32 inc = scan_incl(cnt), tot = (u32)__builtin_amdgcn_readlane((int)inc, 63), exc = inc - cnt;
        if (ne + (tot & 0xffffu) + (tot >> 16) > cap) return ~0u;
#pragma unroll
        for (int i = 0; i < R; ++i) {
            Appender A; A.init(edst + ne + (i ? (tot & 0xffffu) + (exc >> 16) : (exc & 0xffffu)));
#pragma unroll
            for (int j = 0; j < 4; ++j) A.push(perm(0u, q[4 * i + j], l[4 * i + j]), m[4 * i + j]);
            A.flush();
            if ((r + i) * 1024 + lane * 16 < n) *(LAS u32*)(cdst + ((r + i) * 64 + lane) * 4) = c[4 * i] | (c[4 * i + 1] << 2) | (c[4 * i + 2] << 4) | (c[4 * i + 3] << 6);
        }
        ne += (tot & 0xffffu) + (tot >> 16);
    }
    CBAR();
    return ne;
}
template <int N> __device__ __forceinline__ u32 rank_ror(u32 key) { return (dpp<0x120 + N, 0xf, true>(key) > key ? 1u : 0u) + rank_ror<N - 1>(key); }
template <> __device__ __forceinline__ u32 rank_ror<0>(u32) { return 0u; }
__device__ __forceinline__ u32 row_or(u32 x) { x |= dpp<0x128, 0xf, true>(x); x |= dpp<0x124, 0xf, true>(x); x |= dpp<0x122, 0xf, true>(x); x |= dpp<0x121, 0xf, true>(x); return rfl(x); }
__device__ __forceinline__ void split(const u32 (&v)[64], u32 (&Mr)[32], u32 (&Er)[32]) {
#pragma unroll
    for (int g = 0; g < 32; ++g) {
        const u32 a = __builtin_amdgcn_alignbit(v[2 * g], v[2 * g], 31), b = __builtin_amdgcn_alignbit(v[2 * g + 1], v[2 * g + 1], 31);
        Er[g] = perm(b, a, 0x07050301u); Mr[g] = perm(b, a, 0x06040200u);
        asm volatile("" : "+v"(Er[g]), "+v"(Mr[g]));
    }
}
__device__ __forceinline__ u32x4 piece(const u32 (&Mr)[32], const u32 (&Er)[32], int p) {
    const u32 a = perm(Er[2 * p], Mr[2 * p], 0x05010400u), b = perm(Er[2 * p], Mr[2 * p], 0x07030602u);
    const u32 c = perm(Er[2 * p + 1], Mr[2 * p + 1], 0x05010400u), d = perm(Er[2 * p + 1], Mr[2 * p + 1], 0x07030602u);
    return (u32x4){__builtin_amdgcn_alignbit(a, a, 1), __builtin_amdgcn_alignbit(b, b, 1), __builtin_amdgcn_alignbit(c, c, 1), __builtin_amdgcn_alignbit(d, d, 1)};
}
template <int R> __device__ __forceinline__ u32 pack_short(u32 src, u32 n, u32 t, u32 cdst, u32 edst, u32 cap, u32 lane) {
    Classifier K; K.init(t);
    if (lane < 16) *(LAS unsigned char*)(src + n + lane) = (unsigned char)K.s1;
    u32 ne = 0;
    for (u32 r = 0; r * 256 < n; r += R) {
        CBAR();
        u32 q[R], c[R], m[R], l[R], cnt = 0;
#pragma unroll
        for (int i = 0; i < R; ++i) { const u32 p0 = (r + i) * 256 + lane * 4, x = lds32(src + p0); q[i] = p0 < n ? x : K.s1; }
        K.run<R>(q, c);
#pragma unroll
        for (int i = 0; i < R; ++i) { m[i] = escmask(c[i]); l[i] = selector(m[i]); cnt |= (popc(m[i]) >> 3) << (16 * i); }
        const u32 inc = scan_incl(cnt), tot = (u32)__builtin_amdgcn_readlane((int)inc, 63), exc = inc - cnt;
        if (ne + (tot & 0xffffu) + (tot >> 16) > cap) return ~0u;
#pragma unroll
        for (int i = 0; i < R; ++i) {
            Appender A; A.init(edst + ne + (i ? (tot & 0xffffu) + (exc >> 16) : (exc & 0xffffu)));
            A.push(perm(0u, q[i], l[i]), m[i]); A.flush();
            u32 x = c[i] << (2 * (lane & 3)); x |= dpp<0xb1, 0xf, true>(x); x |= dpp<0x4e, 0xf, true>(x);
            if ((lane & 3) == 0 && (r + i) * 256 + lane * 4 < n) *(LAS u32*)(cdst + (r + i) * 64 + lane) = x;
        }
        ne += (tot & 0xffffu) + (tot >> 16);
    }
    CBAR();
    return ne;
}
template <int N, int STEP, int OFF> __device__ __forceinline__ void sample(const u32 (&v)[64], u32 hist, u32 lane) {
#pragma unroll
    for (int i = 0; i < N; ++i) lds_add(hist + (((v[i * STEP + OFF] >> ((i & 1) * 16 + 3)) & 0xff0u) | ((lane & 3) * 4)), 1u);
}
template <bool SAMPLED, class Side>
__device__ __forceinline__ u32 pack_chunk(const u32 (&v)[64], u32 (&Mr)[32], u32 (&Er)[32], u32 (&Cr)[8], u32 scr, u32 lane, bool solo, const Side& side) {
    CBAR();
#define SIDE4(p0) do { if (side.on && !solo) { _Pragma("unroll") for (int p = p0; p < p0 + 4; ++p) side(p, piece(Mr, Er, p)); } } while (0)
    if (side.on && !solo) {
#pragma unroll
        for (int p = 0; p < 4; ++p) side(p, (u32x4){v[4 * p], v[4 * p + 1], v[4 * p + 2], v[4 * p + 3]});
    }
    const u32 hist = solo ? scr + 10240 : LDS_HIST; const u32x4 z4 = zero4();
    if (solo) {
#pragma unroll
        for (int j = 0; j < 4; ++j) lds128w(hist + j * 1024 + lane * 16, z4);
        CBAR();
        sample<12, 5, 2>(v, hist, lane);
    }
    if (!SAMPLED) sample<6, 10, 5>(v, hist, lane);
#pragma unroll
    for (int j = 0; j < 10; ++j) lds128w(scr + j * 1024 + lane * 16, z4);
    if (!SAMPLED) { WAIT_LGKM0(); if (!solo) BAR(); }
    CBAR();
    u32 key = 0;
#pragma unroll
    for (int i = 0; i < 4; ++i) { const u32 bin = i * 64 + lane; const u32x4 h = lds128(hist + bin * 16); key = umax(key, ((h.x + h.y + h.z + h.w) << 8) | bin); }
    split(v, Mr, Er);
    { const auto r = __builtin_amdgcn_permlane32_swap(key, key, false, false); key = umax(r[0], r[1]); }
    { const auto r = __builtin_amdgcn_permlane16_swap(key, key, false, false); key = umax(r[0], r[1]); }
    const u32 rank = rank_ror<15>(key), lev = (rank * 11u) >> 5, sym = (key & 255u) << (8 * (rank - 3 * lev + 1));
    const u32 T[4] = {row_or(lev == 0 ? sym : 0u), row_or(lev == 1 ? sym : 0u), row_or(lev == 2 ? sym : 0u), row_or(lev == 3 ? sym : 0u)};
    SIDE4(4);
    u32 cnt = 0, mk[32], sl[32];
    {
        Classifier K; K.init(T[0]);
        u32 c[32]; K.run<32>(Er, c);
#pragma unroll
        for (int g = 0; g < 32; ++g) { mk[g] = escmask(c[g]); sl[g] = selector(mk[g]); cnt += popc(mk[g]); }
#pragma unroll
        for (int k = 0; k < 8; ++k) Cr[k] = c[4 * k] | (c[4 * k + 1] << 2) | (c[4 * k + 2] << 4) | (c[4 * k + 3] << 6);
    }
    cnt >>= 3;
    const u32 inc = scan_incl(cnt), n2 = (u32)__builtin_amdgcn_readlane((int)inc, 63);
    SIDE4(8);
    if (n2 <= CAP2) {
        Appender A; A.init(scr + E_L2 + inc - cnt);
#pragma unroll
        for (int g = 0; g < 32; ++g) A.push(perm(0u, Er[g], sl[g]), mk[g]);
        A.flush();
    }
    SIDE4(12);
    if (n2 > CAP2) return RAWCODE;
    CBAR();
    const u32 d2 = (n2 + 15) >> 4;
    const u32 n3 = pack_level<2>(scr + E_L2, n2, T[1], scr + E_VAR + 16, scr + E_L3, CAP3, lane);
    if (n3 == ~0u) return RAWCODE;
    const u32 d3 = (n3 + 15) >> 4;
    const u32 n4 = pack_short<2>(scr + E_L3, n3, T[2], scr + E_VAR + 16 + 4 * d2, scr + E_L4, CAP4, lane);
    if (n4 == ~0u) return RAWCODE;
    const u32 d4 = (n4 + 15) >> 4, b5 = 16 + 4 * (d2 + d3 + d4);
    const u32 n5 = pack_short<1>(scr + E_L4, n4, T[3], scr + E_VAR + 16 + 4 * (d2 + d3), scr + E_VAR + b5, CAPV - b5, lane);
    if (n5 == ~0u) return RAWCODE;
    if (lane == 0) lds128w(scr + E_VAR, (u32x4){T[0] | d2, T[1] | d3, T[2] | d4, T[3]});
    CBAR();
    return (b5 + n5 + 15) >> 4;
}
__device__ __forceinline__ void pack_init(u32 wid, u32 lane) {
    if (wid >= 4) lds128w(LDS_HIST + (wid - 4) * 1024 + lane * 16, zero4());
    if ((wid == 1 || wid == 2) && lane < 16) { u32 sel = 0x0c0c0c0cu, k = 0;
#pragma unroll
        for (u32 b = 0; b < 4; ++b) if ((lane >> b) & 1) { sel = (sel & ~(255u << k)) | (b << k); k += 8; }
        *(LAS u32*)(LDS_LUT + lane * 4) = sel; }
}
__device__ __forceinline__ u32 chunk_bytes(u32 code) { return code == RAWCODE ? RAWB : FIXB + 16 * code; }

__device__ __forceinline__ void pack_store(char* dst, u32 code, const u32 (&Mr)[32], const u32 (&Er)[32], const u32 (&Cr)[8], u32 scr, u32 lane) {
    char* dm = dst + (chunk_bytes(code) - 8192);
#pragma unroll
    for (int j = 0; j < 8; ++j) st_sys(dm + j * 1024, (u32x4){Mr[4 * j], Mr[4 * j + 1], Mr[4 * j + 2], Mr[4 * j + 3]});
    if (code == RAWCODE) {
#pragma unroll
        for (int j = 0; j < 8; ++j) st_sys(dst + j * 1024, (u32x4){Er[4 * j], Er[4 * j + 1], Er[4 * j + 2], Er[4 * j + 3]});
    } else {
        st_sys(dst, (u32x4){Cr[0], Cr[1], Cr[2], Cr[3]}); st_sys(dst + 1024, (u32x4){Cr[4], Cr[5], Cr[6], Cr[7]});
        if (lane < code) st_sys(dst + 2048, lds128(scr + E_VAR + lane * 16));
        if (lane + 64 < code) st_sys(dst + 3072, lds128(scr + E_VAR + 1024 + lane * 16));
    }
}
__device__ __forceinline__ void raise_flags(u32* fp, u32 fv) { if (fp) __hip_atomic_store((u32*)(GAS u32*)fp, fv, __ATOMIC_RELAXED, __HIP_MEMORY_SCOPE_SYSTEM); }
template <bool SAMPLED, class Side>
__device__ __forceinline__ void pack_group(const Params& P, const u32 (&v)[64], u32 gid, u32 w, bool solo, u32 peers, u32 call, u32 wid, u32 lane, const Side& side, u32*& fp, u32& fv) {
    u32 Mr[32], Er[32], Cr[8];
    const u32 scr = wid * 16384;
    const u32 code = rfl(pack_chunk<SAMPLED>(v, Mr, Er, Cr, scr, lane, solo, side));
    u32 off = w * RAWB, mine = code;
    if (!solo) {
        if (lane == 0) lds32vw(CTL_SIZE + wid * 4, code | gid << 8);
        for (;;) { mine = lds32v(CTL_SIZE + (lane & 7) * 4); if (__all((mine >> 8) == gid || (lane & 7) > wid)) break; __builtin_amdgcn_s_sleep(1); }
        mine &= 255u; off = 0;
#pragma unroll
        for (u32 j = 0; j < 7; ++j) { const u32 c = (u32)__builtin_amdgcn_readlane((int)mine, j); if (j < wid) off += chunk_bytes(c); }
        if (wid == 7) {
            const u32x4 z4 = zero4();
#pragma unroll
            for (int j = 0; j < 4; ++j) lds128w(LDS_HIST + j * 1024 + lane * 16, z4);
        }
    }
    char* wq = (char*)(GAS char*)__builtin_bit_cast(u64, lds64v(LDS_TAB + (lane >> 3) * 8));
    pack_store(P.me + WS_STG + (call & 1) * STG_PAR + (size_t)gid * GRP_STRIDE + off + lane * 16, code, Mr, Er, Cr, scr, lane);
    fp = (solo ? (lane & 7) == w : wid == 7) && ((peers >> (lane >> 3)) & 1) ? (u32*)(wq + WS_FLAG + (call & 1) * FLAG_PAR + (size_t)P.rank * FLAG_SRC + gid * 32) + (lane & 7) : nullptr;
    fv = (call << 8) | mine;
    WAIT_LGKM0();
}

template <bool WRAP> struct Unp {
    u32 sel[4], lo[4], hi[4], of[4];
    __device__ __forceinline__ void fetch(u32 c, u32& off) {
#pragma unroll
        for (int j = 0; j < 4; ++j) {
            const u32 cj = (c >> (2 * j)) & 0x03030303u, mk = escmask(cj), m2 = mk & 0x02020202u;
            const u32 a = off & ~3u; lo[j] = lds32(WRAP ? (a & RMASK) : a); hi[j] = lds32(WRAP ? ((a + 4) & RMASK) : a + 4); of[j] = off;
            u32 p; asm("v_mad_u32_u24 %0, %1, %2, %3" : "=v"(p) : "v"(m2), "s"(0x808080u), "v"(0x04040404u));
            sel[j] = (mk & p) | cj;
            off += popc(m2);
        }
    }
    __device__ __forceinline__ u32 finish(int j, u32 t) const {
        const u32 w = __builtin_amdgcn_alignbyte(hi[j], lo[j], WRAP ? of[j] & 3u : of[j]);
        return perm(w, t, sel[j]);
    }
    __device__ __forceinline__ u32x4 finish4(u32 t) const { return (u32x4){finish(0, t), finish(1, t), finish(2, t), finish(3, t)}; }
};
#define SCHED() __builtin_amdgcn_sched_barrier(0)
template <bool WRAP> __device__ __forceinline__ u32 unpack_round(u32 c, u32 t, u32 src, u32 dst) {
    const u32 cnt = popc(escbits(c)), inc = scan_incl(cnt);
    u32 off = src + inc - cnt;
    Unp<WRAP> u; u.fetch(c, off); SCHED();
    lds128w(dst, u.finish4(t));
    return (u32)__builtin_amdgcn_readlane((int)inc, 63);
}
__device__ __forceinline__ void ring_read8k(u32 a0, u32 (&r)[32]) {
#pragma unroll
    for (int j = 0; j < 8; ++j) { const u32x4 q = lds128((a0 + j * 1024) & RMASK); r[4 * j] = q.x; r[4 * j + 1] = q.y; r[4 * j + 2] = q.z; r[4 * j + 3] = q.w; }
}
__device__ __forceinline__ void unpack_exp(u32 S, u32 code, u32 scr, u32 lane, u32 (&Er)[32]) {
    const u32 a0 = S + lane * 16;
    if (code == RAWCODE) { ring_read8k(a0, Er); return; }
    const u32x4 ca = lds128(a0 & RMASK), cb = lds128((a0 + 1024) & RMASK);
    const u32 V = S + 2048;
    const u32x4 hd = lds128(V & RMASK);
    const u32 t1 = rfl(hd.x), t2 = rfl(hd.y), t3 = rfl(hd.z), t4 = rfl(hd.w), d2 = t1 & 255u, d3 = t2 & 127u, d4 = t3 & 63u;
    u32 c2[4], c3[2];
#pragma unroll
    for (int r = 0; r < 4; ++r) { const u32 i = r * 64 + lane; const u32 x = lds32((V + 16 + 4 * i) & RMASK); c2[r] = i < d2 ? x : 0x55555555u; }
#pragma unroll
    for (int r = 0; r < 2; ++r) { const u32 i = r * 64 + lane; const u32 x = lds32((V + 16 + 4 * (d2 + i)) & RMASK); c3[r] = i < d3 ? x : 0x55555555u; }
    const u32 x4 = lds32((V + 16 + 4 * (d2 + d3 + lane)) & RMASK), c4 = lane < d4 ? x4 : 0x55555555u;
    unpack_round<true>(c4, t4, V + 16 + 4 * (d2 + d3 + d4), scr + D_L4 + lane * 16);
    CBAR();
    const u32 s4 = scr + D_L4 + unpack_round<false>(c3[0], t3, scr + D_L4, scr + D_L3 + lane * 16);
    if (d3 > 64) unpack_round<false>(c3[1], t3, s4, scr + D_L3 + 1024 + lane * 16);
    CBAR();
    const u32 C[8] = {ca.x, ca.y, ca.z, ca.w, cb.x, cb.y, cb.z, cb.w};
    u32 cnt = 0;
#pragma unroll
    for (int k = 0; k < 8; ++k) cnt += popc(escbits(C[k]));
    {
        const u32 n0 = popc(escbits(c2[0])), n1 = popc(escbits(c2[1]));
        const u32 i01 = scan_incl(n0 | (n1 << 16)), tot = (u32)__builtin_amdgcn_readlane((int)i01, 63);
        cnt = scan_incl(cnt) - cnt;
        u32 o0 = scr + D_L3 + (i01 & 0xffffu) - n0, o1 = scr + D_L3 + (tot & 0xffffu) + (i01 >> 16) - n1;
        Unp<false> u0, u1; u0.fetch(c2[0], o0); u1.fetch(c2[1], o1); SCHED();
        lds128w(scr + D_L2 + lane * 16, u0.finish4(t2)); lds128w(scr + D_L2 + 1024 + lane * 16, u1.finish4(t2));
        if (d2 > 128) {
            u32 s3 = scr + D_L3 + (tot & 0xffffu) + (tot >> 16);
            s3 += unpack_round<false>(c2[2], t2, s3, scr + D_L2 + 2048 + lane * 16);
            if (d2 > 192) unpack_round<false>(c2[3], t2, s3, scr + D_L2 + 3072 + lane * 16);
        }
    }
    CBAR();
    u32 off = scr + D_L2 + cnt;
    Unp<false> u[8];
    u[0].fetch(C[0], off); u[1].fetch(C[1], off);
#pragma unroll
    for (int k = 0; k < 8; ++k) {
        SCHED();
        if (k + 2 < 8) u[k + 2].fetch(C[k + 2], off);
        SCHED();
#pragma unroll
        for (int j = 0; j < 4; ++j) Er[4 * k + j] = u[k].finish(j, t1);
    }
    CBAR();
}
__device__ __forceinline__ u32 x_piece_off(u32 half, int p) { return half * 16384 + p * 1024; }
__device__ __forceinline__ u32 w_piece_off(u32 h, int p) {
    const u32 g32 = p >> 2, n = (p >> 1) & 1, kb = p & 1;
    return (g32 & 1) * 16384 + ((2 * (2 * h + (g32 >> 1)) + n) * 2 + kb) * 1024;
}
__device__ __forceinline__ u16* y_base(const Params& P, u32 pm, u32 j, u32 w, u32 lane) {
    return P.out + (size_t)(pm * 256 + (w >> 2) * 64 + (lane >> 3)) * NN + j * 256 + (w & 3) * 64 + (lane & 7) * 8;
}
__device__ __forceinline__ size_t y_piece_off(int p) { return (size_t)((p >> 3) * 128 + (p & 7) * 8) * NN; }
__device__ __forceinline__ u32 local_panel(u32 j, u32 rank) { return j >= (u32)(NR * DP) ? j - (NR - 1) * DP : j - rank * DP; }
__device__ __forceinline__ u32 global_panel(u32 jj, u32 rank) { return jj >= (u32)DP ? jj + (NR - 1) * DP : jj + rank * DP; }
__device__ __forceinline__ u32* counter(const Params& P, u32 i) { return (u32*)(P.me + WS_CNT + i * 128); }
__device__ __forceinline__ u32* tmo_word(const Params& P) { return (u32*)(P.me + WS_TMO); }
__device__ __forceinline__ bool expired(const Params& P, u64 t0, u32 call) {
    if (__hip_atomic_load(tmo_word(P), __ATOMIC_RELAXED, __HIP_MEMORY_SCOPE_AGENT) == call || now() - t0 > DEADLINE) {
        __hip_atomic_store(tmo_word(P), call, __ATOMIC_RELAXED, __HIP_MEMORY_SCOPE_AGENT); return true; }
    return false;
}

struct ImageStore {
    char* o; u32* cnt; u32 half; bool isx, on;
    __device__ __forceinline__ ImageStore(const Params& P, u32 gid, u32 w, u32 q, u32 lane, bool on_) {
        isx = gid < (u32)GW; half = w & 1; on = on_;
        const u32 pn = isx ? gid & 15 : local_panel((gid - GW) >> 1, P.rank), ktl = (isx ? gid >> 4 : gid & 1) * 4 + (w >> 1);
        const u32 qi = (q - P.rank - 1) & 7, stage = qi == 7 ? 0u : ktl == 7 ? 5u : 1 + (ktl >> 1);
        const u32 slot = stage == 0 ? ktl : 8 + ktl * 7 + qi;
        o = P.me + (isx ? WS_XI : WS_WI) + (size_t)pn * PANEL + (size_t)slot * 32768 + lane * 16;
        cnt = counter(P, isx ? pn : 16 + pn) + stage;
    }
    __device__ __forceinline__ void operator()(int p, u32x4 x) const { st_dev(o + (isx ? x_piece_off(half, p) : w_piece_off(half, p)), x); }
    __device__ __forceinline__ void done(u32 lane) const { WAIT_VM(0); if (lane == 0) __hip_atomic_fetch_add(cnt, 1u, __ATOMIC_RELAXED, __HIP_MEMORY_SCOPE_AGENT); }
};
struct OutStore {
    u16* o; static constexpr bool on = true;
    __device__ __forceinline__ void operator()(int p, u32x4 x) const { st_out(o + y_piece_off(p), x); }
};
__device__ __forceinline__ u32* call_word(const Params& P, u32 wid) { return (u32*)(P.me + WS_CALL + blockIdx.x * 128 + wid * 4); }
__device__ __forceinline__ void call_load(const Params& P, u32 wid, u32 lane) {
    u32 keep;
    if (lane == 0) asm volatile("s_mov_b32 %0, m0\n\ts_mov_b32 m0, %2\n\ts_nop 0\n\tglobal_load_lds_dword %1, off sc1\n\ts_mov_b32 m0, %0" : "=&s"(keep) : "v"(call_word(P, wid)), "s"(CTL_CALL + wid * 4) : "memory");
}
__device__ __forceinline__ u32 call_take(u32 wid) { WAIT_VM(0); __builtin_amdgcn_s_waitcnt(0x0f70);   return rfl(lds32v(CTL_CALL + wid * 4)) + 1; }
__device__ __forceinline__ void call_bump(const Params& P, u32 wid, u32 lane, u32 call) { if (lane == 0) __hip_atomic_store(call_word(P, wid), call, __ATOMIC_RELAXED, __HIP_MEMORY_SCOPE_AGENT); }

__device__ __forceinline__ void hold(u32 ticks) { const u64 t0 = now(); while (now() - t0 < ticks) __builtin_amdgcn_s_sleep(2); }
__device__ __forceinline__ void load_tables(u32 lane) {
    const char* ka = (const char*)(const GAS char*)(u64)__builtin_amdgcn_kernarg_segment_ptr() + lane * 16;
    glds_plain(ka, LDS_TAB); glds_plain(ka + 1024, LDS_TAB + 1024); glds_plain(ka + 2048, LDS_TAB + 2048);
}

__device__ __forceinline__ u32 solo_gid(u32 c) { return (c >> 2) & 15; }
__device__ __forceinline__ u32 solo_w(u32 c) { return (c & 3) | (c >> 6) << 2; }
__device__ __forceinline__ void load_x(const Params& P, u32 gid, u32 w, u32 lane, u32 (&v)[64]) {
    const u16* s = P.x + (size_t)((gid & 15) * 256 + (w & 1) * 128 + (lane >> 3)) * KS + ((gid >> 4) * 4 + (w >> 1)) * 64 + ((lane ^ (lane >> 3)) & 7) * 8;
#pragma unroll
    for (int p = 0; p < 16; ++p) { const u32x4 t = *(const u32x4*)(s + p * 8 * KS); v[4 * p] = t.x; v[4 * p + 1] = t.y; v[4 * p + 2] = t.z; v[4 * p + 3] = t.w; }
}
__device__ __forceinline__ void puller(const Params& P, u32 role, u32 wid, u32 lane) {
    const u32 q = role & 7, sub = (role >> 3) & 15, PW = (role >> 7) & 15;
    if (wid >= 2) for (u32 pc = sub + PW * (7 - wid); pc < (u32)NRAW; pc += 6 * PW) {
        const u32 gid = solo_gid(pc), w = solo_w(pc);
        u32 v[64], Mr[32], Er[32];
        load_x(P, gid, w, lane, v);
        const u32 call = call_take(wid);
        split(v, Mr, Er);
        char* o = P.peer + WS_INB + (size_t)(((call & 1) * 8 + P.rank) * 64 + pc) * 8192 + lane * 16;
#pragma unroll
        for (int j = 0; j < 8; ++j) { WAIT_VM(0);   st_sys(o + j * 1024, (u32x4){Mr[4 * j], Mr[4 * j + 1], Mr[4 * j + 2], Mr[4 * j + 3]}); }
        WAIT_VM(0);
        if (lane == 0) __hip_atomic_store((u32*)(P.peer + WS_FLAG + (call & 1) * FLAG_PAR + (size_t)P.rank * FLAG_SRC + (GM + gid) * 32) + w, call << 8, __ATOMIC_RELAXED, __HIP_MEMORY_SCOPE_SYSTEM);
    }
    if (wid == 1) { hold(100); load_tables(lane); }
    const u32 call = call_take(wid);
    call_bump(P, wid, lane, call);
    if (wid == 0) {
        __builtin_amdgcn_s_setprio(3);
        const char* base = P.peer + WS_STG + (call & 1) * STG_PAR;
        const u32 voff = lane * 16;
        u32 pos = 0, pub = 0, limit = RING / 1024;
        for (u32 ui = 0;; ++ui) {
            u32 nu = rfl(lds32v(CTL_NUNIT));
            if ((nu & ~DONE) <= ui) {
                WAIT_VM(0); lds32vw(CTL_LANDED, pos); pub = pos;
                while (nu <= ui) { __builtin_amdgcn_s_sleep(1); nu = rfl(lds32v(CTL_NUNIT)); }
                if ((nu & ~DONE) <= ui) break;
            }
            CBAR();
            const u32x2 un = lds64v(D_UNIT + ui * 8);
            const u32 ut = rfl(un.y), total = ut & 0x3ffffu, lastc = ut >> 18;
            const char* src = base + rfl(un.x);
            for (u32 o = 0; o < total;) {
                if (pos + 4 > limit) {
                    bool drained = false;
                    for (;;) {
                        const u32 nx = lds32v(CTL_NEXT + (lane & 7) * 4);
                        u32 f = lastc;
#pragma unroll
                        for (u32 j = 0; j < ND; ++j) f = umin(f, (u32)__builtin_amdgcn_readlane((int)nx, j));
                        limit = (rfl(lds32v(D_DESC + f * 8)) >> 14) + RING / 1024;
                        if (pos + 4 <= limit) break;
                        if (!drained) { WAIT_VM(0); lds32vw(CTL_LANDED, pos); pub = pos; drained = true; }
                        __builtin_amdgcn_s_sleep(1);
                    }
                }
                const u32 dst = (pos * 1024) & RMASK;
                if (o + 4096 <= total && dst + 4096 <= RING) {
                    u32 keep;
                    asm volatile("s_mov_b32 %0, m0\n\ts_mov_b32 m0, %3\n\ts_nop 4\n\t"
                                 "s_waitcnt vmcnt(%4)\n\tglobal_load_lds_dwordx4 %1, %2 sc0 sc1\n\t"
                                 "s_waitcnt vmcnt(%4)\n\tglobal_load_lds_dwordx4 %1, %2 offset:1024 sc0 sc1\n\t"
                                 "s_waitcnt vmcnt(%4)\n\tglobal_load_lds_dwordx4 %1, %2 offset:2048 sc0 sc1\n\t"
                                 "s_waitcnt vmcnt(%4)\n\tglobal_load_lds_dwordx4 %1, %2 offset:3072 sc0 sc1\n\t"
                                 "s_mov_b32 m0, %0" : "=&s"(keep) : "v"(voff), "s"(src + o), "s"(dst), "i"(WIN) : "memory");
                    pos += 4; o += 4096;
                } else {
                    u32 keep;
                    if (o + voff < total) asm volatile("s_mov_b32 %0, m0\n\ts_mov_b32 m0, %3\n\ts_nop 4\n\ts_waitcnt vmcnt(%4)\n\tglobal_load_lds_dwordx4 %1, %2 sc0 sc1\n\ts_mov_b32 m0, %0"
                                                       : "=&s"(keep) : "v"(voff), "s"(src + o), "s"(dst), "i"(WIN) : "memory");
                    pos += 1; o += 1024;
                }
                if (pos > pub + WIN + 1) { pub = pos - WIN - 1; lds32vw(CTL_LANDED, pub); }
            }
        }
    } else if (wid == 1) {
        u32 basek = 0, nch = 0, nun = 0;
        u32 held = 0, heldw = 0, helds = 0;
        auto release = [&](u32 c) {
            const u32 a0 = held & ~127u, total = held - a0 + 8192;
            if (lane == 0) { lds64vw(D_UNIT + nun * 8, (u32x2){a0, total | (c << 18)}); lds32vw(D_DESC + c * 8 + 4, heldw | ((basek * 1024 + held - a0 - helds) >> 4) << 13); }
            nun += 1; basek += (total + 1023) >> 10; held = 0;
        };
        u32* ticket = (u32*)(P.me + WS_CLAIM + q * 128);
        const u32 nown = P.nunits & 0xffffu, nall = nown + (P.nunits >> 16);
        u32 tbase = NSTAT * PW - (call - 1) * (nown - NSTAT * PW + PW), tend = nown;
        for (u32 ui = 0;; ++ui) {
            while (rfl(lds32v(CTL_LANDED)) + LEAD < basek + (held ? 8u : 0u)) __builtin_amdgcn_s_sleep(4);
            u32 idx = ui * PW + sub;
            if (ui >= (u32)NSTAT) {
                if (lane == 0) idx = __hip_atomic_fetch_add(ticket, 1u, __ATOMIC_RELAXED, __HIP_MEMORY_SCOPE_AGENT);
                idx = rfl(idx) + tbase;
                if (idx >= tend) {
                    if (tend == nall) break;
                    ticket = (u32*)(P.me + WS_CLAIM + P.rank * 128); tbase = nown - (call - 1) * (nall - nown + NPULL); tend = nall;
                    continue;
                }
            }
            const u32 e = tab16(TAB_STREAM, idx), gid = e & 1023u, part = e >> 10, lv = 31 - __builtin_clz(part), n = 8u >> lv, first = (part - (1u << lv)) * n;
            if (nch + n > (u32)(NS * 8)) break;
            const bool solo = gid < (u32)SOLOG, mine = lane >= first && lane < first + n, sent = gid < (u32)(NRAW / 4) && first < 4;
            const u32* f = (const u32*)(P.me + WS_FLAG + (call & 1) * FLAG_PAR + (size_t)q * FLAG_SRC + (gid + (sent && (lane & 8) ? GM : 0)) * 32) + (lane & 7);
            const u64 t0 = now(); u32 val; bool dead = false;
            for (;;) {
                val = __hip_atomic_load(f, __ATOMIC_RELAXED, __HIP_MEMORY_SCOPE_SYSTEM);
                if (__all((val >> 8) == (call & 0xffffffu) || (lane & 7) >= first + n || (solo && (lane & 7) < first))) break;
                if (__any(expired(P, t0, call))) { dead = true; break; }
                __builtin_amdgcn_s_sleep(1);
            }
            if (dead) {
                const u32 t = gid - GY;
                if (lane == 0 && t < (u32)(16 * PK)) *y_base(P, t & 15, global_panel(t >> 4, q), first, 0u) = 0x7fc0;
                break;
            }
            u32 code = val & 255u; if (code > 127u) code = RAWCODE;
            const u32 cb = chunk_bytes(code) - (sent ? 8192u : 0u), sz = solo ? (mine ? (cb + 1023u) & ~1023u : 0u) : (lane < first + n ? cb : 0u);
            u32 inc = sz; inc += dpp<0x111, 0xf, true>(inc); inc += dpp<0x112, 0xf, true>(inc); inc += dpp<0x114, 0xf, true>(inc);
            const bool two = n == 1;
            const u32 a0 = (u32)__builtin_amdgcn_readlane((int)(inc - sz), (int)first) & ~127u, end = (u32)__builtin_amdgcn_readlane((int)inc, 7) - (two ? 8192u : 0u), total = end - a0;
            const u32 S = basek * 1024 + inc - sz - a0;
            if (mine) lds64vw(D_DESC + (nch + lane - first) * 8, (u32x2){(S << 4) | code, (gid * 8 + lane) | (two ? 0u : (chunk_bytes(code) - 8192) >> 4) << 13});
            if (solo) { if (mine) lds64vw(D_UNIT + (nun + lane - first) * 8, (u32x2){gid * (u32)GRP_STRIDE + lane * RAWB, cb | ((nch + lane - first) << 18)}); nun += n; }
            else { if (lane == 0) lds64vw(D_UNIT + nun * 8, (u32x2){gid * (u32)GRP_STRIDE + a0, total | ((nch + n - 1) << 18)}); nun += 1; }
            basek += (total + 1023) >> 10;
            if (held) release(nch - 1);
            if (two) { held = gid * (u32)GRP_STRIDE + end; heldw = gid * 8 + first; helds = (u32)__builtin_amdgcn_readlane((int)S, (int)first); }
            nch += n;
            WAIT_LGKM0();
            lds32vw(CTL_NUNIT, nun); lds32vw(CTL_NCHUNK, nch);
        }
        if (held) { release(nch - 1); WAIT_LGKM0(); }
        lds32vw(CTL_NUNIT, nun | DONE); lds32vw(CTL_NCHUNK, nch | DONE);
    } else {
        const u32 dw = wid - 2, scr = D_SCR + dw * D_SCRB;
        for (u32 c = dw;; c += ND) {
            u32 nc = rfl(lds32v(CTL_NCHUNK));
            while (nc <= c) { __builtin_amdgcn_s_sleep(2); nc = rfl(lds32v(CTL_NCHUNK)); }
            if ((nc & ~DONE) <= c) break;
            CBAR();
            const u32x2 de = lds64v(D_DESC + c * 8);
            const u32 d = rfl(de.x), code = d & 255u, S = (d >> 8) << 4, gid = (rfl(de.y) >> 3) & 1023u, w = rfl(de.y) & 7;
            while (rfl(lds32v(CTL_LANDED)) < (S + chunk_bytes(code) - 8192 + 1023) >> 10) __builtin_amdgcn_s_sleep(1);
            CBAR();
            u32 Mr[32], Er[32];
            unpack_exp(S, code, scr, lane, Er);
            if (gid < (u32)(NRAW / 4) && w < 4) {
                const char* m = P.me + WS_INB + (size_t)(((call & 1) * 8 + q) * 64 + gid * 4 + w) * 8192 + lane * 16;
                u32x4 r[8];
                asm volatile("global_load_dwordx4 %0, %8, off sc0 sc1\n\tglobal_load_dwordx4 %1, %8, off offset:1024 sc0 sc1\n\t"
                             "global_load_dwordx4 %2, %8, off offset:2048 sc0 sc1\n\tglobal_load_dwordx4 %3, %8, off offset:3072 sc0 sc1\n\t"
                             "global_load_dwordx4 %4, %9, off sc0 sc1\n\tglobal_load_dwordx4 %5, %9, off offset:1024 sc0 sc1\n\t"
                             "global_load_dwordx4 %6, %9, off offset:2048 sc0 sc1\n\tglobal_load_dwordx4 %7, %9, off offset:3072 sc0 sc1\n\ts_waitcnt vmcnt(0)"
                             : "=&v"(r[0]), "=&v"(r[1]), "=&v"(r[2]), "=&v"(r[3]), "=&v"(r[4]), "=&v"(r[5]), "=&v"(r[6]), "=&v"(r[7]) : "v"(m), "v"(m + 4096) : "memory");
#pragma unroll
                for (int j = 0; j < 8; ++j) { Mr[4 * j] = r[j].x; Mr[4 * j + 1] = r[j].y; Mr[4 * j + 2] = r[j].z; Mr[4 * j + 3] = r[j].w; }
            } else {
                u32 dm = rfl(de.y) >> 13;
                while (!dm) { __builtin_amdgcn_s_sleep(1); dm = rfl(lds32v(D_DESC + c * 8 + 4)) >> 13; }
                const u32 SM = S + dm * 16;
                while (rfl(lds32v(CTL_LANDED)) < (SM + 9215) >> 10) __builtin_amdgcn_s_sleep(1);
                CBAR();
                ring_read8k(SM + lane * 16, Mr);
            }
            WAIT_LGKM0(); lds32vw(CTL_NEXT + dw * 4, c + ND); CBAR();
            if (gid >= (u32)GY) {
                const u32 t = gid - GY;
                u16* o = y_base(P, t & 15, global_panel(t >> 4, q), w, lane);
#pragma unroll
                for (int p = 0; p < 16; ++p) st_out(o + y_piece_off(p), piece(Mr, Er, p));
            } else {
                const ImageStore im(P, gid, w, q, lane, true);
#pragma unroll
                for (int p = 0; p < 16; ++p) im(p, piece(Mr, Er, p));
                im.done(lane);
            }
        }
    }
}

__device__ __forceinline__ void pack_inputs(const Params& P, u32 gid, u32 w, bool solo, u32 wid, u32 lane) {
    asm volatile("" : "+v"(lane));
    u32 v[64];
    u32 peers = 255u & ~(1u << P.rank); bool own = true;
    if (gid < (u32)GW) load_x(P, gid, w, lane, v);
    else {
        const u32 j = (gid - GW) >> 1, ktl = (gid & 1) * 4 + (w >> 1);
        if (j < (u32)(NR * DP)) { const u32 dest = j / DP; own = dest == (u32)P.rank; peers = own ? 0u : 1u << dest; }
        else if (j < (u32)(NR * (DP + FX))) own = (j & 7) != P.rank;
        const u16* s = P.w + (size_t)(ktl * 64 + (lane >> 4)) * NN + j * 256 + (w & 1) * 128 + (lane & 15) * 8;
        const u32 scr = wid * 16384;
#pragma unroll
        for (int i = 0; i < 16; ++i) glds_w(s + (size_t)i * 4 * NN, scr + i * 1024);
        WAIT_VM(0);
        const u32 ta = scr + ((lane >> 4) * 8 + ((lane & 15) >> 2)) * 256 + (lane & 3) * 16;
#pragma unroll
        for (int p = 0; p < 16; ++p) {
            const u32 a = ta + (p & 1) * 32 * 256 + (p >> 2) * 64 + ((p >> 1) & 1) * 8;
            const u32x2 lo = __builtin_bit_cast(u32x2, __builtin_amdgcn_ds_read_tr16_b64_v4i16((LAS s16x4*)a));
            const u32x2 hi = __builtin_bit_cast(u32x2, __builtin_amdgcn_ds_read_tr16_b64_v4i16((LAS s16x4*)(a + 4 * 256)));
            v[4 * p] = lo.x; v[4 * p + 1] = lo.y; v[4 * p + 2] = hi.x; v[4 * p + 3] = hi.y;
        }
        WAIT_LGKM0();
    }
    const u32 call = call_take(wid);
    const ImageStore im(P, gid, w, P.rank, lane, own);
    if (peers) {
        u32* fp; u32 fv;
        pack_group<false>(P, v, gid, w, solo, peers, call, wid, lane, im, fp, fv);
        WAIT_VM(0);
        if (!solo) BAR();
        raise_flags(fp, fv);
        if (solo) { load_x(P, gid, w, lane, v); __builtin_amdgcn_s_waitcnt(0x0f70); }
        else if (own && lane == 0) __hip_atomic_fetch_add(im.cnt, 1u, __ATOMIC_RELAXED, __HIP_MEMORY_SCOPE_AGENT);
    }
    if (!peers || solo) {
#pragma unroll
        for (int p = 0; p < 16; ++p) im(p, (u32x4){v[4 * p], v[4 * p + 1], v[4 * p + 2], v[4 * p + 3]});
        im.done(lane);
    }
}

typedef __bf16 bf16x2 __attribute__((ext_vector_type(2)));
typedef float f32x2 __attribute__((ext_vector_type(2)));
__device__ __forceinline__ u32 cvt_pk(float lo, float hi) { return __builtin_bit_cast(u32, __builtin_convertvector((f32x2){lo, hi}, bf16x2)); }
__device__ __forceinline__ void wait_panels(const Params& P, u32 tile, u32 upto, u32 call, u32 wid, u32 lane) {
    if (wid == 0) {
        const u32 st = umin(lane & 7, upto), need = call * (st == 0 ? 16u : st < 4 ? 28u : 14u);
        const u32* c = counter(P, (lane & 8) ? 16 + (tile >> 4) : (tile & 15)) + st;
        const u64 t0 = now();
        for (;;) {
            const u32 x = __hip_atomic_load(c, __ATOMIC_RELAXED, __HIP_MEMORY_SCOPE_AGENT);
            if (__all((int)(x - need) >= 0)) break;
            if (__any(expired(P, t0, call))) { asm volatile("ds_write_b32 %0, %1\n\ts_waitcnt lgkmcnt(0)" :: "v"(CTL_BAD), "v"(1u) : "memory"); break; }
            __builtin_amdgcn_s_sleep(4);
        }
        __builtin_amdgcn_fence(__ATOMIC_ACQUIRE, "agent");
        WAIT_VM(0);
    }
    BAR();
}
__device__ __forceinline__ void stage2(u32 dst, u32 voff, const char* src) {
    asm volatile("s_mov_b32 m0, %2\n\ts_nop 0\n\tglobal_load_lds_dwordx4 %0, %1\n\tglobal_load_lds_dwordx4 %0, %1 offset:1024" :: "v"(voff), "s"(src), "s"(dst) : "memory");
}
__device__ __forceinline__ void stage4(u32 dst, u32 voff, const char* src) {
    asm volatile("s_mov_b32 m0, %2\n\ts_nop 0\n\tglobal_load_lds_dwordx4 %0, %1\n\tglobal_load_lds_dwordx4 %0, %1 offset:1024\n\t"
                 "global_load_lds_dwordx4 %0, %1 offset:2048\n\tglobal_load_lds_dwordx4 %0, %1 offset:3072" :: "v"(voff), "s"(src), "s"(dst) : "memory");
}
__device__ __forceinline__ void gemm_role(const Params& P, u32 place, u32 call, u32 wid, u32 lane) {
    const u32 wr = wid >> 2, wc = wid & 3;
    u32 aoff = wr * 8192 + (lane & 15) * 128 + (((lane >> 4) ^ lane) & 7) * 16, aoff1 = aoff ^ 64, boff = 65536 + wc * 4096 + lane * 16;
    const u32 mA = wid * 2048, mB = 65536 + wid * 4096;
    u32 vA0 = mA + lane * 16, vA1 = vA0 + 16384, vA2 = vA0 + 32768, vA3 = vA0 + 49152, vB0 = wid * 4096 + lane * 16, vB1 = vB0 + 32768;
    asm volatile("" : "+v"(vA0), "+v"(vA1), "+v"(vA2), "+v"(vA3), "+v"(vB0), "+v"(vB1), "+v"(aoff), "+v"(aoff1), "+v"(boff));
#define G_LDA(b, h) do { _Pragma("unroll") for (int m = 0; m < 4; ++m) _Pragma("unroll") for (int k = 0; k < 2; ++k) At[m][k] = *(const LAS bf16x8*)((k ? aoff1 : aoff) + ((b) * 2 + (h)) * 16384 + m * 2048); } while (0)
#define G_LDB(b) do { _Pragma("unroll") for (int j = 0; j < 2; ++j) _Pragma("unroll") for (int n = 0; n < 2; ++n) _Pragma("unroll") for (int k = 0; k < 2; ++k) \
        Bt[j][n][k] = *(const LAS bf16x8*)(boff + ((b) * 2 + j) * 16384 + n * 2048 + k * 1024); } while (0)
#define G_MMA(ai, STEP) do { G_WV(8); __builtin_amdgcn_s_waitcnt(0xc07f);   G_SCHED; G_BAR; __builtin_amdgcn_s_setprio(1); _Pragma("unroll") for (int i = 0; i < 32; ++i) { \
        const int bj = i >> 4, k = (i >> 3) & 1, m = (i >> 1) & 3, n = i & 1; \
        if (i == 8) { G_SCHED; STEP; G_SCHED; } \
        if (i == 32 - 2) { G_SCHED; __builtin_amdgcn_s_setprio(3); G_BAR; G_SCHED; } \
        acc[ai][bj][m][n] = __builtin_amdgcn_mfma_f32_16x16x32_bf16(Bt[bj][n][k], At[m][k], acc[ai][bj][m][n], 0, 0, 0); } \
        __builtin_amdgcn_s_setprio(0); G_SCHED; } while (0)
#define G_WV(n) asm volatile("s_waitcnt vmcnt(" #n ")" ::: "memory")
#define G_BAR __builtin_amdgcn_s_barrier()
#define G_SCHED __builtin_amdgcn_sched_barrier(0)
    const u32x2 cl = lds64v(TAB_COLS + ((blockIdx.x & 4) * 2 + (place >> 2)) * 8);
    const u64 mycols = (u64)rfl(cl.y) << 32 | rfl(cl.x); const u32 pm = (blockIdx.x & 3) * 4 + (place & 3);
    const char* const cA = P.me + WS_XI + (size_t)pm * PANEL; const char* const WI = P.me + WS_WI;
    u32 cur = pm | ((u32)mycols & 255u) << 4;
    bool fresh = true; u32* fp = nullptr; u32 fv = 0;
    bf16x8 At[4][2], Bt[2][2][2];
    for (u32 ui = 0;; ++ui) {
        const u32 nxt = pm | ((u32)(mycols >> (8 * ui + 8)) & 255u) << 4;
        const bool has_next = (nxt >> 4) != 255u, dist = (cur >> 4) < (u32)PK, chain = has_next && !dist;
        const char* cB = WI + (size_t)(cur >> 4) * PANEL;
        if (fresh) {
            wait_panels(P, cur, ui > 1 ? 5u : 0u, call, wid, lane);
            stage4(mB, vB0, cB); stage2(mA, vA0, cA); stage2(mA + 16384, vA1, cA);
            if (wr == 1) G_BAR;
            G_WV(2); G_BAR;
            stage4(mB + 32768, vB1, cB); stage2(mA + 32768, vA2, cA);
            G_WV(6); G_BAR;
            raise_flags(fp, fv); fp = nullptr; fv = 0;
        }
        f32x4 acc[2][2][4][2];
#pragma unroll
        for (int a = 0; a < 2; ++a)
#pragma unroll
            for (int b = 0; b < 2; ++b)
#pragma unroll
                for (int m = 0; m < 4; ++m)
#pragma unroll
                    for (int n = 0; n < 2; ++n) acc[a][b][m][n] = (f32x4){0.f, 0.f, 0.f, 0.f};
        const char *p0 = cA, *pA = cA + 65536, *pB = cB + 65536;
        u32 stops = 0x80000000u | (ui == 0 ? 0x09020408u : ui == 1 ? 0x00020008u : 0u), upto = 0;
        for (u32 trip = 0;;) {
            const u32 end = stops ? (u32)__builtin_ctz(stops) : 32u;
            u32 left = end - trip;
            do {
                G_LDB(0); G_SCHED; G_LDA(0, 0); stage2(mA + 49152, vA3, p0);
                G_MMA(0, );
                G_LDA(0, 1); stage4(mB, vB0, pB); stage2(mA, vA0, pA);
                G_MMA(1, );
                G_LDB(1); G_SCHED; G_LDA(1, 0); stage2(mA + 16384, vA1, pA);
                G_MMA(0, );
                G_LDA(1, 1); stage4(mB + 32768, vB1, pB); stage2(mA + 32768, vA2, pA);
                G_MMA(1, p0 = pA; pA += 65536; pB += 65536; --left; asm volatile("" : "+s"(p0), "+s"(pA), "+s"(pB), "+s"(left)));
            } while (left);
            if (!stops) break;
            trip = end; stops &= stops - 1;
            if (stops) { upto += ui ? (upto ? 3u : 2u) : 1u; wait_panels(P, cur, upto, call, wid, lane); }
            else {
                pA = cA; pB = cB;
                if (chain) { pB = WI + (size_t)(nxt >> 4) * PANEL; wait_panels(P, nxt, ui ? 5u : 0u, call, wid, lane); }
            }
        }
        if (!chain) G_WV(0);
        if (wr == 0) G_BAR;
        u32 v[64];
        {
            int tz = threadIdx.x; asm volatile("" : "+v"(tz));
            const u32 ln = tz & 63, bad = lds32v(CTL_BAD);
            const u32 ep = D_DESC + wid * 2048, to = ep + (ln & 15) * 128 + (((ln >> 4) ^ (ln & 7)) * 16), from = ep + (ln ^ (ln >> 3)) * 16;
            u16* o = y_base(P, cur & 15, global_panel(cur >> 4, P.rank), wid, ln);
#pragma unroll
            for (int p = 0; p < 16; p += 2) {
                const int ai = p >> 3, m = (p >> 1) & 3;
#pragma unroll
                for (int bj = 0; bj < 2; ++bj) {
                    const f32x4 x0 = acc[ai][bj][m][0], x1 = acc[ai][bj][m][1];
                    lds128w(to ^ (bj * 64), (u32x4){bad ? 0x7fc07fc0u : cvt_pk(x0[0], x0[1]), cvt_pk(x0[2], x0[3]), cvt_pk(x1[0], x1[1]), cvt_pk(x1[2], x1[3])});
                }
                CBAR();
#pragma unroll
                for (int j = 0; j < 2; ++j) { const u32x4 t = lds128(from + j * 1024); v[4 * (p + j)] = t.x; v[4 * (p + j) + 1] = t.y; v[4 * (p + j) + 2] = t.z; v[4 * (p + j) + 3] = t.w; }
                CBAR();
            }
            if (!dist) {
#pragma unroll
                for (int p = 0; p < 16; ++p) st_out(o + y_piece_off(p), (u32x4){v[4 * p], v[4 * p + 1], v[4 * p + 2], v[4 * p + 3]});
            }
            if (chain) { if (wr == 1) G_BAR; }
            else {
                if (dist) {
                    sample<6, 10, 5>(v, LDS_HIST, ln);
                    WAIT_LGKM0(); BAR();
                    const OutStore os = {o};
                    pack_group<true>(P, v, GY + cur, wid, false, (cur >> 4) < (u32)DP ? 255u & ~(1u << P.rank) : 1u << (((cur >> 4) - DP) & 7), call, wid, ln, os, fp, fv);
                    if (!has_next) { G_WV(0); BAR(); raise_flags(fp, fv); }
                }
                if (!has_next) break;
            }
        }
        fresh = !chain; cur = nxt;
    }
}

__global__ void __launch_bounds__(512, 2) fused(const Args A) {
    const u32 wid = rfl(threadIdx.x >> 6), lane = threadIdx.x & 63;
    asm volatile("gar_code0:");
    const u32 role = pin((((const u32*)A.role)[blockIdx.x >> 1] >> ((blockIdx.x & 1) * 16)) & 0xffffu);
    char* peer = pin(A.ws[0]); char* lanep = peer;
#pragma unroll
    for (u32 j = 1; j < NR; ++j) { char* c = pin(A.ws[j]); peer = (role & 7) == j ? c : peer; lanep = (lane & 7) == j ? c : lanep; }
    const Params P = {pin(A.me), peer, pin(A.x), pin(A.w), pin(A.out), pin((u32)A.rank), pin((u32)A.nunits)};
    call_load(P, wid, lane);
    if (role & 0x8000u) {
        if (threadIdx.x == 0) { lds32vw(CTL_NUNIT, 0u); lds32vw(CTL_LANDED, 0u); lds32vw(CTL_NCHUNK, 0u); }
        if (wid == 1 && lane < 8) lds32vw(CTL_NEXT + lane * 4, lane);
        WAIT_LGKM0(); BAR();
        puller(P, role, wid, lane); return;
    }
    if (threadIdx.x == 0) lds32vw(CTL_BAD, 0u);
    if (threadIdx.x < 8) lds32vw(CTL_SIZE + threadIdx.x * 4, 0u);
    pack_init(wid, lane);
    const u32 rb = NENC - 1 - (role & 255u), sc = rb + (wid & 2) * 32;
    const bool soloist = (wid == 1 || wid == 2) && rb < 64u;
    if (soloist) { if (lane < 8) lds64vw(LDS_TAB + lane * 8, __builtin_bit_cast(u32x2, (u64)lanep)); }
    else {
        if (rb >= 64u) {
            if (wid == 3) {
                u64 pc; u32 back, len;
                asm volatile("s_getpc_b64 %0\ngar_here:\n\ts_mov_b32 %1, gar_here-gar_code0\n\ts_mov_b32 %2, gar_code1-gar_code0" : "=s"(pc), "=s"(back), "=s"(len));
                for (u32 o = ((blockIdx.x >> 3) & 15) * 1024 + lane * 16; o < len; o += 16384) glds_plain((const char*)(const GAS char*)((pc - back) & ~15ull) + o, LDS_CTL - 1024);
            }
        } else hold(100);
        if (wid == 0) load_tables(lane);
        if (rb >= 64u) hold(150);
    }
    for (u32 it = 0; it < 3; ++it) {
        const u32 gid = it ? tab16(TAB_ENC, (role & 255u) * 2 + it - 1) : solo_gid(sc);
        if (gid == 0xffffu) break;
        if (it || soloist) pack_inputs(P, gid, it ? wid : solo_w(sc), !it, wid, lane);
        WAIT_VM(0); WAIT_LGKM0(); BAR();
    }
    const u32 call = call_take(wid);
    call_bump(P, wid, lane, call);
    if (role >> 8) gemm_role(P, (role >> 8) - 1, call, wid, lane);
    asm volatile("gar_code1:");
}

static void build_tables(Args& P) {
    unsigned char* cols = P.cols; u16 *stream = P.stream, *role = P.role, *job = P.enc, enc[NPACK];
    {
        int peer[2][4], np[2] = {0, 0}, cnt[8] = {}, seen[2] = {0, 0}, n = 0, top = NENC;
        for (int d = 1; d < NR; ++d) { const int q = (P.rank + d) & 7, h = ((0xb2 >> q) ^ (0xb2 >> P.rank)) & 1; peer[h][np[h]++] = q; }
        for (int b = 0; b < NWG; ++b) {
            const int h = (b >> 2) & 1, i = b >> 3, g0 = 32 - 4 * (h ? CH : CL), near = h ? 4 * NH : 3 * NL, far = h ? 3 * FL : 4 * FH, k = seen[h]++;
            if (k >= near + far) { role[b] = i >= g0 ? (u16)(n++ | (i - g0 + 1) << 8) : (u16)--top; continue; }
            const int side = k < near ? h : !h, j = k < near ? k : k - near, q = peer[side][side ? (j + j / 4) & 3 : j % 3];
            role[b] = (u16)(0x8000 | q | cnt[q]++ << 3);
        }
        for (int b = 0; b < NWG; ++b) if (role[b] & 0x8000) role[b] |= (u16)(cnt[role[b] & 7] << 7);
    }
    int col[LC], nc = 0;
    {
        const int rest = LC - (ROUNDS - 1) * CPR, r0 = (rest + 1) / 2, light = rest < CPR ? CPR - rest : CPR;
        int over = PK - FX - light * (ROUNDS - 1), nd = 0, nr = PK;
        for (int i = 0; i < 128; ++i) cols[i] = 0xff;
        for (int r = 0; r < ROUNDS; ++r) for (int c = 0; c < 8; ++c) for (int cg = 0; cg < 2; ++cg) {
            const bool last = c < (cg ? rest - r0 : r0), heavy = last && rest < CPR;
            if (nd >= DP && nd < PK && ((nd - DP) & 7) == P.rank) ++nd;
            if (c >= (cg ? CH : CL) || (r + 1 == ROUNDS && !last)) continue;
            cols[(cg * 8 + c) * 8 + r] = (unsigned char)(col[nc++] = nd < PK && (!heavy || over-- > 0) ? nd++ : nr++);
        }
    }
    auto wgid = [&](int jj, int dest, int h) { return GW + 2 * (jj >= DP ? jj + (NR - 1) * DP : dest * DP + jj) + h; };
    int k = 0;
    auto put = [&](int gid, int lv) { for (int j = 0; j < (1 << lv); ++j) stream[k++] = (u16)(gid | ((1 << lv) + j) << 10); };
    auto opnd = [&](int i, int h) { return i < 16 ? GX + 16 * h + i : wgid(col[i - 16], P.rank, h); };
    auto stage = [&](int st, int i0, int i1) {
        const int h = st >> 1, half = st & 1;
        for (int i = i0; i < i1; ++i) {
            if (i < 16 && h == 0) { for (int j = 0; j < 2; ++j) stream[k++] = (u16)(opnd(i, h) | (4 + 2 * half + j) << 10); }
            else stream[k++] = (u16)(opnd(i, h) | (2 + half) << 10);
        }
    };
    stage(0, NRAW / 4, 16); stage(1, 0, 12);
    {
        u16 t[64]; const int n = k; int e = 0;
        for (int i = 0; i < n; ++i) t[i] = stream[i];
        k = 0;
        for (int i = 0; i < n || e < NRAW / 4; ++i) { if (i < n) stream[k++] = t[i]; if (i >= 18 && e < NRAW / 4) stream[k++] = (u16)((GX + e++) | 2 << 10); }
    }
    const int first = 16 + CPR - 4;
    stage(0, 16, first); stage(1, 12, first); stage(0, first, 16 + CPR); stage(2, 0, first); stage(1, first, 16 + CPR);
    for (int part = 6; part < 8; ++part) for (int i = 0; i < first; ++i) stream[k++] = (u16)(opnd(i, 1) | part << 10);
    stage(2, first, 16 + CPR); stage(3, first, 16 + CPR);
    for (int h = 0; h < 2; ++h) for (int i = CPR; i < 2 * CPR; ++i) put(wgid(col[i], P.rank, h), 0);
    for (int i = 2 * CPR; i < LC; ++i) for (int h = 0; h < 2; ++h) put(wgid(col[i], P.rank, h), 0);
    for (int t = 0; t < 16 * DP; ++t) { const int left = 16 * DP - t; put(GY + t, left <= 8 ? 1 : 0); }
    const int nown = k;
    for (int t = 0; t < 16 * FX; ++t) put(GY + (DP + (t >> 4) * 8 + P.rank) * 16 + (t & 15), 16 * FX - t <= 16 ? 3 : 16 * FX - t <= 30 ? 2 : 1);
    P.nunits = nown | (k - nown) << 16;
    k = 0;
    for (int i = 0; i < CPR; ++i) for (int d = 1; d < (col[i] < DP ? NR : 2); ++d) enc[k++] = (u16)wgid(col[i], (P.rank + d) & 7, 0);
    for (int i = SOLOG; i < 32; ++i) enc[k++] = (u16)(GX + i);
    for (int i = 0; i < CPR; ++i) { if (col[i] < DP) enc[k++] = (u16)wgid(col[i], P.rank, 0); enc[k++] = (u16)wgid(col[i], P.rank, 1); }
    for (int i = 0; i < CPR; ++i) if (col[i] < DP) for (int d = 1; d < NR; ++d) enc[k++] = (u16)wgid(col[i], (P.rank + d) & 7, 1);
    for (int f = 0; f < FX; ++f) for (int h = 0; h < 2; ++h) enc[k++] = (u16)wgid(DP + f * 8 + P.rank, 0, h);
    for (int i = CPR; i < LC; ++i) for (int h = 0; h < 2; ++h) for (int d = 0; d < (col[i] < DP ? NR : 1); ++d) enc[k++] = (u16)wgid(col[i], (P.rank + d) & 7, h);
    int nx = 0, top = NENC;
    for (int i = 0; i < NENC - 64; ++i) { job[2 * i] = enc[nx++]; job[2 * i + 1] = 0xffff; }
    for (int i = 0; i < NENC - 64; ++i) if (job[2 * i] >= GW && (job[2 * i] - GW) / 2 / DP == P.rank) job[2 * i + 1] = enc[nx++];
    for (int i = NENC - 64; i < NENC; ++i) { job[2 * i] = enc[nx++]; job[2 * i + 1] = 0xffff; }
    while (nx < NPACK) { do --top; while (job[2 * top + 1] != 0xffff); job[2 * top + 1] = enc[nx++]; }
}
}

extern "C" void kernel_launch(int rank, int nranks, ncclComm_t comm, hipStream_t stream,
                              void* const* d_in, const int* in_sizes, int n_in,
                              void* d_out, int out_size, const int* in_dtypes, int out_dtype,
                              void* const* d_out_all, void* const* d_ws_all, size_t ws_bytes) {
    using namespace gar;
    if (nranks != NR || (unsigned)rank >= (unsigned)NR || n_in != 2 || ws_bytes < WS_END || in_sizes[0] != MM * KS || in_sizes[1] != KS * NN || out_size != MM * NN) {
        fprintf(stderr, "kernel_launch: unexpected shapes\n"); return; }
    Args P = {};
    for (int p = 0; p < NR; ++p) P.ws[p] = (char*)d_ws_all[p];
    P.me = P.ws[rank];
    P.x = (const u16*)d_in[0]; P.w = (const u16*)d_in[1]; P.out = (u16*)d_out; P.rank = rank;
    build_tables(P);
    HIP_CHECK(hipFuncSetAttribute((const void*)fused, hipFuncAttributeMaxDynamicSharedMemorySize, LDS_BYTES));
    hipLaunchKernelGGL(fused, dim3(NWG), dim3(512), LDS_BYTES, stream, P);
    HIP_CHECK(hipGetLastError());
}
```

```cpp
#include <hip/hip_runtime.h>
#include <rccl/rccl.h>
#include <cstdio>
#include <cstdint>
#include <cstddef>
#pragma clang diagnostic ignored "-Wint-to-pointer-cast"

#define HIP_CHECK(expr) do { hipError_t _e = (expr); if (_e != hipSuccess) { \
    fprintf(stderr, "HIP error %s at %s:%d\n", hipGetErrorString(_e), __FILE__, __LINE__); \
    return; } } while (0)

namespace gar {
#define LAS __attribute__((address_space(3)))
#define GAS __attribute__((address_space(1)))
typedef unsigned u32;
typedef unsigned short u16;
typedef unsigned long long u64;
typedef unsigned u32x4 __attribute__((ext_vector_type(4)));
typedef unsigned u32x2 __attribute__((ext_vector_type(2)));
typedef short s16x4 __attribute__((ext_vector_type(4)));
typedef short bf16x8 __attribute__((ext_vector_type(8)));
typedef float f32x4 __attribute__((ext_vector_type(4)));

constexpr int NR = 8, MM = 4096, NN = 32768, KS = 512;
constexpr int DP = 10;
constexpr int RP = 128 - NR * DP;
constexpr int LP = DP + RP;
constexpr int NWG = 256;
constexpr int NL = 9, FL = 0, NH = 8, FH = 1, CL = 6, CH = 6, CPR = CL + CH;
constexpr int NPULL = 3 * (NL + FL) + 4 * (NH + FH);
constexpr int WIN = 5;
static_assert(3 * NL + 4 * FH + 16 * CL <= 128 && 4 * NH + 3 * FL + 16 * CH <= 128, "roles");
constexpr int SOLOG = 16, NRAW = 52;
constexpr int LEAD = 44;
static_assert(NRAW <= 64 && NRAW % 4 == 0 && 8 * SOLOG <= NWG - NPULL && SOLOG == 16, "solo chunks");
constexpr int NENC = NWG - NPULL;
constexpr int GX = 0, GW = 32, GY = 288;
constexpr int FX = 2, PK = DP + 8 * FX, LC = LP - FX;
constexpr int NGID = GY + 16 * PK, GM = NGID, NGT = GM + 16;
constexpr int NS = 32 + 2 * LC + 16 * DP + 16 * FX;
constexpr int ROUNDS = (LC + CPR - 1) / CPR;
constexpr int NUMAX = 704;
constexpr int NSTAT = 3;
constexpr int NPACK = 288 - SOLOG;
constexpr u64 DEADLINE = 30000000ull;

constexpr size_t WS_CALL = 0;
constexpr size_t WS_TMO = 32768;
constexpr size_t WS_CNT = 36864;
constexpr size_t WS_CLAIM = 57344;
constexpr size_t WS_FLAG = 65536;
constexpr size_t FLAG_SRC = NGT * 32, FLAG_PAR = 8 * FLAG_SRC;
constexpr size_t GRP_STRIDE = 131072;
constexpr size_t WS_STG = 2u << 20;
constexpr size_t STG_PAR = (NGID * GRP_STRIDE + 0x1fffff) & ~(size_t)0x1fffff;
constexpr size_t WS_INB = WS_STG + 2 * STG_PAR;
constexpr size_t WS_XI = WS_INB + 2 * 8 * 64 * 8192;
constexpr size_t PANEL = 64 * 32768;
constexpr size_t WS_WI = WS_XI + 16 * PANEL;
constexpr size_t WS_END = WS_WI + (size_t)LP * PANEL;
static_assert(WS_FLAG + 2 * FLAG_PAR <= WS_STG && NGT <= 1024 && RP >= 8 * FX, "layout");

constexpr u32 LDS_CTL = 131072;
constexpr u32 LDS_LUT = LDS_CTL + 512, LDS_HIST = LDS_CTL + 1024;
constexpr u32 LDS_TAB = LDS_HIST + 4096, D_DESC = LDS_TAB + 3072, LDS_BYTES = D_DESC + NS * 64;
constexpr u32 CTL_CALL = LDS_CTL  , CTL_SIZE = LDS_CTL + 64  , CTL_BAD = LDS_CTL + 128;
constexpr u32 CTL_NUNIT = LDS_CTL + 192, CTL_LANDED = LDS_CTL + 256, CTL_NCHUNK = LDS_CTL + 320, CTL_NEXT = LDS_CTL + 384  ;
constexpr u32 E_L2 = 0, E_L3 = E_L2 + 4160, E_L4 = E_L3 + 2112, E_VAR = E_L4 + 1088, E_END = E_VAR + 2112;
static_assert(E_END <= 10240 && E_L4 % 16 == 0 && E_L3 % 16 == 0 && E_VAR % 16 == 0, "packer scratch");
constexpr u32 CAP2 = 4080, CAP3 = 2032, CAP4 = 1008, CAPV = 2032, FIXB = 10240, RAWB = 16384, RAWCODE = 255;
constexpr u32 RING = 65536, RMASK = RING - 1, ND = 6;
constexpr u32 D_SCR = RING, D_SCRB = 7360, D_L2 = 0, D_L3 = 4160, D_L4 = D_L3 + 2112;
constexpr u32 D_UNIT = D_SCR + ND * D_SCRB, DONE = 0x80000000u;
static_assert(D_L4 + 1088 <= D_SCRB && D_UNIT + (2 * NUMAX + 8 * SOLOG) * 8 <= LDS_CTL - 2048 && LDS_BYTES <= 163840 && D_DESC % 128 == 0 && NS * 64 >= 16384, "puller LDS");

struct Args {
    char* ws[NR];
    char* me; const u16* x; const u16* w; u16* out;
    int rank, nunits;
    u16 role[NWG];
    u16 stream[NUMAX];
    unsigned char cols[128];
    u16 enc[2 * NENC];
    u16 pad[(3072 - 104 - 128) / 2 - NWG - NUMAX - 2 * NENC];
};
static_assert(sizeof(Args) == 3072 && ROUNDS < 8 && offsetof(Args, cols) % 8 == 0 && offsetof(Args, role) % 4 == 0, "kernarg");
constexpr u32 TAB_STREAM = LDS_TAB + offsetof(Args, stream), TAB_ENC = LDS_TAB + offsetof(Args, enc), TAB_COLS = LDS_TAB + offsetof(Args, cols);
struct Params { char* me; char* peer; const u16* x; const u16* w; u16* out; u32 rank, nunits; };
template <class T> __device__ __forceinline__ T* pin(T* p) { u64 v = (u64)p; asm volatile("" : "+s"(v)); return (T*)(GAS T*)v; }
__device__ __forceinline__ u32 pin(u32 v) { asm volatile("" : "+s"(v)); return v; }

__device__ __forceinline__ u32 lds32(u32 a) { return *(LAS u32*)a; }
__device__ __forceinline__ u32 lds32v(u32 a) { return *(volatile LAS u32*)a; }
__device__ __forceinline__ void lds32vw(u32 a, u32 x) { *(volatile LAS u32*)a = x; }
__device__ __forceinline__ u32x2 lds64v(u32 a) { return *(volatile LAS u32x2*)a; }
__device__ __forceinline__ void lds64vw(u32 a, u32x2 x) { *(volatile LAS u32x2*)a = x; }
__device__ __forceinline__ u32x4 lds128(u32 a) { return *(LAS u32x4*)a; }
__device__ __forceinline__ void lds128w(u32 a, u32x4 x) { *(LAS u32x4*)a = x; }
__device__ __forceinline__ u32x4 zero4() { u32 z; asm volatile("v_mov_b32 %0, 0" : "=v"(z)); return (u32x4){z, z, z, z}; }
__device__ __forceinline__ void lds_or(u32 a, u32 x) { __hip_atomic_fetch_or((LAS u32*)a, x, __ATOMIC_RELAXED, __HIP_MEMORY_SCOPE_WORKGROUP); }
__device__ __forceinline__ void lds_add(u32 a, u32 x) { __hip_atomic_fetch_add((LAS u32*)a, x, __ATOMIC_RELAXED, __HIP_MEMORY_SCOPE_WORKGROUP); }
__device__ __forceinline__ u32 rfl(u32 x) { return (u32)__builtin_amdgcn_readfirstlane((int)x); }
__device__ __forceinline__ u32 tab16(u32 t, u32 i) { return rfl(*(volatile LAS u16*)(t + i * 2)); }
#define CBAR() asm volatile("" ::: "memory")
#define WAIT_LGKM0() asm volatile("s_waitcnt lgkmcnt(0)" ::: "memory")
#define WAIT_VM(n) asm volatile("s_waitcnt vmcnt(%0)" :: "i"(n) : "memory")
#define BAR() do { CBAR(); __builtin_amdgcn_s_barrier(); CBAR(); } while (0)
__device__ __forceinline__ void st_sys(void* p, u32x4 x) { asm volatile("global_store_dwordx4 %0, %1, off sc0 sc1\n\ts_nop 1" :: "v"(p), "v"(x) : "memory"); }
__device__ __forceinline__ void st_dev(void* p, u32x4 x) { asm volatile("global_store_dwordx4 %0, %1, off sc1\n\ts_nop 1" :: "v"(p), "v"(x) : "memory"); }
__device__ __forceinline__ void st_out(void* p, u32x4 x) { asm volatile("global_store_dwordx4 %0, %1, off sc1 nt\n\ts_nop 1" :: "v"(p), "v"(x) : "memory"); }
__device__ __forceinline__ void glds_plain(const void* g, u32 dst) {
    u32 keep;
    asm volatile("s_mov_b32 %0, m0\n\ts_mov_b32 m0, %2\n\ts_nop 0\n\tglobal_load_lds_dwordx4 %1, off\n\ts_mov_b32 m0, %0" : "=&s"(keep) : "v"(g), "s"(dst) : "memory");
}
__device__ __forceinline__ void glds_w(const void* g, u32 dst) {
    u32 keep;
    asm volatile("s_mov_b32 %0, m0\n\ts_mov_b32 m0, %2\n\ts_nop 0\n\tglobal_load_lds_dwordx4 %1, off nt\n\ts_mov_b32 m0, %0" : "=&s"(keep) : "v"(g), "s"(dst) : "memory");
}
__device__ __forceinline__ u64 now() { return __builtin_amdgcn_s_memrealtime(); }
template <int CTRL, int RM, bool BC> __device__ __forceinline__ u32 dpp(u32 v) { return (u32)__builtin_amdgcn_update_dpp(0, (int)v, CTRL, RM, 0xf, BC); }
__device__ __forceinline__ u32 scan_incl(u32 v) {
    v += dpp<0x111, 0xf, true>(v); v += dpp<0x112, 0xf, true>(v); v += dpp<0x114, 0xf, true>(v); v += dpp<0x118, 0xf, true>(v);
    v += dpp<0x142, 0xa, false>(v); v += dpp<0x143, 0xc, false>(v); return v;
}
__device__ __forceinline__ u32 umax(u32 a, u32 b) { return a > b ? a : b; }
__device__ __forceinline__ u32 umin(u32 a, u32 b) { return a < b ? a : b; }
__device__ __forceinline__ u32 popc(u32 x) { return (u32)__builtin_popcount(x); }
__device__ __forceinline__ u32 perm(u32 hi, u32 lo, u32 sel) { return __builtin_amdgcn_perm(hi, lo, sel); }


__device__ __forceinline__ u32 nzb(u32 z) { return ((z & 0x7f7f7f7fu) + 0x7f7f7f7fu) | z; }
typedef unsigned short u16x2 __attribute__((ext_vector_type(2)));
struct Classifier {
    u32 s1, s2, s3, nb, tlo, thi; bool win;
    __device__ __forceinline__ void init(u32 t) {
        const u32 a = (t >> 8) & 255u, b = (t >> 16) & 255u, c = t >> 24, lo = rfl(umin(a, umin(b, c)));
        win = umax(a, umax(b, c)) - lo <= 6u;
        const u64 l = (1ull << (((a - lo) & 7u) * 8)) | (2ull << (((b - lo) & 7u) * 8)) | (3ull << (((c - lo) & 7u) * 8));
        tlo = (u32)l; thi = (u32)(l >> 32); nb = ((0u - lo) & 255u) * 0x01000100u;
        s1 = a * 0x01010101u; s2 = b * 0x01010101u; s3 = c * 0x01010101u;
    }
    __device__ __forceinline__ u32 any(u32 e) const {
        const u32 a1 = nzb(e ^ s1), a2 = nzb(e ^ s2), a3 = nzb(e ^ s3);
        return ((~(a1 & a3) >> 7) & 0x01010101u) | ((~(a2 & a3) >> 6) & 0x02020202u);
    }
    __device__ __forceinline__ u32 near(u32 e) const {
        const u16x2 x = __builtin_bit_cast(u16x2, e), n = __builtin_bit_cast(u16x2, nb), top = {0x07ff, 0x07ff};
        u32 sh; asm("v_pk_mad_u16 %0, %1, %2, %3" : "=v"(sh) : "v"(e), "v"(0x01000100u), "s"(nb));
        const u16x2 hi = __builtin_elementwise_min((u16x2)(x + n), top), lo = __builtin_elementwise_min(__builtin_bit_cast(u16x2, sh), top);
        return perm(thi, tlo, perm(__builtin_bit_cast(u32, hi), __builtin_bit_cast(u32, lo), 0x07030501u));
    }
    template <int N> __device__ __forceinline__ void run(const u32* e, u32* c) const {
        if (win) {
#pragma unroll
            for (int i = 0; i < N; ++i) c[i] = near(e[i]);
        } else {
#pragma unroll
            for (int i = 0; i < N; ++i) c[i] = any(e[i]);
        }
    }
};
__device__ __forceinline__ u32 escbits(u32 c) { return ~(c | (c >> 1)) & 0x55555555u; }
struct Appender {
    u32 lo, fill, addr;
    __device__ __forceinline__ void init(u32 off) { lo = 0; fill = (off & 3u) * 8u; addr = off & ~3u; }
    __device__ __forceinline__ void push(u32 k, u32 mk) {
        lo |= k << fill;
        const u32 nf = fill + popc(mk);
        if (nf >= 32u) { lds_or(addr, lo); lo = __builtin_amdgcn_ubfe(k, 0u - fill, fill); addr += 4; }
        fill = nf & 31u;
    }
    __device__ __forceinline__ void flush() { if (lo) lds_or(addr, lo); }
};
__device__ __forceinline__ u32 escmask(u32 c) { return perm(0u, 0xffu, c); }
__device__ __forceinline__ u32 selector(u32 mk) { return lds32(__builtin_amdgcn_udot4(mk & 0x20100804u, 0x01010101u, LDS_LUT, false)); }
template <int R> __device__ __forceinline__ u32 pack_level(u32 src, u32 n, u32 t, u32 cdst, u32 edst, u32 cap, u32 lane) {
    Classifier K; K.init(t);
    if (lane < 16) *(LAS unsigned char*)(src + n + lane) = (unsigned char)K.s1;
    u32 ne = 0;
    for (u32 r = 0; r * 1024 < n; r += R) {
        CBAR();
        u32 q[4 * R], c[4 * R], m[4 * R], l[4 * R], cnt = 0;
#pragma unroll
        for (int i = 0; i < R; ++i) { const u32 p0 = (r + i) * 1024 + lane * 16; const u32x4 x = lds128(src + p0); const bool act = p0 < n;
            q[4 * i] = act ? x.x : K.s1; q[4 * i + 1] = act ? x.y : K.s1; q[4 * i + 2] = act ? x.z : K.s1; q[4 * i + 3] = act ? x.w : K.s1; }
        K.run<4 * R>(q, c);
#pragma unroll
        for (int i = 0; i < 4 * R; ++i) { m[i] = escmask(c[i]); l[i] = selector(m[i]); }
#pragma unroll
        for (int i = 0; i < R; ++i) cnt |= ((popc(m[4 * i]) + popc(m[4 * i + 1]) + popc(m[4 * i + 2]) + popc(m[4 * i + 3])) >> 3) << (16 * i);
        const u32 inc = scan_incl(cnt), tot = (u32)__builtin_amdgcn_readlane((int)inc, 63), exc = inc - cnt;
        if (ne + (tot & 0xffffu) + (tot >> 16) > cap) return ~0u;
#pragma unroll
        for (int i = 0; i < R; ++i) {
            Appender A; A.init(edst + ne + (i ? (tot & 0xffffu) + (exc >> 16) : (exc & 0xffffu)));
#pragma unroll
            for (int j = 0; j < 4; ++j) A.push(perm(0u, q[4 * i + j], l[4 * i + j]), m[4 * i + j]);
            A.flush();
            if ((r + i) * 1024 + lane * 16 < n) *(LAS u32*)(cdst + ((r + i) * 64 + lane) * 4) = c[4 * i] | (c[4 * i + 1] << 2) | (c[4 * i + 2] << 4) | (c[4 * i + 3] << 6);
        }
        ne += (tot & 0xffffu) + (tot >> 16);
    }
    CBAR();
    return ne;
}
template <int N> __device__ __forceinline__ u32 rank_ror(u32 key) { return (dpp<0x120 + N, 0xf, true>(key) > key ? 1u : 0u) + rank_ror<N - 1>(key); }
template <> __device__ __forceinline__ u32 rank_ror<0>(u32) { return 0u; }
__device__ __forceinline__ u32 row_or(u32 x) { x |= dpp<0x128, 0xf, true>(x); x |= dpp<0x124, 0xf, true>(x); x |= dpp<0x122, 0xf, true>(x); x |= dpp<0x121, 0xf, true>(x); return rfl(x); }
__device__ __forceinline__ void split(const u32 (&v)[64], u32 (&Mr)[32], u32 (&Er)[32]) {
#pragma unroll
    for (int g = 0; g < 32; ++g) {
        const u32 a = __builtin_amdgcn_alignbit(v[2 * g], v[2 * g], 31), b = __builtin_amdgcn_alignbit(v[2 * g + 1], v[2 * g + 1], 31);
        Er[g] = perm(b, a, 0x07050301u); Mr[g] = perm(b, a, 0x06040200u);
        asm volatile("" : "+v"(Er[g]), "+v"(Mr[g]));
    }
}
__device__ __forceinline__ u32x4 piece(const u32 (&Mr)[32], const u32 (&Er)[32], int p) {
    const u32 a = perm(Er[2 * p], Mr[2 * p], 0x05010400u), b = perm(Er[2 * p], Mr[2 * p], 0x07030602u);
    const u32 c = perm(Er[2 * p + 1], Mr[2 * p + 1], 0x05010400u), d = perm(Er[2 * p + 1], Mr[2 * p + 1], 0x07030602u);
    return (u32x4){__builtin_amdgcn_alignbit(a, a, 1), __builtin_amdgcn_alignbit(b, b, 1), __builtin_amdgcn_alignbit(c, c, 1), __builtin_amdgcn_alignbit(d, d, 1)};
}
template <int R> __device__ __forceinline__ u32 pack_short(u32 src, u32 n, u32 t, u32 cdst, u32 edst, u32 cap, u32 lane) {
    Classifier K; K.init(t);
    if (lane < 16) *(LAS unsigned char*)(src + n + lane) = (unsigned char)K.s1;
    u32 ne = 0;
    for (u32 r = 0; r * 256 < n; r += R) {
        CBAR();
        u32 q[R], c[R], m[R], l[R], cnt = 0;
#pragma unroll
        for (int i = 0; i < R; ++i) { const u32 p0 = (r + i) * 256 + lane * 4, x = lds32(src + p0); q[i] = p0 < n ? x : K.s1; }
        K.run<R>(q, c);
#pragma unroll
        for (int i = 0; i < R; ++i) { m[i] = escmask(c[i]); l[i] = selector(m[i]); cnt |= (popc(m[i]) >> 3) << (16 * i); }
        const u32 inc = scan_incl(cnt), tot = (u32)__builtin_amdgcn_readlane((int)inc, 63), exc = inc - cnt;
        if (ne + (tot & 0xffffu) + (tot >> 16) > cap) return ~0u;
#pragma unroll
        for (int i = 0; i < R; ++i) {
            Appender A; A.init(edst + ne + (i ? (tot & 0xffffu) + (exc >> 16) : (exc & 0xffffu)));
            A.push(perm(0u, q[i], l[i]), m[i]); A.flush();
            u32 x = c[i] << (2 * (lane & 3)); x |= dpp<0xb1, 0xf, true>(x); x |= dpp<0x4e, 0xf, true>(x);
            if ((lane & 3) == 0 && (r + i) * 256 + lane * 4 < n) *(LAS u32*)(cdst + (r + i) * 64 + lane) = x;
        }
        ne += (tot & 0xffffu) + (tot >> 16);
    }
    CBAR();
    return ne;
}
template <int N, int STEP, int OFF> __device__ __forceinline__ void sample(const u32 (&v)[64], u32 hist, u32 lane) {
#pragma unroll
    for (int i = 0; i < N; ++i) lds_add(hist + (((v[i * STEP + OFF] >> ((i & 1) * 16 + 3)) & 0xff0u) | ((lane & 3) * 4)), 1u);
}
template <bool SAMPLED, class Side>
__device__ __forceinline__ u32 pack_chunk(const u32 (&v)[64], u32 (&Mr)[32], u32 (&Er)[32], u32 (&Cr)[8], u32 scr, u32 lane, bool solo, const Side& side) {
    CBAR();
#define SIDE4(p0) do { if (side.on && !solo) { _Pragma("unroll") for (int p = p0; p < p0 + 4; ++p) side(p, piece(Mr, Er, p)); } } while (0)
    if (side.on && !solo) {
#pragma unroll
        for (int p = 0; p < 4; ++p) side(p, (u32x4){v[4 * p], v[4 * p + 1], v[4 * p + 2], v[4 * p + 3]});
    }
    const u32 hist = solo ? scr + 10240 : LDS_HIST; const u32x4 z4 = zero4();
    if (solo) {
#pragma unroll
        for (int j = 0; j < 4; ++j) lds128w(hist + j * 1024 + lane * 16, z4);
        CBAR();
        sample<12, 5, 2>(v, hist, lane);
    }
    if (!SAMPLED) sample<6, 10, 5>(v, hist, lane);
#pragma unroll
    for (int j = 0; j < 10; ++j) lds128w(scr + j * 1024 + lane * 16, z4);
    if (!SAMPLED) { WAIT_LGKM0(); if (!solo) BAR(); }
    CBAR();
    u32 key = 0;
#pragma unroll
    for (int i = 0; i < 4; ++i) { const u32 bin = i * 64 + lane; const u32x4 h = lds128(hist + bin * 16); key = umax(key, ((h.x + h.y + h.z + h.w) << 8) | bin); }
    split(v, Mr, Er);
    { const auto r = __builtin_amdgcn_permlane32_swap(key, key, false, false); key = umax(r[0], r[1]); }
    { const auto r = __builtin_amdgcn_permlane16_swap(key, key, false, false); key = umax(r[0], r[1]); }
    const u32 rank = rank_ror<15>(key), lev = (rank * 11u) >> 5, sym = (key & 255u) << (8 * (rank - 3 * lev + 1));
    const u32 T[4] = {row_or(lev == 0 ? sym : 0u), row_or(lev == 1 ? sym : 0u), row_or(lev == 2 ? sym : 0u), row_or(lev == 3 ? sym : 0u)};
    SIDE4(4);
    u32 cnt = 0, mk[32], sl[32];
    {
        Classifier K; K.init(T[0]);
        u32 c[32]; K.run<32>(Er, c);
#pragma unroll
        for (int g = 0; g < 32; ++g) { mk[g] = escmask(c[g]); sl[g] = selector(mk[g]); cnt += popc(mk[g]); }
#pragma unroll
        for (int k = 0; k < 8; ++k) Cr[k] = c[4 * k] | (c[4 * k + 1] << 2) | (c[4 * k + 2] << 4) | (c[4 * k + 3] << 6);
    }
    cnt >>= 3;
    const u32 inc = scan_incl(cnt), n2 = (u32)__builtin_amdgcn_readlane((int)inc, 63);
    SIDE4(8);
    if (n2 <= CAP2) {
        Appender A; A.init(scr + E_L2 + inc - cnt);
#pragma unroll
        for (int g = 0; g < 32; ++g) A.push(perm(0u, Er[g], sl[g]), mk[g]);
        A.flush();
    }
    SIDE4(12);
    if (n2 > CAP2) return RAWCODE;
    CBAR();
    const u32 d2 = (n2 + 15) >> 4;
    const u32 n3 = pack_level<2>(scr + E_L2, n2, T[1], scr + E_VAR + 16, scr + E_L3, CAP3, lane);
    if (n3 == ~0u) return RAWCODE;
    const u32 d3 = (n3 + 15) >> 4;
    const u32 n4 = pack_short<2>(scr + E_L3, n3, T[2], scr + E_VAR + 16 + 4 * d2, scr + E_L4, CAP4, lane);
    if (n4 == ~0u) return RAWCODE;
    const u32 d4 = (n4 + 15) >> 4, b5 = 16 + 4 * (d2 + d3 + d4);
    const u32 n5 = pack_short<1>(scr + E_L4, n4, T[3], scr + E_VAR + 16 + 4 * (d2 + d3), scr + E_VAR + b5, CAPV - b5, lane);
    if (n5 == ~0u) return RAWCODE;
    if (lane == 0) lds128w(scr + E_VAR, (u32x4){T[0] | d2, T[1] | d3, T[2] | d4, T[3]});
    CBAR();
    return (b5 + n5 + 15) >> 4;
}
__device__ __forceinline__ void pack_init(u32 wid, u32 lane) {
    if (wid >= 4) lds128w(LDS_HIST + (wid - 4) * 1024 + lane * 16, zero4());
    if ((wid == 1 || wid == 2) && lane < 16) { u32 sel = 0x0c0c0c0cu, k = 0;
#pragma unroll
        for (u32 b = 0; b < 4; ++b) if ((lane >> b) & 1) { sel = (sel & ~(255u << k)) | (b << k); k += 8; }
        *(LAS u32*)(LDS_LUT + lane * 4) = sel; }
}
__device__ __forceinline__ u32 chunk_bytes(u32 code) { return code == RAWCODE ? RAWB : FIXB + 16 * code; }

__device__ __forceinline__ u32 blk0(u32 a, u32 len, u32 lane) { return (a & ~127u) + lane * 16 + (lane * 16 < (a & 127u) ? len : 0u); }
__device__ __forceinline__ void pack_store(char* grp, u32 off, u32 code, const u32 (&Mr)[32], const u32 (&Er)[32], const u32 (&Cr)[8], u32 scr, u32 lane) {
    const u32 om = off + chunk_bytes(code) - 8192;
    char* dm = grp + (om & ~127u) + lane * 16; char* de = grp + (off & ~127u) + lane * 16;
    st_sys(grp + blk0(om, 8192, lane), (u32x4){Mr[0], Mr[1], Mr[2], Mr[3]});
#pragma unroll
    for (int j = 1; j < 8; ++j) st_sys(dm + j * 1024, (u32x4){Mr[4 * j], Mr[4 * j + 1], Mr[4 * j + 2], Mr[4 * j + 3]});
    if (code == RAWCODE) {
        st_sys(grp + blk0(off, 8192, lane), (u32x4){Er[0], Er[1], Er[2], Er[3]});
#pragma unroll
        for (int j = 1; j < 8; ++j) st_sys(de + j * 1024, (u32x4){Er[4 * j], Er[4 * j + 1], Er[4 * j + 2], Er[4 * j + 3]});
    } else {
        st_sys(grp + blk0(off, 2048, lane), (u32x4){Cr[0], Cr[1], Cr[2], Cr[3]}); st_sys(de + 1024, (u32x4){Cr[4], Cr[5], Cr[6], Cr[7]});
        char* dv = grp + off + 2048 + lane * 16;
        if (lane < code) st_sys(dv, lds128(scr + E_VAR + lane * 16));
        if (lane + 64 < code) st_sys(dv + 1024, lds128(scr + E_VAR + 1024 + lane * 16));
    }
}
__device__ __forceinline__ void raise_flags(u32* fp, u32 fv) { if (fp) __hip_atomic_store((u32*)(GAS u32*)fp, fv, __ATOMIC_RELAXED, __HIP_MEMORY_SCOPE_SYSTEM); }
template <bool SAMPLED, class Side>
__device__ __forceinline__ void pack_group(const Params& P, const u32 (&v)[64], u32 gid, u32 w, bool solo, u32 peers, u32 call, u32 wid, u32 lane, const Side& side, u32*& fp, u32& fv) {
    u32 Mr[32], Er[32], Cr[8];
    const u32 scr = wid * 16384;
    const u32 code = rfl(pack_chunk<SAMPLED>(v, Mr, Er, Cr, scr, lane, solo, side));
    u32 off = w * RAWB, mine = code;
    if (!solo) {
        if (lane == 0) lds32vw(CTL_SIZE + wid * 4, code | gid << 8);
        for (;;) { mine = lds32v(CTL_SIZE + (lane & 7) * 4); if (__all((mine >> 8) == gid || (lane & 7) > wid)) break; __builtin_amdgcn_s_sleep(1); }
        mine &= 255u; off = 0;
#pragma unroll
        for (u32 j = 0; j < 7; ++j) { const u32 c = (u32)__builtin_amdgcn_readlane((int)mine, j); if (j < wid) off += chunk_bytes(c); }
        if (wid == 7) {
            const u32x4 z4 = zero4();
#pragma unroll
            for (int j = 0; j < 4; ++j) lds128w(LDS_HIST + j * 1024 + lane * 16, z4);
        }
    }
    char* wq = (char*)(GAS char*)__builtin_bit_cast(u64, lds64v(LDS_TAB + (lane >> 3) * 8));
    pack_store(P.me + WS_STG + (call & 1) * STG_PAR + (size_t)gid * GRP_STRIDE, off, code, Mr, Er, Cr, scr, lane);
    fp = (solo ? (lane & 7) == w : wid == 7) && ((peers >> (lane >> 3)) & 1) ? (u32*)(wq + WS_FLAG + (call & 1) * FLAG_PAR + (size_t)P.rank * FLAG_SRC + gid * 32) + (lane & 7) : nullptr;
    fv = (call << 8) | mine;
    WAIT_LGKM0();
}

template <bool WRAP> struct Unp {
    u32 sel[4], lo[4], hi[4], of[4];
    __device__ __forceinline__ void fetch(u32 c, u32& off) {
#pragma unroll
        for (int j = 0; j < 4; ++j) {
            const u32 cj = (c >> (2 * j)) & 0x03030303u, mk = escmask(cj), m2 = mk & 0x02020202u;
            const u32 a = off & ~3u; lo[j] = lds32(WRAP ? (a & RMASK) : a); hi[j] = lds32(WRAP ? ((a + 4) & RMASK) : a + 4); of[j] = off;
            u32 p; asm("v_mad_u32_u24 %0, %1, %2, %3" : "=v"(p) : "v"(m2), "s"(0x808080u), "v"(0x04040404u));
            sel[j] = (mk & p) | cj;
            off += popc(m2);
        }
    }
    __device__ __forceinline__ u32 finish(int j, u32 t) const {
        const u32 w = __builtin_amdgcn_alignbyte(hi[j], lo[j], WRAP ? of[j] & 3u : of[j]);
        return perm(w, t, sel[j]);
    }
    __device__ __forceinline__ u32x4 finish4(u32 t) const { return (u32x4){finish(0, t), finish(1, t), finish(2, t), finish(3, t)}; }
};
#define SCHED() __builtin_amdgcn_sched_barrier(0)
template <bool WRAP> __device__ __forceinline__ u32 unpack_round(u32 c, u32 t, u32 src, u32 dst) {
    const u32 cnt = popc(escbits(c)), inc = scan_incl(cnt);
    u32 off = src + inc - cnt;
    Unp<WRAP> u; u.fetch(c, off); SCHED();
    lds128w(dst, u.finish4(t));
    return (u32)__builtin_amdgcn_readlane((int)inc, 63);
}
__device__ __forceinline__ void ring_read8k(u32 a, u32 lane, u32 (&r)[32]) {
    const u32 a0 = (a & ~127u) + lane * 16;
#pragma unroll
    for (int j = 0; j < 8; ++j) { const u32x4 q = lds128((j ? a0 + j * 1024 : blk0(a, 8192, lane)) & RMASK); r[4 * j] = q.x; r[4 * j + 1] = q.y; r[4 * j + 2] = q.z; r[4 * j + 3] = q.w; }
}
__device__ __forceinline__ void unpack_exp(u32 S, u32 code, u32 scr, u32 lane, u32 (&Er)[32]) {
    if (code == RAWCODE) { ring_read8k(S, lane, Er); return; }
    const u32x4 ca = lds128(blk0(S, 2048, lane) & RMASK), cb = lds128(((S & ~127u) + lane * 16 + 1024) & RMASK);
    const u32 V = S + 2048;
    const u32x4 hd = lds128(V & RMASK);
    const u32 t1 = rfl(hd.x), t2 = rfl(hd.y), t3 = rfl(hd.z), t4 = rfl(hd.w), d2 = t1 & 255u, d3 = t2 & 127u, d4 = t3 & 63u;
    u32 c2[4], c3[2];
#pragma unroll
    for (int r = 0; r < 4; ++r) { const u32 i = r * 64 + lane; const u32 x = lds32((V + 16 + 4 * i) & RMASK); c2[r] = i < d2 ? x : 0x55555555u; }
#pragma unroll
    for (int r = 0; r < 2; ++r) { const u32 i = r * 64 + lane; const u32 x = lds32((V + 16 + 4 * (d2 + i)) & RMASK); c3[r] = i < d3 ? x : 0x55555555u; }
    const u32 x4 = lds32((V + 16 + 4 * (d2 + d3 + lane)) & RMASK), c4 = lane < d4 ? x4 : 0x55555555u;
    unpack_round<true>(c4, t4, V + 16 + 4 * (d2 + d3 + d4), scr + D_L4 + lane * 16);
    CBAR();
    const u32 s4 = scr + D_L4 + unpack_round<false>(c3[0], t3, scr + D_L4, scr + D_L3 + lane * 16);
    if (d3 > 64) unpack_round<false>(c3[1], t3, s4, scr + D_L3 + 1024 + lane * 16);
    CBAR();
    const u32 C[8] = {ca.x, ca.y, ca.z, ca.w, cb.x, cb.y, cb.z, cb.w};
    u32 cnt = 0;
#pragma unroll
    for (int k = 0; k < 8; ++k) cnt += popc(escbits(C[k]));
    {
        const u32 n0 = popc(escbits(c2[0])), n1 = popc(escbits(c2[1]));
        const u32 i01 = scan_incl(n0 | (n1 << 16)), tot = (u32)__builtin_amdgcn_readlane((int)i01, 63);
        cnt = scan_incl(cnt) - cnt;
        u32 o0 = scr + D_L3 + (i01 & 0xffffu) - n0, o1 = scr + D_L3 + (tot & 0xffffu) + (i01 >> 16) - n1;
        Unp<false> u0, u1; u0.fetch(c2[0], o0); u1.fetch(c2[1], o1); SCHED();
        lds128w(scr + D_L2 + lane * 16, u0.finish4(t2)); lds128w(scr + D_L2 + 1024 + lane * 16, u1.finish4(t2));
        if (d2 > 128) {
            u32 s3 = scr + D_L3 + (tot & 0xffffu) + (tot >> 16);
            s3 += unpack_round<false>(c2[2], t2, s3, scr + D_L2 + 2048 + lane * 16);
            if (d2 > 192) unpack_round<false>(c2[3], t2, s3, scr + D_L2 + 3072 + lane * 16);
        }
    }
    CBAR();
    u32 off = scr + D_L2 + cnt;
    Unp<false> u[8];
    u[0].fetch(C[0], off); u[1].fetch(C[1], off);
#pragma unroll
    for (int k = 0; k < 8; ++k) {
        SCHED();
        if (k + 2 < 8) u[k + 2].fetch(C[k + 2], off);
        SCHED();
#pragma unroll
        for (int j = 0; j < 4; ++j) Er[4 * k + j] = u[k].finish(j, t1);
    }
    CBAR();
}
__device__ __forceinline__ u32 x_piece_off(u32 half, int p) { return half * 16384 + p * 1024; }
__device__ __forceinline__ u32 w_piece_off(u32 h, int p) {
    const u32 g32 = p >> 2, n = (p >> 1) & 1, kb = p & 1;
    return (g32 & 1) * 16384 + ((2 * (2 * h + (g32 >> 1)) + n) * 2 + kb) * 1024;
}
__device__ __forceinline__ u16* y_base(const Params& P, u32 pm, u32 j, u32 w, u32 lane) {
    return P.out + (size_t)(pm * 256 + (w >> 2) * 64 + (lane >> 3)) * NN + j * 256 + (w & 3) * 64 + (lane & 7) * 8;
}
__device__ __forceinline__ size_t y_piece_off(int p) { return (size_t)((p >> 3) * 128 + (p & 7) * 8) * NN; }
__device__ __forceinline__ u32 local_panel(u32 j, u32 rank) { return j >= (u32)(NR * DP) ? j - (NR - 1) * DP : j - rank * DP; }
__device__ __forceinline__ u32 global_panel(u32 jj, u32 rank) { return jj >= (u32)DP ? jj + (NR - 1) * DP : jj + rank * DP; }
__device__ __forceinline__ u32* counter(const Params& P, u32 i) { return (u32*)(P.me + WS_CNT + i * 128); }
__device__ __forceinline__ u32* tmo_word(const Params& P) { return (u32*)(P.me + WS_TMO); }
__device__ __forceinline__ bool expired(const Params& P, u64 t0, u32 call) {
    if (__hip_atomic_load(tmo_word(P), __ATOMIC_RELAXED, __HIP_MEMORY_SCOPE_AGENT) == call || now() - t0 > DEADLINE) {
        __hip_atomic_store(tmo_word(P), call, __ATOMIC_RELAXED, __HIP_MEMORY_SCOPE_AGENT); return true; }
    return false;
}

struct ImageStore {
    char* o; u32* cnt; u32 half; bool isx, on;
    __device__ __forceinline__ ImageStore(const Params& P, u32 gid, u32 w, u32 q, u32 lane, bool on_) {
        isx = gid < (u32)GW; half = w & 1; on = on_;
        const u32 pn = isx ? gid & 15 : local_panel((gid - GW) >> 1, P.rank), ktl = (isx ? gid >> 4 : gid & 1) * 4 + (w >> 1);
        const u32 qi = (q - P.rank - 1) & 7, stage = qi == 7 ? 0u : ktl == 7 ? 5u : 1 + (ktl >> 1);
        const u32 slot = stage == 0 ? ktl : 8 + ktl * 7 + qi;
        o = P.me + (isx ? WS_XI : WS_WI) + (size_t)pn * PANEL + (size_t)slot * 32768 + lane * 16;
        cnt = counter(P, isx ? pn : 16 + pn) + stage;
    }
    __device__ __forceinline__ void operator()(int p, u32x4 x) const { st_dev(o + (isx ? x_piece_off(half, p) : w_piece_off(half, p)), x); }
    __device__ __forceinline__ void done(u32 lane) const { WAIT_VM(0); if (lane == 0) __hip_atomic_fetch_add(cnt, 1u, __ATOMIC_RELAXED, __HIP_MEMORY_SCOPE_AGENT); }
};
struct OutStore {
    u16* o; static constexpr bool on = true;
    __device__ __forceinline__ void operator()(int p, u32x4 x) const { st_out(o + y_piece_off(p), x); }
};
__device__ __forceinline__ u32* call_word(const Params& P, u32 wid) { return (u32*)(P.me + WS_CALL + blockIdx.x * 128 + wid * 4); }
__device__ __forceinline__ void call_load(const Params& P, u32 wid, u32 lane) {
    u32 keep;
    if (lane == 0) asm volatile("s_mov_b32 %0, m0\n\ts_mov_b32 m0, %2\n\ts_nop 0\n\tglobal_load_lds_dword %1, off sc1\n\ts_mov_b32 m0, %0" : "=&s"(keep) : "v"(call_word(P, wid)), "s"(CTL_CALL + wid * 4) : "memory");
}
__device__ __forceinline__ u32 call_take(u32 wid) { WAIT_VM(0); __builtin_amdgcn_s_waitcnt(0x0f70);   return rfl(lds32v(CTL_CALL + wid * 4)) + 1; }
__device__ __forceinline__ void call_bump(const Params& P, u32 wid, u32 lane, u32 call) { if (lane == 0) __hip_atomic_store(call_word(P, wid), call, __ATOMIC_RELAXED, __HIP_MEMORY_SCOPE_AGENT); }

__device__ __forceinline__ void hold(u32 ticks) { const u64 t0 = now(); while (now() - t0 < ticks) __builtin_amdgcn_s_sleep(2); }
__device__ __forceinline__ void load_tables(u32 lane) {
    const char* ka = (const char*)(const GAS char*)(u64)__builtin_amdgcn_kernarg_segment_ptr() + lane * 16;
    glds_plain(ka, LDS_TAB); glds_plain(ka + 1024, LDS_TAB + 1024); glds_plain(ka + 2048, LDS_TAB + 2048);
}

__device__ __forceinline__ u32 solo_gid(u32 c) { return (c >> 2) & 15; }
__device__ __forceinline__ u32 solo_w(u32 c) { return (c & 3) | (c >> 6) << 2; }
__device__ __forceinline__ void load_x(const Params& P, u32 gid, u32 w, u32 lane, u32 (&v)[64]) {
    const u16* s = P.x + (size_t)((gid & 15) * 256 + (w & 1) * 128 + (lane >> 3)) * KS + ((gid >> 4) * 4 + (w >> 1)) * 64 + ((lane ^ (lane >> 3)) & 7) * 8;
#pragma unroll
    for (int p = 0; p < 16; ++p) { const u32x4 t = *(const u32x4*)(s + p * 8 * KS); v[4 * p] = t.x; v[4 * p + 1] = t.y; v[4 * p + 2] = t.z; v[4 * p + 3] = t.w; }
}
__device__ __forceinline__ void puller(const Params& P, u32 role, u32 wid, u32 lane) {
    const u32 q = role & 7, sub = (role >> 3) & 15, PW = (role >> 7) & 15;
    if (wid >= 2) for (u32 pc = sub + PW * (7 - wid); pc < (u32)NRAW; pc += 6 * PW) {
        const u32 gid = solo_gid(pc), w = solo_w(pc);
        u32 v[64], Mr[32], Er[32];
        load_x(P, gid, w, lane, v);
        const u32 call = call_take(wid);
        split(v, Mr, Er);
        char* o = P.peer + WS_INB + (size_t)(((call & 1) * 8 + P.rank) * 64 + pc) * 8192 + lane * 16;
#pragma unroll
        for (int j = 0; j < 8; ++j) { WAIT_VM(0);   st_sys(o + j * 1024, (u32x4){Mr[4 * j], Mr[4 * j + 1], Mr[4 * j + 2], Mr[4 * j + 3]}); }
        WAIT_VM(0);
        if (lane == 0) __hip_atomic_store((u32*)(P.peer + WS_FLAG + (call & 1) * FLAG_PAR + (size_t)P.rank * FLAG_SRC + (GM + gid) * 32) + w, call << 8, __ATOMIC_RELAXED, __HIP_MEMORY_SCOPE_SYSTEM);
    }
    if (wid == 1) { hold(100); load_tables(lane); }
    const u32 call = call_take(wid);
    call_bump(P, wid, lane, call);
    if (wid == 0) {
        __builtin_amdgcn_s_setprio(3);
        const char* base = P.peer + WS_STG + (call & 1) * STG_PAR;
        const u32 voff = lane * 16;
        u32 pos = 0, pub = 0, limit = RING / 1024;
        for (u32 ui = 0;; ++ui) {
            u32 nu = rfl(lds32v(CTL_NUNIT));
            if ((nu & ~DONE) <= ui) {
                WAIT_VM(0); lds32vw(CTL_LANDED, pos); pub = pos;
                while (nu <= ui) { __builtin_amdgcn_s_sleep(1); nu = rfl(lds32v(CTL_NUNIT)); }
                if ((nu & ~DONE) <= ui) break;
            }
            CBAR();
            const u32x2 un = lds64v(D_UNIT + ui * 8);
            const u32 ut = rfl(un.y), total = ut & 0x3ffffu, lastc = ut >> 18;
            const char* src = base + rfl(un.x);
            for (u32 o = 0; o < total;) {
                if (pos + 4 > limit) {
                    bool drained = false;
                    for (;;) {
                        const u32 nx = lds32v(CTL_NEXT + (lane & 7) * 4);
                        u32 f = lastc;
#pragma unroll
                        for (u32 j = 0; j < ND; ++j) f = umin(f, (u32)__builtin_amdgcn_readlane((int)nx, j));
                        limit = (rfl(lds32v(D_DESC + f * 8)) >> 14) + RING / 1024;
                        if (pos + 4 <= limit) break;
                        if (!drained) { WAIT_VM(0); lds32vw(CTL_LANDED, pos); pub = pos; drained = true; }
                        __builtin_amdgcn_s_sleep(1);
                    }
                }
                const u32 dst = (pos * 1024) & RMASK;
                if (o + 4096 <= total && dst + 4096 <= RING) {
                    u32 keep;
                    asm volatile("s_mov_b32 %0, m0\n\ts_mov_b32 m0, %3\n\ts_nop 4\n\t"
                                 "s_waitcnt vmcnt(%4)\n\tglobal_load_lds_dwordx4 %1, %2 sc0 sc1\n\t"
                                 "s_waitcnt vmcnt(%4)\n\tglobal_load_lds_dwordx4 %1, %2 offset:1024 sc0 sc1\n\t"
                                 "s_waitcnt vmcnt(%4)\n\tglobal_load_lds_dwordx4 %1, %2 offset:2048 sc0 sc1\n\t"
                                 "s_waitcnt vmcnt(%4)\n\tglobal_load_lds_dwordx4 %1, %2 offset:3072 sc0 sc1\n\t"
                                 "s_mov_b32 m0, %0" : "=&s"(keep) : "v"(voff), "s"(src + o), "s"(dst), "i"(WIN) : "memory");
                    pos += 4; o += 4096;
                } else {
                    u32 keep;
                    if (o + voff < total) asm volatile("s_mov_b32 %0, m0\n\ts_mov_b32 m0, %3\n\ts_nop 4\n\ts_waitcnt vmcnt(%4)\n\tglobal_load_lds_dwordx4 %1, %2 sc0 sc1\n\ts_mov_b32 m0, %0"
                                                       : "=&s"(keep) : "v"(voff), "s"(src + o), "s"(dst), "i"(WIN) : "memory");
                    pos += 1; o += 1024;
                }
                if (pos > pub + WIN + 1) { pub = pos - WIN - 1; lds32vw(CTL_LANDED, pub); }
            }
        }
    } else if (wid == 1) {
        u32 basek = 0, nch = 0, nun = 0;
        u32 held = 0, heldw = 0, helds = 0;
        auto release = [&](u32 c) {
            const u32 a0 = held & ~127u, total = held - a0 + 8192;
            if (lane == 0) { lds64vw(D_UNIT + nun * 8, (u32x2){a0, total | (c << 18)}); lds32vw(D_DESC + c * 8 + 4, heldw | ((basek * 1024 + held - a0 - helds) >> 4) << 13); }
            nun += 1; basek += (total + 1023) >> 10; held = 0;
        };
        u32* ticket = (u32*)(P.me + WS_CLAIM + q * 128);
        const u32 nown = P.nunits & 0xffffu, nall = nown + (P.nunits >> 16);
        u32 tbase = NSTAT * PW - (call - 1) * (nown - NSTAT * PW + PW), tend = nown;
        for (u32 ui = 0;; ++ui) {
            while (rfl(lds32v(CTL_LANDED)) + LEAD < basek + (held ? 8u : 0u)) __builtin_amdgcn_s_sleep(4);
            u32 idx = ui * PW + sub;
            if (ui >= (u32)NSTAT) {
                if (lane == 0) idx = __hip_atomic_fetch_add(ticket, 1u, __ATOMIC_RELAXED, __HIP_MEMORY_SCOPE_AGENT);
                idx = rfl(idx) + tbase;
                if (idx >= tend) {
                    if (tend == nall) break;
                    ticket = (u32*)(P.me + WS_CLAIM + P.rank * 128); tbase = nown - (call - 1) * (nall - nown + NPULL); tend = nall;
                    continue;
                }
            }
            const u32 e = tab16(TAB_STREAM, idx), gid = e & 1023u, part = e >> 10, lv = 31 - __builtin_clz(part), n = 8u >> lv, first = (part - (1u << lv)) * n;
            if (nch + n > (u32)(NS * 8)) break;
            const bool solo = gid < (u32)SOLOG, mine = lane >= first && lane < first + n, sent = gid < (u32)(NRAW / 4) && first < 4;
            const u32* f = (const u32*)(P.me + WS_FLAG + (call & 1) * FLAG_PAR + (size_t)q * FLAG_SRC + (gid + (sent && (lane & 8) ? GM : 0)) * 32) + (lane & 7);
            const u64 t0 = now(); u32 val; bool dead = false;
            for (;;) {
                val = __hip_atomic_load(f, __ATOMIC_RELAXED, __HIP_MEMORY_SCOPE_SYSTEM);
                if (__all((val >> 8) == (call & 0xffffffu) || (lane & 7) >= first + n || (solo && (lane & 7) < first))) break;
                if (__any(expired(P, t0, call))) { dead = true; break; }
                __builtin_amdgcn_s_sleep(1);
            }
            if (dead) {
                const u32 t = gid - GY;
                if (lane == 0 && t < (u32)(16 * PK)) *y_base(P, t & 15, global_panel(t >> 4, q), first, 0u) = 0x7fc0;
                break;
            }
            u32 code = val & 255u; if (code > 127u) code = RAWCODE;
            const u32 cb = chunk_bytes(code) - (sent ? 8192u : 0u), sz = solo ? (mine ? (cb + 1023u) & ~1023u : 0u) : (lane < first + n ? cb : 0u);
            u32 inc = sz; inc += dpp<0x111, 0xf, true>(inc); inc += dpp<0x112, 0xf, true>(inc); inc += dpp<0x114, 0xf, true>(inc);
            const bool two = n == 1;
            const u32 a0 = (u32)__builtin_amdgcn_readlane((int)(inc - sz), (int)first) & ~127u, end = (u32)__builtin_amdgcn_readlane((int)inc, 7) - (two ? 8192u : 0u), total = end - a0;
            const u32 S = basek * 1024 + inc - sz - a0;
            if (mine) lds64vw(D_DESC + (nch + lane - first) * 8, (u32x2){(S << 4) | code, (gid * 8 + lane) | (two ? 0u : (chunk_bytes(code) - 8192) >> 4) << 13});
            if (solo) { if (mine) lds64vw(D_UNIT + (nun + lane - first) * 8, (u32x2){gid * (u32)GRP_STRIDE + lane * RAWB, cb | ((nch + lane - first) << 18)}); nun += n; }
            else { if (lane == 0) lds64vw(D_UNIT + nun * 8, (u32x2){gid * (u32)GRP_STRIDE + a0, total | ((nch + n - 1) << 18)}); nun += 1; }
            basek += (total + 1023) >> 10;
            if (held) release(nch - 1);
            if (two) { held = gid * (u32)GRP_STRIDE + end; heldw = gid * 8 + first; helds = (u32)__builtin_amdgcn_readlane((int)S, (int)first); }
            nch += n;
            WAIT_LGKM0();
            lds32vw(CTL_NUNIT, nun); lds32vw(CTL_NCHUNK, nch);
        }
        if (held) { release(nch - 1); WAIT_LGKM0(); }
        lds32vw(CTL_NUNIT, nun | DONE); lds32vw(CTL_NCHUNK, nch | DONE);
    } else {
        const u32 dw = wid - 2, scr = D_SCR + dw * D_SCRB;
        for (u32 c = dw;; c += ND) {
            u32 nc = rfl(lds32v(CTL_NCHUNK));
            while (nc <= c) { __builtin_amdgcn_s_sleep(2); nc = rfl(lds32v(CTL_NCHUNK)); }
            if ((nc & ~DONE) <= c) break;
            CBAR();
            const u32x2 de = lds64v(D_DESC + c * 8);
            const u32 d = rfl(de.x), code = d & 255u, S = (d >> 8) << 4, gid = (rfl(de.y) >> 3) & 1023u, w = rfl(de.y) & 7;
            while (rfl(lds32v(CTL_LANDED)) < (S + chunk_bytes(code) - 8192 + 1023) >> 10) __builtin_amdgcn_s_sleep(1);
            CBAR();
            u32 Mr[32], Er[32];
            unpack_exp(S, code, scr, lane, Er);
            if (gid < (u32)(NRAW / 4) && w < 4) {
                const char* m = P.me + WS_INB + (size_t)(((call & 1) * 8 + q) * 64 + gid * 4 + w) * 8192 + lane * 16;
                u32x4 r[8];
                asm volatile("global_load_dwordx4 %0, %8, off sc0 sc1\n\tglobal_load_dwordx4 %1, %8, off offset:1024 sc0 sc1\n\t"
                             "global_load_dwordx4 %2, %8, off offset:2048 sc0 sc1\n\tglobal_load_dwordx4 %3, %8, off offset:3072 sc0 sc1\n\t"
                             "global_load_dwordx4 %4, %9, off sc0 sc1\n\tglobal_load_dwordx4 %5, %9, off offset:1024 sc0 sc1\n\t"
                             "global_load_dwordx4 %6, %9, off offset:2048 sc0 sc1\n\tglobal_load_dwordx4 %7, %9, off offset:3072 sc0 sc1\n\ts_waitcnt vmcnt(0)"
                             : "=&v"(r[0]), "=&v"(r[1]), "=&v"(r[2]), "=&v"(r[3]), "=&v"(r[4]), "=&v"(r[5]), "=&v"(r[6]), "=&v"(r[7]) : "v"(m), "v"(m + 4096) : "memory");
#pragma unroll
                for (int j = 0; j < 8; ++j) { Mr[4 * j] = r[j].x; Mr[4 * j + 1] = r[j].y; Mr[4 * j + 2] = r[j].z; Mr[4 * j + 3] = r[j].w; }
            } else {
                u32 dm = rfl(de.y) >> 13;
                while (!dm) { __builtin_amdgcn_s_sleep(1); dm = rfl(lds32v(D_DESC + c * 8 + 4)) >> 13; }
                const u32 SM = S + dm * 16;
                while (rfl(lds32v(CTL_LANDED)) < (SM + 9215) >> 10) __builtin_amdgcn_s_sleep(1);
                CBAR();
                ring_read8k(SM, lane, Mr);
            }
            WAIT_LGKM0(); lds32vw(CTL_NEXT + dw * 4, c + ND); CBAR();
            if (gid >= (u32)GY) {
                const u32 t = gid - GY;
                u16* o = y_base(P, t & 15, global_panel(t >> 4, q), w, lane);
#pragma unroll
                for (int p = 0; p < 16; ++p) st_out(o + y_piece_off(p), piece(Mr, Er, p));
            } else {
                const ImageStore im(P, gid, w, q, lane, true);
#pragma unroll
                for (int p = 0; p < 16; ++p) im(p, piece(Mr, Er, p));
                im.done(lane);
            }
        }
    }
}

__device__ __forceinline__ void pack_inputs(const Params& P, u32 gid, u32 w, bool solo, u32 wid, u32 lane) {
    asm volatile("" : "+v"(lane));
    u32 v[64];
    u32 peers = 255u & ~(1u << P.rank); bool own = true;
    if (gid < (u32)GW) load_x(P, gid, w, lane, v);
    else {
        const u32 j = (gid - GW) >> 1, ktl = (gid & 1) * 4 + (w >> 1);
        if (j < (u32)(NR * DP)) { const u32 dest = j / DP; own = dest == (u32)P.rank; peers = own ? 0u : 1u << dest; }
        else if (j < (u32)(NR * (DP + FX))) own = (j & 7) != P.rank;
        const u16* s = P.w + (size_t)(ktl * 64 + (lane >> 4)) * NN + j * 256 + (w & 1) * 128 + (lane & 15) * 8;
        const u32 scr = wid * 16384;
#pragma unroll
        for (int i = 0; i < 16; ++i) glds_w(s + (size_t)i * 4 * NN, scr + i * 1024);
        WAIT_VM(0);
        const u32 ta = scr + ((lane >> 4) * 8 + ((lane & 15) >> 2)) * 256 + (lane & 3) * 16;
#pragma unroll
        for (int p = 0; p < 16; ++p) {
            const u32 a = ta + (p & 1) * 32 * 256 + (p >> 2) * 64 + ((p >> 1) & 1) * 8;
            const u32x2 lo = __builtin_bit_cast(u32x2, __builtin_amdgcn_ds_read_tr16_b64_v4i16((LAS s16x4*)a));
            const u32x2 hi = __builtin_bit_cast(u32x2, __builtin_amdgcn_ds_read_tr16_b64_v4i16((LAS s16x4*)(a + 4 * 256)));
            v[4 * p] = lo.x; v[4 * p + 1] = lo.y; v[4 * p + 2] = hi.x; v[4 * p + 3] = hi.y;
        }
        WAIT_LGKM0();
    }
    const u32 call = call_take(wid);
    const ImageStore im(P, gid, w, P.rank, lane, own);
    if (peers) {
        u32* fp; u32 fv;
        pack_group<false>(P, v, gid, w, solo, peers, call, wid, lane, im, fp, fv);
        WAIT_VM(0);
        if (!solo) BAR();
        raise_flags(fp, fv);
        if (solo) { load_x(P, gid, w, lane, v); __builtin_amdgcn_s_waitcnt(0x0f70); }
        else if (own && lane == 0) __hip_atomic_fetch_add(im.cnt, 1u, __ATOMIC_RELAXED, __HIP_MEMORY_SCOPE_AGENT);
    }
    if (!peers || solo) {
#pragma unroll
        for (int p = 0; p < 16; ++p) im(p, (u32x4){v[4 * p], v[4 * p + 1], v[4 * p + 2], v[4 * p + 3]});
        im.done(lane);
    }
}

typedef __bf16 bf16x2 __attribute__((ext_vector_type(2)));
typedef float f32x2 __attribute__((ext_vector_type(2)));
__device__ __forceinline__ u32 cvt_pk(float lo, float hi) { return __builtin_bit_cast(u32, __builtin_convertvector((f32x2){lo, hi}, bf16x2)); }
__device__ __forceinline__ void wait_panels(const Params& P, u32 tile, u32 upto, u32 call, u32 wid, u32 lane) {
    if (wid == 0) {
        const u32 st = umin(lane & 7, upto), need = call * (st == 0 ? 16u : st < 4 ? 28u : 14u);
        const u32* c = counter(P, (lane & 8) ? 16 + (tile >> 4) : (tile & 15)) + st;
        const u64 t0 = now();
        for (;;) {
            const u32 x = __hip_atomic_load(c, __ATOMIC_RELAXED, __HIP_MEMORY_SCOPE_AGENT);
            if (__all((int)(x - need) >= 0)) break;
            if (__any(expired(P, t0, call))) { asm volatile("ds_write_b32 %0, %1\n\ts_waitcnt lgkmcnt(0)" :: "v"(CTL_BAD), "v"(1u) : "memory"); break; }
            __builtin_amdgcn_s_sleep(4);
        }
        __builtin_amdgcn_fence(__ATOMIC_ACQUIRE, "agent");
        WAIT_VM(0);
    }
    BAR();
}
__device__ __forceinline__ void stage2(u32 dst, u32 voff, const char* src) {
    asm volatile("s_mov_b32 m0, %2\n\ts_nop 0\n\tglobal_load_lds_dwordx4 %0, %1\n\tglobal_load_lds_dwordx4 %0, %1 offset:1024" :: "v"(voff), "s"(src), "s"(dst) : "memory");
}
__device__ __forceinline__ void stage4(u32 dst, u32 voff, const char* src) {
    asm volatile("s_mov_b32 m0, %2\n\ts_nop 0\n\tglobal_load_lds_dwordx4 %0, %1\n\tglobal_load_lds_dwordx4 %0, %1 offset:1024\n\t"
                 "global_load_lds_dwordx4 %0, %1 offset:2048\n\tglobal_load_lds_dwordx4 %0, %1 offset:3072" :: "v"(voff), "s"(src), "s"(dst) : "memory");
}
__device__ __forceinline__ void gemm_role(const Params& P, u32 place, u32 call, u32 wid, u32 lane) {
    const u32 wr = wid >> 2, wc = wid & 3;
    u32 aoff = wr * 8192 + (lane & 15) * 128 + (((lane >> 4) ^ lane) & 7) * 16, aoff1 = aoff ^ 64, boff = 65536 + wc * 4096 + lane * 16;
    const u32 mA = wid * 2048, mB = 65536 + wid * 4096;
    u32 vA0 = mA + lane * 16, vA1 = vA0 + 16384, vA2 = vA0 + 32768, vA3 = vA0 + 49152, vB0 = wid * 4096 + lane * 16, vB1 = vB0 + 32768;
    asm volatile("" : "+v"(vA0), "+v"(vA1), "+v"(vA2), "+v"(vA3), "+v"(vB0), "+v"(vB1), "+v"(aoff), "+v"(aoff1), "+v"(boff));
#define G_LDA(b, h) do { _Pragma("unroll") for (int m = 0; m < 4; ++m) _Pragma("unroll") for (int k = 0; k < 2; ++k) At[m][k] = *(const LAS bf16x8*)((k ? aoff1 : aoff) + ((b) * 2 + (h)) * 16384 + m * 2048); } while (0)
#define G_LDB(b) do { _Pragma("unroll") for (int j = 0; j < 2; ++j) _Pragma("unroll") for (int n = 0; n < 2; ++n) _Pragma("unroll") for (int k = 0; k < 2; ++k) \
        Bt[j][n][k] = *(const LAS bf16x8*)(boff + ((b) * 2 + j) * 16384 + n * 2048 + k * 1024); } while (0)
#define G_MMA(ai, STEP) do { G_WV(8); __builtin_amdgcn_s_waitcnt(0xc07f);   G_SCHED; G_BAR; __builtin_amdgcn_s_setprio(1); _Pragma("unroll") for (int i = 0; i < 32; ++i) { \
        const int bj = i >> 4, k = (i >> 3) & 1, m = (i >> 1) & 3, n = i & 1; \
        if (i == 8) { G_SCHED; STEP; G_SCHED; } \
        if (i == 32 - 2) { G_SCHED; __builtin_amdgcn_s_setprio(3); G_BAR; G_SCHED; } \
        acc[ai][bj][m][n] = __builtin_amdgcn_mfma_f32_16x16x32_bf16(Bt[bj][n][k], At[m][k], acc[ai][bj][m][n], 0, 0, 0); } \
        __builtin_amdgcn_s_setprio(0); G_SCHED; } while (0)
#define G_WV(n) asm volatile("s_waitcnt vmcnt(" #n ")" ::: "memory")
#define G_BAR __builtin_amdgcn_s_barrier()
#define G_SCHED __builtin_amdgcn_sched_barrier(0)
    const u32x2 cl = lds64v(TAB_COLS + ((blockIdx.x & 4) * 2 + (place >> 2)) * 8);
    const u64 mycols = (u64)rfl(cl.y) << 32 | rfl(cl.x); const u32 pm = (blockIdx.x & 3) * 4 + (place & 3);
    const char* const cA = P.me + WS_XI + (size_t)pm * PANEL; const char* const WI = P.me + WS_WI;
    u32 cur = pm | ((u32)mycols & 255u) << 4;
    bool fresh = true; u32* fp = nullptr; u32 fv = 0;
    bf16x8 At[4][2], Bt[2][2][2];
    for (u32 ui = 0;; ++ui) {
        const u32 nxt = pm | ((u32)(mycols >> (8 * ui + 8)) & 255u) << 4;
        const bool has_next = (nxt >> 4) != 255u, dist = (cur >> 4) < (u32)PK, chain = has_next && !dist;
        const char* cB = WI + (size_t)(cur >> 4) * PANEL;
        if (fresh) {
            wait_panels(P, cur, ui > 1 ? 5u : 0u, call, wid, lane);
            stage4(mB, vB0, cB); stage2(mA, vA0, cA); stage2(mA + 16384, vA1, cA);
            if (wr == 1) G_BAR;
            G_WV(2); G_BAR;
            stage4(mB + 32768, vB1, cB); stage2(mA + 32768, vA2, cA);
            G_WV(6); G_BAR;
            raise_flags(fp, fv); fp = nullptr; fv = 0;
        }
        f32x4 acc[2][2][4][2];
#pragma unroll
        for (int a = 0; a < 2; ++a)
#pragma unroll
            for (int b = 0; b < 2; ++b)
#pragma unroll
                for (int m = 0; m < 4; ++m)
#pragma unroll
                    for (int n = 0; n < 2; ++n) acc[a][b][m][n] = (f32x4){0.f, 0.f, 0.f, 0.f};
        const char *p0 = cA, *pA = cA + 65536, *pB = cB + 65536;
        u32 stops = 0x80000000u | (ui == 0 ? 0x09020408u : ui == 1 ? 0x00020008u : 0u), upto = 0;
        for (u32 trip = 0;;) {
            const u32 end = stops ? (u32)__builtin_ctz(stops) : 32u;
            u32 left = end - trip;
            do {
                G_LDB(0); G_SCHED; G_LDA(0, 0); stage2(mA + 49152, vA3, p0);
                G_MMA(0, );
                G_LDA(0, 1); stage4(mB, vB0, pB); stage2(mA, vA0, pA);
                G_MMA(1, );
                G_LDB(1); G_SCHED; G_LDA(1, 0); stage2(mA + 16384, vA1, pA);
                G_MMA(0, );
                G_LDA(1, 1); stage4(mB + 32768, vB1, pB); stage2(mA + 32768, vA2, pA);
                G_MMA(1, p0 = pA; pA += 65536; pB += 65536; --left; asm volatile("" : "+s"(p0), "+s"(pA), "+s"(pB), "+s"(left)));
            } while (left);
            if (!stops) break;
            trip = end; stops &= stops - 1;
            if (stops) { upto += ui ? (upto ? 3u : 2u) : 1u; wait_panels(P, cur, upto, call, wid, lane); }
            else {
                pA = cA; pB = cB;
                if (chain) { pB = WI + (size_t)(nxt >> 4) * PANEL; wait_panels(P, nxt, ui ? 5u : 0u, call, wid, lane); }
            }
        }
        if (!chain) G_WV(0);
        if (wr == 0) G_BAR;
        u32 v[64];
        {
            int tz = threadIdx.x; asm volatile("" : "+v"(tz));
            const u32 ln = tz & 63, bad = lds32v(CTL_BAD);
            const u32 ep = D_DESC + wid * 2048, to = ep + (ln & 15) * 128 + (((ln >> 4) ^ (ln & 7)) * 16), from = ep + (ln ^ (ln >> 3)) * 16;
            u16* o = y_base(P, cur & 15, global_panel(cur >> 4, P.rank), wid, ln);
#pragma unroll
            for (int p = 0; p < 16; p += 2) {
                const int ai = p >> 3, m = (p >> 1) & 3;
#pragma unroll
                for (int bj = 0; bj < 2; ++bj) {
                    const f32x4 x0 = acc[ai][bj][m][0], x1 = acc[ai][bj][m][1];
                    lds128w(to ^ (bj * 64), (u32x4){bad ? 0x7fc07fc0u : cvt_pk(x0[0], x0[1]), cvt_pk(x0[2], x0[3]), cvt_pk(x1[0], x1[1]), cvt_pk(x1[2], x1[3])});
                }
                CBAR();
#pragma unroll
                for (int j = 0; j < 2; ++j) { const u32x4 t = lds128(from + j * 1024); v[4 * (p + j)] = t.x; v[4 * (p + j) + 1] = t.y; v[4 * (p + j) + 2] = t.z; v[4 * (p + j) + 3] = t.w; }
                CBAR();
            }
            if (!dist) {
#pragma unroll
                for (int p = 0; p < 16; ++p) st_out(o + y_piece_off(p), (u32x4){v[4 * p], v[4 * p + 1], v[4 * p + 2], v[4 * p + 3]});
            }
            if (chain) { if (wr == 1) G_BAR; }
            else {
                if (dist) {
                    sample<6, 10, 5>(v, LDS_HIST, ln);
                    WAIT_LGKM0(); BAR();
                    const OutStore os = {o};
                    pack_group<true>(P, v, GY + cur, wid, false, (cur >> 4) < (u32)DP ? 255u & ~(1u << P.rank) : 1u << (((cur >> 4) - DP) & 7), call, wid, ln, os, fp, fv);
                    if (!has_next) { G_WV(0); BAR(); raise_flags(fp, fv); }
                }
                if (!has_next) break;
            }
        }
        fresh = !chain; cur = nxt;
    }
}

__global__ void __launch_bounds__(512, 2) fused(const Args A) {
    const u32 wid = rfl(threadIdx.x >> 6), lane = threadIdx.x & 63;
    asm volatile("gar_code0:");
    const u32 role = pin((((const u32*)A.role)[blockIdx.x >> 1] >> ((blockIdx.x & 1) * 16)) & 0xffffu);
    char* peer = pin(A.ws[0]); char* lanep = peer;
#pragma unroll
    for (u32 j = 1; j < NR; ++j) { char* c = pin(A.ws[j]); peer = (role & 7) == j ? c : peer; lanep = (lane & 7) == j ? c : lanep; }
    const Params P = {pin(A.me), peer, pin(A.x), pin(A.w), pin(A.out), pin((u32)A.rank), pin((u32)A.nunits)};
    call_load(P, wid, lane);
    if (role & 0x8000u) {
        if (threadIdx.x == 0) { lds32vw(CTL_NUNIT, 0u); lds32vw(CTL_LANDED, 0u); lds32vw(CTL_NCHUNK, 0u); }
        if (wid == 1 && lane < 8) lds32vw(CTL_NEXT + lane * 4, lane);
        WAIT_LGKM0(); BAR();
        puller(P, role, wid, lane); return;
    }
    if (threadIdx.x == 0) lds32vw(CTL_BAD, 0u);
    if (threadIdx.x < 8) lds32vw(CTL_SIZE + threadIdx.x * 4, 0u);
    pack_init(wid, lane);
    const u32 rb = NENC - 1 - (role & 255u), sc = rb + (wid & 2) * 32;
    const bool soloist = (wid == 1 || wid == 2) && rb < 64u;
    if (soloist) { if (lane < 8) lds64vw(LDS_TAB + lane * 8, __builtin_bit_cast(u32x2, (u64)lanep)); }
    else {
        if (rb >= 64u) {
            if (wid == 3) {
                u64 pc; u32 back, len;
                asm volatile("s_getpc_b64 %0\ngar_here:\n\ts_mov_b32 %1, gar_here-gar_code0\n\ts_mov_b32 %2, gar_code1-gar_code0" : "=s"(pc), "=s"(back), "=s"(len));
                for (u32 o = ((blockIdx.x >> 3) & 15) * 1024 + lane * 16; o < len; o += 16384) glds_plain((const char*)(const GAS char*)((pc - back) & ~15ull) + o, LDS_CTL - 1024);
            }
        } else hold(100);
        if (wid == 0) load_tables(lane);
        if (rb >= 64u) hold(150);
    }
    for (u32 it = 0; it < 3; ++it) {
        const u32 gid = it ? tab16(TAB_ENC, (role & 255u) * 2 + it - 1) : solo_gid(sc);
        if (gid == 0xffffu) break;
        if (it || soloist) pack_inputs(P, gid, it ? wid : solo_w(sc), !it, wid, lane);
        WAIT_VM(0); WAIT_LGKM0(); BAR();
    }
    const u32 call = call_take(wid);
    call_bump(P, wid, lane, call);
    if (role >> 8) gemm_role(P, (role >> 8) - 1, call, wid, lane);
    asm volatile("gar_code1:");
}

static void build_tables(Args& P) {
    unsigned char* cols = P.cols; u16 *stream = P.stream, *role = P.role, *job = P.enc, enc[NPACK];
    {
        int peer[2][4], np[2] = {0, 0}, cnt[8] = {}, seen[2] = {0, 0}, n = 0, top = NENC;
        for (int d = 1; d < NR; ++d) { const int q = (P.rank + d) & 7, h = ((0xb2 >> q) ^ (0xb2 >> P.rank)) & 1; peer[h][np[h]++] = q; }
        for (int b = 0; b < NWG; ++b) {
            const int h = (b >> 2) & 1, i = b >> 3, g0 = 32 - 4 * (h ? CH : CL), near = h ? 4 * NH : 3 * NL, far = h ? 3 * FL : 4 * FH, k = seen[h]++;
            if (k >= near + far) { role[b] = i >= g0 ? (u16)(n++ | (i - g0 + 1) << 8) : (u16)--top; continue; }
            const int side = k < near ? h : !h, j = k < near ? k : k - near, q = peer[side][side ? (j + j / 4) & 3 : j % 3];
            role[b] = (u16)(0x8000 | q | cnt[q]++ << 3);
        }
        for (int b = 0; b < NWG; ++b) if (role[b] & 0x8000) role[b] |= (u16)(cnt[role[b] & 7] << 7);
    }
    int col[LC], nc = 0;
    {
        const int rest = LC - (ROUNDS - 1) * CPR, r0 = (rest + 1) / 2, light = rest < CPR ? CPR - rest : CPR;
        int over = PK - FX - light * (ROUNDS - 1), nd = 0, nr = PK;
        for (int i = 0; i < 128; ++i) cols[i] = 0xff;
        for (int r = 0; r < ROUNDS; ++r) for (int c = 0; c < 8; ++c) for (int cg = 0; cg < 2; ++cg) {
            const bool last = c < (cg ? rest - r0 : r0), heavy = last && rest < CPR;
            if (nd >= DP && nd < PK && ((nd - DP) & 7) == P.rank) ++nd;
            if (c >= (cg ? CH : CL) || (r + 1 == ROUNDS && !last)) continue;
            cols[(cg * 8 + c) * 8 + r] = (unsigned char)(col[nc++] = nd < PK && (!heavy || over-- > 0) ? nd++ : nr++);
        }
    }
    auto wgid = [&](int jj, int dest, int h) { return GW + 2 * (jj >= DP ? jj + (NR - 1) * DP : dest * DP + jj) + h; };
    int k = 0;
    auto put = [&](int gid, int lv) { for (int j = 0; j < (1 << lv); ++j) stream[k++] = (u16)(gid | ((1 << lv) + j) << 10); };
    auto opnd = [&](int i, int h) { return i < 16 ? GX + 16 * h + i : wgid(col[i - 16], P.rank, h); };
    auto stage = [&](int st, int i0, int i1) {
        const int h = st >> 1, half = st & 1;
        for (int i = i0; i < i1; ++i) {
            if ((i < 16 && h == 0) || (st < 1 && i < 16 + CPR - 4)) { for (int j = 0; j < 2; ++j) stream[k++] = (u16)(opnd(i, h) | (4 + 2 * half + j) << 10); }
            else stream[k++] = (u16)(opnd(i, h) | (2 + half) << 10);
        }
    };
    stage(0, NRAW / 4, 16); stage(1, 0, 9);
    {
        u16 t[64]; const int n = k; int e = 0;
        for (int i = 0; i < n; ++i) t[i] = stream[i];
        k = 0;
        for (int i = 0; i < n || e < NRAW / 4; ++i) { if (i < n) stream[k++] = t[i]; if (i >= 18 && e < NRAW / 4) stream[k++] = (u16)((GX + e++) | 2 << 10); }
    }
    const int first = 16 + CPR - 4;
    stage(0, 16, first); stage(1, 9, first); stage(0, first, 16 + CPR); stage(2, 0, first); stage(1, first, 16 + CPR);
    for (int part = 6; part < 8; ++part) for (int i = 0; i < first; ++i) stream[k++] = (u16)(opnd(i, 1) | part << 10);
    stage(2, first, 16 + CPR); stage(3, first, 16 + CPR);
    for (int h = 0; h < 2; ++h) for (int i = CPR; i < 2 * CPR; ++i) put(wgid(col[i], P.rank, h), 0);
    for (int i = 2 * CPR; i < LC; ++i) for (int h = 0; h < 2; ++h) put(wgid(col[i], P.rank, h), 0);
    for (int t = 0; t < 16 * DP; ++t) { const int left = 16 * DP - t; put(GY + t, left <= 8 ? 1 : 0); }
    const int nown = k;
    for (int t = 0; t < 16 * FX; ++t) put(GY + (DP + (t >> 4) * 8 + P.rank) * 16 + (t & 15), 16 * FX - t <= 16 ? 3 : 16 * FX - t <= 30 ? 2 : 1);
    P.nunits = nown | (k - nown) << 16;
    k = 0;
    for (int i = 0; i < CPR; ++i) for (int d = 1; d < (col[i] < DP ? NR : 2); ++d) enc[k++] = (u16)wgid(col[i], (P.rank + d) & 7, 0);
    for (int i = SOLOG; i < 32; ++i) enc[k++] = (u16)(GX + i);
    for (int i = 0; i < CPR; ++i) { if (col[i] < DP) enc[k++] = (u16)wgid(col[i], P.rank, 0); enc[k++] = (u16)wgid(col[i], P.rank, 1); }
    for (int i = 0; i < CPR; ++i) if (col[i] < DP) for (int d = 1; d < NR; ++d) enc[k++] = (u16)wgid(col[i], (P.rank + d) & 7, 1);
    for (int f = 0; f < FX; ++f) for (int h = 0; h < 2; ++h) enc[k++] = (u16)wgid(DP + f * 8 + P.rank, 0, h);
    for (int i = CPR; i < LC; ++i) for (int h = 0; h < 2; ++h) for (int d = 0; d < (col[i] < DP ? NR : 1); ++d) enc[k++] = (u16)wgid(col[i], (P.rank + d) & 7, h);
    int nx = 0, top = NENC;
    for (int i = 0; i < NENC - 64; ++i) { job[2 * i] = enc[nx++]; job[2 * i + 1] = 0xffff; }
    for (int i = 0; i < NENC - 64; ++i) if (job[2 * i] >= GW && (job[2 * i] - GW) / 2 / DP == P.rank) job[2 * i + 1] = enc[nx++];
    for (int i = NENC - 64; i < NENC; ++i) { job[2 * i] = enc[nx++]; job[2 * i + 1] = 0xffff; }
    while (nx < NPACK) { do --top; while (job[2 * top + 1] != 0xffff); job[2 * top + 1] = enc[nx++]; }
}
}

extern "C" void kernel_launch(int rank, int nranks, ncclComm_t comm, hipStream_t stream,
                              void* const* d_in, const int* in_sizes, int n_in,
                              void* d_out, int out_size, const int* in_dtypes, int out_dtype,
                              void* const* d_out_all, void* const* d_ws_all, size_t ws_bytes) {
    using namespace gar;
    if (nranks != NR || (unsigned)rank >= (unsigned)NR || n_in != 2 || ws_bytes < WS_END || in_sizes[0] != MM * KS || in_sizes[1] != KS * NN || out_size != MM * NN) {
        fprintf(stderr, "kernel_launch: unexpected shapes\n"); return; }
    Args P = {};
    for (int p = 0; p < NR; ++p) P.ws[p] = (char*)d_ws_all[p];
    P.me = P.ws[rank];
    P.x = (const u16*)d_in[0]; P.w = (const u16*)d_in[1]; P.out = (u16*)d_out; P.rank = rank;
    build_tables(P);
    HIP_CHECK(hipFuncSetAttribute((const void*)fused, hipFuncAttributeMaxDynamicSharedMemorySize, LDS_BYTES));
    hipLaunchKernelGGL(fused, dim3(NWG), dim3(512), LDS_BYTES, stream, P);
    HIP_CHECK(hipGetLastError());
}
```
